```python
import numpy as np

kernel_name = "gpu_gemm_rs_m4096_k4096_n16384_e5m2e5m2_dequant"


ABSMAX_RTOL = 0.05

COMPUTE_DTYPE = "float32"
EPILOGUE = "dequant"
IN_QUANT = None
OUT_QUANT = None
IN_FP8 = "e5m2e5m2"
IN_DTYPES = {"x": "e5m2", "w_mat": "e5m2",
             "scale_x": "float32", "scale_w": "float32"}
WORLD_SIZE = 8

def _snap_fp8(x, mant_bits, max_abs, min_normal):
    a = np.abs(x.astype(np.float64))
    _m, _e = np.frexp(a)
    q = np.where(a >= min_normal, np.ldexp(1.0, _e - 1 - mant_bits),
                 min_normal * 2.0 ** -mant_bits)
    snapped = np.minimum(np.round(a / q) * q, max_abs)
    return (np.sign(x) * snapped).astype(np.float32)

def setup_inputs():

    x = _snap_fp8(np.random.randn(4096, 4096).astype(np.float32), 2, 57344.0, 2.0 ** -14)
    w_mat = _snap_fp8(np.random.randn(4096, 16384).astype(np.float32), 2, 57344.0, 2.0 ** -14)
    scale_x = (0.02 * (1.0 + np.random.rand(1))).astype(np.float32)
    scale_w = (0.02 * (1.0 + np.random.rand(1))).astype(np.float32)
    return {"x": x, "w_mat": w_mat, "scale_x": scale_x, "scale_w": scale_w}

def reference(x, w_mat, scale_x, scale_w):
    acc = (x @ w_mat).astype(np.float32)
    y = (acc * (scale_x * scale_w)).astype(np.float32)
    return y
```

```cpp
#include <hip/hip_runtime.h>
#include <rccl/rccl.h>
#include <cstdio>
#include <cstdint>

#define HIP_CHECK(expr) do { hipError_t _e = (expr); if (_e != hipSuccess) { \
    fprintf(stderr, "HIP error %s at %s:%d\n", hipGetErrorString(_e), __FILE__, __LINE__); \
    return; } } while (0)

#define ENC_IN 1
#define ENC_OUT 1
#define PULL_WAVES 8
#define PULL_WGS 8
#define STEP_IN 2
#define STEP_OUT 2
#define GEMM_AUX 17
#define LOC_AUX 17
#define OUT_AUX 0

namespace {

typedef unsigned u32;
typedef unsigned long long u64;
typedef unsigned u32x4 __attribute__((ext_vector_type(4)));
typedef unsigned u32x2 __attribute__((ext_vector_type(2)));
typedef int i32x2 __attribute__((ext_vector_type(2)));
typedef int i32x8 __attribute__((ext_vector_type(8)));
typedef float f32x4 __attribute__((ext_vector_type(4)));
typedef __amdgpu_buffer_rsrc_t rsrc_t;
#define LDS __attribute__((address_space(3)))
#define GLB __attribute__((address_space(1)))

constexpr int NR = 8, MM = 4096, NN = 16384, KS = 512, NCOL = 2048;
constexpr int NWG = 256, NTHR = 512;
constexpr int NPULL = 7 * PULL_WGS, NGEMM = NWG - NPULL;
constexpr int AUX_SYS = 17;
constexpr u64 DEADLINE_TICKS = 40000000ull;

constexpr u32 KiB = 1024, MiB = 1024 * 1024;
constexpr u32 OFF_CALL = 0;
constexpr u32 OFF_TMO = 32 * KiB;
constexpr u32 OFF_CNT = 36 * KiB;
constexpr u32 OFF_FIN = 64 * KiB;
constexpr u32 OFF_FOUT = OFF_FIN + 8 * 384 * 128;
constexpr u32 OFF_DBG = 768 * KiB;
constexpr u32 OFF_XA = 1 * MiB;
constexpr u32 OFF_WA = OFF_XA + 16 * MiB;
constexpr u32 OFF_PAR = OFF_WA + 8 * MiB;
constexpr u32 IU_ENC = 7 * KiB, IU_RAW = 8 * KiB;
constexpr u32 OU_ENC = 13824, OU_RAW = 16 * KiB;
constexpr u32 IU_SIDE = 6144, OU_SIDE = 12800;
constexpr u32 EX_INE = 0;
constexpr u32 EX_INR = EX_INE + 1280 * IU_ENC;
constexpr u32 EX_OUE = EX_INR + 1280 * IU_RAW;
constexpr u32 EX_OUR = EX_OUE + 2048 * OU_ENC;
constexpr u32 PAR_BYTES = EX_OUR + 2048 * OU_RAW;
constexpr u32 WS_NEED = OFF_PAR + 2 * PAR_BYTES;
static_assert(OFF_FOUT + 8 * 256 * 128 <= OFF_DBG, "flags");
static_assert(WS_NEED <= 257u * MiB, "workspace");
constexpr u32 IN_CAP0 = 126, IN_CAP = 510;
constexpr u32 OUT_CAP0 = 15, OUT_CAP = 127;

constexpr u32 STAGE = 48 * KiB;
constexpr u32 L_SCR = 64 * KiB;
constexpr u32 L_EPI = 3 * STAGE;
constexpr u32 LDS_BYTES = L_EPI + 8 * KiB;

struct Params {
  const uint8_t* x; const uint8_t* w; const float* sx; const float* sw;
  float* out; char* const* ws_all; char* ws; int rank; u32 ws_bytes;
};

__device__ __forceinline__ rsrc_t mk_rsrc(const void* p, u32 bytes) {
  u64 a = (u64)p;
  u32 lo = __builtin_amdgcn_readfirstlane((u32)a), hi = __builtin_amdgcn_readfirstlane((u32)(a >> 32));
  return __builtin_amdgcn_make_buffer_rsrc((void*)(((u64)hi << 32) | lo), 0, (int)bytes, 0x00020000);
}
#define ST128(v, rs, voff, soff, aux) __builtin_amdgcn_raw_buffer_store_b128((v), (rs), (int)((u32)(voff) + (u32)(soff)), 0, (aux))
__device__ __forceinline__ float as_f(u32 v) { return __builtin_bit_cast(float, v); }
__device__ __forceinline__ u32 as_u(float v) { return __builtin_bit_cast(u32, v); }
__device__ __forceinline__ void vm_drain() { asm volatile("s_waitcnt vmcnt(0)" ::: "memory"); }
__device__ __forceinline__ void wave_sync() {
  asm volatile("s_waitcnt lgkmcnt(0)" ::: "memory");
}
__device__ __forceinline__ u32 bfi(u32 m, u32 a, u32 b) { return (m & a) | (~m & b); }
__device__ __forceinline__ u32 wave_sum_u32(u32 v) {
  #pragma unroll
  for (int o = 32; o; o >>= 1) v += __shfl_xor(v, o);
  return __builtin_amdgcn_readfirstlane(v);
}
__device__ __forceinline__ u32 ld_flag(const char* ws, u32 off) {
  return __hip_atomic_load((const GLB u32*)(ws + off), __ATOMIC_RELAXED, __HIP_MEMORY_SCOPE_SYSTEM);
}
__device__ __forceinline__ bool wait_ge(char* ws, u32 off, u32 target, u32 call) {
  if (__all(ld_flag(ws, off) >= target)) return true;
  const u64 t0 = __builtin_amdgcn_s_memrealtime();
  GLB u32* tmo = (GLB u32*)(ws + OFF_TMO);
  for (;;) {
    __builtin_amdgcn_s_sleep(4);
    if (__all(ld_flag(ws, off) >= target)) return true;
    if (__hip_atomic_load(tmo, __ATOMIC_RELAXED, __HIP_MEMORY_SCOPE_AGENT) == call ||
        __builtin_amdgcn_s_memrealtime() - t0 > DEADLINE_TICKS) {
      __hip_atomic_store(tmo, call, __ATOMIC_RELAXED, __HIP_MEMORY_SCOPE_AGENT);
      return false;
    }
  }
}
__device__ __forceinline__ void cnt_add(char* ws, u32 idx, int lane) {
  if (lane == 0) __hip_atomic_fetch_add((GLB u32*)(ws + OFF_CNT + idx * 128), 1u, __ATOMIC_RELAXED, __HIP_MEMORY_SCOPE_AGENT);
}
#define STAMPC(slot) do {} while (0)
#define STAMP(slot) do {} while (0)

__device__ __forceinline__ void store_raw_unit(const u32x4 (&x)[8], rsrc_t rs, u32 off, int lane) {
  #pragma unroll
  for (int j = 0; j < 8; ++j) ST128(x[j], rs, lane * 16 + j * 1024, off, AUX_SYS);
}

__device__ __forceinline__ void export_unit(const u32x4 (&x)[8], rsrc_t rs, u32 par, u32 uid, LDS char* scr, int lane) {
  const u32 offE = par + EX_INE + uid * IU_ENC, offR = par + EX_INR + uid * IU_RAW;
  if (!ENC_IN) { store_raw_unit(x, rs, offR, lane); return; }
  u32 s = 0;
  #pragma unroll
  for (int j = 0; j < 8; ++j) {
    *(LDS u32x4*)(scr + j * 1024 + lane * 16) = x[j];
    #pragma unroll
    for (int c = 0; c < 4; ++c) s = __builtin_amdgcn_sad_u8(x[j][c] & 0x7C7C7C7Cu, 0u, s);
  }
  s = wave_sum_u32(s);
  int e0 = ((int)s - 136970) >> 15;
  e0 = e0 < 0 ? 0 : (e0 > 24 ? 24 : e0);
  const u32 e4 = (u32)e0 * 0x04040404u;
  u32 mk[4] = {0, 0, 0, 0};
  u32 pk[24];
  #pragma unroll
  for (int j = 0; j < 8; ++j) {
    u32 o[4];
    #pragma unroll
    for (int c = 0; c < 4; ++c) {
      const u32 u = (x[j][c] | 0x80808080u) - e4;
      o[c] = bfi(0x1F1F1F1Fu, u, x[j][c]);
      const u32 t = (((u >> 1) ^ 0x40404040u) & 0x70707070u) + 0x70707070u;
      const int d = j * 4 + c;
      mk[d >> 3] = bfi(0x80808080u >> (d & 7), t >> (d & 7), mk[d >> 3]);
    }
    pk[j * 3 + 0] = bfi(0x60606060u, o[3] << 5, o[0]);
    pk[j * 3 + 1] = bfi(0x60606060u, o[3] << 3, o[1]);
    pk[j * 3 + 2] = bfi(0x60606060u, bfi(0x40404040u, o[3] >> 1, o[3] << 1), o[2]);
  }
  #pragma unroll
  for (int j = 0; j < 6; ++j) {
    const u32x4 v = {pk[j * 4], pk[j * 4 + 1], pk[j * 4 + 2], pk[j * 4 + 3]};
    ST128(v, rs, lane * 16 + j * 1024, offE, AUX_SYS);
  }
  const u32 mine = __popc(mk[0]) + __popc(mk[1]) + __popc(mk[2]) + __popc(mk[3]);
  u32 inc = mine;
  #pragma unroll
  for (int o = 1; o < 64; o <<= 1) { const u32 t = __shfl_up(inc, o); if (lane >= o) inc += t; }
  const u32 total = __builtin_amdgcn_readlane(inc, 63);
  u32 hdr = total | ((u32)e0 << 16);
  LDS unsigned short* list = (LDS unsigned short*)(scr + 8192);
  if (total <= IN_CAP) {
    u32 slot = 2 + inc - mine;
    wave_sync();
    #pragma unroll
    for (int q = 0; q < 4; ++q) {
      u32 m = mk[q];
      while (m) {
        const int bp = __builtin_ctz(m); m &= m - 1;
        const int d = q * 8 + 7 - (bp & 7);
        const u32 pos = (d >> 2) * 1024 + lane * 16 + (d & 3) * 4 + (bp >> 3);
        const u32 by = *(LDS unsigned char*)(scr + pos);
        list[slot++] = (unsigned short)((pos << 3) | ((128u + (by & 127u) - 4u * (u32)e0) >> 5));
      }
    }
    wave_sync();
  } else {
    hdr = 0x80000000u;
    store_raw_unit(x, rs, offR, lane);
  }
  u32 sd = *(LDS u32*)(scr + 8192 + lane * 4);
  if (lane == 0) sd = hdr;
  __builtin_amdgcn_raw_buffer_store_b32(sd, rs, lane * 4, offE + 6144, AUX_SYS);
  if (total > IN_CAP0 && total <= IN_CAP) {
    #pragma unroll
    for (int j = 1; j < 4; ++j)
      __builtin_amdgcn_raw_buffer_store_b32(*(LDS u32*)(scr + 8192 + j * 256 + lane * 4), rs, lane * 4 + j * 256, offE + 6144, AUX_SYS);
  }
}

__device__ __forceinline__ void patch_in(LDS char* scr, u32 ent) {
  const u32 pos = ent >> 3;
  __hip_atomic_fetch_add((LDS u32*)(scr + (pos & ~3u)), (((ent & 7u) - 4u) << 5) << ((pos & 3u) * 8u), __ATOMIC_RELAXED, __HIP_MEMORY_SCOPE_WORKGROUP);
}

__device__ __forceinline__ float unfix(int i, int g) { return __builtin_amdgcn_ldexpf((float)i, -g); }

__global__ void __launch_bounds__(NTHR, 2) gemm_rs_kernel(Params p) {
  extern __shared__ __attribute__((aligned(16))) char smem_[];
  LDS char* smem = (LDS char*)smem_;
  const int tid = threadIdx.x, lane = tid & 63, wave = __builtin_amdgcn_readfirstlane(tid >> 6);
  const int b = blockIdx.x, rank = p.rank;
  const int lg = lane >> 4, li = lane & 15;
  STAMP(0);
  const rsrc_t rsL = mk_rsrc(p.ws, p.ws_bytes);
  const u32 tick = (u32)__builtin_amdgcn_raw_ptr_buffer_atomic_add_i32(1, rsL, lane == 0 ? 0 : (int)0xF0000000u, OFF_CALL + b * 128 + wave * 16, 0);
  char* const peer_ws = p.ws_all[lane & 7];
  u32 call;

  if (b < 128) {
    const rsrc_t rsW = mk_rsrc(p.w, KS * NN);
    #pragma unroll
    for (int i = 0; i < 8; ++i) {
      const int row = (wave * 8 + i) * 8 + (lane >> 3);
      const int sw = ((row >> 1) & 3) | (((row >> 4) & 1) << 2);
      __builtin_amdgcn_raw_ptr_buffer_load_lds(rsW, (LDS void*)(smem + (wave * 8 + i) * 1024), 16, row * NN + (((lane & 7) ^ sw) << 4), b * 128, 0, 0);
    }
    asm volatile("s_waitcnt vmcnt(0)\n\ts_barrier" ::: "memory");
    call = __builtin_amdgcn_readfirstlane(tick) + 1u;
    const int cgl = wave & 1, kb = wave >> 1;
    u32x4 x[8];
    #pragma unroll
    for (int f = 0; f < 4; ++f)
      #pragma unroll
      for (int h = 0; h < 2; ++h)
        #pragma unroll
        for (int t = 0; t < 2; ++t) {
          const int row = kb * 128 + 64 * h + 16 * lg + 8 * t + (li >> 1);
          const int sw = ((row >> 1) & 3) | (((row >> 4) & 1) << 2);
          const i32x2 v = __builtin_amdgcn_ds_read_tr8_b64_v2i32((LDS i32x2*)(smem + row * 128 + (((cgl * 4 + f) ^ sw) << 4) + (li & 1) * 8));
          x[f * 2 + h][t * 2] = v.x; x[f * 2 + h][t * 2 + 1] = v.y;
        }
    const int dest = b >> 4, cg = (b & 15) * 2 + cgl;
    if (dest == rank) {
      store_raw_unit(x, rsL, OFF_WA + ((rank * 4 + kb) * 32 + cg) * IU_RAW, lane);
      vm_drain();
      cnt_add(p.ws, (16 + (b & 15)) * 5 + 4, lane);
    } else {
      export_unit(x, rsL, OFF_PAR + (call & 1) * PAR_BYTES, 256 + (dest * 32 + cg) * 4 + kb, smem + L_SCR + wave * 9 * KiB, lane);
      vm_drain();
      if (lane == dest)
        __hip_atomic_store((GLB u32*)(peer_ws + OFF_FIN + (rank * 384 + 256 + cg * 4 + kb) * 128), call, __ATOMIC_RELAXED, __HIP_MEMORY_SCOPE_SYSTEM);
    }
  } else if (wave < 2) {
    const int u = (b - 128) * 2 + wave, rg = u >> 2, kb = u & 3;
    const rsrc_t rsX = mk_rsrc(p.x, MM * KS);
    u32x4 x[8];
    #pragma unroll
    for (int j = 0; j < 8; ++j)
      x[j] = __builtin_amdgcn_raw_buffer_load_b128(rsX, ((j >> 1) * 16 + li) * KS + (j & 1) * 64 + lg * 16, rg * 64 * KS + kb * 128, 0);
    call = __builtin_amdgcn_readfirstlane(tick) + 1u;
    export_unit(x, rsL, OFF_PAR + (call & 1) * PAR_BYTES, u, smem + L_SCR + wave * 9 * KiB, lane);
    store_raw_unit(x, rsL, OFF_XA + ((rank * 4 + kb) * 64 + rg) * IU_RAW, lane);
    vm_drain();
    if (lane < NR && lane != rank)
      __hip_atomic_store((GLB u32*)(peer_ws + OFF_FIN + (rank * 384 + u) * 128), call, __ATOMIC_RELAXED, __HIP_MEMORY_SCOPE_SYSTEM);
    cnt_add(p.ws, (rg >> 2) * 5 + 4, lane);
  }
  call = __builtin_amdgcn_readfirstlane(tick) + 1u;
  const u32 par = OFF_PAR + (call & 1) * PAR_BYTES;
  const float scale = p.sx[0] * p.sw[0];
  const rsrc_t rsO = mk_rsrc(p.out, 512u * NN * 4u);
  STAMP(1);

  if (b >= NGEMM) {
    if (wave >= PULL_WAVES) return;
    const int pb = b - NGEMM, link = pb / PULL_WGS;
    const int src = (rank + 1 + link) & 7;
    constexpr int NWL = PULL_WGS * PULL_WAVES;
    const int v = wave * PULL_WGS + (pb % PULL_WGS);
    const u64 pw = (u64)peer_ws;
    const u32 pw_lo = (u32)__builtin_amdgcn_readlane((int)(u32)pw, src), pw_hi = (u32)__builtin_amdgcn_readlane((int)(u32)(pw >> 32), src);
    const rsrc_t rsP = mk_rsrc((const void*)(((u64)pw_hi << 32) | (u64)pw_lo), p.ws_bytes);
    LDS char* scr = smem + wave * 8 * KiB;
    int pend_cnt = -1;
    u32x2 pend_ex = {0u, 0u}; bool pend_exv = false, pend_any = false;
    auto flush = [&]() {
      if (pend_cnt >= 0) { cnt_add(p.ws, pend_cnt, lane); pend_cnt = -1; }
      if (pend_any) { if (pend_exv) __builtin_amdgcn_raw_buffer_store_b32(pend_ex.y, rsO, pend_ex.x, 0, OUT_AUX); pend_any = false; }
    };
    auto unit_x = [&](int i, int& kb, int& rg) -> bool {
      if (i < 240) { kb = i / 60; const int t = i - kb * 60; if (t >= 28) { rg = t - 28; return false; }
        rg = 8 * ((rank + 1 + (t >> 2)) & 7) + (t & 3); return true; }
      if (i < 352) { const int t = i - 240; kb = t / 28; const int u = t - kb * 28; rg = 8 * ((rank + 1 + (u >> 2)) & 7) + 4 + (u & 3); return true; }
      const int t = i - 352; kb = t >> 3; rg = 8 * rank + (t & 7); return true;
    };
    auto flag_off = [&](int i) -> u32 {
      if (i >= 384) return OFF_FOUT + (src * 256 + (i < 640 ? i - 384 : 0)) * 128;
      int kb, rg; const bool isx = unit_x(i, kb, rg);
      return OFF_FIN + (src * 384 + (isx ? 0 : 256) + rg * 4 + kb) * 128;
    };
    u32 nextf = 0;
    int i = v;
    for (; i < 384; i += NWL) {
      bool ok = true;
      if (nextf < call) { vm_drain(); flush(); ok = wait_ge(p.ws, flag_off(i), call, call); }
      asm volatile("" ::: "memory");
      nextf = 0;
      if (!ok) continue;
      int kb, rg; const bool isx = unit_x(i, kb, rg);
      const u32 uid = isx ? rg * 4 + kb : 256 + (rank * 32 + rg) * 4 + kb;
      const u32 dst = isx ? OFF_XA + ((src * 4 + kb) * 64 + rg) * IU_RAW : OFF_WA + ((src * 4 + kb) * 32 + rg) * IU_RAW;
      const u32 fo = flag_off(i + NWL);
      u32x4 x[8];
      bool raw = !ENC_IN;
      if (ENC_IN) {
        const u32 offE = par + EX_INE + uid * IU_ENC;
        u32x4 c[6]; u32 sd = 0, nf = 0;
        #pragma unroll
        for (int j = 0; j < 6; ++j) {
          if (j == 0) sd = __builtin_amdgcn_raw_buffer_load_b32(rsP, lane * 4, offE + 6144, AUX_SYS);
          c[j] = __builtin_amdgcn_raw_buffer_load_b128(rsP, lane * 16 + j * 1024, offE, AUX_SYS);
          if (j == 5) nf = ld_flag(p.ws, fo);
          if ((j + 1) % STEP_IN == 0 || j == 5) { vm_drain(); if (j < STEP_IN) flush(); }
        }
        nextf = __builtin_amdgcn_readfirstlane(nf);
        const u32 hdr = __builtin_amdgcn_readfirstlane(sd);
        raw = hdr >> 31;
        if (!raw) {
          const u32 cnt = hdr & 0xFFFFu, e4 = ((hdr >> 16) & 31u) * 0x04040404u;
          #pragma unroll
          for (int j = 0; j < 8; ++j) {
            const u32 p0 = c[(j * 3) >> 2][(j * 3) & 3], p1 = c[(j * 3 + 1) >> 2][(j * 3 + 1) & 3], p2 = c[(j * 3 + 2) >> 2][(j * 3 + 2) & 3];
            const u32 o3 = bfi(0x80808080u, p2 << 1, bfi(0x10101010u, p2 >> 1, bfi(0x0C0C0C0Cu, p1 >> 3, p0 >> 5)));
            const u32x4 y = {(p0 & 0x9F9F9F9Fu) + e4, (p1 & 0x9F9F9F9Fu) + e4, (p2 & 0x9F9F9F9Fu) + e4, (o3 & 0x9F9F9F9Fu) + e4};
            *(LDS u32x4*)(scr + j * 1024 + lane * 16) = y;
          }
          wave_sync();
          if (lane >= 1) {
            if (2u * lane - 2u < cnt) patch_in(scr, sd & 0xFFFFu);
            if (2u * lane - 1u < cnt) patch_in(scr, sd >> 16);
          }
          if (cnt > IN_CAP0) {
            for (int j = 1; j < 4; ++j) {
              const u32 s2 = __builtin_amdgcn_raw_buffer_load_b32(rsP, lane * 4 + j * 256, offE + 6144, AUX_SYS);
              const u32 id = j * 128 + 2 * lane - 2;
              if (id < cnt) patch_in(scr, s2 & 0xFFFFu);
              if (id + 1 < cnt) patch_in(scr, s2 >> 16);
            }
          }
          wave_sync();
          #pragma unroll
          for (int j = 0; j < 8; ++j) x[j] = *(LDS u32x4*)(scr + j * 1024 + lane * 16);
          store_raw_unit(x, rsL, dst, lane);
        }
      }
      if (raw) {
        const u32 offR = par + EX_INR + uid * IU_RAW;
        u32 nf = 0;
        #pragma unroll
        for (int j = 0; j < 8; ++j) {
          x[j] = __builtin_amdgcn_raw_buffer_load_b128(rsP, lane * 16 + j * 1024, offR, AUX_SYS);
          if (j == 7) nf = ld_flag(p.ws, fo);
          if ((j + 1) % STEP_IN == 0 || j == 7) {
            vm_drain(); if (j < STEP_IN) flush();
            #pragma unroll
            for (int t = j / STEP_IN * STEP_IN; t <= j; ++t) ST128(x[t], rsL, lane * 16 + t * 1024, dst, AUX_SYS);
          }
        }
        nextf = __builtin_amdgcn_readfirstlane(nf);
      }
      pend_cnt = (isx ? (rg >> 2) : 16 + (rg >> 1)) * 5 + kb;
    }
    STAMP(2);
    for (; i < 640; i += NWL) {
      bool ok = true;
      if (nextf < call) { vm_drain(); flush(); ok = wait_ge(p.ws, flag_off(i), call, call); }
      asm volatile("" ::: "memory");
      nextf = 0;
      if (i < 384 + NWL) STAMP(3);
      const int t = i - 384, hf = t >> 7, tn = (t >> 3) & 15, wv = t & 7;
      const u32 obase = ((hf * 256 + (wv >> 1) * 64) * NN + src * NCOL + tn * 128 + (wv & 1) * 64) * 4;
      const u32 olane = (li * NN + lg * 4) * 4;
      if (!ok) {
        const u32x4 nan4 = {0x7FC00000u, 0x7FC00000u, 0x7FC00000u, 0x7FC00000u};
        #pragma unroll
        for (int e = 0; e < 16; ++e) ST128(nan4, rsO, olane + (e & 3) * 64, obase + (e >> 2) * 16 * NN * 4, OUT_AUX);
        continue;
      }
      const u32 uid = rank * 256 + t;
      const u32 fo = flag_off(i + NWL);
      bool raw = !ENC_OUT;
      if (ENC_OUT) {
        const u32 offE = par + EX_OUE + uid * OU_ENC;
        u32x4 c[12]; u32x2 sg, sd; u32 nf = 0, cnt = 0; int g = 0;
        #pragma unroll
        for (int j = 0; j < 12; ++j) {
          if (j == 0) {
            sd = __builtin_amdgcn_raw_buffer_load_b64(rsP, li * 8, offE + 12800, AUX_SYS);
            sg = __builtin_amdgcn_raw_buffer_load_b64(rsP, lane * 8, offE + 12288, AUX_SYS);
          }
          c[j] = __builtin_amdgcn_raw_buffer_load_b128(rsP, lane * 16 + j * 1024, offE, AUX_SYS);
          if (j == 11) nf = ld_flag(p.ws, fo);
          if ((j + 1) % STEP_OUT == 0 || j == 11) {
            vm_drain();
            if (j < STEP_OUT) {
              flush();
              const u32 hdr = __builtin_amdgcn_readfirstlane(sd.x);
              raw = hdr >> 31; cnt = hdr & 0xFFFFu; g = (int)((hdr >> 16) & 0x1FFu) - 256;
              if (raw) break;
            }
            #pragma unroll
            for (int mi = (j / STEP_OUT * STEP_OUT) / 3; mi < (j + 1) / 3; ++mi)
              #pragma unroll
              for (int ni = 0; ni < 4; ++ni) {
                const int q = mi * 12 + ni * 3, e = (mi * 4 + ni) * 4;
                const u32 d0 = c[q >> 2][q & 3], d1 = c[(q + 1) >> 2][(q + 1) & 3], d2 = c[(q + 2) >> 2][(q + 2) & 3];
                const u32 lo[4] = {d0, __builtin_amdgcn_alignbit(d1, d0, 24), __builtin_amdgcn_alignbit(d2, d1, 16), d2 >> 8};
                u32x4 y;
                #pragma unroll
                for (int r = 0; r < 4; ++r) {
                  const u32 sm = (u32)__builtin_amdgcn_sbfe((int)sg[(e + r) >> 5], 31 - ((e + r) & 31), 1);
                  y[r] = __builtin_bit_cast(u32, unfix((int)bfi(0x00FFFFFFu, lo[r], sm), g) * scale);
                }
                ST128(y, rsO, olane + ni * 64, obase + mi * 16 * NN * 4, OUT_AUX);
              }
          }
        }
        if (!raw) {
          nextf = __builtin_amdgcn_readfirstlane(nf);
          auto ex_off = [&](u32 pos) -> u32 {
            return obase + (((pos >> 10) * 16 + ((pos >> 2) & 15)) * NN + ((pos >> 8) & 3) * 16 + ((pos >> 6) & 3) * 4 + (pos & 3)) * 4;
          };
          if (cnt > OUT_CAP0) {
            vm_drain();
            for (int j = 0; j < 2; ++j) {
              const u32x2 s2 = __builtin_amdgcn_raw_buffer_load_b64(rsP, lane * 8 + j * 512, offE + 12800, AUX_SYS);
              const u32 id = j * 64 + lane;
              if (id >= 1 && id <= cnt)
                __builtin_amdgcn_raw_buffer_store_b32(as_u(as_f(s2.y) * scale), rsO, ex_off(s2.x), 0, OUT_AUX);
            }
          } else if (cnt) {
            pend_any = true; pend_exv = lane >= 1 && (u32)lane <= cnt;
            pend_ex.x = ex_off(sd.x); pend_ex.y = as_u(as_f(sd.y) * scale);
          }
        }
      }
      if (raw) {
        const u32 offR = par + EX_OUR + uid * OU_RAW;
        u32x4 c[16]; u32 nf = 0;
        #pragma unroll
        for (int j = 0; j < 16; ++j) {
          c[j] = __builtin_amdgcn_raw_buffer_load_b128(rsP, lane * 16 + j * 1024, offR, AUX_SYS);
          if (j == 15) nf = ld_flag(p.ws, fo);
          if ((j + 1) % STEP_OUT == 0 || j == 15) {
            vm_drain(); if (j < STEP_OUT) flush();
            #pragma unroll
            for (int e = j / STEP_OUT * STEP_OUT; e <= j; ++e) {
              const f32x4 y = __builtin_bit_cast(f32x4, c[e]) * scale;
              ST128(__builtin_bit_cast(u32x4, y), rsO, olane + (e & 3) * 64, obase + (e >> 2) * 16 * NN * 4, OUT_AUX);
            }
          }
        }
        nextf = __builtin_amdgcn_readfirstlane(nf);
      }
    }
    vm_drain(); flush();
    STAMP(4);
    return;
  }

  constexpr int SLOTS = NGEMM / 8;
  const int xc = b & 7, slot = b >> 3, wm = wave >> 1, wn = wave & 1;
  for (int ti = slot; ti < 32; ti += SLOTS) {
    int tm, tn;
    if (ti < 28) { const int t = ti < 14 ? ti : ti - 14; tm = 2 * ((rank + 1 + t % 7) & 7) + (ti >= 14); tn = 2 * xc + t / 7; }
    else { tm = 2 * rank + ((ti - 28) >> 1); tn = 2 * xc + (ti & 1); }
    auto ready = [&](int grp) {
      const u32 idx = ((lane & 1) ? 16 + tn : tm) * 5 + grp;
      const u32 per = (lane & 1) ? (grp == 4 ? 8u : 14u) : (grp == 4 ? 16u : 28u);
      (void)wait_ge(p.ws, OFF_CNT + idx * 128, per * call, call);
    };
    auto issue = [&](int s, u32 st) {
      int ks;
      if (s < 4) ks = rank * 4 + s; else { const int t = s - 4, kb = t / 7; ks = ((rank + 1 + t - kb * 7) & 7) * 4 + kb; }
      const u32 xo = OFF_XA + ks * (64 * IU_RAW) + tm * (4 * IU_RAW) + wave * 4096;
      const u32 wo = OFF_WA + ks * (32 * IU_RAW) + tn * (2 * IU_RAW) + wave * 2048;
      LDS char* dx = smem + st + wave * 4096;
      LDS char* dw = smem + st + 32 * KiB + wave * 2048;
      __builtin_amdgcn_raw_ptr_buffer_load_lds(rsL, (LDS void*)dx, 16, lane * 16, xo, 0, GEMM_AUX);
      __builtin_amdgcn_raw_ptr_buffer_load_lds(rsL, (LDS void*)dx, 16, lane * 16, xo, 1024, GEMM_AUX);
      __builtin_amdgcn_raw_ptr_buffer_load_lds(rsL, (LDS void*)dx, 16, lane * 16, xo, 2048, GEMM_AUX);
      __builtin_amdgcn_raw_ptr_buffer_load_lds(rsL, (LDS void*)dx, 16, lane * 16, xo, 3072, GEMM_AUX);
      __builtin_amdgcn_raw_ptr_buffer_load_lds(rsL, (LDS void*)dw, 16, lane * 16, wo, 0, GEMM_AUX);
      __builtin_amdgcn_raw_ptr_buffer_load_lds(rsL, (LDS void*)dw, 16, lane * 16, wo, 1024, GEMM_AUX);
    };
    f32x4 acc[4][4];
    #pragma unroll
    for (int mi = 0; mi < 4; ++mi)
      #pragma unroll
      for (int ni = 0; ni < 4; ++ni) acc[mi][ni] = f32x4{0.f, 0.f, 0.f, 0.f};
    if (wave == 0) ready(4);
    asm volatile("s_waitcnt vmcnt(0) lgkmcnt(0)\n\ts_barrier" ::: "memory");
    if (ti == slot) { STAMP(2); STAMPC(14); }
    issue(0, 0); issue(1, STAGE);
    u32 st = 0;
    for (int s = 0; s < 32; ++s) {
      if (wave == 0 && (s == 2 || s == 9 || s == 16 || s == 23)) { if (ti == slot) STAMP(6 + (s - 2) / 7); ready((s - 2) / 7); if (ti == slot) STAMP(10 + (s - 2) / 7); }
      if (s < 31) asm volatile("s_waitcnt vmcnt(6) lgkmcnt(0)\n\ts_barrier" ::: "memory");
      else asm volatile("s_waitcnt vmcnt(0) lgkmcnt(0)\n\ts_barrier" ::: "memory");
      const u32 st2 = st == 0 ? 2 * STAGE : st - STAGE;
      if (s < 30) issue(s + 2, st2);
      LDS char* ax = smem + st + wm * 8192 + lane * 16;
      LDS char* aw = smem + st + 32 * KiB + wn * 8192 + lane * 16;
      i32x8 fx[4], fw[4];
      #pragma unroll
      for (int f = 0; f < 4; ++f) {
        const u32x4 a0 = *(LDS u32x4*)(aw + f * 2048), a1 = *(LDS u32x4*)(aw + f * 2048 + 1024);
        const u32x4 b0 = *(LDS u32x4*)(ax + f * 2048), b1 = *(LDS u32x4*)(ax + f * 2048 + 1024);
        fw[f] = __builtin_bit_cast(i32x8, __builtin_shufflevector(a0, a1, 0, 1, 2, 3, 4, 5, 6, 7));
        fx[f] = __builtin_bit_cast(i32x8, __builtin_shufflevector(b0, b1, 0, 1, 2, 3, 4, 5, 6, 7));
      }
      #pragma unroll
      for (int mi = 0; mi < 4; ++mi)
        #pragma unroll
        for (int ni = 0; ni < 4; ++ni)
          acc[mi][ni] = __builtin_amdgcn_mfma_scale_f32_16x16x128_f8f6f4(fw[ni], fx[mi], acc[mi][ni], 1, 1, 0, 0, 0, 0);
      st = st == 2 * STAGE ? 0 : st + STAGE;
    }
    if (ti == slot) { STAMP(3); STAMPC(15); }
    int lz = lane; asm volatile("" : "+v"(lz));
    const bool dead = __builtin_amdgcn_readfirstlane(__hip_atomic_load((GLB u32*)(p.ws + OFF_TMO), __ATOMIC_RELAXED, __HIP_MEMORY_SCOPE_AGENT)) == call;
    const int owner = tm >> 1;
    if (owner == rank) {
      const u32 obase = (((tm & 1) * 256 + wm * 64) * NN + rank * NCOL + tn * 128 + wn * 64) * 4, olane = ((lz & 15) * NN + (lz >> 4) * 4) * 4;
      const float sc = dead ? __builtin_nanf("") : scale;
      #pragma unroll
      for (int mi = 0; mi < 4; ++mi)
        #pragma unroll
        for (int ni = 0; ni < 4; ++ni)
          ST128(__builtin_bit_cast(u32x4, acc[mi][ni] * sc), rsO, olane + ni * 64, obase + mi * 16 * NN * 4, OUT_AUX);
    } else if (!dead) {
      const int t = ((tm & 1) * 16 + tn) * 8 + wave;
      const u32 uid = owner * 256 + t;
      bool raw = !ENC_OUT;
      if (ENC_OUT) {
        const u32 offE = par + EX_OUE + uid * OU_ENC;
        float sa = 0.f;
        #pragma unroll
        for (int mi = 0; mi < 4; ++mi)
          #pragma unroll
          for (int ni = 0; ni < 4; ++ni)
            #pragma unroll
            for (int r = 0; r < 4; ++r) sa += __builtin_fabsf(acc[mi][ni][r]);
        #pragma unroll
        for (int o = 32; o; o >>= 1) sa += __shfl_xor(sa, o);
        const int ex = (int)((__builtin_amdgcn_readfirstlane(__builtin_bit_cast(u32, sa * (3.75f / 4096.f))) >> 23) & 255u) - 126;
        const int g = 24 - ex;
        LDS u32x2* list = (LDS u32x2*)(smem + L_EPI + wave * 1024);
        u32 cnt = 0, sg[2] = {0u, 0u};
        #pragma unroll
        for (int mi = 0; mi < 4; ++mi) {
          u32 pk[12];
          #pragma unroll
          for (int ni = 0; ni < 4; ++ni) {
            u32 q[4];
            #pragma unroll
            for (int r = 0; r < 4; ++r) {
              const float vv = acc[mi][ni][r];
              const float tt = __builtin_amdgcn_fmed3f(__builtin_amdgcn_ldexpf(vv, g), -16777216.f, 16777215.f);
              const int iv = (int)tt;
              q[r] = (u32)iv;
              const int e = (mi * 4 + ni) * 4 + r;
              sg[e >> 5] = __builtin_amdgcn_alignbit(sg[e >> 5], (u32)iv, 31);
              const bool bad = __builtin_bit_cast(u32, unfix(iv, g)) != __builtin_bit_cast(u32, vv);
              const u64 bm = __ballot(bad);
              if (bm) {
                const u32 sl = cnt + 1 + __builtin_amdgcn_mbcnt_hi((u32)(bm >> 32), __builtin_amdgcn_mbcnt_lo((u32)bm, 0u));
                if (bad && sl <= OUT_CAP) list[sl] = u32x2{(u32)((mi * 4 + ni) * 256 + lz * 4 + r), __builtin_bit_cast(u32, vv)};
                cnt += __popcll(bm);
              }
            }
            pk[ni * 3 + 0] = __builtin_amdgcn_perm(q[1], q[0], 0x04020100u);
            pk[ni * 3 + 1] = __builtin_amdgcn_perm(q[2], q[1], 0x05040201u);
            pk[ni * 3 + 2] = __builtin_amdgcn_perm(q[3], q[2], 0x06050402u);
          }
          #pragma unroll
          for (int j = 0; j < 3; ++j)
            ST128((u32x4{pk[j * 4], pk[j * 4 + 1], pk[j * 4 + 2], pk[j * 4 + 3]}), rsL, lz * 16 + j * 1024, offE + mi * 3072, AUX_SYS);
        }
        __builtin_amdgcn_raw_buffer_store_b64(u32x2{sg[0], sg[1]}, rsL, lz * 8, offE + 12288, AUX_SYS);
        raw = cnt > OUT_CAP;
        wave_sync();
        u32x2 sd = list[lz];
        if (lz == 0) sd.x = raw ? 0x80000000u : (cnt | ((u32)(g + 256) << 16));
        if (lz < 16 || cnt > OUT_CAP0) __builtin_amdgcn_raw_buffer_store_b64(sd, rsL, lz * 8, offE + 12800, AUX_SYS);
        if (cnt > 63) __builtin_amdgcn_raw_buffer_store_b64(list[64 + lz], rsL, lz * 8 + 512, offE + 12800, AUX_SYS);
      }
      if (raw) {
        const u32 offR = par + EX_OUR + uid * OU_RAW;
        #pragma unroll
        for (int mi = 0; mi < 4; ++mi)
          #pragma unroll
          for (int ni = 0; ni < 4; ++ni)
            ST128(__builtin_bit_cast(u32x4, acc[mi][ni]), rsL, lz * 16 + ni * 1024, offR + mi * 4096, AUX_SYS);
      }
      vm_drain();
      if (lz == owner)
        __hip_atomic_store((GLB u32*)(peer_ws + OFF_FOUT + (rank * 256 + t) * 128), call, __ATOMIC_RELAXED, __HIP_MEMORY_SCOPE_SYSTEM);
    }
    if (ti == slot) STAMP(4); else STAMP(5);
  }
}

}

extern "C" void kernel_launch(int rank, int nranks, ncclComm_t comm, hipStream_t stream,
                              void* const* d_in, const int* in_sizes, int n_in,
                              void* d_out, int out_size,
                              const int* in_dtypes, int out_dtype,
                              void* const* d_out_all,
                              void* const* d_ws_all, size_t ws_bytes) {
  (void)comm; (void)in_sizes; (void)n_in; (void)out_size; (void)in_dtypes; (void)out_dtype; (void)d_out_all;
  if (nranks != NR || (unsigned)rank >= (unsigned)NR || ws_bytes < WS_NEED) { fprintf(stderr, "unexpected launch shape\n"); return; }
  static unsigned attr_set = 0;
  if (!(attr_set >> rank & 1u)) {
    HIP_CHECK(hipFuncSetAttribute((const void*)gemm_rs_kernel, hipFuncAttributeMaxDynamicSharedMemorySize, (int)LDS_BYTES));
    attr_set |= 1u << rank;
  }
  Params p;
  p.x = (const uint8_t*)d_in[0]; p.w = (const uint8_t*)d_in[1]; p.sx = (const float*)d_in[2]; p.sw = (const float*)d_in[3];
  p.out = (float*)d_out; p.ws_all = (char* const*)d_ws_all; p.ws = (char*)d_ws_all[rank]; p.rank = rank; p.ws_bytes = WS_NEED;
  hipLaunchKernelGGL(gemm_rs_kernel, dim3(NWG), dim3(NTHR), LDS_BYTES, stream, p);
  HIP_CHECK(hipGetLastError());
}
```

```cpp
#include <hip/hip_runtime.h>
#include <rccl/rccl.h>
#include <cstdio>
#include <cstdint>

#define HIP_CHECK(expr) do { hipError_t _e = (expr); if (_e != hipSuccess) { \
    fprintf(stderr, "HIP error %s at %s:%d\n", hipGetErrorString(_e), __FILE__, __LINE__); \
    return; } } while (0)

#define PULL_WGS 8
#define WIN 8
#define LOOK 3
#define OUT_AUX 17

namespace {

typedef unsigned u32;
typedef unsigned long long u64;
typedef unsigned u32x4 __attribute__((ext_vector_type(4)));
typedef unsigned u32x2 __attribute__((ext_vector_type(2)));
typedef int i32x2 __attribute__((ext_vector_type(2)));
typedef int i32x8 __attribute__((ext_vector_type(8)));
typedef float f32x4 __attribute__((ext_vector_type(4)));
typedef __amdgpu_buffer_rsrc_t rsrc_t;
#define LDS __attribute__((address_space(3)))
#define GLB __attribute__((address_space(1)))

constexpr int NR = 8, MM = 4096, NN = 16384, KS = 512, NCOL = 2048;
constexpr int NWG = 256, NTHR = 512;
constexpr int NPULL = 7 * PULL_WGS, NGEMM = NWG - NPULL;
constexpr int AUX_SYS = 17;
constexpr u64 DEADLINE_TICKS = 40000000ull;

constexpr u32 KiB = 1024, MiB = 1024 * 1024;
constexpr u32 OFF_CALL = 0;
constexpr u32 OFF_TMO = 32 * KiB;
constexpr u32 OFF_TIC = 33 * KiB;
constexpr u32 OFF_CNT = 36 * KiB;
constexpr u32 OFF_FIN = 64 * KiB;
constexpr u32 OFF_FOUT = OFF_FIN + 8 * 384 * 128;
constexpr u32 OFF_DBG = 768 * KiB;
constexpr u32 OFF_XA = 1 * MiB;
constexpr u32 OFF_WA = OFF_XA + 16 * MiB;
constexpr u32 OFF_PAR = OFF_WA + 8 * MiB;
constexpr u32 IU_ENC = 7 * KiB, IU_RAW = 8 * KiB;
constexpr u32 OU_ENC = 13824, OU_RAW = 16 * KiB;
constexpr u32 IU_SIDE = 6144, OU_SIDE = 12800;
constexpr u32 EX_INE = 0;
constexpr u32 EX_INR = EX_INE + 1280 * IU_ENC;
constexpr u32 EX_OUE = EX_INR + 1280 * IU_RAW;
constexpr u32 EX_OUR = EX_OUE + 2048 * OU_ENC;
constexpr u32 PAR_BYTES = EX_OUR + 2048 * OU_RAW;
constexpr u32 WS_NEED = OFF_PAR + 2 * PAR_BYTES;
static_assert(OFF_FOUT + 8 * 256 * 128 <= OFF_DBG, "flags");
static_assert(WS_NEED <= 257u * MiB, "workspace");
constexpr u32 IN_CAP0 = 126, IN_CAP = 510;
constexpr u32 OUT_CAP0 = 15, OUT_CAP = 127;

constexpr u32 STAGE = 48 * KiB;
constexpr u32 L_SCR = 64 * KiB;
constexpr u32 L_EPI = 3 * STAGE;
constexpr u32 NSLOT = 8, SLOT = 13 * KiB, NCONS = 6;
constexpr u32 L_PSCR = NSLOT * SLOT;
constexpr u32 L_CTL = L_PSCR + NCONS * 8 * KiB;
constexpr u32 LDS_BYTES = L_CTL + 256;
static_assert(L_EPI + 8 * KiB <= LDS_BYTES && L_SCR + 72 * KiB <= LDS_BYTES && LDS_BYTES <= 160 * KiB, "lds");

struct Params {
  const uint8_t* x; const uint8_t* w; const float* sx; const float* sw;
  float* out; char* const* ws_all; char* ws; int rank; u32 ws_bytes;
};

__device__ __forceinline__ rsrc_t mk_rsrc(const void* p, u32 bytes) {
  u64 a = (u64)p;
  u32 lo = __builtin_amdgcn_readfirstlane((u32)a), hi = __builtin_amdgcn_readfirstlane((u32)(a >> 32));
  return __builtin_amdgcn_make_buffer_rsrc((void*)(((u64)hi << 32) | lo), 0, (int)bytes, 0x00020000);
}
#define ST128(v, rs, voff, soff, aux) __builtin_amdgcn_raw_buffer_store_b128((v), (rs), (int)((u32)(voff) + (u32)(soff)), 0, (aux))
__device__ __forceinline__ float as_f(u32 v) { return __builtin_bit_cast(float, v); }
__device__ __forceinline__ u32 as_u(float v) { return __builtin_bit_cast(u32, v); }
__device__ __forceinline__ void vm_drain() { asm volatile("s_waitcnt vmcnt(0)" ::: "memory"); }
__device__ __forceinline__ void wave_sync() {
  asm volatile("s_waitcnt lgkmcnt(0)" ::: "memory");
}
__device__ __forceinline__ u32 bfi(u32 m, u32 a, u32 b) { return (m & a) | (~m & b); }
template <int CTRL, int ROWS> __device__ __forceinline__ u32 dpp0(u32 v) { return (u32)__builtin_amdgcn_update_dpp(0, (int)v, CTRL, ROWS, 0xf, false); }
__device__ __forceinline__ u32 wave_scan_u32(u32 v) {
  v += dpp0<0x111, 0xf>(v); v += dpp0<0x112, 0xf>(v); v += dpp0<0x114, 0xf>(v); v += dpp0<0x118, 0xf>(v);
  v += dpp0<0x142, 0xa>(v); v += dpp0<0x143, 0xc>(v);
  return v;
}
__device__ __forceinline__ float wave_sum_f32(float v) {
  v += as_f(dpp0<0x111, 0xf>(as_u(v))); v += as_f(dpp0<0x112, 0xf>(as_u(v))); v += as_f(dpp0<0x114, 0xf>(as_u(v)));
  v += as_f(dpp0<0x118, 0xf>(as_u(v))); v += as_f(dpp0<0x142, 0xa>(as_u(v))); v += as_f(dpp0<0x143, 0xc>(as_u(v)));
  return as_f((u32)__builtin_amdgcn_readlane((int)as_u(v), 63));
}
__device__ __forceinline__ u32 ctl_ld(u32 addr) {
  u32 v; asm volatile("ds_read_b32 %0, %1\n\ts_waitcnt lgkmcnt(0)" : "=&v"(v) : "v"(addr) : "memory");
  return (u32)__builtin_amdgcn_readfirstlane((int)v);
}
__device__ __forceinline__ void ctl_st(u32 addr, u32 v) { asm volatile("ds_write_b32 %0, %1" :: "v"(addr), "v"(v) : "memory"); }
__device__ __forceinline__ u32 ld_flag(const char* ws, u32 off) {
  return __hip_atomic_load((const GLB u32*)(ws + off), __ATOMIC_RELAXED, __HIP_MEMORY_SCOPE_SYSTEM);
}
__device__ __forceinline__ bool wait_ge(char* ws, u32 off, u32 target, u32 call) {
  if (__all(ld_flag(ws, off) >= target)) return true;
  const u64 t0 = __builtin_amdgcn_s_memrealtime();
  GLB u32* tmo = (GLB u32*)(ws + OFF_TMO);
  for (;;) {
    __builtin_amdgcn_s_sleep(4);
    if (__all(ld_flag(ws, off) >= target)) return true;
    if (__hip_atomic_load(tmo, __ATOMIC_RELAXED, __HIP_MEMORY_SCOPE_AGENT) == call ||
        __builtin_amdgcn_s_memrealtime() - t0 > DEADLINE_TICKS) {
      __hip_atomic_store(tmo, call, __ATOMIC_RELAXED, __HIP_MEMORY_SCOPE_AGENT);
      return false;
    }
  }
}
__device__ __forceinline__ void cnt_add(char* ws, u32 idx, int lane) {
  if (lane == 0) __hip_atomic_fetch_add((GLB u32*)(ws + OFF_CNT + idx * 128), 1u, __ATOMIC_RELAXED, __HIP_MEMORY_SCOPE_AGENT);
}
#define STAMPC(slot) do {} while (0)
#define STAMP(slot) do {} while (0)

__device__ __forceinline__ void store_raw_unit(const u32x4 (&x)[8], rsrc_t rs, u32 off, int lane) {
  #pragma unroll
  for (int j = 0; j < 8; ++j) ST128(x[j], rs, lane * 16 + j * 1024, off, AUX_SYS);
}

__device__ __forceinline__ void export_unit(const u32x4 (&x)[8], rsrc_t rs, u32 par, u32 uid, LDS char* scr, int lane) {
  const u32 offE = par + EX_INE + uid * IU_ENC, offR = par + EX_INR + uid * IU_RAW;
  u32 s = 0;
  #pragma unroll
  for (int j = 0; j < 8; ++j) {
    *(LDS u32x4*)(scr + j * 1024 + lane * 16) = x[j];
    s = __builtin_amdgcn_sad_u8(x[j][j & 3] & 0x7C7C7C7Cu, 0u, s);
  }
  s = (u32)__builtin_amdgcn_readlane((int)wave_scan_u32(s), 63);
  int e0 = ((int)s - 34243) >> 13;
  e0 = e0 < 0 ? 0 : (e0 > 24 ? 24 : e0);
  const u32 e4 = (u32)e0 * 0x04040404u;
  u32 mk[4] = {0, 0, 0, 0};
  u32 pk[24];
  #pragma unroll
  for (int j = 0; j < 8; ++j) {
    u32 o[4];
    #pragma unroll
    for (int c = 0; c < 4; ++c) {
      const u32 u = (x[j][c] | 0x80808080u) - e4;
      o[c] = bfi(0x1F1F1F1Fu, u, x[j][c]);
      const u32 t = (((u >> 1) ^ 0x40404040u) & 0x70707070u) + 0x70707070u;
      const int d = j * 4 + c;
      mk[d >> 3] = bfi(0x80808080u >> (d & 7), t >> (d & 7), mk[d >> 3]);
    }
    pk[j * 3 + 0] = bfi(0x60606060u, o[3] << 5, o[0]);
    pk[j * 3 + 1] = bfi(0x60606060u, o[3] << 3, o[1]);
    pk[j * 3 + 2] = bfi(0x60606060u, bfi(0x40404040u, o[3] >> 1, o[3] << 1), o[2]);
  }
  #pragma unroll
  for (int j = 0; j < 6; ++j) {
    const u32x4 v = {pk[j * 4], pk[j * 4 + 1], pk[j * 4 + 2], pk[j * 4 + 3]};
    ST128(v, rs, lane * 16 + j * 1024, offE, AUX_SYS);
  }
  const u32 mine = __popc(mk[0]) + __popc(mk[1]) + __popc(mk[2]) + __popc(mk[3]);
  const u32 inc = wave_scan_u32(mine);
  const u32 total = (u32)__builtin_amdgcn_readlane((int)inc, 63);
  if (total <= IN_CAP) {
    LDS unsigned short* list = (LDS unsigned short*)(scr + 8192);
    u32 slot = 2 + inc - mine;
    #pragma unroll
    for (int q = 0; q < 4; ++q) {
      u32 m = mk[q];
      while (m) {
        const int bp = __builtin_ctz(m); m &= m - 1;
        const int d = q * 8 + 7 - (bp & 7);
        list[slot++] = (unsigned short)((d >> 2) * 1024 + lane * 16 + (d & 3) * 4 + (bp >> 3));
      }
    }
    wave_sync();
    for (u32 j = 0; j < 4; ++j) {
      const u32 pp = *(LDS u32*)(scr + 8192 + j * 256 + lane * 4);
      const u32 p0 = pp & 8191u, p1 = (pp >> 16) & 8191u;
      const u32 b0 = *(LDS unsigned char*)(scr + p0), b1 = *(LDS unsigned char*)(scr + p1);
      u32 sd = (p0 << 3) | ((128u + (b0 & 127u) - 4u * (u32)e0) >> 5) | (p1 << 19) | (((128u + (b1 & 127u) - 4u * (u32)e0) >> 5) << 16);
      if (j == 0 && lane == 0) sd = total | ((u32)e0 << 16);
      __builtin_amdgcn_raw_buffer_store_b32(sd, rs, lane * 4 + j * 256, offE + IU_SIDE, AUX_SYS);
      if (total <= IN_CAP0 + j * 128) break;
    }
  } else {
    store_raw_unit(x, rs, offR, lane);
    if (lane == 0) __builtin_amdgcn_raw_buffer_store_b32(0x80000000u, rs, 0, offE + IU_SIDE, AUX_SYS);
  }
}

__device__ __forceinline__ void patch_in(LDS char* scr, u32 ent) {
  const u32 pos = ent >> 3;
  __hip_atomic_fetch_add((LDS u32*)(scr + (pos & ~3u)), (((ent & 7u) - 4u) << 5) << ((pos & 3u) * 8u), __ATOMIC_RELAXED, __HIP_MEMORY_SCOPE_WORKGROUP);
}

__device__ __forceinline__ float unfix(int i, int g) { return __builtin_amdgcn_ldexpf((float)i, -g); }

__global__ void __launch_bounds__(NTHR, 2) gemm_rs_kernel(Params p) {
  extern __shared__ __attribute__((aligned(16))) char smem_[];
  LDS char* smem = (LDS char*)smem_;
  const int tid = threadIdx.x, lane = tid & 63, wave = __builtin_amdgcn_readfirstlane(tid >> 6);
  const int b = blockIdx.x, rank = p.rank;
  const int lg = lane >> 4, li = lane & 15;
  STAMP(0);
  const rsrc_t rsL = mk_rsrc(p.ws, p.ws_bytes);
  const u32 tick = (u32)__builtin_amdgcn_raw_ptr_buffer_atomic_add_i32(1, rsL, lane == 0 ? 0 : (int)0xF0000000u, OFF_CALL + b * 128 + wave * 16, 0);
  char* const peer_ws = p.ws_all[lane & 7];
  u32 call;

  if (b < 128) {
    const rsrc_t rsW = mk_rsrc(p.w, KS * NN);
    #pragma unroll
    for (int i = 0; i < 8; ++i) {
      const int row = (wave * 8 + i) * 8 + (lane >> 3);
      const int sw = ((row >> 1) & 3) | (((row >> 4) & 1) << 2);
      __builtin_amdgcn_raw_ptr_buffer_load_lds(rsW, (LDS void*)(smem + (wave * 8 + i) * 1024), 16, row * NN + (((lane & 7) ^ sw) << 4), b * 128, 0, 0);
    }
    asm volatile("s_waitcnt vmcnt(0)\n\ts_barrier" ::: "memory");
    call = __builtin_amdgcn_readfirstlane(tick) + 1u;
    const int cgl = wave & 1, kb = wave >> 1;
    u32x4 x[8];
    #pragma unroll
    for (int f = 0; f < 4; ++f)
      #pragma unroll
      for (int h = 0; h < 2; ++h)
        #pragma unroll
        for (int t = 0; t < 2; ++t) {
          const int row = kb * 128 + 64 * h + 16 * lg + 8 * t + (li >> 1);
          const int sw = ((row >> 1) & 3) | (((row >> 4) & 1) << 2);
          const i32x2 v = __builtin_amdgcn_ds_read_tr8_b64_v2i32((LDS i32x2*)(smem + row * 128 + (((cgl * 4 + f) ^ sw) << 4) + (li & 1) * 8));
          x[f * 2 + h][t * 2] = v.x; x[f * 2 + h][t * 2 + 1] = v.y;
        }
    const int dest = b >> 4, cg = (b & 15) * 2 + cgl;
    if (dest == rank) {
      store_raw_unit(x, rsL, OFF_WA + ((rank * 4 + kb) * 32 + cg) * IU_RAW, lane);
      vm_drain();
      cnt_add(p.ws, (16 + (b & 15)) * 5 + 4, lane);
    } else {
      export_unit(x, rsL, OFF_PAR + (call & 1) * PAR_BYTES, 256 + (dest * 32 + cg) * 4 + kb, smem + L_SCR + wave * 9 * KiB, lane);
      vm_drain();
      if (lane == dest)
        __hip_atomic_store((GLB u32*)(peer_ws + OFF_FIN + (rank * 384 + 256 + cg * 4 + kb) * 128), call, __ATOMIC_RELAXED, __HIP_MEMORY_SCOPE_SYSTEM);
    }
  } else if (b < 192 && wave < 4) {
    const int u = (b - 128) * 4 + wave, rg = u >> 2, kb = u & 3;
    const rsrc_t rsX = mk_rsrc(p.x, MM * KS);
    u32x4 x[8];
    #pragma unroll
    for (int j = 0; j < 8; ++j)
      x[j] = __builtin_amdgcn_raw_buffer_load_b128(rsX, ((j >> 1) * 16 + li) * KS + (j & 1) * 64 + lg * 16, rg * 64 * KS + kb * 128, 0);
    call = __builtin_amdgcn_readfirstlane(tick) + 1u;
    STAMP(6);
    export_unit(x, rsL, OFF_PAR + (call & 1) * PAR_BYTES, u, smem + L_SCR + wave * 9 * KiB, lane);
    STAMP(7);
    vm_drain();
    if (lane < NR && lane != rank)
      __hip_atomic_store((GLB u32*)(peer_ws + OFF_FIN + (rank * 384 + u) * 128), call, __ATOMIC_RELAXED, __HIP_MEMORY_SCOPE_SYSTEM);
    STAMP(8);
    store_raw_unit(x, rsL, OFF_XA + ((rank * 4 + kb) * 64 + rg) * IU_RAW, lane);
    vm_drain();
    cnt_add(p.ws, (rg >> 2) * 5 + 4, lane);
  }
  call = __builtin_amdgcn_readfirstlane(tick) + 1u;
  const u32 par = OFF_PAR + (call & 1) * PAR_BYTES;
  const float scale = p.sx[0] * p.sw[0];
  const rsrc_t rsO = mk_rsrc(p.out, 512u * NN * 4u);
  STAMP(1);

  if (b >= NGEMM) {
    const int pb = b - NGEMM, link = pb / PULL_WGS, q = pb % PULL_WGS;
    const int src = (rank + 1 + link) & 7;
    const u64 pw = (u64)peer_ws;
    const u32 pw_lo = (u32)__builtin_amdgcn_readlane((int)(u32)pw, src), pw_hi = (u32)__builtin_amdgcn_readlane((int)(u32)(pw >> 32), src);
    const rsrc_t rsP = mk_rsrc((const void*)(((u64)pw_hi << 32) | (u64)pw_lo), p.ws_bytes);
    const u32 ctl = (u32)(uintptr_t)(smem + L_CTL);
    const u32 C_READY = ctl, C_LANDED = ctl + 4, C_ISSUED = ctl + 8, C_CLOSED = ctl + 12, C_REL = ctl + 16, C_Q = ctl + 64;
    if (wave == 0) ctl_st(ctl + lane * 4, 0u);
    asm volatile("s_waitcnt lgkmcnt(0)\n\ts_barrier" ::: "memory");
    auto unit_x = [&](int i, int& kb, int& rg) -> bool {
      if (i < 240) { kb = i / 60; const int t = i - kb * 60; if (t >= 28) { rg = t - 28; return false; }
        rg = 8 * ((rank + 1 + (t >> 2)) & 7) + (t & 3); return true; }
      if (i < 352) { const int t = i - 240; kb = t / 28; const int u = t - kb * 28; rg = 8 * ((rank + 1 + (u >> 2)) & 7) + 4 + (u & 3); return true; }
      const int t = i - 352; kb = t >> 3; rg = 8 * rank + (t & 7); return true;
    };
    if (wave == 0) {
      int g = 0, pub = 0;
      #define LANDED(v) do { if ((v) > pub) { pub = (v); ctl_st(C_LANDED, (u32)pub); } } while (0)
      #define PIECE(SZ, VOFF, OFF) do { __builtin_amdgcn_raw_ptr_buffer_load_lds(rsP, (LDS void*)(dst + (OFF)), SZ, VOFF, so + (OFF), 0, AUX_SYS); \
        asm volatile("s_waitcnt vmcnt(%0)" :: "i"(WIN - 1) : "memory"); ++g; LANDED(g - (WIN - 1)); } while (0)
      bool seen_out = false;
      for (u32 n = 0;; ++n) {
        if (ctl_ld(C_READY) <= n) {
          vm_drain(); LANDED(g);
          bool fin = false;
          while (ctl_ld(C_READY) <= n) { if (ctl_ld(C_CLOSED)) { fin = ctl_ld(C_READY) <= n; break; } __builtin_amdgcn_s_sleep(1); }
          if (fin) break;
        }
        const u32 ent = ctl_ld(C_Q + (n & 15) * 4);
        ctl_st(C_ISSUED, n + 1);
        if (ent >> 31) continue;
        const u32 need = n >= NSLOT ? n - NSLOT + 1 : 0, ra = C_REL + (n % NSLOT) * 4;
        if (ctl_ld(ra) < need) {
          vm_drain(); LANDED(g);
          while (ctl_ld(ra) < need) __builtin_amdgcn_s_sleep(1);
        }
        const int i = ent & 1023;
        if (n == 0) STAMP(6);
        if (i >= 384 && !seen_out) { seen_out = true; STAMP(3); }
        LDS char* dst = smem + (n % NSLOT) * SLOT;
        if (i < 384) {
          int kb, rg; const bool isx = unit_x(i, kb, rg);
          const u32 so = par + EX_INE + (isx ? rg * 4 + kb : 256 + (rank * 32 + rg) * 4 + kb) * IU_ENC;
          PIECE(4, lane * 4, IU_SIDE);
          #pragma unroll
          for (int j = 0; j < 6; ++j) PIECE(16, lane * 16, j * 1024);
          STAMP(2);
        } else {
          const u32 so = par + EX_OUE + (rank * 256 + i - 384) * OU_ENC;
          if (lane < 40) __builtin_amdgcn_raw_ptr_buffer_load_lds(rsP, (LDS void*)(dst + 12288), 16, lane * 16, so + 12288, 0, AUX_SYS);
          asm volatile("s_waitcnt vmcnt(%0)" :: "i"(WIN - 1) : "memory"); ++g; LANDED(g - (WIN - 1));
          #pragma unroll
          for (int j = 0; j < 12; ++j) PIECE(16, lane * 16, j * 1024);
        }
      }
      vm_drain(); LANDED(g);
      #undef PIECE
      #undef LANDED
      STAMP(7);
      return;
    }
    if (wave == 1) {
      const u32 tic = OFF_TIC + ((call & 1) * 8 + link) * 128;
      const int only0 = lane == 0 ? 0 : (int)0xF0000000u;
      if (q == 0 && lane == 0)
        __hip_atomic_store((GLB u32*)(p.ws + OFF_TIC + ((~call & 1) * 8 + link) * 128), 0u, __ATOMIC_RELAXED, __HIP_MEMORY_SCOPE_AGENT);
      u32 n = 0, endp = 0; bool ok = true;
      int tnext = __builtin_amdgcn_raw_ptr_buffer_atomic_add_i32(1, rsL, only0, tic, 0);
      for (;;) {
        const u32 t = (u32)__builtin_amdgcn_readfirstlane(tnext);
        if (t >= 448) break;
        while (n >= ctl_ld(C_ISSUED) + LOOK) __builtin_amdgcn_s_sleep(1);
        tnext = __builtin_amdgcn_raw_ptr_buffer_atomic_add_i32(1, rsL, only0, tic, 0);
        const int i = t < 192 ? 2 * t + (lane & 1) : 192 + t;
        u32 fo;
        if (i >= 384) fo = OFF_FOUT + (src * 256 + i - 384) * 128;
        else { int kb, rg; const bool isx = unit_x(i, kb, rg); fo = OFF_FIN + (src * 384 + (isx ? 0 : 256) + rg * 4 + kb) * 128; }
        ok = ok && wait_ge(p.ws, fo, call, call);
        const u32 k = t < 192 ? (lane & 1) : 0, per = ok ? (t < 192 ? 7 : 13) : 0;
        ctl_st(C_Q + ((n + k) & 15) * 4, (u32)i | ((endp + per * (k + 1)) << 10) | (ok ? 0u : 0x80000000u));
        n += t < 192 ? 2 : 1; endp += t < 192 ? 2 * per : per;
        asm volatile("s_waitcnt lgkmcnt(0)" ::: "memory");
        ctl_st(C_READY, n);
      }
      asm volatile("s_waitcnt lgkmcnt(0)" ::: "memory");
      ctl_st(C_CLOSED, 1u);
      return;
    }
    LDS char* scr = smem + L_PSCR + (wave - 2) * 8 * KiB;
    for (u32 n = wave - 2;; n += NCONS) {
      bool fin = false;
      while (ctl_ld(C_READY) <= n) { if (ctl_ld(C_CLOSED)) { fin = ctl_ld(C_READY) <= n; break; } __builtin_amdgcn_s_sleep(1); }
      if (fin) break;
      const u32 ent = ctl_ld(C_Q + (n & 15) * 4);
      const bool ok = !(ent >> 31);
      const int i = ent & 1023;
      if (ok) { while (ctl_ld(C_LANDED) < ((ent >> 10) & 0x1FFFu)) __builtin_amdgcn_s_sleep(1); }
      LDS char* sl = smem + (n % NSLOT) * SLOT;
      if (i < 384) {
        if (!ok) continue;
        u32x4 c[6];
        #pragma unroll
        for (int j = 0; j < 6; ++j) c[j] = *(LDS u32x4*)(sl + j * 1024 + lane * 16);
        const u32 sd = *(LDS u32*)(sl + IU_SIDE + lane * 4);
        wave_sync();
        ctl_st(C_REL + (n % NSLOT) * 4, n + 1);
        int kb, rg; const bool isx = unit_x(i, kb, rg);
        const u32 uid = isx ? rg * 4 + kb : 256 + (rank * 32 + rg) * 4 + kb;
        const u32 dst = isx ? OFF_XA + ((src * 4 + kb) * 64 + rg) * IU_RAW : OFF_WA + ((src * 4 + kb) * 32 + rg) * IU_RAW;
        const u32 hdr = (u32)__builtin_amdgcn_readfirstlane((int)sd);
        u32x4 x[8];
        if (!(hdr >> 31)) {
          const u32 cnt = hdr & 0xFFFFu, e4 = ((hdr >> 16) & 31u) * 0x04040404u;
          #pragma unroll
          for (int j = 0; j < 8; ++j) {
            const u32 p0 = c[(j * 3) >> 2][(j * 3) & 3], p1 = c[(j * 3 + 1) >> 2][(j * 3 + 1) & 3], p2 = c[(j * 3 + 2) >> 2][(j * 3 + 2) & 3];
            const u32 o3 = bfi(0x80808080u, p2 << 1, bfi(0x10101010u, p2 >> 1, bfi(0x0C0C0C0Cu, p1 >> 3, p0 >> 5)));
            const u32x4 y = {(p0 & 0x9F9F9F9Fu) + e4, (p1 & 0x9F9F9F9Fu) + e4, (p2 & 0x9F9F9F9Fu) + e4, (o3 & 0x9F9F9F9Fu) + e4};
            *(LDS u32x4*)(scr + j * 1024 + lane * 16) = y;
          }
          wave_sync();
          if (lane >= 1) {
            if (2u * lane - 2u < cnt) patch_in(scr, sd & 0xFFFFu);
            if (2u * lane - 1u < cnt) patch_in(scr, sd >> 16);
          }
          if (cnt > IN_CAP0) {
            for (int j = 1; j < 4; ++j) {
              const u32 s2 = __builtin_amdgcn_raw_buffer_load_b32(rsP, lane * 4 + j * 256, par + EX_INE + uid * IU_ENC + IU_SIDE, AUX_SYS);
              const u32 id = j * 128 + 2 * lane - 2;
              if (id < cnt) patch_in(scr, s2 & 0xFFFFu);
              if (id + 1 < cnt) patch_in(scr, s2 >> 16);
            }
          }
          wave_sync();
          #pragma unroll
          for (int j = 0; j < 8; ++j) x[j] = *(LDS u32x4*)(scr + j * 1024 + lane * 16);
        } else {
          #pragma unroll
          for (int j = 0; j < 8; ++j) x[j] = __builtin_amdgcn_raw_buffer_load_b128(rsP, lane * 16 + j * 1024, par + EX_INR + uid * IU_RAW, AUX_SYS);
        }
        store_raw_unit(x, rsL, dst, lane);
        vm_drain();
        cnt_add(p.ws, (isx ? (rg >> 2) : 16 + (rg >> 1)) * 5 + kb, lane);
        continue;
      }
      const int t = i - 384, hf = t >> 7, tn = (t >> 3) & 15, wv = t & 7;
      const u32 obase = ((hf * 256 + (wv >> 1) * 64) * NN + src * NCOL + tn * 128 + (wv & 1) * 64) * 4;
      const u32 olane = (li * NN + lg * 4) * 4;
      if (!ok) {
        const u32x4 nan4 = {0x7FC00000u, 0x7FC00000u, 0x7FC00000u, 0x7FC00000u};
        #pragma unroll
        for (int e = 0; e < 16; ++e) ST128(nan4, rsO, olane + (e & 3) * 64, obase + (e >> 2) * 16 * NN * 4, OUT_AUX);
        continue;
      }
      u32x4 c[12];
      #pragma unroll
      for (int j = 0; j < 12; ++j) c[j] = *(LDS u32x4*)(sl + j * 1024 + lane * 16);
      const u32x2 sg = *(LDS u32x2*)(sl + 12288 + lane * 8), sd = *(LDS u32x2*)(sl + OU_SIDE + li * 8);
      wave_sync();
      ctl_st(C_REL + (n % NSLOT) * 4, n + 1);
      const u32 uid = rank * 256 + t;
      const u32 hdr = (u32)__builtin_amdgcn_readfirstlane((int)sd.x);
      if (!(hdr >> 31)) {
        const u32 cnt = hdr & 0xFFFFu; const int g = (int)((hdr >> 16) & 0x1FFu) - 256;
        #pragma unroll
        for (int mi = 0; mi < 4; ++mi)
          #pragma unroll
          for (int ni = 0; ni < 4; ++ni) {
            const int k = mi * 12 + ni * 3, e = (mi * 4 + ni) * 4;
            const u32 d0 = c[k >> 2][k & 3], d1 = c[(k + 1) >> 2][(k + 1) & 3], d2 = c[(k + 2) >> 2][(k + 2) & 3];
            const u32 lo[4] = {d0, __builtin_amdgcn_alignbit(d1, d0, 24), __builtin_amdgcn_alignbit(d2, d1, 16), d2 >> 8};
            u32x4 y;
            #pragma unroll
            for (int r = 0; r < 4; ++r) {
              const u32 sm = (u32)__builtin_amdgcn_sbfe((int)sg[(e + r) >> 5], 31 - ((e + r) & 31), 1);
              y[r] = as_u(unfix((int)bfi(0x00FFFFFFu, lo[r], sm), g) * scale);
            }
            ST128(y, rsO, olane + ni * 64, obase + mi * 16 * NN * 4, OUT_AUX);
          }
        if (cnt) {
          auto ex_off = [&](u32 pos) -> u32 {
            return obase + ((((pos >> 10) & 3) * 16 + ((pos >> 2) & 15)) * NN + ((pos >> 8) & 3) * 16 + ((pos >> 6) & 3) * 4 + (pos & 3)) * 4;
          };
          vm_drain();
          if (cnt <= OUT_CAP0) {
            if (lane >= 1 && (u32)lane <= cnt) __builtin_amdgcn_raw_buffer_store_b32(as_u(as_f(sd.y) * scale), rsO, ex_off(sd.x), 0, OUT_AUX);
          } else {
            for (int j = 0; j < 2; ++j) {
              const u32x2 s2 = __builtin_amdgcn_raw_buffer_load_b64(rsP, lane * 8 + j * 512, par + EX_OUE + uid * OU_ENC + OU_SIDE, AUX_SYS);
              const u32 id = j * 64 + lane;
              if (id >= 1 && id <= cnt) __builtin_amdgcn_raw_buffer_store_b32(as_u(as_f(s2.y) * scale), rsO, ex_off(s2.x), 0, OUT_AUX);
            }
          }
        }
      } else {
        #pragma unroll
        for (int mi = 0; mi < 4; ++mi) {
          u32x4 r4[4];
          #pragma unroll
          for (int ni = 0; ni < 4; ++ni) r4[ni] = __builtin_amdgcn_raw_buffer_load_b128(rsP, lane * 16 + ni * 1024, par + EX_OUR + uid * OU_RAW + mi * 4096, AUX_SYS);
          #pragma unroll
          for (int ni = 0; ni < 4; ++ni) {
            const f32x4 y = __builtin_bit_cast(f32x4, r4[ni]) * scale;
            ST128(__builtin_bit_cast(u32x4, y), rsO, olane + ni * 64, obase + mi * 16 * NN * 4, OUT_AUX);
          }
        }
      }
    }
    STAMP(4);
    return;
  }

  constexpr int SLOTS = NGEMM / 8;
  const int xc = b & 7, slot = b >> 3, wm = wave >> 1, wn = wave & 1;
  for (int ti = slot; ti < 32; ti += SLOTS) {
    int tm, tn;
    if (ti < 28) { const int t = ti < 14 ? ti : ti - 14; tm = 2 * ((rank + 1 + t % 7) & 7) + (ti >= 14); tn = 2 * xc + t / 7; }
    else { tm = 2 * rank + ((ti - 28) >> 1); tn = 2 * xc + (ti & 1); }
    const u32 deadw = (u32)(uintptr_t)(smem + L_CTL);
    auto ready = [&](int grp) {
      const u32 idx = ((lane & 1) ? 16 + tn : tm) * 5 + grp;
      const u32 per = (lane & 1) ? (grp == 4 ? 8u : 14u) : (grp == 4 ? 16u : 28u);
      if (!wait_ge(p.ws, OFF_CNT + idx * 128, per * call, call)) ctl_st(deadw, 1u);
    };
    auto issue = [&](int s, u32 st) {
      int ks;
      if (s < 4) ks = rank * 4 + s; else { const int t = s - 4, kb = t / 7; ks = ((rank + 1 + t - kb * 7) & 7) * 4 + kb; }
      const u32 xo = OFF_XA + ks * (64 * IU_RAW) + tm * (4 * IU_RAW) + wave * 4096;
      const u32 wo = OFF_WA + ks * (32 * IU_RAW) + tn * (2 * IU_RAW) + wave * 2048;
      LDS char* dx = smem + st + wave * 4096;
      LDS char* dw = smem + st + 32 * KiB + wave * 2048;
      __builtin_amdgcn_raw_ptr_buffer_load_lds(rsL, (LDS void*)dx, 16, lane * 16, xo, 0, AUX_SYS);
      __builtin_amdgcn_raw_ptr_buffer_load_lds(rsL, (LDS void*)dx, 16, lane * 16, xo, 1024, AUX_SYS);
      __builtin_amdgcn_raw_ptr_buffer_load_lds(rsL, (LDS void*)dx, 16, lane * 16, xo, 2048, AUX_SYS);
      __builtin_amdgcn_raw_ptr_buffer_load_lds(rsL, (LDS void*)dx, 16, lane * 16, xo, 3072, AUX_SYS);
      __builtin_amdgcn_raw_ptr_buffer_load_lds(rsL, (LDS void*)dw, 16, lane * 16, wo, 0, AUX_SYS);
      __builtin_amdgcn_raw_ptr_buffer_load_lds(rsL, (LDS void*)dw, 16, lane * 16, wo, 1024, AUX_SYS);
    };
    f32x4 acc[4][4];
    #pragma unroll
    for (int mi = 0; mi < 4; ++mi)
      #pragma unroll
      for (int ni = 0; ni < 4; ++ni) acc[mi][ni] = f32x4{0.f, 0.f, 0.f, 0.f};
    if (wave == 0) { if (ti == slot) ctl_st(deadw, 0u); ready(4); }
    asm volatile("s_waitcnt vmcnt(0) lgkmcnt(0)\n\ts_barrier" ::: "memory");
    if (ti == slot) { STAMP(2); STAMPC(14); }
    issue(0, 0); issue(1, STAGE);
    u32 st = 0;
    i32x8 fa[8], fb[8];
    auto mma = [&](const i32x8 (&f)[8]) {
      #pragma unroll
      for (int mi = 0; mi < 4; ++mi)
        #pragma unroll
        for (int ni = 0; ni < 4; ++ni)
          acc[mi][ni] = __builtin_amdgcn_mfma_scale_f32_16x16x128_f8f6f4(f[ni], f[4 + mi], acc[mi][ni], 1, 1, 0, 0, 0, 0);
    };
    auto step = [&](int s, i32x8 (&cur)[8], const i32x8 (&prev)[8], bool first, bool last) {
      if (wave == 0 && (s == 2 || s == 9 || s == 16 || s == 23)) { ready((s - 2) / 7); if (ti == slot) STAMP(10 + (s - 2) / 7); }
      if (!last) asm volatile("s_waitcnt vmcnt(6) lgkmcnt(0)\n\ts_barrier" ::: "memory");
      else asm volatile("s_waitcnt vmcnt(0) lgkmcnt(0)\n\ts_barrier" ::: "memory");
      if (s < 30) issue(s + 2, st == 0 ? 2 * STAGE : st - STAGE);
      LDS char* ax = smem + st + wm * 8192 + lane * 16;
      LDS char* aw = smem + st + 32 * KiB + wn * 8192 + lane * 16;
      #pragma unroll
      for (int f = 0; f < 4; ++f) {
        const u32x4 a0 = *(LDS u32x4*)(aw + f * 2048), a1 = *(LDS u32x4*)(aw + f * 2048 + 1024);
        const u32x4 b0 = *(LDS u32x4*)(ax + f * 2048), b1 = *(LDS u32x4*)(ax + f * 2048 + 1024);
        cur[f] = __builtin_bit_cast(i32x8, __builtin_shufflevector(a0, a1, 0, 1, 2, 3, 4, 5, 6, 7));
        cur[4 + f] = __builtin_bit_cast(i32x8, __builtin_shufflevector(b0, b1, 0, 1, 2, 3, 4, 5, 6, 7));
      }
      if (!first) {
        mma(prev);
        #pragma unroll
        for (int k = 0; k < 16; ++k) { __builtin_amdgcn_sched_group_barrier(0x100, 1, 0); __builtin_amdgcn_sched_group_barrier(0x8, 1, 0); }
      }
      st = st == 2 * STAGE ? 0 : st + STAGE;
    };
    step(0, fa, fb, true, false);
    for (int s = 1; s < 31; s += 2) { step(s, fb, fa, false, false); step(s + 1, fa, fb, false, false); }
    step(31, fb, fa, false, true);
    mma(fb);
    if (ti == slot) { STAMP(3); STAMPC(15); }
    int lz = lane; asm volatile("" : "+v"(lz));
    const bool dead = ctl_ld(deadw) != 0u;
    const int owner = tm >> 1;
    if (owner == rank) {
      const u32 obase = (((tm & 1) * 256 + wm * 64) * NN + rank * NCOL + tn * 128 + wn * 64) * 4, olane = ((lz & 15) * NN + (lz >> 4) * 4) * 4;
      const float sc = dead ? __builtin_nanf("") : scale;
      #pragma unroll
      for (int mi = 0; mi < 4; ++mi)
        #pragma unroll
        for (int ni = 0; ni < 4; ++ni)
          ST128(__builtin_bit_cast(u32x4, acc[mi][ni] * sc), rsO, olane + ni * 64, obase + mi * 16 * NN * 4, OUT_AUX);
    } else if (!dead) {
      const int t = ((tm & 1) * 16 + tn) * 8 + wave;
      const u32 uid = owner * 256 + t;
      const u32 offE = par + EX_OUE + uid * OU_ENC;
      float sa = 0.f;
      #pragma unroll
      for (int mi = 0; mi < 4; ++mi)
        #pragma unroll
        for (int ni = 0; ni < 4; ++ni) sa += __builtin_fabsf(acc[mi][ni][(mi + ni) & 3]);
      const int g = max(0, 24 - ((int)((as_u(wave_sum_f32(sa) * (3.75f / 1024.f)) >> 23) & 255u) - 126));
      LDS u32x2* list = (LDS u32x2*)(smem + L_EPI + wave * 1024);
      u32 cnt = 0, sg[2] = {0u, 0u};
      #pragma unroll
      for (int mi = 0; mi < 4; ++mi) {
        u32 pk[12];
        #pragma unroll
        for (int ni = 0; ni < 4; ++ni) {
          u32 qv[4]; u64 bm[4];
          #pragma unroll
          for (int r = 0; r < 4; ++r) {
            const float tt = __builtin_amdgcn_ldexpf(acc[mi][ni][r], g);
            const int iv = (int)__builtin_amdgcn_fmed3f(tt, -16777216.f, 16777215.f);
            qv[r] = (u32)iv;
            const int e = (mi * 4 + ni) * 4 + r;
            sg[e >> 5] = __builtin_amdgcn_alignbit(sg[e >> 5], (u32)iv, 31);
            bm[r] = __ballot(as_u((float)iv) != as_u(tt));
          }
          if (bm[0] | bm[1] | bm[2] | bm[3]) {
            #pragma unroll
            for (int r = 0; r < 4; ++r) {
              const u32 sl = cnt + 1 + __builtin_amdgcn_mbcnt_hi((u32)(bm[r] >> 32), __builtin_amdgcn_mbcnt_lo((u32)bm[r], 0u));
              const float vv = acc[mi][ni][r];
              if (((bm[r] >> lz) & 1) && sl <= OUT_CAP) list[sl] = u32x2{(u32)((mi * 4 + ni) * 256 + lz * 4 + r), as_u(vv)};
              cnt += __popcll(bm[r]);
            }
          }
          pk[ni * 3 + 0] = __builtin_amdgcn_perm(qv[1], qv[0], 0x04020100u);
          pk[ni * 3 + 1] = __builtin_amdgcn_perm(qv[2], qv[1], 0x05040201u);
          pk[ni * 3 + 2] = __builtin_amdgcn_perm(qv[3], qv[2], 0x06050402u);
        }
        #pragma unroll
        for (int j = 0; j < 3; ++j)
          ST128((u32x4{pk[j * 4], pk[j * 4 + 1], pk[j * 4 + 2], pk[j * 4 + 3]}), rsL, lz * 16 + j * 1024, offE + mi * 3072, AUX_SYS);
      }
      __builtin_amdgcn_raw_buffer_store_b64(u32x2{sg[0], sg[1]}, rsL, lz * 8, offE + 12288, AUX_SYS);
      const bool raw = cnt > OUT_CAP;
      wave_sync();
      u32x2 sd = list[lz];
      if (lz == 0) sd.x = raw ? 0x80000000u : (cnt | ((u32)(g + 256) << 16));
      if (lz < 16 || cnt > OUT_CAP0) __builtin_amdgcn_raw_buffer_store_b64(sd, rsL, lz * 8, offE + OU_SIDE, AUX_SYS);
      if (cnt > 63) __builtin_amdgcn_raw_buffer_store_b64(list[64 + lz], rsL, lz * 8 + 512, offE + OU_SIDE, AUX_SYS);
      if (raw) {
        const u32 offR = par + EX_OUR + uid * OU_RAW;
        #pragma unroll
        for (int mi = 0; mi < 4; ++mi)
          #pragma unroll
          for (int ni = 0; ni < 4; ++ni)
            ST128(__builtin_bit_cast(u32x4, acc[mi][ni]), rsL, lz * 16 + ni * 1024, offR + mi * 4096, AUX_SYS);
      }
      if (ti == slot) STAMP(9);
      vm_drain();
      if (lz == owner)
        __hip_atomic_store((GLB u32*)(peer_ws + OFF_FOUT + (rank * 256 + t) * 128), call, __ATOMIC_RELAXED, __HIP_MEMORY_SCOPE_SYSTEM);
    }
    if (ti == slot) STAMP(4); else STAMP(5);
  }
}

}

extern "C" void kernel_launch(int rank, int nranks, ncclComm_t comm, hipStream_t stream,
                              void* const* d_in, const int* in_sizes, int n_in,
                              void* d_out, int out_size,
                              const int* in_dtypes, int out_dtype,
                              void* const* d_out_all,
                              void* const* d_ws_all, size_t ws_bytes) {
  (void)comm; (void)in_sizes; (void)n_in; (void)out_size; (void)in_dtypes; (void)out_dtype; (void)d_out_all;
  if (nranks != NR || (unsigned)rank >= (unsigned)NR || ws_bytes < WS_NEED) { fprintf(stderr, "unexpected launch shape\n"); return; }
  static unsigned attr_set = 0;
  if (!(attr_set >> rank & 1u)) {
    HIP_CHECK(hipFuncSetAttribute((const void*)gemm_rs_kernel, hipFuncAttributeMaxDynamicSharedMemorySize, (int)LDS_BYTES));
    attr_set |= 1u << rank;
  }
  Params p;
  p.x = (const uint8_t*)d_in[0]; p.w = (const uint8_t*)d_in[1]; p.sx = (const float*)d_in[2]; p.sw = (const float*)d_in[3];
  p.out = (float*)d_out; p.ws_all = (char* const*)d_ws_all; p.ws = (char*)d_ws_all[rank]; p.rank = rank; p.ws_bytes = WS_NEED;
  hipLaunchKernelGGL(gemm_rs_kernel, dim3(NWG), dim3(NTHR), LDS_BYTES, stream, p);
  HIP_CHECK(hipGetLastError());
}
```

```cpp
#include <hip/hip_runtime.h>
#include <rccl/rccl.h>
#include <cstdio>
#include <cstdint>

#define HIP_CHECK(expr) do { hipError_t _e = (expr); if (_e != hipSuccess) { \
    fprintf(stderr, "HIP error %s at %s:%d\n", hipGetErrorString(_e), __FILE__, __LINE__); \
    return; } } while (0)

#define PULL_WGS 8
#define WIN 8
#define LOOK 4
#define NRAW 8
#define SEGORD 0x73625410u
#define QKB 2
#define OUT_AUX 17

namespace {

typedef unsigned u32;
typedef unsigned long long u64;
typedef unsigned u32x4 __attribute__((ext_vector_type(4)));
typedef unsigned u32x2 __attribute__((ext_vector_type(2)));
typedef int i32x2 __attribute__((ext_vector_type(2)));
typedef int i32x8 __attribute__((ext_vector_type(8)));
typedef float f32x4 __attribute__((ext_vector_type(4)));
typedef __amdgpu_buffer_rsrc_t rsrc_t;
#define LDS __attribute__((address_space(3)))
#define GLB __attribute__((address_space(1)))

constexpr int NR = 8, MM = 4096, NN = 16384, KS = 512;
constexpr int NWG = 256, NTHR = 512;
constexpr int NPULL = 7 * PULL_WGS, NGEMM = NWG - NPULL;
constexpr int AUX_SYS = 17;
constexpr u64 DEADLINE_TICKS = 40000000ull;

constexpr u32 KiB = 1024, MiB = 1024 * 1024;
constexpr u32 OFF_CALL = 0;
constexpr u32 OFF_TMO = 32 * KiB;
constexpr u32 OFF_TIC = 33 * KiB;
constexpr u32 OFF_CNT = 36 * KiB;
constexpr u32 OFF_FIN = 128 * KiB;
constexpr u32 OFF_FOUT = OFF_FIN + 8 * 576 * 128;
constexpr u32 OFF_DBG = 1280 * KiB;
constexpr u32 OFF_XA = 2 * MiB;
constexpr u32 OFF_WA = OFF_XA + 16 * MiB;
constexpr u32 OFF_PAR = OFF_WA + 16 * MiB;
constexpr u32 IU_ENC = 8 * KiB, IU_RAW = 8 * KiB;
constexpr u32 OU_ENC = 13056, OU_RAW = 16 * KiB;
constexpr u32 IU_SIDE = 6144, OU_SIDE = 12288;
constexpr u32 EX_INE = 0;
constexpr u32 EX_INR = EX_INE + 1280 * IU_ENC;
constexpr u32 EX_OUE = EX_INR + 1280 * IU_RAW;
constexpr u32 EX_OUR = EX_OUE + 2560 * OU_ENC;
constexpr u32 PAR_BYTES = EX_OUR + 2560 * OU_RAW;
constexpr u32 WS_NEED = OFF_PAR + 2 * PAR_BYTES;
static_assert(OFF_CNT + 96 * 5 * 128 <= OFF_FIN && OFF_FOUT + 8 * 320 * 128 <= OFF_DBG, "flags");
static_assert(WS_NEED <= 257u * MiB, "workspace");
constexpr u32 QV = 5120, QV0 = 896, QV_STR = 64, QV_END = 1792, QV_ALL = 2304;
constexpr u32 Q_ESC0 = (QV0 - QV_STR) * 8 / 3, Q_ESC = (QV_END - QV_STR) * 8 / 3, Q_PAT0 = 30, Q_PAT = 286;
constexpr u32 IN_CAP0 = 126, IN_CAP = 510;
constexpr u32 OUT_CAP0 = 10, OUT_CAP = 64;

constexpr u32 STAGE = 48 * KiB;
constexpr u32 L_SCR = 64 * KiB, SCR_W = 10752;
constexpr u32 L_EPI = 3 * STAGE;
constexpr u32 NSLOT = 8, SLOT = 13 * KiB, NCONS = 6;
constexpr u32 L_PSCR = NSLOT * SLOT;
constexpr u32 L_CTL = L_PSCR + NCONS * 8 * KiB;
constexpr u32 L_TAB = L_CTL + 256;
constexpr u32 LDS_BYTES = L_TAB + 512;
static_assert(L_EPI + 8 * KiB <= LDS_BYTES && L_SCR + 8 * SCR_W <= LDS_BYTES && LDS_BYTES <= 160 * KiB, "lds");

__device__ __forceinline__ int group_of(int rank, int j) { return rank < 4 ? rank * 4 + j : j * 4 + (rank & 3); }
__device__ __forceinline__ int split_of(int rank, int j, int c) { return c < 4 + (((rank & 3) + j) & 1) ? 3 : 4; }

struct Params {
  const uint8_t* x; const uint8_t* w; const float* sx; const float* sw;
  float* out; char* const* ws_all; char* ws; int rank; u32 ws_bytes;
};

__device__ __forceinline__ rsrc_t mk_rsrc(const void* p, u32 bytes) {
  u64 a = (u64)p;
  u32 lo = __builtin_amdgcn_readfirstlane((u32)a), hi = __builtin_amdgcn_readfirstlane((u32)(a >> 32));
  return __builtin_amdgcn_make_buffer_rsrc((void*)(((u64)hi << 32) | lo), 0, (int)bytes, 0x00020000);
}
#define ST128(v, rs, voff, soff, aux) __builtin_amdgcn_raw_buffer_store_b128((v), (rs), (int)((u32)(voff) + (u32)(soff)), 0, (aux))
__device__ __forceinline__ float as_f(u32 v) { return __builtin_bit_cast(float, v); }
__device__ __forceinline__ u32 as_u(float v) { return __builtin_bit_cast(u32, v); }
__device__ __forceinline__ void vm_drain() { asm volatile("s_waitcnt vmcnt(0)" ::: "memory"); }
__device__ __forceinline__ void wave_sync() {
  asm volatile("s_waitcnt lgkmcnt(0)" ::: "memory");
}
__device__ __forceinline__ u32 bfi(u32 m, u32 a, u32 b) { return __builtin_amdgcn_bitop3_b32(m, a, b, 0xCA); }
template <int CTRL, int ROWS> __device__ __forceinline__ u32 dpp0(u32 v) { return (u32)__builtin_amdgcn_update_dpp(0, (int)v, CTRL, ROWS, 0xf, false); }
__device__ __forceinline__ u32 wave_scan_u32(u32 v) {
  v += dpp0<0x111, 0xf>(v); v += dpp0<0x112, 0xf>(v); v += dpp0<0x114, 0xf>(v); v += dpp0<0x118, 0xf>(v);
  v += dpp0<0x142, 0xa>(v); v += dpp0<0x143, 0xc>(v);
  return v;
}
__device__ __forceinline__ float wave_sum_f32(float v) {
  v += as_f(dpp0<0x111, 0xf>(as_u(v))); v += as_f(dpp0<0x112, 0xf>(as_u(v))); v += as_f(dpp0<0x114, 0xf>(as_u(v)));
  v += as_f(dpp0<0x118, 0xf>(as_u(v))); v += as_f(dpp0<0x142, 0xa>(as_u(v))); v += as_f(dpp0<0x143, 0xc>(as_u(v)));
  return as_f((u32)__builtin_amdgcn_readlane((int)as_u(v), 63));
}
__device__ __forceinline__ u32 ctl_ld(u32 addr) {
  u32 v; asm volatile("ds_read_b32 %0, %1\n\ts_waitcnt lgkmcnt(0)" : "=&v"(v) : "v"(addr) : "memory");
  return (u32)__builtin_amdgcn_readfirstlane((int)v);
}
__device__ __forceinline__ void ctl_st(u32 addr, u32 v) { asm volatile("ds_write_b32 %0, %1" :: "v"(addr), "v"(v) : "memory"); }
__device__ __forceinline__ u32 ld_flag(const char* ws, u32 off) {
  return __hip_atomic_load((const GLB u32*)(ws + off), __ATOMIC_RELAXED, __HIP_MEMORY_SCOPE_SYSTEM);
}
__device__ __forceinline__ bool wait_ge(char* ws, u32 off, u32 target, u32 call) {
  if (__all(ld_flag(ws, off) >= target)) return true;
  const u64 t0 = __builtin_amdgcn_s_memrealtime();
  GLB u32* tmo = (GLB u32*)(ws + OFF_TMO);
  for (;;) {
    __builtin_amdgcn_s_sleep(4);
    if (__all(ld_flag(ws, off) >= target)) return true;
    if (__hip_atomic_load(tmo, __ATOMIC_RELAXED, __HIP_MEMORY_SCOPE_AGENT) == call ||
        __builtin_amdgcn_s_memrealtime() - t0 > DEADLINE_TICKS) {
      __hip_atomic_store(tmo, call, __ATOMIC_RELAXED, __HIP_MEMORY_SCOPE_AGENT);
      return false;
    }
  }
}
__device__ __forceinline__ void cnt_add(char* ws, u32 idx, int lane) {
  if (lane == 0) __hip_atomic_fetch_add((GLB u32*)(ws + OFF_CNT + idx * 128), 1u, __ATOMIC_RELAXED, __HIP_MEMORY_SCOPE_AGENT);
}
#define STAMPC(slot) do {} while (0)
#define USTAMP(k) do {} while (0)
#define STAMP(slot) do {} while (0)

__device__ __forceinline__ void store_raw_unit(const u32x4 (&x)[8], rsrc_t rs, u32 off, int lane) {
  #pragma unroll
  for (int j = 0; j < 8; ++j) ST128(x[j], rs, lane * 16 + j * 1024, off, AUX_SYS);
}

__device__ __forceinline__ void export_unit(const u32x4 (&x)[8], rsrc_t rs, u32 par, u32 uid, LDS char* scr, int lane) {
  const u32 offE = par + EX_INE + uid * IU_ENC, offR = par + EX_INR + uid * IU_RAW;
  u32 s = 0;
  #pragma unroll
  for (int j = 0; j < 8; ++j) {
    *(LDS u32x4*)(scr + j * 1024 + lane * 16) = x[j];
    s = __builtin_amdgcn_sad_u8(x[j][j & 3] & 0x7C7C7C7Cu, 0u, s);
  }
  s = (u32)__builtin_amdgcn_readlane((int)wave_scan_u32(s), 63);
  int e0 = ((int)s - 34243) >> 13;
  e0 = e0 < 0 ? 0 : (e0 > 24 ? 24 : e0);
  const u32 e4 = (u32)e0 * 0x04040404u;
  u32 mk[4] = {0, 0, 0, 0};
  u32 pk[24];
  #pragma unroll
  for (int j = 0; j < 8; ++j) {
    u32 o[4];
    #pragma unroll
    for (int c = 0; c < 4; ++c) {
      const u32 u = (x[j][c] | 0x80808080u) - e4;
      o[c] = bfi(0x1F1F1F1Fu, u, x[j][c]);
      const u32 t = (((u >> 1) ^ 0x40404040u) & 0x70707070u) + 0x70707070u;
      const int d = j * 4 + c;
      mk[d >> 3] = bfi(0x80808080u >> (d & 7), t >> (d & 7), mk[d >> 3]);
    }
    pk[j * 3 + 0] = bfi(0x60606060u, o[3] << 5, o[0]);
    pk[j * 3 + 1] = bfi(0x60606060u, o[3] << 3, o[1]);
    pk[j * 3 + 2] = bfi(0x60606060u, bfi(0x40404040u, o[3] >> 1, o[3] << 1), o[2]);
  }
  #pragma unroll
  for (int j = 0; j < 6; ++j) {
    const u32x4 v = {pk[j * 4], pk[j * 4 + 1], pk[j * 4 + 2], pk[j * 4 + 3]};
    ST128(v, rs, lane * 16 + j * 1024, offE, AUX_SYS);
  }
  const u32 mine = __popc(mk[0]) + __popc(mk[1]) + __popc(mk[2]) + __popc(mk[3]);
  const u32 inc = wave_scan_u32(mine);
  const u32 total = (u32)__builtin_amdgcn_readlane((int)inc, 63);
  if (total <= IN_CAP) {
    LDS unsigned short* list = (LDS unsigned short*)(scr + 8192);
    u32 slot = 2 + inc - mine;
    #pragma unroll
    for (int q = 0; q < 4; ++q) {
      u32 m = mk[q];
      while (m) {
        const int bp = __builtin_ctz(m); m &= m - 1;
        const int d = q * 8 + 7 - (bp & 7);
        list[slot++] = (unsigned short)((d >> 2) * 1024 + lane * 16 + (d & 3) * 4 + (bp >> 3));
      }
    }
    wave_sync();
    for (u32 j = 0; j < 4; ++j) {
      const u32 pp = *(LDS u32*)(scr + 8192 + j * 256 + lane * 4);
      const u32 p0 = pp & 8191u, p1 = (pp >> 16) & 8191u;
      const u32 b0 = *(LDS unsigned char*)(scr + p0), b1 = *(LDS unsigned char*)(scr + p1);
      u32 sd = (p0 << 3) | ((128u + (b0 & 127u) - 4u * (u32)e0) >> 5) | (p1 << 19) | (((128u + (b1 & 127u) - 4u * (u32)e0) >> 5) << 16);
      if (j == 0 && lane == 0) sd = total | ((u32)e0 << 16);
      __builtin_amdgcn_raw_buffer_store_b32(sd, rs, lane * 4 + j * 256, offE + IU_SIDE, AUX_SYS);
      if (total <= IN_CAP0 + j * 128) break;
    }
  } else {
    store_raw_unit(x, rs, offR, lane);
    if (lane == 0) __builtin_amdgcn_raw_buffer_store_b32(0x80000000u, rs, 0, offE + IU_SIDE, AUX_SYS);
  }
}

__device__ __forceinline__ void export_short(const u32x4 (&x)[8], rsrc_t rs, u32 par, u32 uid, LDS char* scr, int lane) {
  typedef unsigned short u16x2 __attribute__((ext_vector_type(2)));
  const u32 offE = par + EX_INE + uid * IU_ENC, offR = par + EX_INR + uid * IU_RAW;
  LDS char* var = scr + 8192;
  LDS char* tab = var + QV_ALL;
  u32 s = 0;
  #pragma unroll
  for (int j = 0; j < 8; ++j) {
    *(LDS u32x4*)(scr + j * 1024 + lane * 16) = x[j];
    s = __builtin_amdgcn_sad_u8(x[j][j & 3] & 0x7C7C7C7Cu, 0u, s);
  }
  *(LDS u32x4*)(var + lane * 16) = u32x4{0u, 0u, 0u, 0u};
  *(LDS u32x4*)(var + 1024 + lane * 16) = u32x4{0u, 0u, 0u, 0u};
  {
    u32 sel = 0x0C0C0C0Cu, at = 0;
    #pragma unroll
    for (int y = 0; y < 4; ++y) if ((lane >> y) & 1) { sel = (sel & ~(0xFFu << at)) | ((u32)y << at); at += 8; }
    if (lane < 16) *(LDS u32*)(tab + lane * 4) = sel;
  }
  s = (u32)__builtin_amdgcn_readlane((int)wave_scan_u32(s), 63);
  const int K = min(31, ((int)s >> 13) + 3);
  const u32 kw = (128u + 4u * (u32)K) * 0x01010101u, tabo = (u32)(uintptr_t)tab;
  u32 mk[4] = {0, 0, 0, 0}, zq[32], car[20], bits = 0;
  wave_sync();
  #pragma unroll
  for (int g = 0; g < 4; ++g) {
    u32 a[8];
    #pragma unroll
    for (int k = 0; k < 8; ++k) {
      const int d = g * 8 + k;
      const u32 xv = x[d >> 2][d & 3];
      const u32 w = kw - (xv & 0x7C7C7C7Cu);
      const u32 sel = (w >> 2) & 0x07070707u;
      const u32 hi8 = __builtin_bit_cast(u32, __builtin_bit_cast(u16x2, (w >> 5) & 0x01010101u) * (u16x2){255, 255});
      const u32 l0 = __builtin_amdgcn_perm(0x7D6D5D4Du, 0x0004083Du, sel), l1 = __builtin_amdgcn_perm(0xFDEDDDCDu, 0x802D1D0Du, sel);
      const u32 L = (hi8 & l1) | (~hi8 & l0);
      const u32 pf = L | ~w | (w << 1);
      mk[g] = bfi(0x80808080u >> k, pf >> k, mk[g]);
      a[k] = bfi(0x0C0C0C0Cu, L, xv);
      const u32 F = L & 0x01010101u;
      const u32 cs = __builtin_amdgcn_perm(0u, (L >> 4) & 0x07070707u, *(LDS u32*)(uintptr_t)__builtin_amdgcn_udot4(F, 0x20100804u, tabo, false));
      zq[d] = (__builtin_amdgcn_udot4(cs, 0x00400801u, 0u, false) + ((cs >> 24) << 9)) | (bits << 16);
      bits = __builtin_amdgcn_udot4(F, 0x03030303u, bits, false);
    }
    car[g * 5 + 0] = bfi(0x70707070u, a[5] << 4, a[0]);
    car[g * 5 + 1] = bfi(0x70707070u, a[6] << 4, a[1]);
    car[g * 5 + 2] = bfi(0x70707070u, a[7] << 4, a[2]);
    car[g * 5 + 3] = bfi(0x40404040u, a[7] << 3, bfi(0x20202020u, a[6] << 2, bfi(0x10101010u, a[5] << 1, a[3])));
    car[g * 5 + 4] = bfi(0x40404040u, a[7] >> 1, bfi(0x20202020u, a[6] >> 2, bfi(0x10101010u, a[5] >> 3, a[4])));
  }
  #pragma unroll
  for (int j = 0; j < 5; ++j) ST128((u32x4{car[j * 4], car[j * 4 + 1], car[j * 4 + 2], car[j * 4 + 3]}), rs, lane * 16 + j * 1024, offE, AUX_SYS);
  const u32 mine = __popc(mk[0]) + __popc(mk[1]) + __popc(mk[2]) + __popc(mk[3]);
  const u32 inc = wave_scan_u32(bits | (mine << 16));
  const u32 tot = (u32)__builtin_amdgcn_readlane((int)inc, 63), nbit = tot & 0xFFFFu, npat = tot >> 16;
  if (nbit <= Q_ESC * 3 && npat <= Q_PAT) {
    const u32 base = QV_STR * 8 + (inc & 0xFFFFu) - bits;
    #pragma unroll
    for (int d = 0; d < 32; ++d) {
      const u32 q = base + (zq[d] >> 16);
      const u64 v = (u64)(zq[d] & 0xFFFFu) << (q & 31u);
      LDS u32* at = (LDS u32*)(var + ((q >> 5) << 2));
      __hip_atomic_fetch_or(at, (u32)v, __ATOMIC_RELAXED, __HIP_MEMORY_SCOPE_WORKGROUP);
      __hip_atomic_fetch_or(at + 1, (u32)(v >> 32), __ATOMIC_RELAXED, __HIP_MEMORY_SCOPE_WORKGROUP);
    }
    u32 slot = (inc >> 16) - mine;
    #pragma unroll
    for (int q = 0; q < 4; ++q) {
      u32 m = mk[q];
      while (m) {
        const int bp = __builtin_ctz(m); m &= m - 1;
        const int d = q * 8 + 7 - (bp & 7);
        const u32 pos = (d >> 2) * 1024 + lane * 16 + (d & 3) * 4 + (bp >> 3);
        const int v = K - (int)((*(LDS unsigned char*)(scr + pos) >> 2) & 31u), t = v & 15;
        *(LDS unsigned short*)(var + (slot < Q_PAT0 ? 4 + slot * 2 : QV_END - Q_PAT0 * 2 + slot * 2)) =
            (unsigned short)((pos << 3) | (u32)(((v - (t < 11 ? t : t - 8)) >> 3) + 4));
        ++slot;
      }
    }
    if (lane == 0) *(LDS u32*)var = nbit | (npat << 15) | ((u32)K << 24);
    wave_sync();
    if (lane < (int)QV0 / 16) ST128(*(LDS u32x4*)(var + lane * 16), rs, lane * 16, offE + QV, AUX_SYS);
    if (nbit > Q_ESC0 * 3 && lane < (int)(QV_END - QV0) / 16) ST128(*(LDS u32x4*)(var + QV0 + lane * 16), rs, lane * 16, offE + QV + QV0, AUX_SYS);
    if (npat > Q_PAT0 && lane < (int)(QV_ALL - QV_END) / 16) ST128(*(LDS u32x4*)(var + QV_END + lane * 16), rs, lane * 16, offE + QV + QV_END, AUX_SYS);
  } else {
    store_raw_unit(x, rs, offR, lane);
    if (lane == 0) __builtin_amdgcn_raw_buffer_store_b32(0x80000000u, rs, 0, offE + QV, AUX_SYS);
  }
}

__device__ __forceinline__ void patch_short(LDS char* scr, u32 ent) {
  const u32 pos = ent >> 3;
  __hip_atomic_fetch_add((LDS u32*)(scr + (pos & ~3u)), ((4u - (ent & 7u)) << 5) << ((pos & 3u) * 8u), __ATOMIC_RELAXED, __HIP_MEMORY_SCOPE_WORKGROUP);
}
__device__ __forceinline__ void patch_in(LDS char* scr, u32 ent) {
  const u32 pos = ent >> 3;
  __hip_atomic_fetch_add((LDS u32*)(scr + (pos & ~3u)), (((ent & 7u) - 4u) << 5) << ((pos & 3u) * 8u), __ATOMIC_RELAXED, __HIP_MEMORY_SCOPE_WORKGROUP);
}

__device__ __forceinline__ float unfix(int i, int g) { return __builtin_amdgcn_ldexpf((float)i, -g); }

__global__ void __launch_bounds__(NTHR, 2) gemm_rs_kernel(Params p) {
  extern __shared__ __attribute__((aligned(16))) char smem_[];
  LDS char* smem = (LDS char*)smem_;
  const int tid = threadIdx.x, lane = tid & 63, wave = __builtin_amdgcn_readfirstlane(tid >> 6);
  const int b = blockIdx.x, rank = p.rank;
  const int lg = lane >> 4, li = lane & 15;
  STAMP(0);
  const rsrc_t rsL = mk_rsrc(p.ws, p.ws_bytes);
  const u32 tick = (u32)__builtin_amdgcn_raw_ptr_buffer_atomic_add_i32(1, rsL, lane == 0 ? 0 : (int)0xF0000000u, OFF_CALL + b * 128 + wave * 16, 0);
  char* const peer_ws = p.ws_all[lane & 7];
  u32 call;

  if (b < 128) {
    const rsrc_t rsW = mk_rsrc(p.w, KS * NN);
    #pragma unroll
    for (int i = 0; i < 8; ++i) {
      const int row = (wave * 8 + i) * 8 + (lane >> 3);
      const int sw = ((row >> 1) & 3) | (((row >> 4) & 1) << 2);
      __builtin_amdgcn_raw_ptr_buffer_load_lds(rsW, (LDS void*)(smem + (wave * 8 + i) * 1024), 16, row * NN + (((lane & 7) ^ sw) << 4), b * 128, 0, 0);
    }
    asm volatile("s_waitcnt vmcnt(0)\n\ts_barrier" ::: "memory");
    call = __builtin_amdgcn_readfirstlane(tick) + 1u;
    const int cgl = wave & 1, kb = wave >> 1;
    u32x4 x[8];
    #pragma unroll
    for (int f = 0; f < 4; ++f)
      #pragma unroll
      for (int h = 0; h < 2; ++h)
        #pragma unroll
        for (int t = 0; t < 2; ++t) {
          const int row = kb * 128 + 64 * h + 16 * lg + 8 * t + (li >> 1);
          const int sw = ((row >> 1) & 3) | (((row >> 4) & 1) << 2);
          const i32x2 v = __builtin_amdgcn_ds_read_tr8_b64_v2i32((LDS i32x2*)(smem + row * 128 + (((cgl * 4 + f) ^ sw) << 4) + (li & 1) * 8));
          x[f * 2 + h][t * 2] = v.x; x[f * 2 + h][t * 2 + 1] = v.y;
        }
    const int gl = b >> 5, gr = 4 + ((b >> 3) & 3), c = b & 7;
    export_short(x, rsL, OFF_PAR + (call & 1) * PAR_BYTES, 256 + (b * 2 + cgl) * 4 + kb, smem + L_SCR + wave * SCR_W, lane);
    vm_drain();
    if ((lane == gl || lane == gr) && lane != rank) {
      const int cb = (lane == gl ? gr & 3 : gl) * 8 + c;
      __hip_atomic_store((GLB u32*)(peer_ws + OFF_FIN + (rank * 576 + 256 + (cb * 2 + cgl) * 4 + kb) * 128), call, __ATOMIC_RELAXED, __HIP_MEMORY_SCOPE_SYSTEM);
    }
    if (rank == gl || rank == gr) {
      const int cb = (rank == gl ? gr & 3 : gl) * 8 + c;
      store_raw_unit(x, rsL, OFF_WA + ((rank * 4 + kb) * 64 + cb * 2 + cgl) * IU_RAW, lane);
      vm_drain();
      cnt_add(p.ws, (64 + cb) * 5 + 4, lane);
    }
  } else if (b < 192 && wave < 4) {
    const int u = (b - 128) * 4 + wave, rg = u >> 2, kb = u & 3;
    const rsrc_t rsX = mk_rsrc(p.x, MM * KS);
    u32x4 x[8];
    #pragma unroll
    for (int j = 0; j < 8; ++j)
      x[j] = __builtin_amdgcn_raw_buffer_load_b128(rsX, ((j >> 1) * 16 + li) * KS + (j & 1) * 64 + lg * 16, rg * 64 * KS + kb * 128, 0);
    call = __builtin_amdgcn_readfirstlane(tick) + 1u;
    STAMP(6);
    const u32 parw = OFF_PAR + (call & 1) * PAR_BYTES;
    const int ow = rg >> 3, r8 = rg & 7;
    const u32 users = ((r8 <= 3 ? 0xFu << (ow & 4) : 0u) | (r8 >= 3 ? 0xFu << (~ow & 4) : 0u) | (1u << ow)) & ~(1u << rank);
    const bool tell = lane < NR && ((users >> lane) & 1);
    if (NRAW && kb == 0) {
      store_raw_unit(x, rsL, parw + EX_INR + u * IU_RAW, lane);
      vm_drain();
      if (tell) __hip_atomic_store((GLB u32*)(peer_ws + OFF_FIN + (rank * 576 + 512 + rg) * 128), call, __ATOMIC_RELAXED, __HIP_MEMORY_SCOPE_SYSTEM);
    }
    if (kb < QKB) export_unit(x, rsL, parw, u, smem + L_SCR + wave * SCR_W, lane);
    else export_short(x, rsL, parw, u, smem + L_SCR + wave * SCR_W, lane);
    STAMP(7);
    vm_drain();
    if (tell) __hip_atomic_store((GLB u32*)(peer_ws + OFF_FIN + (rank * 576 + u) * 128), call, __ATOMIC_RELAXED, __HIP_MEMORY_SCOPE_SYSTEM);
    STAMP(8);
    store_raw_unit(x, rsL, OFF_XA + ((rank * 4 + kb) * 64 + rg) * IU_RAW, lane);
    vm_drain();
    cnt_add(p.ws, rg * 5 + 4, lane);
  }
  call = __builtin_amdgcn_readfirstlane(tick) + 1u;
  const u32 par = OFF_PAR + (call & 1) * PAR_BYTES;
  const float scale = p.sx[0] * p.sw[0];
  const rsrc_t rsO = mk_rsrc(p.out, 512u * NN * 4u);
  STAMP(1);

  if (b >= NGEMM) {
    const int pb = b - NGEMM, link = pb / PULL_WGS, q = pb % PULL_WGS;
    const int src = (rank + 1 + link) & 7;
    const u64 pw = (u64)peer_ws;
    const u32 pw_lo = (u32)__builtin_amdgcn_readlane((int)(u32)pw, src), pw_hi = (u32)__builtin_amdgcn_readlane((int)(u32)(pw >> 32), src);
    const rsrc_t rsP = mk_rsrc((const void*)(((u64)pw_hi << 32) | (u64)pw_lo), p.ws_bytes);
    const u32 ctl = (u32)(uintptr_t)(smem + L_CTL);
    const u32 C_READY = ctl, C_LANDED = ctl + 4, C_ISSUED = ctl + 8, C_CLOSED = ctl + 12, C_REL = ctl + 16, C_Q = ctl + 64;
    const bool same = (src >> 2) == (rank >> 2);
    if (wave == 0) ctl_st(ctl + lane * 4, 0u);
    u32 nout = 0;
    if (wave == 1) {
      const int j = (lane >> 3) & 3, c = ((lane >> 1) & 3) * 2 + (lane >> 5), sp = split_of(src, j, c);
      const u32 k0 = same ? 0 : sp - 3, mine = same ? sp : (j == (rank & 3) ? 0 : 8 - sp);
      const u32 inc = wave_scan_u32(mine);
      for (u32 k = 0; k < mine; ++k) ((LDS unsigned short*)(smem + L_TAB))[inc - mine + k] = (unsigned short)(((j * 8 + c) * 2 + (lane & 1)) * 5 + k0 + k);
      nout = (u32)__builtin_amdgcn_readlane((int)inc, 63);
    }
    asm volatile("s_waitcnt lgkmcnt(0)\n\ts_barrier" ::: "memory");
    if (wave == 0) {
      int g = 0, pub = 0;
      #define LANDED(v) do { if ((v) > pub) { pub = (v); ctl_st(C_LANDED, (u32)pub); } } while (0)
      #define PIECE(SZ, VOFF, OFF) do { __builtin_amdgcn_raw_ptr_buffer_load_lds(rsP, (LDS void*)(dst + (OFF)), SZ, VOFF, so + (OFF), 0, AUX_SYS); \
        asm volatile("s_waitcnt vmcnt(%0)" :: "i"(WIN - 1) : "memory"); ++g; LANDED(g - (WIN - 1)); } while (0)
      bool seen_out = false;
      for (u32 n = 0;; ++n) {
        if (ctl_ld(C_READY) <= n) {
          vm_drain(); LANDED(g);
          bool fin = false;
          while (ctl_ld(C_READY) <= n) { if (ctl_ld(C_CLOSED)) { fin = ctl_ld(C_READY) <= n; break; } __builtin_amdgcn_s_sleep(1); }
          if (fin) break;
        }
        const u32 ent = ctl_ld(C_Q + (n & 15) * 4);
        ctl_st(C_ISSUED, n + 1);
        if (ent >> 31) continue;
        const u32 need = n >= NSLOT ? n - NSLOT + 1 : 0, ra = C_REL + (n % NSLOT) * 4;
        if (ctl_ld(ra) < need) {
          vm_drain(); LANDED(g);
          while (ctl_ld(ra) < need) __builtin_amdgcn_s_sleep(1);
        }
        const int i = ent & 1023;
        if (n == 0) STAMP(6);
        USTAMP(0);
        if (i >= 512 && !seen_out) { seen_out = true; STAMP(3); }
        LDS char* dst = smem + (n % NSLOT) * SLOT;
        if (ent & 1024) {
          const u32 so = par + EX_INR + i * IU_RAW;
          #pragma unroll
          for (int j = 0; j < 8; ++j) PIECE(16, lane * 16, j * 1024);
        } else if (i < 256 && (i & 3) < QKB) {
          const u32 so = par + EX_INE + i * IU_ENC;
          PIECE(4, lane * 4, IU_SIDE);
          #pragma unroll
          for (int j = 0; j < 6; ++j) PIECE(16, lane * 16, j * 1024);
        } else if (i < 512) {
          const u32 so = par + EX_INE + (i < 256 ? i : 256 + group_of(rank, (i >> 6) & 3) * 64 + (i & 63)) * IU_ENC;
          if (lane < (int)QV0 / 16) __builtin_amdgcn_raw_ptr_buffer_load_lds(rsP, (LDS void*)(dst + QV), 16, lane * 16, so + QV, 0, AUX_SYS);
          asm volatile("s_waitcnt vmcnt(%0)" :: "i"(WIN - 1) : "memory"); ++g; LANDED(g - (WIN - 1));
          #pragma unroll
          for (int j = 0; j < 5; ++j) PIECE(16, lane * 16, j * 1024);
          STAMP(2);
        } else {
          const u32 so = par + EX_OUE + (rank * 320 + i - 512) * OU_ENC;
          PIECE(4, lane * 4, OU_SIDE);
          #pragma unroll
          for (int j = 0; j < 12; ++j) PIECE(16, lane * 16, j * 1024);
        }
      }
      vm_drain(); LANDED(g);
      #undef PIECE
      #undef LANDED
      STAMP(7);
      return;
    }
    if (wave == 1) {
      const u32 tic = OFF_TIC + ((call & 1) * 8 + link) * 128;
      const int only0 = lane == 0 ? 0 : (int)0xF0000000u;
      if (q == 0 && lane == 0)
        __hip_atomic_store((GLB u32*)(p.ws + OFF_TIC + ((~call & 1) * 8 + link) * 128), 0u, __ATOMIC_RELAXED, __HIP_MEMORY_SCOPE_AGENT);
      u32 n = 0, endp = 0; bool ok = true;
      int tnext = __builtin_amdgcn_raw_ptr_buffer_atomic_add_i32(1, rsL, only0, tic, 0);
      for (;;) {
        const u32 t = (u32)__builtin_amdgcn_readfirstlane(tnext);
        constexpr u32 NPAIR = 208;
        if (t >= NPAIR + nout) break;
        while (n >= ctl_ld(C_ISSUED) + LOOK) __builtin_amdgcn_s_sleep(1);
        tnext = __builtin_amdgcn_raw_ptr_buffer_atomic_add_i32(1, rsL, only0, tic, 0);
        int i;
        if (t < NPAIR) {
          const int e = 2 * t + (lane & 1);
          if (e < 256) {
            const int sg = (SEGORD >> ((e >> 5) * 4)) & 7;
            int u = (e & 31) + (sg >> 2) * 32, kb = sg & 3;
            if (e < 64) { if (e < NRAW) kb = 0; else if (e < NRAW + 32) { u = e - NRAW; kb = 1; } else { u = e - 32; kb = 0; } }
            const int v = u - 12;
            if (u < 12) i = (8 * ((rank & 4) | ((rank + 1 + (u >> 2)) & 3)) + (u & 3)) * 4;
            else if (u < 32) i = (8 * ((~rank & 4) | (v / 5)) + 3 + v % 5) * 4;
            else i = 256 + (((u - 32) >> 1) * 4 + (u & 1)) * 4;
            i += kb;
          } else if (e < 384) i = 256 + ((((e & 31) >> 1) * 2 + 1) * 2 + (e & 1)) * 4 + ((e - 256) >> 5);
          else i = (8 * rank + (e & 7)) * 4 + ((e - 384) >> 3);
        } else i = 512 + ((LDS unsigned short*)(smem + L_TAB))[t - NPAIR];
        const bool unp = 2 * t < NRAW;
        const u32 fo = i >= 512 ? OFF_FOUT + (src * 320 + i - 512) * 128 : OFF_FIN + (src * 576 + (unp ? 512 + (i >> 2) : i)) * 128;
        ok = ok && wait_ge(p.ws, fo, call, call);
        const u32 k = t < NPAIR ? (lane & 1) : 0, per = !ok ? 0 : t >= NPAIR ? 13 : unp ? 8 : (u32)__builtin_amdgcn_readfirstlane(i < 256 && (i & 3) < QKB ? 7 : 6);
        ctl_st(C_Q + ((n + k) & 15) * 4, (u32)i | (unp ? 1024u : 0u) | ((endp + per * (k + 1)) << 11) | (ok ? 0u : 0x80000000u));
        n += t < NPAIR ? 2 : 1; endp += t < NPAIR ? 2 * per : per;
        asm volatile("s_waitcnt lgkmcnt(0)" ::: "memory");
        ctl_st(C_READY, n);
      }
      asm volatile("s_waitcnt lgkmcnt(0)" ::: "memory");
      ctl_st(C_CLOSED, 1u);
      return;
    }
    LDS char* scr = smem + L_PSCR + (wave - 2) * 8 * KiB;
    for (u32 n = wave - 2;; n += NCONS) {
      bool fin = false;
      while (ctl_ld(C_READY) <= n) { if (ctl_ld(C_CLOSED)) { fin = ctl_ld(C_READY) <= n; break; } __builtin_amdgcn_s_sleep(1); }
      if (fin) break;
      const u32 ent = ctl_ld(C_Q + (n & 15) * 4);
      const bool ok = !(ent >> 31);
      const int i = ent & 1023;
      const u32 endp = (ent >> 11) & 0x1FFFu;
      auto landed = [&](u32 upto) { while (ctl_ld(C_LANDED) < upto) __builtin_amdgcn_s_sleep(1); };
      LDS char* sl = smem + (n % NSLOT) * SLOT;
      if (i < 512) {
        if (!ok) continue;
        landed(endp);
        USTAMP(1);
        const int kb = i & 3, rg = (i >> 2) & 63; const bool isx = i < 256;
        const u32 uid = isx ? i : 256 + group_of(rank, (i >> 6) & 3) * 64 + (i & 63);
        const u32 dst = (isx ? OFF_XA : OFF_WA) + ((src * 4 + kb) * 64 + rg) * IU_RAW;
        const u32 rel = C_REL + (n % NSLOT) * 4, offE = par + EX_INE + uid * IU_ENC;
        u32x4 x[8];
        bool raw = false;
        if (ent & 1024) {
          #pragma unroll
          for (int j = 0; j < 8; ++j) x[j] = *(LDS u32x4*)(sl + j * 1024 + lane * 16);
          wave_sync();
          ctl_st(rel, n + 1);
        } else if (isx && kb < QKB) {
          u32x4 c[6];
          #pragma unroll
          for (int j = 0; j < 6; ++j) c[j] = *(LDS u32x4*)(sl + j * 1024 + lane * 16);
          const u32 sd = *(LDS u32*)(sl + IU_SIDE + lane * 4);
          wave_sync();
          ctl_st(rel, n + 1);
          const u32 hdr = (u32)__builtin_amdgcn_readfirstlane((int)sd);
          raw = hdr >> 31;
          if (!raw) {
            const u32 cnt = hdr & 0xFFFFu, e4 = ((hdr >> 16) & 31u) * 0x04040404u;
            #pragma unroll
            for (int j = 0; j < 8; ++j) {
              const u32 p0 = c[(j * 3) >> 2][(j * 3) & 3], p1 = c[(j * 3 + 1) >> 2][(j * 3 + 1) & 3], p2 = c[(j * 3 + 2) >> 2][(j * 3 + 2) & 3];
              const u32 o3 = bfi(0x80808080u, p2 << 1, bfi(0x10101010u, p2 >> 1, bfi(0x0C0C0C0Cu, p1 >> 3, p0 >> 5)));
              const u32x4 y = {(p0 & 0x9F9F9F9Fu) + e4, (p1 & 0x9F9F9F9Fu) + e4, (p2 & 0x9F9F9F9Fu) + e4, (o3 & 0x9F9F9F9Fu) + e4};
              *(LDS u32x4*)(scr + j * 1024 + lane * 16) = y;
            }
            wave_sync();
            if (lane >= 1) {
              if (2u * lane - 2u < cnt) patch_in(scr, sd & 0xFFFFu);
              if (2u * lane - 1u < cnt) patch_in(scr, sd >> 16);
            }
            if (cnt > IN_CAP0) {
              for (int j = 1; j < 4; ++j) {
                const u32 s2 = __builtin_amdgcn_raw_buffer_load_b32(rsP, lane * 4 + j * 256, offE + IU_SIDE, AUX_SYS);
                const u32 id = j * 128 + 2 * lane - 2;
                if (id < cnt) patch_in(scr, s2 & 0xFFFFu);
                if (id + 1 < cnt) patch_in(scr, s2 >> 16);
              }
            }
            wave_sync();
            #pragma unroll
            for (int j = 0; j < 8; ++j) x[j] = *(LDS u32x4*)(scr + j * 1024 + lane * 16);
          }
        } else {
          LDS char* var = sl + QV;
          u32x4 c[5];
          #pragma unroll
          for (int j = 0; j < 5; ++j) c[j] = *(LDS u32x4*)(sl + j * 1024 + lane * 16);
          const u32 hdr = (u32)__builtin_amdgcn_readfirstlane((int)*(LDS u32*)var);
          const u32 pe = *(LDS unsigned short*)(var + 4 + (lane & 31) * 2);
          raw = hdr >> 31;
          if (!raw) {
            const u32 nbit = hdr & 0x7FFFu, npat = (hdr >> 15) & 0x1FFu, bias = (4u * ((hdr >> 24) & 31u) - 52u) * 0x01010101u;
            if (nbit > Q_ESC0 * 3 && lane < (int)(QV_END - QV0) / 16)
              *(LDS u32x4*)(var + QV0 + lane * 16) = __builtin_amdgcn_raw_buffer_load_b128(rsP, lane * 16, offE + QV + QV0, AUX_SYS);
            u32 a[32], qo[32], bits = 0;
            #pragma unroll
            for (int g = 0; g < 4; ++g) {
              u32 k5[5];
              #pragma unroll
              for (int k = 0; k < 5; ++k) { k5[k] = c[(g * 5 + k) >> 2][(g * 5 + k) & 3]; a[g * 8 + k] = k5[k]; }
              a[g * 8 + 5] = bfi(0x80808080u, k5[4] << 3, bfi(0x08080808u, k5[3] >> 1, k5[0] >> 4));
              a[g * 8 + 6] = bfi(0x80808080u, k5[4] << 2, bfi(0x08080808u, k5[3] >> 2, k5[1] >> 4));
              a[g * 8 + 7] = bfi(0x80808080u, k5[4] << 1, bfi(0x08080808u, k5[3] >> 3, k5[2] >> 4));
            }
            u32 es[32];
            #pragma unroll
            for (int d = 0; d < 32; ++d) {
              es[d] = a[d] & (a[d] >> 1) & 0x04040404u;
              qo[d] = bits;
              bits = __builtin_amdgcn_udot4(es[d], 0x03030303u, bits, false);
            }
            const u32 base4 = (((u32)(uintptr_t)var + QV_STR) * 8 + wave_scan_u32(bits >> 2)) * 4 - bits;
            wave_sync();
            #pragma unroll
            for (int d = 0; d < 32; ++d) {
              const u32 q4 = base4 + qo[d];
              const LDS u32* at = (const LDS u32*)(uintptr_t)((q4 >> 5) & ~3u);
              qo[d] = __builtin_amdgcn_alignbit(at[1], at[0], q4 >> 2);
            }
            wave_sync();
            ctl_st(rel, n + 1);
            #pragma unroll
            for (int j = 0; j < 8; ++j) {
              u32x4 y;
              #pragma unroll
              for (int k = 0; k < 4; ++k) {
                const u32 z = qo[j * 4 + k], z2 = bfi(0x0000003Fu, z, z << 10);
                const u32 sp = bfi(0x00070007u, z2, z2 << 5) & 0x07070707u;
                const u32 sel = __umul24(es[j * 4 + k], 0x404040u) + es[j * 4 + k];
                const u32 code = __builtin_amdgcn_perm(sp, 0x03030303u, sel);
                y[k] = (a[j * 4 + k] & 0x8F8F8F8Fu) + __builtin_amdgcn_perm(0x0C101418u, 0x28000408u, code) + bias;
              }
              *(LDS u32x4*)(scr + j * 1024 + lane * 16) = y;
            }
            wave_sync();
            if ((u32)lane < npat && lane < (int)Q_PAT0) patch_short(scr, pe);
            if (npat > Q_PAT0) {
              for (int j = 0; j < 2; ++j) {
                const u32 s2 = __builtin_amdgcn_raw_buffer_load_b32(rsP, lane * 4 + j * 256, offE + QV + QV_END, AUX_SYS);
                const u32 id = Q_PAT0 + j * 128 + 2 * lane;
                if (id < npat) patch_short(scr, s2 & 0xFFFFu);
                if (id + 1 < npat) patch_short(scr, s2 >> 16);
              }
            }
            wave_sync();
            #pragma unroll
            for (int j = 0; j < 8; ++j) x[j] = *(LDS u32x4*)(scr + j * 1024 + lane * 16);
          } else {
            wave_sync();
            ctl_st(rel, n + 1);
          }
        }
        if (raw) {
          #pragma unroll
          for (int j = 0; j < 8; ++j) x[j] = __builtin_amdgcn_raw_buffer_load_b128(rsP, lane * 16 + j * 1024, par + EX_INR + uid * IU_RAW, AUX_SYS);
        }
        store_raw_unit(x, rsL, dst, lane);
        USTAMP(2);
        vm_drain();
        cnt_add(p.ws, (isx ? rg : 64 + (rg >> 1)) * 5 + kb, lane);
        USTAMP(3);
        continue;
      }
      const int t = i - 512, cg = t / 5, rg = t - cg * 5 + (same ? 0 : 3);
      const u32 obase = (rg * 64 * NN + group_of(src, cg >> 4) * 1024 + (cg & 15) * 64) * 4;
      const u32 olane = (li * NN + lg * 4) * 4;
      if (!ok) {
        const u32x4 nan4 = {0x7FC00000u, 0x7FC00000u, 0x7FC00000u, 0x7FC00000u};
        #pragma unroll
        for (int e = 0; e < 16; ++e) ST128(nan4, rsO, olane + (e & 3) * 64, obase + (e >> 2) * 16 * NN * 4, OUT_AUX);
        continue;
      }
      landed(endp - 9);
      const u32 odd = *(LDS unsigned short*)(sl + OU_SIDE + lane * 2) | ((u32)*(LDS unsigned char*)(sl + OU_SIDE + 128 + lane) << 16);
      const u32 hdr = (u32)__builtin_amdgcn_readfirstlane((int)*(LDS u32*)(sl + OU_SIDE + 192));
      u32x2 sd = {*(LDS unsigned short*)(sl + OU_SIDE + 196 + li * 2), *(LDS u32*)(sl + OU_SIDE + 216 + (li < 10 ? li : 0) * 4)};
      const u32 uid = rank * 320 + t;
      if (!(hdr >> 31)) {
        const u32 cnt = hdr & 0xFFFFu; const int g = (int)((hdr >> 16) & 0x1FFu) - 256;
        if (cnt > OUT_CAP0) {
          const u32x2 s2 = __builtin_amdgcn_raw_buffer_load_b64(rsP, lane * 8, par + EX_OUE + uid * OU_ENC + OU_SIDE + 256 - 80, AUX_SYS);
          if (lane >= (int)OUT_CAP0) sd = s2;
        }
        const u32 exg = (u32)lane < cnt ? (sd.x >> 8) & 15u : 16u;
        u64 om = (1ull << (odd & 63)) | (1ull << ((odd >> 6) & 63)) | (1ull << ((odd >> 12) & 63)) | (1ull << ((odd >> 18) & 63));
        if ((odd & 63) > ((odd >> 6) & 63)) om = 0;
        const u32 ob[2] = {(u32)om, (u32)(om >> 32)};
        const float half = unfix(1, g);
        #pragma unroll
        for (int mi = 0; mi < 4; ++mi) {
          if (mi) landed(endp - 9 + 3 * mi);
          u32x4 c[3];
          #pragma unroll
          for (int j = 0; j < 3; ++j) c[j] = *(LDS u32x4*)(sl + (mi * 3 + j) * 1024 + lane * 16);
          if (mi == 3) { wave_sync(); ctl_st(C_REL + (n % NSLOT) * 4, n + 1); }
          #pragma unroll
          for (int ni = 0; ni < 4; ++ni) {
            const int k = ni * 3, e = (mi * 4 + ni) * 4;
            const u32 d0 = c[k >> 2][k & 3], d1 = c[(k + 1) >> 2][(k + 1) & 3], d2 = c[(k + 2) >> 2][(k + 2) & 3];
            const u32 hi[4] = {d0 << 8, __builtin_amdgcn_perm(d1, d0, 0x0504030Cu), __builtin_amdgcn_perm(d2, d1, 0x0403020Cu), d2 & 0xFFFFFF00u};
            float y[4];
            #pragma unroll
            for (int r = 0; r < 4; ++r)
              y[r] = unfix(((int)hi[r] >> 7) | (int)__builtin_amdgcn_ubfe(ob[(e + r) >> 5], (e + r) & 31, 1), g);
            u64 em = __ballot(exg == (u32)(mi * 4 + ni));
            while (em) {
              const int src_l = __builtin_ctzll(em); em &= em - 1;
              const u32 pos = (u32)__builtin_amdgcn_readlane((int)sd.x, src_l), rel = (pos & 255u) - (u32)lane * 4u;
              const float vv = as_f((u32)__builtin_amdgcn_readlane((int)sd.y, src_l));
              #pragma unroll
              for (int r = 0; r < 4; ++r) y[r] = rel == (u32)r ? (pos >> 15 ? y[r] + half : vv) : y[r];
            }
            ST128((u32x4{as_u(y[0] * scale), as_u(y[1] * scale), as_u(y[2] * scale), as_u(y[3] * scale)}), rsO, olane + ni * 64, obase + mi * 16 * NN * 4, OUT_AUX);
          }
        }
      } else {
        landed(endp);
        ctl_st(C_REL + (n % NSLOT) * 4, n + 1);
        #pragma unroll
        for (int mi = 0; mi < 4; ++mi) {
          u32x4 r4[4];
          #pragma unroll
          for (int ni = 0; ni < 4; ++ni) r4[ni] = __builtin_amdgcn_raw_buffer_load_b128(rsP, lane * 16 + ni * 1024, par + EX_OUR + uid * OU_RAW + mi * 4096, AUX_SYS);
          #pragma unroll
          for (int ni = 0; ni < 4; ++ni) {
            const f32x4 y = __builtin_bit_cast(f32x4, r4[ni]) * scale;
            ST128(__builtin_bit_cast(u32x4, y), rsO, olane + ni * 64, obase + mi * 16 * NN * 4, OUT_AUX);
          }
        }
      }
    }
    STAMP(4);
    return;
  }

  constexpr int SLOTS = NGEMM / 8;
  const int xc = b & 7, slot = b >> 3, wm = wave >> 1, wn = wave & 1;
  for (int ti = slot; ti < 32; ti += SLOTS) {
    const int j = xc >> 1;
    const int c = 4 * (xc & 1) + (ti < 24 ? 2 * ((ti % 12) / 6) + ti / 12 : (ti - 24) >> 1), cb = j * 8 + c;
    auto row_group = [&](int k) -> int {
      const int r = 4 * (ti & 1) + k, t = (ti % 6) >> 1;
      if (ti >= 24) return 8 * rank + r;
      return 8 * (r < split_of(rank, j, c) ? (rank & 4) | ((rank + 1 + t) & 3) : (~rank & 4) | ((j + 1 + t) & 3)) + r;
    };
    const int rgw = row_group(wm);
    const u32 deadw = (u32)(uintptr_t)(smem + L_CTL);
    auto ready = [&](int grp) {
      const bool isx = lane < 4;
      const int rgl = row_group(lane & 3);
      const u32 off = OFF_CNT + ((isx ? rgl : 64 + cb) * 5 + grp) * 128, want = (isx ? 1u : 2u) * (grp == 4 ? 4u : 7u) * call;
      if (!wait_ge(p.ws, off, want, call)) ctl_st(deadw, 1u);
    };
    const u32 xo0 = OFF_XA + rgw * IU_RAW + wn * 4096, wo0 = OFF_WA + cb * 2 * IU_RAW + wave * 2048;
    auto dma = [&](int k, int ks, u32 st) {
      const u32 xo = xo0 + ks * (64 * IU_RAW), wo = wo0 + ks * (64 * IU_RAW);
      LDS char* dx = smem + st + wave * 4096;
      LDS char* dw = smem + st + 32 * KiB + wave * 2048;
      if (k == 0) __builtin_amdgcn_raw_ptr_buffer_load_lds(rsL, (LDS void*)dx, 16, lane * 16, xo, 0, AUX_SYS);
      if (k == 1) __builtin_amdgcn_raw_ptr_buffer_load_lds(rsL, (LDS void*)dx, 16, lane * 16, xo, 1024, AUX_SYS);
      if (k == 2) __builtin_amdgcn_raw_ptr_buffer_load_lds(rsL, (LDS void*)dx, 16, lane * 16, xo, 2048, AUX_SYS);
      if (k == 3) __builtin_amdgcn_raw_ptr_buffer_load_lds(rsL, (LDS void*)dx, 16, lane * 16, xo, 3072, AUX_SYS);
      if (k == 4) __builtin_amdgcn_raw_ptr_buffer_load_lds(rsL, (LDS void*)dw, 16, lane * 16, wo, 0, AUX_SYS);
      if (k == 5) __builtin_amdgcn_raw_ptr_buffer_load_lds(rsL, (LDS void*)dw, 16, lane * 16, wo, 1024, AUX_SYS);
    };
    f32x4 acc[4][4];
    #pragma unroll
    for (int mi = 0; mi < 4; ++mi)
      #pragma unroll
      for (int ni = 0; ni < 4; ++ni) acc[mi][ni] = f32x4{0.f, 0.f, 0.f, 0.f};
    if (wave == 0) { if (ti == slot) ctl_st(deadw, 0u); ready(4); }
    asm volatile("s_waitcnt vmcnt(0) lgkmcnt(0)\n\ts_barrier" ::: "memory");
    if (ti == slot) { STAMP(2); STAMPC(14); }
    #pragma unroll
    for (int k = 0; k < 12; ++k) dma(k % 6, rank * 4 + k / 6, k / 6 * STAGE);
    u32 st = 0;
    int ksn = rank * 4 + 2, jn = -2, kbn = 0;
    i32x8 fa[8], fb[8];
    auto step = [&](i32x8 (&cur)[8], const i32x8 (&prev)[8], bool first, bool more, bool last) {
      if (more && wave == 0 && jn == 0) { ready(kbn); if (ti == slot) STAMP(10 + kbn); }
      if (!last) asm volatile("s_waitcnt vmcnt(6) lgkmcnt(0)\n\ts_barrier" ::: "memory");
      else asm volatile("s_waitcnt vmcnt(0) lgkmcnt(0)\n\ts_barrier" ::: "memory");
      const u32 st2 = st == 0 ? 2 * STAGE : st - STAGE;
      LDS char* ax = smem + st + wm * 8192 + lane * 16;
      LDS char* aw = smem + st + 32 * KiB + wn * 8192 + lane * 16;
      u32x4 h[16];
      auto M = [&](int k) {
        if (!first) acc[k >> 2][k & 3] = __builtin_amdgcn_mfma_scale_f32_16x16x128_f8f6f4(prev[k & 3], prev[4 + (k >> 2)], acc[k >> 2][k & 3], 1, 1, 0, 0, 0, 0);
        __builtin_amdgcn_sched_barrier(0);
      };
      auto R = [&](int k) { h[k] = *(LDS u32x4*)(((k >> 1) & 1 ? ax : aw) + (k >> 2) * 2048 + (k & 1) * 1024); __builtin_amdgcn_sched_barrier(0); };
      __builtin_amdgcn_sched_barrier(0);
      #pragma unroll
      for (int k = 0; k < 6; ++k) {
        M(2 * k); R(2 * k); R(2 * k + 1); M(2 * k + 1);
        if (more) dma(k, ksn, st2);
        __builtin_amdgcn_sched_barrier(0);
      }
      M(12); R(12); R(13); M(13); R(14); R(15); M(14); M(15);
      #pragma unroll
      for (int f = 0; f < 4; ++f) {
        cur[f] = __builtin_bit_cast(i32x8, __builtin_shufflevector(h[4 * f], h[4 * f + 1], 0, 1, 2, 3, 4, 5, 6, 7));
        cur[4 + f] = __builtin_bit_cast(i32x8, __builtin_shufflevector(h[4 * f + 2], h[4 * f + 3], 0, 1, 2, 3, 4, 5, 6, 7));
      }
      if (++jn == 7) { jn = 0; ++kbn; }
      ksn = jn < 0 ? ksn + 1 : ((rank + 1 + jn) & 7) * 4 + kbn;
      st = st == 2 * STAGE ? 0 : st + STAGE;
    };
    step(fa, fb, true, true, false);
    #pragma unroll 1
    for (int s = 1; s < 29; s += 2) { step(fb, fa, false, true, false); step(fa, fb, false, true, false); }
    step(fb, fa, false, true, false);
    step(fa, fb, false, false, false);
    step(fb, fa, false, false, true);
    #pragma unroll
    for (int k = 0; k < 16; ++k)
      acc[k >> 2][k & 3] = __builtin_amdgcn_mfma_scale_f32_16x16x128_f8f6f4(fb[k & 3], fb[4 + (k >> 2)], acc[k >> 2][k & 3], 1, 1, 0, 0, 0, 0);
    if (ti == slot) { STAMP(3); STAMPC(15); }
    int lz = lane; asm volatile("" : "+v"(lz));
    const bool dead = ctl_ld(deadw) != 0u;
    const int owner = rgw >> 3;
    if (owner == rank) {
      const u32 obase = ((rgw & 7) * 64 * NN + group_of(rank, j) * 1024 + c * 128 + wn * 64) * 4, olane = ((lz & 15) * NN + (lz >> 4) * 4) * 4;
      const float sc = dead ? __builtin_nanf("") : scale;
      #pragma unroll
      for (int mi = 0; mi < 4; ++mi)
        #pragma unroll
        for (int ni = 0; ni < 4; ++ni)
          ST128(__builtin_bit_cast(u32x4, acc[mi][ni] * sc), rsO, olane + ni * 64, obase + mi * 16 * NN * 4, OUT_AUX);
    } else if (!dead) {
      const int t = (cb * 2 + wn) * 5 + (rgw & 7) - ((owner >> 2) == (rank >> 2) ? 0 : 3);
      const u32 uid = owner * 320 + t;
      const u32 offE = par + EX_OUE + uid * OU_ENC;
      float sa = 0.f;
      #pragma unroll
      for (int mi = 0; mi < 4; ++mi)
        #pragma unroll
        for (int ni = 0; ni < 4; ++ni) sa += __builtin_fabsf(acc[mi][ni][(mi + ni) & 3]);
      const int g = max(0, 24 - ((int)((as_u(wave_sum_f32(sa) * (3.75f / 1024.f)) >> 23) & 255u) - 126));
      LDS char* side = smem + L_EPI + wave * 1024;
      u32 cnt = 0, sg[2] = {0u, 0u}, skip[2] = {0u, 0u};
      auto put = [&](bool on, u64 bm, u32 pos, u32 val) {
        const u32 sl = cnt + __builtin_amdgcn_mbcnt_hi((u32)(bm >> 32), __builtin_amdgcn_mbcnt_lo((u32)bm, 0u));
        if (on && sl < OUT_CAP0) { *(LDS unsigned short*)(side + 196 + sl * 2) = (unsigned short)pos; *(LDS u32*)(side + 216 + sl * 4) = val; }
        else if (on && sl < OUT_CAP) *(LDS u32x2*)(side + 256 - 80 + sl * 8) = u32x2{pos, val};
        cnt += __popcll(bm);
      };
      #pragma unroll
      for (int mi = 0; mi < 4; ++mi) {
        u32 pk[12];
        #pragma unroll
        for (int ni = 0; ni < 4; ++ni) {
          u32 qv[4]; u64 bm[4];
          #pragma unroll
          for (int r = 0; r < 4; ++r) {
            const float tt = __builtin_amdgcn_ldexpf(acc[mi][ni][r], g);
            const int iv = (int)__builtin_amdgcn_fmed3f(tt, -16777216.f, 16777215.f);
            qv[r] = (u32)iv >> 1;
            const int e = (mi * 4 + ni) * 4 + r;
            sg[e >> 5] = __builtin_amdgcn_alignbit((u32)iv, sg[e >> 5], 1);
            bm[r] = __ballot(as_u((float)iv) != as_u(tt));
          }
          if (bm[0] | bm[1] | bm[2] | bm[3]) {
            #pragma unroll
            for (int r = 0; r < 4; ++r) {
              const bool on = (bm[r] >> lz) & 1;
              const float vv = acc[mi][ni][r];
              const int e = (mi * 4 + ni) * 4 + r;
              put(on, bm[r], (u32)((mi * 4 + ni) * 256 + lz * 4 + r), as_u(vv));
              if (on) skip[e >> 5] |= 1u << (e & 31);
            }
          }
          pk[ni * 3 + 0] = __builtin_amdgcn_perm(qv[1], qv[0], 0x04020100u);
          pk[ni * 3 + 1] = __builtin_amdgcn_perm(qv[2], qv[1], 0x05040201u);
          pk[ni * 3 + 2] = __builtin_amdgcn_perm(qv[3], qv[2], 0x06050402u);
        }
        #pragma unroll
        for (int j = 0; j < 3; ++j)
          ST128((u32x4{pk[j * 4], pk[j * 4 + 1], pk[j * 4 + 2], pk[j * 4 + 3]}), rsL, lz * 16 + j * 1024, offE + mi * 3072, AUX_SYS);
      }
      u64 om = ((u64)(sg[1] & ~skip[1]) << 32) | (sg[0] & ~skip[0]);
      u32 odd = om ? 0u : 1u, last = 0;
      #pragma unroll
      for (int k = 0; k < 4; ++k) { if (om) last = (u32)__builtin_ctzll(om); om &= om - 1; odd |= last << (6 * k); }
      for (u64 bm; (bm = __ballot(om != 0)) != 0 && cnt <= OUT_CAP; om &= om - 1) {
        const u32 e = (u32)__builtin_ctzll(om | (1ull << 63));
        put(om != 0, bm, 0x8000u | ((e >> 2) << 8) | (lz * 4 + (e & 3)), 0u);
      }
      const bool raw = cnt > OUT_CAP;
      *(LDS unsigned short*)(side + lz * 2) = (unsigned short)odd;
      *(LDS unsigned char*)(side + 128 + lz) = (unsigned char)(odd >> 16);
      if (lz == 0) *(LDS u32*)(side + 192) = raw ? 0x80000000u : (cnt | ((u32)(g + 256) << 16));
      wave_sync();
      __builtin_amdgcn_raw_buffer_store_b32(*(LDS u32*)(side + lz * 4), rsL, lz * 4, offE + OU_SIDE, AUX_SYS);
      if (cnt > OUT_CAP0) __builtin_amdgcn_raw_buffer_store_b64(*(LDS u32x2*)(side + 256 + lz * 8), rsL, lz * 8 + 256, offE + OU_SIDE, AUX_SYS);
      if (raw) {
        const u32 offR = par + EX_OUR + uid * OU_RAW;
        #pragma unroll
        for (int mi = 0; mi < 4; ++mi)
          #pragma unroll
          for (int ni = 0; ni < 4; ++ni)
            ST128(__builtin_bit_cast(u32x4, acc[mi][ni]), rsL, lz * 16 + ni * 1024, offR + mi * 4096, AUX_SYS);
      }
      if (ti == slot && b < 96) STAMP(9);
      vm_drain();
      if (lz == owner)
        __hip_atomic_store((GLB u32*)(peer_ws + OFF_FOUT + (rank * 320 + t) * 128), call, __ATOMIC_RELAXED, __HIP_MEMORY_SCOPE_SYSTEM);
    }
    if (ti == slot) STAMP(4); else STAMP(5);
  }
}

}

extern "C" void kernel_launch(int rank, int nranks, ncclComm_t comm, hipStream_t stream,
                              void* const* d_in, const int* in_sizes, int n_in,
                              void* d_out, int out_size,
                              const int* in_dtypes, int out_dtype,
                              void* const* d_out_all,
                              void* const* d_ws_all, size_t ws_bytes) {
  (void)comm; (void)in_sizes; (void)n_in; (void)out_size; (void)in_dtypes; (void)out_dtype; (void)d_out_all;
  if (nranks != NR || (unsigned)rank >= (unsigned)NR || ws_bytes < WS_NEED) { fprintf(stderr, "unexpected launch shape\n"); return; }
  static unsigned attr_set = 0;
  if (!(attr_set >> rank & 1u)) {
    HIP_CHECK(hipFuncSetAttribute((const void*)gemm_rs_kernel, hipFuncAttributeMaxDynamicSharedMemorySize, (int)LDS_BYTES));
    attr_set |= 1u << rank;
  }
  Params p;
  p.x = (const uint8_t*)d_in[0]; p.w = (const uint8_t*)d_in[1]; p.sx = (const float*)d_in[2]; p.sw = (const float*)d_in[3];
  p.out = (float*)d_out; p.ws_all = (char* const*)d_ws_all; p.ws = (char*)d_ws_all[rank]; p.rank = rank; p.ws_bytes = WS_NEED;
  hipLaunchKernelGGL(gemm_rs_kernel, dim3(NWG), dim3(NTHR), LDS_BYTES, stream, p);
  HIP_CHECK(hipGetLastError());
}
```

```cpp
#include <hip/hip_runtime.h>
#include <rccl/rccl.h>
#include <cstdio>
#include <cstdint>

#define HIP_CHECK(expr) do { hipError_t _e = (expr); if (_e != hipSuccess) { \
    fprintf(stderr, "HIP error %s at %s:%d\n", hipGetErrorString(_e), __FILE__, __LINE__); \
    return; } } while (0)

#define PULL_WGS 8
#define WIN 8
#define LOOK 4
#define NRAW 8
#define NA 3
#define LOOK_OUT 2
#define EPI_AUX AUX_SYS
#define WIN_OUT 8
#define OWN_AUX 17
#define PULL_AUX 17
#define GEMM_AUX 17
#define OUT_AUX 17

namespace {

typedef unsigned u32;
typedef unsigned long long u64;
typedef unsigned u32x4 __attribute__((ext_vector_type(4)));
typedef unsigned u32x2 __attribute__((ext_vector_type(2)));
typedef int i32x2 __attribute__((ext_vector_type(2)));
typedef int i32x8 __attribute__((ext_vector_type(8)));
typedef float f32x4 __attribute__((ext_vector_type(4)));
typedef __amdgpu_buffer_rsrc_t rsrc_t;
#define LDS __attribute__((address_space(3)))
#define GLB __attribute__((address_space(1)))

constexpr int NR = 8, MM = 4096, NN = 16384, KS = 512;
constexpr int NWG = 256, NTHR = 512;
constexpr int NPULL = 7 * PULL_WGS, NGEMM = NWG - NPULL;
constexpr int AUX_SYS = 17;
constexpr u64 DEADLINE_TICKS = 40000000ull;

constexpr u32 KiB = 1024, MiB = 1024 * 1024;
constexpr u32 OFF_CALL = 0;
constexpr u32 OFF_TMO = 32 * KiB;
constexpr u32 OFF_TIC = 33 * KiB;
constexpr u32 OFF_CNT = 40 * KiB;
constexpr u32 OFF_FIN = 128 * KiB;
constexpr u32 OFF_FOUT = OFF_FIN + 8 * 576 * 128;
constexpr u32 OFF_DBG = 1280 * KiB;
constexpr u32 OFF_XA = 2 * MiB;
constexpr u32 OFF_WA = OFF_XA + 16 * MiB;
constexpr u32 OFF_PAR = OFF_WA + 16 * MiB;
constexpr u32 IU_ENC = 8 * KiB, IU_RAW = 8 * KiB;
constexpr u32 OU_ENC = 13056, OU_RAW = 16 * KiB;
constexpr u32 IU_SIDE = 6144, OU_SIDE = 12288;
constexpr u32 EX_INE = 0;
constexpr u32 EX_INR = EX_INE + 1280 * IU_ENC;
constexpr u32 EX_OUE = EX_INR + 1280 * IU_RAW;
constexpr u32 EX_OUR = EX_OUE + 2560 * OU_ENC;
constexpr u32 PAR_BYTES = EX_OUR + 2560 * OU_RAW;
constexpr u32 WS_NEED = OFF_PAR + 2 * PAR_BYTES;
static_assert(OFF_CNT + 96 * 5 * 128 <= OFF_FIN && OFF_FOUT + 8 * 320 * 128 <= OFF_DBG, "flags");
static_assert(WS_NEED <= 257u * MiB, "workspace");
constexpr u32 QV = 5120, QV0 = 896, QV_STR = 64, QV_END = 1792, QV_ALL = 2304;
constexpr u32 Q_ESC0 = (QV0 - QV_STR) * 8 / 3, Q_ESC = (QV_END - QV_STR) * 8 / 3, Q_PAT0 = 30, Q_PAT = 286;
constexpr u32 IN_CAP0 = 62, IN_CAP = 254;
constexpr u32 OUT_CAP0 = 10, OUT_CAP = 64;

constexpr u32 STAGE = 48 * KiB;
constexpr u32 L_SCR = 64 * KiB, SCR_W = 10752;
constexpr u32 NSLOT = 8, SLOT = 13 * KiB, NCONS = 6;
constexpr u32 L_PSCR = NSLOT * SLOT;
constexpr u32 L_CTL = L_PSCR + NCONS * 8 * KiB;
constexpr u32 LDS_BYTES = L_CTL + 256;
static_assert(3 * STAGE <= L_CTL && L_SCR + 8 * SCR_W <= LDS_BYTES && LDS_BYTES <= 160 * KiB, "lds");

__device__ __forceinline__ int group_of(int rank, int j) { return rank < 4 ? rank * 4 + j : j * 4 + (rank & 3); }

struct Params {
  const uint8_t* x; const uint8_t* w; const float* sx; const float* sw;
  float* out; char* const* ws_all; char* ws; int rank; u32 ws_bytes;
};

__device__ __forceinline__ rsrc_t mk_rsrc(const void* p, u32 bytes) {
  u64 a = (u64)p;
  u32 lo = __builtin_amdgcn_readfirstlane((u32)a), hi = __builtin_amdgcn_readfirstlane((u32)(a >> 32));
  return __builtin_amdgcn_make_buffer_rsrc((void*)(((u64)hi << 32) | lo), 0, (int)bytes, 0x00020000);
}
#define ST128(v, rs, voff, soff, aux) __builtin_amdgcn_raw_buffer_store_b128((v), (rs), (int)((u32)(voff) + (u32)(soff)), 0, (aux))
__device__ __forceinline__ float as_f(u32 v) { return __builtin_bit_cast(float, v); }
__device__ __forceinline__ u32 as_u(float v) { return __builtin_bit_cast(u32, v); }
__device__ __forceinline__ void vm_drain() { asm volatile("s_waitcnt vmcnt(0)" ::: "memory"); }
__device__ __forceinline__ void wave_sync() {
  asm volatile("s_waitcnt lgkmcnt(0)" ::: "memory");
}
__device__ __forceinline__ u32 bfi(u32 m, u32 a, u32 b) { return __builtin_amdgcn_bitop3_b32(m, a, b, 0xCA); }
template <int CTRL, int ROWS> __device__ __forceinline__ u32 dpp0(u32 v) { return (u32)__builtin_amdgcn_update_dpp(0, (int)v, CTRL, ROWS, 0xf, false); }
__device__ __forceinline__ u32 wave_scan_u32(u32 v) {
  v += dpp0<0x111, 0xf>(v); v += dpp0<0x112, 0xf>(v); v += dpp0<0x114, 0xf>(v); v += dpp0<0x118, 0xf>(v);
  v += dpp0<0x142, 0xa>(v); v += dpp0<0x143, 0xc>(v);
  return v;
}
__device__ __forceinline__ float wave_sum_f32(float v) {
  v += as_f(dpp0<0x111, 0xf>(as_u(v))); v += as_f(dpp0<0x112, 0xf>(as_u(v))); v += as_f(dpp0<0x114, 0xf>(as_u(v)));
  v += as_f(dpp0<0x118, 0xf>(as_u(v))); v += as_f(dpp0<0x142, 0xa>(as_u(v))); v += as_f(dpp0<0x143, 0xc>(as_u(v)));
  return as_f((u32)__builtin_amdgcn_readlane((int)as_u(v), 63));
}
__device__ __forceinline__ u32 ctl_ld(u32 addr) {
  u32 v; asm volatile("ds_read_b32 %0, %1\n\ts_waitcnt lgkmcnt(0)" : "=&v"(v) : "v"(addr) : "memory");
  return (u32)__builtin_amdgcn_readfirstlane((int)v);
}
__device__ __forceinline__ void ctl_st(u32 addr, u32 v) { asm volatile("ds_write_b32 %0, %1" :: "v"(addr), "v"(v) : "memory"); }
__device__ __forceinline__ u32 ld_flag(const char* ws, u32 off) {
  return __hip_atomic_load((const GLB u32*)(ws + off), __ATOMIC_RELAXED, __HIP_MEMORY_SCOPE_SYSTEM);
}
__device__ __forceinline__ bool wait_ge(char* ws, u32 off, u32 target, u32 call) {
  if (__all(ld_flag(ws, off) >= target)) return true;
  const u64 t0 = __builtin_amdgcn_s_memrealtime();
  GLB u32* tmo = (GLB u32*)(ws + OFF_TMO);
  for (;;) {
    __builtin_amdgcn_s_sleep(4);
    if (__all(ld_flag(ws, off) >= target)) return true;
    if (__hip_atomic_load(tmo, __ATOMIC_RELAXED, __HIP_MEMORY_SCOPE_AGENT) == call ||
        __builtin_amdgcn_s_memrealtime() - t0 > DEADLINE_TICKS) {
      __hip_atomic_store(tmo, call, __ATOMIC_RELAXED, __HIP_MEMORY_SCOPE_AGENT);
      return false;
    }
  }
}
__device__ __forceinline__ void cnt_add(char* ws, u32 idx, int lane) {
  if (lane == 0) __hip_atomic_fetch_add((GLB u32*)(ws + OFF_CNT + idx * 128), 1u, __ATOMIC_RELAXED, __HIP_MEMORY_SCOPE_AGENT);
}
#define STAMPC(slot) do {} while (0)
#define USTAMP(k) do {} while (0)
#define STAMP(slot) do {} while (0)

__device__ __forceinline__ void store_raw_unit(const u32x4 (&x)[8], rsrc_t rs, u32 off, int lane) {
  #pragma unroll
  for (int j = 0; j < 8; ++j) ST128(x[j], rs, lane * 16 + j * 1024, off, AUX_SYS);
}

__device__ __forceinline__ void export_half(const u32x4 (&x)[4], rsrc_t rs, u32 par, u32 uid, int hf, LDS char* scr, int lane) {
  const u32 offE = par + EX_INE + uid * IU_ENC;
  u32 s = 0;
  #pragma unroll
  for (int j = 0; j < 4; ++j) {
    *(LDS u32x4*)(scr + j * 1024 + lane * 16) = x[j];
    s = __builtin_amdgcn_sad_u8(x[j][j] & 0x7C7C7C7Cu, 0u, s);
  }
  s = (u32)__builtin_amdgcn_readlane((int)wave_scan_u32(s), 63);
  int e0 = ((int)s - 17121) >> 12;
  e0 = e0 < 0 ? 0 : (e0 > 24 ? 24 : e0);
  const u32 e4 = (u32)e0 * 0x04040404u;
  u32 mk[2] = {0, 0};
  u32 pk[12];
  #pragma unroll
  for (int j = 0; j < 4; ++j) {
    u32 o[4];
    #pragma unroll
    for (int c = 0; c < 4; ++c) {
      const u32 u = (x[j][c] | 0x80808080u) - e4;
      o[c] = bfi(0x1F1F1F1Fu, u, x[j][c]);
      const u32 t = (((u >> 1) ^ 0x40404040u) & 0x70707070u) + 0x70707070u;
      const int d = j * 4 + c;
      mk[d >> 3] = bfi(0x80808080u >> (d & 7), t >> (d & 7), mk[d >> 3]);
    }
    pk[j * 3 + 0] = bfi(0x60606060u, o[3] << 5, o[0]);
    pk[j * 3 + 1] = bfi(0x60606060u, o[3] << 3, o[1]);
    pk[j * 3 + 2] = bfi(0x60606060u, bfi(0x40404040u, o[3] >> 1, o[3] << 1), o[2]);
  }
  #pragma unroll
  for (int j = 0; j < 3; ++j)
    ST128((u32x4{pk[j * 4], pk[j * 4 + 1], pk[j * 4 + 2], pk[j * 4 + 3]}), rs, lane * 16 + j * 1024, offE + hf * 3072, AUX_SYS);
  const u32 mine = __popc(mk[0]) + __popc(mk[1]);
  const u32 inc = wave_scan_u32(mine);
  const u32 total = (u32)__builtin_amdgcn_readlane((int)inc, 63);
  if (total <= IN_CAP) {
    LDS unsigned short* list = (LDS unsigned short*)(scr + 8192);
    u32 slot = 2 + inc - mine;
    #pragma unroll
    for (int q = 0; q < 2; ++q) {
      u32 m = mk[q];
      while (m) {
        const int bp = __builtin_ctz(m); m &= m - 1;
        const int d = q * 8 + 7 - (bp & 7);
        list[slot++] = (unsigned short)((d >> 2) * 1024 + lane * 16 + (d & 3) * 4 + (bp >> 3));
      }
    }
    wave_sync();
    for (u32 j = 0; j < 2; ++j) {
      const u32 pp = *(LDS u32*)(scr + 8192 + j * 256 + lane * 4);
      const u32 p0 = pp & 4095u, p1 = (pp >> 16) & 4095u;
      const u32 b0 = *(LDS unsigned char*)(scr + p0), b1 = *(LDS unsigned char*)(scr + p1);
      u32 sd = (((p0 << 3) | ((128u + (b0 & 127u) - 4u * (u32)e0) >> 5) | (p1 << 19) | (((128u + (b1 & 127u) - 4u * (u32)e0) >> 5) << 16))) + hf * 0x80008000u;
      if (j == 0 && lane == 0) sd = total | ((u32)e0 << 16);
      const u32 at = j * 64 + lane;
      if (at < 32 || total > IN_CAP0)
        __builtin_amdgcn_raw_buffer_store_b32(sd, rs, at < 32 ? hf * 128 + at * 4 : 128 + hf * 384 + at * 4, offE + IU_SIDE, AUX_SYS);
      if (total <= 126) break;
    }
  } else {
    #pragma unroll
    for (int j = 0; j < 4; ++j) ST128(x[j], rs, lane * 16 + j * 1024, par + EX_INR + uid * IU_RAW + hf * 4096, AUX_SYS);
    if (lane == 0) __builtin_amdgcn_raw_buffer_store_b32(0x80000000u, rs, hf * 128, offE + IU_SIDE, AUX_SYS);
  }
}

__device__ __forceinline__ void export_short(const u32x4 (&x)[8], rsrc_t rs, u32 par, u32 uid, LDS char* scr, int lane) {
  typedef unsigned short u16x2 __attribute__((ext_vector_type(2)));
  const u32 offE = par + EX_INE + uid * IU_ENC, offR = par + EX_INR + uid * IU_RAW;
  LDS char* var = scr + 8192;
  LDS char* tab = var + QV_ALL;
  u32 s = 0;
  #pragma unroll
  for (int j = 0; j < 8; ++j) {
    *(LDS u32x4*)(scr + j * 1024 + lane * 16) = x[j];
    s = __builtin_amdgcn_sad_u8(x[j][j & 3] & 0x7C7C7C7Cu, 0u, s);
  }
  *(LDS u32x4*)(var + lane * 16) = u32x4{0u, 0u, 0u, 0u};
  *(LDS u32x4*)(var + 1024 + lane * 16) = u32x4{0u, 0u, 0u, 0u};
  {
    u32 sel = 0x0C0C0C0Cu, at = 0;
    #pragma unroll
    for (int y = 0; y < 4; ++y) if ((lane >> y) & 1) { sel = (sel & ~(0xFFu << at)) | ((u32)y << at); at += 8; }
    if (lane < 16) *(LDS u32*)(tab + lane * 4) = sel;
  }
  s = (u32)__builtin_amdgcn_readlane((int)wave_scan_u32(s), 63);
  const int K = min(31, ((int)s >> 13) + 3);
  const u32 kw = (128u + 4u * (u32)K) * 0x01010101u, tabo = (u32)(uintptr_t)tab;
  u32 mk[4] = {0, 0, 0, 0}, zq[32], car[20], bits = 0;
  wave_sync();
  #pragma unroll
  for (int g = 0; g < 4; ++g) {
    u32 a[8];
    #pragma unroll
    for (int k = 0; k < 8; ++k) {
      const int d = g * 8 + k;
      const u32 xv = x[d >> 2][d & 3];
      const u32 w = kw - (xv & 0x7C7C7C7Cu);
      const u32 sel = (w >> 2) & 0x07070707u;
      const u32 hi8 = __builtin_bit_cast(u32, __builtin_bit_cast(u16x2, (w >> 5) & 0x01010101u) * (u16x2){255, 255});
      const u32 l0 = __builtin_amdgcn_perm(0x7D6D5D4Du, 0x0004083Du, sel), l1 = __builtin_amdgcn_perm(0xFDEDDDCDu, 0x802D1D0Du, sel);
      const u32 L = (hi8 & l1) | (~hi8 & l0);
      const u32 pf = L | ~w | (w << 1);
      mk[g] = bfi(0x80808080u >> k, pf >> k, mk[g]);
      a[k] = bfi(0x0C0C0C0Cu, L, xv);
      const u32 F = L & 0x01010101u;
      const u32 cs = __builtin_amdgcn_perm(0u, (L >> 4) & 0x07070707u, *(LDS u32*)(uintptr_t)__builtin_amdgcn_udot4(F, 0x20100804u, tabo, false));
      zq[d] = (__builtin_amdgcn_udot4(cs, 0x00400801u, 0u, false) + ((cs >> 24) << 9)) | (bits << 16);
      bits = __builtin_amdgcn_udot4(F, 0x03030303u, bits, false);
    }
    car[g * 5 + 0] = bfi(0x70707070u, a[5] << 4, a[0]);
    car[g * 5 + 1] = bfi(0x70707070u, a[6] << 4, a[1]);
    car[g * 5 + 2] = bfi(0x70707070u, a[7] << 4, a[2]);
    car[g * 5 + 3] = bfi(0x40404040u, a[7] << 3, bfi(0x20202020u, a[6] << 2, bfi(0x10101010u, a[5] << 1, a[3])));
    car[g * 5 + 4] = bfi(0x40404040u, a[7] >> 1, bfi(0x20202020u, a[6] >> 2, bfi(0x10101010u, a[5] >> 3, a[4])));
  }
  #pragma unroll
  for (int j = 0; j < 5; ++j) ST128((u32x4{car[j * 4], car[j * 4 + 1], car[j * 4 + 2], car[j * 4 + 3]}), rs, lane * 16 + j * 1024, offE, AUX_SYS);
  const u32 mine = __popc(mk[0]) + __popc(mk[1]) + __popc(mk[2]) + __popc(mk[3]);
  const u32 inc = wave_scan_u32(bits | (mine << 16));
  const u32 tot = (u32)__builtin_amdgcn_readlane((int)inc, 63), nbit = tot & 0xFFFFu, npat = tot >> 16;
  if (nbit <= Q_ESC * 3 && npat <= Q_PAT) {
    const u32 base = QV_STR * 8 + (inc & 0xFFFFu) - bits;
    #pragma unroll
    for (int d = 0; d < 32; ++d) {
      const u32 q = base + (zq[d] >> 16);
      const u64 v = (u64)(zq[d] & 0xFFFFu) << (q & 31u);
      LDS u32* at = (LDS u32*)(var + ((q >> 5) << 2));
      __hip_atomic_fetch_or(at, (u32)v, __ATOMIC_RELAXED, __HIP_MEMORY_SCOPE_WORKGROUP);
      __hip_atomic_fetch_or(at + 1, (u32)(v >> 32), __ATOMIC_RELAXED, __HIP_MEMORY_SCOPE_WORKGROUP);
    }
    u32 slot = (inc >> 16) - mine;
    #pragma unroll
    for (int q = 0; q < 4; ++q) {
      u32 m = mk[q];
      while (m) {
        const int bp = __builtin_ctz(m); m &= m - 1;
        const int d = q * 8 + 7 - (bp & 7);
        const u32 pos = (d >> 2) * 1024 + lane * 16 + (d & 3) * 4 + (bp >> 3);
        const int v = K - (int)((*(LDS unsigned char*)(scr + pos) >> 2) & 31u), t = v & 15;
        *(LDS unsigned short*)(var + (slot < Q_PAT0 ? 4 + slot * 2 : QV_END - Q_PAT0 * 2 + slot * 2)) =
            (unsigned short)((pos << 3) | (u32)(((v - (t < 11 ? t : t - 8)) >> 3) + 4));
        ++slot;
      }
    }
    if (lane == 0) *(LDS u32*)var = nbit | (npat << 15) | ((u32)K << 24);
    wave_sync();
    if (lane < (int)QV0 / 16) ST128(*(LDS u32x4*)(var + lane * 16), rs, lane * 16, offE + QV, AUX_SYS);
    if (nbit > Q_ESC0 * 3 && lane < (int)(QV_END - QV0) / 16) ST128(*(LDS u32x4*)(var + QV0 + lane * 16), rs, lane * 16, offE + QV + QV0, AUX_SYS);
    if (npat > Q_PAT0 && lane < (int)(QV_ALL - QV_END) / 16) ST128(*(LDS u32x4*)(var + QV_END + lane * 16), rs, lane * 16, offE + QV + QV_END, AUX_SYS);
  } else {
    store_raw_unit(x, rs, offR, lane);
    if (lane == 0) __builtin_amdgcn_raw_buffer_store_b32(0x80000000u, rs, 0, offE + QV, AUX_SYS);
  }
}

__device__ __forceinline__ void patch_short(LDS char* scr, u32 ent) {
  const u32 pos = ent >> 3;
  __hip_atomic_fetch_add((LDS u32*)(scr + (pos & ~3u)), ((4u - (ent & 7u)) << 5) << ((pos & 3u) * 8u), __ATOMIC_RELAXED, __HIP_MEMORY_SCOPE_WORKGROUP);
}
__device__ __forceinline__ void patch_in(LDS char* scr, u32 ent) {
  const u32 pos = ent >> 3;
  __hip_atomic_fetch_add((LDS u32*)(scr + (pos & ~3u)), (((ent & 7u) - 4u) << 5) << ((pos & 3u) * 8u), __ATOMIC_RELAXED, __HIP_MEMORY_SCOPE_WORKGROUP);
}

__device__ __forceinline__ float unfix(int i, int g) { return __builtin_amdgcn_ldexpf((float)i, -g); }

__global__ void __launch_bounds__(NTHR, 2) gemm_rs_kernel(Params p) {
  extern __shared__ __attribute__((aligned(16))) char smem_[];
  LDS char* smem = (LDS char*)smem_;
  const int tid = threadIdx.x, lane = tid & 63, wave = __builtin_amdgcn_readfirstlane(tid >> 6);
  const int b = blockIdx.x, rank = p.rank;
  const int lg = lane >> 4, li = lane & 15;
  STAMP(0);
  const rsrc_t rsL = mk_rsrc(p.ws, p.ws_bytes);
  const u32 tick = (u32)__builtin_amdgcn_raw_ptr_buffer_atomic_add_i32(1, rsL, lane == 0 ? 0 : (int)0xF0000000u, OFF_CALL + b * 128 + wave * 16, 0);
  char* const peer_ws = p.ws_all[lane & 7];
  u32 call;

  if (b < 128) {
    const rsrc_t rsW = mk_rsrc(p.w, KS * NN);
    #pragma unroll
    for (int i = 0; i < 8; ++i) {
      const int row = (wave * 8 + i) * 8 + (lane >> 3);
      const int sw = ((row >> 1) & 3) | (((row >> 4) & 1) << 2);
      __builtin_amdgcn_raw_ptr_buffer_load_lds(rsW, (LDS void*)(smem + (wave * 8 + i) * 1024), 16, row * NN + (((lane & 7) ^ sw) << 4), b * 128, 0, 0);
    }
    asm volatile("s_waitcnt vmcnt(0)\n\ts_barrier" ::: "memory");
    call = __builtin_amdgcn_readfirstlane(tick) + 1u;
    const int cgl = wave & 1, kb = wave >> 1;
    u32x4 x[8];
    #pragma unroll
    for (int f = 0; f < 4; ++f)
      #pragma unroll
      for (int h = 0; h < 2; ++h)
        #pragma unroll
        for (int t = 0; t < 2; ++t) {
          const int row = kb * 128 + 64 * h + 16 * lg + 8 * t + (li >> 1);
          const int sw = ((row >> 1) & 3) | (((row >> 4) & 1) << 2);
          const i32x2 v = __builtin_amdgcn_ds_read_tr8_b64_v2i32((LDS i32x2*)(smem + row * 128 + (((cgl * 4 + f) ^ sw) << 4) + (li & 1) * 8));
          x[f * 2 + h][t * 2] = v.x; x[f * 2 + h][t * 2 + 1] = v.y;
        }
    const int gl = b >> 5, gr = 4 + ((b >> 3) & 3), c = b & 7;
    export_short(x, rsL, OFF_PAR + (call & 1) * PAR_BYTES, 256 + (b * 2 + cgl) * 4 + kb, smem + L_SCR + wave * SCR_W, lane);
    vm_drain();
    if ((lane == gl || lane == gr) && lane != rank) {
      const int cb = (lane == gl ? gr & 3 : gl) * 8 + c;
      __hip_atomic_store((GLB u32*)(peer_ws + OFF_FIN + (rank * 576 + 256 + (cb * 2 + cgl) * 4 + kb) * 128), call, __ATOMIC_RELAXED, __HIP_MEMORY_SCOPE_SYSTEM);
    }
    if (rank == gl || rank == gr) {
      const int cb = (rank == gl ? gr & 3 : gl) * 8 + c;
      store_raw_unit(x, rsL, OFF_WA + ((rank * 4 + kb) * 64 + cb * 2 + cgl) * IU_RAW, lane);
      vm_drain();
      cnt_add(p.ws, (64 + cb) * 5 + 4, lane);
    }
  } else if (b < 192) {
    const int rg = b - 128, kb = wave & 3, hf = wave >> 2, u = rg * 4 + kb;
    const rsrc_t rsX = mk_rsrc(p.x, MM * KS);
    const u32 both = (u32)(uintptr_t)(smem + L_CTL) + 16;
    const u32 xoff = OFF_XA + ((rank * 4 + kb) * 64 + rg) * IU_RAW;
    const int ow = rg >> 3, r8 = rg & 7;
    const u32 users = ((r8 <= 3 ? 0xFu << (ow & 4) : 0u) | (r8 >= 3 ? 0xFu << (~ow & 4) : 0u) | (1u << ow)) & ~(1u << rank);
    const bool tell = lane < NR && ((users >> lane) & 1);
    if (kb < 2) {
      u32x4 x[4];
      #pragma unroll
      for (int j = 0; j < 4; ++j)
        x[j] = __builtin_amdgcn_raw_buffer_load_b128(rsX, ((j >> 1) * 16 + li) * KS + (j & 1) * 64 + lg * 16, (rg * 64 + hf * 32) * KS + kb * 128, 0);
      if (wave == 0) ctl_st(both + (lane & 3) * 4, 0u);
      asm volatile("s_waitcnt lgkmcnt(0)\n\ts_barrier" ::: "memory");
      call = __builtin_amdgcn_readfirstlane(tick) + 1u;
      const u32 parw = OFF_PAR + (call & 1) * PAR_BYTES;
      auto second = [&](int k) -> bool {
        u32 old = 0;
        if (lane == 0) old = __hip_atomic_fetch_add((LDS u32*)(uintptr_t)(both + k * 4), 1u, __ATOMIC_RELAXED, __HIP_MEMORY_SCOPE_WORKGROUP);
        return __builtin_amdgcn_readfirstlane((int)old) == 1;
      };
      STAMP(6);
      if (NRAW && kb == 0) {
        #pragma unroll
        for (int j = 0; j < 4; ++j) ST128(x[j], rsL, lane * 16 + j * 1024, parw + EX_INR + u * IU_RAW + hf * 4096, AUX_SYS);
        vm_drain();
        if (second(2) && tell) __hip_atomic_store((GLB u32*)(peer_ws + OFF_FIN + (rank * 576 + 512 + rg) * 128), call, __ATOMIC_RELAXED, __HIP_MEMORY_SCOPE_SYSTEM);
      }
      export_half(x, rsL, parw, u, hf, smem + L_SCR + wave * SCR_W, lane);
      STAMP(7);
      vm_drain();
      if (second(kb) && tell) __hip_atomic_store((GLB u32*)(peer_ws + OFF_FIN + (rank * 576 + u) * 128), call, __ATOMIC_RELAXED, __HIP_MEMORY_SCOPE_SYSTEM);
      STAMP(8);
      #pragma unroll
      for (int j = 0; j < 4; ++j) ST128(x[j], rsL, lane * 16 + j * 1024, xoff + hf * 4096, AUX_SYS);
      vm_drain();
      cnt_add(p.ws, rg * 5 + 4, lane);
    } else if (hf == 0) {
      u32x4 x[8];
      #pragma unroll
      for (int j = 0; j < 8; ++j)
        x[j] = __builtin_amdgcn_raw_buffer_load_b128(rsX, ((j >> 1) * 16 + li) * KS + (j & 1) * 64 + lg * 16, rg * 64 * KS + kb * 128, 0);
      asm volatile("s_barrier" ::: "memory");
      call = __builtin_amdgcn_readfirstlane(tick) + 1u;
      export_short(x, rsL, OFF_PAR + (call & 1) * PAR_BYTES, u, smem + L_SCR + wave * SCR_W, lane);
      vm_drain();
      if (tell) __hip_atomic_store((GLB u32*)(peer_ws + OFF_FIN + (rank * 576 + u) * 128), call, __ATOMIC_RELAXED, __HIP_MEMORY_SCOPE_SYSTEM);
      store_raw_unit(x, rsL, xoff, lane);
      vm_drain();
      cnt_add(p.ws, rg * 5 + 4, lane);
    } else {
      asm volatile("s_barrier" ::: "memory");
    }
  }
  call = __builtin_amdgcn_readfirstlane(tick) + 1u;
  const u32 par = OFF_PAR + (call & 1) * PAR_BYTES;
  const float scale = p.sx[0] * p.sw[0];
  const rsrc_t rsO = mk_rsrc(p.out, 512u * NN * 4u);
  STAMP(1);

  if (b >= NGEMM) {
    const int pb = b - NGEMM, link = pb / PULL_WGS, q = pb % PULL_WGS;
    const int src = (rank + 1 + link) & 7;
    const u64 pw = (u64)peer_ws;
    const u32 pw_lo = (u32)__builtin_amdgcn_readlane((int)(u32)pw, src), pw_hi = (u32)__builtin_amdgcn_readlane((int)(u32)(pw >> 32), src);
    const rsrc_t rsP = mk_rsrc((const void*)(((u64)pw_hi << 32) | (u64)pw_lo), p.ws_bytes);
    const u32 ctl = (u32)(uintptr_t)(smem + L_CTL);
    const u32 C_READY = ctl, C_LANDED = ctl + 4, C_ISSUED = ctl + 8, C_CLOSED = ctl + 12, C_REL = ctl + 16, C_Q = ctl + 64;
    const bool same = (src >> 2) == (rank >> 2);
    if (wave == 0) ctl_st(ctl + lane * 4, 0u);
    asm volatile("s_waitcnt lgkmcnt(0)\n\ts_barrier" ::: "memory");
    if (wave == 0) {
      int g = 0, pub = 0;
      #define LANDED(v) do { if ((v) > pub) { pub = (v); ctl_st(C_LANDED, (u32)pub); } } while (0)
      #define PIECEW(W, SZ, VOFF, OFF) do { __builtin_amdgcn_raw_ptr_buffer_load_lds(rsP, (LDS void*)(dst + (OFF)), SZ, VOFF, so + (OFF), 0, PULL_AUX); \
        asm volatile("s_waitcnt vmcnt(%0)" :: "i"((W) - 1) : "memory"); ++g; LANDED(g - ((W) - 1)); } while (0)
      #define PIECE(SZ, VOFF, OFF) PIECEW(WIN, SZ, VOFF, OFF)
      bool seen_out = false;
      for (u32 n = 0;; ++n) {
        if (ctl_ld(C_READY) <= n) {
          vm_drain(); LANDED(g);
          bool fin = false;
          while (ctl_ld(C_READY) <= n) { if (ctl_ld(C_CLOSED)) { fin = ctl_ld(C_READY) <= n; break; } __builtin_amdgcn_s_sleep(1); }
          if (fin) break;
        }
        const u32 ent = ctl_ld(C_Q + (n & 15) * 4);
        ctl_st(C_ISSUED, n + 1);
        if (ent >> 31) continue;
        const u32 need = n >= NSLOT ? n - NSLOT + 1 : 0, ra = C_REL + (n % NSLOT) * 4;
        if (ctl_ld(ra) < need) {
          vm_drain(); LANDED(g);
          while (ctl_ld(ra) < need) __builtin_amdgcn_s_sleep(1);
        }
        const int i = ent & 1023;
        if (n == 0) STAMP(6);
        USTAMP(0);
        if (i >= 512 && !seen_out) { seen_out = true; STAMP(3); }
        LDS char* dst = smem + (n % NSLOT) * SLOT;
        if (ent & 1024) {
          const u32 so = par + EX_INR + i * IU_RAW;
          #pragma unroll
          for (int j = 0; j < 8; ++j) PIECE(16, lane * 16, j * 1024);
        } else if (i < 256 && (i & 3) < 2) {
          const u32 so = par + EX_INE + i * IU_ENC;
          PIECE(4, lane * 4, IU_SIDE);
          #pragma unroll
          for (int j = 0; j < 6; ++j) PIECE(16, lane * 16, j * 1024);
        } else if (i < 512) {
          const u32 so = par + EX_INE + (i < 256 ? i : 256 + group_of(rank, (i >> 6) & 3) * 64 + (i & 63)) * IU_ENC;
          if (lane < (int)QV0 / 16) __builtin_amdgcn_raw_ptr_buffer_load_lds(rsP, (LDS void*)(dst + QV), 16, lane * 16, so + QV, 0, AUX_SYS);
          asm volatile("s_waitcnt vmcnt(%0)" :: "i"(WIN - 1) : "memory"); ++g; LANDED(g - (WIN - 1));
          #pragma unroll
          for (int j = 0; j < 5; ++j) PIECE(16, lane * 16, j * 1024);
          STAMP(2);
        } else {
          const u32 so = par + EX_OUE + (rank * 320 + i - 512) * OU_ENC;
          PIECEW(WIN_OUT, 4, lane * 4, OU_SIDE);
          #pragma unroll
          for (int j = 0; j < 12; ++j) PIECEW(WIN_OUT, 16, lane * 16, j * 1024);
        }
      }
      vm_drain(); LANDED(g);
      #undef PIECE
      #undef PIECEW
      #undef LANDED
      STAMP(7);
      return;
    }
    if (wave == 1) {
      const u32 tic0 = OFF_TIC + (call & 1) * 24 * 128;
      const int only0 = lane == 0 ? 0 : (int)0xF0000000u;
      if (q == 0 && (lane == link || (link == 0 && lane >= 8 && lane < 24)))
        __hip_atomic_store((GLB u32*)(p.ws + OFF_TIC + ((~call & 1) * 24 + lane) * 128), 0u, __ATOMIC_RELAXED, __HIP_MEMORY_SCOPE_AGENT);
      constexpr u32 NPAIR = 208 + NRAW / 2, NFIX = 192;
      u32 n = 0, endp = 0; bool ok = true;
      int fj = -1, fnext = -1;
      const u32 gtic = tic0 + (8 + group_of(src, lane & 3)) * 128;
      auto fullest = [&](u32 v) -> int {
        v = v >= 16u || (!same && (lane & 3) == (rank & 3)) ? 0xFFFFu : (v << 4) | ((((lane - q) & 3) << 2) | (lane & 3));
        v = min(v, (u32)__builtin_amdgcn_mov_dpp((int)v, 0xB1, 0xf, 0xf, true));
        v = min(v, (u32)__builtin_amdgcn_mov_dpp((int)v, 0x4E, 0xf, 0xf, true));
        v = (u32)__builtin_amdgcn_readfirstlane((int)v);
        return v == 0xFFFFu ? -2 : (int)(v & 3u);
      };
      auto draw = [&](int g) { return __builtin_amdgcn_raw_ptr_buffer_atomic_add_i32(1, rsL, only0, tic0 + (g < 0 ? link : 8 + group_of(src, g)) * 128, 0); };
      int tnext = draw(-1);
      u32 seen = 0;
      for (;;) {
        u32 t = (u32)__builtin_amdgcn_readfirstlane(tnext);
        fj = fnext;
        if (t >= (fj < 0 ? NPAIR + NFIX : 16u)) {
          if ((fnext = fullest(ld_flag(p.ws, gtic))) < 0) break;
          tnext = draw(fnext);
          continue;
        }
        while (n >= ctl_ld(C_ISSUED) + (t < NPAIR && fj < 0 ? LOOK : LOOK_OUT)) __builtin_amdgcn_s_sleep(1);
        bool last = false;
        if (fj >= 0 || t + PULL_WGS >= NPAIR + NFIX) last = (fnext = fullest(seen)) < 0;
        if (last && fj < 0) { fnext = -1; last = false; }
        if (!last) tnext = draw(fnext);
        if (t >= NPAIR || fj >= 0) seen = ld_flag(p.ws, gtic);
        if (fj >= 0) t += NPAIR + NFIX;
        int i;
        if (t < NPAIR) {
          constexpr int WA_ = NA * 8;
          const int e = t < NRAW ? t : 2 * t - NRAW + (lane & 1);
          int u = -1, m = 0, kb;
          if (e < 64) { if (e < NRAW) { u = e; kb = 0; } else if (e < NRAW + 32) { u = e - NRAW; kb = 1; } else { u = e - 32; kb = 0; } }
          else if (e < 64 + 2 * WA_) { kb = e >= 64 + WA_; m = e - 64 - kb * WA_; }
          else if (e < 128 + 4 * WA_) {
            const int r = e - 64 - 2 * WA_;
            kb = 2 + (r >= 32 + WA_); m = r - (kb - 2) * (32 + WA_) - 32;
            if (m < 0) u = m + 32;
          } else { const int r = e - 128 - 4 * WA_; kb = r & 3; m = WA_ + (r >> 2); }
          const int v = u - 12;
          if (e >= 384) i = (8 * rank + (e & 7)) * 4 + ((e - 384) >> 3);
          else if (u < 0) i = 256 + ((((m >> 1) & 3) * 8 + (m >> 3)) * 2 + (m & 1)) * 4 + kb;
          else if (u < 12) i = (8 * ((rank & 4) | ((rank + 1 + (u >> 2)) & 3)) + (u & 3)) * 4 + kb;
          else i = (8 * ((~rank & 4) | (v / 5)) + 3 + v % 5) * 4 + kb;
        } else if (fj < 0) {
          const int o = t - NPAIR, cm = same ? o / 3 : o >> 2, k = same ? o - cm * 3 : 1 + (o & 3);
          const int c = same ? cm >> 3 : cm / 6, j3 = (cm - c * 6) >> 1, j = same ? (cm >> 1) & 3 : j3 + (j3 >= (rank & 3));
          i = 512 + ((j * 8 + c) * 2 + (cm & 1)) * 5 + k;
        } else i = 512 + (fj * 16 + (int)(t - NPAIR - NFIX)) * 5 + (same ? 3 : 0);
        const bool unp = t < NRAW, two = !unp && t < NPAIR;
        const u32 fo = i >= 512 ? OFF_FOUT + (src * 320 + i - 512) * 128 : OFF_FIN + (src * 576 + (unp ? 512 + (i >> 2) : i)) * 128;
        ok = ok && wait_ge(p.ws, fo, call, call);
        const u32 k = two ? (lane & 1) : 0, per = !ok ? 0 : t >= NPAIR ? 13 : unp ? 8 : (u32)__builtin_amdgcn_readfirstlane(i < 256 && (i & 3) < 2 ? 7 : 6);
        ctl_st(C_Q + ((n + k) & 15) * 4, (u32)i | (unp ? 1024u : 0u) | ((endp + per * (k + 1)) << 11) | (ok ? 0u : 0x80000000u));
        n += two ? 2 : 1; endp += two ? 2 * per : per;
        asm volatile("s_waitcnt lgkmcnt(0)" ::: "memory");
        ctl_st(C_READY, n);
        if (last) break;
      }
      asm volatile("s_waitcnt lgkmcnt(0)" ::: "memory");
      ctl_st(C_CLOSED, 1u);
      return;
    }
    LDS char* scr = smem + L_PSCR + (wave - 2) * 8 * KiB;
    for (u32 n = wave - 2;; n += NCONS) {
      bool fin = false;
      while (ctl_ld(C_READY) <= n) { if (ctl_ld(C_CLOSED)) { fin = ctl_ld(C_READY) <= n; break; } __builtin_amdgcn_s_sleep(1); }
      if (fin) break;
      const u32 ent = ctl_ld(C_Q + (n & 15) * 4);
      const bool ok = !(ent >> 31);
      const int i = ent & 1023;
      const u32 endp = (ent >> 11) & 0x1FFFu;
      auto landed = [&](u32 upto) { while (ctl_ld(C_LANDED) < upto) __builtin_amdgcn_s_sleep(1); };
      LDS char* sl = smem + (n % NSLOT) * SLOT;
      if (i < 512) {
        if (!ok) continue;
        landed(endp);
        USTAMP(1);
        const int kb = i & 3, rg = (i >> 2) & 63; const bool isx = i < 256;
        const u32 uid = isx ? i : 256 + group_of(rank, (i >> 6) & 3) * 64 + (i & 63);
        const u32 dst = (isx ? OFF_XA : OFF_WA) + ((src * 4 + kb) * 64 + rg) * IU_RAW;
        const u32 rel = C_REL + (n % NSLOT) * 4, offE = par + EX_INE + uid * IU_ENC;
        u32x4 x[8];
        bool raw = false;
        if (ent & 1024) {
          #pragma unroll
          for (int j = 0; j < 8; ++j) x[j] = *(LDS u32x4*)(sl + j * 1024 + lane * 16);
          wave_sync();
          ctl_st(rel, n + 1);
        } else if (isx && kb < 2) {
          u32x4 c[6];
          #pragma unroll
          for (int j = 0; j < 6; ++j) c[j] = *(LDS u32x4*)(sl + j * 1024 + lane * 16);
          const u32 sd = *(LDS u32*)(sl + IU_SIDE + lane * 4);
          wave_sync();
          ctl_st(rel, n + 1);
          const u32 hd[2] = {(u32)__builtin_amdgcn_readlane((int)sd, 0), (u32)__builtin_amdgcn_readlane((int)sd, 32)};
          #pragma unroll
          for (int j = 0; j < 8; ++j) {
            const u32 e4 = ((hd[j >> 2] >> 16) & 31u) * 0x04040404u;
            const u32 p0 = c[(j * 3) >> 2][(j * 3) & 3], p1 = c[(j * 3 + 1) >> 2][(j * 3 + 1) & 3], p2 = c[(j * 3 + 2) >> 2][(j * 3 + 2) & 3];
            const u32 o3 = bfi(0x80808080u, p2 << 1, bfi(0x10101010u, p2 >> 1, bfi(0x0C0C0C0Cu, p1 >> 3, p0 >> 5)));
            const u32x4 y = {(p0 & 0x9F9F9F9Fu) + e4, (p1 & 0x9F9F9F9Fu) + e4, (p2 & 0x9F9F9F9Fu) + e4, (o3 & 0x9F9F9F9Fu) + e4};
            *(LDS u32x4*)(scr + j * 1024 + lane * 16) = y;
          }
          wave_sync();
          const u32 cnt = (lane < 32 ? hd[0] : hd[1]) & 0xFFFFu, id0 = 2u * (lane & 31) - 2u;
          if (lane & 31) {
            if (id0 < cnt) patch_in(scr, sd & 0xFFFFu);
            if (id0 + 1 < cnt) patch_in(scr, sd >> 16);
          }
          if (max(hd[0] & 0xFFFFu, hd[1] & 0xFFFFu) > IN_CAP0) {
            for (int j = 0; j < 3; ++j) {
              const u32 s2 = __builtin_amdgcn_raw_buffer_load_b32(rsP, 256 + (lane >> 5) * 384 + (j * 32 + (lane & 31)) * 4, offE + IU_SIDE, AUX_SYS);
              const u32 id = IN_CAP0 + j * 64 + 2 * (lane & 31);
              if (id < cnt) patch_in(scr, s2 & 0xFFFFu);
              if (id + 1 < cnt) patch_in(scr, s2 >> 16);
            }
          }
          wave_sync();
          #pragma unroll
          for (int j = 0; j < 8; ++j) x[j] = *(LDS u32x4*)(scr + j * 1024 + lane * 16);
          #pragma unroll
          for (int h = 0; h < 2; ++h)
            if (hd[h] >> 31) {
              #pragma unroll
              for (int j = h * 4; j < h * 4 + 4; ++j) x[j] = __builtin_amdgcn_raw_buffer_load_b128(rsP, lane * 16 + j * 1024, par + EX_INR + uid * IU_RAW, AUX_SYS);
            }
        } else {
          LDS char* var = sl + QV;
          u32x4 c[5];
          #pragma unroll
          for (int j = 0; j < 5; ++j) c[j] = *(LDS u32x4*)(sl + j * 1024 + lane * 16);
          const u32 hdr = (u32)__builtin_amdgcn_readfirstlane((int)*(LDS u32*)var);
          const u32 pe = *(LDS unsigned short*)(var + 4 + (lane & 31) * 2);
          raw = hdr >> 31;
          if (!raw) {
            const u32 nbit = hdr & 0x7FFFu, npat = (hdr >> 15) & 0x1FFu, bias = (4u * ((hdr >> 24) & 31u) - 52u) * 0x01010101u;
            if (nbit > Q_ESC0 * 3 && lane < (int)(QV_END - QV0) / 16)
              *(LDS u32x4*)(var + QV0 + lane * 16) = __builtin_amdgcn_raw_buffer_load_b128(rsP, lane * 16, offE + QV + QV0, AUX_SYS);
            u32 a[32], qo[32], bits = 0;
            #pragma unroll
            for (int g = 0; g < 4; ++g) {
              u32 k5[5];
              #pragma unroll
              for (int k = 0; k < 5; ++k) { k5[k] = c[(g * 5 + k) >> 2][(g * 5 + k) & 3]; a[g * 8 + k] = k5[k]; }
              a[g * 8 + 5] = bfi(0x80808080u, k5[4] << 3, bfi(0x08080808u, k5[3] >> 1, k5[0] >> 4));
              a[g * 8 + 6] = bfi(0x80808080u, k5[4] << 2, bfi(0x08080808u, k5[3] >> 2, k5[1] >> 4));
              a[g * 8 + 7] = bfi(0x80808080u, k5[4] << 1, bfi(0x08080808u, k5[3] >> 3, k5[2] >> 4));
            }
            u32 es[32];
            #pragma unroll
            for (int d = 0; d < 32; ++d) {
              es[d] = a[d] & (a[d] >> 1) & 0x04040404u;
              qo[d] = bits;
              bits = __builtin_amdgcn_udot4(es[d], 0x03030303u, bits, false);
            }
            const u32 base4 = (((u32)(uintptr_t)var + QV_STR) * 8 + wave_scan_u32(bits >> 2)) * 4 - bits;
            wave_sync();
            #pragma unroll
            for (int d = 0; d < 32; ++d) {
              const u32 q4 = base4 + qo[d];
              const LDS u32* at = (const LDS u32*)(uintptr_t)((q4 >> 5) & ~3u);
              qo[d] = __builtin_amdgcn_alignbit(at[1], at[0], q4 >> 2);
            }
            wave_sync();
            ctl_st(rel, n + 1);
            #pragma unroll
            for (int j = 0; j < 8; ++j) {
              u32x4 y;
              #pragma unroll
              for (int k = 0; k < 4; ++k) {
                const u32 z = qo[j * 4 + k], z2 = bfi(0x0000003Fu, z, z << 10);
                const u32 sp = bfi(0x00070007u, z2, z2 << 5) & 0x07070707u;
                const u32 sel = __umul24(es[j * 4 + k], 0x404040u) + es[j * 4 + k];
                const u32 code = __builtin_amdgcn_perm(sp, 0x03030303u, sel);
                y[k] = (a[j * 4 + k] & 0x8F8F8F8Fu) + __builtin_amdgcn_perm(0x0C101418u, 0x28000408u, code) + bias;
              }
              *(LDS u32x4*)(scr + j * 1024 + lane * 16) = y;
            }
            wave_sync();
            if ((u32)lane < npat && lane < (int)Q_PAT0) patch_short(scr, pe);
            if (npat > Q_PAT0) {
              for (int j = 0; j < 2; ++j) {
                const u32 s2 = __builtin_amdgcn_raw_buffer_load_b32(rsP, lane * 4 + j * 256, offE + QV + QV_END, AUX_SYS);
                const u32 id = Q_PAT0 + j * 128 + 2 * lane;
                if (id < npat) patch_short(scr, s2 & 0xFFFFu);
                if (id + 1 < npat) patch_short(scr, s2 >> 16);
              }
            }
            wave_sync();
            #pragma unroll
            for (int j = 0; j < 8; ++j) x[j] = *(LDS u32x4*)(scr + j * 1024 + lane * 16);
          } else {
            wave_sync();
            ctl_st(rel, n + 1);
          }
        }
        if (raw) {
          #pragma unroll
          for (int j = 0; j < 8; ++j) x[j] = __builtin_amdgcn_raw_buffer_load_b128(rsP, lane * 16 + j * 1024, par + EX_INR + uid * IU_RAW, AUX_SYS);
        }
        store_raw_unit(x, rsL, dst, lane);
        USTAMP(2);
        vm_drain();
        cnt_add(p.ws, (isx ? rg : 64 + (rg >> 1)) * 5 + kb, lane);
        USTAMP(3);
        continue;
      }
      const int t = i - 512, cg = t / 5, rg = t - cg * 5 + (same ? 0 : 3);
      const u32 obase = (rg * 64 * NN + group_of(src, cg >> 4) * 1024 + (cg & 15) * 64) * 4;
      const u32 olane = (li * NN + lg * 4) * 4;
      if (!ok) {
        const u32x4 nan4 = {0x7FC00000u, 0x7FC00000u, 0x7FC00000u, 0x7FC00000u};
        #pragma unroll
        for (int e = 0; e < 16; ++e) ST128(nan4, rsO, olane + (e & 3) * 64, obase + (e >> 2) * 16 * NN * 4, OUT_AUX);
        continue;
      }
      landed(endp - 9);
      const u32 odd = *(LDS unsigned short*)(sl + OU_SIDE + lane * 2) | ((u32)*(LDS unsigned char*)(sl + OU_SIDE + 128 + lane) << 16);
      const u32 hdr = (u32)__builtin_amdgcn_readfirstlane((int)*(LDS u32*)(sl + OU_SIDE + 192));
      u32x2 sd = {*(LDS unsigned short*)(sl + OU_SIDE + 196 + li * 2), *(LDS u32*)(sl + OU_SIDE + 216 + (li < 10 ? li : 0) * 4)};
      const u32 uid = rank * 320 + t;
      if (!(hdr >> 31)) {
        const u32 cnt = hdr & 0xFFFFu; const int g = (int)((hdr >> 16) & 0x1FFu) - 256;
        if (cnt > OUT_CAP0) {
          const u32x2 s2 = __builtin_amdgcn_raw_buffer_load_b64(rsP, lane * 8, par + EX_OUE + uid * OU_ENC + OU_SIDE + 256 - 80, AUX_SYS);
          if (lane >= (int)OUT_CAP0) sd = s2;
        }
        const u32 exg = (u32)lane < cnt ? (sd.x >> 8) & 15u : 16u;
        u64 om = (1ull << (odd & 63)) | (1ull << ((odd >> 6) & 63)) | (1ull << ((odd >> 12) & 63)) | (1ull << ((odd >> 18) & 63));
        if ((odd & 63) > ((odd >> 6) & 63)) om = 0;
        const u32 ob[2] = {(u32)om, (u32)(om >> 32)};
        const float half = unfix(1, g);
        #pragma unroll
        for (int mi = 0; mi < 4; ++mi) {
          if (mi) landed(endp - 9 + 3 * mi);
          u32x4 c[3];
          #pragma unroll
          for (int j = 0; j < 3; ++j) c[j] = *(LDS u32x4*)(sl + (mi * 3 + j) * 1024 + lane * 16);
          if (mi == 3) { wave_sync(); ctl_st(C_REL + (n % NSLOT) * 4, n + 1); }
          #pragma unroll
          for (int ni = 0; ni < 4; ++ni) {
            const int k = ni * 3;
            const u32 d0 = c[k >> 2][k & 3], d1 = c[(k + 1) >> 2][(k + 1) & 3], d2 = c[(k + 2) >> 2][(k + 2) & 3];
            const u32 hi[4] = {d0 << 8, __builtin_amdgcn_perm(d1, d0, 0x0504030Cu), __builtin_amdgcn_perm(d2, d1, 0x0403020Cu), d2 & 0xFFFFFF00u};
            float y[4];
            #pragma unroll
            for (int r = 0; r < 4; ++r)
              y[r] = unfix(((int)hi[r] >> 7) | (int)__builtin_amdgcn_ubfe(ob[mi >> 1], r * 8 + 7 - ((mi * 4 + ni) & 7), 1), g);
            u64 em = __ballot(exg == (u32)(mi * 4 + ni));
            while (em) {
              const int src_l = __builtin_ctzll(em); em &= em - 1;
              const u32 pos = (u32)__builtin_amdgcn_readlane((int)sd.x, src_l), rel = (pos & 255u) - (u32)lane * 4u;
              const float vv = as_f((u32)__builtin_amdgcn_readlane((int)sd.y, src_l));
              #pragma unroll
              for (int r = 0; r < 4; ++r) y[r] = rel == (u32)r ? (pos >> 15 ? y[r] + half : vv) : y[r];
            }
            ST128((u32x4{as_u(y[0] * scale), as_u(y[1] * scale), as_u(y[2] * scale), as_u(y[3] * scale)}), rsO, olane + ni * 64, obase + mi * 16 * NN * 4, OUT_AUX);
          }
        }
      } else {
        landed(endp);
        ctl_st(C_REL + (n % NSLOT) * 4, n + 1);
        #pragma unroll
        for (int mi = 0; mi < 4; ++mi) {
          u32x4 r4[4];
          #pragma unroll
          for (int ni = 0; ni < 4; ++ni) r4[ni] = __builtin_amdgcn_raw_buffer_load_b128(rsP, lane * 16 + ni * 1024, par + EX_OUR + uid * OU_RAW + mi * 4096, AUX_SYS);
          #pragma unroll
          for (int ni = 0; ni < 4; ++ni) {
            const f32x4 y = __builtin_bit_cast(f32x4, r4[ni]) * scale;
            ST128(__builtin_bit_cast(u32x4, y), rsO, olane + ni * 64, obase + mi * 16 * NN * 4, OUT_AUX);
          }
        }
      }
    }
    STAMP(4);
    return;
  }

  constexpr int SLOTS = NGEMM / 8;
  const int xc = b & 7, slot = b >> 3, wm = wave >> 1, wn = wave & 1;
  for (int ti = slot; ti < 36; ti += SLOTS) {
    const int j = xc >> 1, ft = ti * 2 + (xc & 1);
    const int c = ti < 24 ? ft / 6 : 4 * (xc & 1) + (ti < 28 ? ti - 24 : (ti - 28) >> 1), cb = j * 8 + c;
    auto row_group = [&](int k) -> int {
      if (ti >= 28) return 8 * rank + 4 * (ti & 1) + k;
      if (ti >= 24) return 8 * ((~rank & 4) | ((j + 1 + min(k, 2)) & 3)) + 3;
      const int t = ti % 3;
      return 8 * (xc & 1 ? (~rank & 4) | ((j + 1 + t) & 3) : (rank & 4) | ((rank + 1 + t) & 3)) + 4 * (xc & 1) + k;
    };
    const int rgw = row_group(wm);
    const u32 deadw = (u32)(uintptr_t)(smem + L_CTL);
    auto ready = [&](int grp) {
      const bool isx = lane < 4;
      const int rgl = row_group(lane & 3);
      const u32 off = OFF_CNT + ((isx ? rgl : 64 + cb) * 5 + grp) * 128, want = (grp == 4 ? (isx ? 6u : 8u) : (isx ? 7u : 14u)) * call;
      if (!wait_ge(p.ws, off, want, call)) ctl_st(deadw, 1u);
    };
    const u32 xo0 = OFF_XA + rgw * IU_RAW + wn * 4096, wo0 = OFF_WA + cb * 2 * IU_RAW + wave * 2048;
    auto dma = [&](int k, int ks, u32 st) {
      const u32 xo = xo0 + ks * (64 * IU_RAW), wo = wo0 + ks * (64 * IU_RAW);
      LDS char* dx = smem + st + wave * 4096;
      LDS char* dw = smem + st + 32 * KiB + wave * 2048;
      if (k == 0) __builtin_amdgcn_raw_ptr_buffer_load_lds(rsL, (LDS void*)dx, 16, lane * 16, xo, 0, GEMM_AUX);
      if (k == 1) __builtin_amdgcn_raw_ptr_buffer_load_lds(rsL, (LDS void*)dx, 16, lane * 16, xo, 1024, GEMM_AUX);
      if (k == 2) __builtin_amdgcn_raw_ptr_buffer_load_lds(rsL, (LDS void*)dx, 16, lane * 16, xo, 2048, GEMM_AUX);
      if (k == 3) __builtin_amdgcn_raw_ptr_buffer_load_lds(rsL, (LDS void*)dx, 16, lane * 16, xo, 3072, GEMM_AUX);
      if (k == 4) __builtin_amdgcn_raw_ptr_buffer_load_lds(rsL, (LDS void*)dw, 16, lane * 16, wo, 0, GEMM_AUX);
      if (k == 5) __builtin_amdgcn_raw_ptr_buffer_load_lds(rsL, (LDS void*)dw, 16, lane * 16, wo, 1024, GEMM_AUX);
    };
    f32x4 acc[4][4];
    #pragma unroll
    for (int mi = 0; mi < 4; ++mi)
      #pragma unroll
      for (int ni = 0; ni < 4; ++ni) acc[mi][ni] = f32x4{0.f, 0.f, 0.f, 0.f};
    const bool twin = ti >= 24 && ti < 28;
    const int kr = twin ? (~rank & 4) | j : rank;
    if (wave == 0) {
      if (ti == slot) ctl_st(deadw, 0u);
      for (int grp = twin ? 0 : 4; grp < 5; ++grp) ready(grp);
    }
    asm volatile("s_waitcnt vmcnt(0) lgkmcnt(0)\n\ts_barrier" ::: "memory");
    if (ti == slot) { STAMP(2); STAMPC(14); }
    #pragma unroll
    for (int k = 0; k < 12; ++k) dma(k % 6, kr * 4 + k / 6, k / 6 * STAGE);
    u32 st = 0;
    int ksn = kr * 4 + 2, jn = -2, kbn = 0;
    i32x8 fa[8], fb[8];
    auto step = [&](i32x8 (&cur)[8], const i32x8 (&prev)[8], bool first, bool more, bool last) {
      if (more && wave == 0 && jn == 0) { ready(kbn); if (ti == slot) STAMP(10 + kbn); }
      if (!last) asm volatile("s_waitcnt vmcnt(6) lgkmcnt(0)\n\ts_barrier" ::: "memory");
      else asm volatile("s_waitcnt vmcnt(0) lgkmcnt(0)\n\ts_barrier" ::: "memory");
      const u32 st2 = st == 0 ? 2 * STAGE : st - STAGE;
      LDS char* ax = smem + st + wm * 8192 + lane * 16;
      LDS char* aw = smem + st + 32 * KiB + wn * 8192 + lane * 16;
      u32x4 h[16];
      auto M = [&](int k) {
        if (!first) acc[k >> 2][k & 3] = __builtin_amdgcn_mfma_scale_f32_16x16x128_f8f6f4(prev[k & 3], prev[4 + (k >> 2)], acc[k >> 2][k & 3], 1, 1, 0, 0, 0, 0);
        __builtin_amdgcn_sched_barrier(0);
      };
      auto R = [&](int k) { h[k] = *(LDS u32x4*)(((k >> 1) & 1 ? ax : aw) + (k >> 2) * 2048 + (k & 1) * 1024); __builtin_amdgcn_sched_barrier(0); };
      __builtin_amdgcn_sched_barrier(0);
      #pragma unroll
      for (int k = 0; k < 6; ++k) {
        M(2 * k); R(2 * k); R(2 * k + 1); M(2 * k + 1);
        if (more) dma(k, ksn, st2);
        __builtin_amdgcn_sched_barrier(0);
      }
      M(12); R(12); R(13); M(13); R(14); R(15); M(14); M(15);
      #pragma unroll
      for (int f = 0; f < 4; ++f) {
        cur[f] = __builtin_bit_cast(i32x8, __builtin_shufflevector(h[4 * f], h[4 * f + 1], 0, 1, 2, 3, 4, 5, 6, 7));
        cur[4 + f] = __builtin_bit_cast(i32x8, __builtin_shufflevector(h[4 * f + 2], h[4 * f + 3], 0, 1, 2, 3, 4, 5, 6, 7));
      }
      if (++jn == 7) { jn = 0; ++kbn; }
      ksn = jn < 0 ? ksn + 1 : ((kr + 1 + jn) & 7) * 4 + kbn;
      st = st == 2 * STAGE ? 0 : st + STAGE;
    };
    step(fa, fb, true, true, false);
    #pragma unroll 1
    for (int s = 1; s < 29; s += 2) { step(fb, fa, false, true, false); step(fa, fb, false, true, false); }
    step(fb, fa, false, true, false);
    step(fa, fb, false, false, false);
    step(fb, fa, false, false, true);
    #pragma unroll
    for (int k = 0; k < 16; ++k)
      acc[k >> 2][k & 3] = __builtin_amdgcn_mfma_scale_f32_16x16x128_f8f6f4(fb[k & 3], fb[4 + (k >> 2)], acc[k >> 2][k & 3], 1, 1, 0, 0, 0, 0);
    if (ti == slot) { STAMP(3); STAMPC(15); }
    int lz = lane; asm volatile("" : "+v"(lz));
    const bool dead = ctl_ld(deadw) != 0u;
    const int owner = rgw >> 3;
    if (owner == rank) {
      const u32 obase = ((rgw & 7) * 64 * NN + group_of(rank, j) * 1024 + c * 128 + wn * 64) * 4, olane = ((lz & 15) * NN + (lz >> 4) * 4) * 4;
      const float sc = dead ? __builtin_nanf("") : scale;
      #pragma unroll
      for (int mi = 0; mi < 4; ++mi)
        #pragma unroll
        for (int ni = 0; ni < 4; ++ni)
          ST128(__builtin_bit_cast(u32x4, acc[mi][ni] * sc), rsO, olane + ni * 64, obase + mi * 16 * NN * 4, OWN_AUX);
    } else if (!dead && !(ti >= 24 && wm == 3)) {
      const int t = (cb * 2 + wn) * 5 + (rgw & 7) - ((owner >> 2) == (rank >> 2) ? 0 : 3);
      const u32 uid = owner * 320 + t;
      const u32 offE = par + EX_OUE + uid * OU_ENC;
      float sa = 0.f;
      #pragma unroll
      for (int mi = 0; mi < 4; ++mi)
        #pragma unroll
        for (int ni = 0; ni < 4; ++ni) sa += __builtin_fabsf(acc[mi][ni][(mi + ni) & 3]);
      const int g = max(0, 24 - ((int)((as_u(wave_sum_f32(sa) * (3.75f / 1024.f)) >> 23) & 255u) - 126));
      if (ti == slot && b < 72) STAMP(7);
      LDS char* side = smem + wave * 4096;
      LDS char* pend = side + 1024;
      u32 npend = 0, sg[2] = {0u, 0u};
      auto fix = [&](float v, float& tt) -> u32 {
        tt = __builtin_amdgcn_ldexpf(v, g + 7);
        return (u32)(int)__builtin_amdgcn_fmed3f(tt, -2147483648.f, 2147483520.f);
      };
      #pragma unroll
      for (int mi = 0; mi < 4; ++mi) {
        u32 pk[12];
        #pragma unroll
        for (int ni = 0; ni < 4; ++ni) {
          u32 qv[4]; u64 bm = 0;
          #pragma unroll
          for (int r = 0; r < 4; ++r) {
            float tt;
            qv[r] = fix(acc[mi][ni][r], tt);
            bm |= __ballot(as_u((float)(int)qv[r]) != as_u(tt));
          }
          const u32 low = __builtin_amdgcn_perm(qv[1], qv[0], 0x0C0C0400u) | __builtin_amdgcn_perm(qv[3], qv[2], 0x04000C0Cu);
          const int gq = mi * 4 + ni;
          sg[gq >> 3] = bfi(0x80808080u >> (gq & 7), low >> (gq & 7), sg[gq >> 3]);
          bm |= __ballot((low & 0x7F7F7F7Fu) != 0u);
          if (bm) {
            const u32 sl = npend + __builtin_amdgcn_mbcnt_hi((u32)(bm >> 32), __builtin_amdgcn_mbcnt_lo((u32)bm, 0u));
            if (((bm >> lz) & 1) && sl < 64u) { *(LDS f32x4*)(pend + sl * 16) = acc[mi][ni]; *(LDS u32*)(pend + 1024 + sl * 4) = gq * 256 + lz * 4; }
            npend += __popcll(bm);
          }
          pk[ni * 3 + 0] = __builtin_amdgcn_perm(qv[1], qv[0], 0x05030201u);
          pk[ni * 3 + 1] = __builtin_amdgcn_perm(qv[2], qv[1], 0x06050302u);
          pk[ni * 3 + 2] = __builtin_amdgcn_perm(qv[3], qv[2], 0x07060503u);
        }
        #pragma unroll
        for (int j = 0; j < 3; ++j)
          ST128((u32x4{pk[j * 4], pk[j * 4 + 1], pk[j * 4 + 2], pk[j * 4 + 3]}), rsL, lz * 16 + j * 1024, offE + mi * 3072, EPI_AUX);
      }
      if (ti == slot && b < 72) STAMP(8);
      u32 cnt = npend > 64u ? OUT_CAP + 1 : 0u;
      auto put = [&](bool on, u32 pos, u32 val) {
        const u64 bm = __ballot(on);
        const u32 sl = cnt + __builtin_amdgcn_mbcnt_hi((u32)(bm >> 32), __builtin_amdgcn_mbcnt_lo((u32)bm, 0u));
        if (on && sl < OUT_CAP0) { *(LDS unsigned short*)(side + 196 + sl * 2) = (unsigned short)pos; *(LDS u32*)(side + 216 + sl * 4) = val; }
        else if (on && sl < OUT_CAP) *(LDS u32x2*)(side + 256 - 80 + sl * 8) = u32x2{pos, val};
        cnt += __popcll(bm);
      };
      u64 om = ((u64)sg[1] << 32) | sg[0];
      u32 odd = om ? 0u : 1u, last = 0;
      #pragma unroll
      for (int k = 0; k < 4; ++k) { if (om) last = (u32)__builtin_ctzll(om); om &= om - 1; odd |= last << (6 * k); }
      for (; __any(om != 0) && cnt <= OUT_CAP; om &= om - 1) {
        const u32 e = (u32)__builtin_ctzll(om | (1ull << 63));
        put(om != 0, 0x8000u | (((e >> 5) * 8 + 7 - (e & 7)) << 8) | (lz * 4 + ((e >> 3) & 3)), 0u);
      }
      if (npend) {
        wave_sync();
        const f32x4 pv = *(LDS f32x4*)(pend + lz * 16);
        const u32 pp = *(LDS u32*)(pend + 1024 + lz * 4);
        for (int r = 0; r < 4; ++r) {
          const float v = r == 0 ? pv[0] : r == 1 ? pv[1] : r == 2 ? pv[2] : pv[3];
          float tt;
          const u32 iv = fix(v, tt);
          put((u32)lz < npend && (as_u((float)(int)iv) != as_u(tt) || (iv & 127u)), pp + r, as_u(v));
        }
      }
      const bool raw = cnt > OUT_CAP;
      *(LDS unsigned short*)(side + lz * 2) = (unsigned short)odd;
      *(LDS unsigned char*)(side + 128 + lz) = (unsigned char)(odd >> 16);
      if (lz == 0) *(LDS u32*)(side + 192) = raw ? 0x80000000u : (cnt | ((u32)(g + 256) << 16));
      wave_sync();
      __builtin_amdgcn_raw_buffer_store_b32(*(LDS u32*)(side + lz * 4), rsL, lz * 4, offE + OU_SIDE, AUX_SYS);
      if (cnt > OUT_CAP0) __builtin_amdgcn_raw_buffer_store_b64(*(LDS u32x2*)(side + 256 + lz * 8), rsL, lz * 8 + 256, offE + OU_SIDE, AUX_SYS);
      if (raw) {
        const u32 offR = par + EX_OUR + uid * OU_RAW;
        #pragma unroll
        for (int mi = 0; mi < 4; ++mi)
          #pragma unroll
          for (int ni = 0; ni < 4; ++ni)
            ST128(__builtin_bit_cast(u32x4, acc[mi][ni]), rsL, lz * 16 + ni * 1024, offR + mi * 4096, AUX_SYS);
      }
      if (ti == slot && b < 72) STAMP(9);
      vm_drain();
      if (lz == owner)
        __hip_atomic_store((GLB u32*)(peer_ws + OFF_FOUT + (rank * 320 + t) * 128), call, __ATOMIC_RELAXED, __HIP_MEMORY_SCOPE_SYSTEM);
    }
    if (ti == slot) STAMP(4); else STAMP(5);
  }
}

}

extern "C" void kernel_launch(int rank, int nranks, ncclComm_t comm, hipStream_t stream,
                              void* const* d_in, const int* in_sizes, int n_in,
                              void* d_out, int out_size,
                              const int* in_dtypes, int out_dtype,
                              void* const* d_out_all,
                              void* const* d_ws_all, size_t ws_bytes) {
  (void)comm; (void)in_sizes; (void)n_in; (void)out_size; (void)in_dtypes; (void)out_dtype; (void)d_out_all;
  if (nranks != NR || (unsigned)rank >= (unsigned)NR || ws_bytes < WS_NEED) { fprintf(stderr, "unexpected launch shape\n"); return; }
  static unsigned attr_set = 0;
  if (!(attr_set >> rank & 1u)) {
    HIP_CHECK(hipFuncSetAttribute((const void*)gemm_rs_kernel, hipFuncAttributeMaxDynamicSharedMemorySize, (int)LDS_BYTES));
    attr_set |= 1u << rank;
  }
  Params p;
  p.x = (const uint8_t*)d_in[0]; p.w = (const uint8_t*)d_in[1]; p.sx = (const float*)d_in[2]; p.sw = (const float*)d_in[3];
  p.out = (float*)d_out; p.ws_all = (char* const*)d_ws_all; p.ws = (char*)d_ws_all[rank]; p.rank = rank; p.ws_bytes = WS_NEED;
  hipLaunchKernelGGL(gemm_rs_kernel, dim3(NWG), dim3(NTHR), LDS_BYTES, stream, p);
  HIP_CHECK(hipGetLastError());
}
```

```cpp
#include <hip/hip_runtime.h>
#include <rccl/rccl.h>
#include <cstdio>
#include <cstdint>

#define HIP_CHECK(expr) do { hipError_t _e = (expr); if (_e != hipSuccess) { \
    fprintf(stderr, "HIP error %s at %s:%d\n", hipGetErrorString(_e), __FILE__, __LINE__); \
    return; } } while (0)

#define PULL_WGS 8
#define WIN 8
#define LOOK 4
#define NRAW 8
#define NA 3
#define LOOK_OUT 2
#define EPI_AUX AUX_SYS
#define NE 1
#define WIN_OUT 8
#define OWN_AUX 17
#define PULL_AUX 17
#define GEMM_AUX 17
#define OUT_AUX 17

namespace {

typedef unsigned u32;
typedef unsigned long long u64;
typedef unsigned u32x4 __attribute__((ext_vector_type(4)));
typedef unsigned u32x2 __attribute__((ext_vector_type(2)));
typedef int i32x2 __attribute__((ext_vector_type(2)));
typedef int i32x8 __attribute__((ext_vector_type(8)));
typedef float f32x4 __attribute__((ext_vector_type(4)));
typedef __amdgpu_buffer_rsrc_t rsrc_t;
#define LDS __attribute__((address_space(3)))
#define GLB __attribute__((address_space(1)))

constexpr int NR = 8, MM = 4096, NN = 16384, KS = 512;
constexpr int NWG = 256, NTHR = 512;
constexpr int NPULL = 7 * PULL_WGS, NGEMM = NWG - NPULL;
constexpr int AUX_SYS = 17;
constexpr u64 DEADLINE_TICKS = 40000000ull;

constexpr u32 KiB = 1024, MiB = 1024 * 1024;
constexpr u32 OFF_CALL = 0;
constexpr u32 OFF_TMO = 32 * KiB;
constexpr u32 OFF_TIC = 33 * KiB;
constexpr u32 OFF_CNT = 40 * KiB;
constexpr u32 OFF_FIN = 128 * KiB;
constexpr u32 OFF_FOUT = OFF_FIN + 8 * 576 * 128;
constexpr u32 OFF_DBG = 1088 * KiB;
constexpr u32 OFF_XA = 2 * MiB;
constexpr u32 OFF_WA = OFF_XA + 16 * MiB;
constexpr u32 OFF_PAR = OFF_WA + 16 * MiB;
constexpr u32 IU_ENC = 8 * KiB, IU_RAW = 8 * KiB;
constexpr u32 OU_ENC = 12672, OU_RAW = 16 * KiB;
constexpr u32 IU_SIDE = 6144, OU_SIDE = 12032;
constexpr u32 EX_INE = 0;
constexpr u32 EX_INR = EX_INE + 1280 * IU_ENC;
constexpr u32 EX_OUE = EX_INR + 1280 * IU_RAW;
constexpr u32 EX_OUR = EX_OUE + 2560 * OU_ENC;
constexpr u32 PAR_BYTES = EX_OUR + 2560 * OU_RAW;
constexpr u32 WS_NEED = OFF_PAR + 2 * PAR_BYTES;
static_assert(OFF_CNT + 96 * 5 * 128 <= OFF_FIN && OFF_FOUT + 8 * 320 * 128 <= OFF_DBG, "flags");
static_assert(WS_NEED <= 257u * MiB, "workspace");
constexpr u32 QV = 5120, QV0 = 896, QV_STR = 64, QV_END = 1792, QV_ALL = 2304;
constexpr u32 Q_ESC0 = (QV0 - QV_STR) * 8 / 3, Q_ESC = (QV_END - QV_STR) * 8 / 3, Q_PAT0 = 30, Q_PAT = 286;
constexpr u32 IN_CAP0 = 62, IN_CAP = 254;
constexpr u32 OS_CAP0 = 24, OS_CAP = 64, OF_CAP0 = 2, OF_CAP = 32;

constexpr u32 STAGE = 48 * KiB;
constexpr u32 L_SCR = 64 * KiB, SCR_W = 10752;
constexpr u32 NSLOT = 8, SLOT = 13 * KiB, NCONS = 6;
constexpr u32 L_PSCR = NSLOT * SLOT;
constexpr u32 L_CTL = L_PSCR + NCONS * 8 * KiB;
constexpr u32 LDS_BYTES = L_CTL + 256;
static_assert(3 * STAGE <= L_CTL && L_SCR + 8 * SCR_W <= LDS_BYTES && LDS_BYTES <= 160 * KiB, "lds");

__device__ __forceinline__ int group_of(int rank, int j) { return rank < 4 ? rank * 4 + j : j * 4 + (rank & 3); }

struct Params {
  const uint8_t* x; const uint8_t* w; const float* sx; const float* sw;
  float* out; char* const* ws_all; char* ws; int rank; u32 ws_bytes;
};

__device__ __forceinline__ rsrc_t mk_rsrc(const void* p, u32 bytes) {
  u64 a = (u64)p;
  u32 lo = __builtin_amdgcn_readfirstlane((u32)a), hi = __builtin_amdgcn_readfirstlane((u32)(a >> 32));
  return __builtin_amdgcn_make_buffer_rsrc((void*)(((u64)hi << 32) | lo), 0, (int)bytes, 0x00020000);
}
#define ST128(v, rs, voff, soff, aux) __builtin_amdgcn_raw_buffer_store_b128((v), (rs), (int)((u32)(voff) + (u32)(soff)), 0, (aux))
__device__ __forceinline__ float as_f(u32 v) { return __builtin_bit_cast(float, v); }
__device__ __forceinline__ u32 as_u(float v) { return __builtin_bit_cast(u32, v); }
__device__ __forceinline__ int f2i_sat(float v) {
  int r; asm("v_cvt_i32_f32 %0, %1" : "=v"(r) : "v"(v)); return r;
}
__device__ __forceinline__ void vm_drain() { asm volatile("s_waitcnt vmcnt(0)" ::: "memory"); }
__device__ __forceinline__ void wave_sync() {
  asm volatile("s_waitcnt lgkmcnt(0)" ::: "memory");
}
__device__ __forceinline__ u32 bfi(u32 m, u32 a, u32 b) { return __builtin_amdgcn_bitop3_b32(m, a, b, 0xCA); }
template <int CTRL, int ROWS> __device__ __forceinline__ u32 dpp0(u32 v) { return (u32)__builtin_amdgcn_update_dpp(0, (int)v, CTRL, ROWS, 0xf, false); }
__device__ __forceinline__ u32 wave_scan_u32(u32 v) {
  v += dpp0<0x111, 0xf>(v); v += dpp0<0x112, 0xf>(v); v += dpp0<0x114, 0xf>(v); v += dpp0<0x118, 0xf>(v);
  v += dpp0<0x142, 0xa>(v); v += dpp0<0x143, 0xc>(v);
  return v;
}
__device__ __forceinline__ float wave_sum_f32(float v) {
  v += as_f(dpp0<0x111, 0xf>(as_u(v))); v += as_f(dpp0<0x112, 0xf>(as_u(v))); v += as_f(dpp0<0x114, 0xf>(as_u(v)));
  v += as_f(dpp0<0x118, 0xf>(as_u(v))); v += as_f(dpp0<0x142, 0xa>(as_u(v))); v += as_f(dpp0<0x143, 0xc>(as_u(v)));
  return as_f((u32)__builtin_amdgcn_readlane((int)as_u(v), 63));
}
__device__ __forceinline__ u32 ctl_ld(u32 addr) {
  u32 v; asm volatile("ds_read_b32 %0, %1\n\ts_waitcnt lgkmcnt(0)" : "=&v"(v) : "v"(addr) : "memory");
  return (u32)__builtin_amdgcn_readfirstlane((int)v);
}
__device__ __forceinline__ void ctl_st(u32 addr, u32 v) { asm volatile("ds_write_b32 %0, %1" :: "v"(addr), "v"(v) : "memory"); }
__device__ __forceinline__ u32 ld_flag(const char* ws, u32 off) {
  return __hip_atomic_load((const GLB u32*)(ws + off), __ATOMIC_RELAXED, __HIP_MEMORY_SCOPE_SYSTEM);
}
__device__ __forceinline__ bool wait_ge(char* ws, u32 off, u32 target, u32 call) {
  if (__all(ld_flag(ws, off) >= target)) return true;
  const u64 t0 = __builtin_amdgcn_s_memrealtime();
  GLB u32* tmo = (GLB u32*)(ws + OFF_TMO);
  for (;;) {
    __builtin_amdgcn_s_sleep(4);
    if (__all(ld_flag(ws, off) >= target)) return true;
    if (__hip_atomic_load(tmo, __ATOMIC_RELAXED, __HIP_MEMORY_SCOPE_AGENT) == call ||
        __builtin_amdgcn_s_memrealtime() - t0 > DEADLINE_TICKS) {
      __hip_atomic_store(tmo, call, __ATOMIC_RELAXED, __HIP_MEMORY_SCOPE_AGENT);
      return false;
    }
  }
}
__device__ __forceinline__ void cnt_add(char* ws, u32 idx, int lane) {
  if (lane == 0) __hip_atomic_fetch_add((GLB u32*)(ws + OFF_CNT + idx * 128), 1u, __ATOMIC_RELAXED, __HIP_MEMORY_SCOPE_AGENT);
}
#define STAMPC(slot) do {} while (0)
#define USTAMP(k) do {} while (0)
#define TSTAMP(k) do {} while (0)
#define STAMP(slot) do {} while (0)

__device__ __forceinline__ void store_raw_unit(const u32x4 (&x)[8], rsrc_t rs, u32 off, int lane) {
  #pragma unroll
  for (int j = 0; j < 8; ++j) ST128(x[j], rs, lane * 16 + j * 1024, off, AUX_SYS);
}

__device__ __forceinline__ void export_half(const u32x4 (&x)[4], rsrc_t rs, u32 par, u32 uid, int hf, LDS char* scr, int lane) {
  const u32 offE = par + EX_INE + uid * IU_ENC;
  u32 s = 0;
  #pragma unroll
  for (int j = 0; j < 4; ++j) {
    *(LDS u32x4*)(scr + j * 1024 + lane * 16) = x[j];
    s = __builtin_amdgcn_sad_u8(x[j][j] & 0x7C7C7C7Cu, 0u, s);
  }
  s = (u32)__builtin_amdgcn_readlane((int)wave_scan_u32(s), 63);
  int e0 = ((int)s - 17121) >> 12;
  e0 = e0 < 0 ? 0 : (e0 > 24 ? 24 : e0);
  const u32 e4 = (u32)e0 * 0x04040404u;
  u32 mk[2] = {0, 0};
  u32 pk[12];
  #pragma unroll
  for (int j = 0; j < 4; ++j) {
    u32 o[4];
    #pragma unroll
    for (int c = 0; c < 4; ++c) {
      const u32 u = (x[j][c] | 0x80808080u) - e4;
      o[c] = bfi(0x1F1F1F1Fu, u, x[j][c]);
      const u32 t = (((u >> 1) ^ 0x40404040u) & 0x70707070u) + 0x70707070u;
      const int d = j * 4 + c;
      mk[d >> 3] = bfi(0x80808080u >> (d & 7), t >> (d & 7), mk[d >> 3]);
    }
    pk[j * 3 + 0] = bfi(0x60606060u, o[3] << 5, o[0]);
    pk[j * 3 + 1] = bfi(0x60606060u, o[3] << 3, o[1]);
    pk[j * 3 + 2] = bfi(0x60606060u, bfi(0x40404040u, o[3] >> 1, o[3] << 1), o[2]);
  }
  #pragma unroll
  for (int j = 0; j < 3; ++j)
    ST128((u32x4{pk[j * 4], pk[j * 4 + 1], pk[j * 4 + 2], pk[j * 4 + 3]}), rs, lane * 16 + j * 1024, offE + hf * 3072, AUX_SYS);
  const u32 mine = __popc(mk[0]) + __popc(mk[1]);
  const u32 inc = wave_scan_u32(mine);
  const u32 total = (u32)__builtin_amdgcn_readlane((int)inc, 63);
  if (total <= IN_CAP) {
    LDS unsigned short* list = (LDS unsigned short*)(scr + 8192);
    u32 slot = 2 + inc - mine;
    #pragma unroll
    for (int q = 0; q < 2; ++q) {
      u32 m = mk[q];
      while (m) {
        const int bp = __builtin_ctz(m); m &= m - 1;
        const int d = q * 8 + 7 - (bp & 7);
        list[slot++] = (unsigned short)((d >> 2) * 1024 + lane * 16 + (d & 3) * 4 + (bp >> 3));
      }
    }
    wave_sync();
    for (u32 j = 0; j < 2; ++j) {
      const u32 pp = *(LDS u32*)(scr + 8192 + j * 256 + lane * 4);
      const u32 p0 = pp & 4095u, p1 = (pp >> 16) & 4095u;
      const u32 b0 = *(LDS unsigned char*)(scr + p0), b1 = *(LDS unsigned char*)(scr + p1);
      u32 sd = (((p0 << 3) | ((128u + (b0 & 127u) - 4u * (u32)e0) >> 5) | (p1 << 19) | (((128u + (b1 & 127u) - 4u * (u32)e0) >> 5) << 16))) + hf * 0x80008000u;
      if (j == 0 && lane == 0) sd = total | ((u32)e0 << 16);
      const u32 at = j * 64 + lane;
      if (at < 32 || total > IN_CAP0)
        __builtin_amdgcn_raw_buffer_store_b32(sd, rs, at < 32 ? hf * 128 + at * 4 : 128 + hf * 384 + at * 4, offE + IU_SIDE, AUX_SYS);
      if (total <= 126) break;
    }
  } else {
    #pragma unroll
    for (int j = 0; j < 4; ++j) ST128(x[j], rs, lane * 16 + j * 1024, par + EX_INR + uid * IU_RAW + hf * 4096, AUX_SYS);
    if (lane == 0) __builtin_amdgcn_raw_buffer_store_b32(0x80000000u, rs, hf * 128, offE + IU_SIDE, AUX_SYS);
  }
}

__device__ __forceinline__ void export_short(const u32x4 (&x)[8], rsrc_t rs, u32 par, u32 uid, LDS char* scr, int lane) {
  typedef unsigned short u16x2 __attribute__((ext_vector_type(2)));
  const u32 offE = par + EX_INE + uid * IU_ENC, offR = par + EX_INR + uid * IU_RAW;
  LDS char* var = scr + 8192;
  LDS char* tab = var + QV_ALL;
  u32 s = 0;
  #pragma unroll
  for (int j = 0; j < 8; ++j) {
    *(LDS u32x4*)(scr + j * 1024 + lane * 16) = x[j];
    s = __builtin_amdgcn_sad_u8(x[j][j & 3] & 0x7C7C7C7Cu, 0u, s);
  }
  *(LDS u32x4*)(var + lane * 16) = u32x4{0u, 0u, 0u, 0u};
  *(LDS u32x4*)(var + 1024 + lane * 16) = u32x4{0u, 0u, 0u, 0u};
  {
    u32 sel = 0x0C0C0C0Cu, at = 0;
    #pragma unroll
    for (int y = 0; y < 4; ++y) if ((lane >> y) & 1) { sel = (sel & ~(0xFFu << at)) | ((u32)y << at); at += 8; }
    if (lane < 16) *(LDS u32*)(tab + lane * 4) = sel;
  }
  s = (u32)__builtin_amdgcn_readlane((int)wave_scan_u32(s), 63);
  const int K = min(31, ((int)s >> 13) + 3);
  const u32 kw = (128u + 4u * (u32)K) * 0x01010101u, tabo = (u32)(uintptr_t)tab;
  u32 mk[4] = {0, 0, 0, 0}, zq[32], car[20], bits = 0;
  wave_sync();
  #pragma unroll
  for (int g = 0; g < 4; ++g) {
    u32 a[8];
    #pragma unroll
    for (int k = 0; k < 8; ++k) {
      const int d = g * 8 + k;
      const u32 xv = x[d >> 2][d & 3];
      const u32 w = kw - (xv & 0x7C7C7C7Cu);
      const u32 sel = (w >> 2) & 0x07070707u;
      const u32 hi8 = __builtin_bit_cast(u32, __builtin_bit_cast(u16x2, (w >> 5) & 0x01010101u) * (u16x2){255, 255});
      const u32 l0 = __builtin_amdgcn_perm(0x7D6D5D4Du, 0x0004083Du, sel), l1 = __builtin_amdgcn_perm(0xFDEDDDCDu, 0x802D1D0Du, sel);
      const u32 L = (hi8 & l1) | (~hi8 & l0);
      const u32 pf = L | ~w | (w << 1);
      mk[g] = bfi(0x80808080u >> k, pf >> k, mk[g]);
      a[k] = bfi(0x0C0C0C0Cu, L, xv);
      const u32 F = L & 0x01010101u;
      const u32 cs = __builtin_amdgcn_perm(0u, (L >> 4) & 0x07070707u, *(LDS u32*)(uintptr_t)__builtin_amdgcn_udot4(F, 0x20100804u, tabo, false));
      zq[d] = (__builtin_amdgcn_udot4(cs, 0x00400801u, 0u, false) + ((cs >> 24) << 9)) | (bits << 16);
      bits = __builtin_amdgcn_udot4(F, 0x03030303u, bits, false);
    }
    car[g * 5 + 0] = bfi(0x70707070u, a[5] << 4, a[0]);
    car[g * 5 + 1] = bfi(0x70707070u, a[6] << 4, a[1]);
    car[g * 5 + 2] = bfi(0x70707070u, a[7] << 4, a[2]);
    car[g * 5 + 3] = bfi(0x40404040u, a[7] << 3, bfi(0x20202020u, a[6] << 2, bfi(0x10101010u, a[5] << 1, a[3])));
    car[g * 5 + 4] = bfi(0x40404040u, a[7] >> 1, bfi(0x20202020u, a[6] >> 2, bfi(0x10101010u, a[5] >> 3, a[4])));
  }
  #pragma unroll
  for (int j = 0; j < 5; ++j) ST128((u32x4{car[j * 4], car[j * 4 + 1], car[j * 4 + 2], car[j * 4 + 3]}), rs, lane * 16 + j * 1024, offE, AUX_SYS);
  const u32 mine = __popc(mk[0]) + __popc(mk[1]) + __popc(mk[2]) + __popc(mk[3]);
  const u32 inc = wave_scan_u32(bits | (mine << 16));
  const u32 tot = (u32)__builtin_amdgcn_readlane((int)inc, 63), nbit = tot & 0xFFFFu, npat = tot >> 16;
  if (nbit <= Q_ESC * 3 && npat <= Q_PAT) {
    const u32 base = QV_STR * 8 + (inc & 0xFFFFu) - bits;
    #pragma unroll
    for (int d = 0; d < 32; ++d) {
      const u32 q = base + (zq[d] >> 16);
      const u64 v = (u64)(zq[d] & 0xFFFFu) << (q & 31u);
      LDS u32* at = (LDS u32*)(var + ((q >> 5) << 2));
      __hip_atomic_fetch_or(at, (u32)v, __ATOMIC_RELAXED, __HIP_MEMORY_SCOPE_WORKGROUP);
      __hip_atomic_fetch_or(at + 1, (u32)(v >> 32), __ATOMIC_RELAXED, __HIP_MEMORY_SCOPE_WORKGROUP);
    }
    u32 slot = (inc >> 16) - mine;
    #pragma unroll
    for (int q = 0; q < 4; ++q) {
      u32 m = mk[q];
      while (m) {
        const int bp = __builtin_ctz(m); m &= m - 1;
        const int d = q * 8 + 7 - (bp & 7);
        const u32 pos = (d >> 2) * 1024 + lane * 16 + (d & 3) * 4 + (bp >> 3);
        const int v = K - (int)((*(LDS unsigned char*)(scr + pos) >> 2) & 31u), t = v & 15;
        *(LDS unsigned short*)(var + (slot < Q_PAT0 ? 4 + slot * 2 : QV_END - Q_PAT0 * 2 + slot * 2)) =
            (unsigned short)((pos << 3) | (u32)(((v - (t < 11 ? t : t - 8)) >> 3) + 4));
        ++slot;
      }
    }
    if (lane == 0) *(LDS u32*)var = nbit | (npat << 15) | ((u32)K << 24);
    wave_sync();
    if (lane < (int)QV0 / 16) ST128(*(LDS u32x4*)(var + lane * 16), rs, lane * 16, offE + QV, AUX_SYS);
    if (nbit > Q_ESC0 * 3 && lane < (int)(QV_END - QV0) / 16) ST128(*(LDS u32x4*)(var + QV0 + lane * 16), rs, lane * 16, offE + QV + QV0, AUX_SYS);
    if (npat > Q_PAT0 && lane < (int)(QV_ALL - QV_END) / 16) ST128(*(LDS u32x4*)(var + QV_END + lane * 16), rs, lane * 16, offE + QV + QV_END, AUX_SYS);
  } else {
    store_raw_unit(x, rs, offR, lane);
    if (lane == 0) __builtin_amdgcn_raw_buffer_store_b32(0x80000000u, rs, 0, offE + QV, AUX_SYS);
  }
}

__device__ __forceinline__ void patch_short(LDS char* scr, u32 ent) {
  const u32 pos = ent >> 3;
  __hip_atomic_fetch_add((LDS u32*)(scr + (pos & ~3u)), ((4u - (ent & 7u)) << 5) << ((pos & 3u) * 8u), __ATOMIC_RELAXED, __HIP_MEMORY_SCOPE_WORKGROUP);
}
__device__ __forceinline__ void patch_in(LDS char* scr, u32 ent) {
  const u32 pos = ent >> 3;
  __hip_atomic_fetch_add((LDS u32*)(scr + (pos & ~3u)), (((ent & 7u) - 4u) << 5) << ((pos & 3u) * 8u), __ATOMIC_RELAXED, __HIP_MEMORY_SCOPE_WORKGROUP);
}

__device__ __forceinline__ float unfix(int i, int g) { return __builtin_amdgcn_ldexpf((float)i, -g); }
__device__ __forceinline__ u32 sign_flip(u32 t) {
  return __builtin_amdgcn_bitop3_b32(__builtin_amdgcn_bitop3_b32(t, t >> 1, 0x40404040u, 0x78), t >> 2, 0x20202020u, 0x78);
}
__device__ __forceinline__ u32 top_pack(u32 t, u32& miss) {
  const u32 u = sign_flip(t), v = u >> 8, v1 = u >> 7, w = u | v;
  miss = (u & ((u << 1) | v)) | (v & v1);
  return bfi(0x20002000u, w << 7, bfi(0x00200020u, u | (v & (w >> 1)), bfi(0x00400040u, v | (v1 & ~u), t)));
}
__device__ __forceinline__ u32 top_unpack(u32 x) {
  const u32 r = __builtin_amdgcn_perm(0x24044240u, 0x22022000u, bfi(0x00030003u, x >> 5, x >> 11) & 0x00070007u);
  return sign_flip(bfi(0x60006000u, r << 12, bfi(0x00600060u, r, x)));
}

__global__ void __launch_bounds__(NTHR, 2) gemm_rs_kernel(Params p) {
  extern __shared__ __attribute__((aligned(16))) char smem_[];
  LDS char* smem = (LDS char*)smem_;
  const int tid = threadIdx.x, lane = tid & 63, wave = __builtin_amdgcn_readfirstlane(tid >> 6);
  const int b = blockIdx.x, rank = p.rank;
  const int lg = lane >> 4, li = lane & 15;
  STAMP(0);
  const rsrc_t rsL = mk_rsrc(p.ws, p.ws_bytes);
  const u32 tick = (u32)__builtin_amdgcn_raw_ptr_buffer_atomic_add_i32(1, rsL, lane == 0 ? 0 : (int)0xF0000000u, OFF_CALL + b * 128 + wave * 16, 0);
  char* const peer_ws = p.ws_all[lane & 7];
  u32 call;

  if (b < 128) {
    const rsrc_t rsW = mk_rsrc(p.w, KS * NN);
    #pragma unroll
    for (int i = 0; i < 8; ++i) {
      const int row = (wave * 8 + i) * 8 + (lane >> 3);
      const int sw = ((row >> 1) & 3) | (((row >> 4) & 1) << 2);
      __builtin_amdgcn_raw_ptr_buffer_load_lds(rsW, (LDS void*)(smem + (wave * 8 + i) * 1024), 16, row * NN + (((lane & 7) ^ sw) << 4), b * 128, 0, 0);
    }
    asm volatile("s_waitcnt vmcnt(0)\n\ts_barrier" ::: "memory");
    call = __builtin_amdgcn_readfirstlane(tick) + 1u;
    const int cgl = wave & 1, kb = wave >> 1;
    u32x4 x[8];
    #pragma unroll
    for (int f = 0; f < 4; ++f)
      #pragma unroll
      for (int h = 0; h < 2; ++h)
        #pragma unroll
        for (int t = 0; t < 2; ++t) {
          const int row = kb * 128 + 64 * h + 16 * lg + 8 * t + (li >> 1);
          const int sw = ((row >> 1) & 3) | (((row >> 4) & 1) << 2);
          const i32x2 v = __builtin_amdgcn_ds_read_tr8_b64_v2i32((LDS i32x2*)(smem + row * 128 + (((cgl * 4 + f) ^ sw) << 4) + (li & 1) * 8));
          x[f * 2 + h][t * 2] = v.x; x[f * 2 + h][t * 2 + 1] = v.y;
        }
    const int gl = b >> 5, gr = 4 + ((b >> 3) & 3), c = b & 7;
    export_short(x, rsL, OFF_PAR + (call & 1) * PAR_BYTES, 256 + (b * 2 + cgl) * 4 + kb, smem + L_SCR + wave * SCR_W, lane);
    vm_drain();
    if ((lane == gl || lane == gr) && lane != rank) {
      const int cb = (lane == gl ? gr & 3 : gl) * 8 + c;
      __hip_atomic_store((GLB u32*)(peer_ws + OFF_FIN + (rank * 576 + 256 + (cb * 2 + cgl) * 4 + kb) * 128), call, __ATOMIC_RELAXED, __HIP_MEMORY_SCOPE_SYSTEM);
    }
    if (rank == gl || rank == gr) {
      const int cb = (rank == gl ? gr & 3 : gl) * 8 + c;
      store_raw_unit(x, rsL, OFF_WA + ((rank * 4 + kb) * 64 + cb * 2 + cgl) * IU_RAW, lane);
      vm_drain();
      cnt_add(p.ws, (64 + cb) * 5 + 4, lane);
    }
  } else if (b < 192) {
    const int rg = b - 128, kb = wave & 3, hf = wave >> 2, u = rg * 4 + kb;
    const rsrc_t rsX = mk_rsrc(p.x, MM * KS);
    const u32 both = (u32)(uintptr_t)(smem + L_CTL) + 16;
    const u32 xoff = OFF_XA + ((rank * 4 + kb) * 64 + rg) * IU_RAW;
    const int ow = rg >> 3, r8 = rg & 7;
    const u32 users = ((r8 <= 3 ? 0xFu << (ow & 4) : 0u) | (r8 >= 3 ? 0xFu << (~ow & 4) : 0u) | (1u << ow)) & ~(1u << rank);
    const bool tell = lane < NR && ((users >> lane) & 1);
    if (kb < 2) {
      u32x4 x[4];
      #pragma unroll
      for (int j = 0; j < 4; ++j)
        x[j] = __builtin_amdgcn_raw_buffer_load_b128(rsX, ((j >> 1) * 16 + li) * KS + (j & 1) * 64 + lg * 16, (rg * 64 + hf * 32) * KS + kb * 128, 0);
      if (wave == 0) ctl_st(both + (lane & 3) * 4, 0u);
      asm volatile("s_waitcnt lgkmcnt(0)\n\ts_barrier" ::: "memory");
      call = __builtin_amdgcn_readfirstlane(tick) + 1u;
      const u32 parw = OFF_PAR + (call & 1) * PAR_BYTES;
      auto second = [&](int k) -> bool {
        u32 old = 0;
        if (lane == 0) old = __hip_atomic_fetch_add((LDS u32*)(uintptr_t)(both + k * 4), 1u, __ATOMIC_RELAXED, __HIP_MEMORY_SCOPE_WORKGROUP);
        return __builtin_amdgcn_readfirstlane((int)old) == 1;
      };
      STAMP(6);
      if (NRAW && kb == 0) {
        #pragma unroll
        for (int j = 0; j < 4; ++j) ST128(x[j], rsL, lane * 16 + j * 1024, parw + EX_INR + u * IU_RAW + hf * 4096, AUX_SYS);
        vm_drain();
        if (second(2) && tell) __hip_atomic_store((GLB u32*)(peer_ws + OFF_FIN + (rank * 576 + 512 + rg) * 128), call, __ATOMIC_RELAXED, __HIP_MEMORY_SCOPE_SYSTEM);
      }
      export_half(x, rsL, parw, u, hf, smem + L_SCR + wave * SCR_W, lane);
      STAMP(7);
      vm_drain();
      if (second(kb) && tell) __hip_atomic_store((GLB u32*)(peer_ws + OFF_FIN + (rank * 576 + u) * 128), call, __ATOMIC_RELAXED, __HIP_MEMORY_SCOPE_SYSTEM);
      STAMP(8);
      #pragma unroll
      for (int j = 0; j < 4; ++j) ST128(x[j], rsL, lane * 16 + j * 1024, xoff + hf * 4096, AUX_SYS);
      vm_drain();
      cnt_add(p.ws, rg * 5 + 4, lane);
    } else if (hf == 0) {
      u32x4 x[8];
      #pragma unroll
      for (int j = 0; j < 8; ++j)
        x[j] = __builtin_amdgcn_raw_buffer_load_b128(rsX, ((j >> 1) * 16 + li) * KS + (j & 1) * 64 + lg * 16, rg * 64 * KS + kb * 128, 0);
      asm volatile("s_barrier" ::: "memory");
      call = __builtin_amdgcn_readfirstlane(tick) + 1u;
      export_short(x, rsL, OFF_PAR + (call & 1) * PAR_BYTES, u, smem + L_SCR + wave * SCR_W, lane);
      vm_drain();
      if (tell) __hip_atomic_store((GLB u32*)(peer_ws + OFF_FIN + (rank * 576 + u) * 128), call, __ATOMIC_RELAXED, __HIP_MEMORY_SCOPE_SYSTEM);
      store_raw_unit(x, rsL, xoff, lane);
      vm_drain();
      cnt_add(p.ws, rg * 5 + 4, lane);
    } else {
      asm volatile("s_barrier" ::: "memory");
    }
  }
  call = __builtin_amdgcn_readfirstlane(tick) + 1u;
  const u32 par = OFF_PAR + (call & 1) * PAR_BYTES;
  const float scale = p.sx[0] * p.sw[0];
  const rsrc_t rsO = mk_rsrc(p.out, 512u * NN * 4u);
  STAMP(1);

  if (b >= NGEMM) {
    const int pb = b - NGEMM, link = pb / PULL_WGS, q = pb % PULL_WGS;
    const int src = (rank + 1 + link) & 7;
    const u64 pw = (u64)peer_ws;
    const u32 pw_lo = (u32)__builtin_amdgcn_readlane((int)(u32)pw, src), pw_hi = (u32)__builtin_amdgcn_readlane((int)(u32)(pw >> 32), src);
    const rsrc_t rsP = mk_rsrc((const void*)(((u64)pw_hi << 32) | (u64)pw_lo), p.ws_bytes);
    const u32 ctl = (u32)(uintptr_t)(smem + L_CTL);
    const u32 C_READY = ctl, C_LANDED = ctl + 4, C_CLOSED = ctl + 12, C_REL = ctl + 16, C_ISSUED = ctl + 8, C_Q = ctl + 64;
    const bool same = (src >> 2) == (rank >> 2);
    if (wave == 0) ctl_st(ctl + lane * 4, 0u);
    asm volatile("s_waitcnt lgkmcnt(0)\n\ts_barrier" ::: "memory");
    if (wave == 0) {
      int g = 0, pub = 0;
      #define LANDED(v) do { if ((v) > pub) { pub = (v); ctl_st(C_LANDED, (u32)pub); } } while (0)
      #define PIECEW(W, SZ, VOFF, OFF) do { __builtin_amdgcn_raw_ptr_buffer_load_lds(rsP, (LDS void*)(dst + (OFF)), SZ, VOFF, so + (OFF), 0, PULL_AUX); \
        asm volatile("s_waitcnt vmcnt(%0)" :: "i"((W) - 1) : "memory"); ++g; LANDED(g - ((W) - 1)); } while (0)
      #define PIECE(SZ, VOFF, OFF) PIECEW(WIN, SZ, VOFF, OFF)
      bool seen_out = false;
      for (u32 n = 0;; ++n) {
        if (ctl_ld(C_READY) <= n) {
          vm_drain(); LANDED(g);
          bool fin = false;
          while (ctl_ld(C_READY) <= n) { if (ctl_ld(C_CLOSED)) { fin = ctl_ld(C_READY) <= n; break; } __builtin_amdgcn_s_sleep(1); }
          if (fin) break;
        }
        const u32 ent = ctl_ld(C_Q + (n & 15) * 4);
        ctl_st(C_ISSUED, n + 1);
        if (ent >> 31) continue;
        const u32 need = n >= NSLOT ? n - NSLOT + 1 : 0, ra = C_REL + (n % NSLOT) * 4;
        if (ctl_ld(ra) < need) {
          vm_drain(); LANDED(g);
          while (ctl_ld(ra) < need) __builtin_amdgcn_s_sleep(1);
        }
        const int i = ent & 1023;
        if (n == 0) STAMP(6);
        USTAMP(0);
        if (i >= 512 && !seen_out) { seen_out = true; STAMP(3); }
        LDS char* dst = smem + (n % NSLOT) * SLOT;
        if (ent & 1024) {
          const u32 so = par + EX_INR + i * IU_RAW;
          #pragma unroll
          for (int j = 0; j < 8; ++j) PIECE(16, lane * 16, j * 1024);
        } else if (i < 256 && (i & 3) < 2) {
          const u32 so = par + EX_INE + i * IU_ENC;
          PIECE(4, lane * 4, IU_SIDE);
          #pragma unroll
          for (int j = 0; j < 6; ++j) PIECE(16, lane * 16, j * 1024);
        } else if (i < 512) {
          const u32 so = par + EX_INE + (i < 256 ? i : 256 + group_of(rank, (i >> 6) & 3) * 64 + (i & 63)) * IU_ENC;
          if (lane < (int)QV0 / 16) __builtin_amdgcn_raw_ptr_buffer_load_lds(rsP, (LDS void*)(dst + QV), 16, lane * 16, so + QV, 0, AUX_SYS);
          asm volatile("s_waitcnt vmcnt(%0)" :: "i"(WIN - 1) : "memory"); ++g; LANDED(g - (WIN - 1));
          #pragma unroll
          for (int j = 0; j < 5; ++j) PIECE(16, lane * 16, j * 1024);
          STAMP(2);
        } else {
          const u32 so = par + EX_OUE + (rank * 320 + i - 512) * OU_ENC;
          #pragma unroll
          for (int j = 0; j < 12; ++j) PIECEW(WIN_OUT, 16, lane * 16, (j + 11) % 12 * 1024);
        }
      }
      vm_drain(); LANDED(g);
      #undef PIECE
      #undef PIECEW
      #undef LANDED
      STAMP(7);
      return;
    }
    if (wave == 1) {
      const u32 tic0 = OFF_TIC + (call & 1) * 24 * 128;
      const int only0 = lane == 0 ? 0 : (int)0xF0000000u;
      if (q == 0 && (lane == link || (link == 0 && lane >= 8 && lane < 24)))
        __hip_atomic_store((GLB u32*)(p.ws + OFF_TIC + ((~call & 1) * 24 + lane) * 128), 0u, __ATOMIC_RELAXED, __HIP_MEMORY_SCOPE_AGENT);
      constexpr u32 NPAIR = 208 + NRAW / 2, NFIX = 192;
      u32 n = 0, endp = 0; bool ok = true;
      int fj = -1, fnext = -1;
      const u32 gtic = tic0 + (8 + group_of(src, lane & 3)) * 128;
      auto fullest = [&](u32 v) -> int {
        v = v >= 16u || (!same && (lane & 3) == (rank & 3)) ? 0xFFFFu : (v << 4) | ((((lane - q) & 3) << 2) | (lane & 3));
        v = min(v, (u32)__builtin_amdgcn_mov_dpp((int)v, 0xB1, 0xf, 0xf, true));
        v = min(v, (u32)__builtin_amdgcn_mov_dpp((int)v, 0x4E, 0xf, 0xf, true));
        v = (u32)__builtin_amdgcn_readfirstlane((int)v);
        return v == 0xFFFFu ? -2 : (int)(v & 3u);
      };
      auto draw = [&](int g) { return __builtin_amdgcn_raw_ptr_buffer_atomic_add_i32(1, rsL, only0, tic0 + (g < 0 ? link : 8 + group_of(src, g)) * 128, 0); };
      int tnext = draw(-1);
      u32 seen = 0;
      for (;;) {
        u32 t = (u32)__builtin_amdgcn_readfirstlane(tnext);
        fj = fnext;
        if (t >= (fj < 0 ? NPAIR + NFIX : 16u)) {
          if ((fnext = fullest(ld_flag(p.ws, gtic))) < 0) break;
          tnext = draw(fnext);
          continue;
        }
        while (n >= ctl_ld(C_ISSUED) + (t < NPAIR && fj < 0 ? LOOK : LOOK_OUT)) __builtin_amdgcn_s_sleep(1);
        bool last = false;
        if (fj >= 0 || t + PULL_WGS >= NPAIR + NFIX) last = (fnext = fullest(seen)) < 0;
        if (last && fj < 0) { fnext = -1; last = false; }
        if (!last) tnext = draw(fnext);
        if (t >= NPAIR || fj >= 0) seen = ld_flag(p.ws, gtic);
        if (fj >= 0) t += NPAIR + NFIX;
        int i;
        if (t < NPAIR) {
          constexpr int WA_ = NA * 8, WE_ = NE ? 8 : 0;
          const int e = t < NRAW ? t : 2 * t - NRAW + (lane & 1);
          int u = -1, m = 0, kb;
          if (e < 64) { if (e < NRAW) { u = e; kb = 0; } else if (e < NRAW + 32) { u = e - NRAW; kb = 1; } else { u = e - 32; kb = 0; } }
          else if (e < 64 + 2 * WA_ + WE_) { kb = e >= 64 + WA_ + WE_; m = e - 64 - kb * (WA_ + WE_); }
          else if (e < 128 + 4 * WA_ + WE_) {
            const int r = e - 64 - 2 * WA_ - WE_;
            kb = 2 + (r >= 32 + WA_); m = r - (kb - 2) * (32 + WA_) - 32;
            if (m < 0) u = m + 32;
          } else if (e < 128 + 4 * WA_ + 4 * WE_) { const int r = e - 128 - 4 * WA_ - WE_; kb = 1 + (r >> 3); m = WA_ + (r & 7); }
          else { const int r = e - 128 - 4 * WA_; kb = r & 3; m = WA_ + (r >> 2); }
          const int v = u - 12;
          if (e >= 384) i = (8 * rank + (e & 7)) * 4 + ((e - 384) >> 3);
          else if (u < 0) i = 256 + ((((m >> 1) & 3) * 8 + (m >> 3)) * 2 + (m & 1)) * 4 + kb;
          else if (u < 12) i = (8 * ((rank & 4) | ((rank + 1 + (u >> 2)) & 3)) + (u & 3)) * 4 + kb;
          else i = (8 * ((~rank & 4) | (v / 5)) + 3 + v % 5) * 4 + kb;
        } else if (fj < 0) {
          const int o = t - NPAIR, cm = same ? o / 3 : o >> 2, k = same ? o - cm * 3 : 1 + (o & 3);
          const int c = same ? cm >> 3 : cm / 6, j3 = (cm - c * 6) >> 1, j = same ? (cm >> 1) & 3 : j3 + (j3 >= (rank & 3));
          i = 512 + ((j * 8 + c) * 2 + (cm & 1)) * 5 + k;
        } else i = 512 + (fj * 16 + (int)(t - NPAIR - NFIX)) * 5 + (same ? 3 : 0);
        const bool unp = t < NRAW, two = !unp && t < NPAIR;
        const u32 fo = i >= 512 ? OFF_FOUT + (src * 320 + i - 512) * 128 : OFF_FIN + (src * 576 + (unp ? 512 + (i >> 2) : i)) * 128;
        ok = ok && wait_ge(p.ws, fo, call, call);
        const u32 k = two ? (lane & 1) : 0, per = !ok ? 0 : t >= NPAIR ? 12 : unp ? 8 : (u32)__builtin_amdgcn_readfirstlane(i < 256 && (i & 3) < 2 ? 7 : 6);
        ctl_st(C_Q + ((n + k) & 15) * 4, (u32)i | (unp ? 1024u : 0u) | ((endp + per * (k + 1)) << 11) | (ok ? 0u : 0x80000000u));
        n += two ? 2 : 1; endp += two ? 2 * per : per;
        asm volatile("s_waitcnt lgkmcnt(0)" ::: "memory");
        ctl_st(C_READY, n);
        if (last) break;
      }
      asm volatile("s_waitcnt lgkmcnt(0)" ::: "memory");
      ctl_st(C_CLOSED, 1u);
      return;
    }
    LDS char* scr = smem + L_PSCR + (wave - 2) * 8 * KiB;
    for (u32 n = wave - 2;; n += NCONS) {
      bool fin = false;
      while (ctl_ld(C_READY) <= n) { if (ctl_ld(C_CLOSED)) { fin = ctl_ld(C_READY) <= n; break; } __builtin_amdgcn_s_sleep(1); }
      if (fin) break;
      const u32 ent = ctl_ld(C_Q + (n & 15) * 4);
      const bool ok = !(ent >> 31);
      const int i = ent & 1023;
      const u32 endp = (ent >> 11) & 0x1FFFu;
      auto landed = [&](u32 upto) { while (ctl_ld(C_LANDED) < upto) __builtin_amdgcn_s_sleep(1); };
      LDS char* sl = smem + (n % NSLOT) * SLOT;
      if (i < 512) {
        if (!ok) continue;
        landed(endp);
        USTAMP(1);
        const int kb = i & 3, rg = (i >> 2) & 63; const bool isx = i < 256;
        const u32 uid = isx ? i : 256 + group_of(rank, (i >> 6) & 3) * 64 + (i & 63);
        const u32 dst = (isx ? OFF_XA : OFF_WA) + ((src * 4 + kb) * 64 + rg) * IU_RAW;
        const u32 rel = C_REL + (n % NSLOT) * 4, offE = par + EX_INE + uid * IU_ENC;
        u32x4 x[8];
        bool raw = false;
        if (ent & 1024) {
          #pragma unroll
          for (int j = 0; j < 8; ++j) x[j] = *(LDS u32x4*)(sl + j * 1024 + lane * 16);
          wave_sync();
          ctl_st(rel, n + 1);
        } else if (isx && kb < 2) {
          u32x4 c[6];
          #pragma unroll
          for (int j = 0; j < 6; ++j) c[j] = *(LDS u32x4*)(sl + j * 1024 + lane * 16);
          const u32 sd = *(LDS u32*)(sl + IU_SIDE + lane * 4);
          wave_sync();
          ctl_st(rel, n + 1);
          const u32 hd[2] = {(u32)__builtin_amdgcn_readlane((int)sd, 0), (u32)__builtin_amdgcn_readlane((int)sd, 32)};
          #pragma unroll
          for (int j = 0; j < 8; ++j) {
            const u32 e4 = ((hd[j >> 2] >> 16) & 31u) * 0x04040404u;
            const u32 p0 = c[(j * 3) >> 2][(j * 3) & 3], p1 = c[(j * 3 + 1) >> 2][(j * 3 + 1) & 3], p2 = c[(j * 3 + 2) >> 2][(j * 3 + 2) & 3];
            const u32 o3 = bfi(0x80808080u, p2 << 1, bfi(0x10101010u, p2 >> 1, bfi(0x0C0C0C0Cu, p1 >> 3, p0 >> 5)));
            const u32x4 y = {(p0 & 0x9F9F9F9Fu) + e4, (p1 & 0x9F9F9F9Fu) + e4, (p2 & 0x9F9F9F9Fu) + e4, (o3 & 0x9F9F9F9Fu) + e4};
            *(LDS u32x4*)(scr + j * 1024 + lane * 16) = y;
          }
          wave_sync();
          const u32 cnt = (lane < 32 ? hd[0] : hd[1]) & 0xFFFFu, id0 = 2u * (lane & 31) - 2u;
          if (lane & 31) {
            if (id0 < cnt) patch_in(scr, sd & 0xFFFFu);
            if (id0 + 1 < cnt) patch_in(scr, sd >> 16);
          }
          if (max(hd[0] & 0xFFFFu, hd[1] & 0xFFFFu) > IN_CAP0) {
            for (int j = 0; j < 3; ++j) {
              const u32 s2 = __builtin_amdgcn_raw_buffer_load_b32(rsP, 256 + (lane >> 5) * 384 + (j * 32 + (lane & 31)) * 4, offE + IU_SIDE, AUX_SYS);
              const u32 id = IN_CAP0 + j * 64 + 2 * (lane & 31);
              if (id < cnt) patch_in(scr, s2 & 0xFFFFu);
              if (id + 1 < cnt) patch_in(scr, s2 >> 16);
            }
          }
          wave_sync();
          #pragma unroll
          for (int j = 0; j < 8; ++j) x[j] = *(LDS u32x4*)(scr + j * 1024 + lane * 16);
          #pragma unroll
          for (int h = 0; h < 2; ++h)
            if (hd[h] >> 31) {
              #pragma unroll
              for (int j = h * 4; j < h * 4 + 4; ++j) x[j] = __builtin_amdgcn_raw_buffer_load_b128(rsP, lane * 16 + j * 1024, par + EX_INR + uid * IU_RAW, AUX_SYS);
            }
        } else {
          LDS char* var = sl + QV;
          u32x4 c[5];
          #pragma unroll
          for (int j = 0; j < 5; ++j) c[j] = *(LDS u32x4*)(sl + j * 1024 + lane * 16);
          const u32 hdr = (u32)__builtin_amdgcn_readfirstlane((int)*(LDS u32*)var);
          const u32 pe = *(LDS unsigned short*)(var + 4 + (lane & 31) * 2);
          raw = hdr >> 31;
          if (!raw) {
            const u32 nbit = hdr & 0x7FFFu, npat = (hdr >> 15) & 0x1FFu, bias = (4u * ((hdr >> 24) & 31u) - 52u) * 0x01010101u;
            if (nbit > Q_ESC0 * 3 && lane < (int)(QV_END - QV0) / 16)
              *(LDS u32x4*)(var + QV0 + lane * 16) = __builtin_amdgcn_raw_buffer_load_b128(rsP, lane * 16, offE + QV + QV0, AUX_SYS);
            u32 a[32], qo[32], bits = 0;
            #pragma unroll
            for (int g = 0; g < 4; ++g) {
              u32 k5[5];
              #pragma unroll
              for (int k = 0; k < 5; ++k) { k5[k] = c[(g * 5 + k) >> 2][(g * 5 + k) & 3]; a[g * 8 + k] = k5[k]; }
              a[g * 8 + 5] = bfi(0x80808080u, k5[4] << 3, bfi(0x08080808u, k5[3] >> 1, k5[0] >> 4));
              a[g * 8 + 6] = bfi(0x80808080u, k5[4] << 2, bfi(0x08080808u, k5[3] >> 2, k5[1] >> 4));
              a[g * 8 + 7] = bfi(0x80808080u, k5[4] << 1, bfi(0x08080808u, k5[3] >> 3, k5[2] >> 4));
            }
            u32 es[32];
            #pragma unroll
            for (int d = 0; d < 32; ++d) {
              es[d] = a[d] & (a[d] >> 1) & 0x04040404u;
              qo[d] = bits;
              bits = __builtin_amdgcn_udot4(es[d], 0x03030303u, bits, false);
            }
            const u32 base4 = (((u32)(uintptr_t)var + QV_STR) * 8 + wave_scan_u32(bits >> 2)) * 4 - bits;
            wave_sync();
            #pragma unroll
            for (int d = 0; d < 32; ++d) {
              const u32 q4 = base4 + qo[d];
              const LDS u32* at = (const LDS u32*)(uintptr_t)((q4 >> 5) & ~3u);
              qo[d] = __builtin_amdgcn_alignbit(at[1], at[0], q4 >> 2);
            }
            wave_sync();
            ctl_st(rel, n + 1);
            #pragma unroll
            for (int j = 0; j < 8; ++j) {
              u32x4 y;
              #pragma unroll
              for (int k = 0; k < 4; ++k) {
                const u32 z = qo[j * 4 + k], z2 = bfi(0x0000003Fu, z, z << 10);
                const u32 sp = bfi(0x00070007u, z2, z2 << 5) & 0x07070707u;
                const u32 sel = __umul24(es[j * 4 + k], 0x404040u) + es[j * 4 + k];
                const u32 code = __builtin_amdgcn_perm(sp, 0x03030303u, sel);
                y[k] = (a[j * 4 + k] & 0x8F8F8F8Fu) + __builtin_amdgcn_perm(0x0C101418u, 0x28000408u, code) + bias;
              }
              *(LDS u32x4*)(scr + j * 1024 + lane * 16) = y;
            }
            wave_sync();
            if ((u32)lane < npat && lane < (int)Q_PAT0) patch_short(scr, pe);
            if (npat > Q_PAT0) {
              for (int j = 0; j < 2; ++j) {
                const u32 s2 = __builtin_amdgcn_raw_buffer_load_b32(rsP, lane * 4 + j * 256, offE + QV + QV_END, AUX_SYS);
                const u32 id = Q_PAT0 + j * 128 + 2 * lane;
                if (id < npat) patch_short(scr, s2 & 0xFFFFu);
                if (id + 1 < npat) patch_short(scr, s2 >> 16);
              }
            }
            wave_sync();
            #pragma unroll
            for (int j = 0; j < 8; ++j) x[j] = *(LDS u32x4*)(scr + j * 1024 + lane * 16);
          } else {
            wave_sync();
            ctl_st(rel, n + 1);
          }
        }
        if (raw) {
          #pragma unroll
          for (int j = 0; j < 8; ++j) x[j] = __builtin_amdgcn_raw_buffer_load_b128(rsP, lane * 16 + j * 1024, par + EX_INR + uid * IU_RAW, AUX_SYS);
        }
        store_raw_unit(x, rsL, dst, lane);
        USTAMP(2);
        vm_drain();
        cnt_add(p.ws, (isx ? rg : 64 + (rg >> 1)) * 5 + kb, lane);
        USTAMP(3);
        continue;
      }
      const int t = i - 512, cg = t / 5, rg = t - cg * 5 + (same ? 0 : 3);
      const u32 obase = (rg * 64 * NN + group_of(src, cg >> 4) * 1024 + (cg & 15) * 64) * 4;
      const u32 olane = (li * NN + lg * 4) * 4;
      if (!ok) {
        const u32x4 nan4 = {0x7FC00000u, 0x7FC00000u, 0x7FC00000u, 0x7FC00000u};
        #pragma unroll
        for (int e = 0; e < 16; ++e) ST128(nan4, rsO, olane + (e & 3) * 64, obase + (e >> 2) * 16 * NN * 4, OUT_AUX);
        continue;
      }
      #pragma unroll
      for (int j = 0; j < 4; ++j) *(LDS u32x4*)(scr + j * 1024 + lane * 16) = u32x4{0u, 0u, 0u, 0u};
      TSTAMP(0);
      landed(endp - 11);
      TSTAMP(1);
      const u32 odd = *(LDS unsigned short*)(sl + OU_SIDE + lane * 2) | ((u32)*(LDS unsigned char*)(sl + OU_SIDE + 128 + lane) << 16);
      const u32 hdr = (u32)__builtin_amdgcn_readfirstlane((int)*(LDS u32*)(sl + OU_SIDE + 192));
      u32 se = *(LDS unsigned short*)(sl + OU_SIDE + 196 + (lane < 24 ? lane : 0) * 2);
      u32x2 fe = {*(LDS unsigned short*)(sl + OU_SIDE + 244 + (lane & 1) * 2), *(LDS u32*)(sl + OU_SIDE + 248 + (lane & 1) * 4)};
      const u32 tl[3] = {*(LDS u32*)(sl + 11264 + lane * 4), *(LDS u32*)(sl + 11520 + lane * 4), *(LDS u32*)(sl + 11776 + lane * 4)};
      const u32 uid = rank * 320 + t;
      if (!(hdr >> 31)) {
        const u32 ns = hdr & 0xFFu, nf = (hdr >> 8) & 0xFFu; const int g = (int)((hdr >> 16) & 0x1FFu) - 256;
        const u32 tail = par + EX_OUE + uid * OU_ENC + 12288;
        if (ns > OS_CAP0) {
          const u32 s2 = __builtin_amdgcn_raw_buffer_load_b16(rsP, lane >= (int)OS_CAP0 ? lane * 2 : OS_CAP0 * 2, tail - OS_CAP0 * 2, AUX_SYS);
          se = lane >= (int)OS_CAP0 ? s2 : se;
          asm volatile("" : "+v"(se));
        }
        if (nf > OF_CAP0) {
          const u32x2 f2 = __builtin_amdgcn_raw_buffer_load_b64(rsP, max((u32)OF_CAP0, min((u32)lane, nf - 1)) * 8, tail + 128 - OF_CAP0 * 8, AUX_SYS);
          fe = lane >= (int)OF_CAP0 ? f2 : fe;
          asm volatile("" : "+v"(fe));
        }
        if ((u32)lane < ns)
          __hip_atomic_fetch_add((LDS u32*)(scr + ((se >> 13) & 3u) * 1024 + ((se >> 5) & 63u) * 16 + ((se >> 11) & 3u) * 4),
                                 se >> 15 ? ((se & 3u) | ((se & 12u) << 6)) << (se & 16u) : ((se & 7u) << 5) << (se & 24u),
                                 __ATOMIC_RELAXED, __HIP_MEMORY_SCOPE_WORKGROUP);
        const u32 fgq = (u32)lane < nf ? fe.x >> 8 : 16u;
        u64 om = (1ull << (odd & 63)) | (1ull << ((odd >> 6) & 63)) | (1ull << ((odd >> 12) & 63)) | (1ull << ((odd >> 18) & 63));
        if ((odd & 63) > ((odd >> 6) & 63)) om = 0;
        const u32 ob[2] = {(u32)om, (u32)(om >> 32)};
        u32 t15 = tl[0] << 1;
        wave_sync();
        TSTAMP(2);
        #pragma unroll
        for (int mi = 0; mi < 4; ++mi) {
          landed(mi < 3 ? endp - 8 + 3 * mi : endp);
          if (mi == 3) TSTAMP(3);
          u32x4 c[3];
          #pragma unroll
          for (int j = 0; j < (mi < 3 ? 3 : 2); ++j) c[j] = *(LDS u32x4*)(sl + (mi * 3 + j) * 1024 + lane * 16);
          const u32x4 adj = *(LDS u32x4*)(scr + mi * 1024 + lane * 16);
          if (mi == 3) { wave_sync(); ctl_st(C_REL + (n % NSLOT) * 4, n + 1); c[2] = u32x4{tl[0], tl[1], tl[2], 0u}; }
          #pragma unroll
          for (int ni = 0; ni < 4; ++ni) {
            const int k = ni * 3, gq = mi * 4 + ni;
            const u32 l0 = c[k >> 2][k & 3], l1 = c[(k + 1) >> 2][(k + 1) & 3], tp = gq < 15 ? c[(k + 2) >> 2][(k + 2) & 3] : t15;
            if (gq < 14) t15 = bfi(0x00010001u << gq, tp >> (14 - gq), t15);
            const u32 top = __builtin_amdgcn_bitop3_b32(top_unpack(tp), adj[ni] << 5, 0x60606060u, 0x78);
            const u32 low = ((ob[gq >> 3] << (gq & 7)) & 0x80808080u) + (adj[ni] & 0xE0E0E0E0u);
            const u32 q01 = __builtin_amdgcn_perm(top, low, 0x05010400u), q23 = __builtin_amdgcn_perm(top, low, 0x07030602u);
            float y[4] = {unfix((int)__builtin_amdgcn_perm(q01, l0, 0x05010004u), g + 7), unfix((int)__builtin_amdgcn_perm(q01, l0, 0x07030206u), g + 7),
                          unfix((int)__builtin_amdgcn_perm(q23, l1, 0x05010004u), g + 7), unfix((int)__builtin_amdgcn_perm(q23, l1, 0x07030206u), g + 7)};
            if (nf)
              for (u64 em = __ballot(fgq == (u32)gq); em; em &= em - 1) {
                const int from = __builtin_ctzll(em);
                const u32 rel = ((u32)__builtin_amdgcn_readlane((int)fe.x, from) & 255u) - (u32)lane * 4u;
                const float vv = as_f((u32)__builtin_amdgcn_readlane((int)fe.y, from));
                #pragma unroll
                for (int r = 0; r < 4; ++r) y[r] = rel == (u32)r ? vv : y[r];
              }
            ST128((u32x4{as_u(y[0] * scale), as_u(y[1] * scale), as_u(y[2] * scale), as_u(y[3] * scale)}), rsO, olane + ni * 64, obase + mi * 16 * NN * 4, OUT_AUX);
          }
        }
        TSTAMP(4);
      } else {
        landed(endp);
        ctl_st(C_REL + (n % NSLOT) * 4, n + 1);
        #pragma unroll
        for (int mi = 0; mi < 4; ++mi) {
          u32x4 r4[4];
          #pragma unroll
          for (int ni = 0; ni < 4; ++ni) r4[ni] = __builtin_amdgcn_raw_buffer_load_b128(rsP, lane * 16 + ni * 1024, par + EX_OUR + uid * OU_RAW + mi * 4096, AUX_SYS);
          #pragma unroll
          for (int ni = 0; ni < 4; ++ni) {
            const f32x4 y = __builtin_bit_cast(f32x4, r4[ni]) * scale;
            ST128(__builtin_bit_cast(u32x4, y), rsO, olane + ni * 64, obase + mi * 16 * NN * 4, OUT_AUX);
          }
        }
      }
    }
    STAMP(4);
    return;
  }

  constexpr int SLOTS = NGEMM / 8;
  const int xc = b & 7, slot = b >> 3, wm = wave >> 1, wn = wave & 1;
  for (int ti = slot; ti < 36; ti += SLOTS) {
    const int j = xc >> 1, ft = ti * 2 + (xc & 1);
    const int c = ti < 24 ? ft / 6 : 4 * (xc & 1) + (ti < 28 ? ti - 24 : (ti - 28) >> 1), cb = j * 8 + c;
    auto row_group = [&](int k) -> int {
      if (ti >= 28) return 8 * rank + 4 * (ti & 1) + k;
      if (ti >= 24) return 8 * ((~rank & 4) | ((j + 1 + min(k, 2)) & 3)) + 3;
      const int t = ti % 3;
      return 8 * (xc & 1 ? (~rank & 4) | ((j + 1 + t) & 3) : (rank & 4) | ((rank + 1 + t) & 3)) + 4 * (xc & 1) + k;
    };
    const int rgw = row_group(wm);
    const u32 deadw = (u32)(uintptr_t)(smem + L_CTL);
    auto ready = [&](int grp) {
      const bool isx = lane < 4;
      const int rgl = row_group(lane & 3);
      const u32 off = OFF_CNT + ((isx ? rgl : 64 + cb) * 5 + grp) * 128, want = (grp == 4 ? (isx ? 6u : 8u) : (isx ? 7u : 14u)) * call;
      if (!wait_ge(p.ws, off, want, call)) ctl_st(deadw, 1u);
    };
    const u32 xo0 = OFF_XA + rgw * IU_RAW + wn * 4096, wo0 = OFF_WA + cb * 2 * IU_RAW + wave * 2048;
    auto dma = [&](int k, int ks, u32 st) {
      const u32 xo = xo0 + ks * (64 * IU_RAW), wo = wo0 + ks * (64 * IU_RAW);
      LDS char* dx = smem + st + wave * 4096;
      LDS char* dw = smem + st + 32 * KiB + wave * 2048;
      if (k == 0) __builtin_amdgcn_raw_ptr_buffer_load_lds(rsL, (LDS void*)dx, 16, lane * 16, xo, 0, GEMM_AUX);
      if (k == 1) __builtin_amdgcn_raw_ptr_buffer_load_lds(rsL, (LDS void*)dx, 16, lane * 16, xo, 1024, GEMM_AUX);
      if (k == 2) __builtin_amdgcn_raw_ptr_buffer_load_lds(rsL, (LDS void*)dx, 16, lane * 16, xo, 2048, GEMM_AUX);
      if (k == 3) __builtin_amdgcn_raw_ptr_buffer_load_lds(rsL, (LDS void*)dx, 16, lane * 16, xo, 3072, GEMM_AUX);
      if (k == 4) __builtin_amdgcn_raw_ptr_buffer_load_lds(rsL, (LDS void*)dw, 16, lane * 16, wo, 0, GEMM_AUX);
      if (k == 5) __builtin_amdgcn_raw_ptr_buffer_load_lds(rsL, (LDS void*)dw, 16, lane * 16, wo, 1024, GEMM_AUX);
    };
    f32x4 acc[4][4];
    #pragma unroll
    for (int mi = 0; mi < 4; ++mi)
      #pragma unroll
      for (int ni = 0; ni < 4; ++ni) acc[mi][ni] = f32x4{0.f, 0.f, 0.f, 0.f};
    const bool twin = ti >= 24 && ti < 28;
    const int kr = twin ? (~rank & 4) | j : rank;
    if (wave == 0) {
      if (ti == slot) ctl_st(deadw, 0u);
      for (int grp = twin ? 0 : 4; grp < 5; ++grp) ready(grp);
    }
    asm volatile("s_waitcnt vmcnt(0) lgkmcnt(0)\n\ts_barrier" ::: "memory");
    if (ti == slot) { STAMP(2); STAMPC(14); }
    #pragma unroll
    for (int k = 0; k < 12; ++k) dma(k % 6, kr * 4 + k / 6, k / 6 * STAGE);
    u32 st = 0;
    int ksn = kr * 4 + 2, jn = -2, kbn = 0;
    i32x8 fa[8], fb[8];
    auto step = [&](i32x8 (&cur)[8], const i32x8 (&prev)[8], bool first, bool more, bool last) {
      if (more && wave == 0 && jn == 0) { ready(kbn); if (ti == slot) STAMP(10 + kbn); }
      if (!last) asm volatile("s_waitcnt vmcnt(6) lgkmcnt(0)\n\ts_barrier" ::: "memory");
      else asm volatile("s_waitcnt vmcnt(0) lgkmcnt(0)\n\ts_barrier" ::: "memory");
      const u32 st2 = st == 0 ? 2 * STAGE : st - STAGE;
      LDS char* ax = smem + st + wm * 8192 + lane * 16;
      LDS char* aw = smem + st + 32 * KiB + wn * 8192 + lane * 16;
      u32x4 h[16];
      auto M = [&](int k) {
        if (!first) acc[k >> 2][k & 3] = __builtin_amdgcn_mfma_scale_f32_16x16x128_f8f6f4(prev[k & 3], prev[4 + (k >> 2)], acc[k >> 2][k & 3], 1, 1, 0, 0, 0, 0);
        __builtin_amdgcn_sched_barrier(0);
      };
      auto R = [&](int k) { h[k] = *(LDS u32x4*)(((k >> 1) & 1 ? ax : aw) + (k >> 2) * 2048 + (k & 1) * 1024); __builtin_amdgcn_sched_barrier(0); };
      __builtin_amdgcn_sched_barrier(0);
      #pragma unroll
      for (int k = 0; k < 6; ++k) {
        M(2 * k); R(2 * k); R(2 * k + 1); M(2 * k + 1);
        if (more) dma(k, ksn, st2);
        __builtin_amdgcn_sched_barrier(0);
      }
      M(12); R(12); R(13); M(13); R(14); R(15); M(14); M(15);
      #pragma unroll
      for (int f = 0; f < 4; ++f) {
        cur[f] = __builtin_bit_cast(i32x8, __builtin_shufflevector(h[4 * f], h[4 * f + 1], 0, 1, 2, 3, 4, 5, 6, 7));
        cur[4 + f] = __builtin_bit_cast(i32x8, __builtin_shufflevector(h[4 * f + 2], h[4 * f + 3], 0, 1, 2, 3, 4, 5, 6, 7));
      }
      if (++jn == 7) { jn = 0; ++kbn; }
      ksn = jn < 0 ? ksn + 1 : ((kr + 1 + jn) & 7) * 4 + kbn;
      st = st == 2 * STAGE ? 0 : st + STAGE;
    };
    step(fa, fb, true, true, false);
    #pragma unroll 1
    for (int s = 1; s < 29; s += 2) { step(fb, fa, false, true, false); step(fa, fb, false, true, false); }
    step(fb, fa, false, true, false);
    step(fa, fb, false, false, false);
    step(fb, fa, false, false, true);
    #pragma unroll
    for (int k = 0; k < 16; ++k)
      acc[k >> 2][k & 3] = __builtin_amdgcn_mfma_scale_f32_16x16x128_f8f6f4(fb[k & 3], fb[4 + (k >> 2)], acc[k >> 2][k & 3], 1, 1, 0, 0, 0, 0);
    if (ti == slot) { STAMP(3); STAMPC(15); } else STAMP(6);
    int lz = lane; asm volatile("" : "+v"(lz));
    const bool dead = ctl_ld(deadw) != 0u;
    const int owner = rgw >> 3;
    if (owner == rank) {
      const u32 obase = ((rgw & 7) * 64 * NN + group_of(rank, j) * 1024 + c * 128 + wn * 64) * 4, olane = ((lz & 15) * NN + (lz >> 4) * 4) * 4;
      const float sc = dead ? __builtin_nanf("") : scale;
      #pragma unroll
      for (int mi = 0; mi < 4; ++mi)
        #pragma unroll
        for (int ni = 0; ni < 4; ++ni)
          ST128(__builtin_bit_cast(u32x4, acc[mi][ni] * sc), rsO, olane + ni * 64, obase + mi * 16 * NN * 4, OWN_AUX);
    } else if (!dead && !(ti >= 24 && wm == 3)) {
      const int t = (cb * 2 + wn) * 5 + (rgw & 7) - ((owner >> 2) == (rank >> 2) ? 0 : 3);
      const u32 uid = owner * 320 + t;
      const u32 offE = par + EX_OUE + uid * OU_ENC;
      float sa = 0.f;
      #pragma unroll
      for (int mi = 0; mi < 4; ++mi)
        #pragma unroll
        for (int ni = 0; ni < 4; ++ni) sa += __builtin_fabsf(acc[mi][ni][(mi + ni) & 3]);
      const int g = max(0, 24 - ((int)((as_u(wave_sum_f32(sa) * (4.2f / 1024.f)) >> 23) & 255u) - 126));
      if (ti == slot && b < 72) STAMP(7);
      LDS char* last1k = smem + wave * 4096;
      LDS char* side = last1k + 768;
      LDS char* pend = last1k + 1536;
      u32 npend = 0, sg[2] = {0u, 0u};
      auto quad = [&](int gq, u32& l0, u32& l1, u32& tp) {
        u32 qv[4], fr[4];
        #pragma unroll
        for (int r = 0; r < 4; ++r) {
          const float tt = __builtin_amdgcn_ldexpf(acc[gq >> 2][gq & 3][r], g + 7);
          qv[r] = (u32)f2i_sat(tt);
          fr[r] = as_u(__builtin_amdgcn_fractf(tt));
        }
        const u32 p01 = __builtin_amdgcn_perm(qv[1], qv[0], 0x07030400u), p23 = __builtin_amdgcn_perm(qv[3], qv[2], 0x07030400u);
        const u32 low = __builtin_amdgcn_perm(p23, p01, 0x05040100u);
        u32 miss;
        tp = top_pack(__builtin_amdgcn_perm(p23, p01, 0x07060302u), miss);
        l0 = __builtin_amdgcn_perm(qv[1], qv[0], 0x06050201u);
        l1 = __builtin_amdgcn_perm(qv[3], qv[2], 0x06050201u);
        sg[gq >> 3] = bfi(0x80808080u >> (gq & 7), low >> (gq & 7), sg[gq >> 3]);
        const bool odd_one = (fr[0] | fr[1] | fr[2] | fr[3] | (low & 0x7F7F7F7Fu) | (miss & 0x00400040u)) != 0u;
        const u64 bm = __ballot(odd_one);
        if (odd_one) {
          const u32 sl = min(63u, npend + __builtin_amdgcn_mbcnt_hi((u32)(bm >> 32), __builtin_amdgcn_mbcnt_lo((u32)bm, 0u)));
          *(LDS f32x4*)(pend + sl * 16) = acc[gq >> 2][gq & 3];
          *(LDS u32*)(pend + 1024 + sl * 4) = gq * 64 + lz;
        }
        npend += __popcll(bm);
      };
      u32 z0, z1, t15;
      quad(15, z0, z1, t15);
      *(LDS u32*)(last1k + 256 + lz * 4) = z0;
      *(LDS u32*)(last1k + 512 + lz * 4) = z1;
      #pragma unroll
      for (int mi = 0; mi < 4; ++mi) {
        u32 pk[12];
        #pragma unroll
        for (int ni = 0; ni < 4; ++ni) {
          const int gq = mi * 4 + ni;
          if (gq == 15) break;
          quad(gq, pk[ni * 3], pk[ni * 3 + 1], pk[ni * 3 + 2]);
          pk[ni * 3 + 2] = bfi(0x40004000u, gq < 14 ? t15 << (14 - gq) : t15 >> 1, pk[ni * 3 + 2]);
        }
        #pragma unroll
        for (int j = 0; j < (mi < 3 ? 3 : 2); ++j)
          ST128((u32x4{pk[j * 4], pk[j * 4 + 1], pk[j * 4 + 2], pk[j * 4 + 3]}), rsL, lz * 16 + j * 1024, offE + mi * 3072, EPI_AUX);
        if (mi == 3) *(LDS u32*)(last1k + lz * 4) = pk[8];
      }
      if (ti == slot && b < 72) STAMP(8);
      u32 ns = npend > 64u ? OS_CAP + 1 : 0u, nf = 0;
      auto put = [&](bool on, u32 e) {
        const u64 bm = __ballot(on);
        const u32 sl = ns + __builtin_amdgcn_mbcnt_hi((u32)(bm >> 32), __builtin_amdgcn_mbcnt_lo((u32)bm, 0u));
        if (on && sl < OS_CAP) *(LDS unsigned short*)(side + (sl < OS_CAP0 ? 196 : 256 - OS_CAP0 * 2) + sl * 2) = (unsigned short)e;
        ns += __popcll(bm);
      };
      u64 om = ((u64)sg[1] << 32) | sg[0];
      u32 odd = om ? 0u : 1u, last = 0;
      #pragma unroll
      for (int k = 0; k < 4; ++k) { if (om) last = (u32)__builtin_ctzll(om); om &= om - 1; odd |= last << (6 * k); }
      for (; __any(om != 0) && ns <= OS_CAP; om &= om - 1) {
        const u32 e = (u32)__builtin_ctzll(om | (1ull << 63));
        put(om != 0, (((e >> 5) * 8 + 7 - (e & 7)) << 11) | (lz << 5) | (e & 24u) | 4u);
      }
      if (npend) {
        wave_sync();
        const f32x4 pv = *(LDS f32x4*)(pend + lz * 16);
        const u32 pp = *(LDS u32*)(pend + 1024 + lz * 4);
        const bool on = (u32)lz < npend;
        u32 top = 0;
        for (int r = 0; r < 4; ++r) {
          const float v = r == 0 ? pv[0] : r == 1 ? pv[1] : r == 2 ? pv[2] : pv[3];
          const float tt = __builtin_amdgcn_ldexpf(v, g + 7);
          const u32 iv = (u32)f2i_sat(tt);
          top |= (iv >> 24) << (8 * r);
          const bool full = on && (as_u((float)(int)iv) != as_u(tt) || (iv & 31u));
          put(on && !full && (iv & 96u), (pp << 5) | (r << 3) | ((iv >> 5) & 3u));
          const u64 bm = __ballot(full);
          const u32 sl = nf + __builtin_amdgcn_mbcnt_hi((u32)(bm >> 32), __builtin_amdgcn_mbcnt_lo((u32)bm, 0u));
          if (full && sl < OF_CAP0) { *(LDS unsigned short*)(side + 244 + sl * 2) = (unsigned short)(pp * 4 + r); *(LDS u32*)(side + 248 + sl * 4) = as_u(v); }
          else if (full && sl < OF_CAP) *(LDS u32x2*)(side + 384 - OF_CAP0 * 8 + sl * 8) = u32x2{pp * 4 + r, as_u(v)};
          nf += __popcll(bm);
        }
        u32 miss;
        const u32 fl = (top_unpack(top_pack(top, miss)) ^ top) >> 5;
        for (int h = 0; h < 2; ++h) {
          const u32 d4 = ((fl >> (16 * h)) & 3u) | ((fl >> (16 * h + 6)) & 12u);
          put(on && d4, 0x8000u | (pp << 5) | (h << 4) | d4);
        }
      }
      const bool raw = ns > OS_CAP || nf > OF_CAP;
      *(LDS unsigned short*)(side + lz * 2) = (unsigned short)odd;
      *(LDS unsigned char*)(side + 128 + lz) = (unsigned char)(odd >> 16);
      if (lz == 0) *(LDS u32*)(side + 192) = raw ? 0x80000000u : (ns | (nf << 8) | ((u32)(g + 256) << 16));
      wave_sync();
      ST128(*(LDS u32x4*)(last1k + lz * 16), rsL, lz * 16, offE + 11264, AUX_SYS);
      if ((ns > OS_CAP0 || nf > OF_CAP0) && lz < 24) ST128(*(LDS u32x4*)(last1k + 1024 + lz * 16), rsL, lz * 16, offE + 12288, AUX_SYS);
      if (raw) {
        const u32 offR = par + EX_OUR + uid * OU_RAW;
        #pragma unroll
        for (int mi = 0; mi < 4; ++mi)
          #pragma unroll
          for (int ni = 0; ni < 4; ++ni)
            ST128(__builtin_bit_cast(u32x4, acc[mi][ni]), rsL, lz * 16 + ni * 1024, offR + mi * 4096, AUX_SYS);
      }
      if (ti == slot && b < 72) STAMP(9);
      vm_drain();
      if (lz == owner)
        __hip_atomic_store((GLB u32*)(peer_ws + OFF_FOUT + (rank * 320 + t) * 128), call, __ATOMIC_RELAXED, __HIP_MEMORY_SCOPE_SYSTEM);
    }
    if (ti == slot) STAMP(4); else STAMP(5);
  }
}

}

extern "C" void kernel_launch(int rank, int nranks, ncclComm_t comm, hipStream_t stream,
                              void* const* d_in, const int* in_sizes, int n_in,
                              void* d_out, int out_size,
                              const int* in_dtypes, int out_dtype,
                              void* const* d_out_all,
                              void* const* d_ws_all, size_t ws_bytes) {
  (void)comm; (void)in_sizes; (void)n_in; (void)out_size; (void)in_dtypes; (void)out_dtype; (void)d_out_all;
  if (nranks != NR || (unsigned)rank >= (unsigned)NR || ws_bytes < WS_NEED) { fprintf(stderr, "unexpected launch shape\n"); return; }
  static unsigned attr_set = 0;
  if (!(attr_set >> rank & 1u)) {
    HIP_CHECK(hipFuncSetAttribute((const void*)gemm_rs_kernel, hipFuncAttributeMaxDynamicSharedMemorySize, (int)LDS_BYTES));
    attr_set |= 1u << rank;
  }
  Params p;
  p.x = (const uint8_t*)d_in[0]; p.w = (const uint8_t*)d_in[1]; p.sx = (const float*)d_in[2]; p.sw = (const float*)d_in[3];
  p.out = (float*)d_out; p.ws_all = (char* const*)d_ws_all; p.ws = (char*)d_ws_all[rank]; p.rank = rank; p.ws_bytes = WS_NEED;
  hipLaunchKernelGGL(gemm_rs_kernel, dim3(NWG), dim3(NTHR), LDS_BYTES, stream, p);
  HIP_CHECK(hipGetLastError());
}
```

```cpp
#include <hip/hip_runtime.h>
#include <rccl/rccl.h>
#include <cstdio>
#include <cstdint>

#define HIP_CHECK(expr) do { hipError_t _e = (expr); if (_e != hipSuccess) { \
    fprintf(stderr, "HIP error %s at %s:%d\n", hipGetErrorString(_e), __FILE__, __LINE__); \
    return; } } while (0)

namespace {

typedef unsigned u32;
typedef unsigned long long u64;
typedef unsigned u32x4 __attribute__((ext_vector_type(4)));
typedef unsigned u32x2 __attribute__((ext_vector_type(2)));
typedef int i32x2 __attribute__((ext_vector_type(2)));
typedef int i32x8 __attribute__((ext_vector_type(8)));
typedef float f32x4 __attribute__((ext_vector_type(4)));
typedef __amdgpu_buffer_rsrc_t rsrc_t;
#define LDS __attribute__((address_space(3)))
#define GLB __attribute__((address_space(1)))

constexpr int NR = 8, MM = 4096, NN = 16384, KS = 512;
constexpr int NWG = 256, NTHR = 512;
constexpr int PULL_WGS = 8;
constexpr int WIN = 8;
constexpr u32 LOOK = 4, LOOK_OUT = 3;
constexpr int NRAW = 8, X1FIRST = 16;
constexpr int NPULL = 7 * PULL_WGS, NGEMM = NWG - NPULL;
constexpr int AUX_SYS = 17;
constexpr u64 DEADLINE_TICKS = 40000000ull;

constexpr u32 KiB = 1024, MiB = 1024 * 1024;
constexpr u32 OFF_CALL = 0;
constexpr u32 OFF_TMO = 32 * KiB;
constexpr u32 OFF_TIC = 33 * KiB;
constexpr u32 OFF_CNT = 40 * KiB;
constexpr u32 OFF_FIN = 128 * KiB;
constexpr u32 OFF_FOUT = OFF_FIN + 8 * 576 * 128;
constexpr u32 OFF_XA = 2 * MiB;
constexpr u32 OFF_WA = OFF_XA + 16 * MiB;
constexpr u32 OFF_PAR = OFF_WA + 16 * MiB;
constexpr u32 IU_ENC = 8 * KiB, IU_RAW = 8 * KiB;
constexpr u32 OU_ENC = 12672, OU_RAW = 16 * KiB;
constexpr u32 IU_SIDE = 6144, OU_SIDE = 12032;
constexpr u32 EX_INE = 0;
constexpr u32 EX_INR = EX_INE + 1280 * IU_ENC;
constexpr u32 EX_OUE = EX_INR + 1280 * IU_RAW;
constexpr u32 EX_OUR = EX_OUE + 2560 * OU_ENC;
constexpr u32 PAR_BYTES = EX_OUR + 2560 * OU_RAW;
constexpr u32 WS_NEED = OFF_PAR + 2 * PAR_BYTES;
static_assert(OFF_CNT + 96 * 5 * 128 <= OFF_FIN && OFF_FOUT + 8 * 320 * 128 <= OFF_XA, "flags");
static_assert(WS_NEED <= 257u * MiB, "workspace");
constexpr u32 QV = 5120, QV0 = 896, QV_STR = 64, QV_END = 1792, QV_ALL = 2304;
constexpr u32 Q_ESC0 = (QV0 - QV_STR) * 8 / 3, Q_ESC = (QV_END - QV_STR) * 8 / 3, Q_PAT0 = 30, Q_PAT = 286;
constexpr u32 IN_CAP0 = 62, IN_CAP = 254;
constexpr u32 OS_CAP0 = 24, OS_CAP = 64, OF_CAP0 = 2, OF_CAP = 32;

constexpr u32 STAGE = 48 * KiB;
constexpr u32 L_SCR = 64 * KiB, SCR_W = 10752;
constexpr u32 NSLOT = 8, SLOT = 13 * KiB, NCONS = 6;
constexpr u32 L_PSCR = NSLOT * SLOT;
constexpr u32 L_CTL = L_PSCR + NCONS * 8 * KiB;
constexpr u32 LDS_BYTES = L_CTL + 256;
static_assert(3 * STAGE <= L_CTL && L_SCR + 8 * SCR_W <= LDS_BYTES && LDS_BYTES <= 160 * KiB, "lds");

__device__ __forceinline__ bool quick_form(int i) { return i < 256 ? (i & 3) < 2 : (i & 0x3B) == 0; }
__device__ __forceinline__ int group_of(int rank, int j) { return rank < 4 ? rank * 4 + j : j * 4 + (rank & 3); }

struct Params {
  const uint8_t* x; const uint8_t* w; const float* sx; const float* sw;
  float* out; char* const* ws_all; char* ws; int rank; u32 ws_bytes;
};

__device__ __forceinline__ rsrc_t mk_rsrc(const void* p, u32 bytes) {
  u64 a = (u64)p;
  u32 lo = __builtin_amdgcn_readfirstlane((u32)a), hi = __builtin_amdgcn_readfirstlane((u32)(a >> 32));
  return __builtin_amdgcn_make_buffer_rsrc((void*)(((u64)hi << 32) | lo), 0, (int)bytes, 0x00020000);
}
#define ST128(v, rs, voff, soff, aux) __builtin_amdgcn_raw_buffer_store_b128((v), (rs), (int)((u32)(voff) + (u32)(soff)), 0, (aux))
__device__ __forceinline__ float as_f(u32 v) { return __builtin_bit_cast(float, v); }
__device__ __forceinline__ u32 as_u(float v) { return __builtin_bit_cast(u32, v); }
__device__ __forceinline__ int f2i_sat(float v) {
  int r; asm("v_cvt_i32_f32 %0, %1" : "=v"(r) : "v"(v)); return r;
}
__device__ __forceinline__ void vm_drain() { asm volatile("s_waitcnt vmcnt(0)" ::: "memory"); }
__device__ __forceinline__ void wave_sync() {
  asm volatile("s_waitcnt lgkmcnt(0)" ::: "memory");
}
__device__ __forceinline__ u32 bfi(u32 m, u32 a, u32 b) { return __builtin_amdgcn_bitop3_b32(m, a, b, 0xCA); }
template <int CTRL, int ROWS> __device__ __forceinline__ u32 dpp0(u32 v) { return (u32)__builtin_amdgcn_update_dpp(0, (int)v, CTRL, ROWS, 0xf, false); }
__device__ __forceinline__ u32 wave_scan_u32(u32 v) {
  v += dpp0<0x111, 0xf>(v); v += dpp0<0x112, 0xf>(v); v += dpp0<0x114, 0xf>(v); v += dpp0<0x118, 0xf>(v);
  v += dpp0<0x142, 0xa>(v); v += dpp0<0x143, 0xc>(v);
  return v;
}
__device__ __forceinline__ float wave_sum_f32(float v) {
  v += as_f(dpp0<0x111, 0xf>(as_u(v))); v += as_f(dpp0<0x112, 0xf>(as_u(v))); v += as_f(dpp0<0x114, 0xf>(as_u(v)));
  v += as_f(dpp0<0x118, 0xf>(as_u(v))); v += as_f(dpp0<0x142, 0xa>(as_u(v))); v += as_f(dpp0<0x143, 0xc>(as_u(v)));
  return as_f((u32)__builtin_amdgcn_readlane((int)as_u(v), 63));
}
__device__ __forceinline__ u32 ctl_ld(u32 addr) {
  u32 v; asm volatile("ds_read_b32 %0, %1\n\ts_waitcnt lgkmcnt(0)" : "=&v"(v) : "v"(addr) : "memory");
  return (u32)__builtin_amdgcn_readfirstlane((int)v);
}
__device__ __forceinline__ void ctl_st(u32 addr, u32 v) { asm volatile("ds_write_b32 %0, %1" :: "v"(addr), "v"(v) : "memory"); }
__device__ __forceinline__ u32 ld_flag(const char* ws, u32 off) {
  return __hip_atomic_load((const GLB u32*)(ws + off), __ATOMIC_RELAXED, __HIP_MEMORY_SCOPE_SYSTEM);
}
__device__ __forceinline__ bool wait_ge(char* ws, u32 off, u32 target, u32 call) {
  if (__all(ld_flag(ws, off) >= target)) return true;
  const u64 t0 = __builtin_amdgcn_s_memrealtime();
  GLB u32* tmo = (GLB u32*)(ws + OFF_TMO);
  for (u32 it = 1;; ++it) {
    __builtin_amdgcn_s_sleep(2);
    if (__all(ld_flag(ws, off) >= target)) return true;
    if ((it & 15u) == 0u &&
        (__hip_atomic_load(tmo, __ATOMIC_RELAXED, __HIP_MEMORY_SCOPE_AGENT) == call ||
         __builtin_amdgcn_s_memrealtime() - t0 > DEADLINE_TICKS)) {
      __hip_atomic_store(tmo, call, __ATOMIC_RELAXED, __HIP_MEMORY_SCOPE_AGENT);
      return false;
    }
  }
}
__device__ __forceinline__ void cnt_add(char* ws, u32 idx, int lane) {
  if (lane == 0) __hip_atomic_fetch_add((GLB u32*)(ws + OFF_CNT + idx * 128), 1u, __ATOMIC_RELAXED, __HIP_MEMORY_SCOPE_AGENT);
}

__device__ __forceinline__ void store_raw_unit(const u32x4 (&x)[8], rsrc_t rs, u32 off, int lane) {
  #pragma unroll
  for (int j = 0; j < 8; ++j) ST128(x[j], rs, lane * 16 + j * 1024, off, AUX_SYS);
}

__device__ __forceinline__ void export_half(const u32x4 (&x)[4], rsrc_t rs, u32 par, u32 uid, int hf, LDS char* scr, int lane) {
  const u32 offE = par + EX_INE + uid * IU_ENC;
  u32 s = 0;
  #pragma unroll
  for (int j = 0; j < 4; ++j) {
    *(LDS u32x4*)(scr + j * 1024 + lane * 16) = x[j];
    s = __builtin_amdgcn_sad_u8(x[j][j] & 0x7C7C7C7Cu, 0u, s);
  }
  s = (u32)__builtin_amdgcn_readlane((int)wave_scan_u32(s), 63);
  int e0 = ((int)s - 17121) >> 12;
  e0 = e0 < 0 ? 0 : (e0 > 24 ? 24 : e0);
  const u32 e4 = (u32)e0 * 0x04040404u;
  u32 mk[2] = {0, 0};
  u32 pk[12];
  #pragma unroll
  for (int j = 0; j < 4; ++j) {
    u32 o[4];
    #pragma unroll
    for (int c = 0; c < 4; ++c) {
      const u32 u = (x[j][c] | 0x80808080u) - e4;
      o[c] = bfi(0x1F1F1F1Fu, u, x[j][c]);
      const u32 t = (((u >> 1) ^ 0x40404040u) & 0x70707070u) + 0x70707070u;
      const int d = j * 4 + c;
      mk[d >> 3] = bfi(0x80808080u >> (d & 7), t >> (d & 7), mk[d >> 3]);
    }
    pk[j * 3 + 0] = bfi(0x60606060u, o[3] << 5, o[0]);
    pk[j * 3 + 1] = bfi(0x60606060u, o[3] << 3, o[1]);
    pk[j * 3 + 2] = bfi(0x60606060u, bfi(0x40404040u, o[3] >> 1, o[3] << 1), o[2]);
  }
  #pragma unroll
  for (int j = 0; j < 3; ++j)
    ST128((u32x4{pk[j * 4], pk[j * 4 + 1], pk[j * 4 + 2], pk[j * 4 + 3]}), rs, lane * 16 + j * 1024, offE + hf * 3072, AUX_SYS);
  const u32 mine = __popc(mk[0]) + __popc(mk[1]);
  const u32 inc = wave_scan_u32(mine);
  const u32 total = (u32)__builtin_amdgcn_readlane((int)inc, 63);
  if (total <= IN_CAP) {
    LDS unsigned short* list = (LDS unsigned short*)(scr + 8192);
    u32 slot = 2 + inc - mine;
    #pragma unroll
    for (int q = 0; q < 2; ++q) {
      u32 m = mk[q];
      while (m) {
        const int bp = __builtin_ctz(m); m &= m - 1;
        const int d = q * 8 + 7 - (bp & 7);
        list[slot++] = (unsigned short)((d >> 2) * 1024 + lane * 16 + (d & 3) * 4 + (bp >> 3));
      }
    }
    wave_sync();
    for (u32 j = 0; j < 2; ++j) {
      const u32 pp = *(LDS u32*)(scr + 8192 + j * 256 + lane * 4);
      const u32 p0 = pp & 4095u, p1 = (pp >> 16) & 4095u;
      const u32 b0 = *(LDS unsigned char*)(scr + p0), b1 = *(LDS unsigned char*)(scr + p1);
      u32 sd = (((p0 << 3) | ((128u + (b0 & 127u) - 4u * (u32)e0) >> 5) | (p1 << 19) | (((128u + (b1 & 127u) - 4u * (u32)e0) >> 5) << 16))) + hf * 0x80008000u;
      if (j == 0 && lane == 0) sd = total | ((u32)e0 << 16);
      const u32 at = j * 64 + lane;
      if (at < 32 || total > IN_CAP0)
        __builtin_amdgcn_raw_buffer_store_b32(sd, rs, at < 32 ? hf * 128 + at * 4 : 128 + hf * 384 + at * 4, offE + IU_SIDE, AUX_SYS);
      if (total <= 126) break;
    }
  } else {
    #pragma unroll
    for (int j = 0; j < 4; ++j) ST128(x[j], rs, lane * 16 + j * 1024, par + EX_INR + uid * IU_RAW + hf * 4096, AUX_SYS);
    if (lane == 0) __builtin_amdgcn_raw_buffer_store_b32(0x80000000u, rs, hf * 128, offE + IU_SIDE, AUX_SYS);
  }
}

__device__ __forceinline__ void export_short(const u32x4 (&x)[8], rsrc_t rs, u32 par, u32 uid, LDS char* scr, int lane) {
  typedef unsigned short u16x2 __attribute__((ext_vector_type(2)));
  const u32 offE = par + EX_INE + uid * IU_ENC, offR = par + EX_INR + uid * IU_RAW;
  LDS char* var = scr + 8192;
  LDS char* tab = var + QV_ALL;
  u32 s = 0;
  #pragma unroll
  for (int j = 0; j < 8; ++j) {
    *(LDS u32x4*)(scr + j * 1024 + lane * 16) = x[j];
    s = __builtin_amdgcn_sad_u8(x[j][j & 3] & 0x7C7C7C7Cu, 0u, s);
  }
  *(LDS u32x4*)(var + lane * 16) = u32x4{0u, 0u, 0u, 0u};
  *(LDS u32x4*)(var + 1024 + lane * 16) = u32x4{0u, 0u, 0u, 0u};
  {
    u32 sel = 0x0C0C0C0Cu, at = 0;
    #pragma unroll
    for (int y = 0; y < 4; ++y) if ((lane >> y) & 1) { sel = (sel & ~(0xFFu << at)) | ((u32)y << at); at += 8; }
    if (lane < 16) *(LDS u32*)(tab + lane * 4) = sel;
  }
  s = (u32)__builtin_amdgcn_readlane((int)wave_scan_u32(s), 63);
  const int K = min(31, ((int)s >> 13) + 3);
  const u32 kw = (128u + 4u * (u32)K) * 0x01010101u, tabo = (u32)(uintptr_t)tab;
  u32 mk[4] = {0, 0, 0, 0}, zq[32], car[20], bits = 0;
  wave_sync();
  #pragma unroll
  for (int g = 0; g < 4; ++g) {
    u32 a[8];
    #pragma unroll
    for (int k = 0; k < 8; ++k) {
      const int d = g * 8 + k;
      const u32 xv = x[d >> 2][d & 3];
      const u32 w = kw - (xv & 0x7C7C7C7Cu);
      const u32 sel = (w >> 2) & 0x07070707u;
      const u32 hi8 = __builtin_bit_cast(u32, __builtin_bit_cast(u16x2, (w >> 5) & 0x01010101u) * (u16x2){255, 255});
      const u32 l0 = __builtin_amdgcn_perm(0x7D6D5D4Du, 0x0004083Du, sel), l1 = __builtin_amdgcn_perm(0xFDEDDDCDu, 0x802D1D0Du, sel);
      const u32 L = (hi8 & l1) | (~hi8 & l0);
      const u32 pf = L | ~w | (w << 1);
      mk[g] = bfi(0x80808080u >> k, pf >> k, mk[g]);
      a[k] = bfi(0x0C0C0C0Cu, L, xv);
      const u32 F = L & 0x01010101u;
      const u32 cs = __builtin_amdgcn_perm(0u, (L >> 4) & 0x07070707u, *(LDS u32*)(uintptr_t)__builtin_amdgcn_udot4(F, 0x20100804u, tabo, false));
      zq[d] = (__builtin_amdgcn_udot4(cs, 0x00400801u, 0u, false) + ((cs >> 24) << 9)) | (bits << 16);
      bits = __builtin_amdgcn_udot4(F, 0x03030303u, bits, false);
    }
    car[g * 5 + 0] = bfi(0x70707070u, a[5] << 4, a[0]);
    car[g * 5 + 1] = bfi(0x70707070u, a[6] << 4, a[1]);
    car[g * 5 + 2] = bfi(0x70707070u, a[7] << 4, a[2]);
    car[g * 5 + 3] = bfi(0x40404040u, a[7] << 3, bfi(0x20202020u, a[6] << 2, bfi(0x10101010u, a[5] << 1, a[3])));
    car[g * 5 + 4] = bfi(0x40404040u, a[7] >> 1, bfi(0x20202020u, a[6] >> 2, bfi(0x10101010u, a[5] >> 3, a[4])));
  }
  #pragma unroll
  for (int j = 0; j < 5; ++j) ST128((u32x4{car[j * 4], car[j * 4 + 1], car[j * 4 + 2], car[j * 4 + 3]}), rs, lane * 16 + j * 1024, offE, AUX_SYS);
  const u32 mine = __popc(mk[0]) + __popc(mk[1]) + __popc(mk[2]) + __popc(mk[3]);
  const u32 inc = wave_scan_u32(bits | (mine << 16));
  const u32 tot = (u32)__builtin_amdgcn_readlane((int)inc, 63), nbit = tot & 0xFFFFu, npat = tot >> 16;
  if (nbit <= Q_ESC * 3 && npat <= Q_PAT) {
    const u32 base = QV_STR * 8 + (inc & 0xFFFFu) - bits;
    #pragma unroll
    for (int d = 0; d < 32; ++d) {
      const u32 q = base + (zq[d] >> 16);
      const u64 v = (u64)(zq[d] & 0xFFFFu) << (q & 31u);
      LDS u32* at = (LDS u32*)(var + ((q >> 5) << 2));
      __hip_atomic_fetch_or(at, (u32)v, __ATOMIC_RELAXED, __HIP_MEMORY_SCOPE_WORKGROUP);
      __hip_atomic_fetch_or(at + 1, (u32)(v >> 32), __ATOMIC_RELAXED, __HIP_MEMORY_SCOPE_WORKGROUP);
    }
    u32 slot = (inc >> 16) - mine;
    #pragma unroll
    for (int q = 0; q < 4; ++q) {
      u32 m = mk[q];
      while (m) {
        const int bp = __builtin_ctz(m); m &= m - 1;
        const int d = q * 8 + 7 - (bp & 7);
        const u32 pos = (d >> 2) * 1024 + lane * 16 + (d & 3) * 4 + (bp >> 3);
        const int v = K - (int)((*(LDS unsigned char*)(scr + pos) >> 2) & 31u), t = v & 15;
        *(LDS unsigned short*)(var + (slot < Q_PAT0 ? 4 + slot * 2 : QV_END - Q_PAT0 * 2 + slot * 2)) =
            (unsigned short)((pos << 3) | (u32)(((v - (t < 11 ? t : t - 8)) >> 3) + 4));
        ++slot;
      }
    }
    if (lane == 0) *(LDS u32*)var = nbit | (npat << 15) | ((u32)K << 24);
    wave_sync();
    if (lane < (int)QV0 / 16) ST128(*(LDS u32x4*)(var + lane * 16), rs, lane * 16, offE + QV, AUX_SYS);
    if (nbit > Q_ESC0 * 3 && lane < (int)(QV_END - QV0) / 16) ST128(*(LDS u32x4*)(var + QV0 + lane * 16), rs, lane * 16, offE + QV + QV0, AUX_SYS);
    if (npat > Q_PAT0 && lane < (int)(QV_ALL - QV_END) / 16) ST128(*(LDS u32x4*)(var + QV_END + lane * 16), rs, lane * 16, offE + QV + QV_END, AUX_SYS);
  } else {
    store_raw_unit(x, rs, offR, lane);
    if (lane == 0) __builtin_amdgcn_raw_buffer_store_b32(0x80000000u, rs, 0, offE + QV, AUX_SYS);
  }
}

__device__ __forceinline__ void patch_short(LDS char* scr, u32 ent) {
  const u32 pos = ent >> 3;
  __hip_atomic_fetch_add((LDS u32*)(scr + (pos & ~3u)), ((4u - (ent & 7u)) << 5) << ((pos & 3u) * 8u), __ATOMIC_RELAXED, __HIP_MEMORY_SCOPE_WORKGROUP);
}
__device__ __forceinline__ void patch_in(LDS char* scr, u32 ent) {
  const u32 pos = ent >> 3;
  __hip_atomic_fetch_add((LDS u32*)(scr + (pos & ~3u)), (((ent & 7u) - 4u) << 5) << ((pos & 3u) * 8u), __ATOMIC_RELAXED, __HIP_MEMORY_SCOPE_WORKGROUP);
}

__device__ __forceinline__ float unfix(int i, int g) { return __builtin_amdgcn_ldexpf((float)i, -g); }
__device__ __forceinline__ u32 top_pack(u32 t, u32& miss) {
  const u32 m = __builtin_amdgcn_perm(0x00010208u, 0x08020100u, (t >> 5) & 0x07070707u);
  miss = __umul24(m >> 8, 3u) + m;
  return bfi(0x20002000u, miss << 11, bfi(0x00600060u, miss << 5, t));
}
__device__ __forceinline__ u32 top_unpack(u32 x) {
  const u32 r = __builtin_amdgcn_perm(0x24044222u, 0x02402000u, bfi(0x00030003u, x >> 5, x >> 11) & 0x00070007u);
  const u32 u = bfi(0x60006000u, r << 12, bfi(0x00600060u, r, x));
  return __builtin_amdgcn_bitop3_b32(__builtin_amdgcn_bitop3_b32(u, x >> 1, 0x40404040u, 0x78), x >> 2, 0x20202020u, 0x78);
}

__global__ void __launch_bounds__(NTHR, 2) gemm_rs_kernel(Params p) {
  extern __shared__ __attribute__((aligned(16))) char smem_[];
  LDS char* smem = (LDS char*)smem_;
  const int tid = threadIdx.x, lane = tid & 63, wave = __builtin_amdgcn_readfirstlane(tid >> 6);
  const int b = blockIdx.x, rank = p.rank;
  const int lg = lane >> 4, li = lane & 15;
  const rsrc_t rsL = mk_rsrc(p.ws, p.ws_bytes);
  const u32 tick = (u32)__builtin_amdgcn_raw_ptr_buffer_atomic_add_i32(1, rsL, lane == 0 ? 0 : (int)0xF0000000u, OFF_CALL + b * 128 + wave * 16, 0);
  char* const peer_ws = p.ws_all[lane & 7];
  u32 call;

  if (b < 128) {
    const rsrc_t rsW = mk_rsrc(p.w, KS * NN);
    #pragma unroll
    for (int i = 0; i < 8; ++i) {
      const int row = (wave * 8 + i) * 8 + (lane >> 3);
      const int sw = ((row >> 1) & 3) | (((row >> 4) & 1) << 2);
      __builtin_amdgcn_raw_ptr_buffer_load_lds(rsW, (LDS void*)(smem + (wave * 8 + i) * 1024), 16, row * NN + (((lane & 7) ^ sw) << 4), b * 128, 0, 0);
    }
    asm volatile("s_waitcnt vmcnt(0)\n\ts_barrier" ::: "memory");
    call = __builtin_amdgcn_readfirstlane(tick) + 1u;
    const int cgl = wave & 1, kb = wave >> 1;
    u32x4 x[8];
    #pragma unroll
    for (int f = 0; f < 4; ++f)
      #pragma unroll
      for (int h = 0; h < 2; ++h)
        #pragma unroll
        for (int t = 0; t < 2; ++t) {
          const int row = kb * 128 + 64 * h + 16 * lg + 8 * t + (li >> 1);
          const int sw = ((row >> 1) & 3) | (((row >> 4) & 1) << 2);
          const i32x2 v = __builtin_amdgcn_ds_read_tr8_b64_v2i32((LDS i32x2*)(smem + row * 128 + (((cgl * 4 + f) ^ sw) << 4) + (li & 1) * 8));
          x[f * 2 + h][t * 2] = v.x; x[f * 2 + h][t * 2 + 1] = v.y;
        }
    const int gl = b >> 5, gr = 4 + ((b >> 3) & 3), c = b & 7;
    if (kb == 0 && c == 0) {
      #pragma unroll 1
      for (int hf = 0; hf < 2; ++hf) {
        const u32x4 xh[4] = {hf ? x[4] : x[0], hf ? x[5] : x[1], hf ? x[6] : x[2], hf ? x[7] : x[3]};
        export_half(xh, rsL, OFF_PAR + (call & 1) * PAR_BYTES, 256 + (b * 2 + cgl) * 4, hf, smem + L_SCR + wave * SCR_W, lane);
      }
    } else export_short(x, rsL, OFF_PAR + (call & 1) * PAR_BYTES, 256 + (b * 2 + cgl) * 4 + kb, smem + L_SCR + wave * SCR_W, lane);
    vm_drain();
    if ((lane == gl || lane == gr) && lane != rank) {
      const int cb = (lane == gl ? gr & 3 : gl) * 8 + c;
      __hip_atomic_store((GLB u32*)(peer_ws + OFF_FIN + (rank * 576 + 256 + (cb * 2 + cgl) * 4 + kb) * 128), call, __ATOMIC_RELAXED, __HIP_MEMORY_SCOPE_SYSTEM);
    }
    if (rank == gl || rank == gr) {
      const int cb = (rank == gl ? gr & 3 : gl) * 8 + c;
      store_raw_unit(x, rsL, OFF_WA + ((rank * 4 + kb) * 64 + cb * 2 + cgl) * IU_RAW, lane);
      vm_drain();
      cnt_add(p.ws, (64 + cb) * 5 + 4, lane);
    }
  } else if (b < 192) {
    const int rg = b - 128, kb = wave & 3, hf = wave >> 2, u = rg * 4 + kb;
    const rsrc_t rsX = mk_rsrc(p.x, MM * KS);
    const u32 both = (u32)(uintptr_t)(smem + L_CTL) + 16;
    const u32 xoff = OFF_XA + ((rank * 4 + kb) * 64 + rg) * IU_RAW;
    const int ow = rg >> 3, r8 = rg & 7;
    const u32 users = ((r8 <= 3 ? 0xFu << (ow & 4) : 0u) | (r8 >= 3 ? 0xFu << (~ow & 4) : 0u) | (1u << ow)) & ~(1u << rank);
    const bool tell = lane < NR && ((users >> lane) & 1);
    if (kb < 2) {
      u32x4 x[4];
      #pragma unroll
      for (int j = 0; j < 4; ++j)
        x[j] = __builtin_amdgcn_raw_buffer_load_b128(rsX, ((j >> 1) * 16 + li) * KS + (j & 1) * 64 + lg * 16, (rg * 64 + hf * 32) * KS + kb * 128, 0);
      if (wave == 0) ctl_st(both + (lane & 3) * 4, 0u);
      asm volatile("s_waitcnt lgkmcnt(0)\n\ts_barrier" ::: "memory");
      call = __builtin_amdgcn_readfirstlane(tick) + 1u;
      const u32 parw = OFF_PAR + (call & 1) * PAR_BYTES;
      auto second = [&](int k) -> bool {
        u32 old = 0;
        if (lane == 0) old = __hip_atomic_fetch_add((LDS u32*)(uintptr_t)(both + k * 4), 1u, __ATOMIC_RELAXED, __HIP_MEMORY_SCOPE_WORKGROUP);
        return __builtin_amdgcn_readfirstlane((int)old) == 1;
      };
      if (NRAW && kb == 0) {
        #pragma unroll
        for (int j = 0; j < 4; ++j) ST128(x[j], rsL, lane * 16 + j * 1024, parw + EX_INR + u * IU_RAW + hf * 4096, AUX_SYS);
        vm_drain();
        if (second(2) && tell) __hip_atomic_store((GLB u32*)(peer_ws + OFF_FIN + (rank * 576 + 512 + rg) * 128), call, __ATOMIC_RELAXED, __HIP_MEMORY_SCOPE_SYSTEM);
      }
      export_half(x, rsL, parw, u, hf, smem + L_SCR + wave * SCR_W, lane);
      vm_drain();
      if (second(kb) && tell) __hip_atomic_store((GLB u32*)(peer_ws + OFF_FIN + (rank * 576 + u) * 128), call, __ATOMIC_RELAXED, __HIP_MEMORY_SCOPE_SYSTEM);
      #pragma unroll
      for (int j = 0; j < 4; ++j) ST128(x[j], rsL, lane * 16 + j * 1024, xoff + hf * 4096, AUX_SYS);
      vm_drain();
      cnt_add(p.ws, rg * 5 + 4, lane);
    } else if (hf == 0) {
      u32x4 x[8];
      #pragma unroll
      for (int j = 0; j < 8; ++j)
        x[j] = __builtin_amdgcn_raw_buffer_load_b128(rsX, ((j >> 1) * 16 + li) * KS + (j & 1) * 64 + lg * 16, rg * 64 * KS + kb * 128, 0);
      asm volatile("s_barrier" ::: "memory");
      call = __builtin_amdgcn_readfirstlane(tick) + 1u;
      export_short(x, rsL, OFF_PAR + (call & 1) * PAR_BYTES, u, smem + L_SCR + wave * SCR_W, lane);
      vm_drain();
      if (tell) __hip_atomic_store((GLB u32*)(peer_ws + OFF_FIN + (rank * 576 + u) * 128), call, __ATOMIC_RELAXED, __HIP_MEMORY_SCOPE_SYSTEM);
      store_raw_unit(x, rsL, xoff, lane);
      vm_drain();
      cnt_add(p.ws, rg * 5 + 4, lane);
    } else {
      asm volatile("s_barrier" ::: "memory");
    }
  }
  call = __builtin_amdgcn_readfirstlane(tick) + 1u;
  const u32 par = OFF_PAR + (call & 1) * PAR_BYTES;
  const float scale = p.sx[0] * p.sw[0];
  const rsrc_t rsO = mk_rsrc(p.out, 512u * NN * 4u);

  if (b >= NGEMM) {
    const int pb = b - NGEMM, link = pb / PULL_WGS, q = pb % PULL_WGS;
    const int src = (rank + 1 + link) & 7;
    const u64 pw = (u64)peer_ws;
    const u32 pw_lo = (u32)__builtin_amdgcn_readlane((int)(u32)pw, src), pw_hi = (u32)__builtin_amdgcn_readlane((int)(u32)(pw >> 32), src);
    const rsrc_t rsP = mk_rsrc((const void*)(((u64)pw_hi << 32) | (u64)pw_lo), p.ws_bytes);
    const u32 ctl = (u32)(uintptr_t)(smem + L_CTL);
    const u32 C_READY = ctl, C_LANDED = ctl + 4, C_CLOSED = ctl + 12, C_REL = ctl + 16, C_ISSUED = ctl + 8, C_Q = ctl + 64;
    const bool same = (src >> 2) == (rank >> 2);
    if (wave == 0) ctl_st(ctl + lane * 4, 0u);
    asm volatile("s_waitcnt lgkmcnt(0)\n\ts_barrier" ::: "memory");
    if (wave == 0) {
      int g = 0, pub = 0;
      #define LANDED(v) do { if ((v) > pub) { pub = (v); ctl_st(C_LANDED, (u32)pub); } } while (0)
      #define PIECE(SZ, VOFF, OFF) do { __builtin_amdgcn_raw_ptr_buffer_load_lds(rsP, (LDS void*)(dst + (OFF)), SZ, VOFF, so + (OFF), 0, AUX_SYS); \
        asm volatile("s_waitcnt vmcnt(%0)" :: "i"(WIN - 1) : "memory"); ++g; LANDED(g - (WIN - 1)); } while (0)
      for (u32 n = 0;; ++n) {
        if (ctl_ld(C_READY) <= n) {
          vm_drain(); LANDED(g);
          bool fin = false;
          while (ctl_ld(C_READY) <= n) { if (ctl_ld(C_CLOSED)) { fin = ctl_ld(C_READY) <= n; break; } __builtin_amdgcn_s_sleep(1); }
          if (fin) break;
        }
        const u32 ent = ctl_ld(C_Q + (n & 15) * 4);
        ctl_st(C_ISSUED, n + 1);
        if (ent >> 31) continue;
        const u32 need = n >= NSLOT ? n - NSLOT + 1 : 0, ra = C_REL + (n % NSLOT) * 4;
        if (ctl_ld(ra) < need) {
          vm_drain(); LANDED(g);
          while (ctl_ld(ra) < need) __builtin_amdgcn_s_sleep(1);
        }
        const int i = ent & 1023;
        LDS char* dst = smem + (n % NSLOT) * SLOT;
        if (ent & 1024) {
          const u32 so = par + EX_INR + i * IU_RAW;
          #pragma unroll
          for (int j = 0; j < 8; ++j) PIECE(16, lane * 16, j * 1024);
        } else if (i < 512 && quick_form(i)) {
          const u32 so = par + EX_INE + (i < 256 ? i : 256 + group_of(rank, (i >> 6) & 3) * 64 + (i & 63)) * IU_ENC;
          PIECE(4, lane * 4, IU_SIDE);
          #pragma unroll
          for (int j = 0; j < 6; ++j) PIECE(16, lane * 16, j * 1024);
        } else if (i < 512) {
          const u32 so = par + EX_INE + (i < 256 ? i : 256 + group_of(rank, (i >> 6) & 3) * 64 + (i & 63)) * IU_ENC;
          if (lane < (int)QV0 / 16) __builtin_amdgcn_raw_ptr_buffer_load_lds(rsP, (LDS void*)(dst + QV), 16, lane * 16, so + QV, 0, AUX_SYS);
          asm volatile("s_waitcnt vmcnt(%0)" :: "i"(WIN - 1) : "memory"); ++g; LANDED(g - (WIN - 1));
          #pragma unroll
          for (int j = 0; j < 5; ++j) PIECE(16, lane * 16, j * 1024);
        } else {
          const u32 so = par + EX_OUE + (rank * 320 + i - 512) * OU_ENC;
          #pragma unroll
          for (int j = 0; j < 12; ++j) PIECE(16, lane * 16, (j + 11) % 12 * 1024);
        }
      }
      vm_drain(); LANDED(g);
      #undef PIECE
      #undef LANDED
      return;
    }
    if (wave == 1) {
      const u32 tic0 = OFF_TIC + (call & 1) * 24 * 128;
      const int only0 = lane == 0 ? 0 : (int)0xF0000000u;
      if (q == 0 && (lane == link || (link == 0 && lane >= 8 && lane < 24)))
        __hip_atomic_store((GLB u32*)(p.ws + OFF_TIC + ((~call & 1) * 24 + lane) * 128), 0u, __ATOMIC_RELAXED, __HIP_MEMORY_SCOPE_AGENT);
      constexpr u32 NPAIR = 208 + NRAW / 2, NFIX = 192;
      u32 n = 0, endp = 0; bool ok = true;
      int fj = -1, fnext = -1;
      const u32 gtic = tic0 + (8 + group_of(src, lane & 3)) * 128;
      auto fullest = [&](u32 v) -> int {
        v = v >= 16u || (!same && (lane & 3) == (rank & 3)) ? 0xFFFFu : (v << 4) | ((((lane - q) & 3) << 2) | (lane & 3));
        v = min(v, (u32)__builtin_amdgcn_mov_dpp((int)v, 0xB1, 0xf, 0xf, true));
        v = min(v, (u32)__builtin_amdgcn_mov_dpp((int)v, 0x4E, 0xf, 0xf, true));
        v = (u32)__builtin_amdgcn_readfirstlane((int)v);
        return v == 0xFFFFu ? -2 : (int)(v & 3u);
      };
      auto draw = [&](int g) { return __builtin_amdgcn_raw_ptr_buffer_atomic_add_i32(1, rsL, only0, tic0 + (g < 0 ? link : 8 + group_of(src, g)) * 128, 0); };
      int tnext = draw(-1);
      u32 seen = 0;
      for (;;) {
        u32 t = (u32)__builtin_amdgcn_readfirstlane(tnext);
        fj = fnext;
        if (t >= (fj < 0 ? NPAIR + NFIX : 16u)) {
          if ((fnext = fullest(ld_flag(p.ws, gtic))) < 0) break;
          tnext = draw(fnext);
          continue;
        }
        while (n >= ctl_ld(C_ISSUED) + (t < NPAIR && fj < 0 ? LOOK : LOOK_OUT)) __builtin_amdgcn_s_sleep(1);
        bool last = false;
        if (fj >= 0 || t + PULL_WGS >= NPAIR + NFIX) last = (fnext = fullest(seen)) < 0;
        if (last && fj < 0) { fnext = -1; last = false; }
        if (!last) tnext = draw(fnext);
        if (t >= NPAIR || fj >= 0) seen = ld_flag(p.ws, gtic);
        if (fj >= 0) t += NPAIR + NFIX;
        int i;
        if (t < NPAIR) {
          const int e = t < NRAW ? t : 2 * t - NRAW + (lane & 1);
          int u = -1, sg = 0, kb = 0;
          if (e < 32 + X1FIRST) { kb = e >= NRAW && e < NRAW + X1FIRST; u = e < NRAW ? e : kb ? e - NRAW : e - X1FIRST; }
          else if (e < 40 + X1FIRST) sg = 0;
          else if (e < 72) { kb = 1; u = e - 40; }
          else if (e < 88) sg = (e - 64) >> 3;
          else if (e < 120) { kb = 2; u = e - 88; }
          else if (e < 144) sg = (e - 96) >> 3;
          else if (e < 176) { kb = 3; u = e - 144; }
          else sg = (e - 128) >> 3;
          int c = 0;
          if (u < 0) {
            if (sg < 6) { c = (0x210100 >> (sg * 4)) & 7; kb = (0x012010 >> (sg * 4)) & 3; }
            else if (sg < 26) { c = ((sg - 6) >> 2) + ((sg - 6) & 3); kb = 3 - ((sg - 6) & 3); }
            else { c = (0x776765 >> ((sg - 26) * 4)) & 7; kb = (0x323123 >> ((sg - 26) * 4)) & 3; }
          }
          const int m = c * 8 + (e & 7);
          const int v = u - 12;
          if (e >= 384) i = (8 * rank + (e & 7)) * 4 + ((e - 384) >> 3);
          else if (u < 0) i = 256 + ((((m >> 1) & 3) * 8 + (m >> 3)) * 2 + (m & 1)) * 4 + kb;
          else if (u < 12) i = (8 * ((rank & 4) | ((rank + 1 + (u >> 2)) & 3)) + (u & 3)) * 4 + kb;
          else i = (8 * ((~rank & 4) | (v / 5)) + 3 + v % 5) * 4 + kb;
        } else if (fj < 0) {
          const int o = t - NPAIR, cm = same ? o / 3 : o >> 2, k = same ? o - cm * 3 : 1 + (o & 3);
          const int c = same ? cm >> 3 : cm / 6, j3 = (cm - c * 6) >> 1, j = same ? (cm >> 1) & 3 : j3 + (j3 >= (rank & 3));
          i = 512 + ((j * 8 + c) * 2 + (cm & 1)) * 5 + k;
        } else i = 512 + (fj * 16 + (int)(t - NPAIR - NFIX)) * 5 + (same ? 3 : 0);
        const bool unp = t < NRAW, two = !unp && t < NPAIR;
        const u32 fo = i >= 512 ? OFF_FOUT + (src * 320 + i - 512) * 128 : OFF_FIN + (src * 576 + (unp ? 512 + (i >> 2) : i)) * 128;
        ok = ok && wait_ge(p.ws, fo, call, call);
        const u32 k = two ? (lane & 1) : 0, per = !ok ? 0 : t >= NPAIR ? 12 : unp ? 8 : (u32)__builtin_amdgcn_readfirstlane(quick_form(i) ? 7 : 6);
        ctl_st(C_Q + ((n + k) & 15) * 4, (u32)i | (unp ? 1024u : 0u) | ((endp + per * (k + 1)) << 11) | (ok ? 0u : 0x80000000u));
        n += two ? 2 : 1; endp += two ? 2 * per : per;
        asm volatile("s_waitcnt lgkmcnt(0)" ::: "memory");
        ctl_st(C_READY, n);
        if (last) break;
      }
      asm volatile("s_waitcnt lgkmcnt(0)" ::: "memory");
      ctl_st(C_CLOSED, 1u);
      return;
    }
    LDS char* scr = smem + L_PSCR + (wave - 2) * 8 * KiB;
    for (u32 n = wave - 2;; n += NCONS) {
      bool fin = false;
      while (ctl_ld(C_READY) <= n) { if (ctl_ld(C_CLOSED)) { fin = ctl_ld(C_READY) <= n; break; } __builtin_amdgcn_s_sleep(1); }
      if (fin) break;
      const u32 ent = ctl_ld(C_Q + (n & 15) * 4);
      const bool ok = !(ent >> 31);
      const int i = ent & 1023;
      const u32 endp = (ent >> 11) & 0x1FFFu;
      auto landed = [&](u32 upto) { while (ctl_ld(C_LANDED) < upto) __builtin_amdgcn_s_sleep(1); };
      LDS char* sl = smem + (n % NSLOT) * SLOT;
      if (i < 512) {
        if (!ok) continue;
        landed(endp);
        const int kb = i & 3, rg = (i >> 2) & 63; const bool isx = i < 256;
        const u32 uid = isx ? i : 256 + group_of(rank, (i >> 6) & 3) * 64 + (i & 63);
        const u32 dst = (isx ? OFF_XA : OFF_WA) + ((src * 4 + kb) * 64 + rg) * IU_RAW;
        const u32 rel = C_REL + (n % NSLOT) * 4, offE = par + EX_INE + uid * IU_ENC;
        u32x4 x[8];
        bool raw = false;
        if (ent & 1024) {
          #pragma unroll
          for (int j = 0; j < 8; ++j) x[j] = *(LDS u32x4*)(sl + j * 1024 + lane * 16);
          wave_sync();
          ctl_st(rel, n + 1);
        } else if (quick_form(i)) {
          u32x4 c[6];
          #pragma unroll
          for (int j = 0; j < 6; ++j) c[j] = *(LDS u32x4*)(sl + j * 1024 + lane * 16);
          const u32 sd = *(LDS u32*)(sl + IU_SIDE + lane * 4);
          wave_sync();
          ctl_st(rel, n + 1);
          const u32 hd[2] = {(u32)__builtin_amdgcn_readlane((int)sd, 0), (u32)__builtin_amdgcn_readlane((int)sd, 32)};
          #pragma unroll
          for (int j = 0; j < 8; ++j) {
            const u32 e4 = ((hd[j >> 2] >> 16) & 31u) * 0x04040404u;
            const u32 p0 = c[(j * 3) >> 2][(j * 3) & 3], p1 = c[(j * 3 + 1) >> 2][(j * 3 + 1) & 3], p2 = c[(j * 3 + 2) >> 2][(j * 3 + 2) & 3];
            const u32 o3 = bfi(0x80808080u, p2 << 1, bfi(0x10101010u, p2 >> 1, bfi(0x0C0C0C0Cu, p1 >> 3, p0 >> 5)));
            const u32x4 y = {(p0 & 0x9F9F9F9Fu) + e4, (p1 & 0x9F9F9F9Fu) + e4, (p2 & 0x9F9F9F9Fu) + e4, (o3 & 0x9F9F9F9Fu) + e4};
            *(LDS u32x4*)(scr + j * 1024 + lane * 16) = y;
          }
          wave_sync();
          const u32 cnt = (lane < 32 ? hd[0] : hd[1]) & 0xFFFFu, id0 = 2u * (lane & 31) - 2u;
          if (lane & 31) {
            if (id0 < cnt) patch_in(scr, sd & 0xFFFFu);
            if (id0 + 1 < cnt) patch_in(scr, sd >> 16);
          }
          if (max(hd[0] & 0xFFFFu, hd[1] & 0xFFFFu) > IN_CAP0) {
            for (int j = 0; j < 3; ++j) {
              const u32 s2 = __builtin_amdgcn_raw_buffer_load_b32(rsP, 256 + (lane >> 5) * 384 + (j * 32 + (lane & 31)) * 4, offE + IU_SIDE, AUX_SYS);
              const u32 id = IN_CAP0 + j * 64 + 2 * (lane & 31);
              if (id < cnt) patch_in(scr, s2 & 0xFFFFu);
              if (id + 1 < cnt) patch_in(scr, s2 >> 16);
            }
          }
          wave_sync();
          #pragma unroll
          for (int j = 0; j < 8; ++j) x[j] = *(LDS u32x4*)(scr + j * 1024 + lane * 16);
          #pragma unroll
          for (int h = 0; h < 2; ++h)
            if (hd[h] >> 31) {
              #pragma unroll
              for (int j = h * 4; j < h * 4 + 4; ++j) x[j] = __builtin_amdgcn_raw_buffer_load_b128(rsP, lane * 16 + j * 1024, par + EX_INR + uid * IU_RAW, AUX_SYS);
            }
        } else {
          LDS char* var = sl + QV;
          u32x4 c[5];
          #pragma unroll
          for (int j = 0; j < 5; ++j) c[j] = *(LDS u32x4*)(sl + j * 1024 + lane * 16);
          const u32 hdr = (u32)__builtin_amdgcn_readfirstlane((int)*(LDS u32*)var);
          const u32 pe = *(LDS unsigned short*)(var + 4 + (lane & 31) * 2);
          raw = hdr >> 31;
          if (!raw) {
            const u32 nbit = hdr & 0x7FFFu, npat = (hdr >> 15) & 0x1FFu, bias = (4u * ((hdr >> 24) & 31u) - 52u) * 0x01010101u;
            if (nbit > Q_ESC0 * 3 && lane < (int)(QV_END - QV0) / 16)
              *(LDS u32x4*)(var + QV0 + lane * 16) = __builtin_amdgcn_raw_buffer_load_b128(rsP, lane * 16, offE + QV + QV0, AUX_SYS);
            u32 a[32], qo[32], bits = 0;
            #pragma unroll
            for (int g = 0; g < 4; ++g) {
              u32 k5[5];
              #pragma unroll
              for (int k = 0; k < 5; ++k) { k5[k] = c[(g * 5 + k) >> 2][(g * 5 + k) & 3]; a[g * 8 + k] = k5[k]; }
              a[g * 8 + 5] = bfi(0x80808080u, k5[4] << 3, bfi(0x08080808u, k5[3] >> 1, k5[0] >> 4));
              a[g * 8 + 6] = bfi(0x80808080u, k5[4] << 2, bfi(0x08080808u, k5[3] >> 2, k5[1] >> 4));
              a[g * 8 + 7] = bfi(0x80808080u, k5[4] << 1, bfi(0x08080808u, k5[3] >> 3, k5[2] >> 4));
            }
            u32 es[32];
            #pragma unroll
            for (int d = 0; d < 32; ++d) {
              es[d] = a[d] & (a[d] >> 1) & 0x04040404u;
              qo[d] = bits;
              bits = __builtin_amdgcn_udot4(es[d], 0x03030303u, bits, false);
            }
            const u32 base4 = (((u32)(uintptr_t)var + QV_STR) * 8 + wave_scan_u32(bits >> 2)) * 4 - bits;
            wave_sync();
            #pragma unroll
            for (int d = 0; d < 32; ++d) {
              const u32 q4 = base4 + qo[d];
              const LDS u32* at = (const LDS u32*)(uintptr_t)((q4 >> 5) & ~3u);
              qo[d] = __builtin_amdgcn_alignbit(at[1], at[0], q4 >> 2);
            }
            wave_sync();
            ctl_st(rel, n + 1);
            #pragma unroll
            for (int j = 0; j < 8; ++j) {
              u32x4 y;
              #pragma unroll
              for (int k = 0; k < 4; ++k) {
                const u32 z = qo[j * 4 + k], z2 = bfi(0x0000003Fu, z, z << 10);
                const u32 sp = bfi(0x00070007u, z2, z2 << 5) & 0x07070707u;
                const u32 sel = __umul24(es[j * 4 + k], 0x404040u) + es[j * 4 + k];
                const u32 code = __builtin_amdgcn_perm(sp, 0x03030303u, sel);
                y[k] = (a[j * 4 + k] & 0x8F8F8F8Fu) + __builtin_amdgcn_perm(0x0C101418u, 0x28000408u, code) + bias;
              }
              *(LDS u32x4*)(scr + j * 1024 + lane * 16) = y;
            }
            wave_sync();
            if ((u32)lane < npat && lane < (int)Q_PAT0) patch_short(scr, pe);
            if (npat > Q_PAT0) {
              for (int j = 0; j < 2; ++j) {
                const u32 s2 = __builtin_amdgcn_raw_buffer_load_b32(rsP, lane * 4 + j * 256, offE + QV + QV_END, AUX_SYS);
                const u32 id = Q_PAT0 + j * 128 + 2 * lane;
                if (id < npat) patch_short(scr, s2 & 0xFFFFu);
                if (id + 1 < npat) patch_short(scr, s2 >> 16);
              }
            }
            wave_sync();
            #pragma unroll
            for (int j = 0; j < 8; ++j) x[j] = *(LDS u32x4*)(scr + j * 1024 + lane * 16);
          } else {
            wave_sync();
            ctl_st(rel, n + 1);
          }
        }
        if (raw) {
          #pragma unroll
          for (int j = 0; j < 8; ++j) x[j] = __builtin_amdgcn_raw_buffer_load_b128(rsP, lane * 16 + j * 1024, par + EX_INR + uid * IU_RAW, AUX_SYS);
        }
        store_raw_unit(x, rsL, dst, lane);
        vm_drain();
        cnt_add(p.ws, (isx ? rg : 64 + (rg >> 1)) * 5 + kb, lane);
        continue;
      }
      const int t = i - 512, cg = t / 5, rg = t - cg * 5 + (same ? 0 : 3);
      const u32 obase = (rg * 64 * NN + group_of(src, cg >> 4) * 1024 + (cg & 15) * 64) * 4;
      const u32 olane = (li * NN + lg * 4) * 4;
      if (!ok) {
        const u32x4 nan4 = {0x7FC00000u, 0x7FC00000u, 0x7FC00000u, 0x7FC00000u};
        #pragma unroll
        for (int e = 0; e < 16; ++e) ST128(nan4, rsO, olane + (e & 3) * 64, obase + (e >> 2) * 16 * NN * 4, AUX_SYS);
        continue;
      }
      #pragma unroll
      for (int j = 0; j < 4; ++j) *(LDS u32x4*)(scr + j * 1024 + lane * 16) = u32x4{0u, 0u, 0u, 0u};
      landed(endp - 11);
      const u32 odd = *(LDS unsigned short*)(sl + OU_SIDE + lane * 2) | ((u32)*(LDS unsigned char*)(sl + OU_SIDE + 128 + lane) << 16);
      const u32 hdr = (u32)__builtin_amdgcn_readfirstlane((int)*(LDS u32*)(sl + OU_SIDE + 192));
      u32 se = *(LDS unsigned short*)(sl + OU_SIDE + 196 + (lane < 24 ? lane : 0) * 2);
      u32x2 fe = {*(LDS unsigned short*)(sl + OU_SIDE + 244 + (lane & 1) * 2), *(LDS u32*)(sl + OU_SIDE + 248 + (lane & 1) * 4)};
      const u32 tl[3] = {*(LDS u32*)(sl + 11264 + lane * 4), *(LDS u32*)(sl + 11520 + lane * 4), *(LDS u32*)(sl + 11776 + lane * 4)};
      const u32 uid = rank * 320 + t;
      if (!(hdr >> 31)) {
        const u32 ns = hdr & 0xFFu, nf = (hdr >> 8) & 0xFFu; const int g = (int)((hdr >> 16) & 0x1FFu) - 256;
        const u32 tail = par + EX_OUE + uid * OU_ENC + 12288;
        if (ns > OS_CAP0) {
          const u32 s2 = __builtin_amdgcn_raw_buffer_load_b16(rsP, lane >= (int)OS_CAP0 ? lane * 2 : OS_CAP0 * 2, tail - OS_CAP0 * 2, AUX_SYS);
          se = lane >= (int)OS_CAP0 ? s2 : se;
          asm volatile("" : "+v"(se));
        }
        if (nf > OF_CAP0) {
          const u32x2 f2 = __builtin_amdgcn_raw_buffer_load_b64(rsP, max((u32)OF_CAP0, min((u32)lane, nf - 1)) * 8, tail + 128 - OF_CAP0 * 8, AUX_SYS);
          fe = lane >= (int)OF_CAP0 ? f2 : fe;
          asm volatile("" : "+v"(fe));
        }
        if ((u32)lane < ns)
          __hip_atomic_fetch_add((LDS u32*)(scr + ((se >> 13) & 3u) * 1024 + ((se >> 5) & 63u) * 16 + ((se >> 11) & 3u) * 4),
                                 se >> 15 ? ((se & 3u) | ((se & 12u) << 6)) << (se & 16u) : ((se & 7u) << 5) << (se & 24u),
                                 __ATOMIC_RELAXED, __HIP_MEMORY_SCOPE_WORKGROUP);
        const u32 fgq = (u32)lane < nf ? fe.x >> 8 : 16u;
        u64 om = (1ull << (odd & 63)) | (1ull << ((odd >> 6) & 63)) | (1ull << ((odd >> 12) & 63)) | (1ull << ((odd >> 18) & 63));
        if ((odd & 63) > ((odd >> 6) & 63)) om = 0;
        const u32 ob[2] = {(u32)om, (u32)(om >> 32)};
        u32 t15 = tl[0] << 1;
        wave_sync();
        #pragma unroll
        for (int mi = 0; mi < 4; ++mi) {
          landed(mi < 3 ? endp - 8 + 3 * mi : endp);
          u32x4 c[3];
          #pragma unroll
          for (int j = 0; j < (mi < 3 ? 3 : 2); ++j) c[j] = *(LDS u32x4*)(sl + (mi * 3 + j) * 1024 + lane * 16);
          const u32x4 adj = *(LDS u32x4*)(scr + mi * 1024 + lane * 16);
          if (mi == 3) { wave_sync(); ctl_st(C_REL + (n % NSLOT) * 4, n + 1); c[2] = u32x4{tl[0], tl[1], tl[2], 0u}; }
          #pragma unroll
          for (int ni = 0; ni < 4; ++ni) {
            const int k = ni * 3, gq = mi * 4 + ni;
            const u32 l0 = c[k >> 2][k & 3], l1 = c[(k + 1) >> 2][(k + 1) & 3], tp = gq < 15 ? c[(k + 2) >> 2][(k + 2) & 3] : t15;
            if (gq < 14) t15 = bfi(0x00010001u << gq, tp >> (14 - gq), t15);
            const u32 top = __builtin_amdgcn_bitop3_b32(top_unpack(tp), adj[ni] << 5, 0x60606060u, 0x78);
            const u32 low = ((ob[gq >> 3] << (gq & 7)) & 0x80808080u) + (adj[ni] & 0xE0E0E0E0u);
            const u32 q01 = __builtin_amdgcn_perm(top, low, 0x05010400u), q23 = __builtin_amdgcn_perm(top, low, 0x07030602u);
            float y[4] = {unfix((int)__builtin_amdgcn_perm(q01, l0, 0x05010004u), g + 7), unfix((int)__builtin_amdgcn_perm(q01, l0, 0x07030206u), g + 7),
                          unfix((int)__builtin_amdgcn_perm(q23, l1, 0x05010004u), g + 7), unfix((int)__builtin_amdgcn_perm(q23, l1, 0x07030206u), g + 7)};
            if (nf)
              for (u64 em = __ballot(fgq == (u32)gq); em; em &= em - 1) {
                const int from = __builtin_ctzll(em);
                const u32 rel = ((u32)__builtin_amdgcn_readlane((int)fe.x, from) & 255u) - (u32)lane * 4u;
                const float vv = as_f((u32)__builtin_amdgcn_readlane((int)fe.y, from));
                #pragma unroll
                for (int r = 0; r < 4; ++r) y[r] = rel == (u32)r ? vv : y[r];
              }
            ST128((u32x4{as_u(y[0] * scale), as_u(y[1] * scale), as_u(y[2] * scale), as_u(y[3] * scale)}), rsO, olane + ni * 64, obase + mi * 16 * NN * 4, AUX_SYS);
          }
        }
      } else {
        landed(endp);
        ctl_st(C_REL + (n % NSLOT) * 4, n + 1);
        #pragma unroll
        for (int mi = 0; mi < 4; ++mi) {
          u32x4 r4[4];
          #pragma unroll
          for (int ni = 0; ni < 4; ++ni) r4[ni] = __builtin_amdgcn_raw_buffer_load_b128(rsP, lane * 16 + ni * 1024, par + EX_OUR + uid * OU_RAW + mi * 4096, AUX_SYS);
          #pragma unroll
          for (int ni = 0; ni < 4; ++ni) {
            const f32x4 y = __builtin_bit_cast(f32x4, r4[ni]) * scale;
            ST128(__builtin_bit_cast(u32x4, y), rsO, olane + ni * 64, obase + mi * 16 * NN * 4, AUX_SYS);
          }
        }
      }
    }
    return;
  }

  constexpr int SLOTS = NGEMM / 8;
  const int xc = b & 7, slot = b >> 3, wm = wave >> 1, wn = wave & 1;
  for (int ti = slot; ti < 36; ti += SLOTS) {
    const int j = xc >> 1, ft = ti * 2 + (xc & 1);
    const int c = ti < 24 ? ft / 6 : 4 * (xc & 1) + (ti < 28 ? ti - 24 : (ti - 28) >> 1), cb = j * 8 + c;
    auto row_group = [&](int k) -> int {
      if (ti >= 28) return 8 * rank + 4 * (ti & 1) + k;
      if (ti >= 24) return 8 * ((~rank & 4) | ((j + 1 + min(k, 2)) & 3)) + 3;
      const int t = ti % 3;
      return 8 * (xc & 1 ? (~rank & 4) | ((j + 1 + t) & 3) : (rank & 4) | ((rank + 1 + t) & 3)) + 4 * (xc & 1) + k;
    };
    const int rgw = row_group(wm);
    const u32 deadw = (u32)(uintptr_t)(smem + L_CTL);
    auto ready = [&](int grp) {
      const bool isx = lane < 4;
      const int rgl = row_group(lane & 3);
      const u32 off = OFF_CNT + ((isx ? rgl : 64 + cb) * 5 + grp) * 128, want = (grp == 4 ? (isx ? 6u : 8u) : (isx ? 7u : 14u)) * call;
      if (!wait_ge(p.ws, off, want, call)) ctl_st(deadw, 1u);
    };
    const u32 xo0 = OFF_XA + rgw * IU_RAW + wn * 4096, wo0 = OFF_WA + cb * 2 * IU_RAW + wave * 2048;
    auto dma = [&](int k, int ks, u32 st) {
      const u32 xo = xo0 + ks * (64 * IU_RAW), wo = wo0 + ks * (64 * IU_RAW);
      LDS char* dx = smem + st + wave * 4096;
      LDS char* dw = smem + st + 32 * KiB + wave * 2048;
      if (k == 0) __builtin_amdgcn_raw_ptr_buffer_load_lds(rsL, (LDS void*)dx, 16, lane * 16, xo, 0, AUX_SYS);
      if (k == 1) __builtin_amdgcn_raw_ptr_buffer_load_lds(rsL, (LDS void*)dx, 16, lane * 16, xo, 1024, AUX_SYS);
      if (k == 2) __builtin_amdgcn_raw_ptr_buffer_load_lds(rsL, (LDS void*)dx, 16, lane * 16, xo, 2048, AUX_SYS);
      if (k == 3) __builtin_amdgcn_raw_ptr_buffer_load_lds(rsL, (LDS void*)dx, 16, lane * 16, xo, 3072, AUX_SYS);
      if (k == 4) __builtin_amdgcn_raw_ptr_buffer_load_lds(rsL, (LDS void*)dw, 16, lane * 16, wo, 0, AUX_SYS);
      if (k == 5) __builtin_amdgcn_raw_ptr_buffer_load_lds(rsL, (LDS void*)dw, 16, lane * 16, wo, 1024, AUX_SYS);
    };
    f32x4 acc[4][4];
    #pragma unroll
    for (int mi = 0; mi < 4; ++mi)
      #pragma unroll
      for (int ni = 0; ni < 4; ++ni) acc[mi][ni] = f32x4{0.f, 0.f, 0.f, 0.f};
    const bool twin = ti >= 24 && ti < 28;
    const int kr = twin ? (~rank & 4) | j : rank;
    if (wave == 0) {
      if (ti == slot) ctl_st(deadw, 0u);
      for (int grp = twin ? 0 : 4; grp < 5; ++grp) ready(grp);
    }
    asm volatile("s_waitcnt vmcnt(0) lgkmcnt(0)\n\ts_barrier" ::: "memory");
    #pragma unroll
    for (int k = 0; k < 12; ++k) dma(k % 6, kr * 4 + k / 6, k / 6 * STAGE);
    u32 st = 0;
    int ksn = kr * 4 + 2, jn = -2, kbn = 0;
    i32x8 fa[8], fb[8];
    auto step = [&](i32x8 (&cur)[8], const i32x8 (&prev)[8], bool first, bool more, bool last) {
      if (more && wave == 0 && jn == 0) ready(kbn);
      if (!last) asm volatile("s_waitcnt vmcnt(6) lgkmcnt(0)\n\ts_barrier" ::: "memory");
      else asm volatile("s_waitcnt vmcnt(0) lgkmcnt(0)\n\ts_barrier" ::: "memory");
      const u32 st2 = st == 0 ? 2 * STAGE : st - STAGE;
      LDS char* ax = smem + st + wm * 8192 + lane * 16;
      LDS char* aw = smem + st + 32 * KiB + wn * 8192 + lane * 16;
      u32x4 h[16];
      auto M = [&](int k) {
        if (!first) acc[k >> 2][k & 3] = __builtin_amdgcn_mfma_scale_f32_16x16x128_f8f6f4(prev[k & 3], prev[4 + (k >> 2)], acc[k >> 2][k & 3], 1, 1, 0, 0, 0, 0);
        __builtin_amdgcn_sched_barrier(0);
      };
      auto R = [&](int k) { h[k] = *(LDS u32x4*)(((k >> 1) & 1 ? ax : aw) + (k >> 2) * 2048 + (k & 1) * 1024); __builtin_amdgcn_sched_barrier(0); };
      __builtin_amdgcn_sched_barrier(0);
      #pragma unroll
      for (int k = 0; k < 6; ++k) {
        M(2 * k); R(2 * k); R(2 * k + 1); M(2 * k + 1);
        if (more) dma(k, ksn, st2);
        __builtin_amdgcn_sched_barrier(0);
      }
      M(12); R(12); R(13); M(13); R(14); R(15); M(14); M(15);
      #pragma unroll
      for (int f = 0; f < 4; ++f) {
        cur[f] = __builtin_bit_cast(i32x8, __builtin_shufflevector(h[4 * f], h[4 * f + 1], 0, 1, 2, 3, 4, 5, 6, 7));
        cur[4 + f] = __builtin_bit_cast(i32x8, __builtin_shufflevector(h[4 * f + 2], h[4 * f + 3], 0, 1, 2, 3, 4, 5, 6, 7));
      }
      if (++jn == 7) { jn = 0; ++kbn; }
      ksn = jn < 0 ? ksn + 1 : ((kr + 1 + jn) & 7) * 4 + kbn;
      st = st == 2 * STAGE ? 0 : st + STAGE;
    };
    step(fa, fb, true, true, false);
    #pragma unroll 1
    for (int s = 1; s < 29; s += 2) { step(fb, fa, false, true, false); step(fa, fb, false, true, false); }
    step(fb, fa, false, true, false);
    step(fa, fb, false, false, false);
    step(fb, fa, false, false, true);
    #pragma unroll
    for (int k = 0; k < 16; ++k)
      acc[k >> 2][k & 3] = __builtin_amdgcn_mfma_scale_f32_16x16x128_f8f6f4(fb[k & 3], fb[4 + (k >> 2)], acc[k >> 2][k & 3], 1, 1, 0, 0, 0, 0);
    int lz = lane; asm volatile("" : "+v"(lz));
    const bool dead = ctl_ld(deadw) != 0u;
    const int owner = rgw >> 3;
    if (owner == rank) {
      const u32 obase = ((rgw & 7) * 64 * NN + group_of(rank, j) * 1024 + c * 128 + wn * 64) * 4, olane = ((lz & 15) * NN + (lz >> 4) * 4) * 4;
      const float sc = dead ? __builtin_nanf("") : scale;
      #pragma unroll
      for (int mi = 0; mi < 4; ++mi)
        #pragma unroll
        for (int ni = 0; ni < 4; ++ni)
          ST128(__builtin_bit_cast(u32x4, acc[mi][ni] * sc), rsO, olane + ni * 64, obase + mi * 16 * NN * 4, AUX_SYS);
    } else if (!dead && !(ti >= 24 && wm == 3)) {
      const int t = (cb * 2 + wn) * 5 + (rgw & 7) - ((owner >> 2) == (rank >> 2) ? 0 : 3);
      const u32 uid = owner * 320 + t;
      const u32 offE = par + EX_OUE + uid * OU_ENC;
      float sa = 0.f;
      #pragma unroll
      for (int mi = 0; mi < 4; ++mi)
        #pragma unroll
        for (int ni = 0; ni < 4; ++ni) sa += __builtin_fabsf(acc[mi][ni][(mi + ni) & 3]);
      const int g = max(0, 24 - ((int)((as_u(wave_sum_f32(sa) * (4.2f / 1024.f)) >> 23) & 255u) - 126));
      LDS char* last1k = smem + wave * 4096;
      LDS char* side = last1k + 768;
      LDS char* pend = last1k + 1536;
      u32 npend = 0, sg[2] = {0u, 0u};
      auto quad = [&](int gq, u32& l0, u32& l1, u32& tp) {
        u32 qv[4], fr[4];
        #pragma unroll
        for (int r = 0; r < 4; ++r) {
          const float tt = __builtin_amdgcn_ldexpf(acc[gq >> 2][gq & 3][r], g + 7);
          qv[r] = (u32)f2i_sat(tt);
          fr[r] = as_u(__builtin_amdgcn_fractf(tt));
        }
        const u32 p01 = __builtin_amdgcn_perm(qv[1], qv[0], 0x07030400u), p23 = __builtin_amdgcn_perm(qv[3], qv[2], 0x07030400u);
        const u32 low = __builtin_amdgcn_perm(p23, p01, 0x05040100u);
        u32 miss;
        tp = top_pack(__builtin_amdgcn_perm(p23, p01, 0x07060302u), miss);
        l0 = __builtin_amdgcn_perm(qv[1], qv[0], 0x06050201u);
        l1 = __builtin_amdgcn_perm(qv[3], qv[2], 0x06050201u);
        sg[gq >> 3] = bfi(0x80808080u >> (gq & 7), low >> (gq & 7), sg[gq >> 3]);
        const bool odd_one = (fr[0] | fr[1] | fr[2] | fr[3] | (low & 0x7F7F7F7Fu) | (miss & 0x00380038u)) != 0u;
        const u64 bm = __ballot(odd_one);
        if (odd_one) {
          const u32 sl = min(63u, npend + __builtin_amdgcn_mbcnt_hi((u32)(bm >> 32), __builtin_amdgcn_mbcnt_lo((u32)bm, 0u)));
          *(LDS f32x4*)(pend + sl * 16) = acc[gq >> 2][gq & 3];
          *(LDS u32*)(pend + 1024 + sl * 4) = gq * 64 + lz;
        }
        npend += __popcll(bm);
      };
      u32 z0, z1, t15;
      quad(15, z0, z1, t15);
      *(LDS u32*)(last1k + 256 + lz * 4) = z0;
      *(LDS u32*)(last1k + 512 + lz * 4) = z1;
      #pragma unroll
      for (int mi = 0; mi < 4; ++mi) {
        u32 pk[12];
        #pragma unroll
        for (int ni = 0; ni < 4; ++ni) {
          const int gq = mi * 4 + ni;
          if (gq == 15) break;
          quad(gq, pk[ni * 3], pk[ni * 3 + 1], pk[ni * 3 + 2]);
          pk[ni * 3 + 2] = bfi(0x40004000u, gq < 14 ? t15 << (14 - gq) : t15 >> 1, pk[ni * 3 + 2]);
        }
        #pragma unroll
        for (int j = 0; j < (mi < 3 ? 3 : 2); ++j)
          ST128((u32x4{pk[j * 4], pk[j * 4 + 1], pk[j * 4 + 2], pk[j * 4 + 3]}), rsL, lz * 16 + j * 1024, offE + mi * 3072, AUX_SYS);
        if (mi == 3) *(LDS u32*)(last1k + lz * 4) = pk[8];
      }
      u32 ns = npend > 64u ? OS_CAP + 1 : 0u, nf = 0;
      auto put = [&](bool on, u32 e) {
        const u64 bm = __ballot(on);
        const u32 sl = ns + __builtin_amdgcn_mbcnt_hi((u32)(bm >> 32), __builtin_amdgcn_mbcnt_lo((u32)bm, 0u));
        if (on && sl < OS_CAP) *(LDS unsigned short*)(side + (sl < OS_CAP0 ? 196 : 256 - OS_CAP0 * 2) + sl * 2) = (unsigned short)e;
        ns += __popcll(bm);
      };
      u64 om = ((u64)sg[1] << 32) | sg[0];
      u32 odd = om ? 0u : 1u, last = 0;
      #pragma unroll
      for (int k = 0; k < 4; ++k) { if (om) last = (u32)__builtin_ctzll(om); om &= om - 1; odd |= last << (6 * k); }
      for (; __any(om != 0) && ns <= OS_CAP; om &= om - 1) {
        const u32 e = (u32)__builtin_ctzll(om | (1ull << 63));
        put(om != 0, (((e >> 5) * 8 + 7 - (e & 7)) << 11) | (lz << 5) | (e & 24u) | 4u);
      }
      wave_sync();
      for (u32 p0 = 0; p0 < min(npend, 64u); p0 += 16) {
        const u32 pi = p0 + (lz >> 2), r = lz & 3;
        const bool on = pi < npend;
        const float v = *(LDS float*)(pend + p0 * 16 + lz * 4);
        const u32 pp = *(LDS u32*)(pend + 1024 + pi * 4);
        const float tt = __builtin_amdgcn_ldexpf(v, g + 7);
        const u32 iv = (u32)f2i_sat(tt);
        const bool full = on && (as_u((float)(int)iv) != as_u(tt) || (iv & 31u));
        const u32 top = __builtin_amdgcn_perm((u32)__builtin_amdgcn_mov_dpp((int)iv, 0xB1, 0xf, 0xf, true), iv, 0x0C0C0703u);
        u32 miss;
        const u32 fl = (top_unpack(top_pack(top, miss)) ^ top) >> 5, d4 = (fl & 3u) | ((fl >> 6) & 12u);
        put(on && !full && (iv & 96u), (pp << 5) | (r << 3) | ((iv >> 5) & 3u));
        put(on && !(r & 1) && d4, 0x8000u | (pp << 5) | (r << 3) | d4);
        const u64 bm = __ballot(full);
        if (bm) {
          const u32 sl = nf + __builtin_amdgcn_mbcnt_hi((u32)(bm >> 32), __builtin_amdgcn_mbcnt_lo((u32)bm, 0u));
          if (full && sl < OF_CAP0) { *(LDS unsigned short*)(side + 244 + sl * 2) = (unsigned short)(pp * 4 + r); *(LDS u32*)(side + 248 + sl * 4) = as_u(v); }
          else if (full && sl < OF_CAP) *(LDS u32x2*)(side + 384 - OF_CAP0 * 8 + sl * 8) = u32x2{pp * 4 + r, as_u(v)};
          nf += __popcll(bm);
        }
      }
      const bool raw = ns > OS_CAP || nf > OF_CAP;
      *(LDS unsigned short*)(side + lz * 2) = (unsigned short)odd;
      *(LDS unsigned char*)(side + 128 + lz) = (unsigned char)(odd >> 16);
      if (lz == 0) *(LDS u32*)(side + 192) = raw ? 0x80000000u : (ns | (nf << 8) | ((u32)(g + 256) << 16));
      wave_sync();
      ST128(*(LDS u32x4*)(last1k + lz * 16), rsL, lz * 16, offE + 11264, AUX_SYS);
      if ((ns > OS_CAP0 || nf > OF_CAP0) && lz < 24) ST128(*(LDS u32x4*)(last1k + 1024 + lz * 16), rsL, lz * 16, offE + 12288, AUX_SYS);
      if (raw) {
        const u32 offR = par + EX_OUR + uid * OU_RAW;
        #pragma unroll
        for (int mi = 0; mi < 4; ++mi)
          #pragma unroll
          for (int ni = 0; ni < 4; ++ni)
            ST128(__builtin_bit_cast(u32x4, acc[mi][ni]), rsL, lz * 16 + ni * 1024, offR + mi * 4096, AUX_SYS);
      }
      vm_drain();
      if (lz == owner)
        __hip_atomic_store((GLB u32*)(peer_ws + OFF_FOUT + (rank * 320 + t) * 128), call, __ATOMIC_RELAXED, __HIP_MEMORY_SCOPE_SYSTEM);
    }
  }
}

}

extern "C" void kernel_launch(int rank, int nranks, ncclComm_t comm, hipStream_t stream,
                              void* const* d_in, const int* in_sizes, int n_in,
                              void* d_out, int out_size,
                              const int* in_dtypes, int out_dtype,
                              void* const* d_out_all,
                              void* const* d_ws_all, size_t ws_bytes) {
  (void)comm; (void)in_sizes; (void)n_in; (void)out_size; (void)in_dtypes; (void)out_dtype; (void)d_out_all;
  if (nranks != NR || (unsigned)rank >= (unsigned)NR || ws_bytes < WS_NEED) { fprintf(stderr, "unexpected launch shape\n"); return; }
  static unsigned attr_set = 0;
  if (!(attr_set >> rank & 1u)) {
    HIP_CHECK(hipFuncSetAttribute((const void*)gemm_rs_kernel, hipFuncAttributeMaxDynamicSharedMemorySize, (int)LDS_BYTES));
    attr_set |= 1u << rank;
  }
  Params p;
  p.x = (const uint8_t*)d_in[0]; p.w = (const uint8_t*)d_in[1]; p.sx = (const float*)d_in[2]; p.sw = (const float*)d_in[3];
  p.out = (float*)d_out; p.ws_all = (char* const*)d_ws_all; p.ws = (char*)d_ws_all[rank]; p.rank = rank; p.ws_bytes = WS_NEED;
  hipLaunchKernelGGL(gemm_rs_kernel, dim3(NWG), dim3(NTHR), LDS_BYTES, stream, p);
  HIP_CHECK(hipGetLastError());
}
```

```cpp
#include <hip/hip_runtime.h>
#include <rccl/rccl.h>
#include <cstdio>
#include <cstdint>

#define HIP_CHECK(expr) do { hipError_t _e = (expr); if (_e != hipSuccess) { \
    fprintf(stderr, "HIP error %s at %s:%d\n", hipGetErrorString(_e), __FILE__, __LINE__); \
    return; } } while (0)

namespace {

typedef unsigned u32;
typedef unsigned long long u64;
typedef unsigned u32x4 __attribute__((ext_vector_type(4)));
typedef unsigned u32x2 __attribute__((ext_vector_type(2)));
typedef int i32x2 __attribute__((ext_vector_type(2)));
typedef int i32x8 __attribute__((ext_vector_type(8)));
typedef float f32x4 __attribute__((ext_vector_type(4)));
typedef __amdgpu_buffer_rsrc_t rsrc_t;
#define LDS __attribute__((address_space(3)))
#define GLB __attribute__((address_space(1)))

constexpr int NR = 8, MM = 4096, NN = 16384, KS = 512;
constexpr int NWG = 256, NTHR = 512;
constexpr int PULL_WGS = 8;
constexpr int WIN = 8, WIN_OUT = 7;
constexpr u32 LOOK = 5, LOOK_OUT = 3;
constexpr int NRAW = 8, X1FIRST = 16;
constexpr int NPULL = 7 * PULL_WGS, NGEMM = NWG - NPULL;
constexpr int AUX_SYS = 17;
constexpr u64 DEADLINE_TICKS = 40000000ull;

constexpr u32 KiB = 1024, MiB = 1024 * 1024;
constexpr u32 OFF_CALL = 0;
constexpr u32 OFF_TMO = 32 * KiB;
constexpr u32 OFF_TIC = 33 * KiB;
constexpr u32 OFF_CNT = 40 * KiB;
constexpr u32 OFF_FIN = 128 * KiB;
constexpr u32 OFF_FOUT = OFF_FIN + 8 * 576 * 128;
constexpr u32 OFF_XA = 2 * MiB;
constexpr u32 OFF_WA = OFF_XA + 16 * MiB;
constexpr u32 OFF_PAR = OFF_WA + 16 * MiB;
constexpr u32 IU_ENC = 8 * KiB, IU_RAW = 8 * KiB;
constexpr u32 OU_ENC = 12672, OU_RAW = 16 * KiB;
constexpr u32 IU_SIDE = 6144, OU_SIDE = 12032;
constexpr u32 EX_INE = 0;
constexpr u32 EX_INR = EX_INE + 1280 * IU_ENC;
constexpr u32 EX_OUE = EX_INR + 1280 * IU_RAW;
constexpr u32 EX_OUR = EX_OUE + 2560 * OU_ENC;
constexpr u32 PAR_BYTES = EX_OUR + 2560 * OU_RAW;
constexpr u32 WS_NEED = OFF_PAR + 2 * PAR_BYTES;
static_assert(OFF_CNT + 96 * 5 * 128 <= OFF_FIN && OFF_FOUT + 8 * 320 * 128 <= OFF_XA, "flags");
static_assert(WS_NEED <= 257u * MiB, "workspace");
constexpr u32 QV = 5120, QV0 = 768, QV_SEC = 24, QV_END = 1792, QV_ALL = 2304;
constexpr u32 Q_ESC = 6144, Q_PAT0 = 10, Q_PAT = 266;
constexpr u32 IN_CAP0 = 62, IN_CAP = 254;
constexpr u32 OS_CAP0 = 24, OS_CAP = 64, OF_CAP0 = 2, OF_CAP = 32;

constexpr u32 STAGE = 48 * KiB;
constexpr u32 L_SCR = 64 * KiB, SCR_W = 10752;
constexpr u32 NSLOT = 8, SLOT = 13 * KiB, NCONS = 6;
constexpr u32 L_PSCR = NSLOT * SLOT;
constexpr u32 L_CTL = L_PSCR + NCONS * 8 * KiB;
constexpr u32 LDS_BYTES = L_CTL + 256;
static_assert(3 * STAGE <= L_CTL && L_SCR + 8 * SCR_W <= LDS_BYTES && LDS_BYTES <= 160 * KiB, "lds");

__device__ __forceinline__ bool quick_form(int i) { return i < 256 ? (i & 3) < 2 : (i & 0x3B) == 0; }
__device__ __forceinline__ int group_of(int rank, int j) { return rank < 4 ? rank * 4 + j : j * 4 + (rank & 3); }

struct Params {
  const uint8_t* x; const uint8_t* w; const float* sx; const float* sw;
  float* out; char* const* ws_all; char* ws; int rank; u32 ws_bytes;
};

__device__ __forceinline__ rsrc_t mk_rsrc(const void* p, u32 bytes) {
  u64 a = (u64)p;
  u32 lo = __builtin_amdgcn_readfirstlane((u32)a), hi = __builtin_amdgcn_readfirstlane((u32)(a >> 32));
  return __builtin_amdgcn_make_buffer_rsrc((void*)(((u64)hi << 32) | lo), 0, (int)bytes, 0x00020000);
}
#define ST128(v, rs, voff, soff, aux) __builtin_amdgcn_raw_buffer_store_b128((v), (rs), (int)((u32)(voff) + (u32)(soff)), 0, (aux))
__device__ __forceinline__ float as_f(u32 v) { return __builtin_bit_cast(float, v); }
__device__ __forceinline__ u32 as_u(float v) { return __builtin_bit_cast(u32, v); }
__device__ __forceinline__ int f2i_sat(float v) {
  int r; asm("v_cvt_i32_f32 %0, %1" : "=v"(r) : "v"(v)); return r;
}
__device__ __forceinline__ void vm_drain() { asm volatile("s_waitcnt vmcnt(0)" ::: "memory"); }
__device__ __forceinline__ void wave_sync() {
  asm volatile("s_waitcnt lgkmcnt(0)" ::: "memory");
}
__device__ __forceinline__ u32 bfi(u32 m, u32 a, u32 b) { return __builtin_amdgcn_bitop3_b32(m, a, b, 0xCA); }
template <int CTRL, int ROWS> __device__ __forceinline__ u32 dpp0(u32 v) { return (u32)__builtin_amdgcn_update_dpp(0, (int)v, CTRL, ROWS, 0xf, false); }
__device__ __forceinline__ u32 wave_scan_u32(u32 v) {
  v += dpp0<0x111, 0xf>(v); v += dpp0<0x112, 0xf>(v); v += dpp0<0x114, 0xf>(v); v += dpp0<0x118, 0xf>(v);
  v += dpp0<0x142, 0xa>(v); v += dpp0<0x143, 0xc>(v);
  return v;
}
__device__ __forceinline__ float wave_sum_f32(float v) {
  v += as_f(dpp0<0x111, 0xf>(as_u(v))); v += as_f(dpp0<0x112, 0xf>(as_u(v))); v += as_f(dpp0<0x114, 0xf>(as_u(v)));
  v += as_f(dpp0<0x118, 0xf>(as_u(v))); v += as_f(dpp0<0x142, 0xa>(as_u(v))); v += as_f(dpp0<0x143, 0xc>(as_u(v)));
  return as_f((u32)__builtin_amdgcn_readlane((int)as_u(v), 63));
}
__device__ __forceinline__ u32 ctl_ld(u32 addr) {
  u32 v; asm volatile("ds_read_b32 %0, %1\n\ts_waitcnt lgkmcnt(0)" : "=&v"(v) : "v"(addr) : "memory");
  return (u32)__builtin_amdgcn_readfirstlane((int)v);
}
__device__ __forceinline__ void ctl_st(u32 addr, u32 v) { asm volatile("ds_write_b32 %0, %1" :: "v"(addr), "v"(v) : "memory"); }
__device__ __forceinline__ u32 ld_flag(const char* ws, u32 off) {
  return __hip_atomic_load((const GLB u32*)(ws + off), __ATOMIC_RELAXED, __HIP_MEMORY_SCOPE_SYSTEM);
}
__device__ __forceinline__ bool wait_ge(char* ws, u32 off, u32 target, u32 call) {
  if (__all(ld_flag(ws, off) >= target)) return true;
  const u64 t0 = __builtin_amdgcn_s_memrealtime();
  GLB u32* tmo = (GLB u32*)(ws + OFF_TMO);
  for (u32 it = 1;; ++it) {
    __builtin_amdgcn_s_sleep(2);
    if (__all(ld_flag(ws, off) >= target)) return true;
    if ((it & 15u) == 0u &&
        (__hip_atomic_load(tmo, __ATOMIC_RELAXED, __HIP_MEMORY_SCOPE_AGENT) == call ||
         __builtin_amdgcn_s_memrealtime() - t0 > DEADLINE_TICKS)) {
      __hip_atomic_store(tmo, call, __ATOMIC_RELAXED, __HIP_MEMORY_SCOPE_AGENT);
      return false;
    }
  }
}
__device__ __forceinline__ void cnt_add(char* ws, u32 idx, int lane) {
  if (lane == 0) __hip_atomic_fetch_add((GLB u32*)(ws + OFF_CNT + idx * 128), 1u, __ATOMIC_RELAXED, __HIP_MEMORY_SCOPE_AGENT);
}

__device__ __forceinline__ void store_raw_unit(const u32x4 (&x)[8], rsrc_t rs, u32 off, int lane) {
  #pragma unroll
  for (int j = 0; j < 8; ++j) ST128(x[j], rs, lane * 16 + j * 1024, off, AUX_SYS);
}

__device__ __forceinline__ void export_half(const u32x4 (&x)[4], rsrc_t rs, u32 par, u32 uid, int hf, LDS char* scr, int lane) {
  const u32 offE = par + EX_INE + uid * IU_ENC;
  u32 s = 0;
  #pragma unroll
  for (int j = 0; j < 4; ++j) {
    *(LDS u32x4*)(scr + j * 1024 + lane * 16) = x[j];
    s = __builtin_amdgcn_sad_u8(x[j][j] & 0x7C7C7C7Cu, 0u, s);
  }
  s = (u32)__builtin_amdgcn_readlane((int)wave_scan_u32(s), 63);
  int e0 = ((int)s - 17121) >> 12;
  e0 = e0 < 0 ? 0 : (e0 > 24 ? 24 : e0);
  const u32 e4 = (u32)e0 * 0x04040404u;
  u32 mk[2] = {0, 0};
  u32 pk[12];
  #pragma unroll
  for (int j = 0; j < 4; ++j) {
    u32 o[4];
    #pragma unroll
    for (int c = 0; c < 4; ++c) {
      const u32 u = (x[j][c] | 0x80808080u) - e4;
      o[c] = bfi(0x1F1F1F1Fu, u, x[j][c]);
      const u32 t = (((u >> 1) ^ 0x40404040u) & 0x70707070u) + 0x70707070u;
      const int d = j * 4 + c;
      mk[d >> 3] = bfi(0x80808080u >> (d & 7), t >> (d & 7), mk[d >> 3]);
    }
    pk[j * 3 + 0] = bfi(0x60606060u, o[3] << 5, o[0]);
    pk[j * 3 + 1] = bfi(0x60606060u, o[3] << 3, o[1]);
    pk[j * 3 + 2] = bfi(0x60606060u, bfi(0x40404040u, o[3] >> 1, o[3] << 1), o[2]);
  }
  #pragma unroll
  for (int j = 0; j < 3; ++j)
    ST128((u32x4{pk[j * 4], pk[j * 4 + 1], pk[j * 4 + 2], pk[j * 4 + 3]}), rs, lane * 16 + j * 1024, offE + hf * 3072, AUX_SYS);
  const u32 mine = __popc(mk[0]) + __popc(mk[1]);
  const u32 inc = wave_scan_u32(mine);
  const u32 total = (u32)__builtin_amdgcn_readlane((int)inc, 63);
  if (total <= IN_CAP) {
    LDS unsigned short* list = (LDS unsigned short*)(scr + 8192);
    u32 slot = 2 + inc - mine;
    #pragma unroll
    for (int q = 0; q < 2; ++q) {
      u32 m = mk[q];
      while (m) {
        const int bp = __builtin_ctz(m); m &= m - 1;
        const int d = q * 8 + 7 - (bp & 7);
        list[slot++] = (unsigned short)((d >> 2) * 1024 + lane * 16 + (d & 3) * 4 + (bp >> 3));
      }
    }
    wave_sync();
    for (u32 j = 0; j < 2; ++j) {
      const u32 pp = *(LDS u32*)(scr + 8192 + j * 256 + lane * 4);
      const u32 p0 = pp & 4095u, p1 = (pp >> 16) & 4095u;
      const u32 b0 = *(LDS unsigned char*)(scr + p0), b1 = *(LDS unsigned char*)(scr + p1);
      u32 sd = (((p0 << 3) | ((128u + (b0 & 127u) - 4u * (u32)e0) >> 5) | (p1 << 19) | (((128u + (b1 & 127u) - 4u * (u32)e0) >> 5) << 16))) + hf * 0x80008000u;
      if (j == 0 && lane == 0) sd = total | ((u32)e0 << 16);
      const u32 at = j * 64 + lane;
      if (at < 32 || total > IN_CAP0)
        __builtin_amdgcn_raw_buffer_store_b32(sd, rs, at < 32 ? hf * 128 + at * 4 : 128 + hf * 384 + at * 4, offE + IU_SIDE, AUX_SYS);
      if (total <= 126) break;
    }
  } else {
    #pragma unroll
    for (int j = 0; j < 4; ++j) ST128(x[j], rs, lane * 16 + j * 1024, par + EX_INR + uid * IU_RAW + hf * 4096, AUX_SYS);
    if (lane == 0) __builtin_amdgcn_raw_buffer_store_b32(0x80000000u, rs, hf * 128, offE + IU_SIDE, AUX_SYS);
  }
}

__device__ __forceinline__ void export_short(const u32x4 (&x)[8], rsrc_t rs, u32 par, u32 uid, LDS char* scr, int lane) {
  typedef unsigned short u16x2 __attribute__((ext_vector_type(2)));
  const u32 offE = par + EX_INE + uid * IU_ENC, offR = par + EX_INR + uid * IU_RAW;
  LDS char* var = scr + 8192;
  LDS char* tab = var + QV_ALL;
  u32 s = 0;
  #pragma unroll
  for (int j = 0; j < 8; ++j) s = __builtin_amdgcn_sad_u8(x[j][j & 3] & 0x7C7C7C7Cu, 0u, s);
  #pragma unroll
  for (int j = 0; j < 7; ++j) *(LDS u32x4*)(scr + j * 1024 + lane * 16) = u32x4{0u, 0u, 0u, 0u};
  *(LDS u32x4*)(var + lane * 16) = u32x4{0u, 0u, 0u, 0u};
  *(LDS u32x4*)(var + 1024 + lane * 16) = u32x4{0u, 0u, 0u, 0u};
  {
    u32 sel = 0x0C0C0C0Cu, at = 0;
    #pragma unroll
    for (int y = 0; y < 4; ++y) if ((lane >> y) & 1) { sel = (sel & ~(0xFFu << at)) | ((u32)y << at); at += 8; }
    if (lane < 16) *(LDS u32*)(tab + lane * 4) = sel;
  }
  s = (u32)__builtin_amdgcn_readlane((int)wave_scan_u32(s), 63);
  const int K = max(5, min(31, ((int)s >> 13) + 3));
  const u32 kw = (128u + 4u * (u32)K) * 0x01010101u, tabo = (u32)(uintptr_t)tab;
  auto front = [&](u32 F) { return *(LDS u32*)(uintptr_t)__builtin_amdgcn_udot4(F, 0x20100804u, tabo, false); };
  u32 cs[32], cn[32], car[20], mine = 0;
  wave_sync();
  #pragma unroll
  for (int g = 0; g < 4; ++g) {
    u32 a[8], uu[8];
    #pragma unroll
    for (int k = 0; k < 8; ++k) {
      const int d = g * 8 + k;
      const u32 xv = x[d >> 2][d & 3];
      const u32 u = uu[k] = (kw - (xv & 0x7C7C7C7Cu)) ^ 0x80808080u;
      const u32 esc = (((u >> 1) & 0x78787878u) + 0x78787878u) | ~((u | (u << 1)) << 4);
      const u32 F = (esc >> 7) & 0x01010101u;
      a[k] = bfi(0x0C0C0C0Cu, ~u | __builtin_bit_cast(u32, __builtin_bit_cast(u16x2, F) * (u16x2){12, 12}), xv);
      cs[d] = front(F);
      cn[d] = mine;
      mine = __builtin_amdgcn_udot4(F, 0x01010101u, mine, false);
    }
    car[g * 5 + 0] = bfi(0x70707070u, a[5] << 4, a[0]);
    car[g * 5 + 1] = bfi(0x70707070u, a[6] << 4, a[1]);
    car[g * 5 + 2] = bfi(0x70707070u, a[7] << 4, a[2]);
    car[g * 5 + 3] = bfi(0x40404040u, a[7] << 3, bfi(0x20202020u, a[6] << 2, bfi(0x10101010u, a[5] << 1, a[3])));
    car[g * 5 + 4] = bfi(0x40404040u, a[7] >> 1, bfi(0x20202020u, a[6] >> 2, bfi(0x10101010u, a[5] >> 3, a[4])));
    __builtin_amdgcn_sched_barrier(0);
    #pragma unroll
    for (int k = 0; k < 8; ++k) cs[g * 8 + k] = __builtin_amdgcn_perm(0u, uu[k], cs[g * 8 + k]);
  }
  #pragma unroll
  for (int j = 0; j < 5; ++j) ST128((u32x4{car[j * 4], car[j * 4 + 1], car[j * 4 + 2], car[j * 4 + 3]}), rs, lane * 16 + j * 1024, offE, AUX_SYS);
  const u32 inc = wave_scan_u32(mine);
  const u32 nesc = (u32)__builtin_amdgcn_readlane((int)inc, 63);
  bool fits = nesc <= Q_ESC;
  if (fits) {
    const u32 base = inc - mine;
    #pragma unroll
    for (int d = 0; d < 32; ++d) {
      const u32 q = base + cn[d];
      const u64 v = (u64)cs[d] << ((q & 3u) * 8u);
      LDS u32* at = (LDS u32*)(scr + (q & ~3u));
      __hip_atomic_fetch_or(at, (u32)v, __ATOMIC_RELAXED, __HIP_MEMORY_SCOPE_WORKGROUP);
      __hip_atomic_fetch_or(at + 1, (u32)(v >> 32), __ATOMIC_RELAXED, __HIP_MEMORY_SCOPE_WORKGROUP);
    }
    wave_sync();
    u32 tbit = (QV_SEC + (nesc + 31) / 32 * 8) * 8, npat = 0;
    for (u32 r0 = 0; r0 < nesc && fits; r0 += 2048) {
      const u32x4 e0 = *(LDS u32x4*)(scr + r0 + lane * 32), e1 = *(LDS u32x4*)(scr + r0 + lane * 32 + 16);
      u32 two[2] = {0u, 0u}, zq[8], fs[8], t3[8], mk = 0, bits = 0;
      #pragma unroll
      for (int k = 0; k < 8; ++k) {
        const u32 u = k < 4 ? e0[k & 3] : e1[k & 3], sel = (u >> 2) & 0x07070707u;
        const u32 far = ((((u >> 1) & 0x70707070u) + 0x70707070u) >> 7) & 0x01010101u;
        const u32 c2 = __builtin_amdgcn_perm(0x03030201u, 0x03030300u, sel) | far | (far << 1);
        const u32 F = c2 & (c2 >> 1) & 0x01010101u;
        two[k >> 2] |= __builtin_amdgcn_udot4(c2, 0x40100401u, 0u, false) << (8 * (k & 3));
        fs[k] = front(F); t3[k] = (sel + 0x02020202u) & 0x07070707u;
        zq[k] = bits << 16;
        bits = __builtin_amdgcn_udot4(F, 0x03030303u, bits, false);
        const u32 lo7 = u & 0x7F7F7F7Fu;
        const u32 out = (F << 7) & (~(lo7 + 0x68686868u) | (lo7 + 0x48484848u) | u);
        mk |= out >> k;
      }
      __builtin_amdgcn_sched_barrier(0);
      #pragma unroll
      for (int k = 0; k < 8; ++k) {
        const u32 c3 = __builtin_amdgcn_perm(0u, t3[k], fs[k]);
        zq[k] |= __builtin_amdgcn_udot4(c3, 0x00400801u, 0u, false) + ((c3 >> 24) << 9);
      }
      if (r0 + lane * 32 < nesc) *(LDS u32x2*)(var + QV_SEC + r0 / 4 + lane * 8) = u32x2{two[0], two[1]};
      const u32 np = __popc(mk), inc2 = wave_scan_u32(bits | (np << 16)), tot = (u32)__builtin_amdgcn_readlane((int)inc2, 63);
      fits = tbit + (tot & 0xFFFFu) + 64 <= QV_END * 8 && npat + (tot >> 16) <= Q_PAT;
      if (!fits) break;
      const u32 b0 = tbit + (inc2 & 0xFFFFu) - bits;
      if (bits) {
        #pragma unroll
        for (int k = 0; k < 8; ++k) {
          const u32 q = b0 + (zq[k] >> 16);
          const u64 v = (u64)(zq[k] & 0xFFFFu) << (q & 31u);
          LDS u32* at = (LDS u32*)(var + ((q >> 5) << 2));
          __hip_atomic_fetch_or(at, (u32)v, __ATOMIC_RELAXED, __HIP_MEMORY_SCOPE_WORKGROUP);
          __hip_atomic_fetch_or(at + 1, (u32)(v >> 32), __ATOMIC_RELAXED, __HIP_MEMORY_SCOPE_WORKGROUP);
        }
      }
      u32 slot = npat + (inc2 >> 16) - np;
      for (u32 m = mk; m; m &= m - 1) {
        const int bp = __builtin_ctz(m);
        const u32 sn = r0 + lane * 32 + (7 - (bp & 7)) * 4 + (bp >> 3);
        const int v = (int)*(LDS signed char*)(scr + sn) >> 2;
        *(LDS unsigned short*)(var + (slot < Q_PAT0 ? 4 + slot * 2 : QV_END - Q_PAT0 * 2 + slot * 2)) = (unsigned short)((sn << 3) | (u32)(((v - 6) >> 3) + 4));
        ++slot;
      }
      tbit += tot & 0xFFFFu; npat += tot >> 16;
    }
    if (fits) {
      const bool more = tbit > QV0 * 8;
      if (lane == 0) *(LDS u32*)var = nesc | (npat << 13) | ((u32)K << 24) | (more ? 1u << 30 : 0u);
      wave_sync();
      if (lane < (int)QV0 / 16) ST128(*(LDS u32x4*)(var + lane * 16), rs, lane * 16, offE + QV, AUX_SYS);
      if (more) ST128(*(LDS u32x4*)(var + QV0 + lane * 16), rs, lane * 16, offE + QV + QV0, AUX_SYS);
      if (npat > Q_PAT0 && lane < (int)(QV_ALL - QV_END) / 16) ST128(*(LDS u32x4*)(var + QV_END + lane * 16), rs, lane * 16, offE + QV + QV_END, AUX_SYS);
    }
  }
  if (!fits) {
    store_raw_unit(x, rs, offR, lane);
    if (lane == 0) __builtin_amdgcn_raw_buffer_store_b32(0x80000000u, rs, 0, offE + QV, AUX_SYS);
  }
}

__device__ __forceinline__ void patch_short(LDS char* scr, u32 ent) {
  const u32 pos = ent >> 3;
  __hip_atomic_fetch_add((LDS u32*)(scr + (pos & ~3u)), ((4u - (ent & 7u)) << 5) << ((pos & 3u) * 8u), __ATOMIC_RELAXED, __HIP_MEMORY_SCOPE_WORKGROUP);
}
__device__ __forceinline__ void patch_in(LDS char* scr, u32 ent) {
  const u32 pos = ent >> 3;
  __hip_atomic_fetch_add((LDS u32*)(scr + (pos & ~3u)), (((ent & 7u) - 4u) << 5) << ((pos & 3u) * 8u), __ATOMIC_RELAXED, __HIP_MEMORY_SCOPE_WORKGROUP);
}

__device__ __forceinline__ float unfix(int i, int g) { return __builtin_amdgcn_ldexpf((float)i, -g); }
__device__ __forceinline__ u32 top_pack(u32 t, u32& miss) {
  const u32 m = __builtin_amdgcn_perm(0x00010208u, 0x08020100u, (t >> 5) & 0x07070707u);
  miss = __umul24(m >> 8, 3u) + m;
  return bfi(0x20002000u, miss << 11, bfi(0x00600060u, miss << 5, t));
}
__device__ __forceinline__ u32 top_unpack(u32 x) {
  const u32 r = __builtin_amdgcn_perm(0x24044222u, 0x02402000u, bfi(0x00030003u, x >> 5, x >> 11) & 0x00070007u);
  const u32 u = bfi(0x60006000u, r << 12, bfi(0x00600060u, r, x));
  return __builtin_amdgcn_bitop3_b32(__builtin_amdgcn_bitop3_b32(u, x >> 1, 0x40404040u, 0x78), x >> 2, 0x20202020u, 0x78);
}

__global__ void __launch_bounds__(NTHR, 2) gemm_rs_kernel(Params p) {
  extern __shared__ __attribute__((aligned(16))) char smem_[];
  LDS char* smem = (LDS char*)smem_;
  const int tid = threadIdx.x, lane = tid & 63, wave = __builtin_amdgcn_readfirstlane(tid >> 6);
  const int b = blockIdx.x, rank = p.rank;
  const int lg = lane >> 4, li = lane & 15;
  const rsrc_t rsL = mk_rsrc(p.ws, p.ws_bytes);
  const u32 tick = (u32)__builtin_amdgcn_raw_ptr_buffer_atomic_add_i32(1, rsL, lane == 0 ? 0 : (int)0xF0000000u, OFF_CALL + b * 128 + wave * 16, 0);
  char* const peer_ws = p.ws_all[lane & 7];
  u32 call;

  if (b < 128) {
    const rsrc_t rsW = mk_rsrc(p.w, KS * NN);
    __builtin_amdgcn_s_sleep(12);
    #pragma unroll
    for (int i = 0; i < 8; ++i) {
      const int row = (wave * 8 + i) * 8 + (lane >> 3);
      const int sw = ((row >> 1) & 3) | (((row >> 4) & 1) << 2);
      __builtin_amdgcn_raw_ptr_buffer_load_lds(rsW, (LDS void*)(smem + (wave * 8 + i) * 1024), 16, row * NN + (((lane & 7) ^ sw) << 4), b * 128, 0, 0);
    }
    asm volatile("s_waitcnt vmcnt(0)\n\ts_barrier" ::: "memory");
    call = __builtin_amdgcn_readfirstlane(tick) + 1u;
    const int cgl = wave & 1, kb = wave >> 1;
    u32x4 x[8];
    #pragma unroll
    for (int f = 0; f < 4; ++f)
      #pragma unroll
      for (int h = 0; h < 2; ++h)
        #pragma unroll
        for (int t = 0; t < 2; ++t) {
          const int row = kb * 128 + 64 * h + 16 * lg + 8 * t + (li >> 1);
          const int sw = ((row >> 1) & 3) | (((row >> 4) & 1) << 2);
          const i32x2 v = __builtin_amdgcn_ds_read_tr8_b64_v2i32((LDS i32x2*)(smem + row * 128 + (((cgl * 4 + f) ^ sw) << 4) + (li & 1) * 8));
          x[f * 2 + h][t * 2] = v.x; x[f * 2 + h][t * 2 + 1] = v.y;
        }
    const int gl = b >> 5, gr = 4 + ((b >> 3) & 3), c = b & 7;
    if (kb == 0 && c == 0) {
      #pragma unroll 1
      for (int hf = 0; hf < 2; ++hf) {
        const u32x4 xh[4] = {hf ? x[4] : x[0], hf ? x[5] : x[1], hf ? x[6] : x[2], hf ? x[7] : x[3]};
        export_half(xh, rsL, OFF_PAR + (call & 1) * PAR_BYTES, 256 + (b * 2 + cgl) * 4, hf, smem + L_SCR + wave * SCR_W, lane);
      }
    } else export_short(x, rsL, OFF_PAR + (call & 1) * PAR_BYTES, 256 + (b * 2 + cgl) * 4 + kb, smem + L_SCR + wave * SCR_W, lane);
    vm_drain();
    if ((lane == gl || lane == gr) && lane != rank) {
      const int cb = (lane == gl ? gr & 3 : gl) * 8 + c;
      __hip_atomic_store((GLB u32*)(peer_ws + OFF_FIN + (rank * 576 + 256 + (cb * 2 + cgl) * 4 + kb) * 128), call, __ATOMIC_RELAXED, __HIP_MEMORY_SCOPE_SYSTEM);
    }
    if (rank == gl || rank == gr) {
      const int cb = (rank == gl ? gr & 3 : gl) * 8 + c;
      store_raw_unit(x, rsL, OFF_WA + ((rank * 4 + kb) * 64 + cb * 2 + cgl) * IU_RAW, lane);
      vm_drain();
      cnt_add(p.ws, (64 + cb) * 5 + 4, lane);
    }
  } else if (b < 192) {
    const int rg = b - 128, kb = wave & 3, hf = wave >> 2, u = rg * 4 + kb;
    const rsrc_t rsX = mk_rsrc(p.x, MM * KS);
    const u32 both = (u32)(uintptr_t)(smem + L_CTL) + 16;
    const u32 xoff = OFF_XA + ((rank * 4 + kb) * 64 + rg) * IU_RAW;
    const int ow = rg >> 3, r8 = rg & 7;
    const u32 users = ((r8 <= 3 ? 0xFu << (ow & 4) : 0u) | (r8 >= 3 ? 0xFu << (~ow & 4) : 0u) | (1u << ow)) & ~(1u << rank);
    const bool tell = lane < NR && ((users >> lane) & 1);
    if (kb < 2) {
      u32x4 x[4];
      #pragma unroll
      for (int j = 0; j < 4; ++j)
        x[j] = __builtin_amdgcn_raw_buffer_load_b128(rsX, ((j >> 1) * 16 + li) * KS + (j & 1) * 64 + lg * 16, (rg * 64 + hf * 32) * KS + kb * 128, 0);
      if (wave == 0) ctl_st(both + (lane & 3) * 4, 0u);
      asm volatile("s_waitcnt lgkmcnt(0)\n\ts_barrier" ::: "memory");
      call = __builtin_amdgcn_readfirstlane(tick) + 1u;
      const u32 parw = OFF_PAR + (call & 1) * PAR_BYTES;
      auto second = [&](int k) -> bool {
        u32 old = 0;
        if (lane == 0) old = __hip_atomic_fetch_add((LDS u32*)(uintptr_t)(both + k * 4), 1u, __ATOMIC_RELAXED, __HIP_MEMORY_SCOPE_WORKGROUP);
        return __builtin_amdgcn_readfirstlane((int)old) == 1;
      };
      if (NRAW && kb == 0) {
        #pragma unroll
        for (int j = 0; j < 4; ++j) ST128(x[j], rsL, lane * 16 + j * 1024, parw + EX_INR + u * IU_RAW + hf * 4096, AUX_SYS);
        vm_drain();
        if (second(2) && tell) __hip_atomic_store((GLB u32*)(peer_ws + OFF_FIN + (rank * 576 + 512 + rg) * 128), call, __ATOMIC_RELAXED, __HIP_MEMORY_SCOPE_SYSTEM);
      }
      export_half(x, rsL, parw, u, hf, smem + L_SCR + wave * SCR_W, lane);
      vm_drain();
      if (second(kb) && tell) __hip_atomic_store((GLB u32*)(peer_ws + OFF_FIN + (rank * 576 + u) * 128), call, __ATOMIC_RELAXED, __HIP_MEMORY_SCOPE_SYSTEM);
      #pragma unroll
      for (int j = 0; j < 4; ++j) ST128(x[j], rsL, lane * 16 + j * 1024, xoff + hf * 4096, AUX_SYS);
      vm_drain();
      cnt_add(p.ws, rg * 5 + 4, lane);
    } else if (hf == 0) {
      u32x4 x[8];
      #pragma unroll
      for (int j = 0; j < 8; ++j)
        x[j] = __builtin_amdgcn_raw_buffer_load_b128(rsX, ((j >> 1) * 16 + li) * KS + (j & 1) * 64 + lg * 16, rg * 64 * KS + kb * 128, 0);
      asm volatile("s_barrier" ::: "memory");
      call = __builtin_amdgcn_readfirstlane(tick) + 1u;
      export_short(x, rsL, OFF_PAR + (call & 1) * PAR_BYTES, u, smem + L_SCR + wave * SCR_W, lane);
      vm_drain();
      if (tell) __hip_atomic_store((GLB u32*)(peer_ws + OFF_FIN + (rank * 576 + u) * 128), call, __ATOMIC_RELAXED, __HIP_MEMORY_SCOPE_SYSTEM);
      store_raw_unit(x, rsL, xoff, lane);
      vm_drain();
      cnt_add(p.ws, rg * 5 + 4, lane);
    } else {
      asm volatile("s_barrier" ::: "memory");
    }
  }
  call = __builtin_amdgcn_readfirstlane(tick) + 1u;
  const u32 par = OFF_PAR + (call & 1) * PAR_BYTES;
  const float scale = p.sx[0] * p.sw[0];
  const rsrc_t rsO = mk_rsrc(p.out, 512u * NN * 4u);

  if (b >= NGEMM) {
    const int pb = b - NGEMM, link = pb / PULL_WGS, q = pb % PULL_WGS;
    const int src = (rank + 1 + link) & 7;
    const u64 pw = (u64)peer_ws;
    const u32 pw_lo = (u32)__builtin_amdgcn_readlane((int)(u32)pw, src), pw_hi = (u32)__builtin_amdgcn_readlane((int)(u32)(pw >> 32), src);
    const rsrc_t rsP = mk_rsrc((const void*)(((u64)pw_hi << 32) | (u64)pw_lo), p.ws_bytes);
    const u32 ctl = (u32)(uintptr_t)(smem + L_CTL);
    const u32 C_READY = ctl, C_LANDED = ctl + 4, C_CLOSED = ctl + 12, C_REL = ctl + 16, C_ISSUED = ctl + 8, C_Q = ctl + 64;
    const bool same = (src >> 2) == (rank >> 2);
    if (wave == 0) ctl_st(ctl + lane * 4, 0u);
    asm volatile("s_waitcnt lgkmcnt(0)\n\ts_barrier" ::: "memory");
    if (wave == 0) {
      int g = 0, pub = 0;
      #define LANDED(v) do { if ((v) > pub) { pub = (v); ctl_st(C_LANDED, (u32)pub); } } while (0)
      #define PIECEW(W, SZ, VOFF, OFF) do { __builtin_amdgcn_raw_ptr_buffer_load_lds(rsP, (LDS void*)(dst + (OFF)), SZ, VOFF, so + (OFF), 0, AUX_SYS); \
        asm volatile("s_waitcnt vmcnt(%0)" :: "i"((W) - 1) : "memory"); ++g; LANDED(g - ((W) - 1)); } while (0)
      #define PIECE(SZ, VOFF, OFF) PIECEW(WIN, SZ, VOFF, OFF)
      for (u32 n = 0;; ++n) {
        if (ctl_ld(C_READY) <= n) {
          vm_drain(); LANDED(g);
          bool fin = false;
          while (ctl_ld(C_READY) <= n) { if (ctl_ld(C_CLOSED)) { fin = ctl_ld(C_READY) <= n; break; } __builtin_amdgcn_s_sleep(1); }
          if (fin) break;
        }
        const u32 ent = ctl_ld(C_Q + (n & 15) * 4);
        ctl_st(C_ISSUED, n + 1);
        if (ent >> 31) continue;
        const u32 need = n >= NSLOT ? n - NSLOT + 1 : 0, ra = C_REL + (n % NSLOT) * 4;
        if (ctl_ld(ra) < need) {
          vm_drain(); LANDED(g);
          while (ctl_ld(ra) < need) __builtin_amdgcn_s_sleep(1);
        }
        const int i = ent & 1023;
        LDS char* dst = smem + (n % NSLOT) * SLOT;
        if (ent & 1024) {
          const u32 so = par + EX_INR + i * IU_RAW;
          #pragma unroll
          for (int j = 0; j < 8; ++j) PIECE(16, lane * 16, j * 1024);
        } else if (i < 512 && quick_form(i)) {
          const u32 so = par + EX_INE + (i < 256 ? i : 256 + group_of(rank, (i >> 6) & 3) * 64 + (i & 63)) * IU_ENC;
          PIECE(4, lane * 4, IU_SIDE);
          #pragma unroll
          for (int j = 0; j < 6; ++j) PIECE(16, lane * 16, j * 1024);
        } else if (i < 512) {
          const u32 so = par + EX_INE + (i < 256 ? i : 256 + group_of(rank, (i >> 6) & 3) * 64 + (i & 63)) * IU_ENC;
          if (lane < (int)QV0 / 16) __builtin_amdgcn_raw_ptr_buffer_load_lds(rsP, (LDS void*)(dst + QV), 16, lane * 16, so + QV, 0, AUX_SYS);
          asm volatile("s_waitcnt vmcnt(%0)" :: "i"(WIN - 1) : "memory"); ++g; LANDED(g - (WIN - 1));
          #pragma unroll
          for (int j = 0; j < 5; ++j) PIECE(16, lane * 16, j * 1024);
        } else {
          const u32 so = par + EX_OUE + (rank * 320 + i - 512) * OU_ENC;
          #pragma unroll
          for (int j = 0; j < 12; ++j) PIECEW(WIN_OUT, 16, lane * 16, (j + 11) % 12 * 1024);
        }
      }
      vm_drain(); LANDED(g);
      #undef PIECE
      #undef PIECEW
      #undef LANDED
      return;
    }
    if (wave == 1) {
      const u32 tic0 = OFF_TIC + (call & 1) * 24 * 128;
      const int only0 = lane == 0 ? 0 : (int)0xF0000000u;
      if (q == 0 && (lane == link || (link == 0 && lane >= 8 && lane < 24)))
        __hip_atomic_store((GLB u32*)(p.ws + OFF_TIC + ((~call & 1) * 24 + lane) * 128), 0u, __ATOMIC_RELAXED, __HIP_MEMORY_SCOPE_AGENT);
      constexpr u32 NPAIR = 208 + NRAW / 2, NFIX = 192;
      u32 n = 0, endp = 0; bool ok = true;
      int fj = -1, fnext = -1;
      const u32 gtic = tic0 + (8 + group_of(src, lane & 3)) * 128;
      auto fullest = [&](u32 v) -> int {
        v = v >= 16u || (!same && (lane & 3) == (rank & 3)) ? 0xFFFFu : (v << 4) | ((((lane - q) & 3) << 2) | (lane & 3));
        v = min(v, (u32)__builtin_amdgcn_mov_dpp((int)v, 0xB1, 0xf, 0xf, true));
        v = min(v, (u32)__builtin_amdgcn_mov_dpp((int)v, 0x4E, 0xf, 0xf, true));
        v = (u32)__builtin_amdgcn_readfirstlane((int)v);
        return v == 0xFFFFu ? -2 : (int)(v & 3u);
      };
      auto draw = [&](int g) { return __builtin_amdgcn_raw_ptr_buffer_atomic_add_i32(1, rsL, only0, tic0 + (g < 0 ? link : 8 + group_of(src, g)) * 128, 0); };
      int tnext = draw(-1);
      u32 seen = 0;
      for (;;) {
        u32 t = (u32)__builtin_amdgcn_readfirstlane(tnext);
        fj = fnext;
        if (t >= (fj < 0 ? NPAIR + NFIX : 16u)) {
          if ((fnext = fullest(ld_flag(p.ws, gtic))) < 0) break;
          tnext = draw(fnext);
          continue;
        }
        while (n >= ctl_ld(C_ISSUED) + (t < NPAIR && fj < 0 ? LOOK : LOOK_OUT)) __builtin_amdgcn_s_sleep(1);
        bool last = false;
        if (fj >= 0 || t + PULL_WGS >= NPAIR + NFIX) last = (fnext = fullest(seen)) < 0;
        if (last && fj < 0) { fnext = -1; last = false; }
        if (!last) tnext = draw(fnext);
        if (t >= NPAIR || fj >= 0) seen = ld_flag(p.ws, gtic);
        if (fj >= 0) t += NPAIR + NFIX;
        int i;
        if (t < NPAIR) {
          const int e = t < NRAW ? t : 2 * t - NRAW + (lane & 1);
          int u = -1, sg = 0, kb = 0;
          if (e < 32 + X1FIRST) { kb = e >= NRAW && e < NRAW + X1FIRST; u = e < NRAW ? e : kb ? e - NRAW : e - X1FIRST; }
          else if (e < 40 + X1FIRST) sg = 0;
          else if (e < 72) { kb = 1; u = e - 40; }
          else if (e < 88) sg = (e - 64) >> 3;
          else if (e < 120) { kb = 2; u = e - 88; }
          else if (e < 144) sg = (e - 96) >> 3;
          else if (e < 176) { kb = 3; u = e - 144; }
          else sg = (e - 128) >> 3;
          int c = 0;
          if (u < 0) {
            if (sg < 6) { c = (0x210100 >> (sg * 4)) & 7; kb = (0x012010 >> (sg * 4)) & 3; }
            else if (sg < 26) { c = ((sg - 6) >> 2) + ((sg - 6) & 3); kb = 3 - ((sg - 6) & 3); }
            else { c = (0x776765 >> ((sg - 26) * 4)) & 7; kb = (0x323123 >> ((sg - 26) * 4)) & 3; }
          }
          const int m = c * 8 + (e & 7);
          const int v = u - 12;
          if (e >= 384) i = (8 * rank + (e & 7)) * 4 + ((e - 384) >> 3);
          else if (u < 0) i = 256 + ((((m >> 1) & 3) * 8 + (m >> 3)) * 2 + (m & 1)) * 4 + kb;
          else if (u < 12) i = (8 * ((rank & 4) | ((rank + 1 + (u >> 2)) & 3)) + (u & 3)) * 4 + kb;
          else i = (8 * ((~rank & 4) | (v / 5)) + 3 + v % 5) * 4 + kb;
        } else if (fj < 0) {
          const int o = t - NPAIR, cm = same ? o / 3 : o >> 2, k = same ? o - cm * 3 : 1 + (o & 3);
          const int c = same ? cm >> 3 : cm / 6, j3 = (cm - c * 6) >> 1, j = same ? (cm >> 1) & 3 : j3 + (j3 >= (rank & 3));
          i = 512 + ((j * 8 + c) * 2 + (cm & 1)) * 5 + k;
        } else i = 512 + (fj * 16 + (int)(t - NPAIR - NFIX)) * 5 + (same ? 3 : 0);
        const bool unp = t < NRAW, two = !unp && t < NPAIR;
        const u32 fo = i >= 512 ? OFF_FOUT + (src * 320 + i - 512) * 128 : OFF_FIN + (src * 576 + (unp ? 512 + (i >> 2) : i)) * 128;
        ok = ok && wait_ge(p.ws, fo, call, call);
        const u32 k = two ? (lane & 1) : 0, per = !ok ? 0 : t >= NPAIR ? 12 : unp ? 8 : (u32)__builtin_amdgcn_readfirstlane(quick_form(i) ? 7 : 6);
        ctl_st(C_Q + ((n + k) & 15) * 4, (u32)i | (unp ? 1024u : 0u) | ((endp + per * (k + 1)) << 11) | (ok ? 0u : 0x80000000u));
        n += two ? 2 : 1; endp += two ? 2 * per : per;
        asm volatile("s_waitcnt lgkmcnt(0)" ::: "memory");
        ctl_st(C_READY, n);
        if (last) break;
      }
      asm volatile("s_waitcnt lgkmcnt(0)" ::: "memory");
      ctl_st(C_CLOSED, 1u);
      return;
    }
    LDS char* scr = smem + L_PSCR + (wave - 2) * 8 * KiB;
    for (u32 n = wave - 2;; n += NCONS) {
      bool fin = false;
      while (ctl_ld(C_READY) <= n) { if (ctl_ld(C_CLOSED)) { fin = ctl_ld(C_READY) <= n; break; } __builtin_amdgcn_s_sleep(1); }
      if (fin) break;
      const u32 ent = ctl_ld(C_Q + (n & 15) * 4);
      const bool ok = !(ent >> 31);
      const int i = ent & 1023;
      const u32 endp = (ent >> 11) & 0x1FFFu;
      auto landed = [&](u32 upto) { while (ctl_ld(C_LANDED) < upto) __builtin_amdgcn_s_sleep(1); };
      LDS char* sl = smem + (n % NSLOT) * SLOT;
      if (i < 512) {
        if (!ok) continue;
        landed(endp);
        const int kb = i & 3, rg = (i >> 2) & 63; const bool isx = i < 256;
        const u32 uid = isx ? i : 256 + group_of(rank, (i >> 6) & 3) * 64 + (i & 63);
        const u32 dst = (isx ? OFF_XA : OFF_WA) + ((src * 4 + kb) * 64 + rg) * IU_RAW;
        const u32 rel = C_REL + (n % NSLOT) * 4, offE = par + EX_INE + uid * IU_ENC;
        u32x4 x[8];
        bool raw = false;
        if (ent & 1024) {
          #pragma unroll
          for (int j = 0; j < 8; ++j) x[j] = *(LDS u32x4*)(sl + j * 1024 + lane * 16);
          wave_sync();
          ctl_st(rel, n + 1);
        } else if (quick_form(i)) {
          u32x4 c[6];
          #pragma unroll
          for (int j = 0; j < 6; ++j) c[j] = *(LDS u32x4*)(sl + j * 1024 + lane * 16);
          const u32 sd = *(LDS u32*)(sl + IU_SIDE + lane * 4);
          wave_sync();
          ctl_st(rel, n + 1);
          const u32 hd[2] = {(u32)__builtin_amdgcn_readlane((int)sd, 0), (u32)__builtin_amdgcn_readlane((int)sd, 32)};
          #pragma unroll
          for (int j = 0; j < 8; ++j) {
            const u32 e4 = ((hd[j >> 2] >> 16) & 31u) * 0x04040404u;
            const u32 p0 = c[(j * 3) >> 2][(j * 3) & 3], p1 = c[(j * 3 + 1) >> 2][(j * 3 + 1) & 3], p2 = c[(j * 3 + 2) >> 2][(j * 3 + 2) & 3];
            const u32 o3 = bfi(0x80808080u, p2 << 1, bfi(0x10101010u, p2 >> 1, bfi(0x0C0C0C0Cu, p1 >> 3, p0 >> 5)));
            const u32x4 y = {(p0 & 0x9F9F9F9Fu) + e4, (p1 & 0x9F9F9F9Fu) + e4, (p2 & 0x9F9F9F9Fu) + e4, (o3 & 0x9F9F9F9Fu) + e4};
            *(LDS u32x4*)(scr + j * 1024 + lane * 16) = y;
          }
          wave_sync();
          const u32 cnt = (lane < 32 ? hd[0] : hd[1]) & 0xFFFFu, id0 = 2u * (lane & 31) - 2u;
          if (lane & 31) {
            if (id0 < cnt) patch_in(scr, sd & 0xFFFFu);
            if (id0 + 1 < cnt) patch_in(scr, sd >> 16);
          }
          if (max(hd[0] & 0xFFFFu, hd[1] & 0xFFFFu) > IN_CAP0) {
            for (int j = 0; j < 3; ++j) {
              const u32 s2 = __builtin_amdgcn_raw_buffer_load_b32(rsP, 256 + (lane >> 5) * 384 + (j * 32 + (lane & 31)) * 4, offE + IU_SIDE, AUX_SYS);
              const u32 id = IN_CAP0 + j * 64 + 2 * (lane & 31);
              if (id < cnt) patch_in(scr, s2 & 0xFFFFu);
              if (id + 1 < cnt) patch_in(scr, s2 >> 16);
            }
          }
          wave_sync();
          #pragma unroll
          for (int j = 0; j < 8; ++j) x[j] = *(LDS u32x4*)(scr + j * 1024 + lane * 16);
          #pragma unroll
          for (int h = 0; h < 2; ++h)
            if (hd[h] >> 31) {
              #pragma unroll
              for (int j = h * 4; j < h * 4 + 4; ++j) x[j] = __builtin_amdgcn_raw_buffer_load_b128(rsP, lane * 16 + j * 1024, par + EX_INR + uid * IU_RAW, AUX_SYS);
            }
        } else {
          LDS char* var = sl + QV;
          u32x4 c[5];
          #pragma unroll
          for (int j = 0; j < 5; ++j) c[j] = *(LDS u32x4*)(sl + j * 1024 + lane * 16);
          const u32 hdr = (u32)__builtin_amdgcn_readfirstlane((int)*(LDS u32*)var);
          const u32 pe = *(LDS unsigned short*)(var + 4 + li * 2);
          raw = hdr >> 31;
          if (!raw) {
            const u32 nesc = hdr & 0x1FFFu, npat = (hdr >> 13) & 0x1FFu, bias = (4u * ((hdr >> 24) & 31u) - 136u) * 0x01010101u;
            if ((hdr >> 30) & 1u)
              *(LDS u32x4*)(var + QV0 + lane * 16) = __builtin_amdgcn_raw_buffer_load_b128(rsP, lane * 16, offE + QV + QV0, AUX_SYS);
            u32 tbit = ((u32)(uintptr_t)var + QV_SEC + (nesc + 31) / 32 * 8) * 8;
            wave_sync();
            for (u32 r0 = 0; r0 < nesc; r0 += 2048) {
              const u32x2 two = *(LDS u32x2*)(var + QV_SEC + r0 / 4 + lane * 8);
              u32 c2[8], e2[8], zo[8], bits = 0;
              #pragma unroll
              for (int k = 0; k < 8; ++k) {
                const u32 z = two[k >> 2] >> (8 * (k & 3)), z2 = bfi(0x0000000Fu, z, z << 12);
                c2[k] = bfi(0x00030003u, z2, z2 << 6) & 0x03030303u;
                e2[k] = (c2[k] << 2) & (c2[k] << 1) & 0x04040404u;
                zo[k] = bits;
                bits = __builtin_amdgcn_udot4(e2[k], 0x03030303u, bits, false);
              }
              const u32 inc = wave_scan_u32(bits >> 2), b4 = (tbit + inc) * 4 - bits;
              tbit += (u32)__builtin_amdgcn_readlane((int)inc, 63);
              u32 g0[8], g1[8];
              #pragma unroll
              for (int k = 0; k < 8; ++k) {
                zo[k] += b4;
                const LDS u32* at = (const LDS u32*)(uintptr_t)((zo[k] >> 5) & ~3u);
                g0[k] = at[0]; g1[k] = at[1];
              }
              __builtin_amdgcn_sched_barrier(0);
              #pragma unroll
              for (int k = 0; k < 8; ++k) {
                const u32 z = __builtin_amdgcn_alignbit(g1[k], g0[k], zo[k] >> 2), z2 = bfi(0x0000003Fu, z, z << 10);
                const u32 sp = bfi(0x00070007u, z2, z2 << 5) & 0x07070707u;
                const u32 c3 = __builtin_amdgcn_perm(sp, 0u, __umul24(e2[k], 0x404040u) + e2[k]);
                zo[k] = __builtin_amdgcn_perm(0u, 0x64686C7Cu, c2[k]) - (c3 << 2);
              }
              *(LDS u32x4*)(scr + r0 + lane * 32) = u32x4{zo[0], zo[1], zo[2], zo[3]};
              *(LDS u32x4*)(scr + r0 + lane * 32 + 16) = u32x4{zo[4], zo[5], zo[6], zo[7]};
            }
            wave_sync();
            ctl_st(rel, n + 1);
            if ((u32)lane < npat && lane < (int)Q_PAT0) patch_short(scr, pe);
            if (npat > Q_PAT0) {
              for (int j = 0; j < 2; ++j) {
                const u32 s2 = __builtin_amdgcn_raw_buffer_load_b32(rsP, lane * 4 + j * 256, offE + QV + QV_END, AUX_SYS);
                const u32 id = Q_PAT0 + j * 128 + 2 * lane;
                if (id < npat) patch_short(scr, s2 & 0xFFFFu);
                if (id + 1 < npat) patch_short(scr, s2 >> 16);
              }
            }
            u32 a[32], es[32], qo[32], cnt4 = 0;
            #pragma unroll
            for (int g = 0; g < 4; ++g) {
              u32 k5[5];
              #pragma unroll
              for (int k = 0; k < 5; ++k) { k5[k] = c[(g * 5 + k) >> 2][(g * 5 + k) & 3]; a[g * 8 + k] = k5[k]; }
              a[g * 8 + 5] = bfi(0x80808080u, k5[4] << 3, bfi(0x08080808u, k5[3] >> 1, k5[0] >> 4));
              a[g * 8 + 6] = bfi(0x80808080u, k5[4] << 2, bfi(0x08080808u, k5[3] >> 2, k5[1] >> 4));
              a[g * 8 + 7] = bfi(0x80808080u, k5[4] << 1, bfi(0x08080808u, k5[3] >> 3, k5[2] >> 4));
            }
            #pragma unroll
            for (int d = 0; d < 32; ++d) {
              es[d] = a[d] & (a[d] >> 1) & 0x04040404u;
              qo[d] = cnt4;
              cnt4 = __builtin_amdgcn_udot4(es[d], 0x01010101u, cnt4, false);
            }
            const u32 base4 = ((u32)(uintptr_t)scr + wave_scan_u32(cnt4 >> 2)) * 4 - cnt4;
            wave_sync();
            #pragma unroll
            for (int h = 0; h < 2; ++h) {
              u32 g0[16], g1[16];
              #pragma unroll
              for (int k = 0; k < 16; ++k) {
                qo[h * 16 + k] = (base4 + qo[h * 16 + k]) >> 2;
                const LDS u32* at = (const LDS u32*)(uintptr_t)(qo[h * 16 + k] & ~3u);
                g0[k] = at[0]; g1[k] = at[1];
              }
              __builtin_amdgcn_sched_barrier(0);
              #pragma unroll
              for (int k = 0; k < 16; ++k) {
                const int d = h * 16 + k;
                const u32 nx = __builtin_amdgcn_alignbyte(g1[k], g0[k], qo[d]);
                const u32 sel = __umul24(es[d], 0x404040u) + es[d];
                x[d >> 2][d & 3] = (a[d] & 0x8F8F8F8Fu) + __builtin_amdgcn_perm(nx, 0x7C7C7C7Cu, sel) + bias;
              }
              __builtin_amdgcn_sched_barrier(0);
            }
          } else {
            wave_sync();
            ctl_st(rel, n + 1);
          }
        }
        if (raw) {
          #pragma unroll
          for (int j = 0; j < 8; ++j) x[j] = __builtin_amdgcn_raw_buffer_load_b128(rsP, lane * 16 + j * 1024, par + EX_INR + uid * IU_RAW, AUX_SYS);
        }
        store_raw_unit(x, rsL, dst, lane);
        vm_drain();
        cnt_add(p.ws, (isx ? rg : 64 + (rg >> 1)) * 5 + kb, lane);
        continue;
      }
      const int t = i - 512, cg = t / 5, rg = t - cg * 5 + (same ? 0 : 3);
      const u32 obase = (rg * 64 * NN + group_of(src, cg >> 4) * 1024 + (cg & 15) * 64) * 4;
      const u32 olane = (li * NN + lg * 4) * 4;
      if (!ok) {
        const u32x4 nan4 = {0x7FC00000u, 0x7FC00000u, 0x7FC00000u, 0x7FC00000u};
        #pragma unroll
        for (int e = 0; e < 16; ++e) ST128(nan4, rsO, olane + (e & 3) * 64, obase + (e >> 2) * 16 * NN * 4, AUX_SYS);
        continue;
      }
      #pragma unroll
      for (int j = 0; j < 4; ++j) *(LDS u32x4*)(scr + j * 1024 + lane * 16) = u32x4{0u, 0u, 0u, 0u};
      landed(endp - 11);
      const u32 odd = *(LDS unsigned short*)(sl + OU_SIDE + lane * 2) | ((u32)*(LDS unsigned char*)(sl + OU_SIDE + 128 + lane) << 16);
      const u32 hdr = (u32)__builtin_amdgcn_readfirstlane((int)*(LDS u32*)(sl + OU_SIDE + 192));
      u32 se = *(LDS unsigned short*)(sl + OU_SIDE + 196 + (lane < 24 ? lane : 0) * 2);
      u32x2 fe = {*(LDS unsigned short*)(sl + OU_SIDE + 244 + (lane & 1) * 2), *(LDS u32*)(sl + OU_SIDE + 248 + (lane & 1) * 4)};
      const u32 tl[3] = {*(LDS u32*)(sl + 11264 + lane * 4), *(LDS u32*)(sl + 11520 + lane * 4), *(LDS u32*)(sl + 11776 + lane * 4)};
      const u32 uid = rank * 320 + t;
      if (!(hdr >> 31)) {
        const u32 ns = hdr & 0xFFu, nf = (hdr >> 8) & 0xFFu; const int g = (int)((hdr >> 16) & 0x1FFu) - 256;
        const u32 tail = par + EX_OUE + uid * OU_ENC + 12288;
        if (ns > OS_CAP0) {
          const u32 s2 = __builtin_amdgcn_raw_buffer_load_b16(rsP, lane >= (int)OS_CAP0 ? lane * 2 : OS_CAP0 * 2, tail - OS_CAP0 * 2, AUX_SYS);
          se = lane >= (int)OS_CAP0 ? s2 : se;
          asm volatile("" : "+v"(se));
        }
        if (nf > OF_CAP0) {
          const u32x2 f2 = __builtin_amdgcn_raw_buffer_load_b64(rsP, max((u32)OF_CAP0, min((u32)lane, nf - 1)) * 8, tail + 128 - OF_CAP0 * 8, AUX_SYS);
          fe = lane >= (int)OF_CAP0 ? f2 : fe;
          asm volatile("" : "+v"(fe));
        }
        if ((u32)lane < ns)
          __hip_atomic_fetch_add((LDS u32*)(scr + ((se >> 13) & 3u) * 1024 + ((se >> 5) & 63u) * 16 + ((se >> 11) & 3u) * 4),
                                 se >> 15 ? ((se & 3u) | ((se & 12u) << 6)) << (se & 16u) : ((se & 7u) << 5) << (se & 24u),
                                 __ATOMIC_RELAXED, __HIP_MEMORY_SCOPE_WORKGROUP);
        const u32 fgq = (u32)lane < nf ? fe.x >> 8 : 16u;
        u64 om = (1ull << (odd & 63)) | (1ull << ((odd >> 6) & 63)) | (1ull << ((odd >> 12) & 63)) | (1ull << ((odd >> 18) & 63));
        if ((odd & 63) > ((odd >> 6) & 63)) om = 0;
        const u32 ob[2] = {(u32)om, (u32)(om >> 32)};
        u32 t15 = tl[0] << 1;
        wave_sync();
        #pragma unroll
        for (int mi = 0; mi < 4; ++mi) {
          landed(mi < 3 ? endp - 8 + 3 * mi : endp);
          u32x4 c[3];
          #pragma unroll
          for (int j = 0; j < (mi < 3 ? 3 : 2); ++j) c[j] = *(LDS u32x4*)(sl + (mi * 3 + j) * 1024 + lane * 16);
          const u32x4 adj = *(LDS u32x4*)(scr + mi * 1024 + lane * 16);
          if (mi == 3) { wave_sync(); ctl_st(C_REL + (n % NSLOT) * 4, n + 1); c[2] = u32x4{tl[0], tl[1], tl[2], 0u}; }
          #pragma unroll
          for (int ni = 0; ni < 4; ++ni) {
            const int k = ni * 3, gq = mi * 4 + ni;
            const u32 l0 = c[k >> 2][k & 3], l1 = c[(k + 1) >> 2][(k + 1) & 3], tp = gq < 15 ? c[(k + 2) >> 2][(k + 2) & 3] : t15;
            if (gq < 14) t15 = bfi(0x00010001u << gq, tp >> (14 - gq), t15);
            const u32 top = __builtin_amdgcn_bitop3_b32(top_unpack(tp), adj[ni] << 5, 0x60606060u, 0x78);
            const u32 low = ((ob[gq >> 3] << (gq & 7)) & 0x80808080u) + (adj[ni] & 0xE0E0E0E0u);
            const u32 q01 = __builtin_amdgcn_perm(top, low, 0x05010400u), q23 = __builtin_amdgcn_perm(top, low, 0x07030602u);
            float y[4] = {unfix((int)__builtin_amdgcn_perm(q01, l0, 0x05010004u), g + 7), unfix((int)__builtin_amdgcn_perm(q01, l0, 0x07030206u), g + 7),
                          unfix((int)__builtin_amdgcn_perm(q23, l1, 0x05010004u), g + 7), unfix((int)__builtin_amdgcn_perm(q23, l1, 0x07030206u), g + 7)};
            if (nf)
              for (u64 em = __ballot(fgq == (u32)gq); em; em &= em - 1) {
                const int from = __builtin_ctzll(em);
                const u32 rel = ((u32)__builtin_amdgcn_readlane((int)fe.x, from) & 255u) - (u32)lane * 4u;
                const float vv = as_f((u32)__builtin_amdgcn_readlane((int)fe.y, from));
                #pragma unroll
                for (int r = 0; r < 4; ++r) y[r] = rel == (u32)r ? vv : y[r];
              }
            ST128((u32x4{as_u(y[0] * scale), as_u(y[1] * scale), as_u(y[2] * scale), as_u(y[3] * scale)}), rsO, olane + ni * 64, obase + mi * 16 * NN * 4, AUX_SYS);
          }
        }
      } else {
        landed(endp);
        ctl_st(C_REL + (n % NSLOT) * 4, n + 1);
        #pragma unroll
        for (int mi = 0; mi < 4; ++mi) {
          u32x4 r4[4];
          #pragma unroll
          for (int ni = 0; ni < 4; ++ni) r4[ni] = __builtin_amdgcn_raw_buffer_load_b128(rsP, lane * 16 + ni * 1024, par + EX_OUR + uid * OU_RAW + mi * 4096, AUX_SYS);
          #pragma unroll
          for (int ni = 0; ni < 4; ++ni) {
            const f32x4 y = __builtin_bit_cast(f32x4, r4[ni]) * scale;
            ST128(__builtin_bit_cast(u32x4, y), rsO, olane + ni * 64, obase + mi * 16 * NN * 4, AUX_SYS);
          }
        }
      }
    }
    return;
  }

  constexpr int SLOTS = NGEMM / 8;
  const int xc = b & 7, slot = b >> 3, wm = wave >> 1, wn = wave & 1;
  for (int ti = slot; ti < 36; ti += SLOTS) {
    const int j = xc >> 1, ft = ti * 2 + (xc & 1);
    const int c = ti < 24 ? ft / 6 : 4 * (xc & 1) + (ti < 28 ? ti - 24 : (ti - 28) >> 1), cb = j * 8 + c;
    auto row_group = [&](int k) -> int {
      if (ti >= 28) return 8 * rank + 4 * (ti & 1) + k;
      if (ti >= 24) return 8 * ((~rank & 4) | ((j + 1 + min(k, 2)) & 3)) + 3;
      const int t = ti % 3;
      return 8 * (xc & 1 ? (~rank & 4) | ((j + 1 + t) & 3) : (rank & 4) | ((rank + 1 + t) & 3)) + 4 * (xc & 1) + k;
    };
    const int rgw = row_group(wm);
    const u32 deadw = (u32)(uintptr_t)(smem + L_CTL);
    auto ready = [&](int grp) {
      const bool isx = lane < 4;
      const int rgl = row_group(lane & 3);
      const u32 off = OFF_CNT + ((isx ? rgl : 64 + cb) * 5 + grp) * 128, want = (grp == 4 ? (isx ? 6u : 8u) : (isx ? 7u : 14u)) * call;
      if (!wait_ge(p.ws, off, want, call)) ctl_st(deadw, 1u);
    };
    const u32 xo0 = OFF_XA + rgw * IU_RAW + wn * 4096, wo0 = OFF_WA + cb * 2 * IU_RAW + wave * 2048;
    auto dma = [&](int k, int ks, u32 st) {
      const u32 xo = xo0 + ks * (64 * IU_RAW), wo = wo0 + ks * (64 * IU_RAW);
      LDS char* dx = smem + st + wave * 4096;
      LDS char* dw = smem + st + 32 * KiB + wave * 2048;
      if (k == 0) __builtin_amdgcn_raw_ptr_buffer_load_lds(rsL, (LDS void*)dx, 16, lane * 16, xo, 0, AUX_SYS);
      if (k == 1) __builtin_amdgcn_raw_ptr_buffer_load_lds(rsL, (LDS void*)dx, 16, lane * 16, xo, 1024, AUX_SYS);
      if (k == 2) __builtin_amdgcn_raw_ptr_buffer_load_lds(rsL, (LDS void*)dx, 16, lane * 16, xo, 2048, AUX_SYS);
      if (k == 3) __builtin_amdgcn_raw_ptr_buffer_load_lds(rsL, (LDS void*)dx, 16, lane * 16, xo, 3072, AUX_SYS);
      if (k == 4) __builtin_amdgcn_raw_ptr_buffer_load_lds(rsL, (LDS void*)dw, 16, lane * 16, wo, 0, AUX_SYS);
      if (k == 5) __builtin_amdgcn_raw_ptr_buffer_load_lds(rsL, (LDS void*)dw, 16, lane * 16, wo, 1024, AUX_SYS);
    };
    f32x4 acc[4][4];
    #pragma unroll
    for (int mi = 0; mi < 4; ++mi)
      #pragma unroll
      for (int ni = 0; ni < 4; ++ni) acc[mi][ni] = f32x4{0.f, 0.f, 0.f, 0.f};
    const bool twin = ti >= 24 && ti < 28;
    const int kr = twin ? (~rank & 4) | j : rank;
    if (wave == 0) {
      if (ti == slot) ctl_st(deadw, 0u);
      for (int grp = twin ? 0 : 4; grp < 5; ++grp) ready(grp);
    }
    asm volatile("s_waitcnt vmcnt(0) lgkmcnt(0)\n\ts_barrier" ::: "memory");
    #pragma unroll
    for (int k = 0; k < 12; ++k) dma(k % 6, kr * 4 + k / 6, k / 6 * STAGE);
    u32 st = 0;
    int ksn = kr * 4 + 2, jn = -2, kbn = 0;
    i32x8 fa[8], fb[8];
    auto step = [&](i32x8 (&cur)[8], const i32x8 (&prev)[8], bool first, bool more, bool last) {
      if (more && wave == 0 && jn == 0) ready(kbn);
      if (!last) asm volatile("s_waitcnt vmcnt(6) lgkmcnt(0)\n\ts_barrier" ::: "memory");
      else asm volatile("s_waitcnt vmcnt(0) lgkmcnt(0)\n\ts_barrier" ::: "memory");
      const u32 st2 = st == 0 ? 2 * STAGE : st - STAGE;
      LDS char* ax = smem + st + wm * 8192 + lane * 16;
      LDS char* aw = smem + st + 32 * KiB + wn * 8192 + lane * 16;
      u32x4 h[16];
      auto M = [&](int k) {
        if (!first) acc[k >> 2][k & 3] = __builtin_amdgcn_mfma_scale_f32_16x16x128_f8f6f4(prev[k & 3], prev[4 + (k >> 2)], acc[k >> 2][k & 3], 1, 1, 0, 0, 0, 0);
        __builtin_amdgcn_sched_barrier(0);
      };
      auto R = [&](int k) { h[k] = *(LDS u32x4*)(((k >> 1) & 1 ? ax : aw) + (k >> 2) * 2048 + (k & 1) * 1024); __builtin_amdgcn_sched_barrier(0); };
      __builtin_amdgcn_sched_barrier(0);
      #pragma unroll
      for (int k = 0; k < 6; ++k) {
        M(2 * k); R(2 * k); R(2 * k + 1); M(2 * k + 1);
        if (more) dma(k, ksn, st2);
        __builtin_amdgcn_sched_barrier(0);
      }
      M(12); R(12); R(13); M(13); R(14); R(15); M(14); M(15);
      #pragma unroll
      for (int f = 0; f < 4; ++f) {
        cur[f] = __builtin_bit_cast(i32x8, __builtin_shufflevector(h[4 * f], h[4 * f + 1], 0, 1, 2, 3, 4, 5, 6, 7));
        cur[4 + f] = __builtin_bit_cast(i32x8, __builtin_shufflevector(h[4 * f + 2], h[4 * f + 3], 0, 1, 2, 3, 4, 5, 6, 7));
      }
      if (++jn == 7) { jn = 0; ++kbn; }
      ksn = jn < 0 ? ksn + 1 : ((kr + 1 + jn) & 7) * 4 + kbn;
      st = st == 2 * STAGE ? 0 : st + STAGE;
    };
    step(fa, fb, true, true, false);
    #pragma unroll 1
    for (int s = 1; s < 29; s += 2) { step(fb, fa, false, true, false); step(fa, fb, false, true, false); }
    step(fb, fa, false, true, false);
    step(fa, fb, false, false, false);
    step(fb, fa, false, false, true);
    #pragma unroll
    for (int k = 0; k < 16; ++k)
      acc[k >> 2][k & 3] = __builtin_amdgcn_mfma_scale_f32_16x16x128_f8f6f4(fb[k & 3], fb[4 + (k >> 2)], acc[k >> 2][k & 3], 1, 1, 0, 0, 0, 0);
    int lz = lane; asm volatile("" : "+v"(lz));
    const bool dead = ctl_ld(deadw) != 0u;
    const int owner = rgw >> 3;
    if (owner == rank) {
      const u32 obase = ((rgw & 7) * 64 * NN + group_of(rank, j) * 1024 + c * 128 + wn * 64) * 4, olane = ((lz & 15) * NN + (lz >> 4) * 4) * 4;
      const float sc = dead ? __builtin_nanf("") : scale;
      #pragma unroll
      for (int mi = 0; mi < 4; ++mi)
        #pragma unroll
        for (int ni = 0; ni < 4; ++ni)
          ST128(__builtin_bit_cast(u32x4, acc[mi][ni] * sc), rsO, olane + ni * 64, obase + mi * 16 * NN * 4, AUX_SYS);
    } else if (!dead && !(ti >= 24 && wm == 3)) {
      const int t = (cb * 2 + wn) * 5 + (rgw & 7) - ((owner >> 2) == (rank >> 2) ? 0 : 3);
      const u32 uid = owner * 320 + t;
      const u32 offE = par + EX_OUE + uid * OU_ENC;
      float sa = 0.f;
      #pragma unroll
      for (int mi = 0; mi < 4; ++mi)
        #pragma unroll
        for (int ni = 0; ni < 4; ++ni) sa += __builtin_fabsf(acc[mi][ni][(mi + ni) & 3]);
      const int g = max(0, 24 - ((int)((as_u(wave_sum_f32(sa) * (4.2f / 1024.f)) >> 23) & 255u) - 126));
      LDS char* last1k = smem + wave * 4096;
      LDS char* side = last1k + 768;
      LDS char* pend = last1k + 1536;
      u32 npend = 0, sg[2] = {0u, 0u};
      auto quad = [&](int gq, u32& l0, u32& l1, u32& tp) {
        u32 qv[4], fr[4];
        #pragma unroll
        for (int r = 0; r < 4; ++r) {
          const float tt = __builtin_amdgcn_ldexpf(acc[gq >> 2][gq & 3][r], g + 7);
          qv[r] = (u32)f2i_sat(tt);
          fr[r] = as_u(__builtin_amdgcn_fractf(tt));
        }
        const u32 p01 = __builtin_amdgcn_perm(qv[1], qv[0], 0x07030400u), p23 = __builtin_amdgcn_perm(qv[3], qv[2], 0x07030400u);
        const u32 low = __builtin_amdgcn_perm(p23, p01, 0x05040100u);
        u32 miss;
        tp = top_pack(__builtin_amdgcn_perm(p23, p01, 0x07060302u), miss);
        l0 = __builtin_amdgcn_perm(qv[1], qv[0], 0x06050201u);
        l1 = __builtin_amdgcn_perm(qv[3], qv[2], 0x06050201u);
        sg[gq >> 3] = bfi(0x80808080u >> (gq & 7), low >> (gq & 7), sg[gq >> 3]);
        const bool odd_one = (fr[0] | fr[1] | fr[2] | fr[3] | (low & 0x7F7F7F7Fu) | (miss & 0x00380038u)) != 0u;
        const u64 bm = __ballot(odd_one);
        if (odd_one) {
          const u32 sl = min(63u, npend + __builtin_amdgcn_mbcnt_hi((u32)(bm >> 32), __builtin_amdgcn_mbcnt_lo((u32)bm, 0u)));
          *(LDS f32x4*)(pend + sl * 16) = acc[gq >> 2][gq & 3];
          *(LDS u32*)(pend + 1024 + sl * 4) = gq * 64 + lz;
        }
        npend += __popcll(bm);
      };
      u32 z0, z1, t15;
      quad(15, z0, z1, t15);
      *(LDS u32*)(last1k + 256 + lz * 4) = z0;
      *(LDS u32*)(last1k + 512 + lz * 4) = z1;
      #pragma unroll
      for (int mi = 0; mi < 4; ++mi) {
        u32 pk[12];
        #pragma unroll
        for (int ni = 0; ni < 4; ++ni) {
          const int gq = mi * 4 + ni;
          if (gq == 15) break;
          quad(gq, pk[ni * 3], pk[ni * 3 + 1], pk[ni * 3 + 2]);
          pk[ni * 3 + 2] = bfi(0x40004000u, gq < 14 ? t15 << (14 - gq) : t15 >> 1, pk[ni * 3 + 2]);
        }
        #pragma unroll
        for (int j = 0; j < (mi < 3 ? 3 : 2); ++j)
          ST128((u32x4{pk[j * 4], pk[j * 4 + 1], pk[j * 4 + 2], pk[j * 4 + 3]}), rsL, lz * 16 + j * 1024, offE + mi * 3072, AUX_SYS);
        if (mi == 3) *(LDS u32*)(last1k + lz * 4) = pk[8];
      }
      u32 ns = npend > 64u ? OS_CAP + 1 : 0u, nf = 0;
      auto put = [&](bool on, u32 e) {
        const u64 bm = __ballot(on);
        const u32 sl = ns + __builtin_amdgcn_mbcnt_hi((u32)(bm >> 32), __builtin_amdgcn_mbcnt_lo((u32)bm, 0u));
        if (on && sl < OS_CAP) *(LDS unsigned short*)(side + (sl < OS_CAP0 ? 196 : 256 - OS_CAP0 * 2) + sl * 2) = (unsigned short)e;
        ns += __popcll(bm);
      };
      u64 om = ((u64)sg[1] << 32) | sg[0];
      u32 odd = om ? 0u : 1u, last = 0;
      #pragma unroll
      for (int k = 0; k < 4; ++k) { if (om) last = (u32)__builtin_ctzll(om); om &= om - 1; odd |= last << (6 * k); }
      for (; __any(om != 0) && ns <= OS_CAP; om &= om - 1) {
        const u32 e = (u32)__builtin_ctzll(om | (1ull << 63));
        put(om != 0, (((e >> 5) * 8 + 7 - (e & 7)) << 11) | (lz << 5) | (e & 24u) | 4u);
      }
      wave_sync();
      for (u32 p0 = 0; p0 < min(npend, 64u); p0 += 16) {
        const u32 pi = p0 + (lz >> 2), r = lz & 3;
        const bool on = pi < npend;
        const float v = *(LDS float*)(pend + p0 * 16 + lz * 4);
        const u32 pp = *(LDS u32*)(pend + 1024 + pi * 4);
        const float tt = __builtin_amdgcn_ldexpf(v, g + 7);
        const u32 iv = (u32)f2i_sat(tt);
        const bool full = on && (as_u((float)(int)iv) != as_u(tt) || (iv & 31u));
        const u32 top = __builtin_amdgcn_perm((u32)__builtin_amdgcn_mov_dpp((int)iv, 0xB1, 0xf, 0xf, true), iv, 0x0C0C0703u);
        u32 miss;
        const u32 fl = (top_unpack(top_pack(top, miss)) ^ top) >> 5, d4 = (fl & 3u) | ((fl >> 6) & 12u);
        put(on && !full && (iv & 96u), (pp << 5) | (r << 3) | ((iv >> 5) & 3u));
        put(on && !(r & 1) && d4, 0x8000u | (pp << 5) | (r << 3) | d4);
        const u64 bm = __ballot(full);
        if (bm) {
          const u32 sl = nf + __builtin_amdgcn_mbcnt_hi((u32)(bm >> 32), __builtin_amdgcn_mbcnt_lo((u32)bm, 0u));
          if (full && sl < OF_CAP0) { *(LDS unsigned short*)(side + 244 + sl * 2) = (unsigned short)(pp * 4 + r); *(LDS u32*)(side + 248 + sl * 4) = as_u(v); }
          else if (full && sl < OF_CAP) *(LDS u32x2*)(side + 384 - OF_CAP0 * 8 + sl * 8) = u32x2{pp * 4 + r, as_u(v)};
          nf += __popcll(bm);
        }
      }
      const bool raw = ns > OS_CAP || nf > OF_CAP;
      *(LDS unsigned short*)(side + lz * 2) = (unsigned short)odd;
      *(LDS unsigned char*)(side + 128 + lz) = (unsigned char)(odd >> 16);
      if (lz == 0) *(LDS u32*)(side + 192) = raw ? 0x80000000u : (ns | (nf << 8) | ((u32)(g + 256) << 16));
      wave_sync();
      ST128(*(LDS u32x4*)(last1k + lz * 16), rsL, lz * 16, offE + 11264, AUX_SYS);
      if ((ns > OS_CAP0 || nf > OF_CAP0) && lz < 24) ST128(*(LDS u32x4*)(last1k + 1024 + lz * 16), rsL, lz * 16, offE + 12288, AUX_SYS);
      if (raw) {
        const u32 offR = par + EX_OUR + uid * OU_RAW;
        #pragma unroll
        for (int mi = 0; mi < 4; ++mi)
          #pragma unroll
          for (int ni = 0; ni < 4; ++ni)
            ST128(__builtin_bit_cast(u32x4, acc[mi][ni]), rsL, lz * 16 + ni * 1024, offR + mi * 4096, AUX_SYS);
      }
      vm_drain();
      if (lz == owner)
        __hip_atomic_store((GLB u32*)(peer_ws + OFF_FOUT + (rank * 320 + t) * 128), call, __ATOMIC_RELAXED, __HIP_MEMORY_SCOPE_SYSTEM);
    }
  }
}

}

extern "C" void kernel_launch(int rank, int nranks, ncclComm_t comm, hipStream_t stream,
                              void* const* d_in, const int* in_sizes, int n_in,
                              void* d_out, int out_size,
                              const int* in_dtypes, int out_dtype,
                              void* const* d_out_all,
                              void* const* d_ws_all, size_t ws_bytes) {
  (void)comm; (void)in_sizes; (void)n_in; (void)out_size; (void)in_dtypes; (void)out_dtype; (void)d_out_all;
  if (nranks != NR || (unsigned)rank >= (unsigned)NR || ws_bytes < WS_NEED) { fprintf(stderr, "unexpected launch shape\n"); return; }
  static unsigned attr_set = 0;
  if (!(attr_set >> rank & 1u)) {
    HIP_CHECK(hipFuncSetAttribute((const void*)gemm_rs_kernel, hipFuncAttributeMaxDynamicSharedMemorySize, (int)LDS_BYTES));
    attr_set |= 1u << rank;
  }
  Params p;
  p.x = (const uint8_t*)d_in[0]; p.w = (const uint8_t*)d_in[1]; p.sx = (const float*)d_in[2]; p.sw = (const float*)d_in[3];
  p.out = (float*)d_out; p.ws_all = (char* const*)d_ws_all; p.ws = (char*)d_ws_all[rank]; p.rank = rank; p.ws_bytes = WS_NEED;
  hipLaunchKernelGGL(gemm_rs_kernel, dim3(NWG), dim3(NTHR), LDS_BYTES, stream, p);
  HIP_CHECK(hipGetLastError());
}
```

```cpp
#include <hip/hip_runtime.h>
#include <rccl/rccl.h>
#include <cstdio>
#include <cstdint>

#define HIP_CHECK(expr) do { hipError_t _e = (expr); if (_e != hipSuccess) { \
    fprintf(stderr, "HIP error %s at %s:%d\n", hipGetErrorString(_e), __FILE__, __LINE__); \
    return; } } while (0)

namespace {

typedef unsigned u32;
typedef unsigned long long u64;
typedef unsigned u32x4 __attribute__((ext_vector_type(4)));
typedef unsigned u32x2 __attribute__((ext_vector_type(2)));
typedef int i32x2 __attribute__((ext_vector_type(2)));
typedef int i32x8 __attribute__((ext_vector_type(8)));
typedef float f32x4 __attribute__((ext_vector_type(4)));
typedef __amdgpu_buffer_rsrc_t rsrc_t;
#define LDS __attribute__((address_space(3)))
#define GLB __attribute__((address_space(1)))

constexpr int NR = 8, MM = 4096, NN = 16384, KS = 512;
constexpr int NWG = 256, NTHR = 512;
constexpr int PULL_WGS = 8;
constexpr int WIN = 8, WIN_OUT = 7;
constexpr u32 LOOK = 5, LOOK_OUT = 3;
constexpr int NRAW = 8, X1FIRST = 16;
constexpr int NPULL = 7 * PULL_WGS, NGEMM = NWG - NPULL;
constexpr int AUX_SYS = 17;
constexpr u64 DEADLINE_TICKS = 40000000ull;

constexpr u32 KiB = 1024, MiB = 1024 * 1024;
constexpr u32 OFF_CALL = 0;
constexpr u32 OFF_TMO = 32 * KiB;
constexpr u32 OFF_TIC = 33 * KiB;
constexpr u32 OFF_CNT = 40 * KiB;
constexpr u32 OFF_FIN = 128 * KiB;
constexpr u32 OFF_FOUT = OFF_FIN + 8 * 576 * 128;
constexpr u32 OFF_XA = 2 * MiB;
constexpr u32 OFF_WA = OFF_XA + 16 * MiB;
constexpr u32 OFF_PAR = OFF_WA + 16 * MiB;
constexpr u32 IU_ENC = 8 * KiB, IU_RAW = 8 * KiB;
constexpr u32 OU_ENC = 12672, OU_RAW = 16 * KiB;
constexpr u32 IU_SIDE = 6144, OU_SIDE = 12032;
constexpr u32 EX_INE = 0;
constexpr u32 EX_INR = EX_INE + 1280 * IU_ENC;
constexpr u32 EX_OUE = EX_INR + 1280 * IU_RAW;
constexpr u32 EX_OUR = EX_OUE + 2560 * OU_ENC;
constexpr u32 PAR_BYTES = EX_OUR + 2560 * OU_RAW;
constexpr u32 WS_NEED = OFF_PAR + 2 * PAR_BYTES;
static_assert(OFF_CNT + 96 * 5 * 128 <= OFF_FIN && OFF_FOUT + 8 * 320 * 128 <= OFF_XA, "flags");
static_assert(WS_NEED <= 257u * MiB, "workspace");
constexpr u32 QV = 5120, QV0 = 1024, QV_SEC = 24, QV_END = 1792, QV_ALL = 2304;
constexpr u32 Q_ESC = 6144, Q_TER = 6400, Q_TMAX = 1760, Q_PAT0 = 8, Q_PAT = 264;
constexpr u32 IN_CAP0 = 62, IN_CAP = 254;
constexpr u32 OS_CAP0 = 24, OS_CAP = 64, OF_CAP0 = 2, OF_CAP = 32;

constexpr u32 STAGE = 48 * KiB;
constexpr u32 L_SCR = 64 * KiB, SCR_W = 10752;
constexpr u32 NSLOT = 8, SLOT = 13 * KiB, NCONS = 6;
constexpr u32 L_PSCR = NSLOT * SLOT;
constexpr u32 L_CTL = L_PSCR + NCONS * 8 * KiB;
constexpr u32 LDS_BYTES = L_CTL + 256;
static_assert(3 * STAGE <= L_CTL && L_SCR + 8 * SCR_W <= LDS_BYTES && LDS_BYTES <= 160 * KiB, "lds");

__device__ __forceinline__ bool quick_form(int i) { return i < 256 ? (i & 3) < 2 : (i & 0x3B) == 0; }
__device__ __forceinline__ int group_of(int rank, int j) { return rank < 4 ? rank * 4 + j : j * 4 + (rank & 3); }

struct Params {
  const uint8_t* x; const uint8_t* w; const float* sx; const float* sw;
  float* out; char* const* ws_all; char* ws; int rank; u32 ws_bytes;
};

__device__ __forceinline__ rsrc_t mk_rsrc(const void* p, u32 bytes) {
  u64 a = (u64)p;
  u32 lo = __builtin_amdgcn_readfirstlane((u32)a), hi = __builtin_amdgcn_readfirstlane((u32)(a >> 32));
  return __builtin_amdgcn_make_buffer_rsrc((void*)(((u64)hi << 32) | lo), 0, (int)bytes, 0x00020000);
}
#define ST128(v, rs, voff, soff, aux) __builtin_amdgcn_raw_buffer_store_b128((v), (rs), (int)((u32)(voff) + (u32)(soff)), 0, (aux))
__device__ __forceinline__ float as_f(u32 v) { return __builtin_bit_cast(float, v); }
__device__ __forceinline__ u32 as_u(float v) { return __builtin_bit_cast(u32, v); }
__device__ __forceinline__ int f2i_sat(float v) {
  int r; asm("v_cvt_i32_f32 %0, %1" : "=v"(r) : "v"(v)); return r;
}
__device__ __forceinline__ void vm_drain() { asm volatile("s_waitcnt vmcnt(0)" ::: "memory"); }
__device__ __forceinline__ void wave_sync() {
  asm volatile("s_waitcnt lgkmcnt(0)" ::: "memory");
}
__device__ __forceinline__ u32 bfi(u32 m, u32 a, u32 b) { return __builtin_amdgcn_bitop3_b32(m, a, b, 0xCA); }
template <int CTRL, int ROWS> __device__ __forceinline__ u32 dpp0(u32 v) { return (u32)__builtin_amdgcn_update_dpp(0, (int)v, CTRL, ROWS, 0xf, false); }
__device__ __forceinline__ u32 wave_scan_u32(u32 v) {
  v += dpp0<0x111, 0xf>(v); v += dpp0<0x112, 0xf>(v); v += dpp0<0x114, 0xf>(v); v += dpp0<0x118, 0xf>(v);
  v += dpp0<0x142, 0xa>(v); v += dpp0<0x143, 0xc>(v);
  return v;
}
__device__ __forceinline__ float wave_sum_f32(float v) {
  v += as_f(dpp0<0x111, 0xf>(as_u(v))); v += as_f(dpp0<0x112, 0xf>(as_u(v))); v += as_f(dpp0<0x114, 0xf>(as_u(v)));
  v += as_f(dpp0<0x118, 0xf>(as_u(v))); v += as_f(dpp0<0x142, 0xa>(as_u(v))); v += as_f(dpp0<0x143, 0xc>(as_u(v)));
  return as_f((u32)__builtin_amdgcn_readlane((int)as_u(v), 63));
}
__device__ __forceinline__ u32 ctl_ld(u32 addr) {
  u32 v; asm volatile("ds_read_b32 %0, %1\n\ts_waitcnt lgkmcnt(0)" : "=&v"(v) : "v"(addr) : "memory");
  return (u32)__builtin_amdgcn_readfirstlane((int)v);
}
__device__ __forceinline__ void ctl_st(u32 addr, u32 v) { asm volatile("ds_write_b32 %0, %1" :: "v"(addr), "v"(v) : "memory"); }
__device__ __forceinline__ u32 ld_flag(const char* ws, u32 off) {
  return __hip_atomic_load((const GLB u32*)(ws + off), __ATOMIC_RELAXED, __HIP_MEMORY_SCOPE_SYSTEM);
}
__device__ __forceinline__ bool wait_ge(char* ws, u32 off, u32 target, u32 call, u32& got) {
  if (__all((got = ld_flag(ws, off)) >= target)) return true;
  const u64 t0 = __builtin_amdgcn_s_memrealtime();
  GLB u32* tmo = (GLB u32*)(ws + OFF_TMO);
  for (u32 it = 1;; ++it) {
    __builtin_amdgcn_s_sleep(2);
    if (__all((got = ld_flag(ws, off)) >= target)) return true;
    if ((it & 15u) == 0u &&
        (__hip_atomic_load(tmo, __ATOMIC_RELAXED, __HIP_MEMORY_SCOPE_AGENT) == call ||
         __builtin_amdgcn_s_memrealtime() - t0 > DEADLINE_TICKS)) {
      __hip_atomic_store(tmo, call, __ATOMIC_RELAXED, __HIP_MEMORY_SCOPE_AGENT);
      return false;
    }
  }
}
__device__ __forceinline__ void cnt_add(char* ws, u32 idx, int lane) {
  if (lane == 0) __hip_atomic_fetch_add((GLB u32*)(ws + OFF_CNT + idx * 128), 1u, __ATOMIC_RELAXED, __HIP_MEMORY_SCOPE_AGENT);
}

__device__ __forceinline__ void store_raw_unit(const u32x4 (&x)[8], rsrc_t rs, u32 off, int lane) {
  #pragma unroll
  for (int j = 0; j < 8; ++j) ST128(x[j], rs, lane * 16 + j * 1024, off, AUX_SYS);
}

__device__ __forceinline__ void export_half(const u32x4 (&x)[4], rsrc_t rs, u32 par, u32 uid, int hf, LDS char* scr, int lane) {
  const u32 offE = par + EX_INE + uid * IU_ENC;
  u32 s = 0;
  #pragma unroll
  for (int j = 0; j < 4; ++j) {
    *(LDS u32x4*)(scr + j * 1024 + lane * 16) = x[j];
    s = __builtin_amdgcn_sad_u8(x[j][j] & 0x7C7C7C7Cu, 0u, s);
  }
  s = (u32)__builtin_amdgcn_readlane((int)wave_scan_u32(s), 63);
  int e0 = ((int)s - 17121) >> 12;
  e0 = e0 < 0 ? 0 : (e0 > 24 ? 24 : e0);
  const u32 e4 = (u32)e0 * 0x04040404u;
  u32 mk[2] = {0, 0};
  u32 pk[12];
  #pragma unroll
  for (int j = 0; j < 4; ++j) {
    u32 o[4];
    #pragma unroll
    for (int c = 0; c < 4; ++c) {
      const u32 u = (x[j][c] | 0x80808080u) - e4;
      o[c] = bfi(0x1F1F1F1Fu, u, x[j][c]);
      const u32 t = (((u >> 1) ^ 0x40404040u) & 0x70707070u) + 0x70707070u;
      const int d = j * 4 + c;
      mk[d >> 3] = bfi(0x80808080u >> (d & 7), t >> (d & 7), mk[d >> 3]);
    }
    pk[j * 3 + 0] = bfi(0x60606060u, o[3] << 5, o[0]);
    pk[j * 3 + 1] = bfi(0x60606060u, o[3] << 3, o[1]);
    pk[j * 3 + 2] = bfi(0x60606060u, bfi(0x40404040u, o[3] >> 1, o[3] << 1), o[2]);
  }
  #pragma unroll
  for (int j = 0; j < 3; ++j)
    ST128((u32x4{pk[j * 4], pk[j * 4 + 1], pk[j * 4 + 2], pk[j * 4 + 3]}), rs, lane * 16 + j * 1024, offE + hf * 3072, AUX_SYS);
  const u32 mine = __popc(mk[0]) + __popc(mk[1]);
  const u32 inc = wave_scan_u32(mine);
  const u32 total = (u32)__builtin_amdgcn_readlane((int)inc, 63);
  if (total <= IN_CAP) {
    LDS unsigned short* list = (LDS unsigned short*)(scr + 8192);
    u32 slot = 2 + inc - mine;
    #pragma unroll
    for (int q = 0; q < 2; ++q) {
      u32 m = mk[q];
      while (m) {
        const int bp = __builtin_ctz(m); m &= m - 1;
        const int d = q * 8 + 7 - (bp & 7);
        list[slot++] = (unsigned short)((d >> 2) * 1024 + lane * 16 + (d & 3) * 4 + (bp >> 3));
      }
    }
    wave_sync();
    for (u32 j = 0; j < 2; ++j) {
      const u32 pp = *(LDS u32*)(scr + 8192 + j * 256 + lane * 4);
      const u32 p0 = pp & 4095u, p1 = (pp >> 16) & 4095u;
      const u32 b0 = *(LDS unsigned char*)(scr + p0), b1 = *(LDS unsigned char*)(scr + p1);
      u32 sd = (((p0 << 3) | ((128u + (b0 & 127u) - 4u * (u32)e0) >> 5) | (p1 << 19) | (((128u + (b1 & 127u) - 4u * (u32)e0) >> 5) << 16))) + hf * 0x80008000u;
      if (j == 0 && lane == 0) sd = total | ((u32)e0 << 16);
      const u32 at = j * 64 + lane;
      if (at < 32 || total > IN_CAP0)
        __builtin_amdgcn_raw_buffer_store_b32(sd, rs, at < 32 ? hf * 128 + at * 4 : 128 + hf * 384 + at * 4, offE + IU_SIDE, AUX_SYS);
      if (total <= 126) break;
    }
  } else {
    #pragma unroll
    for (int j = 0; j < 4; ++j) ST128(x[j], rs, lane * 16 + j * 1024, par + EX_INR + uid * IU_RAW + hf * 4096, AUX_SYS);
    if (lane == 0) __builtin_amdgcn_raw_buffer_store_b32(0x80000000u, rs, hf * 128, offE + IU_SIDE, AUX_SYS);
  }
}

__device__ __forceinline__ u32 export_short(const u32x4 (&x)[8], rsrc_t rs, u32 par, u32 uid, LDS char* scr, int lane) {
  typedef unsigned short u16x2 __attribute__((ext_vector_type(2)));
  const u32 offE = par + EX_INE + uid * IU_ENC, offR = par + EX_INR + uid * IU_RAW;
  LDS char* var = scr + 8192;
  LDS char* tab = var + QV_ALL;
  u32 s = 0;
  #pragma unroll
  for (int j = 0; j < 8; ++j) s = __builtin_amdgcn_sad_u8(x[j][j & 3] & 0x7C7C7C7Cu, 0u, s);
  #pragma unroll
  for (int j = 0; j < 7; ++j) *(LDS u32x4*)(scr + j * 1024 + lane * 16) = u32x4{0u, 0u, 0u, 0u};
  *(LDS u32x4*)(var + lane * 16) = u32x4{0u, 0u, 0u, 0u};
  *(LDS u32x4*)(var + 1024 + lane * 16) = u32x4{0u, 0u, 0u, 0u};
  {
    u32 sel = 0x0C0C0C0Cu, at = 0;
    #pragma unroll
    for (int y = 0; y < 4; ++y) if ((lane >> y) & 1) { sel = (sel & ~(0xFFu << at)) | ((u32)y << at); at += 8; }
    if (lane < 16) *(LDS u32*)(tab + lane * 4) = sel;
  }
  s = (u32)__builtin_amdgcn_readlane((int)wave_scan_u32(s), 63);
  const int K = max(5, min(31, ((int)s >> 13) + 3));
  const u32 kw = (128u + 4u * (u32)K) * 0x01010101u, tabo = (u32)(uintptr_t)tab;
  auto front = [&](u32 F) { return *(LDS u32*)(uintptr_t)__builtin_amdgcn_udot4(F, 0x20100804u, tabo, false); };
  u32 cs[32], cn[32], car[20], mine = 0;
  wave_sync();
  #pragma unroll
  for (int g = 0; g < 4; ++g) {
    u32 a[8], uu[8];
    #pragma unroll
    for (int k = 0; k < 8; ++k) {
      const int d = g * 8 + k;
      const u32 xv = x[d >> 2][d & 3];
      const u32 u = uu[k] = (kw - (xv & 0x7C7C7C7Cu)) ^ 0x80808080u;
      const u32 esc = (((u >> 1) & 0x78787878u) + 0x78787878u) | ~((u | (u << 1)) << 4);
      const u32 F = (esc >> 7) & 0x01010101u;
      a[k] = bfi(0x0C0C0C0Cu, ~u | __builtin_bit_cast(u32, __builtin_bit_cast(u16x2, F) * (u16x2){12, 12}), xv);
      cs[d] = front(F);
      cn[d] = mine;
      mine = __builtin_amdgcn_udot4(F, 0x01010101u, mine, false);
    }
    car[g * 5 + 0] = bfi(0x70707070u, a[5] << 4, a[0]);
    car[g * 5 + 1] = bfi(0x70707070u, a[6] << 4, a[1]);
    car[g * 5 + 2] = bfi(0x70707070u, a[7] << 4, a[2]);
    car[g * 5 + 3] = bfi(0x40404040u, a[7] << 3, bfi(0x20202020u, a[6] << 2, bfi(0x10101010u, a[5] << 1, a[3])));
    car[g * 5 + 4] = bfi(0x40404040u, a[7] >> 1, bfi(0x20202020u, a[6] >> 2, bfi(0x10101010u, a[5] >> 3, a[4])));
    __builtin_amdgcn_sched_barrier(0);
    #pragma unroll
    for (int k = 0; k < 8; ++k) cs[g * 8 + k] = __builtin_amdgcn_perm(0u, uu[k], cs[g * 8 + k]);
  }
  #pragma unroll
  for (int j = 0; j < 5; ++j) ST128((u32x4{car[j * 4], car[j * 4 + 1], car[j * 4 + 2], car[j * 4 + 3]}), rs, lane * 16 + j * 1024, offE, AUX_SYS);
  const u32 inc = wave_scan_u32(mine);
  const u32 nesc = (u32)__builtin_amdgcn_readlane((int)inc, 63);
  bool fits = nesc <= Q_ESC;
  u32 lines = 1;
  if (fits) {
    const u32 base = inc - mine;
    #pragma unroll
    for (int d = 0; d < 32; ++d) {
      const u32 q = base + cn[d];
      const u64 v = (u64)cs[d] << ((q & 3u) * 8u);
      LDS u32* at = (LDS u32*)(scr + (q & ~3u));
      __hip_atomic_fetch_or(at, (u32)v, __ATOMIC_RELAXED, __HIP_MEMORY_SCOPE_WORKGROUP);
      __hip_atomic_fetch_or(at + 1, (u32)(v >> 32), __ATOMIC_RELAXED, __HIP_MEMORY_SCOPE_WORKGROUP);
    }
    wave_sync();
    const u32 t0 = (QV_SEC + (nesc + 31) / 32 * 8) * 8;
    u32 tbit = t0, npat = 0, nter = 0;
    for (u32 r0 = 0; r0 < nesc && fits; r0 += 2048) {
      const u32x4 e0 = *(LDS u32x4*)(scr + r0 + lane * 32), e1 = *(LDS u32x4*)(scr + r0 + lane * 32 + 16);
      u32 two[2] = {0u, 0u}, zq[8], zl[8], fs[8], t3[8], mk = 0, bits = 0, nt = 0;
      #pragma unroll
      for (int k = 0; k < 8; ++k) {
        const u32 u = k < 4 ? e0[k & 3] : e1[k & 3], sel = (u >> 2) & 0x07070707u;
        const u32 far = ((((u >> 1) & 0x70707070u) + 0x70707070u) >> 7) & 0x01010101u;
        const u32 c2 = __builtin_amdgcn_perm(0x03030201u, 0x03030300u, sel) | far | (far << 1);
        const u32 F = c2 & (c2 >> 1) & 0x01010101u;
        two[k >> 2] |= __builtin_amdgcn_udot4(c2, 0x40100401u, 0u, false) << (8 * (k & 3));
        fs[k] = front(F); t3[k] = ((sel + 0x02020202u) & 0x07070707u) + 0x01010101u;
        nt = __builtin_amdgcn_udot4(F, 0x01010101u, nt, false);
        const u32 lo7 = u & 0x7F7F7F7Fu;
        const u32 out = (F << 7) & (~(lo7 + 0x68686868u) | (lo7 + 0x48484848u) | u);
        mk |= out >> k;
      }
      __builtin_amdgcn_sched_barrier(0);
      #pragma unroll
      for (int k = 0; k < 8; ++k) {
        const u32 cum = __builtin_amdgcn_perm(0u, t3[k], fs[k]) * 0x01010101u;
        zl[k] = (1u << (cum & 31u)) | (1u << ((cum >> 8) & 31u)) | (1u << ((cum >> 16) & 31u)) | (1u << ((cum >> 24) & 31u));
        zq[k] = bits | ((cum >> 29) << 16);
        bits += cum >> 24;
      }
      if (r0 + lane * 32 < nesc) *(LDS u32x2*)(var + QV_SEC + r0 / 4 + lane * 8) = u32x2{two[0], two[1]};
      const u32 np = __popc(mk), inc2 = wave_scan_u32(bits | (np << 16)), tot = (u32)__builtin_amdgcn_readlane((int)inc2, 63);
      nter += (u32)__builtin_amdgcn_readlane((int)wave_scan_u32(nt), 63);
      fits = tbit + (tot & 0xFFFFu) + 96 <= QV_END * 8 && npat + (tot >> 16) <= Q_PAT && nter <= Q_TMAX;
      if (!fits) break;
      const u32 b0 = tbit + (inc2 & 0xFFFFu) - bits;
      if (bits) {
        #pragma unroll
        for (int k = 0; k < 8; ++k) {
          const u32 q = b0 + (zq[k] & 0xFFFFu);
          const u64 v = (((u64)(zq[k] >> 16) << 32) | zl[k]) << (q & 31u);
          LDS u32* at = (LDS u32*)(var + ((q >> 5) << 2));
          __hip_atomic_fetch_or(at, (u32)v, __ATOMIC_RELAXED, __HIP_MEMORY_SCOPE_WORKGROUP);
          __hip_atomic_fetch_or(at + 1, (u32)(v >> 32), __ATOMIC_RELAXED, __HIP_MEMORY_SCOPE_WORKGROUP);
        }
      }
      u32 slot = npat + (inc2 >> 16) - np;
      for (u32 m = mk; m; m &= m - 1) {
        const int bp = __builtin_ctz(m);
        const u32 sn = r0 + lane * 32 + (7 - (bp & 7)) * 4 + (bp >> 3);
        const int v = (int)*(LDS signed char*)(scr + sn) >> 2;
        *(LDS unsigned short*)(var + (slot < Q_PAT0 ? 8 + slot * 2 : QV_END - Q_PAT0 * 2 + slot * 2)) = (unsigned short)((sn << 3) | (u32)(((v - 6) >> 3) + 4));
        ++slot;
      }
      tbit += tot & 0xFFFFu; npat += tot >> 16;
    }
    if (fits) {
      if (lane == 0) *(LDS u32x2*)var = u32x2{nesc | (npat << 13) | ((u32)K << 24), tbit - t0 + 1};
      lines = tbit / 1024 + 1;
      wave_sync();
      if ((u32)lane < lines * 8) ST128(*(LDS u32x4*)(var + lane * 16), rs, lane * 16, offE + QV, AUX_SYS);
      if (lines > 8 && lane < (int)(QV_END - QV0) / 16) ST128(*(LDS u32x4*)(var + QV0 + lane * 16), rs, lane * 16, offE + QV + QV0, AUX_SYS);
      if (npat > Q_PAT0 && lane < (int)(QV_ALL - QV_END) / 16) ST128(*(LDS u32x4*)(var + QV_END + lane * 16), rs, lane * 16, offE + QV + QV_END, AUX_SYS);
    }
  }
  if (!fits) {
    store_raw_unit(x, rs, offR, lane);
    if (lane == 0) __builtin_amdgcn_raw_buffer_store_b32(0x80000000u, rs, 0, offE + QV, AUX_SYS);
  }
  return min(lines, 8u);
}

__device__ __forceinline__ void patch_short(LDS char* scr, u32 ent) {
  const u32 pos = ent >> 3;
  __hip_atomic_fetch_add((LDS u32*)(scr + (pos & ~3u)), ((4u - (ent & 7u)) << 5) << ((pos & 3u) * 8u), __ATOMIC_RELAXED, __HIP_MEMORY_SCOPE_WORKGROUP);
}
__device__ __forceinline__ void patch_in(LDS char* scr, u32 ent) {
  const u32 pos = ent >> 3;
  __hip_atomic_fetch_add((LDS u32*)(scr + (pos & ~3u)), (((ent & 7u) - 4u) << 5) << ((pos & 3u) * 8u), __ATOMIC_RELAXED, __HIP_MEMORY_SCOPE_WORKGROUP);
}

__device__ __forceinline__ float unfix(int i, int g) { return __builtin_amdgcn_ldexpf((float)i, -g); }
__device__ __forceinline__ u32 top_pack(u32 t, u32& miss) {
  const u32 m = __builtin_amdgcn_perm(0x00010208u, 0x08020100u, (t >> 5) & 0x07070707u);
  miss = __umul24(m >> 8, 3u) + m;
  return bfi(0x20002000u, miss << 11, bfi(0x00600060u, miss << 5, t));
}
__device__ __forceinline__ u32 top_unpack(u32 x) {
  const u32 r = __builtin_amdgcn_perm(0x24044222u, 0x02402000u, bfi(0x00030003u, x >> 5, x >> 11) & 0x00070007u);
  const u32 u = bfi(0x60006000u, r << 12, bfi(0x00600060u, r, x));
  return __builtin_amdgcn_bitop3_b32(__builtin_amdgcn_bitop3_b32(u, x >> 1, 0x40404040u, 0x78), x >> 2, 0x20202020u, 0x78);
}

__global__ void __launch_bounds__(NTHR, 2) gemm_rs_kernel(Params p) {
  extern __shared__ __attribute__((aligned(16))) char smem_[];
  LDS char* smem = (LDS char*)smem_;
  const int tid = threadIdx.x, lane = tid & 63, wave = __builtin_amdgcn_readfirstlane(tid >> 6);
  const int b = blockIdx.x, rank = p.rank;
  const int lg = lane >> 4, li = lane & 15;
  const rsrc_t rsL = mk_rsrc(p.ws, p.ws_bytes);
  const u32 tick = (u32)__builtin_amdgcn_raw_ptr_buffer_atomic_add_i32(1, rsL, lane == 0 ? 0 : (int)0xF0000000u, OFF_CALL + b * 128 + wave * 16, 0);
  char* const peer_ws = p.ws_all[lane & 7];
  u32 call;

  if (b < 128) {
    const rsrc_t rsW = mk_rsrc(p.w, KS * NN);
    __builtin_amdgcn_s_sleep(12);
    #pragma unroll
    for (int i = 0; i < 8; ++i) {
      const int row = (wave * 8 + i) * 8 + (lane >> 3);
      const int sw = ((row >> 1) & 3) | (((row >> 4) & 1) << 2);
      __builtin_amdgcn_raw_ptr_buffer_load_lds(rsW, (LDS void*)(smem + (wave * 8 + i) * 1024), 16, row * NN + (((lane & 7) ^ sw) << 4), b * 128, 0, 0);
    }
    asm volatile("s_waitcnt vmcnt(0)\n\ts_barrier" ::: "memory");
    call = __builtin_amdgcn_readfirstlane(tick) + 1u;
    const int cgl = wave & 1, kb = wave >> 1;
    u32x4 x[8];
    #pragma unroll
    for (int f = 0; f < 4; ++f)
      #pragma unroll
      for (int h = 0; h < 2; ++h)
        #pragma unroll
        for (int t = 0; t < 2; ++t) {
          const int row = kb * 128 + 64 * h + 16 * lg + 8 * t + (li >> 1);
          const int sw = ((row >> 1) & 3) | (((row >> 4) & 1) << 2);
          const i32x2 v = __builtin_amdgcn_ds_read_tr8_b64_v2i32((LDS i32x2*)(smem + row * 128 + (((cgl * 4 + f) ^ sw) << 4) + (li & 1) * 8));
          x[f * 2 + h][t * 2] = v.x; x[f * 2 + h][t * 2 + 1] = v.y;
        }
    const int gl = b >> 5, gr = 4 + ((b >> 3) & 3), c = b & 7;
    u32 lines = 0;
    if (kb == 0 && c == 0) {
      #pragma unroll 1
      for (int hf = 0; hf < 2; ++hf) {
        const u32x4 xh[4] = {hf ? x[4] : x[0], hf ? x[5] : x[1], hf ? x[6] : x[2], hf ? x[7] : x[3]};
        export_half(xh, rsL, OFF_PAR + (call & 1) * PAR_BYTES, 256 + (b * 2 + cgl) * 4, hf, smem + L_SCR + wave * SCR_W, lane);
      }
    } else lines = export_short(x, rsL, OFF_PAR + (call & 1) * PAR_BYTES, 256 + (b * 2 + cgl) * 4 + kb, smem + L_SCR + wave * SCR_W, lane);
    vm_drain();
    if ((lane == gl || lane == gr) && lane != rank) {
      const int cb = (lane == gl ? gr & 3 : gl) * 8 + c;
      __hip_atomic_store((GLB u32*)(peer_ws + OFF_FIN + (rank * 576 + 256 + (cb * 2 + cgl) * 4 + kb) * 128), call << 4 | lines, __ATOMIC_RELAXED, __HIP_MEMORY_SCOPE_SYSTEM);
    }
    if (rank == gl || rank == gr) {
      const int cb = (rank == gl ? gr & 3 : gl) * 8 + c;
      store_raw_unit(x, rsL, OFF_WA + ((rank * 4 + kb) * 64 + cb * 2 + cgl) * IU_RAW, lane);
      vm_drain();
      cnt_add(p.ws, (64 + cb) * 5 + 4, lane);
    }
  } else if (b < 192) {
    const int rg = b - 128, kb = wave & 3, hf = wave >> 2, u = rg * 4 + kb;
    const rsrc_t rsX = mk_rsrc(p.x, MM * KS);
    const u32 both = (u32)(uintptr_t)(smem + L_CTL) + 16;
    const u32 xoff = OFF_XA + ((rank * 4 + kb) * 64 + rg) * IU_RAW;
    const int ow = rg >> 3, r8 = rg & 7;
    const u32 users = ((r8 <= 3 ? 0xFu << (ow & 4) : 0u) | (r8 >= 3 ? 0xFu << (~ow & 4) : 0u) | (1u << ow)) & ~(1u << rank);
    const bool tell = lane < NR && ((users >> lane) & 1);
    if (kb < 2) {
      u32x4 x[4];
      #pragma unroll
      for (int j = 0; j < 4; ++j)
        x[j] = __builtin_amdgcn_raw_buffer_load_b128(rsX, ((j >> 1) * 16 + li) * KS + (j & 1) * 64 + lg * 16, (rg * 64 + hf * 32) * KS + kb * 128, 0);
      if (wave == 0) ctl_st(both + (lane & 3) * 4, 0u);
      asm volatile("s_waitcnt lgkmcnt(0)\n\ts_barrier" ::: "memory");
      call = __builtin_amdgcn_readfirstlane(tick) + 1u;
      const u32 parw = OFF_PAR + (call & 1) * PAR_BYTES;
      auto second = [&](int k) -> bool {
        u32 old = 0;
        if (lane == 0) old = __hip_atomic_fetch_add((LDS u32*)(uintptr_t)(both + k * 4), 1u, __ATOMIC_RELAXED, __HIP_MEMORY_SCOPE_WORKGROUP);
        return __builtin_amdgcn_readfirstlane((int)old) == 1;
      };
      if (NRAW && kb == 0) {
        #pragma unroll
        for (int j = 0; j < 4; ++j) ST128(x[j], rsL, lane * 16 + j * 1024, parw + EX_INR + u * IU_RAW + hf * 4096, AUX_SYS);
        vm_drain();
        if (second(2) && tell) __hip_atomic_store((GLB u32*)(peer_ws + OFF_FIN + (rank * 576 + 512 + rg) * 128), call << 4, __ATOMIC_RELAXED, __HIP_MEMORY_SCOPE_SYSTEM);
      }
      export_half(x, rsL, parw, u, hf, smem + L_SCR + wave * SCR_W, lane);
      vm_drain();
      if (second(kb) && tell) __hip_atomic_store((GLB u32*)(peer_ws + OFF_FIN + (rank * 576 + u) * 128), call << 4, __ATOMIC_RELAXED, __HIP_MEMORY_SCOPE_SYSTEM);
      #pragma unroll
      for (int j = 0; j < 4; ++j) ST128(x[j], rsL, lane * 16 + j * 1024, xoff + hf * 4096, AUX_SYS);
      vm_drain();
      cnt_add(p.ws, rg * 5 + 4, lane);
    } else if (hf == 0) {
      u32x4 x[8];
      #pragma unroll
      for (int j = 0; j < 8; ++j)
        x[j] = __builtin_amdgcn_raw_buffer_load_b128(rsX, ((j >> 1) * 16 + li) * KS + (j & 1) * 64 + lg * 16, rg * 64 * KS + kb * 128, 0);
      asm volatile("s_barrier" ::: "memory");
      call = __builtin_amdgcn_readfirstlane(tick) + 1u;
      const u32 lines = export_short(x, rsL, OFF_PAR + (call & 1) * PAR_BYTES, u, smem + L_SCR + wave * SCR_W, lane);
      vm_drain();
      if (tell) __hip_atomic_store((GLB u32*)(peer_ws + OFF_FIN + (rank * 576 + u) * 128), call << 4 | lines, __ATOMIC_RELAXED, __HIP_MEMORY_SCOPE_SYSTEM);
      store_raw_unit(x, rsL, xoff, lane);
      vm_drain();
      cnt_add(p.ws, rg * 5 + 4, lane);
    } else {
      asm volatile("s_barrier" ::: "memory");
    }
  }
  call = __builtin_amdgcn_readfirstlane(tick) + 1u;
  const u32 par = OFF_PAR + (call & 1) * PAR_BYTES;
  const float scale = p.sx[0] * p.sw[0];
  const rsrc_t rsO = mk_rsrc(p.out, 512u * NN * 4u);

  if (b >= NGEMM) {
    const int pb = b - NGEMM, link = pb / PULL_WGS, q = pb % PULL_WGS;
    const int src = (rank + 1 + link) & 7;
    const u64 pw = (u64)peer_ws;
    const u32 pw_lo = (u32)__builtin_amdgcn_readlane((int)(u32)pw, src), pw_hi = (u32)__builtin_amdgcn_readlane((int)(u32)(pw >> 32), src);
    const rsrc_t rsP = mk_rsrc((const void*)(((u64)pw_hi << 32) | (u64)pw_lo), p.ws_bytes);
    const u32 ctl = (u32)(uintptr_t)(smem + L_CTL);
    const u32 C_READY = ctl, C_LANDED = ctl + 4, C_CLOSED = ctl + 12, C_REL = ctl + 16, C_ISSUED = ctl + 8, C_Q = ctl + 64;
    const bool same = (src >> 2) == (rank >> 2);
    if (wave == 0) ctl_st(ctl + lane * 4, 0u);
    asm volatile("s_waitcnt lgkmcnt(0)\n\ts_barrier" ::: "memory");
    if (wave == 0) {
      int g = 0, pub = 0;
      #define LANDED(v) do { if ((v) > pub) { pub = (v); ctl_st(C_LANDED, (u32)pub); } } while (0)
      #define PIECEW(W, SZ, VOFF, OFF) do { __builtin_amdgcn_raw_ptr_buffer_load_lds(rsP, (LDS void*)(dst + (OFF)), SZ, VOFF, so + (OFF), 0, AUX_SYS); \
        asm volatile("s_waitcnt vmcnt(%0)" :: "i"((W) - 1) : "memory"); ++g; LANDED(g - ((W) - 1)); } while (0)
      #define PIECE(SZ, VOFF, OFF) PIECEW(WIN, SZ, VOFF, OFF)
      for (u32 n = 0;; ++n) {
        if (ctl_ld(C_READY) <= n) {
          vm_drain(); LANDED(g);
          bool fin = false;
          while (ctl_ld(C_READY) <= n) { if (ctl_ld(C_CLOSED)) { fin = ctl_ld(C_READY) <= n; break; } __builtin_amdgcn_s_sleep(1); }
          if (fin) break;
        }
        const u32 ent = ctl_ld(C_Q + (n & 15) * 4);
        ctl_st(C_ISSUED, n + 1);
        if (ent >> 31) continue;
        const u32 need = n >= NSLOT ? n - NSLOT + 1 : 0, ra = C_REL + (n % NSLOT) * 4;
        if (ctl_ld(ra) < need) {
          vm_drain(); LANDED(g);
          while (ctl_ld(ra) < need) __builtin_amdgcn_s_sleep(1);
        }
        const int i = ent & 1023;
        LDS char* dst = smem + (n % NSLOT) * SLOT;
        if (ent & 1024) {
          const u32 so = par + EX_INR + i * IU_RAW;
          #pragma unroll
          for (int j = 0; j < 8; ++j) PIECE(16, lane * 16, j * 1024);
        } else if (i < 512 && quick_form(i)) {
          const u32 so = par + EX_INE + (i < 256 ? i : 256 + group_of(rank, (i >> 6) & 3) * 64 + (i & 63)) * IU_ENC;
          PIECE(4, lane * 4, IU_SIDE);
          #pragma unroll
          for (int j = 0; j < 6; ++j) PIECE(16, lane * 16, j * 1024);
        } else if (i < 512) {
          const u32 so = par + EX_INE + (i < 256 ? i : 256 + group_of(rank, (i >> 6) & 3) * 64 + (i & 63)) * IU_ENC;
          if ((u32)lane < ((ent >> 24) & 15u) * 8) __builtin_amdgcn_raw_ptr_buffer_load_lds(rsP, (LDS void*)(dst + QV), 16, lane * 16, so + QV, 0, AUX_SYS);
          asm volatile("s_waitcnt vmcnt(%0)" :: "i"(WIN - 1) : "memory"); ++g; LANDED(g - (WIN - 1));
          #pragma unroll
          for (int j = 0; j < 5; ++j) PIECE(16, lane * 16, j * 1024);
        } else {
          const u32 so = par + EX_OUE + (rank * 320 + i - 512) * OU_ENC;
          #pragma unroll
          for (int j = 0; j < 12; ++j) PIECEW(WIN_OUT, 16, lane * 16, (j + 11) % 12 * 1024);
        }
      }
      vm_drain(); LANDED(g);
      #undef PIECE
      #undef PIECEW
      #undef LANDED
      return;
    }
    if (wave == 1) {
      const u32 tic0 = OFF_TIC + (call & 1) * 24 * 128;
      const int only0 = lane == 0 ? 0 : (int)0xF0000000u;
      if (q == 0 && (lane == link || (link == 0 && lane >= 8 && lane < 24)))
        __hip_atomic_store((GLB u32*)(p.ws + OFF_TIC + ((~call & 1) * 24 + lane) * 128), 0u, __ATOMIC_RELAXED, __HIP_MEMORY_SCOPE_AGENT);
      constexpr u32 NPAIR = 208 + NRAW / 2, NFIX = 192;
      u32 n = 0, endp = 0; bool ok = true;
      int fj = -1, fnext = -1;
      const u32 gtic = tic0 + (8 + group_of(src, lane & 3)) * 128;
      auto fullest = [&](u32 v) -> int {
        v = v >= 16u || (!same && (lane & 3) == (rank & 3)) ? 0xFFFFu : (v << 4) | ((((lane - q) & 3) << 2) | (lane & 3));
        v = min(v, (u32)__builtin_amdgcn_mov_dpp((int)v, 0xB1, 0xf, 0xf, true));
        v = min(v, (u32)__builtin_amdgcn_mov_dpp((int)v, 0x4E, 0xf, 0xf, true));
        v = (u32)__builtin_amdgcn_readfirstlane((int)v);
        return v == 0xFFFFu ? -2 : (int)(v & 3u);
      };
      auto draw = [&](int g) { return __builtin_amdgcn_raw_ptr_buffer_atomic_add_i32(1, rsL, only0, tic0 + (g < 0 ? link : 8 + group_of(src, g)) * 128, 0); };
      int tnext = draw(-1);
      u32 seen = 0;
      for (;;) {
        u32 t = (u32)__builtin_amdgcn_readfirstlane(tnext);
        fj = fnext;
        if (t >= (fj < 0 ? NPAIR + NFIX : 16u)) {
          if ((fnext = fullest(ld_flag(p.ws, gtic))) < 0) break;
          tnext = draw(fnext);
          continue;
        }
        while (n >= ctl_ld(C_ISSUED) + (t < NPAIR && fj < 0 ? LOOK : LOOK_OUT)) __builtin_amdgcn_s_sleep(1);
        bool last = false;
        if (fj >= 0 || t + PULL_WGS >= NPAIR + NFIX) last = (fnext = fullest(seen)) < 0;
        if (last && fj < 0) { fnext = -1; last = false; }
        if (!last) tnext = draw(fnext);
        if (t >= NPAIR || fj >= 0) seen = ld_flag(p.ws, gtic);
        if (fj >= 0) t += NPAIR + NFIX;
        int i;
        if (t < NPAIR) {
          const int e = t < NRAW ? t : 2 * t - NRAW + (lane & 1);
          int u = -1, sg = 0, kb = 0;
          if (e < 32 + X1FIRST) { kb = e >= NRAW && e < NRAW + X1FIRST; u = e < NRAW ? e : kb ? e - NRAW : e - X1FIRST; }
          else if (e < 40 + X1FIRST) sg = 0;
          else if (e < 72) { kb = 1; u = e - 40; }
          else if (e < 88) sg = (e - 64) >> 3;
          else if (e < 120) { kb = 2; u = e - 88; }
          else if (e < 144) sg = (e - 96) >> 3;
          else if (e < 176) { kb = 3; u = e - 144; }
          else sg = (e - 128) >> 3;
          int c = 0;
          if (u < 0) {
            if (sg < 6) { c = (0x210100 >> (sg * 4)) & 7; kb = (0x012010 >> (sg * 4)) & 3; }
            else if (sg < 26) { c = ((sg - 6) >> 2) + ((sg - 6) & 3); kb = 3 - ((sg - 6) & 3); }
            else { c = (0x776765 >> ((sg - 26) * 4)) & 7; kb = (0x323123 >> ((sg - 26) * 4)) & 3; }
          }
          const int m = c * 8 + (e & 7);
          const int v = u - 12;
          if (e >= 384) i = (8 * rank + (e & 7)) * 4 + ((e - 384) >> 3);
          else if (u < 0) i = 256 + ((((m >> 1) & 3) * 8 + (m >> 3)) * 2 + (m & 1)) * 4 + kb;
          else if (u < 12) i = (8 * ((rank & 4) | ((rank + 1 + (u >> 2)) & 3)) + (u & 3)) * 4 + kb;
          else i = (8 * ((~rank & 4) | (v / 5)) + 3 + v % 5) * 4 + kb;
        } else if (fj < 0) {
          const int o = t - NPAIR, cm = same ? o / 3 : o >> 2, k = same ? o - cm * 3 : 1 + (o & 3);
          const int c = same ? cm >> 3 : cm / 6, j3 = (cm - c * 6) >> 1, j = same ? (cm >> 1) & 3 : j3 + (j3 >= (rank & 3));
          i = 512 + ((j * 8 + c) * 2 + (cm & 1)) * 5 + k;
        } else i = 512 + (fj * 16 + (int)(t - NPAIR - NFIX)) * 5 + (same ? 3 : 0);
        const bool unp = t < NRAW, two = !unp && t < NPAIR;
        const u32 fo = i >= 512 ? OFF_FOUT + (src * 320 + i - 512) * 128 : OFF_FIN + (src * 576 + (unp ? 512 + (i >> 2) : i)) * 128;
        u32 got = 0;
        ok = ok && wait_ge(p.ws, fo, call << 4, call, got);
        const u32 lines = got >> 4 == call ? got & 15u : 8u;
        const u32 k = two ? (lane & 1) : 0, per = !ok ? 0 : t >= NPAIR ? 12 : unp ? 8 : (u32)__builtin_amdgcn_readfirstlane(quick_form(i) ? 7 : 6);
        ctl_st(C_Q + ((n + k) & 15) * 4, (u32)i | (unp ? 1024u : 0u) | ((endp + per * (k + 1)) << 11) | (lines << 24) | (ok ? 0u : 0x80000000u));
        n += two ? 2 : 1; endp += two ? 2 * per : per;
        asm volatile("s_waitcnt lgkmcnt(0)" ::: "memory");
        ctl_st(C_READY, n);
        if (last) break;
      }
      asm volatile("s_waitcnt lgkmcnt(0)" ::: "memory");
      ctl_st(C_CLOSED, 1u);
      return;
    }
    LDS char* scr = smem + L_PSCR + (wave - 2) * 8 * KiB;
    for (u32 n = wave - 2;; n += NCONS) {
      bool fin = false;
      while (ctl_ld(C_READY) <= n) { if (ctl_ld(C_CLOSED)) { fin = ctl_ld(C_READY) <= n; break; } __builtin_amdgcn_s_sleep(1); }
      if (fin) break;
      const u32 ent = ctl_ld(C_Q + (n & 15) * 4);
      const bool ok = !(ent >> 31);
      const int i = ent & 1023;
      const u32 endp = (ent >> 11) & 0x1FFFu;
      auto landed = [&](u32 upto) { while (ctl_ld(C_LANDED) < upto) __builtin_amdgcn_s_sleep(1); };
      LDS char* sl = smem + (n % NSLOT) * SLOT;
      if (i < 512) {
        if (!ok) continue;
        const bool brief = !(ent & 1024) && !quick_form(i);
        landed(brief ? endp - 5 : endp);
        const int kb = i & 3, rg = (i >> 2) & 63; const bool isx = i < 256;
        const u32 uid = isx ? i : 256 + group_of(rank, (i >> 6) & 3) * 64 + (i & 63);
        const u32 dst = (isx ? OFF_XA : OFF_WA) + ((src * 4 + kb) * 64 + rg) * IU_RAW;
        const u32 rel = C_REL + (n % NSLOT) * 4, offE = par + EX_INE + uid * IU_ENC;
        u32x4 x[8];
        bool raw = false;
        if (ent & 1024) {
          #pragma unroll
          for (int j = 0; j < 8; ++j) x[j] = *(LDS u32x4*)(sl + j * 1024 + lane * 16);
          wave_sync();
          ctl_st(rel, n + 1);
        } else if (quick_form(i)) {
          u32x4 c[6];
          #pragma unroll
          for (int j = 0; j < 6; ++j) c[j] = *(LDS u32x4*)(sl + j * 1024 + lane * 16);
          const u32 sd = *(LDS u32*)(sl + IU_SIDE + lane * 4);
          wave_sync();
          ctl_st(rel, n + 1);
          const u32 hd[2] = {(u32)__builtin_amdgcn_readlane((int)sd, 0), (u32)__builtin_amdgcn_readlane((int)sd, 32)};
          #pragma unroll
          for (int j = 0; j < 8; ++j) {
            const u32 e4 = ((hd[j >> 2] >> 16) & 31u) * 0x04040404u;
            const u32 p0 = c[(j * 3) >> 2][(j * 3) & 3], p1 = c[(j * 3 + 1) >> 2][(j * 3 + 1) & 3], p2 = c[(j * 3 + 2) >> 2][(j * 3 + 2) & 3];
            const u32 o3 = bfi(0x80808080u, p2 << 1, bfi(0x10101010u, p2 >> 1, bfi(0x0C0C0C0Cu, p1 >> 3, p0 >> 5)));
            const u32x4 y = {(p0 & 0x9F9F9F9Fu) + e4, (p1 & 0x9F9F9F9Fu) + e4, (p2 & 0x9F9F9F9Fu) + e4, (o3 & 0x9F9F9F9Fu) + e4};
            *(LDS u32x4*)(scr + j * 1024 + lane * 16) = y;
          }
          wave_sync();
          const u32 cnt = (lane < 32 ? hd[0] : hd[1]) & 0xFFFFu, id0 = 2u * (lane & 31) - 2u;
          if (lane & 31) {
            if (id0 < cnt) patch_in(scr, sd & 0xFFFFu);
            if (id0 + 1 < cnt) patch_in(scr, sd >> 16);
          }
          if (max(hd[0] & 0xFFFFu, hd[1] & 0xFFFFu) > IN_CAP0) {
            for (int j = 0; j < 3; ++j) {
              const u32 s2 = __builtin_amdgcn_raw_buffer_load_b32(rsP, 256 + (lane >> 5) * 384 + (j * 32 + (lane & 31)) * 4, offE + IU_SIDE, AUX_SYS);
              const u32 id = IN_CAP0 + j * 64 + 2 * (lane & 31);
              if (id < cnt) patch_in(scr, s2 & 0xFFFFu);
              if (id + 1 < cnt) patch_in(scr, s2 >> 16);
            }
          }
          wave_sync();
          #pragma unroll
          for (int j = 0; j < 8; ++j) x[j] = *(LDS u32x4*)(scr + j * 1024 + lane * 16);
          #pragma unroll
          for (int h = 0; h < 2; ++h)
            if (hd[h] >> 31) {
              #pragma unroll
              for (int j = h * 4; j < h * 4 + 4; ++j) x[j] = __builtin_amdgcn_raw_buffer_load_b128(rsP, lane * 16 + j * 1024, par + EX_INR + uid * IU_RAW, AUX_SYS);
            }
        } else {
          LDS char* var = sl + QV;
          const u32 hdr = (u32)__builtin_amdgcn_readfirstlane((int)*(LDS u32*)var);
          const u32 pe = *(LDS unsigned short*)(var + 8 + (lane & 7) * 2);
          const u32 tlen = (u32)__builtin_amdgcn_readfirstlane((int)*(LDS u32*)(var + 4));
          raw = hdr >> 31;
          if (!raw) {
            const u32 nesc = hdr & 0x1FFFu, npat = (hdr >> 13) & 0x1FFu, bias = (4u * ((hdr >> 24) & 31u) - 136u) * 0x01010101u;
            const u32 t0 = QV_SEC + (nesc + 31) / 32 * 8;
            if (t0 * 8 + tlen > QV0 * 8 && lane < (int)(QV_END - QV0) / 16)
              *(LDS u32x4*)(var + QV0 + lane * 16) = __builtin_amdgcn_raw_buffer_load_b128(rsP, lane * 16, offE + QV + QV0, AUX_SYS);
            wave_sync();
            u32 tn = (u32)(uintptr_t)scr + Q_TER;
            for (u32 r0 = 0; r0 < tlen; r0 += 1024) {
              const u32 bit0 = r0 + lane * 16;
              const u32 before = *(LDS unsigned short*)(var + t0 + bit0 / 8 - 2);
              u32 m = bit0 < tlen ? *(LDS unsigned short*)(var + t0 + bit0 / 8) : 0u;
              int prev = 15 - (int)__clz(before);
              if (bit0 == 0) { m &= ~1u; prev = 0; }
              const u32 mine = __popc(m), inc = wave_scan_u32(mine);
              LDS unsigned char* at = (LDS unsigned char*)(uintptr_t)(tn + inc - mine);
              for (; m; m &= m - 1) { const int pl = __builtin_ctz(m); *at++ = (unsigned char)((pl - prev) * 4 - 4); prev = pl; }
              tn += (u32)__builtin_amdgcn_readlane((int)inc, 63);
            }
            tn = (u32)(uintptr_t)scr + Q_TER;
            wave_sync();
            for (u32 r0 = 0; r0 < nesc; r0 += 2048) {
              const u32x2 two = *(LDS u32x2*)(var + QV_SEC + r0 / 4 + lane * 8);
              u32 c2[8], e2[8], zo[8], bits = 0;
              #pragma unroll
              for (int k = 0; k < 8; ++k) {
                const u32 z = two[k >> 2] >> (8 * (k & 3)), z2 = bfi(0x0000000Fu, z, z << 12);
                c2[k] = bfi(0x00030003u, z2, z2 << 6) & 0x03030303u;
                e2[k] = (c2[k] << 2) & (c2[k] << 1) & 0x04040404u;
                zo[k] = bits;
                bits = __builtin_amdgcn_udot4(e2[k], 0x01010101u, bits, false);
              }
              const u32 inc = wave_scan_u32(bits >> 2), b4 = (tn + inc) * 4 - bits;
              tn += (u32)__builtin_amdgcn_readlane((int)inc, 63);
              u32 g0[8], g1[8];
              #pragma unroll
              for (int k = 0; k < 8; ++k) {
                zo[k] = (zo[k] + b4) >> 2;
                const LDS u32* at = (const LDS u32*)(uintptr_t)(zo[k] & ~3u);
                g0[k] = at[0]; g1[k] = at[1];
              }
              __builtin_amdgcn_sched_barrier(0);
              #pragma unroll
              for (int k = 0; k < 8; ++k) {
                const u32 nx = __builtin_amdgcn_alignbyte(g1[k], g0[k], zo[k]);
                zo[k] = __builtin_amdgcn_perm(0u, 0x64686C7Cu, c2[k]) - __builtin_amdgcn_perm(nx, 0u, __umul24(e2[k], 0x404040u) + e2[k]);
              }
              *(LDS u32x4*)(scr + r0 + lane * 32) = u32x4{zo[0], zo[1], zo[2], zo[3]};
              *(LDS u32x4*)(scr + r0 + lane * 32 + 16) = u32x4{zo[4], zo[5], zo[6], zo[7]};
            }
            wave_sync();
            if ((u32)lane < npat && lane < (int)Q_PAT0) patch_short(scr, pe);
            if (npat > Q_PAT0) {
              for (int j = 0; j < 2; ++j) {
                const u32 s2 = __builtin_amdgcn_raw_buffer_load_b32(rsP, lane * 4 + j * 256, offE + QV + QV_END, AUX_SYS);
                const u32 id = Q_PAT0 + j * 128 + 2 * lane;
                if (id < npat) patch_short(scr, s2 & 0xFFFFu);
                if (id + 1 < npat) patch_short(scr, s2 >> 16);
              }
            }
            landed(endp);
            u32x4 c[5];
            #pragma unroll
            for (int j = 0; j < 5; ++j) c[j] = *(LDS u32x4*)(sl + j * 1024 + lane * 16);
            wave_sync();
            ctl_st(rel, n + 1);
            u32 a[32], es[32], qo[32], cnt4 = 0;
            #pragma unroll
            for (int g = 0; g < 4; ++g) {
              u32 k5[5];
              #pragma unroll
              for (int k = 0; k < 5; ++k) { k5[k] = c[(g * 5 + k) >> 2][(g * 5 + k) & 3]; a[g * 8 + k] = k5[k]; }
              a[g * 8 + 5] = bfi(0x80808080u, k5[4] << 3, bfi(0x08080808u, k5[3] >> 1, k5[0] >> 4));
              a[g * 8 + 6] = bfi(0x80808080u, k5[4] << 2, bfi(0x08080808u, k5[3] >> 2, k5[1] >> 4));
              a[g * 8 + 7] = bfi(0x80808080u, k5[4] << 1, bfi(0x08080808u, k5[3] >> 3, k5[2] >> 4));
            }
            #pragma unroll
            for (int d = 0; d < 32; ++d) {
              es[d] = a[d] & (a[d] >> 1) & 0x04040404u;
              qo[d] = cnt4;
              cnt4 = __builtin_amdgcn_udot4(es[d], 0x01010101u, cnt4, false);
            }
            const u32 base4 = ((u32)(uintptr_t)scr + wave_scan_u32(cnt4 >> 2)) * 4 - cnt4;
            wave_sync();
            #pragma unroll
            for (int h = 0; h < 2; ++h) {
              u32 g0[16], g1[16];
              #pragma unroll
              for (int k = 0; k < 16; ++k) {
                qo[h * 16 + k] = (base4 + qo[h * 16 + k]) >> 2;
                const LDS u32* at = (const LDS u32*)(uintptr_t)(qo[h * 16 + k] & ~3u);
                g0[k] = at[0]; g1[k] = at[1];
              }
              __builtin_amdgcn_sched_barrier(0);
              #pragma unroll
              for (int k = 0; k < 16; ++k) {
                const int d = h * 16 + k;
                const u32 nx = __builtin_amdgcn_alignbyte(g1[k], g0[k], qo[d]);
                const u32 sel = __umul24(es[d], 0x404040u) + es[d];
                x[d >> 2][d & 3] = (a[d] & 0x8F8F8F8Fu) + __builtin_amdgcn_perm(nx, 0x7C7C7C7Cu, sel) + bias;
              }
              __builtin_amdgcn_sched_barrier(0);
            }
          } else {
            landed(endp);
            ctl_st(rel, n + 1);
          }
        }
        if (raw) {
          #pragma unroll
          for (int j = 0; j < 8; ++j) x[j] = __builtin_amdgcn_raw_buffer_load_b128(rsP, lane * 16 + j * 1024, par + EX_INR + uid * IU_RAW, AUX_SYS);
        }
        store_raw_unit(x, rsL, dst, lane);
        vm_drain();
        cnt_add(p.ws, (isx ? rg : 64 + (rg >> 1)) * 5 + kb, lane);
        continue;
      }
      const int t = i - 512, cg = t / 5, rg = t - cg * 5 + (same ? 0 : 3);
      const u32 obase = (rg * 64 * NN + group_of(src, cg >> 4) * 1024 + (cg & 15) * 64) * 4;
      const u32 olane = (li * NN + lg * 4) * 4;
      if (!ok) {
        const u32x4 nan4 = {0x7FC00000u, 0x7FC00000u, 0x7FC00000u, 0x7FC00000u};
        #pragma unroll
        for (int e = 0; e < 16; ++e) ST128(nan4, rsO, olane + (e & 3) * 64, obase + (e >> 2) * 16 * NN * 4, AUX_SYS);
        continue;
      }
      #pragma unroll
      for (int j = 0; j < 4; ++j) *(LDS u32x4*)(scr + j * 1024 + lane * 16) = u32x4{0u, 0u, 0u, 0u};
      landed(endp - 11);
      const u32 odd = *(LDS unsigned short*)(sl + OU_SIDE + lane * 2) | ((u32)*(LDS unsigned char*)(sl + OU_SIDE + 128 + lane) << 16);
      const u32 hdr = (u32)__builtin_amdgcn_readfirstlane((int)*(LDS u32*)(sl + OU_SIDE + 192));
      u32 se = *(LDS unsigned short*)(sl + OU_SIDE + 196 + (lane < 24 ? lane : 0) * 2);
      u32x2 fe = {*(LDS unsigned short*)(sl + OU_SIDE + 244 + (lane & 1) * 2), *(LDS u32*)(sl + OU_SIDE + 248 + (lane & 1) * 4)};
      const u32 tl[3] = {*(LDS u32*)(sl + 11264 + lane * 4), *(LDS u32*)(sl + 11520 + lane * 4), *(LDS u32*)(sl + 11776 + lane * 4)};
      const u32 uid = rank * 320 + t;
      if (!(hdr >> 31)) {
        const u32 ns = hdr & 0xFFu, nf = (hdr >> 8) & 0xFFu; const int g = (int)((hdr >> 16) & 0x1FFu) - 256;
        const u32 tail = par + EX_OUE + uid * OU_ENC + 12288;
        if (ns > OS_CAP0) {
          const u32 s2 = __builtin_amdgcn_raw_buffer_load_b16(rsP, lane >= (int)OS_CAP0 ? lane * 2 : OS_CAP0 * 2, tail - OS_CAP0 * 2, AUX_SYS);
          se = lane >= (int)OS_CAP0 ? s2 : se;
          asm volatile("" : "+v"(se));
        }
        if (nf > OF_CAP0) {
          const u32x2 f2 = __builtin_amdgcn_raw_buffer_load_b64(rsP, max((u32)OF_CAP0, min((u32)lane, nf - 1)) * 8, tail + 128 - OF_CAP0 * 8, AUX_SYS);
          fe = lane >= (int)OF_CAP0 ? f2 : fe;
          asm volatile("" : "+v"(fe));
        }
        if ((u32)lane < ns)
          __hip_atomic_fetch_add((LDS u32*)(scr + ((se >> 13) & 3u) * 1024 + ((se >> 5) & 63u) * 16 + ((se >> 11) & 3u) * 4),
                                 se >> 15 ? ((se & 3u) | ((se & 12u) << 6)) << (se & 16u) : ((se & 7u) << 5) << (se & 24u),
                                 __ATOMIC_RELAXED, __HIP_MEMORY_SCOPE_WORKGROUP);
        const u32 fgq = (u32)lane < nf ? fe.x >> 8 : 16u;
        u64 om = (1ull << (odd & 63)) | (1ull << ((odd >> 6) & 63)) | (1ull << ((odd >> 12) & 63)) | (1ull << ((odd >> 18) & 63));
        if ((odd & 63) > ((odd >> 6) & 63)) om = 0;
        const u32 ob[2] = {(u32)om, (u32)(om >> 32)};
        u32 t15 = tl[0] << 1;
        wave_sync();
        #pragma unroll
        for (int mi = 0; mi < 4; ++mi) {
          landed(mi < 3 ? endp - 8 + 3 * mi : endp);
          u32x4 c[3];
          #pragma unroll
          for (int j = 0; j < (mi < 3 ? 3 : 2); ++j) c[j] = *(LDS u32x4*)(sl + (mi * 3 + j) * 1024 + lane * 16);
          const u32x4 adj = *(LDS u32x4*)(scr + mi * 1024 + lane * 16);
          if (mi == 3) { wave_sync(); ctl_st(C_REL + (n % NSLOT) * 4, n + 1); c[2] = u32x4{tl[0], tl[1], tl[2], 0u}; }
          #pragma unroll
          for (int ni = 0; ni < 4; ++ni) {
            const int k = ni * 3, gq = mi * 4 + ni;
            const u32 l0 = c[k >> 2][k & 3], l1 = c[(k + 1) >> 2][(k + 1) & 3], tp = gq < 15 ? c[(k + 2) >> 2][(k + 2) & 3] : t15;
            if (gq < 14) t15 = bfi(0x00010001u << gq, tp >> (14 - gq), t15);
            const u32 top = __builtin_amdgcn_bitop3_b32(top_unpack(tp), adj[ni] << 5, 0x60606060u, 0x78);
            const u32 low = ((ob[gq >> 3] << (gq & 7)) & 0x80808080u) + (adj[ni] & 0xE0E0E0E0u);
            const u32 q01 = __builtin_amdgcn_perm(top, low, 0x05010400u), q23 = __builtin_amdgcn_perm(top, low, 0x07030602u);
            float y[4] = {unfix((int)__builtin_amdgcn_perm(q01, l0, 0x05010004u), g + 7), unfix((int)__builtin_amdgcn_perm(q01, l0, 0x07030206u), g + 7),
                          unfix((int)__builtin_amdgcn_perm(q23, l1, 0x05010004u), g + 7), unfix((int)__builtin_amdgcn_perm(q23, l1, 0x07030206u), g + 7)};
            if (nf)
              for (u64 em = __ballot(fgq == (u32)gq); em; em &= em - 1) {
                const int from = __builtin_ctzll(em);
                const u32 rel = ((u32)__builtin_amdgcn_readlane((int)fe.x, from) & 255u) - (u32)lane * 4u;
                const float vv = as_f((u32)__builtin_amdgcn_readlane((int)fe.y, from));
                #pragma unroll
                for (int r = 0; r < 4; ++r) y[r] = rel == (u32)r ? vv : y[r];
              }
            ST128((u32x4{as_u(y[0] * scale), as_u(y[1] * scale), as_u(y[2] * scale), as_u(y[3] * scale)}), rsO, olane + ni * 64, obase + mi * 16 * NN * 4, AUX_SYS);
          }
        }
      } else {
        landed(endp);
        ctl_st(C_REL + (n % NSLOT) * 4, n + 1);
        #pragma unroll
        for (int mi = 0; mi < 4; ++mi) {
          u32x4 r4[4];
          #pragma unroll
          for (int ni = 0; ni < 4; ++ni) r4[ni] = __builtin_amdgcn_raw_buffer_load_b128(rsP, lane * 16 + ni * 1024, par + EX_OUR + uid * OU_RAW + mi * 4096, AUX_SYS);
          #pragma unroll
          for (int ni = 0; ni < 4; ++ni) {
            const f32x4 y = __builtin_bit_cast(f32x4, r4[ni]) * scale;
            ST128(__builtin_bit_cast(u32x4, y), rsO, olane + ni * 64, obase + mi * 16 * NN * 4, AUX_SYS);
          }
        }
      }
    }
    return;
  }

  constexpr int SLOTS = NGEMM / 8;
  const int xc = b & 7, slot = b >> 3, wm = wave >> 1, wn = wave & 1;
  for (int ti = slot; ti < 36; ti += SLOTS) {
    const int j = xc >> 1, ft = ti * 2 + (xc & 1);
    const int c = ti < 24 ? ft / 6 : 4 * (xc & 1) + (ti < 28 ? ti - 24 : (ti - 28) >> 1), cb = j * 8 + c;
    auto row_group = [&](int k) -> int {
      if (ti >= 28) return 8 * rank + 4 * (ti & 1) + k;
      if (ti >= 24) return 8 * ((~rank & 4) | ((j + 1 + min(k, 2)) & 3)) + 3;
      const int t = ti % 3;
      return 8 * (xc & 1 ? (~rank & 4) | ((j + 1 + t) & 3) : (rank & 4) | ((rank + 1 + t) & 3)) + 4 * (xc & 1) + k;
    };
    const int rgw = row_group(wm);
    const u32 deadw = (u32)(uintptr_t)(smem + L_CTL);
    auto ready = [&](int grp) {
      const bool isx = lane < 4;
      const int rgl = row_group(lane & 3);
      const u32 off = OFF_CNT + ((isx ? rgl : 64 + cb) * 5 + grp) * 128, want = (grp == 4 ? (isx ? 6u : 8u) : (isx ? 7u : 14u)) * call;
      u32 got;
      if (!wait_ge(p.ws, off, want, call, got)) ctl_st(deadw, 1u);
    };
    const u32 xo0 = OFF_XA + rgw * IU_RAW + wn * 4096, wo0 = OFF_WA + cb * 2 * IU_RAW + wave * 2048;
    auto dma = [&](int k, int ks, u32 st) {
      const u32 xo = xo0 + ks * (64 * IU_RAW), wo = wo0 + ks * (64 * IU_RAW);
      LDS char* dx = smem + st + wave * 4096;
      LDS char* dw = smem + st + 32 * KiB + wave * 2048;
      if (k == 0) __builtin_amdgcn_raw_ptr_buffer_load_lds(rsL, (LDS void*)dx, 16, lane * 16, xo, 0, AUX_SYS);
      if (k == 1) __builtin_amdgcn_raw_ptr_buffer_load_lds(rsL, (LDS void*)dx, 16, lane * 16, xo, 1024, AUX_SYS);
      if (k == 2) __builtin_amdgcn_raw_ptr_buffer_load_lds(rsL, (LDS void*)dx, 16, lane * 16, xo, 2048, AUX_SYS);
      if (k == 3) __builtin_amdgcn_raw_ptr_buffer_load_lds(rsL, (LDS void*)dx, 16, lane * 16, xo, 3072, AUX_SYS);
      if (k == 4) __builtin_amdgcn_raw_ptr_buffer_load_lds(rsL, (LDS void*)dw, 16, lane * 16, wo, 0, AUX_SYS);
      if (k == 5) __builtin_amdgcn_raw_ptr_buffer_load_lds(rsL, (LDS void*)dw, 16, lane * 16, wo, 1024, AUX_SYS);
    };
    f32x4 acc[4][4];
    #pragma unroll
    for (int mi = 0; mi < 4; ++mi)
      #pragma unroll
      for (int ni = 0; ni < 4; ++ni) acc[mi][ni] = f32x4{0.f, 0.f, 0.f, 0.f};
    const bool twin = ti >= 24 && ti < 28;
    const int kr = twin ? (~rank & 4) | j : rank;
    if (wave == 0) {
      if (ti == slot) ctl_st(deadw, 0u);
      for (int grp = twin ? 0 : 4; grp < 5; ++grp) ready(grp);
    }
    asm volatile("s_waitcnt vmcnt(0) lgkmcnt(0)\n\ts_barrier" ::: "memory");
    #pragma unroll
    for (int k = 0; k < 12; ++k) dma(k % 6, kr * 4 + k / 6, k / 6 * STAGE);
    u32 st = 0;
    int ksn = kr * 4 + 2, jn = -2, kbn = 0;
    i32x8 fa[8], fb[8];
    auto step = [&](i32x8 (&cur)[8], const i32x8 (&prev)[8], bool first, bool more, bool last) {
      if (more && wave == 0 && jn == 0) ready(kbn);
      if (!last) asm volatile("s_waitcnt vmcnt(6) lgkmcnt(0)\n\ts_barrier" ::: "memory");
      else asm volatile("s_waitcnt vmcnt(0) lgkmcnt(0)\n\ts_barrier" ::: "memory");
      const u32 st2 = st == 0 ? 2 * STAGE : st - STAGE;
      LDS char* ax = smem + st + wm * 8192 + lane * 16;
      LDS char* aw = smem + st + 32 * KiB + wn * 8192 + lane * 16;
      u32x4 h[16];
      auto M = [&](int k) {
        if (!first) acc[k >> 2][k & 3] = __builtin_amdgcn_mfma_scale_f32_16x16x128_f8f6f4(prev[k & 3], prev[4 + (k >> 2)], acc[k >> 2][k & 3], 1, 1, 0, 0, 0, 0);
        __builtin_amdgcn_sched_barrier(0);
      };
      auto R = [&](int k) { h[k] = *(LDS u32x4*)(((k >> 1) & 1 ? ax : aw) + (k >> 2) * 2048 + (k & 1) * 1024); __builtin_amdgcn_sched_barrier(0); };
      __builtin_amdgcn_sched_barrier(0);
      #pragma unroll
      for (int k = 0; k < 6; ++k) {
        M(2 * k); R(2 * k); R(2 * k + 1); M(2 * k + 1);
        if (more) dma(k, ksn, st2);
        __builtin_amdgcn_sched_barrier(0);
      }
      M(12); R(12); R(13); M(13); R(14); R(15); M(14); M(15);
      #pragma unroll
      for (int f = 0; f < 4; ++f) {
        cur[f] = __builtin_bit_cast(i32x8, __builtin_shufflevector(h[4 * f], h[4 * f + 1], 0, 1, 2, 3, 4, 5, 6, 7));
        cur[4 + f] = __builtin_bit_cast(i32x8, __builtin_shufflevector(h[4 * f + 2], h[4 * f + 3], 0, 1, 2, 3, 4, 5, 6, 7));
      }
      if (++jn == 7) { jn = 0; ++kbn; }
      ksn = jn < 0 ? ksn + 1 : ((kr + 1 + jn) & 7) * 4 + kbn;
      st = st == 2 * STAGE ? 0 : st + STAGE;
    };
    step(fa, fb, true, true, false);
    #pragma unroll 1
    for (int s = 1; s < 29; s += 2) { step(fb, fa, false, true, false); step(fa, fb, false, true, false); }
    step(fb, fa, false, true, false);
    step(fa, fb, false, false, false);
    step(fb, fa, false, false, true);
    #pragma unroll
    for (int k = 0; k < 16; ++k)
      acc[k >> 2][k & 3] = __builtin_amdgcn_mfma_scale_f32_16x16x128_f8f6f4(fb[k & 3], fb[4 + (k >> 2)], acc[k >> 2][k & 3], 1, 1, 0, 0, 0, 0);
    int lz = lane; asm volatile("" : "+v"(lz));
    const bool dead = ctl_ld(deadw) != 0u;
    const int owner = rgw >> 3;
    if (owner == rank) {
      const u32 obase = ((rgw & 7) * 64 * NN + group_of(rank, j) * 1024 + c * 128 + wn * 64) * 4, olane = ((lz & 15) * NN + (lz >> 4) * 4) * 4;
      const float sc = dead ? __builtin_nanf("") : scale;
      #pragma unroll
      for (int mi = 0; mi < 4; ++mi)
        #pragma unroll
        for (int ni = 0; ni < 4; ++ni)
          ST128(__builtin_bit_cast(u32x4, acc[mi][ni] * sc), rsO, olane + ni * 64, obase + mi * 16 * NN * 4, AUX_SYS);
    } else if (!dead && !(ti >= 24 && wm == 3)) {
      const int t = (cb * 2 + wn) * 5 + (rgw & 7) - ((owner >> 2) == (rank >> 2) ? 0 : 3);
      const u32 uid = owner * 320 + t;
      const u32 offE = par + EX_OUE + uid * OU_ENC;
      float sa = 0.f;
      #pragma unroll
      for (int mi = 0; mi < 4; ++mi)
        #pragma unroll
        for (int ni = 0; ni < 4; ++ni) sa += __builtin_fabsf(acc[mi][ni][(mi + ni) & 3]);
      const int g = max(0, 24 - ((int)((as_u(wave_sum_f32(sa) * (4.2f / 1024.f)) >> 23) & 255u) - 126));
      LDS char* last1k = smem + wave * 4096;
      LDS char* side = last1k + 768;
      LDS char* pend = last1k + 1536;
      u32 npend = 0, sg[2] = {0u, 0u};
      auto quad = [&](int gq, u32& l0, u32& l1, u32& tp) {
        u32 qv[4], fr[4];
        #pragma unroll
        for (int r = 0; r < 4; ++r) {
          const float tt = __builtin_amdgcn_ldexpf(acc[gq >> 2][gq & 3][r], g + 7);
          qv[r] = (u32)f2i_sat(tt);
          fr[r] = as_u(__builtin_amdgcn_fractf(tt));
        }
        const u32 p01 = __builtin_amdgcn_perm(qv[1], qv[0], 0x07030400u), p23 = __builtin_amdgcn_perm(qv[3], qv[2], 0x07030400u);
        const u32 low = __builtin_amdgcn_perm(p23, p01, 0x05040100u);
        u32 miss;
        tp = top_pack(__builtin_amdgcn_perm(p23, p01, 0x07060302u), miss);
        l0 = __builtin_amdgcn_perm(qv[1], qv[0], 0x06050201u);
        l1 = __builtin_amdgcn_perm(qv[3], qv[2], 0x06050201u);
        sg[gq >> 3] = bfi(0x80808080u >> (gq & 7), low >> (gq & 7), sg[gq >> 3]);
        const bool odd_one = (fr[0] | fr[1] | fr[2] | fr[3] | (low & 0x7F7F7F7Fu) | (miss & 0x00380038u)) != 0u;
        const u64 bm = __ballot(odd_one);
        if (odd_one) {
          const u32 sl = min(63u, npend + __builtin_amdgcn_mbcnt_hi((u32)(bm >> 32), __builtin_amdgcn_mbcnt_lo((u32)bm, 0u)));
          *(LDS f32x4*)(pend + sl * 16) = acc[gq >> 2][gq & 3];
          *(LDS u32*)(pend + 1024 + sl * 4) = gq * 64 + lz;
        }
        npend += __popcll(bm);
      };
      u32 z0, z1, t15;
      quad(15, z0, z1, t15);
      *(LDS u32*)(last1k + 256 + lz * 4) = z0;
      *(LDS u32*)(last1k + 512 + lz * 4) = z1;
      #pragma unroll
      for (int mi = 0; mi < 4; ++mi) {
        u32 pk[12];
        #pragma unroll
        for (int ni = 0; ni < 4; ++ni) {
          const int gq = mi * 4 + ni;
          if (gq == 15) break;
          quad(gq, pk[ni * 3], pk[ni * 3 + 1], pk[ni * 3 + 2]);
          pk[ni * 3 + 2] = bfi(0x40004000u, gq < 14 ? t15 << (14 - gq) : t15 >> 1, pk[ni * 3 + 2]);
        }
        #pragma unroll
        for (int j = 0; j < (mi < 3 ? 3 : 2); ++j)
          ST128((u32x4{pk[j * 4], pk[j * 4 + 1], pk[j * 4 + 2], pk[j * 4 + 3]}), rsL, lz * 16 + j * 1024, offE + mi * 3072, AUX_SYS);
        if (mi == 3) *(LDS u32*)(last1k + lz * 4) = pk[8];
      }
      u32 ns = npend > 64u ? OS_CAP + 1 : 0u, nf = 0;
      auto put = [&](bool on, u32 e) {
        const u64 bm = __ballot(on);
        const u32 sl = ns + __builtin_amdgcn_mbcnt_hi((u32)(bm >> 32), __builtin_amdgcn_mbcnt_lo((u32)bm, 0u));
        if (on && sl < OS_CAP) *(LDS unsigned short*)(side + (sl < OS_CAP0 ? 196 : 256 - OS_CAP0 * 2) + sl * 2) = (unsigned short)e;
        ns += __popcll(bm);
      };
      u64 om = ((u64)sg[1] << 32) | sg[0];
      u32 odd = om ? 0u : 1u, last = 0;
      #pragma unroll
      for (int k = 0; k < 4; ++k) { if (om) last = (u32)__builtin_ctzll(om); om &= om - 1; odd |= last << (6 * k); }
      for (; __any(om != 0) && ns <= OS_CAP; om &= om - 1) {
        const u32 e = (u32)__builtin_ctzll(om | (1ull << 63));
        put(om != 0, (((e >> 5) * 8 + 7 - (e & 7)) << 11) | (lz << 5) | (e & 24u) | 4u);
      }
      wave_sync();
      for (u32 p0 = 0; p0 < min(npend, 64u); p0 += 16) {
        const u32 pi = p0 + (lz >> 2), r = lz & 3;
        const bool on = pi < npend;
        const float v = *(LDS float*)(pend + p0 * 16 + lz * 4);
        const u32 pp = *(LDS u32*)(pend + 1024 + pi * 4);
        const float tt = __builtin_amdgcn_ldexpf(v, g + 7);
        const u32 iv = (u32)f2i_sat(tt);
        const bool full = on && (as_u((float)(int)iv) != as_u(tt) || (iv & 31u));
        const u32 top = __builtin_amdgcn_perm((u32)__builtin_amdgcn_mov_dpp((int)iv, 0xB1, 0xf, 0xf, true), iv, 0x0C0C0703u);
        u32 miss;
        const u32 fl = (top_unpack(top_pack(top, miss)) ^ top) >> 5, d4 = (fl & 3u) | ((fl >> 6) & 12u);
        put(on && !full && (iv & 96u), (pp << 5) | (r << 3) | ((iv >> 5) & 3u));
        put(on && !(r & 1) && d4, 0x8000u | (pp << 5) | (r << 3) | d4);
        const u64 bm = __ballot(full);
        if (bm) {
          const u32 sl = nf + __builtin_amdgcn_mbcnt_hi((u32)(bm >> 32), __builtin_amdgcn_mbcnt_lo((u32)bm, 0u));
          if (full && sl < OF_CAP0) { *(LDS unsigned short*)(side + 244 + sl * 2) = (unsigned short)(pp * 4 + r); *(LDS u32*)(side + 248 + sl * 4) = as_u(v); }
          else if (full && sl < OF_CAP) *(LDS u32x2*)(side + 384 - OF_CAP0 * 8 + sl * 8) = u32x2{pp * 4 + r, as_u(v)};
          nf += __popcll(bm);
        }
      }
      const bool raw = ns > OS_CAP || nf > OF_CAP;
      *(LDS unsigned short*)(side + lz * 2) = (unsigned short)odd;
      *(LDS unsigned char*)(side + 128 + lz) = (unsigned char)(odd >> 16);
      if (lz == 0) *(LDS u32*)(side + 192) = raw ? 0x80000000u : (ns | (nf << 8) | ((u32)(g + 256) << 16));
      wave_sync();
      ST128(*(LDS u32x4*)(last1k + lz * 16), rsL, lz * 16, offE + 11264, AUX_SYS);
      if ((ns > OS_CAP0 || nf > OF_CAP0) && lz < 24) ST128(*(LDS u32x4*)(last1k + 1024 + lz * 16), rsL, lz * 16, offE + 12288, AUX_SYS);
      if (raw) {
        const u32 offR = par + EX_OUR + uid * OU_RAW;
        #pragma unroll
        for (int mi = 0; mi < 4; ++mi)
          #pragma unroll
          for (int ni = 0; ni < 4; ++ni)
            ST128(__builtin_bit_cast(u32x4, acc[mi][ni]), rsL, lz * 16 + ni * 1024, offR + mi * 4096, AUX_SYS);
      }
      vm_drain();
      if (lz == owner)
        __hip_atomic_store((GLB u32*)(peer_ws + OFF_FOUT + (rank * 320 + t) * 128), call << 4, __ATOMIC_RELAXED, __HIP_MEMORY_SCOPE_SYSTEM);
    }
  }
}

}

extern "C" void kernel_launch(int rank, int nranks, ncclComm_t comm, hipStream_t stream,
                              void* const* d_in, const int* in_sizes, int n_in,
                              void* d_out, int out_size,
                              const int* in_dtypes, int out_dtype,
                              void* const* d_out_all,
                              void* const* d_ws_all, size_t ws_bytes) {
  (void)comm; (void)in_sizes; (void)n_in; (void)out_size; (void)in_dtypes; (void)out_dtype; (void)d_out_all;
  if (nranks != NR || (unsigned)rank >= (unsigned)NR || ws_bytes < WS_NEED) { fprintf(stderr, "unexpected launch shape\n"); return; }
  static unsigned attr_set = 0;
  if (!(attr_set >> rank & 1u)) {
    HIP_CHECK(hipFuncSetAttribute((const void*)gemm_rs_kernel, hipFuncAttributeMaxDynamicSharedMemorySize, (int)LDS_BYTES));
    attr_set |= 1u << rank;
  }
  Params p;
  p.x = (const uint8_t*)d_in[0]; p.w = (const uint8_t*)d_in[1]; p.sx = (const float*)d_in[2]; p.sw = (const float*)d_in[3];
  p.out = (float*)d_out; p.ws_all = (char* const*)d_ws_all; p.ws = (char*)d_ws_all[rank]; p.rank = rank; p.ws_bytes = WS_NEED;
  hipLaunchKernelGGL(gemm_rs_kernel, dim3(NWG), dim3(NTHR), LDS_BYTES, stream, p);
  HIP_CHECK(hipGetLastError());
}
```

```cpp
#include <hip/hip_runtime.h>
#include <rccl/rccl.h>
#include <cstdio>
#include <cstdint>

#define HIP_CHECK(expr) do { hipError_t _e = (expr); if (_e != hipSuccess) { \
    fprintf(stderr, "HIP error %s at %s:%d\n", hipGetErrorString(_e), __FILE__, __LINE__); \
    return; } } while (0)

namespace {

typedef unsigned u32;
typedef unsigned long long u64;
typedef unsigned u32x4 __attribute__((ext_vector_type(4)));
typedef unsigned u32x2 __attribute__((ext_vector_type(2)));
typedef int i32x2 __attribute__((ext_vector_type(2)));
typedef int i32x8 __attribute__((ext_vector_type(8)));
typedef float f32x4 __attribute__((ext_vector_type(4)));
typedef __amdgpu_buffer_rsrc_t rsrc_t;
#define LDS __attribute__((address_space(3)))
#define GLB __attribute__((address_space(1)))

constexpr int NR = 8, MM = 4096, NN = 16384, KS = 512;
constexpr int NWG = 256, NTHR = 512;
constexpr int PULL_WGS = 8;
constexpr int WIN = 8, WIN_OUT = 7;
constexpr u32 LOOK = 5, LOOK_OUT = 3, LOOK_END = 2;
constexpr int NRAW = 8, X1FIRST = 16;
constexpr int NPULL = 7 * PULL_WGS, NGEMM = NWG - NPULL;
constexpr int AUX_SYS = 17;
constexpr u64 DEADLINE_TICKS = 40000000ull;

constexpr u32 KiB = 1024, MiB = 1024 * 1024;
constexpr u32 OFF_CALL = 0;
constexpr u32 OFF_TMO = 32 * KiB;
constexpr u32 OFF_TIC = 33 * KiB;
constexpr u32 OFF_CNT = 40 * KiB;
constexpr u32 OFF_FIN = 128 * KiB;
constexpr u32 OFF_FOUT = OFF_FIN + 8 * 576 * 128;
constexpr u32 OFF_XA = 2 * MiB;
constexpr u32 OFF_WA = OFF_XA + 16 * MiB;
constexpr u32 OFF_PAR = OFF_WA + 16 * MiB;
constexpr u32 IU_ENC = 8 * KiB, IU_RAW = 8 * KiB;
constexpr u32 OU_ENC = 12 * KiB, OU_RAW = 16 * KiB;
constexpr u32 IU_SIDE = 6144, OU_SIDE = 11776;
constexpr u32 EX_INE = 0;
constexpr u32 EX_INR = EX_INE + 1280 * IU_ENC;
constexpr u32 EX_OUE = EX_INR + 1280 * IU_RAW;
constexpr u32 EX_OUR = EX_OUE + 2560 * OU_ENC;
constexpr u32 PAR_BYTES = EX_OUR + 2560 * OU_RAW;
constexpr u32 WS_NEED = OFF_PAR + 2 * PAR_BYTES;
static_assert(OFF_CNT + 96 * 5 * 128 <= OFF_FIN && OFF_FOUT + 8 * 320 * 128 <= OFF_XA, "flags");
static_assert(WS_NEED <= 257u * MiB, "workspace");
constexpr u32 QV = 5120, QV0 = 1024, QV_SEC = 24, QV_END = 1792, QV_ALL = 2304;
constexpr u32 Q_ESC = 6144, Q_TER = 6400, Q_TMAX = 1760, Q_PAT0 = 8, Q_PAT = 264;
constexpr u32 IN_CAP0 = 62, IN_CAP = 254;
constexpr u32 OS_CAP = 64, OF_CAP = 32;

constexpr u32 STAGE = 48 * KiB;
constexpr u32 L_SCR = 64 * KiB, SCR_W = 10752;
constexpr u32 NSLOT = 8, SLOT = 13 * KiB, NCONS = 6;
constexpr u32 L_PSCR = NSLOT * SLOT;
constexpr u32 L_CTL = L_PSCR + NCONS * 8 * KiB;
constexpr u32 LDS_BYTES = L_CTL + 256;
static_assert(3 * STAGE <= L_CTL && L_SCR + 8 * SCR_W <= LDS_BYTES && LDS_BYTES <= 160 * KiB, "lds");

__device__ __forceinline__ bool quick_form(int i) { return i < 256 ? (i & 3) < 2 : (i & 0x3B) == 0; }
__device__ __forceinline__ int group_of(int rank, int j) { return rank < 4 ? rank * 4 + j : j * 4 + (rank & 3); }

struct Params {
  const uint8_t* x; const uint8_t* w; const float* sx; const float* sw;
  float* out; char* const* ws_all; char* ws; int rank; u32 ws_bytes;
};

__device__ __forceinline__ rsrc_t mk_rsrc(const void* p, u32 bytes) {
  u64 a = (u64)p;
  u32 lo = __builtin_amdgcn_readfirstlane((u32)a), hi = __builtin_amdgcn_readfirstlane((u32)(a >> 32));
  return __builtin_amdgcn_make_buffer_rsrc((void*)(((u64)hi << 32) | lo), 0, (int)bytes, 0x00020000);
}
#define ST128(v, rs, voff, soff, aux) __builtin_amdgcn_raw_buffer_store_b128((v), (rs), (int)((u32)(voff) + (u32)(soff)), 0, (aux))
__device__ __forceinline__ float as_f(u32 v) { return __builtin_bit_cast(float, v); }
__device__ __forceinline__ u32 as_u(float v) { return __builtin_bit_cast(u32, v); }
__device__ __forceinline__ int f2i_sat(float v) {
  int r; asm("v_cvt_i32_f32 %0, %1" : "=v"(r) : "v"(v)); return r;
}
__device__ __forceinline__ void vm_drain() { asm volatile("s_waitcnt vmcnt(0)" ::: "memory"); }
__device__ __forceinline__ void wave_sync() {
  asm volatile("s_waitcnt lgkmcnt(0)" ::: "memory");
}
__device__ __forceinline__ u32 bfi(u32 m, u32 a, u32 b) { return __builtin_amdgcn_bitop3_b32(m, a, b, 0xCA); }
template <int CTRL, int ROWS> __device__ __forceinline__ u32 dpp0(u32 v) { return (u32)__builtin_amdgcn_update_dpp(0, (int)v, CTRL, ROWS, 0xf, false); }
__device__ __forceinline__ u32 wave_scan_u32(u32 v) {
  v += dpp0<0x111, 0xf>(v); v += dpp0<0x112, 0xf>(v); v += dpp0<0x114, 0xf>(v); v += dpp0<0x118, 0xf>(v);
  v += dpp0<0x142, 0xa>(v); v += dpp0<0x143, 0xc>(v);
  return v;
}
__device__ __forceinline__ float wave_sum_f32(float v) {
  v += as_f(dpp0<0x111, 0xf>(as_u(v))); v += as_f(dpp0<0x112, 0xf>(as_u(v))); v += as_f(dpp0<0x114, 0xf>(as_u(v)));
  v += as_f(dpp0<0x118, 0xf>(as_u(v))); v += as_f(dpp0<0x142, 0xa>(as_u(v))); v += as_f(dpp0<0x143, 0xc>(as_u(v)));
  return as_f((u32)__builtin_amdgcn_readlane((int)as_u(v), 63));
}
__device__ __forceinline__ u32 ctl_ld(u32 addr) {
  u32 v; asm volatile("ds_read_b32 %0, %1\n\ts_waitcnt lgkmcnt(0)" : "=&v"(v) : "v"(addr) : "memory");
  return (u32)__builtin_amdgcn_readfirstlane((int)v);
}
__device__ __forceinline__ void ctl_st(u32 addr, u32 v) { asm volatile("ds_write_b32 %0, %1" :: "v"(addr), "v"(v) : "memory"); }
__device__ __forceinline__ u32 ld_flag(const char* ws, u32 off) {
  return __hip_atomic_load((const GLB u32*)(ws + off), __ATOMIC_RELAXED, __HIP_MEMORY_SCOPE_SYSTEM);
}
__device__ __forceinline__ bool wait_ge(char* ws, u32 off, u32 target, u32 call, u32& got) {
  if (__all((got = ld_flag(ws, off)) >= target)) return true;
  const u64 t0 = __builtin_amdgcn_s_memrealtime();
  GLB u32* tmo = (GLB u32*)(ws + OFF_TMO);
  for (u32 it = 1;; ++it) {
    __builtin_amdgcn_s_sleep(2);
    if (__all((got = ld_flag(ws, off)) >= target)) return true;
    if ((it & 15u) == 0u &&
        (__hip_atomic_load(tmo, __ATOMIC_RELAXED, __HIP_MEMORY_SCOPE_AGENT) == call ||
         __builtin_amdgcn_s_memrealtime() - t0 > DEADLINE_TICKS)) {
      __hip_atomic_store(tmo, call, __ATOMIC_RELAXED, __HIP_MEMORY_SCOPE_AGENT);
      return false;
    }
  }
}
__device__ __forceinline__ void cnt_add(char* ws, u32 idx, int lane) {
  if (lane == 0) __hip_atomic_fetch_add((GLB u32*)(ws + OFF_CNT + idx * 128), 1u, __ATOMIC_RELAXED, __HIP_MEMORY_SCOPE_AGENT);
}

__device__ __forceinline__ void store_raw_unit(const u32x4 (&x)[8], rsrc_t rs, u32 off, int lane) {
  #pragma unroll
  for (int j = 0; j < 8; ++j) ST128(x[j], rs, lane * 16 + j * 1024, off, AUX_SYS);
}

__device__ __forceinline__ void export_half(const u32x4 (&x)[4], rsrc_t rs, u32 par, u32 uid, int hf, LDS char* scr, int lane) {
  const u32 offE = par + EX_INE + uid * IU_ENC;
  u32 s = 0;
  #pragma unroll
  for (int j = 0; j < 4; ++j) {
    *(LDS u32x4*)(scr + j * 1024 + lane * 16) = x[j];
    s = __builtin_amdgcn_sad_u8(x[j][j] & 0x7C7C7C7Cu, 0u, s);
  }
  s = (u32)__builtin_amdgcn_readlane((int)wave_scan_u32(s), 63);
  int e0 = ((int)s - 17121) >> 12;
  e0 = e0 < 0 ? 0 : (e0 > 24 ? 24 : e0);
  const u32 e4 = (u32)e0 * 0x04040404u;
  u32 mk[2] = {0, 0};
  u32 pk[12];
  #pragma unroll
  for (int j = 0; j < 4; ++j) {
    u32 o[4];
    #pragma unroll
    for (int c = 0; c < 4; ++c) {
      const u32 u = (x[j][c] | 0x80808080u) - e4;
      o[c] = bfi(0x1F1F1F1Fu, u, x[j][c]);
      const u32 t = (((u >> 1) ^ 0x40404040u) & 0x70707070u) + 0x70707070u;
      const int d = j * 4 + c;
      mk[d >> 3] = bfi(0x80808080u >> (d & 7), t >> (d & 7), mk[d >> 3]);
    }
    pk[j * 3 + 0] = bfi(0x60606060u, o[3] << 5, o[0]);
    pk[j * 3 + 1] = bfi(0x60606060u, o[3] << 3, o[1]);
    pk[j * 3 + 2] = bfi(0x60606060u, bfi(0x40404040u, o[3] >> 1, o[3] << 1), o[2]);
  }
  #pragma unroll
  for (int j = 0; j < 3; ++j)
    ST128((u32x4{pk[j * 4], pk[j * 4 + 1], pk[j * 4 + 2], pk[j * 4 + 3]}), rs, lane * 16 + j * 1024, offE + hf * 3072, AUX_SYS);
  const u32 mine = __popc(mk[0]) + __popc(mk[1]);
  const u32 inc = wave_scan_u32(mine);
  const u32 total = (u32)__builtin_amdgcn_readlane((int)inc, 63);
  if (total <= IN_CAP) {
    LDS unsigned short* list = (LDS unsigned short*)(scr + 8192);
    u32 slot = 2 + inc - mine;
    #pragma unroll
    for (int q = 0; q < 2; ++q) {
      u32 m = mk[q];
      while (m) {
        const int bp = __builtin_ctz(m); m &= m - 1;
        const int d = q * 8 + 7 - (bp & 7);
        list[slot++] = (unsigned short)((d >> 2) * 1024 + lane * 16 + (d & 3) * 4 + (bp >> 3));
      }
    }
    wave_sync();
    for (u32 j = 0; j < 2; ++j) {
      const u32 pp = *(LDS u32*)(scr + 8192 + j * 256 + lane * 4);
      const u32 p0 = pp & 4095u, p1 = (pp >> 16) & 4095u;
      const u32 b0 = *(LDS unsigned char*)(scr + p0), b1 = *(LDS unsigned char*)(scr + p1);
      u32 sd = (((p0 << 3) | ((128u + (b0 & 127u) - 4u * (u32)e0) >> 5) | (p1 << 19) | (((128u + (b1 & 127u) - 4u * (u32)e0) >> 5) << 16))) + hf * 0x80008000u;
      if (j == 0 && lane == 0) sd = total | ((u32)e0 << 16);
      const u32 at = j * 64 + lane;
      if (at < 32 || total > IN_CAP0)
        __builtin_amdgcn_raw_buffer_store_b32(sd, rs, at < 32 ? hf * 128 + at * 4 : 128 + hf * 384 + at * 4, offE + IU_SIDE, AUX_SYS);
      if (total <= 126) break;
    }
  } else {
    #pragma unroll
    for (int j = 0; j < 4; ++j) ST128(x[j], rs, lane * 16 + j * 1024, par + EX_INR + uid * IU_RAW + hf * 4096, AUX_SYS);
    if (lane == 0) __builtin_amdgcn_raw_buffer_store_b32(0x80000000u, rs, hf * 128, offE + IU_SIDE, AUX_SYS);
  }
}

__device__ __forceinline__ u32 export_short(const u32x4 (&x)[8], rsrc_t rs, u32 par, u32 uid, LDS char* scr, int lane) {
  typedef unsigned short u16x2 __attribute__((ext_vector_type(2)));
  const u32 offE = par + EX_INE + uid * IU_ENC, offR = par + EX_INR + uid * IU_RAW;
  LDS char* var = scr + 8192;
  LDS char* tab = var + QV_ALL;
  u32 s = 0;
  #pragma unroll
  for (int j = 0; j < 8; ++j) s = __builtin_amdgcn_sad_u8(x[j][j & 3] & 0x7C7C7C7Cu, 0u, s);
  #pragma unroll
  for (int j = 0; j < 7; ++j) *(LDS u32x4*)(scr + j * 1024 + lane * 16) = u32x4{0u, 0u, 0u, 0u};
  *(LDS u32x4*)(var + lane * 16) = u32x4{0u, 0u, 0u, 0u};
  *(LDS u32x4*)(var + 1024 + lane * 16) = u32x4{0u, 0u, 0u, 0u};
  {
    u32 sel = 0x0C0C0C0Cu, at = 0;
    #pragma unroll
    for (int y = 0; y < 4; ++y) if ((lane >> y) & 1) { sel = (sel & ~(0xFFu << at)) | ((u32)y << at); at += 8; }
    if (lane < 16) *(LDS u32*)(tab + lane * 4) = sel;
  }
  s = (u32)__builtin_amdgcn_readlane((int)wave_scan_u32(s), 63);
  const int K = max(5, min(31, ((int)s >> 13) + 3));
  const u32 kw = (128u + 4u * (u32)K) * 0x01010101u, tabo = (u32)(uintptr_t)tab;
  auto front = [&](u32 F) { return *(LDS u32*)(uintptr_t)__builtin_amdgcn_udot4(F, 0x20100804u, tabo, false); };
  u32 cs[32], cn[32], car[20], mine = 0;
  wave_sync();
  #pragma unroll
  for (int g = 0; g < 4; ++g) {
    u32 a[8], uu[8];
    #pragma unroll
    for (int k = 0; k < 8; ++k) {
      const int d = g * 8 + k;
      const u32 xv = x[d >> 2][d & 3];
      const u32 u = uu[k] = (kw - (xv & 0x7C7C7C7Cu)) ^ 0x80808080u;
      const u32 esc = (((u >> 1) & 0x78787878u) + 0x78787878u) | ~((u | (u << 1)) << 4);
      const u32 F = (esc >> 7) & 0x01010101u;
      a[k] = bfi(0x0C0C0C0Cu, ~u | __builtin_bit_cast(u32, __builtin_bit_cast(u16x2, F) * (u16x2){12, 12}), xv);
      cs[d] = front(F);
      cn[d] = mine;
      mine = __builtin_amdgcn_udot4(F, 0x01010101u, mine, false);
    }
    car[g * 5 + 0] = bfi(0x70707070u, a[5] << 4, a[0]);
    car[g * 5 + 1] = bfi(0x70707070u, a[6] << 4, a[1]);
    car[g * 5 + 2] = bfi(0x70707070u, a[7] << 4, a[2]);
    car[g * 5 + 3] = bfi(0x40404040u, a[7] << 3, bfi(0x20202020u, a[6] << 2, bfi(0x10101010u, a[5] << 1, a[3])));
    car[g * 5 + 4] = bfi(0x40404040u, a[7] >> 1, bfi(0x20202020u, a[6] >> 2, bfi(0x10101010u, a[5] >> 3, a[4])));
    __builtin_amdgcn_sched_barrier(0);
    #pragma unroll
    for (int k = 0; k < 8; ++k) cs[g * 8 + k] = __builtin_amdgcn_perm(0u, uu[k], cs[g * 8 + k]);
  }
  #pragma unroll
  for (int j = 0; j < 5; ++j) ST128((u32x4{car[j * 4], car[j * 4 + 1], car[j * 4 + 2], car[j * 4 + 3]}), rs, lane * 16 + j * 1024, offE, AUX_SYS);
  const u32 inc = wave_scan_u32(mine);
  const u32 nesc = (u32)__builtin_amdgcn_readlane((int)inc, 63);
  bool fits = nesc <= Q_ESC;
  u32 lines = 1;
  if (fits) {
    const u32 base = inc - mine;
    #pragma unroll
    for (int d = 0; d < 32; ++d) {
      const u32 q = base + cn[d];
      const u64 v = (u64)cs[d] << ((q & 3u) * 8u);
      LDS u32* at = (LDS u32*)(scr + (q & ~3u));
      __hip_atomic_fetch_or(at, (u32)v, __ATOMIC_RELAXED, __HIP_MEMORY_SCOPE_WORKGROUP);
      __hip_atomic_fetch_or(at + 1, (u32)(v >> 32), __ATOMIC_RELAXED, __HIP_MEMORY_SCOPE_WORKGROUP);
    }
    wave_sync();
    const u32 t0 = (QV_SEC + (nesc + 31) / 32 * 8) * 8;
    u32 tbit = t0, npat = 0, nter = 0;
    for (u32 r0 = 0; r0 < nesc && fits; r0 += 2048) {
      const u32x4 e0 = *(LDS u32x4*)(scr + r0 + lane * 32), e1 = *(LDS u32x4*)(scr + r0 + lane * 32 + 16);
      u32 two[2] = {0u, 0u}, zq[8], zl[8], fs[8], t3[8], mk = 0, bits = 0, nt = 0;
      #pragma unroll
      for (int k = 0; k < 8; ++k) {
        const u32 u = k < 4 ? e0[k & 3] : e1[k & 3], sel = (u >> 2) & 0x07070707u;
        const u32 far = ((((u >> 1) & 0x70707070u) + 0x70707070u) >> 7) & 0x01010101u;
        const u32 c2 = __builtin_amdgcn_perm(0x03030201u, 0x03030300u, sel) | far | (far << 1);
        const u32 F = c2 & (c2 >> 1) & 0x01010101u;
        two[k >> 2] |= __builtin_amdgcn_udot4(c2, 0x40100401u, 0u, false) << (8 * (k & 3));
        fs[k] = front(F); t3[k] = ((sel + 0x02020202u) & 0x07070707u) + 0x01010101u;
        nt = __builtin_amdgcn_udot4(F, 0x01010101u, nt, false);
        const u32 lo7 = u & 0x7F7F7F7Fu;
        const u32 out = (F << 7) & (~(lo7 + 0x68686868u) | (lo7 + 0x48484848u) | u);
        mk |= out >> k;
      }
      __builtin_amdgcn_sched_barrier(0);
      #pragma unroll
      for (int k = 0; k < 8; ++k) {
        const u32 cum = __builtin_amdgcn_perm(0u, t3[k], fs[k]) * 0x01010101u;
        zl[k] = (1u << (cum & 31u)) | (1u << ((cum >> 8) & 31u)) | (1u << ((cum >> 16) & 31u)) | (1u << ((cum >> 24) & 31u));
        zq[k] = bits | ((cum >> 29) << 16);
        bits += cum >> 24;
      }
      if (r0 + lane * 32 < nesc) *(LDS u32x2*)(var + QV_SEC + r0 / 4 + lane * 8) = u32x2{two[0], two[1]};
      const u32 np = __popc(mk), inc2 = wave_scan_u32(bits | (np << 16)), tot = (u32)__builtin_amdgcn_readlane((int)inc2, 63);
      nter += (u32)__builtin_amdgcn_readlane((int)wave_scan_u32(nt), 63);
      fits = tbit + (tot & 0xFFFFu) + 96 <= QV_END * 8 && npat + (tot >> 16) <= Q_PAT && nter <= Q_TMAX;
      if (!fits) break;
      const u32 b0 = tbit + (inc2 & 0xFFFFu) - bits;
      if (bits) {
        #pragma unroll
        for (int k = 0; k < 8; ++k) {
          const u32 q = b0 + (zq[k] & 0xFFFFu);
          const u64 v = (((u64)(zq[k] >> 16) << 32) | zl[k]) << (q & 31u);
          LDS u32* at = (LDS u32*)(var + ((q >> 5) << 2));
          __hip_atomic_fetch_or(at, (u32)v, __ATOMIC_RELAXED, __HIP_MEMORY_SCOPE_WORKGROUP);
          __hip_atomic_fetch_or(at + 1, (u32)(v >> 32), __ATOMIC_RELAXED, __HIP_MEMORY_SCOPE_WORKGROUP);
        }
      }
      u32 slot = npat + (inc2 >> 16) - np;
      for (u32 m = mk; m; m &= m - 1) {
        const int bp = __builtin_ctz(m);
        const u32 sn = r0 + lane * 32 + (7 - (bp & 7)) * 4 + (bp >> 3);
        const int v = (int)*(LDS signed char*)(scr + sn) >> 2;
        *(LDS unsigned short*)(var + (slot < Q_PAT0 ? 8 + slot * 2 : QV_END - Q_PAT0 * 2 + slot * 2)) = (unsigned short)((sn << 3) | (u32)(((v - 6) >> 3) + 4));
        ++slot;
      }
      tbit += tot & 0xFFFFu; npat += tot >> 16;
    }
    if (fits) {
      if (lane == 0) *(LDS u32x2*)var = u32x2{nesc | (npat << 13) | ((u32)K << 24), tbit - t0 + 1};
      lines = tbit / 1024 + 1;
      wave_sync();
      if ((u32)lane < lines * 8) ST128(*(LDS u32x4*)(var + lane * 16), rs, lane * 16, offE + QV, AUX_SYS);
      if (lines > 8 && lane < (int)(QV_END - QV0) / 16) ST128(*(LDS u32x4*)(var + QV0 + lane * 16), rs, lane * 16, offE + QV + QV0, AUX_SYS);
      if (npat > Q_PAT0 && lane < (int)(QV_ALL - QV_END) / 16) ST128(*(LDS u32x4*)(var + QV_END + lane * 16), rs, lane * 16, offE + QV + QV_END, AUX_SYS);
    }
  }
  if (!fits) {
    store_raw_unit(x, rs, offR, lane);
    if (lane == 0) __builtin_amdgcn_raw_buffer_store_b32(0x80000000u, rs, 0, offE + QV, AUX_SYS);
  }
  return min(lines, 8u);
}

__device__ __forceinline__ void patch_short(LDS char* scr, u32 ent) {
  const u32 pos = ent >> 3;
  __hip_atomic_fetch_add((LDS u32*)(scr + (pos & ~3u)), ((4u - (ent & 7u)) << 5) << ((pos & 3u) * 8u), __ATOMIC_RELAXED, __HIP_MEMORY_SCOPE_WORKGROUP);
}
__device__ __forceinline__ void patch_in(LDS char* scr, u32 ent) {
  const u32 pos = ent >> 3;
  __hip_atomic_fetch_add((LDS u32*)(scr + (pos & ~3u)), (((ent & 7u) - 4u) << 5) << ((pos & 3u) * 8u), __ATOMIC_RELAXED, __HIP_MEMORY_SCOPE_WORKGROUP);
}

__device__ __forceinline__ float unfix(int i, int g) { return __builtin_amdgcn_ldexpf((float)i, -g); }
struct Side { u32 h[2], l[2], s, f, end; };
__device__ __forceinline__ Side side_map(u32 n0, u32 n1, u32 ns, u32 nf) {
  Side m;
  m.h[0] = 6; m.l[0] = m.h[0] + (256 + n0 + 7) / 8; m.h[1] = m.l[0] + (n0 + 1) / 2; m.l[1] = m.h[1] + (256 + n1 + 7) / 8;
  m.s = (m.l[1] + (n1 + 1) / 2 + 1) & ~1u; m.f = m.s + ns * 2; m.end = m.f + nf * 6;
  return m;
}

__global__ void __launch_bounds__(NTHR, 2) gemm_rs_kernel(Params p) {
  extern __shared__ __attribute__((aligned(16))) char smem_[];
  LDS char* smem = (LDS char*)smem_;
  const int tid = threadIdx.x, lane = tid & 63, wave = __builtin_amdgcn_readfirstlane(tid >> 6);
  const int b = blockIdx.x, rank = p.rank;
  const int lg = lane >> 4, li = lane & 15;
  const rsrc_t rsL = mk_rsrc(p.ws, p.ws_bytes);
  const u32 tick = (u32)__builtin_amdgcn_raw_ptr_buffer_atomic_add_i32(1, rsL, lane == 0 ? 0 : (int)0xF0000000u, OFF_CALL + b * 128 + wave * 16, 0);
  char* const peer_ws = p.ws_all[lane & 7];
  u32 call;

  if (b < 128) {
    const rsrc_t rsW = mk_rsrc(p.w, KS * NN);
    __builtin_amdgcn_s_sleep(12);
    #pragma unroll
    for (int i = 0; i < 8; ++i) {
      const int row = (wave * 8 + i) * 8 + (lane >> 3);
      const int sw = ((row >> 1) & 3) | (((row >> 4) & 1) << 2);
      __builtin_amdgcn_raw_ptr_buffer_load_lds(rsW, (LDS void*)(smem + (wave * 8 + i) * 1024), 16, row * NN + (((lane & 7) ^ sw) << 4), b * 128, 0, 0);
    }
    asm volatile("s_waitcnt vmcnt(0)\n\ts_barrier" ::: "memory");
    call = __builtin_amdgcn_readfirstlane(tick) + 1u;
    const int cgl = wave & 1, kb = wave >> 1;
    u32x4 x[8];
    #pragma unroll
    for (int f = 0; f < 4; ++f)
      #pragma unroll
      for (int h = 0; h < 2; ++h)
        #pragma unroll
        for (int t = 0; t < 2; ++t) {
          const int row = kb * 128 + 64 * h + 16 * lg + 8 * t + (li >> 1);
          const int sw = ((row >> 1) & 3) | (((row >> 4) & 1) << 2);
          const i32x2 v = __builtin_amdgcn_ds_read_tr8_b64_v2i32((LDS i32x2*)(smem + row * 128 + (((cgl * 4 + f) ^ sw) << 4) + (li & 1) * 8));
          x[f * 2 + h][t * 2] = v.x; x[f * 2 + h][t * 2 + 1] = v.y;
        }
    const int gl = b >> 5, gr = 4 + ((b >> 3) & 3), c = b & 7;
    u32 lines = 0;
    if (kb == 0 && c == 0) {
      #pragma unroll 1
      for (int hf = 0; hf < 2; ++hf) {
        const u32x4 xh[4] = {hf ? x[4] : x[0], hf ? x[5] : x[1], hf ? x[6] : x[2], hf ? x[7] : x[3]};
        export_half(xh, rsL, OFF_PAR + (call & 1) * PAR_BYTES, 256 + (b * 2 + cgl) * 4, hf, smem + L_SCR + wave * SCR_W, lane);
      }
    } else lines = export_short(x, rsL, OFF_PAR + (call & 1) * PAR_BYTES, 256 + (b * 2 + cgl) * 4 + kb, smem + L_SCR + wave * SCR_W, lane);
    vm_drain();
    if ((lane == gl || lane == gr) && lane != rank) {
      const int cb = (lane == gl ? gr & 3 : gl) * 8 + c;
      __hip_atomic_store((GLB u32*)(peer_ws + OFF_FIN + (rank * 576 + 256 + (cb * 2 + cgl) * 4 + kb) * 128), call << 4 | lines, __ATOMIC_RELAXED, __HIP_MEMORY_SCOPE_SYSTEM);
    }
    if (rank == gl || rank == gr) {
      const int cb = (rank == gl ? gr & 3 : gl) * 8 + c;
      store_raw_unit(x, rsL, OFF_WA + ((rank * 4 + kb) * 64 + cb * 2 + cgl) * IU_RAW, lane);
      vm_drain();
      cnt_add(p.ws, (64 + cb) * 5 + 4, lane);
    }
  } else if (b < 192) {
    const int rg = b - 128, kb = wave & 3, hf = wave >> 2, u = rg * 4 + kb;
    const rsrc_t rsX = mk_rsrc(p.x, MM * KS);
    const u32 both = (u32)(uintptr_t)(smem + L_CTL) + 16;
    const u32 xoff = OFF_XA + ((rank * 4 + kb) * 64 + rg) * IU_RAW;
    const int ow = rg >> 3, r8 = rg & 7;
    const u32 users = ((r8 <= 3 ? 0xFu << (ow & 4) : 0u) | (r8 >= 3 ? 0xFu << (~ow & 4) : 0u) | (1u << ow)) & ~(1u << rank);
    const bool tell = lane < NR && ((users >> lane) & 1);
    if (kb < 2) {
      u32x4 x[4];
      #pragma unroll
      for (int j = 0; j < 4; ++j)
        x[j] = __builtin_amdgcn_raw_buffer_load_b128(rsX, ((j >> 1) * 16 + li) * KS + (j & 1) * 64 + lg * 16, (rg * 64 + hf * 32) * KS + kb * 128, 0);
      if (wave == 0) ctl_st(both + (lane & 3) * 4, 0u);
      asm volatile("s_waitcnt lgkmcnt(0)\n\ts_barrier" ::: "memory");
      call = __builtin_amdgcn_readfirstlane(tick) + 1u;
      const u32 parw = OFF_PAR + (call & 1) * PAR_BYTES;
      auto second = [&](int k) -> bool {
        u32 old = 0;
        if (lane == 0) old = __hip_atomic_fetch_add((LDS u32*)(uintptr_t)(both + k * 4), 1u, __ATOMIC_RELAXED, __HIP_MEMORY_SCOPE_WORKGROUP);
        return __builtin_amdgcn_readfirstlane((int)old) == 1;
      };
      if (NRAW && kb == 0) {
        #pragma unroll
        for (int j = 0; j < 4; ++j) ST128(x[j], rsL, lane * 16 + j * 1024, parw + EX_INR + u * IU_RAW + hf * 4096, AUX_SYS);
        vm_drain();
        if (second(2) && tell) __hip_atomic_store((GLB u32*)(peer_ws + OFF_FIN + (rank * 576 + 512 + rg) * 128), call << 4, __ATOMIC_RELAXED, __HIP_MEMORY_SCOPE_SYSTEM);
      }
      export_half(x, rsL, parw, u, hf, smem + L_SCR + wave * SCR_W, lane);
      vm_drain();
      if (second(kb) && tell) __hip_atomic_store((GLB u32*)(peer_ws + OFF_FIN + (rank * 576 + u) * 128), call << 4, __ATOMIC_RELAXED, __HIP_MEMORY_SCOPE_SYSTEM);
      #pragma unroll
      for (int j = 0; j < 4; ++j) ST128(x[j], rsL, lane * 16 + j * 1024, xoff + hf * 4096, AUX_SYS);
      vm_drain();
      cnt_add(p.ws, rg * 5 + 4, lane);
    } else if (hf == 0) {
      u32x4 x[8];
      #pragma unroll
      for (int j = 0; j < 8; ++j)
        x[j] = __builtin_amdgcn_raw_buffer_load_b128(rsX, ((j >> 1) * 16 + li) * KS + (j & 1) * 64 + lg * 16, rg * 64 * KS + kb * 128, 0);
      asm volatile("s_barrier" ::: "memory");
      call = __builtin_amdgcn_readfirstlane(tick) + 1u;
      const u32 lines = export_short(x, rsL, OFF_PAR + (call & 1) * PAR_BYTES, u, smem + L_SCR + wave * SCR_W, lane);
      vm_drain();
      if (tell) __hip_atomic_store((GLB u32*)(peer_ws + OFF_FIN + (rank * 576 + u) * 128), call << 4 | lines, __ATOMIC_RELAXED, __HIP_MEMORY_SCOPE_SYSTEM);
      store_raw_unit(x, rsL, xoff, lane);
      vm_drain();
      cnt_add(p.ws, rg * 5 + 4, lane);
    } else {
      asm volatile("s_barrier" ::: "memory");
    }
  }
  call = __builtin_amdgcn_readfirstlane(tick) + 1u;
  const u32 par = OFF_PAR + (call & 1) * PAR_BYTES;
  const float scale = p.sx[0] * p.sw[0];
  const rsrc_t rsO = mk_rsrc(p.out, 512u * NN * 4u);

  if (b >= NGEMM) {
    const int pb = b - NGEMM, link = pb / PULL_WGS, q = pb % PULL_WGS;
    const int src = (rank + 1 + link) & 7;
    const u64 pw = (u64)peer_ws;
    const u32 pw_lo = (u32)__builtin_amdgcn_readlane((int)(u32)pw, src), pw_hi = (u32)__builtin_amdgcn_readlane((int)(u32)(pw >> 32), src);
    const rsrc_t rsP = mk_rsrc((const void*)(((u64)pw_hi << 32) | (u64)pw_lo), p.ws_bytes);
    const u32 ctl = (u32)(uintptr_t)(smem + L_CTL);
    const u32 C_READY = ctl, C_LANDED = ctl + 4, C_CLOSED = ctl + 12, C_REL = ctl + 16, C_ISSUED = ctl + 8, C_Q = ctl + 64;
    const bool same = (src >> 2) == (rank >> 2);
    if (wave == 0) ctl_st(ctl + lane * 4, 0u);
    asm volatile("s_waitcnt lgkmcnt(0)\n\ts_barrier" ::: "memory");
    if (wave == 0) {
      int g = 0, pub = 0;
      #define LANDED(v) do { if ((v) > pub) { pub = (v); ctl_st(C_LANDED, (u32)pub); } } while (0)
      #define PIECEW(W, SZ, VOFF, OFF) do { __builtin_amdgcn_raw_ptr_buffer_load_lds(rsP, (LDS void*)(dst + (OFF)), SZ, VOFF, so + (OFF), 0, AUX_SYS); \
        asm volatile("s_waitcnt vmcnt(%0)" :: "i"((W) - 1) : "memory"); ++g; LANDED(g - ((W) - 1)); } while (0)
      #define PIECE(SZ, VOFF, OFF) PIECEW(WIN, SZ, VOFF, OFF)
      for (u32 n = 0;; ++n) {
        if (ctl_ld(C_READY) <= n) {
          vm_drain(); LANDED(g);
          bool fin = false;
          while (ctl_ld(C_READY) <= n) { if (ctl_ld(C_CLOSED)) { fin = ctl_ld(C_READY) <= n; break; } __builtin_amdgcn_s_sleep(1); }
          if (fin) break;
        }
        const u32 ent = ctl_ld(C_Q + (n & 15) * 4);
        ctl_st(C_ISSUED, n + 1);
        if (ent >> 31) continue;
        const u32 need = n >= NSLOT ? n - NSLOT + 1 : 0, ra = C_REL + (n % NSLOT) * 4;
        if (ctl_ld(ra) < need) {
          vm_drain(); LANDED(g);
          while (ctl_ld(ra) < need) __builtin_amdgcn_s_sleep(1);
        }
        const int i = ent & 1023;
        LDS char* dst = smem + (n % NSLOT) * SLOT;
        if (ent & 1024) {
          const u32 so = par + EX_INR + i * IU_RAW;
          #pragma unroll
          for (int j = 0; j < 8; ++j) PIECE(16, lane * 16, j * 1024);
        } else if (i < 512 && quick_form(i)) {
          const u32 so = par + EX_INE + (i < 256 ? i : 256 + group_of(rank, (i >> 6) & 3) * 64 + (i & 63)) * IU_ENC;
          PIECE(4, lane * 4, IU_SIDE);
          #pragma unroll
          for (int j = 0; j < 6; ++j) PIECE(16, lane * 16, j * 1024);
        } else if (i < 512) {
          const u32 so = par + EX_INE + (i < 256 ? i : 256 + group_of(rank, (i >> 6) & 3) * 64 + (i & 63)) * IU_ENC;
          if ((u32)lane < ((ent >> 24) & 15u) * 8) __builtin_amdgcn_raw_ptr_buffer_load_lds(rsP, (LDS void*)(dst + QV), 16, lane * 16, so + QV, 0, AUX_SYS);
          asm volatile("s_waitcnt vmcnt(%0)" :: "i"(WIN - 1) : "memory"); ++g; LANDED(g - (WIN - 1));
          #pragma unroll
          for (int j = 0; j < 5; ++j) PIECE(16, lane * 16, j * 1024);
        } else {
          const u32 so = par + EX_OUE + (rank * 320 + i - 512) * OU_ENC;
          if ((u32)lane < 32 + ((ent >> 24) & 15u) * 8)
            __builtin_amdgcn_raw_ptr_buffer_load_lds(rsP, (LDS void*)(dst + 11264), 16, lane * 16, so + 11264, 0, AUX_SYS);
          asm volatile("s_waitcnt vmcnt(%0)" :: "i"(WIN_OUT - 1) : "memory"); ++g; LANDED(g - (WIN_OUT - 1));
          #pragma unroll
          for (int j = 0; j < 11; ++j) PIECEW(WIN_OUT, 16, lane * 16, j * 1024);
        }
      }
      vm_drain(); LANDED(g);
      #undef PIECE
      #undef PIECEW
      #undef LANDED
      return;
    }
    if (wave == 1) {
      const u32 tic0 = OFF_TIC + (call & 1) * 24 * 128;
      const int only0 = lane == 0 ? 0 : (int)0xF0000000u;
      if (q == 0 && (lane == link || (link == 0 && lane >= 8 && lane < 24)))
        __hip_atomic_store((GLB u32*)(p.ws + OFF_TIC + ((~call & 1) * 24 + lane) * 128), 0u, __ATOMIC_RELAXED, __HIP_MEMORY_SCOPE_AGENT);
      constexpr u32 NPAIR = 208 + NRAW / 2, NFIX = 192;
      u32 n = 0, endp = 0; bool ok = true;
      const u32 gtic = tic0 + (8 + group_of(src, lane & 3)) * 128;
      auto fullest = [&](u32 v) -> int {
        v = v >= 16u || (!same && (lane & 3) == (rank & 3)) ? 0xFFFFu : (v << 4) | ((((lane - q) & 3) << 2) | (lane & 3));
        v = min(v, (u32)__builtin_amdgcn_mov_dpp((int)v, 0xB1, 0xf, 0xf, true));
        v = min(v, (u32)__builtin_amdgcn_mov_dpp((int)v, 0x4E, 0xf, 0xf, true));
        v = (u32)__builtin_amdgcn_readfirstlane((int)v);
        return v == 0xFFFFu ? -2 : (int)(v & 3u);
      };
      auto draw = [&](int g) { return __builtin_amdgcn_raw_ptr_buffer_atomic_add_i32(1, rsL, only0, tic0 + (g < 0 ? link : 8 + group_of(src, g)) * 128, 0); };
      int tnext = draw(-1);
      for (;;) {
        const u32 t = (u32)__builtin_amdgcn_readfirstlane(tnext);
        if (t >= NPAIR + NFIX) break;
        while (n >= ctl_ld(C_ISSUED) + (t < NPAIR ? LOOK : LOOK_OUT)) __builtin_amdgcn_s_sleep(1);
        const bool last = t + PULL_WGS >= NPAIR + NFIX;
        if (!last) tnext = draw(-1);
        int i;
        if (t < NPAIR) {
          const int e = t < NRAW ? t : 2 * t - NRAW + (lane & 1);
          int u = -1, sg = 0, kb = 0;
          if (e < 32 + X1FIRST) { kb = e >= NRAW && e < NRAW + X1FIRST; u = e < NRAW ? e : kb ? e - NRAW : e - X1FIRST; }
          else if (e < 40 + X1FIRST) sg = 0;
          else if (e < 72) { kb = 1; u = e - 40; }
          else if (e < 88) sg = (e - 64) >> 3;
          else if (e < 120) { kb = 2; u = e - 88; }
          else if (e < 144) sg = (e - 96) >> 3;
          else if (e < 176) { kb = 3; u = e - 144; }
          else sg = (e - 128) >> 3;
          int c = 0;
          if (u < 0) {
            if (sg < 6) { c = (0x210100 >> (sg * 4)) & 7; kb = (0x012010 >> (sg * 4)) & 3; }
            else if (sg < 26) { c = ((sg - 6) >> 2) + ((sg - 6) & 3); kb = 3 - ((sg - 6) & 3); }
            else { c = (0x776765 >> ((sg - 26) * 4)) & 7; kb = (0x323123 >> ((sg - 26) * 4)) & 3; }
          }
          const int m = c * 8 + (e & 7);
          const int v = u - 12;
          if (e >= 384) i = (8 * rank + (e & 7)) * 4 + ((e - 384) >> 3);
          else if (u < 0) i = 256 + ((((m >> 1) & 3) * 8 + (m >> 3)) * 2 + (m & 1)) * 4 + kb;
          else if (u < 12) i = (8 * ((rank & 4) | ((rank + 1 + (u >> 2)) & 3)) + (u & 3)) * 4 + kb;
          else i = (8 * ((~rank & 4) | (v / 5)) + 3 + v % 5) * 4 + kb;
        } else {
          const int o = t - NPAIR, cm = same ? o / 3 : o >> 2, k = same ? o - cm * 3 : 1 + (o & 3);
          const int c = same ? cm >> 3 : cm / 6, j3 = (cm - c * 6) >> 1, j = same ? (cm >> 1) & 3 : j3 + (j3 >= (rank & 3));
          i = 512 + ((j * 8 + c) * 2 + (cm & 1)) * 5 + k;
        }
        const bool unp = t < NRAW, two = !unp && t < NPAIR;
        const u32 fo = i >= 512 ? OFF_FOUT + (src * 320 + i - 512) * 128 : OFF_FIN + (src * 576 + (unp ? 512 + (i >> 2) : i)) * 128;
        u32 got = 0;
        ok = ok && wait_ge(p.ws, fo, call << 4, call, got);
        const u32 lines = got >> 4 == call ? got & 15u : 8u;
        const u32 k = two ? (lane & 1) : 0, per = !ok ? 0 : t >= NPAIR ? 12 : unp ? 8 : (u32)__builtin_amdgcn_readfirstlane(quick_form(i) ? 7 : 6);
        ctl_st(C_Q + ((n + k) & 15) * 4, (u32)i | (unp ? 1024u : 0u) | ((endp + per * (k + 1)) << 11) | (lines << 24) | (ok ? 0u : 0x80000000u));
        n += two ? 2 : 1; endp += two ? 2 * per : per;
        asm volatile("s_waitcnt lgkmcnt(0)" ::: "memory");
        ctl_st(C_READY, n);
        if (last) break;
      }
      const u32 fo3 = OFF_FOUT + (src * 320 + lane * 5 + (same ? 3 : 0)) * 128;
      const u32 flags = ld_flag(p.ws, fo3);
      u32 seen = ld_flag(p.ws, gtic);
      for (;;) {
        while (n >= ctl_ld(C_ISSUED) + LOOK_END) __builtin_amdgcn_s_sleep(1);
        const int fj = fullest(seen);
        if (fj < 0) break;
        const int tk = draw(fj);
        seen = ld_flag(p.ws, gtic);
        const u32 t = (u32)__builtin_amdgcn_readfirstlane(tk);
        if (t >= 16u) continue;
        const int cg = fj * 16 + (int)t;
        u32 got = (u32)__builtin_amdgcn_readlane((int)flags, cg);
        if (ok && got < call << 4) ok = wait_ge(p.ws, OFF_FOUT + (src * 320 + cg * 5 + (same ? 3 : 0)) * 128, call << 4, call, got);
        if (ok) endp += 12;
        ctl_st(C_Q + (n & 15) * 4, (u32)(512 + cg * 5 + (same ? 3 : 0)) | (endp << 11) | ((got >> 4 == call ? got & 15u : 8u) << 24) | (ok ? 0u : 0x80000000u));
        ++n;
        asm volatile("s_waitcnt lgkmcnt(0)" ::: "memory");
        ctl_st(C_READY, n);
      }
      asm volatile("s_waitcnt lgkmcnt(0)" ::: "memory");
      ctl_st(C_CLOSED, 1u);
      return;
    }
    LDS char* scr = smem + L_PSCR + (wave - 2) * 8 * KiB;
    for (u32 n = wave - 2;; n += NCONS) {
      bool fin = false;
      while (ctl_ld(C_READY) <= n) { if (ctl_ld(C_CLOSED)) { fin = ctl_ld(C_READY) <= n; break; } __builtin_amdgcn_s_sleep(1); }
      if (fin) break;
      const u32 ent = ctl_ld(C_Q + (n & 15) * 4);
      const bool ok = !(ent >> 31);
      const int i = ent & 1023;
      const u32 endp = (ent >> 11) & 0x1FFFu;
      auto landed = [&](u32 upto) { while (ctl_ld(C_LANDED) < upto) __builtin_amdgcn_s_sleep(1); };
      LDS char* sl = smem + (n % NSLOT) * SLOT;
      if (i < 512) {
        if (!ok) continue;
        const bool brief = !(ent & 1024) && !quick_form(i);
        landed(brief ? endp - 5 : endp);
        const int kb = i & 3, rg = (i >> 2) & 63; const bool isx = i < 256;
        const u32 uid = isx ? i : 256 + group_of(rank, (i >> 6) & 3) * 64 + (i & 63);
        const u32 dst = (isx ? OFF_XA : OFF_WA) + ((src * 4 + kb) * 64 + rg) * IU_RAW;
        const u32 rel = C_REL + (n % NSLOT) * 4, offE = par + EX_INE + uid * IU_ENC;
        u32x4 x[8];
        bool raw = false;
        if (ent & 1024) {
          #pragma unroll
          for (int j = 0; j < 8; ++j) x[j] = *(LDS u32x4*)(sl + j * 1024 + lane * 16);
          wave_sync();
          ctl_st(rel, n + 1);
        } else if (quick_form(i)) {
          u32x4 c[6];
          #pragma unroll
          for (int j = 0; j < 6; ++j) c[j] = *(LDS u32x4*)(sl + j * 1024 + lane * 16);
          const u32 sd = *(LDS u32*)(sl + IU_SIDE + lane * 4);
          wave_sync();
          ctl_st(rel, n + 1);
          const u32 hd[2] = {(u32)__builtin_amdgcn_readlane((int)sd, 0), (u32)__builtin_amdgcn_readlane((int)sd, 32)};
          #pragma unroll
          for (int j = 0; j < 8; ++j) {
            const u32 e4 = ((hd[j >> 2] >> 16) & 31u) * 0x04040404u;
            const u32 p0 = c[(j * 3) >> 2][(j * 3) & 3], p1 = c[(j * 3 + 1) >> 2][(j * 3 + 1) & 3], p2 = c[(j * 3 + 2) >> 2][(j * 3 + 2) & 3];
            const u32 o3 = bfi(0x80808080u, p2 << 1, bfi(0x10101010u, p2 >> 1, bfi(0x0C0C0C0Cu, p1 >> 3, p0 >> 5)));
            const u32x4 y = {(p0 & 0x9F9F9F9Fu) + e4, (p1 & 0x9F9F9F9Fu) + e4, (p2 & 0x9F9F9F9Fu) + e4, (o3 & 0x9F9F9F9Fu) + e4};
            *(LDS u32x4*)(scr + j * 1024 + lane * 16) = y;
          }
          wave_sync();
          const u32 cnt = (lane < 32 ? hd[0] : hd[1]) & 0xFFFFu, id0 = 2u * (lane & 31) - 2u;
          if (lane & 31) {
            if (id0 < cnt) patch_in(scr, sd & 0xFFFFu);
            if (id0 + 1 < cnt) patch_in(scr, sd >> 16);
          }
          if (max(hd[0] & 0xFFFFu, hd[1] & 0xFFFFu) > IN_CAP0) {
            for (int j = 0; j < 3; ++j) {
              const u32 s2 = __builtin_amdgcn_raw_buffer_load_b32(rsP, 256 + (lane >> 5) * 384 + (j * 32 + (lane & 31)) * 4, offE + IU_SIDE, AUX_SYS);
              const u32 id = IN_CAP0 + j * 64 + 2 * (lane & 31);
              if (id < cnt) patch_in(scr, s2 & 0xFFFFu);
              if (id + 1 < cnt) patch_in(scr, s2 >> 16);
            }
          }
          wave_sync();
          #pragma unroll
          for (int j = 0; j < 8; ++j) x[j] = *(LDS u32x4*)(scr + j * 1024 + lane * 16);
          #pragma unroll
          for (int h = 0; h < 2; ++h)
            if (hd[h] >> 31) {
              #pragma unroll
              for (int j = h * 4; j < h * 4 + 4; ++j) x[j] = __builtin_amdgcn_raw_buffer_load_b128(rsP, lane * 16 + j * 1024, par + EX_INR + uid * IU_RAW, AUX_SYS);
            }
        } else {
          LDS char* var = sl + QV;
          const u32 hdr = (u32)__builtin_amdgcn_readfirstlane((int)*(LDS u32*)var);
          const u32 pe = *(LDS unsigned short*)(var + 8 + (lane & 7) * 2);
          const u32 tlen = (u32)__builtin_amdgcn_readfirstlane((int)*(LDS u32*)(var + 4));
          raw = hdr >> 31;
          if (!raw) {
            const u32 nesc = hdr & 0x1FFFu, npat = (hdr >> 13) & 0x1FFu, bias = (4u * ((hdr >> 24) & 31u) - 136u) * 0x01010101u;
            const u32 t0 = QV_SEC + (nesc + 31) / 32 * 8;
            if (t0 * 8 + tlen > QV0 * 8 && lane < (int)(QV_END - QV0) / 16)
              *(LDS u32x4*)(var + QV0 + lane * 16) = __builtin_amdgcn_raw_buffer_load_b128(rsP, lane * 16, offE + QV + QV0, AUX_SYS);
            wave_sync();
            u32 tn = (u32)(uintptr_t)scr + Q_TER;
            for (u32 r0 = 0; r0 < tlen; r0 += 1024) {
              const u32 bit0 = r0 + lane * 16;
              const u32 before = *(LDS unsigned short*)(var + t0 + bit0 / 8 - 2);
              u32 m = bit0 < tlen ? *(LDS unsigned short*)(var + t0 + bit0 / 8) : 0u;
              int prev = 15 - (int)__clz(before);
              if (bit0 == 0) { m &= ~1u; prev = 0; }
              const u32 mine = __popc(m), inc = wave_scan_u32(mine);
              LDS unsigned char* at = (LDS unsigned char*)(uintptr_t)(tn + inc - mine);
              for (; m; m &= m - 1) { const int pl = __builtin_ctz(m); *at++ = (unsigned char)((pl - prev) * 4 - 4); prev = pl; }
              tn += (u32)__builtin_amdgcn_readlane((int)inc, 63);
            }
            tn = (u32)(uintptr_t)scr + Q_TER;
            wave_sync();
            for (u32 r0 = 0; r0 < nesc; r0 += 2048) {
              const u32x2 two = *(LDS u32x2*)(var + QV_SEC + r0 / 4 + lane * 8);
              u32 c2[8], e2[8], zo[8], bits = 0;
              #pragma unroll
              for (int k = 0; k < 8; ++k) {
                const u32 z = two[k >> 2] >> (8 * (k & 3)), z2 = bfi(0x0000000Fu, z, z << 12);
                c2[k] = bfi(0x00030003u, z2, z2 << 6) & 0x03030303u;
                e2[k] = (c2[k] << 2) & (c2[k] << 1) & 0x04040404u;
                zo[k] = bits;
                bits = __builtin_amdgcn_udot4(e2[k], 0x01010101u, bits, false);
              }
              const u32 inc = wave_scan_u32(bits >> 2), b4 = (tn + inc) * 4 - bits;
              tn += (u32)__builtin_amdgcn_readlane((int)inc, 63);
              u32 g0[8], g1[8];
              #pragma unroll
              for (int k = 0; k < 8; ++k) {
                zo[k] = (zo[k] + b4) >> 2;
                const LDS u32* at = (const LDS u32*)(uintptr_t)(zo[k] & ~3u);
                g0[k] = at[0]; g1[k] = at[1];
              }
              __builtin_amdgcn_sched_barrier(0);
              #pragma unroll
              for (int k = 0; k < 8; ++k) {
                const u32 nx = __builtin_amdgcn_alignbyte(g1[k], g0[k], zo[k]);
                zo[k] = __builtin_amdgcn_perm(0u, 0x64686C7Cu, c2[k]) - __builtin_amdgcn_perm(nx, 0u, __umul24(e2[k], 0x404040u) + e2[k]);
              }
              *(LDS u32x4*)(scr + r0 + lane * 32) = u32x4{zo[0], zo[1], zo[2], zo[3]};
              *(LDS u32x4*)(scr + r0 + lane * 32 + 16) = u32x4{zo[4], zo[5], zo[6], zo[7]};
            }
            wave_sync();
            if ((u32)lane < npat && lane < (int)Q_PAT0) patch_short(scr, pe);
            if (npat > Q_PAT0) {
              for (int j = 0; j < 2; ++j) {
                const u32 s2 = __builtin_amdgcn_raw_buffer_load_b32(rsP, lane * 4 + j * 256, offE + QV + QV_END, AUX_SYS);
                const u32 id = Q_PAT0 + j * 128 + 2 * lane;
                if (id < npat) patch_short(scr, s2 & 0xFFFFu);
                if (id + 1 < npat) patch_short(scr, s2 >> 16);
              }
            }
            landed(endp);
            u32x4 c[5];
            #pragma unroll
            for (int j = 0; j < 5; ++j) c[j] = *(LDS u32x4*)(sl + j * 1024 + lane * 16);
            wave_sync();
            ctl_st(rel, n + 1);
            u32 a[32], es[32], qo[32], cnt4 = 0;
            #pragma unroll
            for (int g = 0; g < 4; ++g) {
              u32 k5[5];
              #pragma unroll
              for (int k = 0; k < 5; ++k) { k5[k] = c[(g * 5 + k) >> 2][(g * 5 + k) & 3]; a[g * 8 + k] = k5[k]; }
              a[g * 8 + 5] = bfi(0x80808080u, k5[4] << 3, bfi(0x08080808u, k5[3] >> 1, k5[0] >> 4));
              a[g * 8 + 6] = bfi(0x80808080u, k5[4] << 2, bfi(0x08080808u, k5[3] >> 2, k5[1] >> 4));
              a[g * 8 + 7] = bfi(0x80808080u, k5[4] << 1, bfi(0x08080808u, k5[3] >> 3, k5[2] >> 4));
            }
            #pragma unroll
            for (int d = 0; d < 32; ++d) {
              es[d] = a[d] & (a[d] >> 1) & 0x04040404u;
              qo[d] = cnt4;
              cnt4 = __builtin_amdgcn_udot4(es[d], 0x01010101u, cnt4, false);
            }
            const u32 base4 = ((u32)(uintptr_t)scr + wave_scan_u32(cnt4 >> 2)) * 4 - cnt4;
            wave_sync();
            #pragma unroll
            for (int h = 0; h < 2; ++h) {
              u32 g0[16], g1[16];
              #pragma unroll
              for (int k = 0; k < 16; ++k) {
                qo[h * 16 + k] = (base4 + qo[h * 16 + k]) >> 2;
                const LDS u32* at = (const LDS u32*)(uintptr_t)(qo[h * 16 + k] & ~3u);
                g0[k] = at[0]; g1[k] = at[1];
              }
              __builtin_amdgcn_sched_barrier(0);
              #pragma unroll
              for (int k = 0; k < 16; ++k) {
                const int d = h * 16 + k;
                const u32 nx = __builtin_amdgcn_alignbyte(g1[k], g0[k], qo[d]);
                const u32 sel = __umul24(es[d], 0x404040u) + es[d];
                x[d >> 2][d & 3] = (a[d] & 0x8F8F8F8Fu) + __builtin_amdgcn_perm(nx, 0x7C7C7C7Cu, sel) + bias;
              }
              __builtin_amdgcn_sched_barrier(0);
            }
          } else {
            landed(endp);
            ctl_st(rel, n + 1);
          }
        }
        if (raw) {
          #pragma unroll
          for (int j = 0; j < 8; ++j) x[j] = __builtin_amdgcn_raw_buffer_load_b128(rsP, lane * 16 + j * 1024, par + EX_INR + uid * IU_RAW, AUX_SYS);
        }
        store_raw_unit(x, rsL, dst, lane);
        vm_drain();
        cnt_add(p.ws, (isx ? rg : 64 + (rg >> 1)) * 5 + kb, lane);
        continue;
      }
      const int t = i - 512, cg = t / 5, rg = t - cg * 5 + (same ? 0 : 3);
      const u32 obase = (rg * 64 * NN + group_of(src, cg >> 4) * 1024 + (cg & 15) * 64) * 4;
      const u32 olane = (li * NN + lg * 4) * 4;
      if (!ok) {
        const u32x4 nan4 = {0x7FC00000u, 0x7FC00000u, 0x7FC00000u, 0x7FC00000u};
        #pragma unroll
        for (int e = 0; e < 16; ++e) ST128(nan4, rsO, olane + (e & 3) * 64, obase + (e >> 2) * 16 * NN * 4, AUX_SYS);
        continue;
      }
      #pragma unroll
      for (int j = 0; j < 4; ++j) *(LDS u32x4*)(scr + j * 1024 + lane * 16) = u32x4{0u, 0u, 0u, 0u};
      landed(endp - 11);
      LDS char* side = sl + OU_SIDE;
      const u32 hdr = (u32)__builtin_amdgcn_readfirstlane((int)*(LDS u32*)side);
      const u32 hd2 = (u32)__builtin_amdgcn_readfirstlane((int)*(LDS unsigned short*)(side + 4));
      const u32 tl[2] = {*(LDS u32*)(sl + 11264 + lane * 4), *(LDS u32*)(sl + 11520 + lane * 4)};
      const u32 uid = rank * 320 + t;
      if (!(hdr >> 31)) {
        const u32 ns = hd2 & 0xFFu, nf = hd2 >> 8; const int g = (int)((hdr >> 20) & 0xFFu);
        const Side sm = side_map(hdr & 1023u, (hdr >> 10) & 1023u, ns, nf);
        const u32 se = *(LDS unsigned short*)(side + sm.s + lane * 2);
        const u32x2 fe = {*(LDS unsigned short*)(side + sm.f + (lane & 31) * 6),
                          *(LDS unsigned short*)(side + sm.f + (lane & 31) * 6 + 2) | ((u32)*(LDS unsigned short*)(side + sm.f + (lane & 31) * 6 + 4) << 16)};
        if ((u32)lane < ns)
          __hip_atomic_fetch_add((LDS u32*)(scr + ((se >> 5) & 63u) * 64 + (((se >> 11) & 15u) ^ 7u) * 4),
                                 ((se & 7u) << 5) << (se & 24u), __ATOMIC_RELAXED, __HIP_MEMORY_SCOPE_WORKGROUP);
        u32 done = 0;
        for (u32 c = lane; c - lane < max(sm.l[0] - sm.h[0], sm.l[1] - sm.h[1]); c += 64) {
          u32 m[2], k[2], nib[2];
          #pragma unroll
          for (int z = 0; z < 2; ++z) m[z] = c < sm.l[z] - sm.h[z] ? *(LDS unsigned char*)(side + sm.h[z] + c) : 0u;
          const u32 mine = __popc(m[0]) | (__popc(m[1]) << 16), inc = wave_scan_u32(mine);
          k[0] = (done + inc - mine) & 0xFFFFu; k[1] = (done + inc - mine) >> 16;
          #pragma unroll
          for (int z = 0; z < 2; ++z) {
            const u32 nb = ((u32)(uintptr_t)side + sm.l[z]) * 8 + k[z] * 4;
            const LDS u32* at = (const LDS u32*)(uintptr_t)((nb >> 3) & ~3u);
            nib[z] = __builtin_amdgcn_alignbit(at[1], at[0], nb);
          }
          #pragma unroll
          for (int z = 0; z < 2; ++z)
            for (u32 d = c * 8 - k[z]; m[z]; m[z] &= m[z] - 1, --d, nib[z] >>= 4) {
              const u32 hi = (d + (u32)__builtin_ctz(m[z])) & 255u;
              __hip_atomic_fetch_add((LDS u32*)(scr + (hi >> 1) * 32 + (nib[z] & 7u) * 4), (z ? 0x80u : 0x02u) << ((nib[z] & 8u) | (hi & 1u) << 4),
                                     __ATOMIC_RELAXED, __HIP_MEMORY_SCOPE_WORKGROUP);
            }
          done += (u32)__builtin_amdgcn_readlane((int)inc, 63);
        }
        const u32 fgq = (u32)lane < nf ? fe.x >> 8 : 16u;
        u32 th[2] = {0u, 0u};
        wave_sync();
        #pragma unroll
        for (int mi = 0; mi < 4; ++mi) {
          landed(mi < 3 ? endp - 8 + 3 * mi : endp);
          u32x4 c[3];
          #pragma unroll
          for (int j = 0; j < (mi < 3 ? 3 : 2); ++j) c[j] = *(LDS u32x4*)(sl + (mi * 3 + j) * 1024 + lane * 16);
          const u32x4 adj = *(LDS u32x4*)(scr + lane * 64 + (mi ^ 1) * 16);
          if (mi == 3) { wave_sync(); ctl_st(C_REL + (n % NSLOT) * 4, n + 1); }
          #pragma unroll
          for (int ni = 0; ni < 4; ++ni) {
            const int k = ni * 3, gq = mi * 4 + ni, be = gq % 7;
            const u32 l0 = gq < 15 ? c[k >> 2][k & 3] : tl[0], l1 = gq < 15 ? c[(k + 1) >> 2][(k + 1) & 3] : tl[1];
            const u32 tp = gq < 14 ? c[(k + 2) >> 2][(k + 2) & 3] : th[gq & 1];
            if (gq < 14) th[gq / 7] = be < 6 ? bfi(0x01010101u << be, tp >> (6 - be), th[gq / 7]) : bfi(0x80808080u, tp << 1, th[gq / 7]);
            const u32 top = bfi(0x40404040u, (tp >> 1) ^ (adj[3 - ni] << 5), tp), low = adj[3 - ni] & 0xE0E0E0E0u;
            const u32 q01 = __builtin_amdgcn_perm(top, low, 0x05010400u), q23 = __builtin_amdgcn_perm(top, low, 0x07030602u);
            float y[4] = {unfix((int)__builtin_amdgcn_perm(q01, l0, 0x05010004u), g + 7), unfix((int)__builtin_amdgcn_perm(q01, l0, 0x07030206u), g + 7),
                          unfix((int)__builtin_amdgcn_perm(q23, l1, 0x05010004u), g + 7), unfix((int)__builtin_amdgcn_perm(q23, l1, 0x07030206u), g + 7)};
            if (nf)
              for (u64 em = __ballot(fgq == (u32)gq); em; em &= em - 1) {
                const int from = __builtin_ctzll(em);
                const u32 rel = ((u32)__builtin_amdgcn_readlane((int)fe.x, from) & 255u) - (u32)lane * 4u;
                const float vv = as_f((u32)__builtin_amdgcn_readlane((int)fe.y, from));
                #pragma unroll
                for (int r = 0; r < 4; ++r) y[r] = rel == (u32)r ? vv : y[r];
              }
            ST128((u32x4{as_u(y[0] * scale), as_u(y[1] * scale), as_u(y[2] * scale), as_u(y[3] * scale)}), rsO, olane + ni * 64, obase + mi * 16 * NN * 4, AUX_SYS);
          }
        }
      } else {
        landed(endp);
        ctl_st(C_REL + (n % NSLOT) * 4, n + 1);
        #pragma unroll
        for (int mi = 0; mi < 4; ++mi) {
          u32x4 r4[4];
          #pragma unroll
          for (int ni = 0; ni < 4; ++ni) r4[ni] = __builtin_amdgcn_raw_buffer_load_b128(rsP, lane * 16 + ni * 1024, par + EX_OUR + uid * OU_RAW + mi * 4096, AUX_SYS);
          #pragma unroll
          for (int ni = 0; ni < 4; ++ni) {
            const f32x4 y = __builtin_bit_cast(f32x4, r4[ni]) * scale;
            ST128(__builtin_bit_cast(u32x4, y), rsO, olane + ni * 64, obase + mi * 16 * NN * 4, AUX_SYS);
          }
        }
      }
    }
    return;
  }

  constexpr int SLOTS = NGEMM / 8;
  const int xc = b & 7, slot = b >> 3, wm = wave >> 1, wn = wave & 1;
  for (int ti = slot; ti < 36; ti += SLOTS) {
    const int j = xc >> 1, ft = ti * 2 + (xc & 1);
    const int c = ti < 24 ? ft / 6 : 4 * (xc & 1) + (ti < 28 ? ti - 24 : (ti - 28) >> 1), cb = j * 8 + c;
    auto row_group = [&](int k) -> int {
      if (ti >= 28) return 8 * rank + 4 * (ti & 1) + k;
      if (ti >= 24) return 8 * ((~rank & 4) | ((j + 1 + min(k, 2)) & 3)) + 3;
      const int t = ti % 3;
      return 8 * (xc & 1 ? (~rank & 4) | ((j + 1 + t) & 3) : (rank & 4) | ((rank + 1 + t) & 3)) + 4 * (xc & 1) + k;
    };
    const int rgw = row_group(wm);
    const u32 deadw = (u32)(uintptr_t)(smem + L_CTL);
    auto ready = [&](int grp) {
      const bool isx = lane < 4;
      const int rgl = row_group(lane & 3);
      const u32 off = OFF_CNT + ((isx ? rgl : 64 + cb) * 5 + grp) * 128, want = (grp == 4 ? (isx ? 6u : 8u) : (isx ? 7u : 14u)) * call;
      u32 got;
      if (!wait_ge(p.ws, off, want, call, got)) ctl_st(deadw, 1u);
    };
    const u32 xo0 = OFF_XA + rgw * IU_RAW + wn * 4096, wo0 = OFF_WA + cb * 2 * IU_RAW + wave * 2048;
    auto dma = [&](int k, int ks, u32 st) {
      const u32 xo = xo0 + ks * (64 * IU_RAW), wo = wo0 + ks * (64 * IU_RAW);
      LDS char* dx = smem + st + wave * 4096;
      LDS char* dw = smem + st + 32 * KiB + wave * 2048;
      if (k == 0) __builtin_amdgcn_raw_ptr_buffer_load_lds(rsL, (LDS void*)dx, 16, lane * 16, xo, 0, AUX_SYS);
      if (k == 1) __builtin_amdgcn_raw_ptr_buffer_load_lds(rsL, (LDS void*)dx, 16, lane * 16, xo, 1024, AUX_SYS);
      if (k == 2) __builtin_amdgcn_raw_ptr_buffer_load_lds(rsL, (LDS void*)dx, 16, lane * 16, xo, 2048, AUX_SYS);
      if (k == 3) __builtin_amdgcn_raw_ptr_buffer_load_lds(rsL, (LDS void*)dx, 16, lane * 16, xo, 3072, AUX_SYS);
      if (k == 4) __builtin_amdgcn_raw_ptr_buffer_load_lds(rsL, (LDS void*)dw, 16, lane * 16, wo, 0, AUX_SYS);
      if (k == 5) __builtin_amdgcn_raw_ptr_buffer_load_lds(rsL, (LDS void*)dw, 16, lane * 16, wo, 1024, AUX_SYS);
    };
    f32x4 acc[4][4];
    #pragma unroll
    for (int mi = 0; mi < 4; ++mi)
      #pragma unroll
      for (int ni = 0; ni < 4; ++ni) acc[mi][ni] = f32x4{0.f, 0.f, 0.f, 0.f};
    const bool twin = ti >= 24 && ti < 28;
    const int kr = twin ? (~rank & 4) | j : rank;
    if (wave == 0) {
      if (ti == slot) ctl_st(deadw, 0u);
      for (int grp = twin ? 0 : 4; grp < 5; ++grp) ready(grp);
    }
    asm volatile("s_waitcnt vmcnt(0) lgkmcnt(0)\n\ts_barrier" ::: "memory");
    #pragma unroll
    for (int k = 0; k < 12; ++k) dma(k % 6, kr * 4 + k / 6, k / 6 * STAGE);
    u32 st = 0;
    int ksn = kr * 4 + 2, jn = -2, kbn = 0;
    i32x8 fa[8], fb[8];
    auto step = [&](i32x8 (&cur)[8], const i32x8 (&prev)[8], bool first, bool more, bool last) {
      if (more && wave == 0 && jn == 0) ready(kbn);
      if (!last) asm volatile("s_waitcnt vmcnt(6) lgkmcnt(0)\n\ts_barrier" ::: "memory");
      else asm volatile("s_waitcnt vmcnt(0) lgkmcnt(0)\n\ts_barrier" ::: "memory");
      const u32 st2 = st == 0 ? 2 * STAGE : st - STAGE;
      LDS char* ax = smem + st + wm * 8192 + lane * 16;
      LDS char* aw = smem + st + 32 * KiB + wn * 8192 + lane * 16;
      u32x4 h[16];
      auto M = [&](int k) {
        if (!first) acc[k >> 2][k & 3] = __builtin_amdgcn_mfma_scale_f32_16x16x128_f8f6f4(prev[k & 3], prev[4 + (k >> 2)], acc[k >> 2][k & 3], 1, 1, 0, 0, 0, 0);
        __builtin_amdgcn_sched_barrier(0);
      };
      auto R = [&](int k) { h[k] = *(LDS u32x4*)(((k >> 1) & 1 ? ax : aw) + (k >> 2) * 2048 + (k & 1) * 1024); __builtin_amdgcn_sched_barrier(0); };
      __builtin_amdgcn_sched_barrier(0);
      #pragma unroll
      for (int k = 0; k < 6; ++k) {
        M(2 * k); R(2 * k); R(2 * k + 1); M(2 * k + 1);
        if (more) dma(k, ksn, st2);
        __builtin_amdgcn_sched_barrier(0);
      }
      M(12); R(12); R(13); M(13); R(14); R(15); M(14); M(15);
      #pragma unroll
      for (int f = 0; f < 4; ++f) {
        cur[f] = __builtin_bit_cast(i32x8, __builtin_shufflevector(h[4 * f], h[4 * f + 1], 0, 1, 2, 3, 4, 5, 6, 7));
        cur[4 + f] = __builtin_bit_cast(i32x8, __builtin_shufflevector(h[4 * f + 2], h[4 * f + 3], 0, 1, 2, 3, 4, 5, 6, 7));
      }
      if (++jn == 7) { jn = 0; ++kbn; }
      ksn = jn < 0 ? ksn + 1 : ((kr + 1 + jn) & 7) * 4 + kbn;
      st = st == 2 * STAGE ? 0 : st + STAGE;
    };
    step(fa, fb, true, true, false);
    #pragma unroll 1
    for (int s = 1; s < 29; s += 2) { step(fb, fa, false, true, false); step(fa, fb, false, true, false); }
    step(fb, fa, false, true, false);
    step(fa, fb, false, false, false);
    step(fb, fa, false, false, true);
    #pragma unroll
    for (int k = 0; k < 16; ++k)
      acc[k >> 2][k & 3] = __builtin_amdgcn_mfma_scale_f32_16x16x128_f8f6f4(fb[k & 3], fb[4 + (k >> 2)], acc[k >> 2][k & 3], 1, 1, 0, 0, 0, 0);
    int lz = lane; asm volatile("" : "+v"(lz));
    const bool dead = ctl_ld(deadw) != 0u;
    const int owner = rgw >> 3;
    if (owner == rank) {
      const u32 obase = ((rgw & 7) * 64 * NN + group_of(rank, j) * 1024 + c * 128 + wn * 64) * 4, olane = ((lz & 15) * NN + (lz >> 4) * 4) * 4;
      const float sc = dead ? __builtin_nanf("") : scale;
      #pragma unroll
      for (int mi = 0; mi < 4; ++mi)
        #pragma unroll
        for (int ni = 0; ni < 4; ++ni)
          ST128(__builtin_bit_cast(u32x4, acc[mi][ni] * sc), rsO, olane + ni * 64, obase + mi * 16 * NN * 4, AUX_SYS);
    } else if (!dead && !(ti >= 24 && wm == 3)) {
      const int t = (cb * 2 + wn) * 5 + (rgw & 7) - ((owner >> 2) == (rank >> 2) ? 0 : 3);
      const u32 uid = owner * 320 + t;
      const u32 offE = par + EX_OUE + uid * OU_ENC;
      float sa = 0.f;
      #pragma unroll
      for (int mi = 0; mi < 4; ++mi)
        #pragma unroll
        for (int ni = 0; ni < 4; ++ni) sa += __builtin_fabsf(acc[mi][ni][(mi + ni) & 3]);
      const int g = max(0, 24 - ((int)((as_u(wave_sum_f32(sa) * (4.2f / 1024.f)) >> 23) & 255u) - 126));
      LDS char* last1k = smem + wave * 4096;
      LDS char* side = last1k + 512;
      LDS char* pend = last1k + 1536;
      *(LDS u32x2*)(side + lz * 8) = u32x2{0u, 0u};
      u32 npend = 0, mk[4] = {0u, 0u, 0u, 0u};
      auto quad = [&](int gq, u32& l0, u32& l1, u32& tp) {
        u32 qv[4], fr[4];
        #pragma unroll
        for (int r = 0; r < 4; ++r) {
          const float tt = __builtin_amdgcn_ldexpf(acc[gq >> 2][gq & 3][r], g + 7);
          qv[r] = (u32)f2i_sat(tt);
          fr[r] = as_u(__builtin_amdgcn_fractf(tt));
        }
        const u32 p01 = __builtin_amdgcn_perm(qv[1], qv[0], 0x07030400u), p23 = __builtin_amdgcn_perm(qv[3], qv[2], 0x07030400u);
        const u32 low = __builtin_amdgcn_perm(p23, p01, 0x05040100u);
        tp = __builtin_amdgcn_perm(p23, p01, 0x07060302u);
        l0 = __builtin_amdgcn_perm(qv[1], qv[0], 0x06050201u);
        l1 = __builtin_amdgcn_perm(qv[3], qv[2], 0x06050201u);
        mk[gq >> 3] = bfi(0x80808080u >> (gq & 7), (tp ^ (tp << 1)) >> (gq & 7), mk[gq >> 3]);
        mk[2 + (gq >> 3)] = bfi(0x80808080u >> (gq & 7), low >> (gq & 7), mk[2 + (gq >> 3)]);
        const u32 lowest = tp & ~((tp & 0x7F7F7F7Fu) + 0x7F7F7F7Fu) & 0x80808080u;
        const bool odd_one = (fr[0] | fr[1] | fr[2] | fr[3] | (low & 0x7F7F7F7Fu) | lowest) != 0u;
        const u64 bm = __ballot(odd_one);
        if (odd_one) {
          const u32 sl = min(63u, npend + __builtin_amdgcn_mbcnt_hi((u32)(bm >> 32), __builtin_amdgcn_mbcnt_lo((u32)bm, 0u)));
          *(LDS f32x4*)(pend + sl * 16) = acc[gq >> 2][gq & 3];
          *(LDS u32*)(pend + 1024 + sl * 4) = gq * 64 + lz;
        }
        npend += __popcll(bm);
      };
      u32 y0, y1, z0, z1, th[2];
      quad(14, y0, y1, th[0]);
      quad(15, z0, z1, th[1]);
      *(LDS u32*)(last1k + lz * 4) = z0;
      *(LDS u32*)(last1k + 256 + lz * 4) = z1;
      #pragma unroll
      for (int mi = 0; mi < 4; ++mi) {
        u32 pk[12];
        #pragma unroll
        for (int ni = 0; ni < 4; ++ni) {
          const int gq = mi * 4 + ni, be = gq % 7;
          if (gq == 14) { pk[6] = y0; pk[7] = y1; break; }
          quad(gq, pk[ni * 3], pk[ni * 3 + 1], pk[ni * 3 + 2]);
          pk[ni * 3 + 2] = bfi(0x40404040u, be < 6 ? th[gq / 7] << (6 - be) : th[gq / 7] >> 1, pk[ni * 3 + 2]);
        }
        #pragma unroll
        for (int j = 0; j < (mi < 3 ? 3 : 2); ++j)
          ST128((u32x4{pk[j * 4], pk[j * 4 + 1], pk[j * 4 + 2], pk[j * 4 + 3]}), rsL, lz * 16 + j * 1024, offE + mi * 3072, AUX_SYS);
      }
      const u32 mine = (__popc(mk[0]) + __popc(mk[1])) | ((__popc(mk[2]) + __popc(mk[3])) << 16);
      const u32 inc = wave_scan_u32(mine), tot = (u32)__builtin_amdgcn_readlane((int)inc, 63);
      const u32 n0 = tot & 0xFFFFu, n1 = tot >> 16;
      Side sm = side_map(n0, n1, 0u, 0u);
      bool raw = npend > 64u || sm.end > 512u;
      u32 ns = 0, nf = 0;
      if (!raw) {
        wave_sync();
        u32 kk[2] = {(inc - mine) & 0xFFFFu, (inc - mine) >> 16};
        #pragma unroll
        for (int w = 0; w < 4; ++w) {
          const u32 hb = sm.h[w >> 1] * 8 + lz * 4 + (w & 1) * 2, lb = sm.l[w >> 1] * 8;
          for (u32 m = mk[w]; m; m &= m - 1) {
            const u32 bp = (u32)__builtin_ctz(m), k = kk[w >> 1]++, ph = hb + (bp >> 4) + k, pl = lb + k * 4;
            __hip_atomic_fetch_or((LDS u32*)(side + ((ph >> 5) << 2)), 1u << (ph & 31u), __ATOMIC_RELAXED, __HIP_MEMORY_SCOPE_WORKGROUP);
            __hip_atomic_fetch_or((LDS u32*)(side + ((pl >> 5) << 2)), (bp & 15u) << (pl & 31u), __ATOMIC_RELAXED, __HIP_MEMORY_SCOPE_WORKGROUP);
          }
        }
        LDS char* fl = last1k + 1024;
        for (u32 p0 = 0; p0 < npend; p0 += 16) {
          const u32 pi = p0 + (lz >> 2), r = lz & 3;
          const bool on = pi < npend;
          const float v = *(LDS float*)(pend + p0 * 16 + lz * 4);
          const u32 pp = *(LDS u32*)(pend + 1024 + pi * 4);
          const float tt = __builtin_amdgcn_ldexpf(v, g + 7);
          const u32 iv = (u32)f2i_sat(tt);
          const bool full = on && (as_u((float)(int)iv) != as_u(tt) || (iv & 31u)), brief = on && !full && (iv & 96u);
          const u64 bs = __ballot(brief), bf = __ballot(full);
          const u32 ss = ns + __builtin_amdgcn_mbcnt_hi((u32)(bs >> 32), __builtin_amdgcn_mbcnt_lo((u32)bs, 0u));
          const u32 sf = nf + __builtin_amdgcn_mbcnt_hi((u32)(bf >> 32), __builtin_amdgcn_mbcnt_lo((u32)bf, 0u));
          if (brief && sm.s + ss * 2 < 512u) *(LDS unsigned short*)(side + sm.s + ss * 2) = (unsigned short)((pp << 5) | (r << 3) | ((iv >> 5) & 3u));
          if (full && sf < OF_CAP) *(LDS u32x2*)(fl + sf * 8) = u32x2{pp * 4 + r, as_u(v)};
          ns += __popcll(bs); nf += __popcll(bf);
        }
        sm = side_map(n0, n1, ns, nf);
        raw = ns > OS_CAP || nf > OF_CAP || sm.end > 512u;
        if (nf && !raw) {
          wave_sync();
          const u32x2 e = *(LDS u32x2*)(fl + (lz & 31) * 8);
          if ((u32)lz < nf) {
            LDS unsigned short* at = (LDS unsigned short*)(side + sm.f + lz * 6);
            at[0] = (unsigned short)e.x; at[1] = (unsigned short)e.y; at[2] = (unsigned short)(e.y >> 16);
          }
        }
      }
      wave_sync();
      if (lz == 0) {
        *(LDS u32*)side = raw ? 0x80000000u : (n0 | (n1 << 10) | ((u32)g << 20));
        *(LDS unsigned short*)(side + 4) = (unsigned short)(ns | (nf << 8));
      }
      const u32 lines = raw ? 1u : (sm.end + 127) / 128;
      wave_sync();
      if ((u32)lz < 32 + lines * 8) ST128(*(LDS u32x4*)(last1k + lz * 16), rsL, lz * 16, offE + 11264, AUX_SYS);
      if (raw) {
        const u32 offR = par + EX_OUR + uid * OU_RAW;
        #pragma unroll
        for (int mi = 0; mi < 4; ++mi)
          #pragma unroll
          for (int ni = 0; ni < 4; ++ni)
            ST128(__builtin_bit_cast(u32x4, acc[mi][ni]), rsL, lz * 16 + ni * 1024, offR + mi * 4096, AUX_SYS);
      }
      vm_drain();
      if (lz == owner)
        __hip_atomic_store((GLB u32*)(peer_ws + OFF_FOUT + (rank * 320 + t) * 128), call << 4 | lines, __ATOMIC_RELAXED, __HIP_MEMORY_SCOPE_SYSTEM);
    }
  }
}

}

extern "C" void kernel_launch(int rank, int nranks, ncclComm_t comm, hipStream_t stream,
                              void* const* d_in, const int* in_sizes, int n_in,
                              void* d_out, int out_size,
                              const int* in_dtypes, int out_dtype,
                              void* const* d_out_all,
                              void* const* d_ws_all, size_t ws_bytes) {
  (void)comm; (void)in_sizes; (void)n_in; (void)out_size; (void)in_dtypes; (void)out_dtype; (void)d_out_all;
  if (nranks != NR || (unsigned)rank >= (unsigned)NR || ws_bytes < WS_NEED) { fprintf(stderr, "unexpected launch shape\n"); return; }
  static unsigned attr_set = 0;
  if (!(attr_set >> rank & 1u)) {
    HIP_CHECK(hipFuncSetAttribute((const void*)gemm_rs_kernel, hipFuncAttributeMaxDynamicSharedMemorySize, (int)LDS_BYTES));
    attr_set |= 1u << rank;
  }
  Params p;
  p.x = (const uint8_t*)d_in[0]; p.w = (const uint8_t*)d_in[1]; p.sx = (const float*)d_in[2]; p.sw = (const float*)d_in[3];
  p.out = (float*)d_out; p.ws_all = (char* const*)d_ws_all; p.ws = (char*)d_ws_all[rank]; p.rank = rank; p.ws_bytes = WS_NEED;
  hipLaunchKernelGGL(gemm_rs_kernel, dim3(NWG), dim3(NTHR), LDS_BYTES, stream, p);
  HIP_CHECK(hipGetLastError());
}
```

```cpp
#include <hip/hip_runtime.h>
#include <rccl/rccl.h>
#include <cstdio>
#include <cstdint>

#define HIP_CHECK(expr) do { hipError_t _e = (expr); if (_e != hipSuccess) { \
    fprintf(stderr, "HIP error %s at %s:%d\n", hipGetErrorString(_e), __FILE__, __LINE__); \
    return; } } while (0)

namespace {

typedef unsigned u32;
typedef unsigned long long u64;
typedef unsigned u32x4 __attribute__((ext_vector_type(4)));
typedef unsigned u32x2 __attribute__((ext_vector_type(2)));
typedef int i32x2 __attribute__((ext_vector_type(2)));
typedef int i32x8 __attribute__((ext_vector_type(8)));
typedef float f32x4 __attribute__((ext_vector_type(4)));
typedef __amdgpu_buffer_rsrc_t rsrc_t;
#define LDS __attribute__((address_space(3)))
#define GLB __attribute__((address_space(1)))

constexpr int NR = 8, MM = 4096, NN = 16384, KS = 512;
constexpr int NWG = 256, NTHR = 512;
constexpr int PULL_WGS = 8;
constexpr int WIN = 8, WIN_OUT = 7;
constexpr u32 LOOK = 5, LOOK_OUT = 3, LOOK_END = 2;
constexpr int NRAW = 8;
constexpr int X2FIRST = 8;
constexpr int NPULL = 7 * PULL_WGS, NGEMM = NWG - NPULL;
constexpr int AUX_SYS = 17;
constexpr u64 DEADLINE_TICKS = 40000000ull;

constexpr u32 KiB = 1024, MiB = 1024 * 1024;
constexpr u32 OFF_CALL = 0;
constexpr u32 OFF_TMO = 32 * KiB;
constexpr u32 OFF_TIC = 33 * KiB;
constexpr u32 OFF_CNT = 40 * KiB;
constexpr u32 OFF_FIN = 128 * KiB;
constexpr u32 OFF_FOUT = OFF_FIN + 8 * 576 * 128;
constexpr u32 OFF_XA = 2 * MiB;
constexpr u32 OFF_WA = OFF_XA + 16 * MiB;
constexpr u32 OFF_PAR = OFF_WA + 16 * MiB;
constexpr u32 IU_ENC = 8 * KiB, IU_RAW = 8 * KiB;
constexpr u32 OU_ENC = 12 * KiB, OU_RAW = 16 * KiB;
constexpr u32 IU_SIDE = 6144, OU_SIDE = 11776;
constexpr u32 EX_INE = 0;
constexpr u32 EX_INR = EX_INE + 1280 * IU_ENC;
constexpr u32 EX_OUE = EX_INR + 1280 * IU_RAW;
constexpr u32 EX_OUR = EX_OUE + 2560 * OU_ENC;
constexpr u32 PAR_BYTES = EX_OUR + 2560 * OU_RAW;
constexpr u32 WS_NEED = OFF_PAR + 2 * PAR_BYTES;
static_assert(OFF_CNT + 128 * 128 <= OFF_FIN && OFF_FOUT + 8 * 320 * 128 <= OFF_XA, "flags");
static_assert(WS_NEED <= 257u * MiB, "workspace");
constexpr u32 QV = 5120, QV0 = 1024, QV_SEC = 24, QV_END = 1792, QV_ALL = 2304;
constexpr u32 Q_ESC = 6144, Q_TER = 6400, Q_TMAX = 1760, Q_PAT0 = 8, Q_PAT = 264;
constexpr u32 IN_CAP0 = 62, IN_CAP = 254;
constexpr u32 OS_CAP = 64, OF_CAP = 32;

constexpr u32 STAGE = 48 * KiB;
constexpr u32 L_SCR = 64 * KiB, SCR_W = 10752;
constexpr u32 NSLOT = 8, SLOT = 13 * KiB, NCONS = 6;
constexpr u32 L_PSCR = NSLOT * SLOT;
constexpr u32 L_CTL = L_PSCR + NCONS * 8 * KiB;
constexpr u32 LDS_BYTES = L_CTL + 256;
static_assert(3 * STAGE <= L_CTL && L_SCR + 8 * SCR_W <= LDS_BYTES && LDS_BYTES <= 160 * KiB, "lds");

__device__ __forceinline__ bool quick_form(int i) { return i < 256 && ((i & 3) == 0 || (i & 11) == 1); }
__device__ __forceinline__ int group_of(int rank, int j) { return rank < 4 ? rank * 4 + j : j * 4 + (rank & 3); }

struct Params {
  const uint8_t* x; const uint8_t* w; const float* sx; const float* sw;
  float* out; char* const* ws_all; char* ws; int rank; u32 ws_bytes;
};

__device__ __forceinline__ rsrc_t mk_rsrc(const void* p, u32 bytes) {
  u64 a = (u64)p;
  u32 lo = __builtin_amdgcn_readfirstlane((u32)a), hi = __builtin_amdgcn_readfirstlane((u32)(a >> 32));
  return __builtin_amdgcn_make_buffer_rsrc((void*)(((u64)hi << 32) | lo), 0, (int)bytes, 0x00020000);
}
#define ST128(v, rs, voff, soff, aux) __builtin_amdgcn_raw_buffer_store_b128((v), (rs), (int)((u32)(voff) + (u32)(soff)), 0, (aux))
__device__ __forceinline__ float as_f(u32 v) { return __builtin_bit_cast(float, v); }
__device__ __forceinline__ u32 as_u(float v) { return __builtin_bit_cast(u32, v); }
__device__ __forceinline__ int f2i_sat(float v) {
  int r; asm("v_cvt_i32_f32 %0, %1" : "=v"(r) : "v"(v)); return r;
}
__device__ __forceinline__ void vm_drain() { asm volatile("s_waitcnt vmcnt(0)" ::: "memory"); }
__device__ __forceinline__ void wave_sync() {
  asm volatile("s_waitcnt lgkmcnt(0)" ::: "memory");
}
__device__ __forceinline__ u32 bfi(u32 m, u32 a, u32 b) { return __builtin_amdgcn_bitop3_b32(m, a, b, 0xCA); }
template <int CTRL, int ROWS> __device__ __forceinline__ u32 dpp0(u32 v) { return (u32)__builtin_amdgcn_update_dpp(0, (int)v, CTRL, ROWS, 0xf, false); }
__device__ __forceinline__ u32 wave_scan_u32(u32 v) {
  v += dpp0<0x111, 0xf>(v); v += dpp0<0x112, 0xf>(v); v += dpp0<0x114, 0xf>(v); v += dpp0<0x118, 0xf>(v);
  v += dpp0<0x142, 0xa>(v); v += dpp0<0x143, 0xc>(v);
  return v;
}
__device__ __forceinline__ float wave_sum_f32(float v) {
  v += as_f(dpp0<0x111, 0xf>(as_u(v))); v += as_f(dpp0<0x112, 0xf>(as_u(v))); v += as_f(dpp0<0x114, 0xf>(as_u(v)));
  v += as_f(dpp0<0x118, 0xf>(as_u(v))); v += as_f(dpp0<0x142, 0xa>(as_u(v))); v += as_f(dpp0<0x143, 0xc>(as_u(v)));
  return as_f((u32)__builtin_amdgcn_readlane((int)as_u(v), 63));
}
__device__ __forceinline__ u32 ctl_ld(u32 addr) {
  u32 v; asm volatile("ds_read_b32 %0, %1\n\ts_waitcnt lgkmcnt(0)" : "=&v"(v) : "v"(addr) : "memory");
  return (u32)__builtin_amdgcn_readfirstlane((int)v);
}
__device__ __forceinline__ void ctl_st(u32 addr, u32 v) { asm volatile("ds_write_b32 %0, %1" :: "v"(addr), "v"(v) : "memory"); }
__device__ __forceinline__ u32 ld_flag(const char* ws, u32 off) {
  return __hip_atomic_load((const GLB u32*)(ws + off), __ATOMIC_RELAXED, __HIP_MEMORY_SCOPE_SYSTEM);
}
__device__ __forceinline__ bool wait_ge(char* ws, u32 off, u32 target, u32 call, u32& got) {
  if (__all((got = ld_flag(ws, off)) >= target)) return true;
  const u64 t0 = __builtin_amdgcn_s_memrealtime();
  GLB u32* tmo = (GLB u32*)(ws + OFF_TMO);
  for (u32 it = 1;; ++it) {
    __builtin_amdgcn_s_sleep(2);
    if (__all((got = ld_flag(ws, off)) >= target)) return true;
    if ((it & 15u) == 0u &&
        (__hip_atomic_load(tmo, __ATOMIC_RELAXED, __HIP_MEMORY_SCOPE_AGENT) == call ||
         __builtin_amdgcn_s_memrealtime() - t0 > DEADLINE_TICKS)) {
      __hip_atomic_store(tmo, call, __ATOMIC_RELAXED, __HIP_MEMORY_SCOPE_AGENT);
      return false;
    }
  }
}
__device__ __forceinline__ void in_place(char* ws, u32 line, u32 ks, u32 call, int lane) {
  if (lane == 0) __hip_atomic_store((GLB u32*)(ws + OFF_CNT + line * 128 + ks * 4), call, __ATOMIC_RELAXED, __HIP_MEMORY_SCOPE_SYSTEM);
}

__device__ __forceinline__ void store_raw_unit(const u32x4 (&x)[8], rsrc_t rs, u32 off, int lane) {
  #pragma unroll
  for (int j = 0; j < 8; ++j) ST128(x[j], rs, lane * 16 + j * 1024, off, AUX_SYS);
}

__device__ __forceinline__ void export_half(const u32x4 (&x)[4], rsrc_t rs, u32 par, u32 uid, int hf, LDS char* scr, int lane) {
  const u32 offE = par + EX_INE + uid * IU_ENC;
  u32 s = 0;
  #pragma unroll
  for (int j = 0; j < 4; ++j) {
    *(LDS u32x4*)(scr + j * 1024 + lane * 16) = x[j];
    s = __builtin_amdgcn_sad_u8(x[j][j] & 0x7C7C7C7Cu, 0u, s);
  }
  s = (u32)__builtin_amdgcn_readlane((int)wave_scan_u32(s), 63);
  int e0 = ((int)s - 17121) >> 12;
  e0 = e0 < 0 ? 0 : (e0 > 24 ? 24 : e0);
  const u32 e4 = (u32)e0 * 0x04040404u;
  u32 mk[2] = {0, 0};
  u32 pk[12];
  #pragma unroll
  for (int j = 0; j < 4; ++j) {
    u32 o[4];
    #pragma unroll
    for (int c = 0; c < 4; ++c) {
      const u32 u = (x[j][c] | 0x80808080u) - e4;
      o[c] = bfi(0x1F1F1F1Fu, u, x[j][c]);
      const u32 t = (((u >> 1) ^ 0x40404040u) & 0x70707070u) + 0x70707070u;
      const int d = j * 4 + c;
      mk[d >> 3] = bfi(0x80808080u >> (d & 7), t >> (d & 7), mk[d >> 3]);
    }
    pk[j * 3 + 0] = bfi(0x60606060u, o[3] << 5, o[0]);
    pk[j * 3 + 1] = bfi(0x60606060u, o[3] << 3, o[1]);
    pk[j * 3 + 2] = bfi(0x60606060u, bfi(0x40404040u, o[3] >> 1, o[3] << 1), o[2]);
  }
  #pragma unroll
  for (int j = 0; j < 3; ++j)
    ST128((u32x4{pk[j * 4], pk[j * 4 + 1], pk[j * 4 + 2], pk[j * 4 + 3]}), rs, lane * 16 + j * 1024, offE + hf * 3072, AUX_SYS);
  const u32 mine = __popc(mk[0]) + __popc(mk[1]);
  const u32 inc = wave_scan_u32(mine);
  const u32 total = (u32)__builtin_amdgcn_readlane((int)inc, 63);
  if (total <= IN_CAP) {
    LDS unsigned short* list = (LDS unsigned short*)(scr + 8192);
    u32 slot = 2 + inc - mine;
    #pragma unroll
    for (int q = 0; q < 2; ++q) {
      u32 m = mk[q];
      while (m) {
        const int bp = __builtin_ctz(m); m &= m - 1;
        const int d = q * 8 + 7 - (bp & 7);
        list[slot++] = (unsigned short)((d >> 2) * 1024 + lane * 16 + (d & 3) * 4 + (bp >> 3));
      }
    }
    wave_sync();
    for (u32 j = 0; j < 2; ++j) {
      const u32 pp = *(LDS u32*)(scr + 8192 + j * 256 + lane * 4);
      const u32 p0 = pp & 4095u, p1 = (pp >> 16) & 4095u;
      const u32 b0 = *(LDS unsigned char*)(scr + p0), b1 = *(LDS unsigned char*)(scr + p1);
      u32 sd = (((p0 << 3) | ((128u + (b0 & 127u) - 4u * (u32)e0) >> 5) | (p1 << 19) | (((128u + (b1 & 127u) - 4u * (u32)e0) >> 5) << 16))) + hf * 0x80008000u;
      if (j == 0 && lane == 0) sd = total | ((u32)e0 << 16);
      const u32 at = j * 64 + lane;
      if (at < 32 || total > IN_CAP0)
        __builtin_amdgcn_raw_buffer_store_b32(sd, rs, at < 32 ? hf * 128 + at * 4 : 128 + hf * 384 + at * 4, offE + IU_SIDE, AUX_SYS);
      if (total <= 126) break;
    }
  } else {
    #pragma unroll
    for (int j = 0; j < 4; ++j) ST128(x[j], rs, lane * 16 + j * 1024, par + EX_INR + uid * IU_RAW + hf * 4096, AUX_SYS);
    if (lane == 0) __builtin_amdgcn_raw_buffer_store_b32(0x80000000u, rs, hf * 128, offE + IU_SIDE, AUX_SYS);
  }
}

__device__ __forceinline__ u32 export_short(const u32x4 (&x)[8], rsrc_t rs, u32 par, u32 uid, LDS char* scr, int lane) {
  typedef unsigned short u16x2 __attribute__((ext_vector_type(2)));
  const u32 offE = par + EX_INE + uid * IU_ENC, offR = par + EX_INR + uid * IU_RAW;
  LDS char* var = scr + 8192;
  LDS char* tab = var + QV_ALL;
  u32 s = 0;
  #pragma unroll
  for (int j = 0; j < 8; ++j) s = __builtin_amdgcn_sad_u8(x[j][j & 3] & 0x7C7C7C7Cu, 0u, s);
  #pragma unroll
  for (int j = 0; j < 7; ++j) *(LDS u32x4*)(scr + j * 1024 + lane * 16) = u32x4{0u, 0u, 0u, 0u};
  *(LDS u32x4*)(var + lane * 16) = u32x4{0u, 0u, 0u, 0u};
  *(LDS u32x4*)(var + 1024 + lane * 16) = u32x4{0u, 0u, 0u, 0u};
  {
    u32 sel = 0x0C0C0C0Cu, at = 0;
    #pragma unroll
    for (int y = 0; y < 4; ++y) if ((lane >> y) & 1) { sel = (sel & ~(0xFFu << at)) | ((u32)y << at); at += 8; }
    if (lane < 16) *(LDS u32*)(tab + lane * 4) = sel;
  }
  s = (u32)__builtin_amdgcn_readlane((int)wave_scan_u32(s), 63);
  const int K = max(5, min(31, ((int)s >> 13) + 3));
  const u32 kw = (128u + 4u * (u32)K) * 0x01010101u, tabo = (u32)(uintptr_t)tab;
  auto front = [&](u32 F) { return *(LDS u32*)(uintptr_t)__builtin_amdgcn_udot4(F, 0x20100804u, tabo, false); };
  u32 cs[32], cn[32], car[20], mine = 0;
  wave_sync();
  #pragma unroll
  for (int g = 0; g < 4; ++g) {
    u32 a[8], uu[8];
    #pragma unroll
    for (int k = 0; k < 8; ++k) {
      const int d = g * 8 + k;
      const u32 xv = x[d >> 2][d & 3];
      const u32 u = uu[k] = (kw - (xv & 0x7C7C7C7Cu)) ^ 0x80808080u;
      const u32 esc = (((u >> 1) & 0x78787878u) + 0x78787878u) | ~((u | (u << 1)) << 4);
      const u32 F = (esc >> 7) & 0x01010101u;
      a[k] = bfi(0x0C0C0C0Cu, ~u | __builtin_bit_cast(u32, __builtin_bit_cast(u16x2, F) * (u16x2){12, 12}), xv);
      cs[d] = front(F);
      cn[d] = mine;
      mine = __builtin_amdgcn_udot4(F, 0x01010101u, mine, false);
    }
    car[g * 5 + 0] = bfi(0x70707070u, a[5] << 4, a[0]);
    car[g * 5 + 1] = bfi(0x70707070u, a[6] << 4, a[1]);
    car[g * 5 + 2] = bfi(0x70707070u, a[7] << 4, a[2]);
    car[g * 5 + 3] = bfi(0x40404040u, a[7] << 3, bfi(0x20202020u, a[6] << 2, bfi(0x10101010u, a[5] << 1, a[3])));
    car[g * 5 + 4] = bfi(0x40404040u, a[7] >> 1, bfi(0x20202020u, a[6] >> 2, bfi(0x10101010u, a[5] >> 3, a[4])));
    __builtin_amdgcn_sched_barrier(0);
    #pragma unroll
    for (int k = 0; k < 8; ++k) cs[g * 8 + k] = __builtin_amdgcn_perm(0u, uu[k], cs[g * 8 + k]);
  }
  #pragma unroll
  for (int j = 0; j < 5; ++j) ST128((u32x4{car[j * 4], car[j * 4 + 1], car[j * 4 + 2], car[j * 4 + 3]}), rs, lane * 16 + j * 1024, offE, AUX_SYS);
  const u32 inc = wave_scan_u32(mine);
  const u32 nesc = (u32)__builtin_amdgcn_readlane((int)inc, 63);
  bool fits = nesc <= Q_ESC;
  u32 lines = 1;
  if (fits) {
    const u32 base = inc - mine;
    #pragma unroll
    for (int d = 0; d < 32; ++d) {
      const u32 q = base + cn[d];
      const u64 v = (u64)cs[d] << ((q & 3u) * 8u);
      LDS u32* at = (LDS u32*)(scr + (q & ~3u));
      __hip_atomic_fetch_or(at, (u32)v, __ATOMIC_RELAXED, __HIP_MEMORY_SCOPE_WORKGROUP);
      __hip_atomic_fetch_or(at + 1, (u32)(v >> 32), __ATOMIC_RELAXED, __HIP_MEMORY_SCOPE_WORKGROUP);
    }
    wave_sync();
    const u32 t0 = (QV_SEC + (nesc + 31) / 32 * 8) * 8;
    u32 tbit = t0, npat = 0, nter = 0;
    for (u32 r0 = 0; r0 < nesc && fits; r0 += 2048) {
      const u32x4 e0 = *(LDS u32x4*)(scr + r0 + lane * 32), e1 = *(LDS u32x4*)(scr + r0 + lane * 32 + 16);
      u32 two[2] = {0u, 0u}, zq[8], zl[8], fs[8], t3[8], mk = 0, bits = 0, nt = 0;
      #pragma unroll
      for (int k = 0; k < 8; ++k) {
        const u32 u = k < 4 ? e0[k & 3] : e1[k & 3], sel = (u >> 2) & 0x07070707u;
        const u32 far = ((((u >> 1) & 0x70707070u) + 0x70707070u) >> 7) & 0x01010101u;
        const u32 c2 = __builtin_amdgcn_perm(0x03030201u, 0x03030300u, sel) | far | (far << 1);
        const u32 F = c2 & (c2 >> 1) & 0x01010101u;
        two[k >> 2] |= __builtin_amdgcn_udot4(c2, 0x40100401u, 0u, false) << (8 * (k & 3));
        fs[k] = front(F); t3[k] = ((sel + 0x02020202u) & 0x07070707u) + 0x01010101u;
        nt = __builtin_amdgcn_udot4(F, 0x01010101u, nt, false);
        const u32 lo7 = u & 0x7F7F7F7Fu;
        const u32 out = (F << 7) & (~(lo7 + 0x68686868u) | (lo7 + 0x48484848u) | u);
        mk |= out >> k;
      }
      __builtin_amdgcn_sched_barrier(0);
      #pragma unroll
      for (int k = 0; k < 8; ++k) {
        const u32 cum = __builtin_amdgcn_perm(0u, t3[k], fs[k]) * 0x01010101u;
        zl[k] = (1u << (cum & 31u)) | (1u << ((cum >> 8) & 31u)) | (1u << ((cum >> 16) & 31u)) | (1u << ((cum >> 24) & 31u));
        zq[k] = bits | ((cum >> 29) << 16);
        bits += cum >> 24;
      }
      if (r0 + lane * 32 < nesc) *(LDS u32x2*)(var + QV_SEC + r0 / 4 + lane * 8) = u32x2{two[0], two[1]};
      const u32 np = __popc(mk), inc2 = wave_scan_u32(bits | (np << 16)), tot = (u32)__builtin_amdgcn_readlane((int)inc2, 63);
      nter += (u32)__builtin_amdgcn_readlane((int)wave_scan_u32(nt), 63);
      fits = tbit + (tot & 0xFFFFu) + 96 <= QV_END * 8 && npat + (tot >> 16) <= Q_PAT && nter <= Q_TMAX;
      if (!fits) break;
      const u32 b0 = tbit + (inc2 & 0xFFFFu) - bits;
      if (bits) {
        #pragma unroll
        for (int k = 0; k < 8; ++k) {
          const u32 q = b0 + (zq[k] & 0xFFFFu);
          const u64 v = (((u64)(zq[k] >> 16) << 32) | zl[k]) << (q & 31u);
          LDS u32* at = (LDS u32*)(var + ((q >> 5) << 2));
          __hip_atomic_fetch_or(at, (u32)v, __ATOMIC_RELAXED, __HIP_MEMORY_SCOPE_WORKGROUP);
          __hip_atomic_fetch_or(at + 1, (u32)(v >> 32), __ATOMIC_RELAXED, __HIP_MEMORY_SCOPE_WORKGROUP);
        }
      }
      u32 slot = npat + (inc2 >> 16) - np;
      for (u32 m = mk; m; m &= m - 1) {
        const int bp = __builtin_ctz(m);
        const u32 sn = r0 + lane * 32 + (7 - (bp & 7)) * 4 + (bp >> 3);
        const int v = (int)*(LDS signed char*)(scr + sn) >> 2;
        *(LDS unsigned short*)(var + (slot < Q_PAT0 ? 8 + slot * 2 : QV_END - Q_PAT0 * 2 + slot * 2)) = (unsigned short)((sn << 3) | (u32)(((v - 6) >> 3) + 4));
        ++slot;
      }
      tbit += tot & 0xFFFFu; npat += tot >> 16;
    }
    if (fits) {
      if (lane == 0) *(LDS u32x2*)var = u32x2{nesc | (npat << 13) | ((u32)K << 24), tbit - t0 + 1};
      lines = tbit / 1024 + 1;
      wave_sync();
      if ((u32)lane < lines * 8) ST128(*(LDS u32x4*)(var + lane * 16), rs, lane * 16, offE + QV, AUX_SYS);
      if (lines > 8 && lane < (int)(QV_END - QV0) / 16) ST128(*(LDS u32x4*)(var + QV0 + lane * 16), rs, lane * 16, offE + QV + QV0, AUX_SYS);
      if (npat > Q_PAT0 && lane < (int)(QV_ALL - QV_END) / 16) ST128(*(LDS u32x4*)(var + QV_END + lane * 16), rs, lane * 16, offE + QV + QV_END, AUX_SYS);
    }
  }
  if (!fits) {
    store_raw_unit(x, rs, offR, lane);
    if (lane == 0) __builtin_amdgcn_raw_buffer_store_b32(0x80000000u, rs, 0, offE + QV, AUX_SYS);
  }
  return min(lines, 8u);
}

__device__ __forceinline__ void patch_short(LDS char* scr, u32 ent) {
  const u32 pos = ent >> 3;
  __hip_atomic_fetch_add((LDS u32*)(scr + (pos & ~3u)), ((4u - (ent & 7u)) << 5) << ((pos & 3u) * 8u), __ATOMIC_RELAXED, __HIP_MEMORY_SCOPE_WORKGROUP);
}
__device__ __forceinline__ void patch_in(LDS char* scr, u32 ent) {
  const u32 pos = ent >> 3;
  __hip_atomic_fetch_add((LDS u32*)(scr + (pos & ~3u)), (((ent & 7u) - 4u) << 5) << ((pos & 3u) * 8u), __ATOMIC_RELAXED, __HIP_MEMORY_SCOPE_WORKGROUP);
}

__device__ __forceinline__ float unfix(int i, int g) { return __builtin_amdgcn_ldexpf((float)i, -g); }
struct Side { u32 h[2], l[2], s, f, end; };
__device__ __forceinline__ Side side_map(u32 n0, u32 n1, u32 ns, u32 nf) {
  Side m;
  m.h[0] = 6; m.l[0] = m.h[0] + (256 + n0 + 7) / 8; m.h[1] = m.l[0] + (n0 + 1) / 2; m.l[1] = m.h[1] + (256 + n1 + 7) / 8;
  m.s = (m.l[1] + (n1 + 1) / 2 + 1) & ~1u; m.f = m.s + ns * 2; m.end = m.f + nf * 6;
  return m;
}

__global__ void __launch_bounds__(NTHR, 2) gemm_rs_kernel(Params p) {
  extern __shared__ __attribute__((aligned(16))) char smem_[];
  LDS char* smem = (LDS char*)smem_;
  const int tid = threadIdx.x, lane = tid & 63, wave = __builtin_amdgcn_readfirstlane(tid >> 6);
  const int b = blockIdx.x, rank = p.rank;
  const int lg = lane >> 4, li = lane & 15;
  const rsrc_t rsL = mk_rsrc(p.ws, p.ws_bytes);
  const u32 tick = (u32)__builtin_amdgcn_raw_ptr_buffer_atomic_add_i32(1, rsL, lane == 0 ? 0 : (int)0xF0000000u, OFF_CALL + b * 128 + wave * 16, 0);
  char* const peer_ws = p.ws_all[lane & 7];
  u32 call;

  if (b < 128) {
    const rsrc_t rsW = mk_rsrc(p.w, KS * NN);
    __builtin_amdgcn_s_sleep(12);
    #pragma unroll
    for (int i = 0; i < 8; ++i) {
      const int row = (wave * 8 + i) * 8 + (lane >> 3);
      const int sw = ((row >> 1) & 3) | (((row >> 4) & 1) << 2);
      __builtin_amdgcn_raw_ptr_buffer_load_lds(rsW, (LDS void*)(smem + (wave * 8 + i) * 1024), 16, row * NN + (((lane & 7) ^ sw) << 4), b * 128, 0, 0);
    }
    asm volatile("s_waitcnt vmcnt(0)\n\ts_barrier" ::: "memory");
    call = __builtin_amdgcn_readfirstlane(tick) + 1u;
    const int cgl = wave & 1, kb = wave >> 1;
    u32x4 x[8];
    #pragma unroll
    for (int f = 0; f < 4; ++f)
      #pragma unroll
      for (int h = 0; h < 2; ++h)
        #pragma unroll
        for (int t = 0; t < 2; ++t) {
          const int row = kb * 128 + 64 * h + 16 * lg + 8 * t + (li >> 1);
          const int sw = ((row >> 1) & 3) | (((row >> 4) & 1) << 2);
          const i32x2 v = __builtin_amdgcn_ds_read_tr8_b64_v2i32((LDS i32x2*)(smem + row * 128 + (((cgl * 4 + f) ^ sw) << 4) + (li & 1) * 8));
          x[f * 2 + h][t * 2] = v.x; x[f * 2 + h][t * 2 + 1] = v.y;
        }
    const int gl = b >> 5, gr = 4 + ((b >> 3) & 3), c = b & 7;
    const u32 lines = export_short(x, rsL, OFF_PAR + (call & 1) * PAR_BYTES, 256 + (b * 2 + cgl) * 4 + kb, smem + L_SCR + wave * SCR_W, lane);
    vm_drain();
    if ((lane == gl || lane == gr) && lane != rank) {
      const int cb = (lane == gl ? gr & 3 : gl) * 8 + c;
      __hip_atomic_store((GLB u32*)(peer_ws + OFF_FIN + (rank * 576 + 256 + (cb * 2 + cgl) * 4 + kb) * 128), call << 4 | lines, __ATOMIC_RELAXED, __HIP_MEMORY_SCOPE_SYSTEM);
    }
    if (rank == gl || rank == gr) {
      const int cb = (rank == gl ? gr & 3 : gl) * 8 + c;
      store_raw_unit(x, rsL, OFF_WA + ((rank * 4 + kb) * 64 + cb * 2 + cgl) * IU_RAW, lane);
      vm_drain();
      in_place(p.ws, 64 + cb * 2 + cgl, rank * 4 + kb, call, lane);
    }
  } else if (b < 192) {
    const int rg = b - 128, kb = wave & 3, hf = wave >> 2, u = rg * 4 + kb;
    const rsrc_t rsX = mk_rsrc(p.x, MM * KS);
    const u32 both = (u32)(uintptr_t)(smem + L_CTL) + 32;
    const u32 xoff = OFF_XA + ((rank * 4 + kb) * 64 + rg) * IU_RAW;
    const int ow = rg >> 3, r8 = rg & 7;
    const u32 users = ((r8 <= 3 ? 0xFu << (ow & 4) : 0u) | (r8 >= 3 ? 0xFu << (~ow & 4) : 0u) | (1u << ow)) & ~(1u << rank);
    const bool tell = lane < NR && ((users >> lane) & 1);
    if (quick_form(u)) {
      u32x4 x[4];
      #pragma unroll
      for (int j = 0; j < 4; ++j)
        x[j] = __builtin_amdgcn_raw_buffer_load_b128(rsX, ((j >> 1) * 16 + li) * KS + (j & 1) * 64 + lg * 16, (rg * 64 + hf * 32) * KS + kb * 128, 0);
      if (wave == 0) ctl_st(both + (lane & 7) * 4, 0u);
      asm volatile("s_waitcnt lgkmcnt(0)\n\ts_barrier" ::: "memory");
      call = __builtin_amdgcn_readfirstlane(tick) + 1u;
      const u32 parw = OFF_PAR + (call & 1) * PAR_BYTES;
      auto second = [&](int k) -> bool {
        u32 old = 0;
        if (lane == 0) old = __hip_atomic_fetch_add((LDS u32*)(uintptr_t)(both + k * 4), 1u, __ATOMIC_RELAXED, __HIP_MEMORY_SCOPE_WORKGROUP);
        return __builtin_amdgcn_readfirstlane((int)old) == 1;
      };
      if (NRAW && kb == 0) {
        #pragma unroll
        for (int j = 0; j < 4; ++j) ST128(x[j], rsL, lane * 16 + j * 1024, parw + EX_INR + u * IU_RAW + hf * 4096, AUX_SYS);
        vm_drain();
        if (second(2) && tell) __hip_atomic_store((GLB u32*)(peer_ws + OFF_FIN + (rank * 576 + 512 + rg) * 128), call << 4, __ATOMIC_RELAXED, __HIP_MEMORY_SCOPE_SYSTEM);
      }
      export_half(x, rsL, parw, u, hf, smem + L_SCR + wave * SCR_W, lane);
      vm_drain();
      if (second(kb) && tell) __hip_atomic_store((GLB u32*)(peer_ws + OFF_FIN + (rank * 576 + u) * 128), call << 4, __ATOMIC_RELAXED, __HIP_MEMORY_SCOPE_SYSTEM);
      #pragma unroll
      for (int j = 0; j < 4; ++j) ST128(x[j], rsL, lane * 16 + j * 1024, xoff + hf * 4096, AUX_SYS);
      vm_drain();
      if (second(4 + kb)) in_place(p.ws, rg, rank * 4 + kb, call, lane);
    } else if (hf == 0) {
      u32x4 x[8];
      #pragma unroll
      for (int j = 0; j < 8; ++j)
        x[j] = __builtin_amdgcn_raw_buffer_load_b128(rsX, ((j >> 1) * 16 + li) * KS + (j & 1) * 64 + lg * 16, rg * 64 * KS + kb * 128, 0);
      asm volatile("s_barrier" ::: "memory");
      call = __builtin_amdgcn_readfirstlane(tick) + 1u;
      const u32 lines = export_short(x, rsL, OFF_PAR + (call & 1) * PAR_BYTES, u, smem + L_SCR + wave * SCR_W, lane);
      vm_drain();
      if (tell) __hip_atomic_store((GLB u32*)(peer_ws + OFF_FIN + (rank * 576 + u) * 128), call << 4 | lines, __ATOMIC_RELAXED, __HIP_MEMORY_SCOPE_SYSTEM);
      store_raw_unit(x, rsL, xoff, lane);
      vm_drain();
      in_place(p.ws, rg, rank * 4 + kb, call, lane);
    } else {
      asm volatile("s_barrier" ::: "memory");
    }
  }
  call = __builtin_amdgcn_readfirstlane(tick) + 1u;
  const u32 par = OFF_PAR + (call & 1) * PAR_BYTES;
  const float scale = p.sx[0] * p.sw[0];
  const rsrc_t rsO = mk_rsrc(p.out, 512u * NN * 4u);

  if (b >= NGEMM) {
    const int pb = b - NGEMM, link = pb / PULL_WGS, q = pb % PULL_WGS;
    const int src = (rank + 1 + link) & 7;
    const u64 pw = (u64)peer_ws;
    const u32 pw_lo = (u32)__builtin_amdgcn_readlane((int)(u32)pw, src), pw_hi = (u32)__builtin_amdgcn_readlane((int)(u32)(pw >> 32), src);
    const rsrc_t rsP = mk_rsrc((const void*)(((u64)pw_hi << 32) | (u64)pw_lo), p.ws_bytes);
    const u32 ctl = (u32)(uintptr_t)(smem + L_CTL);
    const u32 C_READY = ctl, C_LANDED = ctl + 4, C_CLOSED = ctl + 12, C_REL = ctl + 16, C_ISSUED = ctl + 8, C_Q = ctl + 64;
    const bool same = (src >> 2) == (rank >> 2);
    if (wave == 0) ctl_st(ctl + lane * 4, 0u);
    asm volatile("s_waitcnt lgkmcnt(0)\n\ts_barrier" ::: "memory");
    if (wave == 0) {
      int g = 0, pub = 0;
      #define LANDED(v) do { if ((v) > pub) { pub = (v); ctl_st(C_LANDED, (u32)pub); } } while (0)
      #define PIECEW(W, SZ, VOFF, OFF) do { __builtin_amdgcn_raw_ptr_buffer_load_lds(rsP, (LDS void*)(dst + (OFF)), SZ, VOFF, so + (OFF), 0, AUX_SYS); \
        asm volatile("s_waitcnt vmcnt(%0)" :: "i"((W) - 1) : "memory"); ++g; LANDED(g - ((W) - 1)); } while (0)
      #define PIECE(SZ, VOFF, OFF) PIECEW(WIN, SZ, VOFF, OFF)
      for (u32 n = 0;; ++n) {
        if (ctl_ld(C_READY) <= n) {
          vm_drain(); LANDED(g);
          bool fin = false;
          while (ctl_ld(C_READY) <= n) { if (ctl_ld(C_CLOSED)) { fin = ctl_ld(C_READY) <= n; break; } __builtin_amdgcn_s_sleep(1); }
          if (fin) break;
        }
        const u32 ent = ctl_ld(C_Q + (n & 15) * 4);
        ctl_st(C_ISSUED, n + 1);
        if (ent >> 31) continue;
        const u32 need = n >= NSLOT ? n - NSLOT + 1 : 0, ra = C_REL + (n % NSLOT) * 4;
        if (ctl_ld(ra) < need) {
          vm_drain(); LANDED(g);
          while (ctl_ld(ra) < need) __builtin_amdgcn_s_sleep(1);
        }
        const int i = ent & 1023;
        LDS char* dst = smem + (n % NSLOT) * SLOT;
        if (ent & 1024) {
          const u32 so = par + EX_INR + i * IU_RAW;
          #pragma unroll
          for (int j = 0; j < 8; ++j) PIECE(16, lane * 16, j * 1024);
        } else if (i < 512 && quick_form(i)) {
          const u32 so = par + EX_INE + (i < 256 ? i : 256 + group_of(rank, (i >> 6) & 3) * 64 + (i & 63)) * IU_ENC;
          PIECE(4, lane * 4, IU_SIDE);
          #pragma unroll
          for (int j = 0; j < 6; ++j) PIECE(16, lane * 16, j * 1024);
        } else if (i < 512) {
          const u32 so = par + EX_INE + (i < 256 ? i : 256 + group_of(rank, (i >> 6) & 3) * 64 + (i & 63)) * IU_ENC;
          if ((u32)lane < ((ent >> 24) & 15u) * 8) __builtin_amdgcn_raw_ptr_buffer_load_lds(rsP, (LDS void*)(dst + QV), 16, lane * 16, so + QV, 0, AUX_SYS);
          asm volatile("s_waitcnt vmcnt(%0)" :: "i"(WIN - 1) : "memory"); ++g; LANDED(g - (WIN - 1));
          #pragma unroll
          for (int j = 0; j < 5; ++j) PIECE(16, lane * 16, j * 1024);
        } else {
          const u32 so = par + EX_OUE + (rank * 320 + i - 512) * OU_ENC;
          if ((u32)lane < 32 + ((ent >> 24) & 15u) * 8)
            __builtin_amdgcn_raw_ptr_buffer_load_lds(rsP, (LDS void*)(dst + 11264), 16, lane * 16, so + 11264, 0, AUX_SYS);
          asm volatile("s_waitcnt vmcnt(%0)" :: "i"(WIN_OUT - 1) : "memory"); ++g; LANDED(g - (WIN_OUT - 1));
          #pragma unroll
          for (int j = 0; j < 11; ++j) PIECEW(WIN_OUT, 16, lane * 16, j * 1024);
        }
      }
      vm_drain(); LANDED(g);
      #undef PIECE
      #undef PIECEW
      #undef LANDED
      return;
    }
    if (wave == 1) {
      const u32 tic0 = OFF_TIC + (call & 1) * 24 * 128;
      const int only0 = lane == 0 ? 0 : (int)0xF0000000u;
      if (q == 0 && (lane == link || (link == 0 && lane >= 8 && lane < 24)))
        __hip_atomic_store((GLB u32*)(p.ws + OFF_TIC + ((~call & 1) * 24 + lane) * 128), 0u, __ATOMIC_RELAXED, __HIP_MEMORY_SCOPE_AGENT);
      constexpr u32 NPAIR = 208 + NRAW / 2, NFIX = 192;
      u32 n = 0, endp = 0; bool ok = true;
      const u32 gtic = tic0 + (8 + group_of(src, lane & 3)) * 128;
      auto fullest = [&](u32 v) -> int {
        v = v >= 16u || (!same && (lane & 3) == (rank & 3)) ? 0xFFFFu : (v << 4) | ((((lane - q) & 3) << 2) | (lane & 3));
        v = min(v, (u32)__builtin_amdgcn_mov_dpp((int)v, 0xB1, 0xf, 0xf, true));
        v = min(v, (u32)__builtin_amdgcn_mov_dpp((int)v, 0x4E, 0xf, 0xf, true));
        v = (u32)__builtin_amdgcn_readfirstlane((int)v);
        return v == 0xFFFFu ? -2 : (int)(v & 3u);
      };
      auto draw = [&](int g) { return __builtin_amdgcn_raw_ptr_buffer_atomic_add_i32(1, rsL, only0, tic0 + (g < 0 ? link : 8 + group_of(src, g)) * 128, 0); };
      int tnext = draw(-1);
      for (;;) {
        const u32 t = (u32)__builtin_amdgcn_readfirstlane(tnext);
        if (t >= NPAIR + NFIX) break;
        while (n >= ctl_ld(C_ISSUED) + (t < NPAIR ? LOOK : LOOK_OUT)) __builtin_amdgcn_s_sleep(1);
        const bool last = t + PULL_WGS >= NPAIR + NFIX;
        if (!last) tnext = draw(-1);
        int i;
        if (t < NPAIR) {
          const int e = t < NRAW ? t : 2 * t - NRAW + (lane & 1);
          int u = -1, sg = 0, kb = 0;
          if (e < NRAW) u = e;
          else if (e < NRAW + 14) { const int z = e - NRAW; kb = 1; u = z < 6 ? (z >> 1) * 4 + (z & 1) : 12 + ((z - 6) >> 1) * 5 + 1 + (z & 1); }
          else if (e < 46) u = e - 14;
          else if (e < 64) { const int z = e - 46; kb = 1; u = z < 6 ? (z >> 1) * 4 + 2 + (z & 1) : 12 + ((z - 6) / 3) * 5 + ((z - 6) % 3 ? 2 + (z - 6) % 3 : 0); }
          else if (e < 64 + X2FIRST) { kb = 2; u = e - 64; }
          else if (e < 88 + X2FIRST) sg = (e - 64 - X2FIRST) >> 3;
          else if (e < 120) { kb = 2; u = e - 88; }
          else if (e < 144) sg = (e - 96) >> 3;
          else if (e < 176) { kb = 3; u = e - 144; }
          else sg = (e - 128) >> 3;
          int c = 0;
          if (u < 0) {
            if (sg < 6) { c = (0x210100 >> (sg * 4)) & 7; kb = (0x012010 >> (sg * 4)) & 3; }
            else if (sg < 26) { c = ((sg - 6) >> 2) + ((sg - 6) & 3); kb = 3 - ((sg - 6) & 3); }
            else { c = (0x776765 >> ((sg - 26) * 4)) & 7; kb = (0x323123 >> ((sg - 26) * 4)) & 3; }
          }
          const int m = c * 8 + (e & 7);
          const int v = u - 12;
          if (e >= 384) i = (8 * rank + (e & 7)) * 4 + ((e - 384) >> 3);
          else if (u < 0) i = 256 + ((((m >> 1) & 3) * 8 + (m >> 3)) * 2 + (m & 1)) * 4 + kb;
          else if (u < 12) i = (8 * ((rank & 4) | ((rank + 1 + (u >> 2)) & 3)) + (u & 3)) * 4 + kb;
          else i = (8 * ((~rank & 4) | (v / 5)) + 3 + v % 5) * 4 + kb;
        } else {
          const int o = t - NPAIR, cm = same ? o / 3 : o >> 2, k = same ? o - cm * 3 : 1 + (o & 3);
          const int c = same ? cm >> 3 : cm / 6, j3 = (cm - c * 6) >> 1, j = same ? (cm >> 1) & 3 : j3 + (j3 >= (rank & 3));
          i = 512 + ((j * 8 + c) * 2 + (cm & 1)) * 5 + k;
        }
        const bool unp = t < NRAW, two = !unp && t < NPAIR;
        const u32 fo = i >= 512 ? OFF_FOUT + (src * 320 + i - 512) * 128 : OFF_FIN + (src * 576 + (unp ? 512 + (i >> 2) : i)) * 128;
        u32 got = 0;
        ok = ok && wait_ge(p.ws, fo, call << 4, call, got);
        const u32 lines = got >> 4 == call ? got & 15u : 8u;
        const u32 k = two ? (lane & 1) : 0, per = !ok ? 0 : t >= NPAIR ? 12 : unp ? 8 : (u32)__builtin_amdgcn_readfirstlane(quick_form(i) ? 7 : 6);
        ctl_st(C_Q + ((n + k) & 15) * 4, (u32)i | (unp ? 1024u : 0u) | ((endp + per * (k + 1)) << 11) | (lines << 24) | (ok ? 0u : 0x80000000u));
        n += two ? 2 : 1; endp += two ? 2 * per : per;
        asm volatile("s_waitcnt lgkmcnt(0)" ::: "memory");
        ctl_st(C_READY, n);
        if (last) break;
      }
      const u32 fo3 = OFF_FOUT + (src * 320 + lane * 5 + (same ? 3 : 0)) * 128;
      const u32 flags = ld_flag(p.ws, fo3);
      u32 seen = ld_flag(p.ws, gtic);
      for (;;) {
        while (n >= ctl_ld(C_ISSUED) + LOOK_END) __builtin_amdgcn_s_sleep(1);
        const int fj = fullest(seen);
        if (fj < 0) break;
        const int tk = draw(fj);
        seen = ld_flag(p.ws, gtic);
        const u32 t = (u32)__builtin_amdgcn_readfirstlane(tk);
        if (t >= 16u) continue;
        const int cg = fj * 16 + (int)t;
        u32 got = (u32)__builtin_amdgcn_readlane((int)flags, cg);
        if (ok && got < call << 4) ok = wait_ge(p.ws, OFF_FOUT + (src * 320 + cg * 5 + (same ? 3 : 0)) * 128, call << 4, call, got);
        if (ok) endp += 12;
        ctl_st(C_Q + (n & 15) * 4, (u32)(512 + cg * 5 + (same ? 3 : 0)) | (endp << 11) | ((got >> 4 == call ? got & 15u : 8u) << 24) | (ok ? 0u : 0x80000000u));
        ++n;
        asm volatile("s_waitcnt lgkmcnt(0)" ::: "memory");
        ctl_st(C_READY, n);
      }
      asm volatile("s_waitcnt lgkmcnt(0)" ::: "memory");
      ctl_st(C_CLOSED, 1u);
      return;
    }
    LDS char* scr = smem + L_PSCR + (wave - 2) * 8 * KiB;
    for (u32 n = wave - 2;; n += NCONS) {
      bool fin = false;
      while (ctl_ld(C_READY) <= n) { if (ctl_ld(C_CLOSED)) { fin = ctl_ld(C_READY) <= n; break; } __builtin_amdgcn_s_sleep(1); }
      if (fin) break;
      const u32 ent = ctl_ld(C_Q + (n & 15) * 4);
      const bool ok = !(ent >> 31);
      const int i = ent & 1023;
      const u32 endp = (ent >> 11) & 0x1FFFu;
      auto landed = [&](u32 upto) { while (ctl_ld(C_LANDED) < upto) __builtin_amdgcn_s_sleep(1); };
      LDS char* sl = smem + (n % NSLOT) * SLOT;
      if (i < 512) {
        if (!ok) continue;
        const bool brief = !(ent & 1024) && !quick_form(i);
        landed(brief ? endp - 5 : endp);
        const int kb = i & 3, rg = (i >> 2) & 63; const bool isx = i < 256;
        const u32 uid = isx ? i : 256 + group_of(rank, (i >> 6) & 3) * 64 + (i & 63);
        const u32 dst = (isx ? OFF_XA : OFF_WA) + ((src * 4 + kb) * 64 + rg) * IU_RAW;
        const u32 rel = C_REL + (n % NSLOT) * 4, offE = par + EX_INE + uid * IU_ENC;
        u32x4 x[8];
        bool raw = false;
        if (ent & 1024) {
          #pragma unroll
          for (int j = 0; j < 8; ++j) x[j] = *(LDS u32x4*)(sl + j * 1024 + lane * 16);
          wave_sync();
          ctl_st(rel, n + 1);
        } else if (quick_form(i)) {
          u32x4 c[6];
          #pragma unroll
          for (int j = 0; j < 6; ++j) c[j] = *(LDS u32x4*)(sl + j * 1024 + lane * 16);
          const u32 sd = *(LDS u32*)(sl + IU_SIDE + lane * 4);
          wave_sync();
          ctl_st(rel, n + 1);
          const u32 hd[2] = {(u32)__builtin_amdgcn_readlane((int)sd, 0), (u32)__builtin_amdgcn_readlane((int)sd, 32)};
          #pragma unroll
          for (int j = 0; j < 8; ++j) {
            const u32 e4 = ((hd[j >> 2] >> 16) & 31u) * 0x04040404u;
            const u32 p0 = c[(j * 3) >> 2][(j * 3) & 3], p1 = c[(j * 3 + 1) >> 2][(j * 3 + 1) & 3], p2 = c[(j * 3 + 2) >> 2][(j * 3 + 2) & 3];
            const u32 o3 = bfi(0x80808080u, p2 << 1, bfi(0x10101010u, p2 >> 1, bfi(0x0C0C0C0Cu, p1 >> 3, p0 >> 5)));
            const u32x4 y = {(p0 & 0x9F9F9F9Fu) + e4, (p1 & 0x9F9F9F9Fu) + e4, (p2 & 0x9F9F9F9Fu) + e4, (o3 & 0x9F9F9F9Fu) + e4};
            *(LDS u32x4*)(scr + j * 1024 + lane * 16) = y;
          }
          wave_sync();
          const u32 cnt = (lane < 32 ? hd[0] : hd[1]) & 0xFFFFu, id0 = 2u * (lane & 31) - 2u;
          if (lane & 31) {
            if (id0 < cnt) patch_in(scr, sd & 0xFFFFu);
            if (id0 + 1 < cnt) patch_in(scr, sd >> 16);
          }
          if (max(hd[0] & 0xFFFFu, hd[1] & 0xFFFFu) > IN_CAP0) {
            for (int j = 0; j < 3; ++j) {
              const u32 s2 = __builtin_amdgcn_raw_buffer_load_b32(rsP, 256 + (lane >> 5) * 384 + (j * 32 + (lane & 31)) * 4, offE + IU_SIDE, AUX_SYS);
              const u32 id = IN_CAP0 + j * 64 + 2 * (lane & 31);
              if (id < cnt) patch_in(scr, s2 & 0xFFFFu);
              if (id + 1 < cnt) patch_in(scr, s2 >> 16);
            }
          }
          wave_sync();
          #pragma unroll
          for (int j = 0; j < 8; ++j) x[j] = *(LDS u32x4*)(scr + j * 1024 + lane * 16);
          #pragma unroll
          for (int h = 0; h < 2; ++h)
            if (hd[h] >> 31) {
              #pragma unroll
              for (int j = h * 4; j < h * 4 + 4; ++j) x[j] = __builtin_amdgcn_raw_buffer_load_b128(rsP, lane * 16 + j * 1024, par + EX_INR + uid * IU_RAW, AUX_SYS);
            }
        } else {
          LDS char* var = sl + QV;
          const u32 hdr = (u32)__builtin_amdgcn_readfirstlane((int)*(LDS u32*)var);
          const u32 pe = *(LDS unsigned short*)(var + 8 + (lane & 7) * 2);
          const u32 tlen = (u32)__builtin_amdgcn_readfirstlane((int)*(LDS u32*)(var + 4));
          raw = hdr >> 31;
          if (!raw) {
            const u32 nesc = hdr & 0x1FFFu, npat = (hdr >> 13) & 0x1FFu, bias = (4u * ((hdr >> 24) & 31u) - 136u) * 0x01010101u;
            const u32 t0 = QV_SEC + (nesc + 31) / 32 * 8;
            if (t0 * 8 + tlen > QV0 * 8 && lane < (int)(QV_END - QV0) / 16)
              *(LDS u32x4*)(var + QV0 + lane * 16) = __builtin_amdgcn_raw_buffer_load_b128(rsP, lane * 16, offE + QV + QV0, AUX_SYS);
            wave_sync();
            u32 tn = (u32)(uintptr_t)scr + Q_TER;
            for (u32 r0 = 0; r0 < tlen; r0 += 1024) {
              const u32 bit0 = r0 + lane * 16;
              const u32 before = *(LDS unsigned short*)(var + t0 + bit0 / 8 - 2);
              u32 m = bit0 < tlen ? *(LDS unsigned short*)(var + t0 + bit0 / 8) : 0u;
              int prev = 15 - (int)__clz(before);
              if (bit0 == 0) { m &= ~1u; prev = 0; }
              const u32 mine = __popc(m), inc = wave_scan_u32(mine);
              LDS unsigned char* at = (LDS unsigned char*)(uintptr_t)(tn + inc - mine);
              for (; m; m &= m - 1) { const int pl = __builtin_ctz(m); *at++ = (unsigned char)((pl - prev) * 4 - 4); prev = pl; }
              tn += (u32)__builtin_amdgcn_readlane((int)inc, 63);
            }
            tn = (u32)(uintptr_t)scr + Q_TER;
            wave_sync();
            for (u32 r0 = 0; r0 < nesc; r0 += 2048) {
              const u32x2 two = *(LDS u32x2*)(var + QV_SEC + r0 / 4 + lane * 8);
              u32 c2[8], e2[8], zo[8], bits = 0;
              #pragma unroll
              for (int k = 0; k < 8; ++k) {
                const u32 z = two[k >> 2] >> (8 * (k & 3)), z2 = bfi(0x0000000Fu, z, z << 12);
                c2[k] = bfi(0x00030003u, z2, z2 << 6) & 0x03030303u;
                e2[k] = (c2[k] << 2) & (c2[k] << 1) & 0x04040404u;
                zo[k] = bits;
                bits = __builtin_amdgcn_udot4(e2[k], 0x01010101u, bits, false);
              }
              const u32 inc = wave_scan_u32(bits >> 2), b4 = (tn + inc) * 4 - bits;
              tn += (u32)__builtin_amdgcn_readlane((int)inc, 63);
              u32 g0[8], g1[8];
              #pragma unroll
              for (int k = 0; k < 8; ++k) {
                zo[k] = (zo[k] + b4) >> 2;
                const LDS u32* at = (const LDS u32*)(uintptr_t)(zo[k] & ~3u);
                g0[k] = at[0]; g1[k] = at[1];
              }
              __builtin_amdgcn_sched_barrier(0);
              #pragma unroll
              for (int k = 0; k < 8; ++k) {
                const u32 nx = __builtin_amdgcn_alignbyte(g1[k], g0[k], zo[k]);
                zo[k] = __builtin_amdgcn_perm(0u, 0x64686C7Cu, c2[k]) - __builtin_amdgcn_perm(nx, 0u, __umul24(e2[k], 0x404040u) + e2[k]);
              }
              *(LDS u32x4*)(scr + r0 + lane * 32) = u32x4{zo[0], zo[1], zo[2], zo[3]};
              *(LDS u32x4*)(scr + r0 + lane * 32 + 16) = u32x4{zo[4], zo[5], zo[6], zo[7]};
            }
            wave_sync();
            if ((u32)lane < npat && lane < (int)Q_PAT0) patch_short(scr, pe);
            if (npat > Q_PAT0) {
              for (int j = 0; j < 2; ++j) {
                const u32 s2 = __builtin_amdgcn_raw_buffer_load_b32(rsP, lane * 4 + j * 256, offE + QV + QV_END, AUX_SYS);
                const u32 id = Q_PAT0 + j * 128 + 2 * lane;
                if (id < npat) patch_short(scr, s2 & 0xFFFFu);
                if (id + 1 < npat) patch_short(scr, s2 >> 16);
              }
            }
            landed(endp);
            u32x4 c[5];
            #pragma unroll
            for (int j = 0; j < 5; ++j) c[j] = *(LDS u32x4*)(sl + j * 1024 + lane * 16);
            wave_sync();
            ctl_st(rel, n + 1);
            u32 a[32], es[32], qo[32], cnt4 = 0;
            #pragma unroll
            for (int g = 0; g < 4; ++g) {
              u32 k5[5];
              #pragma unroll
              for (int k = 0; k < 5; ++k) { k5[k] = c[(g * 5 + k) >> 2][(g * 5 + k) & 3]; a[g * 8 + k] = k5[k]; }
              a[g * 8 + 5] = bfi(0x80808080u, k5[4] << 3, bfi(0x08080808u, k5[3] >> 1, k5[0] >> 4));
              a[g * 8 + 6] = bfi(0x80808080u, k5[4] << 2, bfi(0x08080808u, k5[3] >> 2, k5[1] >> 4));
              a[g * 8 + 7] = bfi(0x80808080u, k5[4] << 1, bfi(0x08080808u, k5[3] >> 3, k5[2] >> 4));
            }
            #pragma unroll
            for (int d = 0; d < 32; ++d) {
              es[d] = a[d] & (a[d] >> 1) & 0x04040404u;
              qo[d] = cnt4;
              cnt4 = __builtin_amdgcn_udot4(es[d], 0x01010101u, cnt4, false);
            }
            const u32 base4 = ((u32)(uintptr_t)scr + wave_scan_u32(cnt4 >> 2)) * 4 - cnt4;
            wave_sync();
            #pragma unroll
            for (int h = 0; h < 2; ++h) {
              u32 g0[16], g1[16];
              #pragma unroll
              for (int k = 0; k < 16; ++k) {
                qo[h * 16 + k] = (base4 + qo[h * 16 + k]) >> 2;
                const LDS u32* at = (const LDS u32*)(uintptr_t)(qo[h * 16 + k] & ~3u);
                g0[k] = at[0]; g1[k] = at[1];
              }
              __builtin_amdgcn_sched_barrier(0);
              #pragma unroll
              for (int k = 0; k < 16; ++k) {
                const int d = h * 16 + k;
                const u32 nx = __builtin_amdgcn_alignbyte(g1[k], g0[k], qo[d]);
                const u32 sel = __umul24(es[d], 0x404040u) + es[d];
                x[d >> 2][d & 3] = (a[d] & 0x8F8F8F8Fu) + __builtin_amdgcn_perm(nx, 0x7C7C7C7Cu, sel) + bias;
              }
              __builtin_amdgcn_sched_barrier(0);
            }
          } else {
            landed(endp);
            ctl_st(rel, n + 1);
          }
        }
        if (raw) {
          #pragma unroll
          for (int j = 0; j < 8; ++j) x[j] = __builtin_amdgcn_raw_buffer_load_b128(rsP, lane * 16 + j * 1024, par + EX_INR + uid * IU_RAW, AUX_SYS);
        }
        store_raw_unit(x, rsL, dst, lane);
        vm_drain();
        in_place(p.ws, (isx ? 0 : 64) + rg, src * 4 + kb, call, lane);
        continue;
      }
      const int t = i - 512, cg = t / 5, rg = t - cg * 5 + (same ? 0 : 3);
      const u32 obase = (rg * 64 * NN + group_of(src, cg >> 4) * 1024 + (cg & 15) * 64) * 4;
      const u32 olane = (li * NN + lg * 4) * 4;
      if (!ok) {
        const u32x4 nan4 = {0x7FC00000u, 0x7FC00000u, 0x7FC00000u, 0x7FC00000u};
        #pragma unroll
        for (int e = 0; e < 16; ++e) ST128(nan4, rsO, olane + (e & 3) * 64, obase + (e >> 2) * 16 * NN * 4, AUX_SYS);
        continue;
      }
      #pragma unroll
      for (int j = 0; j < 4; ++j) *(LDS u32x4*)(scr + j * 1024 + lane * 16) = u32x4{0u, 0u, 0u, 0u};
      landed(endp - 11);
      LDS char* side = sl + OU_SIDE;
      const u32 hdr = (u32)__builtin_amdgcn_readfirstlane((int)*(LDS u32*)side);
      const u32 hd2 = (u32)__builtin_amdgcn_readfirstlane((int)*(LDS unsigned short*)(side + 4));
      const u32 tl[2] = {*(LDS u32*)(sl + 11264 + lane * 4), *(LDS u32*)(sl + 11520 + lane * 4)};
      const u32 uid = rank * 320 + t;
      if (!(hdr >> 31)) {
        const u32 ns = hd2 & 0xFFu, nf = hd2 >> 8; const int g = (int)((hdr >> 20) & 0xFFu);
        const Side sm = side_map(hdr & 1023u, (hdr >> 10) & 1023u, ns, nf);
        const u32 se = *(LDS unsigned short*)(side + sm.s + lane * 2);
        const u32x2 fe = {*(LDS unsigned short*)(side + sm.f + (lane & 31) * 6),
                          *(LDS unsigned short*)(side + sm.f + (lane & 31) * 6 + 2) | ((u32)*(LDS unsigned short*)(side + sm.f + (lane & 31) * 6 + 4) << 16)};
        if ((u32)lane < ns)
          __hip_atomic_fetch_add((LDS u32*)(scr + ((se >> 5) & 63u) * 64 + (((se >> 11) & 15u) ^ 7u) * 4),
                                 ((se & 7u) << 5) << (se & 24u), __ATOMIC_RELAXED, __HIP_MEMORY_SCOPE_WORKGROUP);
        u32 done = 0;
        for (u32 c = lane; c - lane < max(sm.l[0] - sm.h[0], sm.l[1] - sm.h[1]); c += 64) {
          u32 m[2], k[2], nib[2];
          #pragma unroll
          for (int z = 0; z < 2; ++z) m[z] = c < sm.l[z] - sm.h[z] ? *(LDS unsigned char*)(side + sm.h[z] + c) : 0u;
          const u32 mine = __popc(m[0]) | (__popc(m[1]) << 16), inc = wave_scan_u32(mine);
          k[0] = (done + inc - mine) & 0xFFFFu; k[1] = (done + inc - mine) >> 16;
          #pragma unroll
          for (int z = 0; z < 2; ++z) {
            const u32 nb = ((u32)(uintptr_t)side + sm.l[z]) * 8 + k[z] * 4;
            const LDS u32* at = (const LDS u32*)(uintptr_t)((nb >> 3) & ~3u);
            nib[z] = __builtin_amdgcn_alignbit(at[1], at[0], nb);
          }
          #pragma unroll
          for (int z = 0; z < 2; ++z)
            for (u32 d = c * 8 - k[z]; m[z]; m[z] &= m[z] - 1, --d, nib[z] >>= 4) {
              const u32 hi = (d + (u32)__builtin_ctz(m[z])) & 255u;
              __hip_atomic_fetch_add((LDS u32*)(scr + (hi >> 1) * 32 + (nib[z] & 7u) * 4), (z ? 0x80u : 0x02u) << ((nib[z] & 8u) | (hi & 1u) << 4),
                                     __ATOMIC_RELAXED, __HIP_MEMORY_SCOPE_WORKGROUP);
            }
          done += (u32)__builtin_amdgcn_readlane((int)inc, 63);
        }
        const u32 fgq = (u32)lane < nf ? fe.x >> 8 : 16u;
        u32 th[2] = {0u, 0u};
        wave_sync();
        #pragma unroll
        for (int mi = 0; mi < 4; ++mi) {
          landed(mi < 3 ? endp - 8 + 3 * mi : endp);
          u32x4 c[3];
          #pragma unroll
          for (int j = 0; j < (mi < 3 ? 3 : 2); ++j) c[j] = *(LDS u32x4*)(sl + (mi * 3 + j) * 1024 + lane * 16);
          const u32x4 adj = *(LDS u32x4*)(scr + lane * 64 + (mi ^ 1) * 16);
          if (mi == 3) { wave_sync(); ctl_st(C_REL + (n % NSLOT) * 4, n + 1); }
          #pragma unroll
          for (int ni = 0; ni < 4; ++ni) {
            const int k = ni * 3, gq = mi * 4 + ni, be = gq % 7;
            const u32 l0 = gq < 15 ? c[k >> 2][k & 3] : tl[0], l1 = gq < 15 ? c[(k + 1) >> 2][(k + 1) & 3] : tl[1];
            const u32 tp = gq < 14 ? c[(k + 2) >> 2][(k + 2) & 3] : th[gq & 1];
            if (gq < 14) th[gq / 7] = be < 6 ? bfi(0x01010101u << be, tp >> (6 - be), th[gq / 7]) : bfi(0x80808080u, tp << 1, th[gq / 7]);
            const u32 top = bfi(0x40404040u, (tp >> 1) ^ (adj[3 - ni] << 5), tp), low = adj[3 - ni] & 0xE0E0E0E0u;
            const u32 q01 = __builtin_amdgcn_perm(top, low, 0x05010400u), q23 = __builtin_amdgcn_perm(top, low, 0x07030602u);
            float y[4] = {unfix((int)__builtin_amdgcn_perm(q01, l0, 0x05010004u), g + 7), unfix((int)__builtin_amdgcn_perm(q01, l0, 0x07030206u), g + 7),
                          unfix((int)__builtin_amdgcn_perm(q23, l1, 0x05010004u), g + 7), unfix((int)__builtin_amdgcn_perm(q23, l1, 0x07030206u), g + 7)};
            if (nf)
              for (u64 em = __ballot(fgq == (u32)gq); em; em &= em - 1) {
                const int from = __builtin_ctzll(em);
                const u32 rel = ((u32)__builtin_amdgcn_readlane((int)fe.x, from) & 255u) - (u32)lane * 4u;
                const float vv = as_f((u32)__builtin_amdgcn_readlane((int)fe.y, from));
                #pragma unroll
                for (int r = 0; r < 4; ++r) y[r] = rel == (u32)r ? vv : y[r];
              }
            ST128((u32x4{as_u(y[0] * scale), as_u(y[1] * scale), as_u(y[2] * scale), as_u(y[3] * scale)}), rsO, olane + ni * 64, obase + mi * 16 * NN * 4, AUX_SYS);
          }
        }
      } else {
        landed(endp);
        ctl_st(C_REL + (n % NSLOT) * 4, n + 1);
        #pragma unroll
        for (int mi = 0; mi < 4; ++mi) {
          u32x4 r4[4];
          #pragma unroll
          for (int ni = 0; ni < 4; ++ni) r4[ni] = __builtin_amdgcn_raw_buffer_load_b128(rsP, lane * 16 + ni * 1024, par + EX_OUR + uid * OU_RAW + mi * 4096, AUX_SYS);
          #pragma unroll
          for (int ni = 0; ni < 4; ++ni) {
            const f32x4 y = __builtin_bit_cast(f32x4, r4[ni]) * scale;
            ST128(__builtin_bit_cast(u32x4, y), rsO, olane + ni * 64, obase + mi * 16 * NN * 4, AUX_SYS);
          }
        }
      }
    }
    return;
  }

  constexpr int SLOTS = NGEMM / 8;
  const int xc = b & 7, slot = b >> 3, wm = wave >> 1, wn = wave & 1;
  for (int ti = slot; ti < 36; ti += SLOTS) {
    const int j = xc >> 1, ft = ti * 2 + (xc & 1);
    const int c = ti < 24 ? ft / 6 : 4 * (xc & 1) + (ti < 28 ? ti - 24 : (ti - 28) >> 1), cb = j * 8 + c;
    auto row_group = [&](int k) -> int {
      if (ti >= 28) return 8 * rank + 4 * (ti & 1) + k;
      if (ti >= 24) return 8 * ((~rank & 4) | ((j + 1 + min(k, 2)) & 3)) + 3;
      const int t = ti % 3;
      return 8 * (xc & 1 ? (~rank & 4) | ((j + 1 + t) & 3) : (rank & 4) | ((rank + 1 + t) & 3)) + 4 * (xc & 1) + k;
    };
    const int rgw = row_group(wm);
    const u32 deadw = (u32)(uintptr_t)(smem + L_CTL);
    u32 todo = 0xFFFFFFFFu, avail = 0;
    const u32 pick = deadw + 4;
    auto choose = [&](u32 word) {
      if (!avail) {
        const u32 off = OFF_CNT + (lane < 32 ? row_group(lane >> 3) : 64 + cb * 2 + ((lane >> 3) & 1)) * 128 + (lane & 7) * 16;
        const u64 t0 = __builtin_amdgcn_s_memrealtime();
        GLB u32* tmo = (GLB u32*)(p.ws + OFF_TMO);
        for (u32 it = 1;; ++it) {
          const u32x4 w = __builtin_amdgcn_raw_buffer_load_b128(rsL, off, 0, AUX_SYS);
          #pragma unroll
          for (int k = 0; k < 4; ++k) {
            u64 m = __ballot(w[k] >= call);
            m &= m >> 32; m &= m >> 16; m &= m >> 8;
            avail |= ((u32)m & 255u) << (8 * k);
          }
          avail &= todo;
          if (avail) break;
          if ((it & 15u) == 0u && (__hip_atomic_load(tmo, __ATOMIC_RELAXED, __HIP_MEMORY_SCOPE_AGENT) == call ||
                                   __builtin_amdgcn_s_memrealtime() - t0 > DEADLINE_TICKS)) {
            __hip_atomic_store(tmo, call, __ATOMIC_RELAXED, __HIP_MEMORY_SCOPE_AGENT);
            ctl_st(deadw, 1u);
            avail = todo;
            break;
          }
          __builtin_amdgcn_s_sleep(2);
        }
        asm volatile("" ::: "memory");
      }
      u32 bit = (u32)__builtin_ctz(avail);
      avail &= ~(1u << bit); todo &= ~(1u << bit);
      ctl_st(word, (bit & 7u) * 4 + (bit >> 3));
    };
    const u32 xo0 = OFF_XA + rgw * IU_RAW + wn * 4096, wo0 = OFF_WA + cb * 2 * IU_RAW + wave * 2048;
    auto dma = [&](int k, int ks, u32 st) {
      const u32 xo = xo0 + ks * (64 * IU_RAW), wo = wo0 + ks * (64 * IU_RAW);
      LDS char* dx = smem + st + wave * 4096;
      LDS char* dw = smem + st + 32 * KiB + wave * 2048;
      if (k == 0) __builtin_amdgcn_raw_ptr_buffer_load_lds(rsL, (LDS void*)dx, 16, lane * 16, xo, 0, AUX_SYS);
      if (k == 1) __builtin_amdgcn_raw_ptr_buffer_load_lds(rsL, (LDS void*)dx, 16, lane * 16, xo, 1024, AUX_SYS);
      if (k == 2) __builtin_amdgcn_raw_ptr_buffer_load_lds(rsL, (LDS void*)dx, 16, lane * 16, xo, 2048, AUX_SYS);
      if (k == 3) __builtin_amdgcn_raw_ptr_buffer_load_lds(rsL, (LDS void*)dx, 16, lane * 16, xo, 3072, AUX_SYS);
      if (k == 4) __builtin_amdgcn_raw_ptr_buffer_load_lds(rsL, (LDS void*)dw, 16, lane * 16, wo, 0, AUX_SYS);
      if (k == 5) __builtin_amdgcn_raw_ptr_buffer_load_lds(rsL, (LDS void*)dw, 16, lane * 16, wo, 1024, AUX_SYS);
    };
    f32x4 acc[4][4];
    #pragma unroll
    for (int mi = 0; mi < 4; ++mi)
      #pragma unroll
      for (int ni = 0; ni < 4; ++ni) acc[mi][ni] = f32x4{0.f, 0.f, 0.f, 0.f};
    if (wave == 0) {
      if (ti == slot) ctl_st(deadw, 0u);
      choose(pick + 8); choose(pick + 12);
    }
    asm volatile("s_waitcnt vmcnt(0) lgkmcnt(0)\n\ts_barrier" ::: "memory");
    const int ks0 = (int)ctl_ld(pick + 8), ks1 = (int)ctl_ld(pick + 12);
    #pragma unroll
    for (int k = 0; k < 12; ++k) dma(k % 6, k < 6 ? ks0 : ks1, k / 6 * STAGE);
    u32 st = 0;
    u32 flip = 0;
    i32x8 fa[8], fb[8];
    auto step = [&](i32x8 (&cur)[8], const i32x8 (&prev)[8], bool first, bool more, bool last) {
      if (more && wave == 0) choose(pick + flip);
      if (!last) asm volatile("s_waitcnt vmcnt(6) lgkmcnt(0)\n\ts_barrier" ::: "memory");
      else asm volatile("s_waitcnt vmcnt(0) lgkmcnt(0)\n\ts_barrier" ::: "memory");
      const u32 st2 = st == 0 ? 2 * STAGE : st - STAGE;
      LDS char* ax = smem + st + wm * 8192 + lane * 16;
      LDS char* aw = smem + st + 32 * KiB + wn * 8192 + lane * 16;
      u32x4 h[16];
      auto M = [&](int k) {
        if (!first) acc[k >> 2][k & 3] = __builtin_amdgcn_mfma_scale_f32_16x16x128_f8f6f4(prev[k & 3], prev[4 + (k >> 2)], acc[k >> 2][k & 3], 1, 1, 0, 0, 0, 0);
        __builtin_amdgcn_sched_barrier(0);
      };
      auto R = [&](int k) { h[k] = *(LDS u32x4*)(((k >> 1) & 1 ? ax : aw) + (k >> 2) * 2048 + (k & 1) * 1024); __builtin_amdgcn_sched_barrier(0); };
      u32 kv = 0;
      if (more) kv = *(LDS u32*)(uintptr_t)(pick + flip);
      __builtin_amdgcn_sched_barrier(0);
      M(0); R(0); R(1); M(1);
      const int ksn = __builtin_amdgcn_readfirstlane((int)kv);
      #pragma unroll
      for (int k = 1; k < 7; ++k) {
        M(2 * k); R(2 * k); R(2 * k + 1); M(2 * k + 1);
        if (more) dma(k - 1, ksn, st2);
        __builtin_amdgcn_sched_barrier(0);
      }
      R(14); R(15); M(14); M(15);
      #pragma unroll
      for (int f = 0; f < 4; ++f) {
        cur[f] = __builtin_bit_cast(i32x8, __builtin_shufflevector(h[4 * f], h[4 * f + 1], 0, 1, 2, 3, 4, 5, 6, 7));
        cur[4 + f] = __builtin_bit_cast(i32x8, __builtin_shufflevector(h[4 * f + 2], h[4 * f + 3], 0, 1, 2, 3, 4, 5, 6, 7));
      }
      flip ^= 4u;
      st = st == 2 * STAGE ? 0 : st + STAGE;
    };
    step(fa, fb, true, true, false);
    #pragma unroll 1
    for (int s = 1; s < 29; s += 2) { step(fb, fa, false, true, false); step(fa, fb, false, true, false); }
    step(fb, fa, false, true, false);
    step(fa, fb, false, false, false);
    step(fb, fa, false, false, true);
    #pragma unroll
    for (int k = 0; k < 16; ++k)
      acc[k >> 2][k & 3] = __builtin_amdgcn_mfma_scale_f32_16x16x128_f8f6f4(fb[k & 3], fb[4 + (k >> 2)], acc[k >> 2][k & 3], 1, 1, 0, 0, 0, 0);
    int lz = lane; asm volatile("" : "+v"(lz));
    const bool dead = ctl_ld(deadw) != 0u;
    const int owner = rgw >> 3;
    if (owner == rank) {
      const u32 obase = ((rgw & 7) * 64 * NN + group_of(rank, j) * 1024 + c * 128 + wn * 64) * 4, olane = ((lz & 15) * NN + (lz >> 4) * 4) * 4;
      const float sc = dead ? __builtin_nanf("") : scale;
      #pragma unroll
      for (int mi = 0; mi < 4; ++mi)
        #pragma unroll
        for (int ni = 0; ni < 4; ++ni)
          ST128(__builtin_bit_cast(u32x4, acc[mi][ni] * sc), rsO, olane + ni * 64, obase + mi * 16 * NN * 4, AUX_SYS);
    } else if (!dead && !(ti >= 24 && wm == 3)) {
      const int t = (cb * 2 + wn) * 5 + (rgw & 7) - ((owner >> 2) == (rank >> 2) ? 0 : 3);
      const u32 uid = owner * 320 + t;
      const u32 offE = par + EX_OUE + uid * OU_ENC;
      float sa = 0.f;
      #pragma unroll
      for (int mi = 0; mi < 4; ++mi)
        #pragma unroll
        for (int ni = 0; ni < 4; ++ni) sa += __builtin_fabsf(acc[mi][ni][(mi + ni) & 3]);
      const int g = max(0, 24 - ((int)((as_u(wave_sum_f32(sa) * (4.2f / 1024.f)) >> 23) & 255u) - 126));
      LDS char* last1k = smem + wave * 4096;
      LDS char* side = last1k + 512;
      LDS char* pend = last1k + 1536;
      *(LDS u32x2*)(side + lz * 8) = u32x2{0u, 0u};
      u32 npend = 0, mk[4] = {0u, 0u, 0u, 0u};
      auto quad = [&](int gq, u32& l0, u32& l1, u32& tp) {
        u32 qv[4], fr[4];
        #pragma unroll
        for (int r = 0; r < 4; ++r) {
          const float tt = __builtin_amdgcn_ldexpf(acc[gq >> 2][gq & 3][r], g + 7);
          qv[r] = (u32)f2i_sat(tt);
          fr[r] = as_u(__builtin_amdgcn_fractf(tt));
        }
        const u32 p01 = __builtin_amdgcn_perm(qv[1], qv[0], 0x07030400u), p23 = __builtin_amdgcn_perm(qv[3], qv[2], 0x07030400u);
        const u32 low = __builtin_amdgcn_perm(p23, p01, 0x05040100u);
        tp = __builtin_amdgcn_perm(p23, p01, 0x07060302u);
        l0 = __builtin_amdgcn_perm(qv[1], qv[0], 0x06050201u);
        l1 = __builtin_amdgcn_perm(qv[3], qv[2], 0x06050201u);
        mk[gq >> 3] = bfi(0x80808080u >> (gq & 7), (tp ^ (tp << 1)) >> (gq & 7), mk[gq >> 3]);
        mk[2 + (gq >> 3)] = bfi(0x80808080u >> (gq & 7), low >> (gq & 7), mk[2 + (gq >> 3)]);
        const u32 lowest = tp & ~((tp & 0x7F7F7F7Fu) + 0x7F7F7F7Fu) & 0x80808080u;
        const bool odd_one = (fr[0] | fr[1] | fr[2] | fr[3] | (low & 0x7F7F7F7Fu) | lowest) != 0u;
        const u64 bm = __ballot(odd_one);
        if (odd_one) {
          const u32 sl = min(63u, npend + __builtin_amdgcn_mbcnt_hi((u32)(bm >> 32), __builtin_amdgcn_mbcnt_lo((u32)bm, 0u)));
          *(LDS f32x4*)(pend + sl * 16) = acc[gq >> 2][gq & 3];
          *(LDS u32*)(pend + 1024 + sl * 4) = gq * 64 + lz;
        }
        npend += __popcll(bm);
      };
      u32 y0, y1, z0, z1, th[2];
      quad(14, y0, y1, th[0]);
      quad(15, z0, z1, th[1]);
      *(LDS u32*)(last1k + lz * 4) = z0;
      *(LDS u32*)(last1k + 256 + lz * 4) = z1;
      #pragma unroll
      for (int mi = 0; mi < 4; ++mi) {
        u32 pk[12];
        #pragma unroll
        for (int ni = 0; ni < 4; ++ni) {
          const int gq = mi * 4 + ni, be = gq % 7;
          if (gq == 14) { pk[6] = y0; pk[7] = y1; break; }
          quad(gq, pk[ni * 3], pk[ni * 3 + 1], pk[ni * 3 + 2]);
          pk[ni * 3 + 2] = bfi(0x40404040u, be < 6 ? th[gq / 7] << (6 - be) : th[gq / 7] >> 1, pk[ni * 3 + 2]);
        }
        #pragma unroll
        for (int j = 0; j < (mi < 3 ? 3 : 2); ++j)
          ST128((u32x4{pk[j * 4], pk[j * 4 + 1], pk[j * 4 + 2], pk[j * 4 + 3]}), rsL, lz * 16 + j * 1024, offE + mi * 3072, AUX_SYS);
      }
      const u32 mine = (__popc(mk[0]) + __popc(mk[1])) | ((__popc(mk[2]) + __popc(mk[3])) << 16);
      const u32 inc = wave_scan_u32(mine), tot = (u32)__builtin_amdgcn_readlane((int)inc, 63);
      const u32 n0 = tot & 0xFFFFu, n1 = tot >> 16;
      Side sm = side_map(n0, n1, 0u, 0u);
      bool raw = npend > 64u || sm.end > 512u;
      u32 ns = 0, nf = 0;
      if (!raw) {
        wave_sync();
        u32 kk[2] = {(inc - mine) & 0xFFFFu, (inc - mine) >> 16};
        #pragma unroll
        for (int w = 0; w < 4; ++w) {
          const u32 hb = sm.h[w >> 1] * 8 + lz * 4 + (w & 1) * 2, lb = sm.l[w >> 1] * 8;
          for (u32 m = mk[w]; m; m &= m - 1) {
            const u32 bp = (u32)__builtin_ctz(m), k = kk[w >> 1]++, ph = hb + (bp >> 4) + k, pl = lb + k * 4;
            __hip_atomic_fetch_or((LDS u32*)(side + ((ph >> 5) << 2)), 1u << (ph & 31u), __ATOMIC_RELAXED, __HIP_MEMORY_SCOPE_WORKGROUP);
            __hip_atomic_fetch_or((LDS u32*)(side + ((pl >> 5) << 2)), (bp & 15u) << (pl & 31u), __ATOMIC_RELAXED, __HIP_MEMORY_SCOPE_WORKGROUP);
          }
        }
        LDS char* fl = last1k + 1024;
        for (u32 p0 = 0; p0 < npend; p0 += 16) {
          const u32 pi = p0 + (lz >> 2), r = lz & 3;
          const bool on = pi < npend;
          const float v = *(LDS float*)(pend + p0 * 16 + lz * 4);
          const u32 pp = *(LDS u32*)(pend + 1024 + pi * 4);
          const float tt = __builtin_amdgcn_ldexpf(v, g + 7);
          const u32 iv = (u32)f2i_sat(tt);
          const bool full = on && (as_u((float)(int)iv) != as_u(tt) || (iv & 31u)), brief = on && !full && (iv & 96u);
          const u64 bs = __ballot(brief), bf = __ballot(full);
          const u32 ss = ns + __builtin_amdgcn_mbcnt_hi((u32)(bs >> 32), __builtin_amdgcn_mbcnt_lo((u32)bs, 0u));
          const u32 sf = nf + __builtin_amdgcn_mbcnt_hi((u32)(bf >> 32), __builtin_amdgcn_mbcnt_lo((u32)bf, 0u));
          if (brief && sm.s + ss * 2 < 512u) *(LDS unsigned short*)(side + sm.s + ss * 2) = (unsigned short)((pp << 5) | (r << 3) | ((iv >> 5) & 3u));
          if (full && sf < OF_CAP) *(LDS u32x2*)(fl + sf * 8) = u32x2{pp * 4 + r, as_u(v)};
          ns += __popcll(bs); nf += __popcll(bf);
        }
        sm = side_map(n0, n1, ns, nf);
        raw = ns > OS_CAP || nf > OF_CAP || sm.end > 512u;
        if (nf && !raw) {
          wave_sync();
          const u32x2 e = *(LDS u32x2*)(fl + (lz & 31) * 8);
          if ((u32)lz < nf) {
            LDS unsigned short* at = (LDS unsigned short*)(side + sm.f + lz * 6);
            at[0] = (unsigned short)e.x; at[1] = (unsigned short)e.y; at[2] = (unsigned short)(e.y >> 16);
          }
        }
      }
      wave_sync();
      if (lz == 0) {
        *(LDS u32*)side = raw ? 0x80000000u : (n0 | (n1 << 10) | ((u32)g << 20));
        *(LDS unsigned short*)(side + 4) = (unsigned short)(ns | (nf << 8));
      }
      const u32 lines = raw ? 1u : (sm.end + 127) / 128;
      wave_sync();
      if ((u32)lz < 32 + lines * 8) ST128(*(LDS u32x4*)(last1k + lz * 16), rsL, lz * 16, offE + 11264, AUX_SYS);
      if (raw) {
        const u32 offR = par + EX_OUR + uid * OU_RAW;
        #pragma unroll
        for (int mi = 0; mi < 4; ++mi)
          #pragma unroll
          for (int ni = 0; ni < 4; ++ni)
            ST128(__builtin_bit_cast(u32x4, acc[mi][ni]), rsL, lz * 16 + ni * 1024, offR + mi * 4096, AUX_SYS);
      }
      vm_drain();
      if (lz == owner)
        __hip_atomic_store((GLB u32*)(peer_ws + OFF_FOUT + (rank * 320 + t) * 128), call << 4 | lines, __ATOMIC_RELAXED, __HIP_MEMORY_SCOPE_SYSTEM);
    }
  }
}

}

extern "C" void kernel_launch(int rank, int nranks, ncclComm_t comm, hipStream_t stream,
                              void* const* d_in, const int* in_sizes, int n_in,
                              void* d_out, int out_size,
                              const int* in_dtypes, int out_dtype,
                              void* const* d_out_all,
                              void* const* d_ws_all, size_t ws_bytes) {
  (void)comm; (void)in_sizes; (void)n_in; (void)out_size; (void)in_dtypes; (void)out_dtype; (void)d_out_all;
  if (nranks != NR || (unsigned)rank >= (unsigned)NR || ws_bytes < WS_NEED) { fprintf(stderr, "unexpected launch shape\n"); return; }
  static unsigned attr_set = 0;
  if (!(attr_set >> rank & 1u)) {
    HIP_CHECK(hipFuncSetAttribute((const void*)gemm_rs_kernel, hipFuncAttributeMaxDynamicSharedMemorySize, (int)LDS_BYTES));
    attr_set |= 1u << rank;
  }
  Params p;
  p.x = (const uint8_t*)d_in[0]; p.w = (const uint8_t*)d_in[1]; p.sx = (const float*)d_in[2]; p.sw = (const float*)d_in[3];
  p.out = (float*)d_out; p.ws_all = (char* const*)d_ws_all; p.ws = (char*)d_ws_all[rank]; p.rank = rank; p.ws_bytes = WS_NEED;
  hipLaunchKernelGGL(gemm_rs_kernel, dim3(NWG), dim3(NTHR), LDS_BYTES, stream, p);
  HIP_CHECK(hipGetLastError());
}
```

```cpp
#include <hip/hip_runtime.h>
#include <rccl/rccl.h>
#include <cstdio>
#include <cstdint>

#define HIP_CHECK(expr) do { hipError_t _e = (expr); if (_e != hipSuccess) { \
    fprintf(stderr, "HIP error %s at %s:%d\n", hipGetErrorString(_e), __FILE__, __LINE__); \
    return; } } while (0)

namespace {

typedef unsigned u32;
typedef unsigned long long u64;
typedef unsigned u32x4 __attribute__((ext_vector_type(4)));
typedef unsigned u32x2 __attribute__((ext_vector_type(2)));
typedef int i32x2 __attribute__((ext_vector_type(2)));
typedef int i32x8 __attribute__((ext_vector_type(8)));
typedef float f32x4 __attribute__((ext_vector_type(4)));
typedef __amdgpu_buffer_rsrc_t rsrc_t;
#define LDS __attribute__((address_space(3)))
#define GLB __attribute__((address_space(1)))

constexpr int NR = 8, MM = 4096, NN = 16384, KS = 512;
constexpr int NWG = 256, NTHR = 512;
constexpr int PULL_WGS = 8;
constexpr int WIN = 8, WIN_OUT = 7;
constexpr u32 LOOK = 5, LOOK_OUT = 3, LOOK_END = 2;
constexpr int NRAW = 8;
constexpr int X2FIRST = 8;
constexpr int NPULL = 7 * PULL_WGS, NGEMM = NWG - NPULL;
constexpr int AUX_SYS = 17;
constexpr u64 DEADLINE_TICKS = 40000000ull;

constexpr u32 KiB = 1024, MiB = 1024 * 1024;
constexpr u32 OFF_CALL = 0;
constexpr u32 OFF_TMO = 32 * KiB;
constexpr u32 OFF_TIC = 33 * KiB;
constexpr u32 OFF_CNT = 40 * KiB;
constexpr u32 OFF_FIN = 128 * KiB;
constexpr u32 OFF_FOUT = OFF_FIN + 8 * 576 * 128;
constexpr u32 OFF_XA = 2 * MiB;
constexpr u32 OFF_WA = OFF_XA + 16 * MiB;
constexpr u32 OFF_PAR = OFF_WA + 16 * MiB;
constexpr u32 IU_ENC = 8 * KiB, IU_RAW = 8 * KiB;
constexpr u32 OU_ENC = 12 * KiB, OU_RAW = 16 * KiB;
constexpr u32 IU_SIDE = 6144, OU_SIDE = 11776;
constexpr u32 EX_INE = 0;
constexpr u32 EX_INR = EX_INE + 1280 * IU_ENC;
constexpr u32 EX_OUE = EX_INR + 1280 * IU_RAW;
constexpr u32 EX_OUR = EX_OUE + 2560 * OU_ENC;
constexpr u32 PAR_BYTES = EX_OUR + 2560 * OU_RAW;
constexpr u32 WS_NEED = OFF_PAR + 2 * PAR_BYTES;
static_assert(OFF_CNT + 128 * 128 <= OFF_FIN && OFF_FOUT + 8 * 320 * 128 <= OFF_XA, "flags");
static_assert(WS_NEED <= 257u * MiB, "workspace");
constexpr u32 QV = 5120, QV0 = 1024, QV_SEC = 24, QV_END = 1792, QV_ALL = 2304;
constexpr u32 Q_ESC = 6144, Q_TER = 6400, Q_TMAX = 1760, Q_PAT0 = 8, Q_PAT = 264;
constexpr u32 IN_CAP0 = 62, IN_CAP = 254;
constexpr u32 OS_CAP = 64, OF_CAP = 32;

constexpr u32 STAGE = 48 * KiB;
constexpr u32 L_SCR = 64 * KiB, SCR_W = 10752;
constexpr u32 NSLOT = 8, SLOT = 13 * KiB, NCONS = 6;
constexpr u32 L_PSCR = NSLOT * SLOT;
constexpr u32 L_CTL = L_PSCR + NCONS * 8 * KiB;
constexpr u32 LDS_BYTES = L_CTL + 256;
static_assert(3 * STAGE <= L_CTL && L_SCR + 8 * SCR_W <= LDS_BYTES && LDS_BYTES <= 160 * KiB, "lds");

__device__ __forceinline__ bool quick_form(int i) { return i < 256 && ((i & 3) == 0 || (i & 11) == 1); }
__device__ __forceinline__ int group_of(int rank, int j) { return rank < 4 ? rank * 4 + j : j * 4 + (rank & 3); }

struct Params {
  const uint8_t* x; const uint8_t* w; const float* sx; const float* sw;
  float* out; char* const* ws_all; char* ws; int rank; u32 ws_bytes;
};

__device__ __forceinline__ rsrc_t mk_rsrc(const void* p, u32 bytes) {
  u64 a = (u64)p;
  u32 lo = __builtin_amdgcn_readfirstlane((u32)a), hi = __builtin_amdgcn_readfirstlane((u32)(a >> 32));
  return __builtin_amdgcn_make_buffer_rsrc((void*)(((u64)hi << 32) | lo), 0, (int)bytes, 0x00020000);
}
#define ST128(v, rs, voff, soff, aux) __builtin_amdgcn_raw_buffer_store_b128((v), (rs), (int)((u32)(voff) + (u32)(soff)), 0, (aux))
__device__ __forceinline__ float as_f(u32 v) { return __builtin_bit_cast(float, v); }
__device__ __forceinline__ u32 as_u(float v) { return __builtin_bit_cast(u32, v); }
__device__ __forceinline__ int f2i_sat(float v) {
  int r; asm("v_cvt_i32_f32 %0, %1" : "=v"(r) : "v"(v)); return r;
}
__device__ __forceinline__ void vm_drain() { asm volatile("s_waitcnt vmcnt(0)" ::: "memory"); }
__device__ __forceinline__ void wave_sync() {
  asm volatile("s_waitcnt lgkmcnt(0)" ::: "memory");
}
__device__ __forceinline__ u32 bfi(u32 m, u32 a, u32 b) { return __builtin_amdgcn_bitop3_b32(m, a, b, 0xCA); }
template <int CTRL, int ROWS> __device__ __forceinline__ u32 dpp0(u32 v) { return (u32)__builtin_amdgcn_update_dpp(0, (int)v, CTRL, ROWS, 0xf, false); }
__device__ __forceinline__ u32 wave_scan_u32(u32 v) {
  v += dpp0<0x111, 0xf>(v); v += dpp0<0x112, 0xf>(v); v += dpp0<0x114, 0xf>(v); v += dpp0<0x118, 0xf>(v);
  v += dpp0<0x142, 0xa>(v); v += dpp0<0x143, 0xc>(v);
  return v;
}
__device__ __forceinline__ float wave_sum_f32(float v) {
  v += as_f(dpp0<0x111, 0xf>(as_u(v))); v += as_f(dpp0<0x112, 0xf>(as_u(v))); v += as_f(dpp0<0x114, 0xf>(as_u(v)));
  v += as_f(dpp0<0x118, 0xf>(as_u(v))); v += as_f(dpp0<0x142, 0xa>(as_u(v))); v += as_f(dpp0<0x143, 0xc>(as_u(v)));
  return as_f((u32)__builtin_amdgcn_readlane((int)as_u(v), 63));
}
__device__ __forceinline__ u32 ctl_ld(u32 addr) {
  u32 v; asm volatile("ds_read_b32 %0, %1\n\ts_waitcnt lgkmcnt(0)" : "=&v"(v) : "v"(addr) : "memory");
  return (u32)__builtin_amdgcn_readfirstlane((int)v);
}
__device__ __forceinline__ void ctl_st(u32 addr, u32 v) { asm volatile("ds_write_b32 %0, %1" :: "v"(addr), "v"(v) : "memory"); }
__device__ __forceinline__ u32 ld_flag(const char* ws, u32 off) {
  return __hip_atomic_load((const GLB u32*)(ws + off), __ATOMIC_RELAXED, __HIP_MEMORY_SCOPE_SYSTEM);
}
__device__ __forceinline__ bool wait_ge(char* ws, u32 off, u32 target, u32 call, u32& got) {
  if (__all((got = ld_flag(ws, off)) >= target)) return true;
  const u64 t0 = __builtin_amdgcn_s_memrealtime();
  GLB u32* tmo = (GLB u32*)(ws + OFF_TMO);
  for (u32 it = 1;; ++it) {
    __builtin_amdgcn_s_sleep(2);
    if (__all((got = ld_flag(ws, off)) >= target)) return true;
    if ((it & 15u) == 0u &&
        (__hip_atomic_load(tmo, __ATOMIC_RELAXED, __HIP_MEMORY_SCOPE_AGENT) == call ||
         __builtin_amdgcn_s_memrealtime() - t0 > DEADLINE_TICKS)) {
      __hip_atomic_store(tmo, call, __ATOMIC_RELAXED, __HIP_MEMORY_SCOPE_AGENT);
      return false;
    }
  }
}
__device__ __forceinline__ void in_place(char* ws, u32 line, u32 ks, u32 call, int lane) {
  if (lane == 0) __hip_atomic_store((GLB u32*)(ws + OFF_CNT + line * 128 + ks * 4), call, __ATOMIC_RELAXED, __HIP_MEMORY_SCOPE_SYSTEM);
}

__device__ __forceinline__ void store_raw_unit(const u32x4 (&x)[8], rsrc_t rs, u32 off, int lane) {
  #pragma unroll
  for (int j = 0; j < 8; ++j) ST128(x[j], rs, lane * 16 + j * 1024, off, AUX_SYS);
}

__device__ __forceinline__ void export_half(const u32x4 (&x)[4], rsrc_t rs, u32 par, u32 uid, int hf, LDS char* scr, int lane) {
  const u32 offE = par + EX_INE + uid * IU_ENC;
  u32 s = 0;
  #pragma unroll
  for (int j = 0; j < 4; ++j) {
    *(LDS u32x4*)(scr + j * 1024 + lane * 16) = x[j];
    s = __builtin_amdgcn_sad_u8(x[j][j] & 0x7C7C7C7Cu, 0u, s);
  }
  s = (u32)__builtin_amdgcn_readlane((int)wave_scan_u32(s), 63);
  int e0 = ((int)s - 17121) >> 12;
  e0 = e0 < 0 ? 0 : (e0 > 24 ? 24 : e0);
  const u32 e4 = (u32)e0 * 0x04040404u;
  u32 mk[2] = {0, 0};
  u32 pk[12];
  #pragma unroll
  for (int j = 0; j < 4; ++j) {
    u32 o[4];
    #pragma unroll
    for (int c = 0; c < 4; ++c) {
      const u32 u = (x[j][c] | 0x80808080u) - e4;
      o[c] = bfi(0x1F1F1F1Fu, u, x[j][c]);
      const u32 t = (((u >> 1) ^ 0x40404040u) & 0x70707070u) + 0x70707070u;
      const int d = j * 4 + c;
      mk[d >> 3] = bfi(0x80808080u >> (d & 7), t >> (d & 7), mk[d >> 3]);
    }
    pk[j * 3 + 0] = bfi(0x60606060u, o[3] << 5, o[0]);
    pk[j * 3 + 1] = bfi(0x60606060u, o[3] << 3, o[1]);
    pk[j * 3 + 2] = bfi(0x60606060u, bfi(0x40404040u, o[3] >> 1, o[3] << 1), o[2]);
  }
  #pragma unroll
  for (int j = 0; j < 3; ++j)
    ST128((u32x4{pk[j * 4], pk[j * 4 + 1], pk[j * 4 + 2], pk[j * 4 + 3]}), rs, lane * 16 + j * 1024, offE + hf * 3072, AUX_SYS);
  const u32 mine = __popc(mk[0]) + __popc(mk[1]);
  const u32 inc = wave_scan_u32(mine);
  const u32 total = (u32)__builtin_amdgcn_readlane((int)inc, 63);
  if (total <= IN_CAP) {
    LDS unsigned short* list = (LDS unsigned short*)(scr + 8192);
    u32 slot = 2 + inc - mine;
    #pragma unroll
    for (int q = 0; q < 2; ++q) {
      u32 m = mk[q];
      while (m) {
        const int bp = __builtin_ctz(m); m &= m - 1;
        const int d = q * 8 + 7 - (bp & 7);
        list[slot++] = (unsigned short)((d >> 2) * 1024 + lane * 16 + (d & 3) * 4 + (bp >> 3));
      }
    }
    wave_sync();
    for (u32 j = 0; j < 2; ++j) {
      const u32 pp = *(LDS u32*)(scr + 8192 + j * 256 + lane * 4);
      const u32 p0 = pp & 4095u, p1 = (pp >> 16) & 4095u;
      const u32 b0 = *(LDS unsigned char*)(scr + p0), b1 = *(LDS unsigned char*)(scr + p1);
      u32 sd = (((p0 << 3) | ((128u + (b0 & 127u) - 4u * (u32)e0) >> 5) | (p1 << 19) | (((128u + (b1 & 127u) - 4u * (u32)e0) >> 5) << 16))) + hf * 0x80008000u;
      if (j == 0 && lane == 0) sd = total | ((u32)e0 << 16);
      const u32 at = j * 64 + lane;
      if (at < 32 || total > IN_CAP0)
        __builtin_amdgcn_raw_buffer_store_b32(sd, rs, at < 32 ? hf * 128 + at * 4 : 128 + hf * 384 + at * 4, offE + IU_SIDE, AUX_SYS);
      if (total <= 126) break;
    }
  } else {
    #pragma unroll
    for (int j = 0; j < 4; ++j) ST128(x[j], rs, lane * 16 + j * 1024, par + EX_INR + uid * IU_RAW + hf * 4096, AUX_SYS);
    if (lane == 0) __builtin_amdgcn_raw_buffer_store_b32(0x80000000u, rs, hf * 128, offE + IU_SIDE, AUX_SYS);
  }
}

__device__ __forceinline__ u32 export_short(const u32x4 (&x)[8], rsrc_t rs, u32 par, u32 uid, LDS char* scr, int lane) {
  typedef unsigned short u16x2 __attribute__((ext_vector_type(2)));
  const u32 offE = par + EX_INE + uid * IU_ENC, offR = par + EX_INR + uid * IU_RAW;
  LDS char* var = scr + 8192;
  LDS char* tab = var + QV_ALL;
  u32 s = 0;
  #pragma unroll
  for (int j = 0; j < 8; ++j) s = __builtin_amdgcn_sad_u8(x[j][j & 3] & 0x7C7C7C7Cu, 0u, s);
  #pragma unroll
  for (int j = 0; j < 7; ++j) *(LDS u32x4*)(scr + j * 1024 + lane * 16) = u32x4{0u, 0u, 0u, 0u};
  *(LDS u32x4*)(var + lane * 16) = u32x4{0u, 0u, 0u, 0u};
  *(LDS u32x4*)(var + 1024 + lane * 16) = u32x4{0u, 0u, 0u, 0u};
  {
    u32 sel = 0x0C0C0C0Cu, at = 0;
    #pragma unroll
    for (int y = 0; y < 4; ++y) if ((lane >> y) & 1) { sel = (sel & ~(0xFFu << at)) | ((u32)y << at); at += 8; }
    if (lane < 16) *(LDS u32*)(tab + lane * 4) = sel;
  }
  s = (u32)__builtin_amdgcn_readlane((int)wave_scan_u32(s), 63);
  const int K = max(5, min(31, ((int)s >> 13) + 3));
  const u32 kw = (128u + 4u * (u32)K) * 0x01010101u, tabo = (u32)(uintptr_t)tab;
  auto front = [&](u32 F) { return *(LDS u32*)(uintptr_t)__builtin_amdgcn_udot4(F, 0x20100804u, tabo, false); };
  u32 cs[32], cn[32], car[20], mine = 0;
  wave_sync();
  #pragma unroll
  for (int g = 0; g < 4; ++g) {
    u32 a[8], uu[8];
    #pragma unroll
    for (int k = 0; k < 8; ++k) {
      const int d = g * 8 + k;
      const u32 xv = x[d >> 2][d & 3];
      const u32 u = uu[k] = (kw - (xv & 0x7C7C7C7Cu)) ^ 0x80808080u;
      const u32 esc = (((u >> 1) & 0x78787878u) + 0x78787878u) | ~((u | (u << 1)) << 4);
      const u32 F = (esc >> 7) & 0x01010101u;
      a[k] = bfi(0x0C0C0C0Cu, ~u | __builtin_bit_cast(u32, __builtin_bit_cast(u16x2, F) * (u16x2){12, 12}), xv);
      cs[d] = front(F);
      cn[d] = mine;
      mine = __builtin_amdgcn_udot4(F, 0x01010101u, mine, false);
    }
    car[g * 5 + 0] = bfi(0x70707070u, a[5] << 4, a[0]);
    car[g * 5 + 1] = bfi(0x70707070u, a[6] << 4, a[1]);
    car[g * 5 + 2] = bfi(0x70707070u, a[7] << 4, a[2]);
    car[g * 5 + 3] = bfi(0x40404040u, a[7] << 3, bfi(0x20202020u, a[6] << 2, bfi(0x10101010u, a[5] << 1, a[3])));
    car[g * 5 + 4] = bfi(0x40404040u, a[7] >> 1, bfi(0x20202020u, a[6] >> 2, bfi(0x10101010u, a[5] >> 3, a[4])));
    __builtin_amdgcn_sched_barrier(0);
    #pragma unroll
    for (int k = 0; k < 8; ++k) cs[g * 8 + k] = __builtin_amdgcn_perm(0u, uu[k], cs[g * 8 + k]);
  }
  #pragma unroll
  for (int j = 0; j < 5; ++j) ST128((u32x4{car[j * 4], car[j * 4 + 1], car[j * 4 + 2], car[j * 4 + 3]}), rs, lane * 16 + j * 1024, offE, AUX_SYS);
  const u32 inc = wave_scan_u32(mine);
  const u32 nesc = (u32)__builtin_amdgcn_readlane((int)inc, 63);
  bool fits = nesc <= Q_ESC;
  u32 lines = 1;
  if (fits) {
    const u32 base = inc - mine;
    #pragma unroll
    for (int d = 0; d < 32; ++d) {
      const u32 q = base + cn[d];
      const u64 v = (u64)cs[d] << ((q & 3u) * 8u);
      LDS u32* at = (LDS u32*)(scr + (q & ~3u));
      __hip_atomic_fetch_or(at, (u32)v, __ATOMIC_RELAXED, __HIP_MEMORY_SCOPE_WORKGROUP);
      __hip_atomic_fetch_or(at + 1, (u32)(v >> 32), __ATOMIC_RELAXED, __HIP_MEMORY_SCOPE_WORKGROUP);
    }
    wave_sync();
    const u32 t0 = (QV_SEC + (nesc + 31) / 32 * 8) * 8;
    u32 tbit = t0, npat = 0, nter = 0;
    for (u32 r0 = 0; r0 < nesc && fits; r0 += 2048) {
      const u32x4 e0 = *(LDS u32x4*)(scr + r0 + lane * 32), e1 = *(LDS u32x4*)(scr + r0 + lane * 32 + 16);
      u32 two[2] = {0u, 0u}, zq[8], zl[8], fs[8], t3[8], mk = 0, bits = 0, nt = 0;
      #pragma unroll
      for (int k = 0; k < 8; ++k) {
        const u32 u = k < 4 ? e0[k & 3] : e1[k & 3], sel = (u >> 2) & 0x07070707u;
        const u32 far = ((((u >> 1) & 0x70707070u) + 0x70707070u) >> 7) & 0x01010101u;
        const u32 c2 = __builtin_amdgcn_perm(0x03030201u, 0x03030300u, sel) | far | (far << 1);
        const u32 F = c2 & (c2 >> 1) & 0x01010101u;
        two[k >> 2] |= __builtin_amdgcn_udot4(c2, 0x40100401u, 0u, false) << (8 * (k & 3));
        fs[k] = front(F); t3[k] = ((sel + 0x02020202u) & 0x07070707u) + 0x01010101u;
        nt = __builtin_amdgcn_udot4(F, 0x01010101u, nt, false);
        const u32 lo7 = u & 0x7F7F7F7Fu;
        const u32 out = (F << 7) & (~(lo7 + 0x68686868u) | (lo7 + 0x48484848u) | u);
        mk |= out >> k;
      }
      __builtin_amdgcn_sched_barrier(0);
      #pragma unroll
      for (int k = 0; k < 8; ++k) {
        const u32 cum = __builtin_amdgcn_perm(0u, t3[k], fs[k]) * 0x01010101u;
        zl[k] = (1u << (cum & 31u)) | (1u << ((cum >> 8) & 31u)) | (1u << ((cum >> 16) & 31u)) | (1u << ((cum >> 24) & 31u));
        zq[k] = bits | ((cum >> 29) << 16);
        bits += cum >> 24;
      }
      if (r0 + lane * 32 < nesc) *(LDS u32x2*)(var + QV_SEC + r0 / 4 + lane * 8) = u32x2{two[0], two[1]};
      const u32 np = __popc(mk), inc2 = wave_scan_u32(bits | (np << 16)), tot = (u32)__builtin_amdgcn_readlane((int)inc2, 63);
      nter += (u32)__builtin_amdgcn_readlane((int)wave_scan_u32(nt), 63);
      fits = tbit + (tot & 0xFFFFu) + 96 <= QV_END * 8 && npat + (tot >> 16) <= Q_PAT && nter <= Q_TMAX;
      if (!fits) break;
      const u32 b0 = tbit + (inc2 & 0xFFFFu) - bits;
      if (bits) {
        #pragma unroll
        for (int k = 0; k < 8; ++k) {
          const u32 q = b0 + (zq[k] & 0xFFFFu);
          const u64 v = (((u64)(zq[k] >> 16) << 32) | zl[k]) << (q & 31u);
          LDS u32* at = (LDS u32*)(var + ((q >> 5) << 2));
          __hip_atomic_fetch_or(at, (u32)v, __ATOMIC_RELAXED, __HIP_MEMORY_SCOPE_WORKGROUP);
          __hip_atomic_fetch_or(at + 1, (u32)(v >> 32), __ATOMIC_RELAXED, __HIP_MEMORY_SCOPE_WORKGROUP);
        }
      }
      u32 slot = npat + (inc2 >> 16) - np;
      for (u32 m = mk; m; m &= m - 1) {
        const int bp = __builtin_ctz(m);
        const u32 sn = r0 + lane * 32 + (7 - (bp & 7)) * 4 + (bp >> 3);
        const int v = (int)*(LDS signed char*)(scr + sn) >> 2;
        *(LDS unsigned short*)(var + (slot < Q_PAT0 ? 8 + slot * 2 : QV_END - Q_PAT0 * 2 + slot * 2)) = (unsigned short)((sn << 3) | (u32)(((v - 6) >> 3) + 4));
        ++slot;
      }
      tbit += tot & 0xFFFFu; npat += tot >> 16;
    }
    if (fits) {
      if (lane == 0) *(LDS u32x2*)var = u32x2{nesc | (npat << 13) | ((u32)K << 24), tbit - t0 + 1};
      lines = tbit / 1024 + 1;
      wave_sync();
      if ((u32)lane < lines * 8) ST128(*(LDS u32x4*)(var + lane * 16), rs, lane * 16, offE + QV, AUX_SYS);
      if (lines > 8 && lane < (int)(QV_END - QV0) / 16) ST128(*(LDS u32x4*)(var + QV0 + lane * 16), rs, lane * 16, offE + QV + QV0, AUX_SYS);
      if (npat > Q_PAT0 && lane < (int)(QV_ALL - QV_END) / 16) ST128(*(LDS u32x4*)(var + QV_END + lane * 16), rs, lane * 16, offE + QV + QV_END, AUX_SYS);
    }
  }
  if (!fits) {
    store_raw_unit(x, rs, offR, lane);
    if (lane == 0) __builtin_amdgcn_raw_buffer_store_b32(0x80000000u, rs, 0, offE + QV, AUX_SYS);
  }
  return min(lines, 8u);
}

__device__ __forceinline__ void patch_short(LDS char* scr, u32 ent) {
  const u32 pos = ent >> 3;
  __hip_atomic_fetch_add((LDS u32*)(scr + (pos & ~3u)), ((4u - (ent & 7u)) << 5) << ((pos & 3u) * 8u), __ATOMIC_RELAXED, __HIP_MEMORY_SCOPE_WORKGROUP);
}
__device__ __forceinline__ void patch_in(LDS char* scr, u32 ent) {
  const u32 pos = ent >> 3;
  __hip_atomic_fetch_add((LDS u32*)(scr + (pos & ~3u)), (((ent & 7u) - 4u) << 5) << ((pos & 3u) * 8u), __ATOMIC_RELAXED, __HIP_MEMORY_SCOPE_WORKGROUP);
}

struct Side { u32 h[2], l[2], s, f, end; };
__device__ __forceinline__ Side side_map(u32 n0, u32 n1, u32 ns, u32 nf) {
  Side m;
  m.h[0] = 6; m.l[0] = m.h[0] + (256 + n0 + 7) / 8; m.h[1] = m.l[0] + (n0 + 1) / 2; m.l[1] = m.h[1] + (256 + n1 + 7) / 8;
  m.s = (m.l[1] + (n1 + 1) / 2 + 1) & ~1u; m.f = m.s + ns * 2; m.end = m.f + nf * 6;
  return m;
}

__global__ void __launch_bounds__(NTHR, 2) gemm_rs_kernel(Params p) {
  extern __shared__ __attribute__((aligned(16))) char smem_[];
  LDS char* smem = (LDS char*)smem_;
  const int tid = threadIdx.x, lane = tid & 63, wave = __builtin_amdgcn_readfirstlane(tid >> 6);
  const int b = blockIdx.x, rank = p.rank;
  const int lg = lane >> 4, li = lane & 15;
  const rsrc_t rsL = mk_rsrc(p.ws, p.ws_bytes);
  const u32 tick = (u32)__builtin_amdgcn_raw_ptr_buffer_atomic_add_i32(1, rsL, lane == 0 ? 0 : (int)0xF0000000u, OFF_CALL + b * 128 + wave * 16, 0);
  char* const peer_ws = p.ws_all[lane & 7];
  u32 call;

  if (b < 128) {
    const rsrc_t rsW = mk_rsrc(p.w, KS * NN);
    __builtin_amdgcn_s_sleep(12);
    #pragma unroll
    for (int i = 0; i < 8; ++i) {
      const int row = (wave * 8 + i) * 8 + (lane >> 3);
      const int sw = ((row >> 1) & 3) | (((row >> 4) & 1) << 2);
      __builtin_amdgcn_raw_ptr_buffer_load_lds(rsW, (LDS void*)(smem + (wave * 8 + i) * 1024), 16, row * NN + (((lane & 7) ^ sw) << 4), b * 128, 0, 0);
    }
    asm volatile("s_waitcnt vmcnt(0)\n\ts_barrier" ::: "memory");
    call = __builtin_amdgcn_readfirstlane(tick) + 1u;
    const int cgl = wave & 1, kb = wave >> 1;
    u32x4 x[8];
    #pragma unroll
    for (int f = 0; f < 4; ++f)
      #pragma unroll
      for (int h = 0; h < 2; ++h)
        #pragma unroll
        for (int t = 0; t < 2; ++t) {
          const int row = kb * 128 + 64 * h + 16 * lg + 8 * t + (li >> 1);
          const int sw = ((row >> 1) & 3) | (((row >> 4) & 1) << 2);
          const i32x2 v = __builtin_amdgcn_ds_read_tr8_b64_v2i32((LDS i32x2*)(smem + row * 128 + (((cgl * 4 + f) ^ sw) << 4) + (li & 1) * 8));
          x[f * 2 + h][t * 2] = v.x; x[f * 2 + h][t * 2 + 1] = v.y;
        }
    const int gl = b >> 5, gr = 4 + ((b >> 3) & 3), c = b & 7;
    const u32 lines = export_short(x, rsL, OFF_PAR + (call & 1) * PAR_BYTES, 256 + (b * 2 + cgl) * 4 + kb, smem + L_SCR + wave * SCR_W, lane);
    vm_drain();
    if ((lane == gl || lane == gr) && lane != rank) {
      const int cb = (lane == gl ? gr & 3 : gl) * 8 + c;
      __hip_atomic_store((GLB u32*)(peer_ws + OFF_FIN + (rank * 576 + 256 + (cb * 2 + cgl) * 4 + kb) * 128), call << 4 | lines, __ATOMIC_RELAXED, __HIP_MEMORY_SCOPE_SYSTEM);
    }
    if (rank == gl || rank == gr) {
      const int cb = (rank == gl ? gr & 3 : gl) * 8 + c;
      store_raw_unit(x, rsL, OFF_WA + ((rank * 4 + kb) * 64 + cb * 2 + cgl) * IU_RAW, lane);
      vm_drain();
      in_place(p.ws, 64 + cb * 2 + cgl, rank * 4 + kb, call, lane);
    }
  } else if (b < 192) {
    const int rg = b - 128, kb = wave & 3, hf = wave >> 2, u = rg * 4 + kb;
    const rsrc_t rsX = mk_rsrc(p.x, MM * KS);
    const u32 both = (u32)(uintptr_t)(smem + L_CTL) + 32;
    const u32 xoff = OFF_XA + ((rank * 4 + kb) * 64 + rg) * IU_RAW;
    const int ow = rg >> 3, r8 = rg & 7;
    const u32 users = ((r8 <= 3 ? 0xFu << (ow & 4) : 0u) | (r8 >= 3 ? 0xFu << (~ow & 4) : 0u) | (1u << ow)) & ~(1u << rank);
    const bool tell = lane < NR && ((users >> lane) & 1);
    if (quick_form(u)) {
      u32x4 x[4];
      #pragma unroll
      for (int j = 0; j < 4; ++j)
        x[j] = __builtin_amdgcn_raw_buffer_load_b128(rsX, ((j >> 1) * 16 + li) * KS + (j & 1) * 64 + lg * 16, (rg * 64 + hf * 32) * KS + kb * 128, 0);
      if (wave == 0) ctl_st(both + (lane & 7) * 4, 0u);
      asm volatile("s_waitcnt lgkmcnt(0)\n\ts_barrier" ::: "memory");
      call = __builtin_amdgcn_readfirstlane(tick) + 1u;
      const u32 parw = OFF_PAR + (call & 1) * PAR_BYTES;
      auto second = [&](int k) -> bool {
        u32 old = 0;
        if (lane == 0) old = __hip_atomic_fetch_add((LDS u32*)(uintptr_t)(both + k * 4), 1u, __ATOMIC_RELAXED, __HIP_MEMORY_SCOPE_WORKGROUP);
        return __builtin_amdgcn_readfirstlane((int)old) == 1;
      };
      if (NRAW && kb == 0) {
        #pragma unroll
        for (int j = 0; j < 4; ++j) ST128(x[j], rsL, lane * 16 + j * 1024, parw + EX_INR + u * IU_RAW + hf * 4096, AUX_SYS);
        vm_drain();
        if (second(2) && tell) __hip_atomic_store((GLB u32*)(peer_ws + OFF_FIN + (rank * 576 + 512 + rg) * 128), call << 4, __ATOMIC_RELAXED, __HIP_MEMORY_SCOPE_SYSTEM);
      }
      export_half(x, rsL, parw, u, hf, smem + L_SCR + wave * SCR_W, lane);
      vm_drain();
      if (second(kb) && tell) __hip_atomic_store((GLB u32*)(peer_ws + OFF_FIN + (rank * 576 + u) * 128), call << 4, __ATOMIC_RELAXED, __HIP_MEMORY_SCOPE_SYSTEM);
      #pragma unroll
      for (int j = 0; j < 4; ++j) ST128(x[j], rsL, lane * 16 + j * 1024, xoff + hf * 4096, AUX_SYS);
      vm_drain();
      if (second(4 + kb)) in_place(p.ws, rg, rank * 4 + kb, call, lane);
    } else if (hf == 0) {
      u32x4 x[8];
      #pragma unroll
      for (int j = 0; j < 8; ++j)
        x[j] = __builtin_amdgcn_raw_buffer_load_b128(rsX, ((j >> 1) * 16 + li) * KS + (j & 1) * 64 + lg * 16, rg * 64 * KS + kb * 128, 0);
      asm volatile("s_barrier" ::: "memory");
      call = __builtin_amdgcn_readfirstlane(tick) + 1u;
      const u32 lines = export_short(x, rsL, OFF_PAR + (call & 1) * PAR_BYTES, u, smem + L_SCR + wave * SCR_W, lane);
      vm_drain();
      if (tell) __hip_atomic_store((GLB u32*)(peer_ws + OFF_FIN + (rank * 576 + u) * 128), call << 4 | lines, __ATOMIC_RELAXED, __HIP_MEMORY_SCOPE_SYSTEM);
      store_raw_unit(x, rsL, xoff, lane);
      vm_drain();
      in_place(p.ws, rg, rank * 4 + kb, call, lane);
    } else {
      asm volatile("s_barrier" ::: "memory");
    }
  }
  call = __builtin_amdgcn_readfirstlane(tick) + 1u;
  const u32 par = OFF_PAR + (call & 1) * PAR_BYTES;
  const float scale = p.sx[0] * p.sw[0];
  const rsrc_t rsO = mk_rsrc(p.out, 512u * NN * 4u);

  if (b >= NGEMM) {
    const int pb = b - NGEMM, link = pb / PULL_WGS, q = pb % PULL_WGS;
    const int src = (rank + 1 + link) & 7;
    const u64 pw = (u64)peer_ws;
    const u32 pw_lo = (u32)__builtin_amdgcn_readlane((int)(u32)pw, src), pw_hi = (u32)__builtin_amdgcn_readlane((int)(u32)(pw >> 32), src);
    const rsrc_t rsP = mk_rsrc((const void*)(((u64)pw_hi << 32) | (u64)pw_lo), p.ws_bytes);
    const u32 ctl = (u32)(uintptr_t)(smem + L_CTL);
    const u32 C_READY = ctl, C_LANDED = ctl + 4, C_CLOSED = ctl + 12, C_REL = ctl + 16, C_ISSUED = ctl + 8, C_Q = ctl + 64;
    const bool same = (src >> 2) == (rank >> 2);
    if (wave == 0) ctl_st(ctl + lane * 4, 0u);
    asm volatile("s_waitcnt lgkmcnt(0)\n\ts_barrier" ::: "memory");
    if (wave == 0) {
      int g = 0, pub = 0;
      #define LANDED(v) do { if ((v) > pub) { pub = (v); ctl_st(C_LANDED, (u32)pub); } } while (0)
      #define PIECEW(W, SZ, VOFF, OFF) do { __builtin_amdgcn_raw_ptr_buffer_load_lds(rsP, (LDS void*)(dst + (OFF)), SZ, VOFF, so + (OFF), 0, AUX_SYS); \
        asm volatile("s_waitcnt vmcnt(%0)" :: "i"((W) - 1) : "memory"); ++g; LANDED(g - ((W) - 1)); } while (0)
      #define PIECE(SZ, VOFF, OFF) PIECEW(WIN, SZ, VOFF, OFF)
      for (u32 n = 0;; ++n) {
        if (ctl_ld(C_READY) <= n) {
          vm_drain(); LANDED(g);
          bool fin = false;
          while (ctl_ld(C_READY) <= n) { if (ctl_ld(C_CLOSED)) { fin = ctl_ld(C_READY) <= n; break; } __builtin_amdgcn_s_sleep(1); }
          if (fin) break;
        }
        const u32 ent = ctl_ld(C_Q + (n & 15) * 4);
        ctl_st(C_ISSUED, n + 1);
        if (ent >> 31) continue;
        const u32 need = n >= NSLOT ? n - NSLOT + 1 : 0, ra = C_REL + (n % NSLOT) * 4;
        if (ctl_ld(ra) < need) {
          vm_drain(); LANDED(g);
          while (ctl_ld(ra) < need) __builtin_amdgcn_s_sleep(1);
        }
        const int i = ent & 1023;
        LDS char* dst = smem + (n % NSLOT) * SLOT;
        if (ent & 1024) {
          const u32 so = par + EX_INR + i * IU_RAW;
          #pragma unroll
          for (int j = 0; j < 8; ++j) PIECE(16, lane * 16, j * 1024);
        } else if (i < 512 && quick_form(i)) {
          const u32 so = par + EX_INE + (i < 256 ? i : 256 + group_of(rank, (i >> 6) & 3) * 64 + (i & 63)) * IU_ENC;
          PIECE(4, lane * 4, IU_SIDE);
          #pragma unroll
          for (int j = 0; j < 6; ++j) PIECE(16, lane * 16, j * 1024);
        } else if (i < 512) {
          const u32 so = par + EX_INE + (i < 256 ? i : 256 + group_of(rank, (i >> 6) & 3) * 64 + (i & 63)) * IU_ENC;
          if ((u32)lane < ((ent >> 24) & 15u) * 8) __builtin_amdgcn_raw_ptr_buffer_load_lds(rsP, (LDS void*)(dst + QV), 16, lane * 16, so + QV, 0, AUX_SYS);
          asm volatile("s_waitcnt vmcnt(%0)" :: "i"(WIN - 1) : "memory"); ++g; LANDED(g - (WIN - 1));
          #pragma unroll
          for (int j = 0; j < 5; ++j) PIECE(16, lane * 16, j * 1024);
        } else {
          const u32 so = par + EX_OUE + (rank * 320 + i - 512) * OU_ENC;
          if ((u32)lane < 32 + ((ent >> 24) & 15u) * 8)
            __builtin_amdgcn_raw_ptr_buffer_load_lds(rsP, (LDS void*)(dst + 11264), 16, lane * 16, so + 11264, 0, AUX_SYS);
          asm volatile("s_waitcnt vmcnt(%0)" :: "i"(WIN_OUT - 1) : "memory"); ++g; LANDED(g - (WIN_OUT - 1));
          #pragma unroll
          for (int j = 0; j < 11; ++j) PIECEW(WIN_OUT, 16, lane * 16, j * 1024);
        }
      }
      vm_drain(); LANDED(g);
      #undef PIECE
      #undef PIECEW
      #undef LANDED
      return;
    }
    if (wave == 1) {
      const u32 tic0 = OFF_TIC + (call & 1) * 24 * 128;
      const int only0 = lane == 0 ? 0 : (int)0xF0000000u;
      if (q == 0 && (lane == link || (link == 0 && lane >= 8 && lane < 24)))
        __hip_atomic_store((GLB u32*)(p.ws + OFF_TIC + ((~call & 1) * 24 + lane) * 128), 0u, __ATOMIC_RELAXED, __HIP_MEMORY_SCOPE_AGENT);
      constexpr u32 NPAIR = 208 + NRAW / 2, NFIX = 192;
      u32 n = 0, endp = 0; bool ok = true;
      const u32 gtic = tic0 + (8 + group_of(src, lane & 3)) * 128;
      auto fullest = [&](u32 v) -> int {
        v = v >= 16u || (!same && (lane & 3) == (rank & 3)) ? 0xFFFFu : (v << 4) | ((((lane - q) & 3) << 2) | (lane & 3));
        v = min(v, (u32)__builtin_amdgcn_mov_dpp((int)v, 0xB1, 0xf, 0xf, true));
        v = min(v, (u32)__builtin_amdgcn_mov_dpp((int)v, 0x4E, 0xf, 0xf, true));
        v = (u32)__builtin_amdgcn_readfirstlane((int)v);
        return v == 0xFFFFu ? -2 : (int)(v & 3u);
      };
      auto draw = [&](int g) { return __builtin_amdgcn_raw_ptr_buffer_atomic_add_i32(1, rsL, only0, tic0 + (g < 0 ? link : 8 + group_of(src, g)) * 128, 0); };
      int tnext = draw(-1);
      for (;;) {
        const u32 t = (u32)__builtin_amdgcn_readfirstlane(tnext);
        if (t >= NPAIR + NFIX) break;
        while (n >= ctl_ld(C_ISSUED) + (t < NPAIR ? LOOK : LOOK_OUT)) __builtin_amdgcn_s_sleep(1);
        const bool last = t + PULL_WGS >= NPAIR + NFIX;
        if (!last) tnext = draw(-1);
        int i;
        if (t < NPAIR) {
          const int e = t < NRAW ? t : 2 * t - NRAW + (lane & 1);
          int u = -1, sg = 0, kb = 0;
          if (e < NRAW) u = e;
          else if (e < NRAW + 14) { const int z = e - NRAW; kb = 1; u = z < 6 ? (z >> 1) * 4 + (z & 1) : 12 + ((z - 6) >> 1) * 5 + 1 + (z & 1); }
          else if (e < 46) u = e - 14;
          else if (e < 64) { const int z = e - 46; kb = 1; u = z < 6 ? (z >> 1) * 4 + 2 + (z & 1) : 12 + ((z - 6) / 3) * 5 + ((z - 6) % 3 ? 2 + (z - 6) % 3 : 0); }
          else if (e < 64 + X2FIRST) { kb = 2; u = e - 64; }
          else if (e < 88 + X2FIRST) sg = (e - 64 - X2FIRST) >> 3;
          else if (e < 120) { kb = 2; u = e - 88; }
          else if (e < 144) sg = (e - 96) >> 3;
          else if (e < 176) { kb = 3; u = e - 144; }
          else sg = (e - 128) >> 3;
          int c = 0;
          if (u < 0) {
            if (sg < 6) { c = (0x210100 >> (sg * 4)) & 7; kb = (0x012010 >> (sg * 4)) & 3; }
            else if (sg < 26) { c = ((sg - 6) >> 2) + ((sg - 6) & 3); kb = 3 - ((sg - 6) & 3); }
            else { c = (0x776765 >> ((sg - 26) * 4)) & 7; kb = (0x323123 >> ((sg - 26) * 4)) & 3; }
          }
          const int m = c * 8 + (e & 7);
          const int v = u - 12;
          if (e >= 384) i = (8 * rank + (e & 7)) * 4 + ((e - 384) >> 3);
          else if (u < 0) i = 256 + ((((m >> 1) & 3) * 8 + (m >> 3)) * 2 + (m & 1)) * 4 + kb;
          else if (u < 12) i = (8 * ((rank & 4) | ((rank + 1 + (u >> 2)) & 3)) + (u & 3)) * 4 + kb;
          else i = (8 * ((~rank & 4) | (v / 5)) + 3 + v % 5) * 4 + kb;
        } else {
          const int o = t - NPAIR, cm = same ? o / 3 : o >> 2, k = same ? o - cm * 3 : 1 + (o & 3);
          const int c = same ? cm >> 3 : cm / 6, j3 = (cm - c * 6) >> 1, j = same ? (cm >> 1) & 3 : j3 + (j3 >= (rank & 3));
          i = 512 + ((j * 8 + c) * 2 + (cm & 1)) * 5 + k;
        }
        const bool unp = t < NRAW, two = !unp && t < NPAIR;
        const u32 fo = i >= 512 ? OFF_FOUT + (src * 320 + i - 512) * 128 : OFF_FIN + (src * 576 + (unp ? 512 + (i >> 2) : i)) * 128;
        u32 got = 0;
        ok = ok && wait_ge(p.ws, fo, call << 4, call, got);
        const u32 lines = got >> 4 == call ? got & 15u : 8u;
        const u32 k = two ? (lane & 1) : 0, per = !ok ? 0 : t >= NPAIR ? 12 : unp ? 8 : (u32)__builtin_amdgcn_readfirstlane(quick_form(i) ? 7 : 6);
        ctl_st(C_Q + ((n + k) & 15) * 4, (u32)i | (unp ? 1024u : 0u) | ((endp + per * (k + 1)) << 11) | (lines << 24) | (ok ? 0u : 0x80000000u));
        n += two ? 2 : 1; endp += two ? 2 * per : per;
        asm volatile("s_waitcnt lgkmcnt(0)" ::: "memory");
        ctl_st(C_READY, n);
        if (last) break;
      }
      const u32 fo3 = OFF_FOUT + (src * 320 + lane * 5 + (same ? 3 : 0)) * 128;
      const u32 flags = ld_flag(p.ws, fo3);
      u32 seen = ld_flag(p.ws, gtic);
      for (;;) {
        while (n >= ctl_ld(C_ISSUED) + LOOK_END) __builtin_amdgcn_s_sleep(1);
        const int fj = fullest(seen);
        if (fj < 0) break;
        const int tk = draw(fj);
        seen = ld_flag(p.ws, gtic);
        const u32 t = (u32)__builtin_amdgcn_readfirstlane(tk);
        if (t >= 16u) continue;
        const int cg = fj * 16 + (int)t;
        u32 got = (u32)__builtin_amdgcn_readlane((int)flags, cg);
        if (ok && got < call << 4) ok = wait_ge(p.ws, OFF_FOUT + (src * 320 + cg * 5 + (same ? 3 : 0)) * 128, call << 4, call, got);
        if (ok) endp += 12;
        ctl_st(C_Q + (n & 15) * 4, (u32)(512 + cg * 5 + (same ? 3 : 0)) | (endp << 11) | ((got >> 4 == call ? got & 15u : 8u) << 24) | (ok ? 0u : 0x80000000u));
        ++n;
        asm volatile("s_waitcnt lgkmcnt(0)" ::: "memory");
        ctl_st(C_READY, n);
      }
      asm volatile("s_waitcnt lgkmcnt(0)" ::: "memory");
      ctl_st(C_CLOSED, 1u);
      return;
    }
    LDS char* scr = smem + L_PSCR + (wave - 2) * 8 * KiB;
    for (u32 n = wave - 2;; n += NCONS) {
      bool fin = false;
      while (ctl_ld(C_READY) <= n) { if (ctl_ld(C_CLOSED)) { fin = ctl_ld(C_READY) <= n; break; } __builtin_amdgcn_s_sleep(1); }
      if (fin) break;
      const u32 ent = ctl_ld(C_Q + (n & 15) * 4);
      const bool ok = !(ent >> 31);
      const int i = ent & 1023;
      const u32 endp = (ent >> 11) & 0x1FFFu;
      auto landed = [&](u32 upto) { while (ctl_ld(C_LANDED) < upto) __builtin_amdgcn_s_sleep(1); };
      LDS char* sl = smem + (n % NSLOT) * SLOT;
      if (i < 512) {
        if (!ok) continue;
        const bool brief = !(ent & 1024) && !quick_form(i);
        landed(brief ? endp - 5 : endp);
        const int kb = i & 3, rg = (i >> 2) & 63; const bool isx = i < 256;
        const u32 uid = isx ? i : 256 + group_of(rank, (i >> 6) & 3) * 64 + (i & 63);
        const u32 dst = (isx ? OFF_XA : OFF_WA) + ((src * 4 + kb) * 64 + rg) * IU_RAW;
        const u32 rel = C_REL + (n % NSLOT) * 4, offE = par + EX_INE + uid * IU_ENC;
        u32x4 x[8];
        bool raw = false;
        if (ent & 1024) {
          #pragma unroll
          for (int j = 0; j < 8; ++j) x[j] = *(LDS u32x4*)(sl + j * 1024 + lane * 16);
          wave_sync();
          ctl_st(rel, n + 1);
        } else if (quick_form(i)) {
          u32x4 c[6];
          #pragma unroll
          for (int j = 0; j < 6; ++j) c[j] = *(LDS u32x4*)(sl + j * 1024 + lane * 16);
          const u32 sd = *(LDS u32*)(sl + IU_SIDE + lane * 4);
          wave_sync();
          ctl_st(rel, n + 1);
          const u32 hd[2] = {(u32)__builtin_amdgcn_readlane((int)sd, 0), (u32)__builtin_amdgcn_readlane((int)sd, 32)};
          #pragma unroll
          for (int j = 0; j < 8; ++j) {
            const u32 e4 = ((hd[j >> 2] >> 16) & 31u) * 0x04040404u;
            const u32 p0 = c[(j * 3) >> 2][(j * 3) & 3], p1 = c[(j * 3 + 1) >> 2][(j * 3 + 1) & 3], p2 = c[(j * 3 + 2) >> 2][(j * 3 + 2) & 3];
            const u32 o3 = bfi(0x80808080u, p2 << 1, bfi(0x10101010u, p2 >> 1, bfi(0x0C0C0C0Cu, p1 >> 3, p0 >> 5)));
            const u32x4 y = {(p0 & 0x9F9F9F9Fu) + e4, (p1 & 0x9F9F9F9Fu) + e4, (p2 & 0x9F9F9F9Fu) + e4, (o3 & 0x9F9F9F9Fu) + e4};
            *(LDS u32x4*)(scr + j * 1024 + lane * 16) = y;
          }
          wave_sync();
          const u32 cnt = (lane < 32 ? hd[0] : hd[1]) & 0xFFFFu, id0 = 2u * (lane & 31) - 2u;
          if (lane & 31) {
            if (id0 < cnt) patch_in(scr, sd & 0xFFFFu);
            if (id0 + 1 < cnt) patch_in(scr, sd >> 16);
          }
          if (max(hd[0] & 0xFFFFu, hd[1] & 0xFFFFu) > IN_CAP0) {
            for (int j = 0; j < 3; ++j) {
              const u32 s2 = __builtin_amdgcn_raw_buffer_load_b32(rsP, 256 + (lane >> 5) * 384 + (j * 32 + (lane & 31)) * 4, offE + IU_SIDE, AUX_SYS);
              const u32 id = IN_CAP0 + j * 64 + 2 * (lane & 31);
              if (id < cnt) patch_in(scr, s2 & 0xFFFFu);
              if (id + 1 < cnt) patch_in(scr, s2 >> 16);
            }
          }
          wave_sync();
          #pragma unroll
          for (int j = 0; j < 8; ++j) x[j] = *(LDS u32x4*)(scr + j * 1024 + lane * 16);
          #pragma unroll
          for (int h = 0; h < 2; ++h)
            if (hd[h] >> 31) {
              #pragma unroll
              for (int j = h * 4; j < h * 4 + 4; ++j) x[j] = __builtin_amdgcn_raw_buffer_load_b128(rsP, lane * 16 + j * 1024, par + EX_INR + uid * IU_RAW, AUX_SYS);
            }
        } else {
          LDS char* var = sl + QV;
          const u32 hdr = (u32)__builtin_amdgcn_readfirstlane((int)*(LDS u32*)var);
          const u32 pe = *(LDS unsigned short*)(var + 8 + (lane & 7) * 2);
          const u32 tlen = (u32)__builtin_amdgcn_readfirstlane((int)*(LDS u32*)(var + 4));
          raw = hdr >> 31;
          if (!raw) {
            const u32 nesc = hdr & 0x1FFFu, npat = (hdr >> 13) & 0x1FFu, bias = (4u * ((hdr >> 24) & 31u) - 136u) * 0x01010101u;
            const u32 t0 = QV_SEC + (nesc + 31) / 32 * 8;
            if (t0 * 8 + tlen > QV0 * 8 && lane < (int)(QV_END - QV0) / 16)
              *(LDS u32x4*)(var + QV0 + lane * 16) = __builtin_amdgcn_raw_buffer_load_b128(rsP, lane * 16, offE + QV + QV0, AUX_SYS);
            wave_sync();
            u32 tn = (u32)(uintptr_t)scr + Q_TER;
            for (u32 r0 = 0; r0 < tlen; r0 += 1024) {
              const u32 bit0 = r0 + lane * 16;
              const u32 before = *(LDS unsigned short*)(var + t0 + bit0 / 8 - 2);
              u32 m = bit0 < tlen ? *(LDS unsigned short*)(var + t0 + bit0 / 8) : 0u;
              int prev = 15 - (int)__clz(before);
              if (bit0 == 0) { m &= ~1u; prev = 0; }
              const u32 mine = __popc(m), inc = wave_scan_u32(mine);
              LDS unsigned char* at = (LDS unsigned char*)(uintptr_t)(tn + inc - mine);
              for (; m; m &= m - 1) { const int pl = __builtin_ctz(m); *at++ = (unsigned char)((pl - prev) * 4 - 4); prev = pl; }
              tn += (u32)__builtin_amdgcn_readlane((int)inc, 63);
            }
            tn = (u32)(uintptr_t)scr + Q_TER;
            wave_sync();
            for (u32 r0 = 0; r0 < nesc; r0 += 2048) {
              const u32x2 two = *(LDS u32x2*)(var + QV_SEC + r0 / 4 + lane * 8);
              u32 c2[8], e2[8], zo[8], bits = 0;
              #pragma unroll
              for (int k = 0; k < 8; ++k) {
                const u32 z = two[k >> 2] >> (8 * (k & 3)), z2 = bfi(0x0000000Fu, z, z << 12);
                c2[k] = bfi(0x00030003u, z2, z2 << 6) & 0x03030303u;
                e2[k] = (c2[k] << 2) & (c2[k] << 1) & 0x04040404u;
                zo[k] = bits;
                bits = __builtin_amdgcn_udot4(e2[k], 0x01010101u, bits, false);
              }
              const u32 inc = wave_scan_u32(bits >> 2), b4 = (tn + inc) * 4 - bits;
              tn += (u32)__builtin_amdgcn_readlane((int)inc, 63);
              u32 g0[8], g1[8];
              #pragma unroll
              for (int k = 0; k < 8; ++k) {
                zo[k] = (zo[k] + b4) >> 2;
                const LDS u32* at = (const LDS u32*)(uintptr_t)(zo[k] & ~3u);
                g0[k] = at[0]; g1[k] = at[1];
              }
              __builtin_amdgcn_sched_barrier(0);
              #pragma unroll
              for (int k = 0; k < 8; ++k) {
                const u32 nx = __builtin_amdgcn_alignbyte(g1[k], g0[k], zo[k]);
                zo[k] = __builtin_amdgcn_perm(0u, 0x64686C7Cu, c2[k]) - __builtin_amdgcn_perm(nx, 0u, __umul24(e2[k], 0x404040u) + e2[k]);
              }
              *(LDS u32x4*)(scr + r0 + lane * 32) = u32x4{zo[0], zo[1], zo[2], zo[3]};
              *(LDS u32x4*)(scr + r0 + lane * 32 + 16) = u32x4{zo[4], zo[5], zo[6], zo[7]};
            }
            wave_sync();
            if ((u32)lane < npat && lane < (int)Q_PAT0) patch_short(scr, pe);
            if (npat > Q_PAT0) {
              for (int j = 0; j < 2; ++j) {
                const u32 s2 = __builtin_amdgcn_raw_buffer_load_b32(rsP, lane * 4 + j * 256, offE + QV + QV_END, AUX_SYS);
                const u32 id = Q_PAT0 + j * 128 + 2 * lane;
                if (id < npat) patch_short(scr, s2 & 0xFFFFu);
                if (id + 1 < npat) patch_short(scr, s2 >> 16);
              }
            }
            landed(endp);
            u32x4 c[5];
            #pragma unroll
            for (int j = 0; j < 5; ++j) c[j] = *(LDS u32x4*)(sl + j * 1024 + lane * 16);
            wave_sync();
            ctl_st(rel, n + 1);
            u32 a[32], es[32], qo[32], cnt4 = 0;
            #pragma unroll
            for (int g = 0; g < 4; ++g) {
              u32 k5[5];
              #pragma unroll
              for (int k = 0; k < 5; ++k) { k5[k] = c[(g * 5 + k) >> 2][(g * 5 + k) & 3]; a[g * 8 + k] = k5[k]; }
              a[g * 8 + 5] = bfi(0x80808080u, k5[4] << 3, bfi(0x08080808u, k5[3] >> 1, k5[0] >> 4));
              a[g * 8 + 6] = bfi(0x80808080u, k5[4] << 2, bfi(0x08080808u, k5[3] >> 2, k5[1] >> 4));
              a[g * 8 + 7] = bfi(0x80808080u, k5[4] << 1, bfi(0x08080808u, k5[3] >> 3, k5[2] >> 4));
            }
            #pragma unroll
            for (int d = 0; d < 32; ++d) {
              es[d] = a[d] & (a[d] >> 1) & 0x04040404u;
              qo[d] = cnt4;
              cnt4 = __builtin_amdgcn_udot4(es[d], 0x01010101u, cnt4, false);
            }
            const u32 base4 = ((u32)(uintptr_t)scr + wave_scan_u32(cnt4 >> 2)) * 4 - cnt4;
            wave_sync();
            #pragma unroll
            for (int h = 0; h < 2; ++h) {
              u32 g0[16], g1[16];
              #pragma unroll
              for (int k = 0; k < 16; ++k) {
                qo[h * 16 + k] = (base4 + qo[h * 16 + k]) >> 2;
                const LDS u32* at = (const LDS u32*)(uintptr_t)(qo[h * 16 + k] & ~3u);
                g0[k] = at[0]; g1[k] = at[1];
              }
              __builtin_amdgcn_sched_barrier(0);
              #pragma unroll
              for (int k = 0; k < 16; ++k) {
                const int d = h * 16 + k;
                const u32 nx = __builtin_amdgcn_alignbyte(g1[k], g0[k], qo[d]);
                const u32 sel = __umul24(es[d], 0x404040u) + es[d];
                x[d >> 2][d & 3] = (a[d] & 0x8F8F8F8Fu) + __builtin_amdgcn_perm(nx, 0x7C7C7C7Cu, sel) + bias;
              }
              __builtin_amdgcn_sched_barrier(0);
            }
          } else {
            landed(endp);
            ctl_st(rel, n + 1);
          }
        }
        if (raw) {
          #pragma unroll
          for (int j = 0; j < 8; ++j) x[j] = __builtin_amdgcn_raw_buffer_load_b128(rsP, lane * 16 + j * 1024, par + EX_INR + uid * IU_RAW, AUX_SYS);
        }
        store_raw_unit(x, rsL, dst, lane);
        vm_drain();
        in_place(p.ws, (isx ? 0 : 64) + rg, src * 4 + kb, call, lane);
        continue;
      }
      const int t = i - 512, cg = t / 5, rg = t - cg * 5 + (same ? 0 : 3);
      const u32 obase = (rg * 64 * NN + group_of(src, cg >> 4) * 1024 + (cg & 15) * 64) * 4;
      const u32 olane = (li * NN + lg * 4) * 4;
      if (!ok) {
        const u32x4 nan4 = {0x7FC00000u, 0x7FC00000u, 0x7FC00000u, 0x7FC00000u};
        #pragma unroll
        for (int e = 0; e < 16; ++e) ST128(nan4, rsO, olane + (e & 3) * 64, obase + (e >> 2) * 16 * NN * 4, AUX_SYS);
        continue;
      }
      #pragma unroll
      for (int j = 0; j < 4; ++j) *(LDS u32x4*)(scr + j * 1024 + lane * 16) = u32x4{0u, 0u, 0u, 0u};
      landed(endp - 11);
      LDS char* side = sl + OU_SIDE;
      const u32 hdr = (u32)__builtin_amdgcn_readfirstlane((int)*(LDS u32*)side);
      const u32 hd2 = (u32)__builtin_amdgcn_readfirstlane((int)*(LDS unsigned short*)(side + 4));
      const u32 tl[2] = {*(LDS u32*)(sl + 11264 + lane * 4), *(LDS u32*)(sl + 11520 + lane * 4)};
      const u32 uid = rank * 320 + t;
      if (!(hdr >> 31)) {
        const u32 ns = hd2 & 0xFFu, nf = hd2 >> 8; const int g = (int)((hdr >> 20) & 0xFFu);
        const Side sm = side_map(hdr & 1023u, (hdr >> 10) & 1023u, ns, nf);
        const u32 se = *(LDS unsigned short*)(side + sm.s + lane * 2);
        const u32x2 fe = {*(LDS unsigned short*)(side + sm.f + (lane & 31) * 6),
                          *(LDS unsigned short*)(side + sm.f + (lane & 31) * 6 + 2) | ((u32)*(LDS unsigned short*)(side + sm.f + (lane & 31) * 6 + 4) << 16)};
        if ((u32)lane < ns)
          __hip_atomic_fetch_add((LDS u32*)(scr + ((se >> 5) & 63u) * 64 + (((se >> 11) & 15u) ^ 7u) * 4),
                                 ((se & 7u) << 5) << (se & 24u), __ATOMIC_RELAXED, __HIP_MEMORY_SCOPE_WORKGROUP);
        u32 done = 0;
        for (u32 c = lane; c - lane < max(sm.l[0] - sm.h[0], sm.l[1] - sm.h[1]); c += 64) {
          u32 m[2], k[2], nib[2];
          #pragma unroll
          for (int z = 0; z < 2; ++z) m[z] = c < sm.l[z] - sm.h[z] ? *(LDS unsigned char*)(side + sm.h[z] + c) : 0u;
          const u32 mine = __popc(m[0]) | (__popc(m[1]) << 16), inc = wave_scan_u32(mine);
          k[0] = (done + inc - mine) & 0xFFFFu; k[1] = (done + inc - mine) >> 16;
          #pragma unroll
          for (int z = 0; z < 2; ++z) {
            const u32 nb = ((u32)(uintptr_t)side + sm.l[z]) * 8 + k[z] * 4;
            const LDS u32* at = (const LDS u32*)(uintptr_t)((nb >> 3) & ~3u);
            nib[z] = __builtin_amdgcn_alignbit(at[1], at[0], nb);
          }
          #pragma unroll
          for (int z = 0; z < 2; ++z)
            for (u32 d = c * 8 - k[z]; m[z]; m[z] &= m[z] - 1, --d, nib[z] >>= 4) {
              const u32 hi = (d + (u32)__builtin_ctz(m[z])) & 255u;
              __hip_atomic_fetch_add((LDS u32*)(scr + (hi >> 1) * 32 + (nib[z] & 7u) * 4), (z ? 0x80u : 0x02u) << ((nib[z] & 8u) | (hi & 1u) << 4),
                                     __ATOMIC_RELAXED, __HIP_MEMORY_SCOPE_WORKGROUP);
            }
          done += (u32)__builtin_amdgcn_readlane((int)inc, 63);
        }
        const u32 fgq = (u32)lane < nf ? fe.x >> 8 : 16u;
        u32 th[2] = {0u, 0u};
        const float down = as_f((u32)(127 - 7 - g) << 23);
        u32 seen = ctl_ld(C_LANDED);
        u32x4 cq[11], aq[4];
        if (seen < endp - 8) { landed(endp - 8); }
        #pragma unroll
        for (int j = 0; j < 3; ++j) cq[j] = *(LDS u32x4*)(sl + j * 1024 + lane * 16);
        #pragma unroll
        for (int j = 0; j < 4; ++j) aq[j] = *(LDS u32x4*)(scr + lane * 64 + (j ^ 1) * 16);
        #pragma unroll
        for (int mi = 0; mi < 4; ++mi) {
          if (mi < 3) {
            const u32 need = mi < 2 ? endp - 5 + 3 * mi : endp;
            if (seen < need) { landed(need); seen = ctl_ld(C_LANDED); }
            #pragma unroll
            for (int j = mi * 3 + 3; j < (mi < 2 ? mi * 3 + 6 : 11); ++j) cq[j] = *(LDS u32x4*)(sl + j * 1024 + lane * 16);
            __builtin_amdgcn_sched_barrier(0);
          } else { wave_sync(); ctl_st(C_REL + (n % NSLOT) * 4, n + 1); }
          const u32x4* c = cq + mi * 3;
          const u32x4 adj = aq[mi];
          #pragma unroll
          for (int ni = 0; ni < 4; ++ni) {
            const int k = ni * 3, gq = mi * 4 + ni, be = gq % 7;
            const u32 l0 = gq < 15 ? c[k >> 2][k & 3] : tl[0], l1 = gq < 15 ? c[(k + 1) >> 2][(k + 1) & 3] : tl[1];
            const u32 tp = gq < 14 ? c[(k + 2) >> 2][(k + 2) & 3] : th[gq & 1];
            if (gq < 14) th[gq / 7] = be < 6 ? bfi(0x01010101u << be, tp >> (6 - be), th[gq / 7]) : bfi(0x80808080u, tp << 1, th[gq / 7]);
            const u32 top = bfi(0x40404040u, (tp >> 1) ^ (adj[3 - ni] << 5), tp), low = adj[3 - ni] & 0xE0E0E0E0u;
            const u32 q01 = __builtin_amdgcn_perm(top, low, 0x05010400u), q23 = __builtin_amdgcn_perm(top, low, 0x07030602u);
            float y[4] = {(float)(int)__builtin_amdgcn_perm(q01, l0, 0x05010004u), (float)(int)__builtin_amdgcn_perm(q01, l0, 0x07030206u),
                          (float)(int)__builtin_amdgcn_perm(q23, l1, 0x05010004u), (float)(int)__builtin_amdgcn_perm(q23, l1, 0x07030206u)};
            #pragma unroll
            for (int r = 0; r < 4; ++r) y[r] = y[r] * down * scale;
            if (nf)
              for (u64 em = __ballot(fgq == (u32)gq); em; em &= em - 1) {
                const int from = __builtin_ctzll(em);
                const u32 rel = ((u32)__builtin_amdgcn_readlane((int)fe.x, from) & 255u) - (u32)lane * 4u;
                const float vv = as_f((u32)__builtin_amdgcn_readlane((int)fe.y, from)) * scale;
                #pragma unroll
                for (int r = 0; r < 4; ++r) y[r] = rel == (u32)r ? vv : y[r];
              }
            ST128((u32x4{as_u(y[0]), as_u(y[1]), as_u(y[2]), as_u(y[3])}), rsO, olane + ni * 64, obase + mi * 16 * NN * 4, AUX_SYS);
          }
        }
      } else {
        landed(endp);
        ctl_st(C_REL + (n % NSLOT) * 4, n + 1);
        #pragma unroll
        for (int mi = 0; mi < 4; ++mi) {
          u32x4 r4[4];
          #pragma unroll
          for (int ni = 0; ni < 4; ++ni) r4[ni] = __builtin_amdgcn_raw_buffer_load_b128(rsP, lane * 16 + ni * 1024, par + EX_OUR + uid * OU_RAW + mi * 4096, AUX_SYS);
          #pragma unroll
          for (int ni = 0; ni < 4; ++ni) {
            const f32x4 y = __builtin_bit_cast(f32x4, r4[ni]) * scale;
            ST128(__builtin_bit_cast(u32x4, y), rsO, olane + ni * 64, obase + mi * 16 * NN * 4, AUX_SYS);
          }
        }
      }
    }
    return;
  }

  constexpr int SLOTS = NGEMM / 8;
  const int xc = b & 7, slot = b >> 3, wm = wave >> 1, wn = wave & 1;
  for (int ti = slot; ti < 36; ti += SLOTS) {
    const int j = xc >> 1, ft = ti * 2 + (xc & 1);
    const int c = ti < 24 ? ft / 6 : 4 * (xc & 1) + (ti < 28 ? ti - 24 : (ti - 28) >> 1), cb = j * 8 + c;
    auto row_group = [&](int k) -> int {
      if (ti >= 28) return 8 * rank + 4 * (ti & 1) + k;
      if (ti >= 24) return 8 * ((~rank & 4) | ((j + 1 + min(k, 2)) & 3)) + 3;
      const int t = ti % 3;
      return 8 * (xc & 1 ? (~rank & 4) | ((j + 1 + t) & 3) : (rank & 4) | ((rank + 1 + t) & 3)) + 4 * (xc & 1) + k;
    };
    const int rgw = row_group(wm);
    const u32 deadw = (u32)(uintptr_t)(smem + L_CTL);
    u32 todo = 0xFFFFFFFFu, avail = 0;
    const u32 pick = deadw + 4;
    auto choose = [&](u32 word) {
      if (!avail) {
        const u32 off = OFF_CNT + (lane < 32 ? row_group(lane >> 3) : 64 + cb * 2 + ((lane >> 3) & 1)) * 128 + (lane & 7) * 16;
        const u64 t0 = __builtin_amdgcn_s_memrealtime();
        GLB u32* tmo = (GLB u32*)(p.ws + OFF_TMO);
        for (u32 it = 1;; ++it) {
          const u32x4 w = __builtin_amdgcn_raw_buffer_load_b128(rsL, off, 0, AUX_SYS);
          #pragma unroll
          for (int k = 0; k < 4; ++k) {
            u64 m = __ballot(w[k] >= call);
            m &= m >> 32; m &= m >> 16; m &= m >> 8;
            avail |= ((u32)m & 255u) << (8 * k);
          }
          avail &= todo;
          if (avail) break;
          if ((it & 15u) == 0u && (__hip_atomic_load(tmo, __ATOMIC_RELAXED, __HIP_MEMORY_SCOPE_AGENT) == call ||
                                   __builtin_amdgcn_s_memrealtime() - t0 > DEADLINE_TICKS)) {
            __hip_atomic_store(tmo, call, __ATOMIC_RELAXED, __HIP_MEMORY_SCOPE_AGENT);
            ctl_st(deadw, 1u);
            avail = todo;
            break;
          }
          __builtin_amdgcn_s_sleep(2);
        }
        asm volatile("" ::: "memory");
      }
      u32 bit = (u32)__builtin_ctz(avail);
      avail &= ~(1u << bit); todo &= ~(1u << bit);
      ctl_st(word, (bit & 7u) * 4 + (bit >> 3));
    };
    const u32 xo0 = OFF_XA + rgw * IU_RAW + wn * 4096, wo0 = OFF_WA + cb * 2 * IU_RAW + wave * 2048;
    auto dma = [&](int k, int ks, u32 st) {
      const u32 xo = xo0 + ks * (64 * IU_RAW), wo = wo0 + ks * (64 * IU_RAW);
      LDS char* dx = smem + st + wave * 4096;
      LDS char* dw = smem + st + 32 * KiB + wave * 2048;
      if (k == 0) __builtin_amdgcn_raw_ptr_buffer_load_lds(rsL, (LDS void*)dx, 16, lane * 16, xo, 0, AUX_SYS);
      if (k == 1) __builtin_amdgcn_raw_ptr_buffer_load_lds(rsL, (LDS void*)dx, 16, lane * 16, xo, 1024, AUX_SYS);
      if (k == 2) __builtin_amdgcn_raw_ptr_buffer_load_lds(rsL, (LDS void*)dx, 16, lane * 16, xo, 2048, AUX_SYS);
      if (k == 3) __builtin_amdgcn_raw_ptr_buffer_load_lds(rsL, (LDS void*)dx, 16, lane * 16, xo, 3072, AUX_SYS);
      if (k == 4) __builtin_amdgcn_raw_ptr_buffer_load_lds(rsL, (LDS void*)dw, 16, lane * 16, wo, 0, AUX_SYS);
      if (k == 5) __builtin_amdgcn_raw_ptr_buffer_load_lds(rsL, (LDS void*)dw, 16, lane * 16, wo, 1024, AUX_SYS);
    };
    f32x4 acc[4][4];
    #pragma unroll
    for (int mi = 0; mi < 4; ++mi)
      #pragma unroll
      for (int ni = 0; ni < 4; ++ni) acc[mi][ni] = f32x4{0.f, 0.f, 0.f, 0.f};
    if (wave == 0) {
      if (ti == slot) ctl_st(deadw, 0u);
      choose(pick + 8); choose(pick + 12);
    }
    asm volatile("s_waitcnt vmcnt(0) lgkmcnt(0)\n\ts_barrier" ::: "memory");
    const int ks0 = (int)ctl_ld(pick + 8), ks1 = (int)ctl_ld(pick + 12);
    #pragma unroll
    for (int k = 0; k < 12; ++k) dma(k % 6, k < 6 ? ks0 : ks1, k / 6 * STAGE);
    u32 st = 0;
    u32 flip = 0;
    i32x8 fa[8], fb[8];
    auto step = [&](i32x8 (&cur)[8], const i32x8 (&prev)[8], bool first, bool more, bool last) {
      if (more && wave == 0) choose(pick + flip);
      if (!last) asm volatile("s_waitcnt vmcnt(6) lgkmcnt(0)\n\ts_barrier" ::: "memory");
      else asm volatile("s_waitcnt vmcnt(0) lgkmcnt(0)\n\ts_barrier" ::: "memory");
      const u32 st2 = st == 0 ? 2 * STAGE : st - STAGE;
      LDS char* ax = smem + st + wm * 8192 + lane * 16;
      LDS char* aw = smem + st + 32 * KiB + wn * 8192 + lane * 16;
      u32x4 h[16];
      auto M = [&](int k) {
        if (!first) acc[k >> 2][k & 3] = __builtin_amdgcn_mfma_scale_f32_16x16x128_f8f6f4(prev[k & 3], prev[4 + (k >> 2)], acc[k >> 2][k & 3], 1, 1, 0, 0, 0, 0);
        __builtin_amdgcn_sched_barrier(0);
      };
      auto R = [&](int k) { h[k] = *(LDS u32x4*)(((k >> 1) & 1 ? ax : aw) + (k >> 2) * 2048 + (k & 1) * 1024); __builtin_amdgcn_sched_barrier(0); };
      u32 kv = 0;
      if (more) kv = *(LDS u32*)(uintptr_t)(pick + flip);
      __builtin_amdgcn_sched_barrier(0);
      M(0); R(0); R(1); M(1);
      const int ksn = __builtin_amdgcn_readfirstlane((int)kv);
      #pragma unroll
      for (int k = 1; k < 7; ++k) {
        M(2 * k); R(2 * k); R(2 * k + 1); M(2 * k + 1);
        if (more) dma(k - 1, ksn, st2);
        __builtin_amdgcn_sched_barrier(0);
      }
      R(14); R(15); M(14); M(15);
      #pragma unroll
      for (int f = 0; f < 4; ++f) {
        cur[f] = __builtin_bit_cast(i32x8, __builtin_shufflevector(h[4 * f], h[4 * f + 1], 0, 1, 2, 3, 4, 5, 6, 7));
        cur[4 + f] = __builtin_bit_cast(i32x8, __builtin_shufflevector(h[4 * f + 2], h[4 * f + 3], 0, 1, 2, 3, 4, 5, 6, 7));
      }
      flip ^= 4u;
      st = st == 2 * STAGE ? 0 : st + STAGE;
    };
    step(fa, fb, true, true, false);
    #pragma unroll 1
    for (int s = 1; s < 29; s += 2) { step(fb, fa, false, true, false); step(fa, fb, false, true, false); }
    step(fb, fa, false, true, false);
    step(fa, fb, false, false, false);
    step(fb, fa, false, false, true);
    #pragma unroll
    for (int k = 0; k < 16; ++k)
      acc[k >> 2][k & 3] = __builtin_amdgcn_mfma_scale_f32_16x16x128_f8f6f4(fb[k & 3], fb[4 + (k >> 2)], acc[k >> 2][k & 3], 1, 1, 0, 0, 0, 0);
    int lz = lane; asm volatile("" : "+v"(lz));
    const bool dead = ctl_ld(deadw) != 0u;
    const int owner = rgw >> 3;
    if (owner == rank) {
      const u32 obase = ((rgw & 7) * 64 * NN + group_of(rank, j) * 1024 + c * 128 + wn * 64) * 4, olane = ((lz & 15) * NN + (lz >> 4) * 4) * 4;
      const float sc = dead ? __builtin_nanf("") : scale;
      #pragma unroll
      for (int mi = 0; mi < 4; ++mi)
        #pragma unroll
        for (int ni = 0; ni < 4; ++ni)
          ST128(__builtin_bit_cast(u32x4, acc[mi][ni] * sc), rsO, olane + ni * 64, obase + mi * 16 * NN * 4, AUX_SYS);
    } else if (!dead && !(ti >= 24 && wm == 3)) {
      const int t = (cb * 2 + wn) * 5 + (rgw & 7) - ((owner >> 2) == (rank >> 2) ? 0 : 3);
      const u32 uid = owner * 320 + t;
      const u32 offE = par + EX_OUE + uid * OU_ENC;
      float sa = 0.f;
      #pragma unroll
      for (int mi = 0; mi < 4; ++mi)
        #pragma unroll
        for (int ni = 0; ni < 4; ++ni) sa += __builtin_fabsf(acc[mi][ni][(mi + ni) & 3]);
      const int g = min(119, max(0, 24 - ((int)((as_u(wave_sum_f32(sa) * (4.2f / 1024.f)) >> 23) & 255u) - 126)));
      LDS char* last1k = smem + wave * 4096;
      LDS char* side = last1k + 512;
      LDS char* pend = last1k + 1536;
      *(LDS u32x2*)(side + lz * 8) = u32x2{0u, 0u};
      u32 npend = 0, mk[4] = {0u, 0u, 0u, 0u};
      auto quad = [&](int gq, u32& l0, u32& l1, u32& tp) {
        u32 qv[4], fr[4];
        #pragma unroll
        for (int r = 0; r < 4; ++r) {
          const float tt = __builtin_amdgcn_ldexpf(acc[gq >> 2][gq & 3][r], g + 7);
          qv[r] = (u32)f2i_sat(tt);
          fr[r] = as_u(__builtin_amdgcn_fractf(tt));
        }
        const u32 p01 = __builtin_amdgcn_perm(qv[1], qv[0], 0x07030400u), p23 = __builtin_amdgcn_perm(qv[3], qv[2], 0x07030400u);
        const u32 low = __builtin_amdgcn_perm(p23, p01, 0x05040100u);
        tp = __builtin_amdgcn_perm(p23, p01, 0x07060302u);
        l0 = __builtin_amdgcn_perm(qv[1], qv[0], 0x06050201u);
        l1 = __builtin_amdgcn_perm(qv[3], qv[2], 0x06050201u);
        mk[gq >> 3] = bfi(0x80808080u >> (gq & 7), (tp ^ (tp << 1)) >> (gq & 7), mk[gq >> 3]);
        mk[2 + (gq >> 3)] = bfi(0x80808080u >> (gq & 7), low >> (gq & 7), mk[2 + (gq >> 3)]);
        const u32 lowest = tp & ~((tp & 0x7F7F7F7Fu) + 0x7F7F7F7Fu) & 0x80808080u;
        const bool odd_one = (fr[0] | fr[1] | fr[2] | fr[3] | (low & 0x7F7F7F7Fu) | lowest) != 0u;
        const u64 bm = __ballot(odd_one);
        if (odd_one) {
          const u32 sl = min(63u, npend + __builtin_amdgcn_mbcnt_hi((u32)(bm >> 32), __builtin_amdgcn_mbcnt_lo((u32)bm, 0u)));
          *(LDS f32x4*)(pend + sl * 16) = acc[gq >> 2][gq & 3];
          *(LDS u32*)(pend + 1024 + sl * 4) = gq * 64 + lz;
        }
        npend += __popcll(bm);
      };
      u32 y0, y1, z0, z1, th[2];
      quad(14, y0, y1, th[0]);
      quad(15, z0, z1, th[1]);
      *(LDS u32*)(last1k + lz * 4) = z0;
      *(LDS u32*)(last1k + 256 + lz * 4) = z1;
      #pragma unroll
      for (int mi = 0; mi < 4; ++mi) {
        u32 pk[12];
        #pragma unroll
        for (int ni = 0; ni < 4; ++ni) {
          const int gq = mi * 4 + ni, be = gq % 7;
          if (gq == 14) { pk[6] = y0; pk[7] = y1; break; }
          quad(gq, pk[ni * 3], pk[ni * 3 + 1], pk[ni * 3 + 2]);
          pk[ni * 3 + 2] = bfi(0x40404040u, be < 6 ? th[gq / 7] << (6 - be) : th[gq / 7] >> 1, pk[ni * 3 + 2]);
        }
        #pragma unroll
        for (int j = 0; j < (mi < 3 ? 3 : 2); ++j)
          ST128((u32x4{pk[j * 4], pk[j * 4 + 1], pk[j * 4 + 2], pk[j * 4 + 3]}), rsL, lz * 16 + j * 1024, offE + mi * 3072, AUX_SYS);
      }
      const u32 mine = (__popc(mk[0]) + __popc(mk[1])) | ((__popc(mk[2]) + __popc(mk[3])) << 16);
      const u32 inc = wave_scan_u32(mine), tot = (u32)__builtin_amdgcn_readlane((int)inc, 63);
      const u32 n0 = tot & 0xFFFFu, n1 = tot >> 16;
      Side sm = side_map(n0, n1, 0u, 0u);
      bool raw = npend > 64u || sm.end > 512u;
      u32 ns = 0, nf = 0;
      if (!raw) {
        wave_sync();
        u32 kk[2] = {(inc - mine) & 0xFFFFu, (inc - mine) >> 16};
        #pragma unroll
        for (int w = 0; w < 4; ++w) {
          const u32 hb = sm.h[w >> 1] * 8 + lz * 4 + (w & 1) * 2, lb = sm.l[w >> 1] * 8;
          for (u32 m = mk[w]; m; m &= m - 1) {
            const u32 bp = (u32)__builtin_ctz(m), k = kk[w >> 1]++, ph = hb + (bp >> 4) + k, pl = lb + k * 4;
            __hip_atomic_fetch_or((LDS u32*)(side + ((ph >> 5) << 2)), 1u << (ph & 31u), __ATOMIC_RELAXED, __HIP_MEMORY_SCOPE_WORKGROUP);
            __hip_atomic_fetch_or((LDS u32*)(side + ((pl >> 5) << 2)), (bp & 15u) << (pl & 31u), __ATOMIC_RELAXED, __HIP_MEMORY_SCOPE_WORKGROUP);
          }
        }
        LDS char* fl = last1k + 1024;
        for (u32 p0 = 0; p0 < npend; p0 += 16) {
          const u32 pi = p0 + (lz >> 2), r = lz & 3;
          const bool on = pi < npend;
          const float v = *(LDS float*)(pend + p0 * 16 + lz * 4);
          const u32 pp = *(LDS u32*)(pend + 1024 + pi * 4);
          const float tt = __builtin_amdgcn_ldexpf(v, g + 7);
          const u32 iv = (u32)f2i_sat(tt);
          const bool full = on && (as_u((float)(int)iv) != as_u(tt) || (iv & 31u)), brief = on && !full && (iv & 96u);
          const u64 bs = __ballot(brief), bf = __ballot(full);
          const u32 ss = ns + __builtin_amdgcn_mbcnt_hi((u32)(bs >> 32), __builtin_amdgcn_mbcnt_lo((u32)bs, 0u));
          const u32 sf = nf + __builtin_amdgcn_mbcnt_hi((u32)(bf >> 32), __builtin_amdgcn_mbcnt_lo((u32)bf, 0u));
          if (brief && sm.s + ss * 2 < 512u) *(LDS unsigned short*)(side + sm.s + ss * 2) = (unsigned short)((pp << 5) | (r << 3) | ((iv >> 5) & 3u));
          if (full && sf < OF_CAP) *(LDS u32x2*)(fl + sf * 8) = u32x2{pp * 4 + r, as_u(v)};
          ns += __popcll(bs); nf += __popcll(bf);
        }
        sm = side_map(n0, n1, ns, nf);
        raw = ns > OS_CAP || nf > OF_CAP || sm.end > 512u;
        if (nf && !raw) {
          wave_sync();
          const u32x2 e = *(LDS u32x2*)(fl + (lz & 31) * 8);
          if ((u32)lz < nf) {
            LDS unsigned short* at = (LDS unsigned short*)(side + sm.f + lz * 6);
            at[0] = (unsigned short)e.x; at[1] = (unsigned short)e.y; at[2] = (unsigned short)(e.y >> 16);
          }
        }
      }
      wave_sync();
      if (lz == 0) {
        *(LDS u32*)side = raw ? 0x80000000u : (n0 | (n1 << 10) | ((u32)g << 20));
        *(LDS unsigned short*)(side + 4) = (unsigned short)(ns | (nf << 8));
      }
      const u32 lines = raw ? 1u : (sm.end + 127) / 128;
      wave_sync();
      if ((u32)lz < 32 + lines * 8) ST128(*(LDS u32x4*)(last1k + lz * 16), rsL, lz * 16, offE + 11264, AUX_SYS);
      if (raw) {
        const u32 offR = par + EX_OUR + uid * OU_RAW;
        #pragma unroll
        for (int mi = 0; mi < 4; ++mi)
          #pragma unroll
          for (int ni = 0; ni < 4; ++ni)
            ST128(__builtin_bit_cast(u32x4, acc[mi][ni]), rsL, lz * 16 + ni * 1024, offR + mi * 4096, AUX_SYS);
      }
      vm_drain();
      if (lz == owner)
        __hip_atomic_store((GLB u32*)(peer_ws + OFF_FOUT + (rank * 320 + t) * 128), call << 4 | lines, __ATOMIC_RELAXED, __HIP_MEMORY_SCOPE_SYSTEM);
    }
  }
}

}

extern "C" void kernel_launch(int rank, int nranks, ncclComm_t comm, hipStream_t stream,
                              void* const* d_in, const int* in_sizes, int n_in,
                              void* d_out, int out_size,
                              const int* in_dtypes, int out_dtype,
                              void* const* d_out_all,
                              void* const* d_ws_all, size_t ws_bytes) {
  (void)comm; (void)in_sizes; (void)n_in; (void)out_size; (void)in_dtypes; (void)out_dtype; (void)d_out_all;
  if (nranks != NR || (unsigned)rank >= (unsigned)NR || ws_bytes < WS_NEED) { fprintf(stderr, "unexpected launch shape\n"); return; }
  static unsigned attr_set = 0;
  if (!(attr_set >> rank & 1u)) {
    HIP_CHECK(hipFuncSetAttribute((const void*)gemm_rs_kernel, hipFuncAttributeMaxDynamicSharedMemorySize, (int)LDS_BYTES));
    attr_set |= 1u << rank;
  }
  Params p;
  p.x = (const uint8_t*)d_in[0]; p.w = (const uint8_t*)d_in[1]; p.sx = (const float*)d_in[2]; p.sw = (const float*)d_in[3];
  p.out = (float*)d_out; p.ws_all = (char* const*)d_ws_all; p.ws = (char*)d_ws_all[rank]; p.rank = rank; p.ws_bytes = WS_NEED;
  hipLaunchKernelGGL(gemm_rs_kernel, dim3(NWG), dim3(NTHR), LDS_BYTES, stream, p);
  HIP_CHECK(hipGetLastError());
}
```

```cpp
#include <hip/hip_runtime.h>
#include <rccl/rccl.h>
#include <cstdio>
#include <cstdint>

#define HIP_CHECK(expr) do { hipError_t _e = (expr); if (_e != hipSuccess) { \
    fprintf(stderr, "HIP error %s at %s:%d\n", hipGetErrorString(_e), __FILE__, __LINE__); \
    return; } } while (0)

namespace {

typedef unsigned u32;
typedef unsigned long long u64;
typedef unsigned u32x4 __attribute__((ext_vector_type(4)));
typedef unsigned u32x2 __attribute__((ext_vector_type(2)));
typedef int i32x2 __attribute__((ext_vector_type(2)));
typedef int i32x8 __attribute__((ext_vector_type(8)));
typedef float f32x4 __attribute__((ext_vector_type(4)));
typedef __amdgpu_buffer_rsrc_t rsrc_t;
#define LDS __attribute__((address_space(3)))
#define GLB __attribute__((address_space(1)))

constexpr int NR = 8, MM = 4096, NN = 16384, KS = 512;
constexpr int NWG = 256, NTHR = 512;
constexpr int PULL_WGS = 8;
constexpr int WIN = 8, WIN_OUT = 7;
constexpr u32 LOOK = 5, LOOK_OUT = 3, LOOK_END = 2;
constexpr int NRAW = 8;
constexpr int X2FIRST = 8;
constexpr int NPULL = 7 * PULL_WGS, NGEMM = NWG - NPULL;
constexpr int AUX_SYS = 17;
constexpr u64 DEADLINE_TICKS = 40000000ull;

constexpr u32 KiB = 1024, MiB = 1024 * 1024;
constexpr u32 OFF_CALL = 0;
constexpr u32 OFF_TMO = 32 * KiB;
constexpr u32 OFF_TIC = 33 * KiB;
constexpr u32 OFF_CNT = 40 * KiB;
constexpr u32 OFF_FIN = 128 * KiB;
constexpr u32 OFF_FOUT = OFF_FIN + 8 * 576 * 128;
constexpr u32 OFF_XA = 2 * MiB;
constexpr u32 OFF_WA = OFF_XA + 16 * MiB;
constexpr u32 OFF_PAR = OFF_WA + 16 * MiB;
constexpr u32 IU_ENC = 8 * KiB, IU_RAW = 8 * KiB;
constexpr u32 OU_ENC = 12 * KiB, OU_RAW = 16 * KiB;
constexpr u32 IU_SIDE = 6144, OU_SIDE = 11776;
constexpr u32 EX_INE = 0;
constexpr u32 EX_INR = EX_INE + 1280 * IU_ENC;
constexpr u32 EX_OUE = EX_INR + 1280 * IU_RAW;
constexpr u32 EX_OUR = EX_OUE + 2560 * OU_ENC;
constexpr u32 PAR_BYTES = EX_OUR + 2560 * OU_RAW;
constexpr u32 WS_NEED = OFF_PAR + 2 * PAR_BYTES;
static_assert(OFF_CNT + 128 * 128 <= OFF_FIN && OFF_FOUT + 8 * 320 * 128 <= OFF_XA, "flags");
static_assert(WS_NEED <= 257u * MiB, "workspace");
constexpr u32 QV = 5120, QV0 = 1024, QV_SEC = 24, QV_END = 1792, QV_ALL = 2304;
constexpr u32 Q_ESC = 6144, Q_TER = 6400, Q_TMAX = 1760, Q_PAT0 = 8, Q_PAT = 264;
constexpr u32 IN_CAP0 = 62, IN_CAP = 254;
constexpr u32 OS_CAP = 64, OF_CAP = 32;

constexpr u32 STAGE = 48 * KiB;
constexpr u32 L_SCR = 64 * KiB, SCR_W = 10752;
constexpr u32 NSLOT = 8, SLOT = 13 * KiB, NCONS = 6;
constexpr u32 L_PSCR = NSLOT * SLOT;
constexpr u32 L_CTL = L_PSCR + NCONS * 8 * KiB;
constexpr u32 LDS_BYTES = L_CTL + 256;
static_assert(3 * STAGE <= L_CTL && L_SCR + 8 * SCR_W <= LDS_BYTES && LDS_BYTES <= 160 * KiB, "lds");

__device__ __forceinline__ bool quick_form(int i) { return i < 256 && ((i & 3) == 0 || (i & 11) == 1); }
__device__ __forceinline__ int group_of(int rank, int j) { return rank < 4 ? rank * 4 + j : j * 4 + (rank & 3); }

struct Params {
  const uint8_t* x; const uint8_t* w; const float* sx; const float* sw;
  float* out; char* const* ws_all; char* ws; int rank; u32 ws_bytes;
};

__device__ __forceinline__ rsrc_t mk_rsrc(const void* p, u32 bytes) {
  u64 a = (u64)p;
  u32 lo = __builtin_amdgcn_readfirstlane((u32)a), hi = __builtin_amdgcn_readfirstlane((u32)(a >> 32));
  return __builtin_amdgcn_make_buffer_rsrc((void*)(((u64)hi << 32) | lo), 0, (int)bytes, 0x00020000);
}
#define ST128(v, rs, voff, soff, aux) __builtin_amdgcn_raw_buffer_store_b128((v), (rs), (int)((u32)(voff) + (u32)(soff)), 0, (aux))
__device__ __forceinline__ float as_f(u32 v) { return __builtin_bit_cast(float, v); }
__device__ __forceinline__ u32 as_u(float v) { return __builtin_bit_cast(u32, v); }
__device__ __forceinline__ int f2i_sat(float v) {
  int r; asm("v_cvt_i32_f32 %0, %1" : "=v"(r) : "v"(v)); return r;
}
__device__ __forceinline__ void vm_drain() { asm volatile("s_waitcnt vmcnt(0)" ::: "memory"); }
__device__ __forceinline__ void wave_sync() {
  asm volatile("s_waitcnt lgkmcnt(0)" ::: "memory");
}
__device__ __forceinline__ u32 bfi(u32 m, u32 a, u32 b) { return __builtin_amdgcn_bitop3_b32(m, a, b, 0xCA); }
template <int CTRL, int ROWS> __device__ __forceinline__ u32 dpp0(u32 v) { return (u32)__builtin_amdgcn_update_dpp(0, (int)v, CTRL, ROWS, 0xf, false); }
__device__ __forceinline__ u32 wave_scan_u32(u32 v) {
  v += dpp0<0x111, 0xf>(v); v += dpp0<0x112, 0xf>(v); v += dpp0<0x114, 0xf>(v); v += dpp0<0x118, 0xf>(v);
  v += dpp0<0x142, 0xa>(v); v += dpp0<0x143, 0xc>(v);
  return v;
}
__device__ __forceinline__ float wave_sum_f32(float v) {
  v += as_f(dpp0<0x111, 0xf>(as_u(v))); v += as_f(dpp0<0x112, 0xf>(as_u(v))); v += as_f(dpp0<0x114, 0xf>(as_u(v)));
  v += as_f(dpp0<0x118, 0xf>(as_u(v))); v += as_f(dpp0<0x142, 0xa>(as_u(v))); v += as_f(dpp0<0x143, 0xc>(as_u(v)));
  return as_f((u32)__builtin_amdgcn_readlane((int)as_u(v), 63));
}
__device__ __forceinline__ u32 ctl_ld(u32 addr) {
  u32 v; asm volatile("ds_read_b32 %0, %1\n\ts_waitcnt lgkmcnt(0)" : "=&v"(v) : "v"(addr) : "memory");
  return (u32)__builtin_amdgcn_readfirstlane((int)v);
}
__device__ __forceinline__ void ctl_st(u32 addr, u32 v) { asm volatile("ds_write_b32 %0, %1" :: "v"(addr), "v"(v) : "memory"); }
__device__ __forceinline__ u32 ld_flag(const char* ws, u32 off) {
  return __hip_atomic_load((const GLB u32*)(ws + off), __ATOMIC_RELAXED, __HIP_MEMORY_SCOPE_SYSTEM);
}
__device__ __forceinline__ bool wait_ge(char* ws, u32 off, u32 target, u32 call, u32& got) {
  if (__all((got = ld_flag(ws, off)) >= target)) return true;
  const u64 t0 = __builtin_amdgcn_s_memrealtime();
  GLB u32* tmo = (GLB u32*)(ws + OFF_TMO);
  for (u32 it = 1;; ++it) {
    __builtin_amdgcn_s_sleep(2);
    if (__all((got = ld_flag(ws, off)) >= target)) return true;
    if ((it & 15u) == 0u &&
        (__hip_atomic_load(tmo, __ATOMIC_RELAXED, __HIP_MEMORY_SCOPE_AGENT) == call ||
         __builtin_amdgcn_s_memrealtime() - t0 > DEADLINE_TICKS)) {
      __hip_atomic_store(tmo, call, __ATOMIC_RELAXED, __HIP_MEMORY_SCOPE_AGENT);
      return false;
    }
  }
}
__device__ __forceinline__ void in_place(char* ws, u32 line, u32 ks, u32 call, int lane) {
  if (lane == 0) __hip_atomic_store((GLB u32*)(ws + OFF_CNT + line * 128 + ks * 4), call, __ATOMIC_RELAXED, __HIP_MEMORY_SCOPE_SYSTEM);
}

__device__ __forceinline__ void store_raw_unit(const u32x4 (&x)[8], rsrc_t rs, u32 off, int lane) {
  #pragma unroll
  for (int j = 0; j < 8; ++j) ST128(x[j], rs, lane * 16 + j * 1024, off, AUX_SYS);
}

__device__ __forceinline__ void export_half(const u32x4 (&x)[4], rsrc_t rs, u32 par, u32 uid, int hf, LDS char* scr, int lane) {
  const u32 offE = par + EX_INE + uid * IU_ENC;
  u32 s = 0;
  #pragma unroll
  for (int j = 0; j < 4; ++j) {
    *(LDS u32x4*)(scr + j * 1024 + lane * 16) = x[j];
    s = __builtin_amdgcn_sad_u8(x[j][j] & 0x7C7C7C7Cu, 0u, s);
  }
  s = (u32)__builtin_amdgcn_readlane((int)wave_scan_u32(s), 63);
  int e0 = ((int)s - 17121) >> 12;
  e0 = e0 < 0 ? 0 : (e0 > 24 ? 24 : e0);
  const u32 e4 = (u32)e0 * 0x04040404u;
  u32 mk[2] = {0, 0};
  u32 pk[12];
  #pragma unroll
  for (int j = 0; j < 4; ++j) {
    u32 o[4];
    #pragma unroll
    for (int c = 0; c < 4; ++c) {
      const u32 u = (x[j][c] | 0x80808080u) - e4;
      o[c] = bfi(0x1F1F1F1Fu, u, x[j][c]);
      const u32 t = (((u >> 1) ^ 0x40404040u) & 0x70707070u) + 0x70707070u;
      const int d = j * 4 + c;
      mk[d >> 3] = bfi(0x80808080u >> (d & 7), t >> (d & 7), mk[d >> 3]);
    }
    pk[j * 3 + 0] = bfi(0x60606060u, o[3] << 5, o[0]);
    pk[j * 3 + 1] = bfi(0x60606060u, o[3] << 3, o[1]);
    pk[j * 3 + 2] = bfi(0x60606060u, bfi(0x40404040u, o[3] >> 1, o[3] << 1), o[2]);
  }
  #pragma unroll
  for (int j = 0; j < 3; ++j)
    ST128((u32x4{pk[j * 4], pk[j * 4 + 1], pk[j * 4 + 2], pk[j * 4 + 3]}), rs, lane * 16 + j * 1024, offE + hf * 3072, AUX_SYS);
  const u32 mine = __popc(mk[0]) + __popc(mk[1]);
  const u32 inc = wave_scan_u32(mine);
  const u32 total = (u32)__builtin_amdgcn_readlane((int)inc, 63);
  if (total <= IN_CAP) {
    LDS unsigned short* list = (LDS unsigned short*)(scr + 8192);
    u32 slot = 2 + inc - mine;
    #pragma unroll
    for (int q = 0; q < 2; ++q) {
      u32 m = mk[q];
      while (m) {
        const int bp = __builtin_ctz(m); m &= m - 1;
        const int d = q * 8 + 7 - (bp & 7);
        list[slot++] = (unsigned short)((d >> 2) * 1024 + lane * 16 + (d & 3) * 4 + (bp >> 3));
      }
    }
    wave_sync();
    for (u32 j = 0; j < 2; ++j) {
      const u32 pp = *(LDS u32*)(scr + 8192 + j * 256 + lane * 4);
      const u32 p0 = pp & 4095u, p1 = (pp >> 16) & 4095u;
      const u32 b0 = *(LDS unsigned char*)(scr + p0), b1 = *(LDS unsigned char*)(scr + p1);
      u32 sd = (((p0 << 3) | ((128u + (b0 & 127u) - 4u * (u32)e0) >> 5) | (p1 << 19) | (((128u + (b1 & 127u) - 4u * (u32)e0) >> 5) << 16))) + hf * 0x80008000u;
      if (j == 0 && lane == 0) sd = total | ((u32)e0 << 16);
      const u32 at = j * 64 + lane;
      if (at < 32 || total > IN_CAP0)
        __builtin_amdgcn_raw_buffer_store_b32(sd, rs, at < 32 ? hf * 128 + at * 4 : 128 + hf * 384 + at * 4, offE + IU_SIDE, AUX_SYS);
      if (total <= 126) break;
    }
  } else {
    #pragma unroll
    for (int j = 0; j < 4; ++j) ST128(x[j], rs, lane * 16 + j * 1024, par + EX_INR + uid * IU_RAW + hf * 4096, AUX_SYS);
    if (lane == 0) __builtin_amdgcn_raw_buffer_store_b32(0x80000000u, rs, hf * 128, offE + IU_SIDE, AUX_SYS);
  }
}

__device__ __forceinline__ u32 export_short(const u32x4 (&x)[8], rsrc_t rs, u32 par, u32 uid, LDS char* scr, int lane) {
  typedef unsigned short u16x2 __attribute__((ext_vector_type(2)));
  const u32 offE = par + EX_INE + uid * IU_ENC, offR = par + EX_INR + uid * IU_RAW;
  LDS char* var = scr + 8192;
  LDS char* tab = var + QV_ALL;
  u32 s = 0;
  #pragma unroll
  for (int j = 0; j < 8; ++j) s = __builtin_amdgcn_sad_u8(x[j][j & 3] & 0x7C7C7C7Cu, 0u, s);
  #pragma unroll
  for (int j = 0; j < 7; ++j) *(LDS u32x4*)(scr + j * 1024 + lane * 16) = u32x4{0u, 0u, 0u, 0u};
  *(LDS u32x4*)(var + lane * 16) = u32x4{0u, 0u, 0u, 0u};
  *(LDS u32x4*)(var + 1024 + lane * 16) = u32x4{0u, 0u, 0u, 0u};
  {
    u32 sel = 0x0C0C0C0Cu, at = 0;
    #pragma unroll
    for (int y = 0; y < 4; ++y) if ((lane >> y) & 1) { sel = (sel & ~(0xFFu << at)) | ((u32)y << at); at += 8; }
    if (lane < 16) *(LDS u32*)(tab + lane * 4) = sel;
  }
  s = (u32)__builtin_amdgcn_readlane((int)wave_scan_u32(s), 63);
  const int K = max(5, min(31, ((int)s >> 13) + 3));
  const u32 kw = (128u + 4u * (u32)K) * 0x01010101u, tabo = (u32)(uintptr_t)tab;
  auto front = [&](u32 F) { return *(LDS u32*)(uintptr_t)__builtin_amdgcn_udot4(F, 0x20100804u, tabo, false); };
  u32 cs[32], cn[32], car[20], mine = 0;
  wave_sync();
  #pragma unroll
  for (int g = 0; g < 4; ++g) {
    u32 a[8], uu[8];
    #pragma unroll
    for (int k = 0; k < 8; ++k) {
      const int d = g * 8 + k;
      const u32 xv = x[d >> 2][d & 3];
      const u32 u = uu[k] = (kw - (xv & 0x7C7C7C7Cu)) ^ 0x80808080u;
      const u32 esc = (((u >> 1) & 0x78787878u) + 0x78787878u) | ~((u | (u << 1)) << 4);
      const u32 F = (esc >> 7) & 0x01010101u;
      a[k] = bfi(0x0C0C0C0Cu, ~u | __builtin_bit_cast(u32, __builtin_bit_cast(u16x2, F) * (u16x2){12, 12}), xv);
      cs[d] = front(F);
      cn[d] = mine;
      mine = __builtin_amdgcn_udot4(F, 0x01010101u, mine, false);
    }
    car[g * 5 + 0] = bfi(0x70707070u, a[5] << 4, a[0]);
    car[g * 5 + 1] = bfi(0x70707070u, a[6] << 4, a[1]);
    car[g * 5 + 2] = bfi(0x70707070u, a[7] << 4, a[2]);
    car[g * 5 + 3] = bfi(0x40404040u, a[7] << 3, bfi(0x20202020u, a[6] << 2, bfi(0x10101010u, a[5] << 1, a[3])));
    car[g * 5 + 4] = bfi(0x40404040u, a[7] >> 1, bfi(0x20202020u, a[6] >> 2, bfi(0x10101010u, a[5] >> 3, a[4])));
    __builtin_amdgcn_sched_barrier(0);
    #pragma unroll
    for (int k = 0; k < 8; ++k) cs[g * 8 + k] = __builtin_amdgcn_perm(0u, uu[k], cs[g * 8 + k]);
  }
  #pragma unroll
  for (int j = 0; j < 5; ++j) ST128((u32x4{car[j * 4], car[j * 4 + 1], car[j * 4 + 2], car[j * 4 + 3]}), rs, lane * 16 + j * 1024, offE, AUX_SYS);
  const u32 inc = wave_scan_u32(mine);
  const u32 nesc = (u32)__builtin_amdgcn_readlane((int)inc, 63);
  bool fits = nesc <= Q_ESC;
  u32 lines = 1;
  if (fits) {
    const u32 base = inc - mine;
    #pragma unroll
    for (int d = 0; d < 32; ++d) {
      const u32 q = base + cn[d];
      const u64 v = (u64)cs[d] << ((q & 3u) * 8u);
      LDS u32* at = (LDS u32*)(scr + (q & ~3u));
      __hip_atomic_fetch_or(at, (u32)v, __ATOMIC_RELAXED, __HIP_MEMORY_SCOPE_WORKGROUP);
      __hip_atomic_fetch_or(at + 1, (u32)(v >> 32), __ATOMIC_RELAXED, __HIP_MEMORY_SCOPE_WORKGROUP);
    }
    wave_sync();
    const u32 t0 = (QV_SEC + (nesc + 31) / 32 * 8) * 8;
    u32 tbit = t0, npat = 0, nter = 0;
    for (u32 r0 = 0; r0 < nesc && fits; r0 += 2048) {
      const u32x4 e0 = *(LDS u32x4*)(scr + r0 + lane * 32), e1 = *(LDS u32x4*)(scr + r0 + lane * 32 + 16);
      u32 two[2] = {0u, 0u}, zq[8], zl[8], fs[8], t3[8], mk = 0, bits = 0, nt = 0;
      #pragma unroll
      for (int k = 0; k < 8; ++k) {
        const u32 u = k < 4 ? e0[k & 3] : e1[k & 3], sel = (u >> 2) & 0x07070707u;
        const u32 far = ((((u >> 1) & 0x70707070u) + 0x70707070u) >> 7) & 0x01010101u;
        const u32 c2 = __builtin_amdgcn_perm(0x03030201u, 0x03030300u, sel) | far | (far << 1);
        const u32 F = c2 & (c2 >> 1) & 0x01010101u;
        two[k >> 2] |= __builtin_amdgcn_udot4(c2, 0x40100401u, 0u, false) << (8 * (k & 3));
        fs[k] = front(F); t3[k] = ((sel + 0x02020202u) & 0x07070707u) + 0x01010101u;
        nt = __builtin_amdgcn_udot4(F, 0x01010101u, nt, false);
        const u32 lo7 = u & 0x7F7F7F7Fu;
        const u32 out = (F << 7) & (~(lo7 + 0x68686868u) | (lo7 + 0x48484848u) | u);
        mk |= out >> k;
      }
      __builtin_amdgcn_sched_barrier(0);
      #pragma unroll
      for (int k = 0; k < 8; ++k) {
        const u32 cum = __builtin_amdgcn_perm(0u, t3[k], fs[k]) * 0x01010101u;
        zl[k] = (1u << (cum & 31u)) | (1u << ((cum >> 8) & 31u)) | (1u << ((cum >> 16) & 31u)) | (1u << ((cum >> 24) & 31u));
        zq[k] = bits | ((cum >> 29) << 16);
        bits += cum >> 24;
      }
      if (r0 + lane * 32 < nesc) *(LDS u32x2*)(var + QV_SEC + r0 / 4 + lane * 8) = u32x2{two[0], two[1]};
      const u32 np = __popc(mk), inc2 = wave_scan_u32(bits | (np << 16)), tot = (u32)__builtin_amdgcn_readlane((int)inc2, 63);
      nter += (u32)__builtin_amdgcn_readlane((int)wave_scan_u32(nt), 63);
      fits = tbit + (tot & 0xFFFFu) + 96 <= QV_END * 8 && npat + (tot >> 16) <= Q_PAT && nter <= Q_TMAX;
      if (!fits) break;
      const u32 b0 = tbit + (inc2 & 0xFFFFu) - bits;
      if (bits) {
        #pragma unroll
        for (int k = 0; k < 8; ++k) {
          const u32 q = b0 + (zq[k] & 0xFFFFu);
          const u64 v = (((u64)(zq[k] >> 16) << 32) | zl[k]) << (q & 31u);
          LDS u32* at = (LDS u32*)(var + ((q >> 5) << 2));
          __hip_atomic_fetch_or(at, (u32)v, __ATOMIC_RELAXED, __HIP_MEMORY_SCOPE_WORKGROUP);
          __hip_atomic_fetch_or(at + 1, (u32)(v >> 32), __ATOMIC_RELAXED, __HIP_MEMORY_SCOPE_WORKGROUP);
        }
      }
      u32 slot = npat + (inc2 >> 16) - np;
      for (u32 m = mk; m; m &= m - 1) {
        const int bp = __builtin_ctz(m);
        const u32 sn = r0 + lane * 32 + (7 - (bp & 7)) * 4 + (bp >> 3);
        const int v = (int)*(LDS signed char*)(scr + sn) >> 2;
        *(LDS unsigned short*)(var + (slot < Q_PAT0 ? 8 + slot * 2 : QV_END - Q_PAT0 * 2 + slot * 2)) = (unsigned short)((sn << 3) | (u32)(((v - 6) >> 3) + 4));
        ++slot;
      }
      tbit += tot & 0xFFFFu; npat += tot >> 16;
    }
    if (fits) {
      if (lane == 0) *(LDS u32x2*)var = u32x2{nesc | (npat << 13) | ((u32)K << 24), tbit - t0 + 1};
      lines = tbit / 1024 + 1;
      wave_sync();
      if ((u32)lane < lines * 8) ST128(*(LDS u32x4*)(var + lane * 16), rs, lane * 16, offE + QV, AUX_SYS);
      if (lines > 8 && lane < (int)(QV_END - QV0) / 16) ST128(*(LDS u32x4*)(var + QV0 + lane * 16), rs, lane * 16, offE + QV + QV0, AUX_SYS);
      if (npat > Q_PAT0 && lane < (int)(QV_ALL - QV_END) / 16) ST128(*(LDS u32x4*)(var + QV_END + lane * 16), rs, lane * 16, offE + QV + QV_END, AUX_SYS);
    }
  }
  if (!fits) {
    store_raw_unit(x, rs, offR, lane);
    if (lane == 0) __builtin_amdgcn_raw_buffer_store_b32(0x80000000u, rs, 0, offE + QV, AUX_SYS);
  }
  return min(lines, 8u);
}

__device__ __forceinline__ void patch_short(LDS char* scr, u32 ent) {
  const u32 pos = ent >> 3;
  __hip_atomic_fetch_add((LDS u32*)(scr + (pos & ~3u)), ((4u - (ent & 7u)) << 5) << ((pos & 3u) * 8u), __ATOMIC_RELAXED, __HIP_MEMORY_SCOPE_WORKGROUP);
}
__device__ __forceinline__ void patch_in(LDS char* scr, u32 ent) {
  const u32 pos = ent >> 3;
  __hip_atomic_fetch_add((LDS u32*)(scr + (pos & ~3u)), (((ent & 7u) - 4u) << 5) << ((pos & 3u) * 8u), __ATOMIC_RELAXED, __HIP_MEMORY_SCOPE_WORKGROUP);
}

struct Side { u32 h[2], l[2], s, f, end; };
__device__ __forceinline__ Side side_map(u32 n0, u32 n1, u32 ns, u32 nf) {
  Side m;
  m.h[0] = 6; m.l[0] = m.h[0] + (256 + n0 + 7) / 8; m.h[1] = m.l[0] + (n0 + 1) / 2; m.l[1] = m.h[1] + (256 + n1 + 7) / 8;
  m.s = (m.l[1] + (n1 + 1) / 2 + 1) & ~1u; m.f = m.s + ns * 2; m.end = m.f + nf * 6;
  return m;
}

__global__ void __launch_bounds__(NTHR, 2) gemm_rs_kernel(Params p) {
  extern __shared__ __attribute__((aligned(16))) char smem_[];
  LDS char* smem = (LDS char*)smem_;
  const int tid = threadIdx.x, lane = tid & 63, wave = __builtin_amdgcn_readfirstlane(tid >> 6);
  const int b = blockIdx.x, rank = p.rank;
  const int lg = lane >> 4, li = lane & 15;
  const rsrc_t rsL = mk_rsrc(p.ws, p.ws_bytes);
  if (b < 128 || (b >= 192 && b < NGEMM)) __builtin_amdgcn_s_sleep(16);
  const u32 tick = (u32)__builtin_amdgcn_raw_ptr_buffer_atomic_add_i32(1, rsL, lane == 0 ? 0 : (int)0xF0000000u, OFF_CALL + b * 128 + wave * 16, 0);
  char* const peer_ws = p.ws_all[lane & 7];
  u32 call;

  if (b < 128) {
    const rsrc_t rsW = mk_rsrc(p.w, KS * NN);
    #pragma unroll
    for (int i = 0; i < 8; ++i) {
      const int row = (wave * 8 + i) * 8 + (lane >> 3);
      const int sw = ((row >> 1) & 3) | (((row >> 4) & 1) << 2);
      __builtin_amdgcn_raw_ptr_buffer_load_lds(rsW, (LDS void*)(smem + (wave * 8 + i) * 1024), 16, row * NN + (((lane & 7) ^ sw) << 4), b * 128, 0, 0);
    }
    asm volatile("s_waitcnt vmcnt(0)\n\ts_barrier" ::: "memory");
    call = __builtin_amdgcn_readfirstlane(tick) + 1u;
    const int cgl = wave & 1, kb = wave >> 1;
    u32x4 x[8];
    #pragma unroll
    for (int f = 0; f < 4; ++f)
      #pragma unroll
      for (int h = 0; h < 2; ++h)
        #pragma unroll
        for (int t = 0; t < 2; ++t) {
          const int row = kb * 128 + 64 * h + 16 * lg + 8 * t + (li >> 1);
          const int sw = ((row >> 1) & 3) | (((row >> 4) & 1) << 2);
          const i32x2 v = __builtin_amdgcn_ds_read_tr8_b64_v2i32((LDS i32x2*)(smem + row * 128 + (((cgl * 4 + f) ^ sw) << 4) + (li & 1) * 8));
          x[f * 2 + h][t * 2] = v.x; x[f * 2 + h][t * 2 + 1] = v.y;
        }
    const int gl = b >> 5, gr = 4 + ((b >> 3) & 3), c = b & 7;
    const u32 lines = export_short(x, rsL, OFF_PAR + (call & 1) * PAR_BYTES, 256 + (b * 2 + cgl) * 4 + kb, smem + L_SCR + wave * SCR_W, lane);
    vm_drain();
    if ((lane == gl || lane == gr) && lane != rank) {
      const int cb = (lane == gl ? gr & 3 : gl) * 8 + c;
      __hip_atomic_store((GLB u32*)(peer_ws + OFF_FIN + (rank * 576 + 256 + (cb * 2 + cgl) * 4 + kb) * 128), call << 4 | lines, __ATOMIC_RELAXED, __HIP_MEMORY_SCOPE_SYSTEM);
    }
    if (rank == gl || rank == gr) {
      const int cb = (rank == gl ? gr & 3 : gl) * 8 + c;
      store_raw_unit(x, rsL, OFF_WA + ((rank * 4 + kb) * 64 + cb * 2 + cgl) * IU_RAW, lane);
      vm_drain();
      in_place(p.ws, 64 + cb * 2 + cgl, rank * 4 + kb, call, lane);
    }
  } else if (b < 192) {
    const int rg = b - 128, kb = wave & 3, hf = wave >> 2, u = rg * 4 + kb;
    const rsrc_t rsX = mk_rsrc(p.x, MM * KS);
    const u32 both = (u32)(uintptr_t)(smem + L_CTL) + 32;
    const u32 xoff = OFF_XA + ((rank * 4 + kb) * 64 + rg) * IU_RAW;
    const int ow = rg >> 3, r8 = rg & 7;
    const u32 users = ((r8 <= 3 ? 0xFu << (ow & 4) : 0u) | (r8 >= 3 ? 0xFu << (~ow & 4) : 0u) | (1u << ow)) & ~(1u << rank);
    const bool tell = lane < NR && ((users >> lane) & 1);
    if (quick_form(u)) {
      u32x4 x[4];
      #pragma unroll
      for (int j = 0; j < 4; ++j)
        x[j] = __builtin_amdgcn_raw_buffer_load_b128(rsX, ((j >> 1) * 16 + li) * KS + (j & 1) * 64 + lg * 16, (rg * 64 + hf * 32) * KS + kb * 128, 0);
      if (wave == 0) ctl_st(both + (lane & 7) * 4, 0u);
      asm volatile("s_waitcnt lgkmcnt(0)\n\ts_barrier" ::: "memory");
      call = __builtin_amdgcn_readfirstlane(tick) + 1u;
      const u32 parw = OFF_PAR + (call & 1) * PAR_BYTES;
      auto second = [&](int k) -> bool {
        u32 old = 0;
        if (lane == 0) old = __hip_atomic_fetch_add((LDS u32*)(uintptr_t)(both + k * 4), 1u, __ATOMIC_RELAXED, __HIP_MEMORY_SCOPE_WORKGROUP);
        return __builtin_amdgcn_readfirstlane((int)old) == 1;
      };
      if (NRAW && kb == 0) {
        #pragma unroll
        for (int j = 0; j < 4; ++j) ST128(x[j], rsL, lane * 16 + j * 1024, parw + EX_INR + u * IU_RAW + hf * 4096, AUX_SYS);
        vm_drain();
        if (second(2) && tell) __hip_atomic_store((GLB u32*)(peer_ws + OFF_FIN + (rank * 576 + 512 + rg) * 128), call << 4, __ATOMIC_RELAXED, __HIP_MEMORY_SCOPE_SYSTEM);
      }
      export_half(x, rsL, parw, u, hf, smem + L_SCR + wave * SCR_W, lane);
      vm_drain();
      if (second(kb) && tell) __hip_atomic_store((GLB u32*)(peer_ws + OFF_FIN + (rank * 576 + u) * 128), call << 4, __ATOMIC_RELAXED, __HIP_MEMORY_SCOPE_SYSTEM);
      #pragma unroll
      for (int j = 0; j < 4; ++j) ST128(x[j], rsL, lane * 16 + j * 1024, xoff + hf * 4096, AUX_SYS);
      vm_drain();
      if (second(4 + kb)) in_place(p.ws, rg, rank * 4 + kb, call, lane);
    } else if (hf == 0) {
      asm volatile("s_barrier" ::: "memory");
      u32x4 x[8];
      #pragma unroll
      for (int j = 0; j < 8; ++j)
        x[j] = __builtin_amdgcn_raw_buffer_load_b128(rsX, ((j >> 1) * 16 + li) * KS + (j & 1) * 64 + lg * 16, rg * 64 * KS + kb * 128, 0);
      call = __builtin_amdgcn_readfirstlane(tick) + 1u;
      const u32 lines = export_short(x, rsL, OFF_PAR + (call & 1) * PAR_BYTES, u, smem + L_SCR + wave * SCR_W, lane);
      vm_drain();
      if (tell) __hip_atomic_store((GLB u32*)(peer_ws + OFF_FIN + (rank * 576 + u) * 128), call << 4 | lines, __ATOMIC_RELAXED, __HIP_MEMORY_SCOPE_SYSTEM);
      store_raw_unit(x, rsL, xoff, lane);
      vm_drain();
      in_place(p.ws, rg, rank * 4 + kb, call, lane);
    } else {
      asm volatile("s_barrier" ::: "memory");
    }
  }
  call = __builtin_amdgcn_readfirstlane(tick) + 1u;
  const u32 par = OFF_PAR + (call & 1) * PAR_BYTES;
  const float scale = p.sx[0] * p.sw[0];
  const rsrc_t rsO = mk_rsrc(p.out, 512u * NN * 4u);

  if (b >= NGEMM) {
    const int pb = b - NGEMM, link = pb / PULL_WGS, q = pb % PULL_WGS;
    const int src = (rank + 1 + link) & 7;
    const u64 pw = (u64)peer_ws;
    const u32 pw_lo = (u32)__builtin_amdgcn_readlane((int)(u32)pw, src), pw_hi = (u32)__builtin_amdgcn_readlane((int)(u32)(pw >> 32), src);
    const rsrc_t rsP = mk_rsrc((const void*)(((u64)pw_hi << 32) | (u64)pw_lo), p.ws_bytes);
    const u32 ctl = (u32)(uintptr_t)(smem + L_CTL);
    const u32 C_READY = ctl, C_LANDED = ctl + 4, C_CLOSED = ctl + 12, C_REL = ctl + 16, C_ISSUED = ctl + 8, C_Q = ctl + 64;
    const bool same = (src >> 2) == (rank >> 2);
    if (wave == 0) ctl_st(ctl + lane * 4, 0u);
    asm volatile("s_waitcnt lgkmcnt(0)\n\ts_barrier" ::: "memory");
    if (wave == 0) {
      int g = 0, pub = 0;
      #define LANDED(v) do { if ((v) > pub) { pub = (v); ctl_st(C_LANDED, (u32)pub); } } while (0)
      #define PIECEW(W, SZ, VOFF, OFF) do { __builtin_amdgcn_raw_ptr_buffer_load_lds(rsP, (LDS void*)(dst + (OFF)), SZ, VOFF, so + (OFF), 0, AUX_SYS); \
        asm volatile("s_waitcnt vmcnt(%0)" :: "i"((W) - 1) : "memory"); ++g; LANDED(g - ((W) - 1)); } while (0)
      #define PIECE(SZ, VOFF, OFF) PIECEW(WIN, SZ, VOFF, OFF)
      for (u32 n = 0;; ++n) {
        if (ctl_ld(C_READY) <= n) {
          vm_drain(); LANDED(g);
          bool fin = false;
          while (ctl_ld(C_READY) <= n) { if (ctl_ld(C_CLOSED)) { fin = ctl_ld(C_READY) <= n; break; } __builtin_amdgcn_s_sleep(1); }
          if (fin) break;
        }
        const u32 ent = ctl_ld(C_Q + (n & 15) * 4);
        ctl_st(C_ISSUED, n + 1);
        if (ent >> 31) continue;
        const u32 need = n >= NSLOT ? n - NSLOT + 1 : 0, ra = C_REL + (n % NSLOT) * 4;
        if (ctl_ld(ra) < need) {
          vm_drain(); LANDED(g);
          while (ctl_ld(ra) < need) __builtin_amdgcn_s_sleep(1);
        }
        const int i = ent & 1023;
        LDS char* dst = smem + (n % NSLOT) * SLOT;
        if (ent & 1024) {
          const u32 so = par + EX_INR + i * IU_RAW;
          #pragma unroll
          for (int j = 0; j < 8; ++j) PIECE(16, lane * 16, j * 1024);
        } else if (i < 512 && quick_form(i)) {
          const u32 so = par + EX_INE + (i < 256 ? i : 256 + group_of(rank, (i >> 6) & 3) * 64 + (i & 63)) * IU_ENC;
          PIECE(4, lane * 4, IU_SIDE);
          #pragma unroll
          for (int j = 0; j < 6; ++j) PIECE(16, lane * 16, j * 1024);
        } else if (i < 512) {
          const u32 so = par + EX_INE + (i < 256 ? i : 256 + group_of(rank, (i >> 6) & 3) * 64 + (i & 63)) * IU_ENC;
          if ((u32)lane < ((ent >> 24) & 15u) * 8) __builtin_amdgcn_raw_ptr_buffer_load_lds(rsP, (LDS void*)(dst + QV), 16, lane * 16, so + QV, 0, AUX_SYS);
          asm volatile("s_waitcnt vmcnt(%0)" :: "i"(WIN - 1) : "memory"); ++g; LANDED(g - (WIN - 1));
          #pragma unroll
          for (int j = 0; j < 5; ++j) PIECE(16, lane * 16, j * 1024);
        } else {
          const u32 so = par + EX_OUE + (rank * 320 + i - 512) * OU_ENC;
          if ((u32)lane < 32 + ((ent >> 24) & 15u) * 8)
            __builtin_amdgcn_raw_ptr_buffer_load_lds(rsP, (LDS void*)(dst + 11264), 16, lane * 16, so + 11264, 0, AUX_SYS);
          asm volatile("s_waitcnt vmcnt(%0)" :: "i"(WIN_OUT - 1) : "memory"); ++g; LANDED(g - (WIN_OUT - 1));
          #pragma unroll
          for (int j = 0; j < 11; ++j) PIECEW(WIN_OUT, 16, lane * 16, j * 1024);
        }
      }
      vm_drain(); LANDED(g);
      #undef PIECE
      #undef PIECEW
      #undef LANDED
      return;
    }
    if (wave == 1) {
      const u32 tic0 = OFF_TIC + (call & 1) * 24 * 128;
      const int only0 = lane == 0 ? 0 : (int)0xF0000000u;
      if (q == 0 && (lane == link || (link == 0 && lane >= 8 && lane < 24)))
        __hip_atomic_store((GLB u32*)(p.ws + OFF_TIC + ((~call & 1) * 24 + lane) * 128), 0u, __ATOMIC_RELAXED, __HIP_MEMORY_SCOPE_AGENT);
      constexpr u32 NPAIR = 208 + NRAW / 2, NFIX = 192;
      u32 n = 0, endp = 0; bool ok = true;
      const u32 gtic = tic0 + (8 + group_of(src, lane & 3)) * 128;
      auto fullest = [&](u32 v) -> int {
        v = v >= 16u || (!same && (lane & 3) == (rank & 3)) ? 0xFFFFu : (v << 4) | ((((lane - q) & 3) << 2) | (lane & 3));
        v = min(v, (u32)__builtin_amdgcn_mov_dpp((int)v, 0xB1, 0xf, 0xf, true));
        v = min(v, (u32)__builtin_amdgcn_mov_dpp((int)v, 0x4E, 0xf, 0xf, true));
        v = (u32)__builtin_amdgcn_readfirstlane((int)v);
        return v == 0xFFFFu ? -2 : (int)(v & 3u);
      };
      auto draw = [&](int g) { return __builtin_amdgcn_raw_ptr_buffer_atomic_add_i32(1, rsL, only0, tic0 + (g < 0 ? link : 8 + group_of(src, g)) * 128, 0); };
      int tnext = q - PULL_WGS;
      for (;;) {
        const u32 t = (u32)(__builtin_amdgcn_readfirstlane(tnext) + PULL_WGS);
        if (t >= NPAIR + NFIX) break;
        while (n >= ctl_ld(C_ISSUED) + (t < NPAIR ? LOOK : LOOK_OUT)) __builtin_amdgcn_s_sleep(1);
        const bool last = t + PULL_WGS >= NPAIR + NFIX;
        if (!last) tnext = draw(-1);
        int i;
        if (t < NPAIR) {
          const int e = t < NRAW ? t : 2 * t - NRAW + (lane & 1);
          int u = -1, sg = 0, kb = 0;
          if (e < NRAW) u = e;
          else if (e < NRAW + 14) { const int z = e - NRAW; kb = 1; u = z < 6 ? (z >> 1) * 4 + (z & 1) : 12 + ((z - 6) >> 1) * 5 + 1 + (z & 1); }
          else if (e < 46) u = e - 14;
          else if (e < 64) { const int z = e - 46; kb = 1; u = z < 6 ? (z >> 1) * 4 + 2 + (z & 1) : 12 + ((z - 6) / 3) * 5 + ((z - 6) % 3 ? 2 + (z - 6) % 3 : 0); }
          else if (e < 64 + X2FIRST) { kb = 2; u = e - 64; }
          else if (e < 88 + X2FIRST) sg = (e - 64 - X2FIRST) >> 3;
          else if (e < 120) { kb = 2; u = e - 88; }
          else if (e < 144) sg = (e - 96) >> 3;
          else if (e < 176) { kb = 3; u = e - 144; }
          else sg = (e - 128) >> 3;
          int c = 0;
          if (u < 0) {
            if (sg < 6) { c = (0x210100 >> (sg * 4)) & 7; kb = (0x012010 >> (sg * 4)) & 3; }
            else if (sg < 26) { c = ((sg - 6) >> 2) + ((sg - 6) & 3); kb = 3 - ((sg - 6) & 3); }
            else { c = (0x776765 >> ((sg - 26) * 4)) & 7; kb = (0x323123 >> ((sg - 26) * 4)) & 3; }
          }
          const int m = c * 8 + (e & 7);
          const int v = u - 12;
          if (e >= 384) i = (8 * rank + (e & 7)) * 4 + ((e - 384) >> 3);
          else if (u < 0) i = 256 + ((((m >> 1) & 3) * 8 + (m >> 3)) * 2 + (m & 1)) * 4 + kb;
          else if (u < 12) i = (8 * ((rank & 4) | ((rank + 1 + (u >> 2)) & 3)) + (u & 3)) * 4 + kb;
          else i = (8 * ((~rank & 4) | (v / 5)) + 3 + v % 5) * 4 + kb;
        } else {
          const int o = t - NPAIR, cm = same ? o / 3 : o >> 2, k = same ? o - cm * 3 : 1 + (o & 3);
          const int c = same ? cm >> 3 : cm / 6, j3 = (cm - c * 6) >> 1, j = same ? (cm >> 1) & 3 : j3 + (j3 >= (rank & 3));
          i = 512 + ((j * 8 + c) * 2 + (cm & 1)) * 5 + k;
        }
        const bool unp = t < NRAW, two = !unp && t < NPAIR;
        const u32 fo = i >= 512 ? OFF_FOUT + (src * 320 + i - 512) * 128 : OFF_FIN + (src * 576 + (unp ? 512 + (i >> 2) : i)) * 128;
        u32 got = 0;
        ok = ok && wait_ge(p.ws, fo, call << 4, call, got);
        const u32 lines = got >> 4 == call ? got & 15u : 8u;
        const u32 k = two ? (lane & 1) : 0, per = !ok ? 0 : t >= NPAIR ? 12 : unp ? 8 : (u32)__builtin_amdgcn_readfirstlane(quick_form(i) ? 7 : 6);
        ctl_st(C_Q + ((n + k) & 15) * 4, (u32)i | (unp ? 1024u : 0u) | ((endp + per * (k + 1)) << 11) | (lines << 24) | (ok ? 0u : 0x80000000u));
        n += two ? 2 : 1; endp += two ? 2 * per : per;
        asm volatile("s_waitcnt lgkmcnt(0)" ::: "memory");
        ctl_st(C_READY, n);
        if (last) break;
      }
      const u32 fo3 = OFF_FOUT + (src * 320 + lane * 5 + (same ? 3 : 0)) * 128;
      const u32 flags = ld_flag(p.ws, fo3);
      u32 seen = ld_flag(p.ws, gtic);
      for (;;) {
        while (n >= ctl_ld(C_ISSUED) + LOOK_END) __builtin_amdgcn_s_sleep(1);
        const int fj = fullest(seen);
        if (fj < 0) break;
        const int tk = draw(fj);
        seen = ld_flag(p.ws, gtic);
        const u32 t = (u32)__builtin_amdgcn_readfirstlane(tk);
        if (t >= 16u) continue;
        const int cg = fj * 16 + (int)t;
        u32 got = (u32)__builtin_amdgcn_readlane((int)flags, cg);
        if (ok && got < call << 4) ok = wait_ge(p.ws, OFF_FOUT + (src * 320 + cg * 5 + (same ? 3 : 0)) * 128, call << 4, call, got);
        if (ok) endp += 12;
        ctl_st(C_Q + (n & 15) * 4, (u32)(512 + cg * 5 + (same ? 3 : 0)) | (endp << 11) | ((got >> 4 == call ? got & 15u : 8u) << 24) | (ok ? 0u : 0x80000000u));
        ++n;
        asm volatile("s_waitcnt lgkmcnt(0)" ::: "memory");
        ctl_st(C_READY, n);
      }
      asm volatile("s_waitcnt lgkmcnt(0)" ::: "memory");
      ctl_st(C_CLOSED, 1u);
      return;
    }
    LDS char* scr = smem + L_PSCR + (wave - 2) * 8 * KiB;
    for (u32 n = wave - 2;; n += NCONS) {
      bool fin = false;
      while (ctl_ld(C_READY) <= n) { if (ctl_ld(C_CLOSED)) { fin = ctl_ld(C_READY) <= n; break; } __builtin_amdgcn_s_sleep(1); }
      if (fin) break;
      const u32 ent = ctl_ld(C_Q + (n & 15) * 4);
      const bool ok = !(ent >> 31);
      const int i = ent & 1023;
      const u32 endp = (ent >> 11) & 0x1FFFu;
      auto landed = [&](u32 upto) { while (ctl_ld(C_LANDED) < upto) __builtin_amdgcn_s_sleep(1); };
      LDS char* sl = smem + (n % NSLOT) * SLOT;
      if (i < 512) {
        if (!ok) continue;
        const bool brief = !(ent & 1024) && !quick_form(i);
        landed(brief ? endp - 5 : endp);
        const int kb = i & 3, rg = (i >> 2) & 63; const bool isx = i < 256;
        const u32 uid = isx ? i : 256 + group_of(rank, (i >> 6) & 3) * 64 + (i & 63);
        const u32 dst = (isx ? OFF_XA : OFF_WA) + ((src * 4 + kb) * 64 + rg) * IU_RAW;
        const u32 rel = C_REL + (n % NSLOT) * 4, offE = par + EX_INE + uid * IU_ENC;
        u32x4 x[8];
        bool raw = false;
        if (ent & 1024) {
          #pragma unroll
          for (int j = 0; j < 8; ++j) x[j] = *(LDS u32x4*)(sl + j * 1024 + lane * 16);
          wave_sync();
          ctl_st(rel, n + 1);
        } else if (quick_form(i)) {
          u32x4 c[6];
          #pragma unroll
          for (int j = 0; j < 6; ++j) c[j] = *(LDS u32x4*)(sl + j * 1024 + lane * 16);
          const u32 sd = *(LDS u32*)(sl + IU_SIDE + lane * 4);
          wave_sync();
          ctl_st(rel, n + 1);
          const u32 hd[2] = {(u32)__builtin_amdgcn_readlane((int)sd, 0), (u32)__builtin_amdgcn_readlane((int)sd, 32)};
          #pragma unroll
          for (int j = 0; j < 8; ++j) {
            const u32 e4 = ((hd[j >> 2] >> 16) & 31u) * 0x04040404u;
            const u32 p0 = c[(j * 3) >> 2][(j * 3) & 3], p1 = c[(j * 3 + 1) >> 2][(j * 3 + 1) & 3], p2 = c[(j * 3 + 2) >> 2][(j * 3 + 2) & 3];
            const u32 o3 = bfi(0x80808080u, p2 << 1, bfi(0x10101010u, p2 >> 1, bfi(0x0C0C0C0Cu, p1 >> 3, p0 >> 5)));
            const u32x4 y = {(p0 & 0x9F9F9F9Fu) + e4, (p1 & 0x9F9F9F9Fu) + e4, (p2 & 0x9F9F9F9Fu) + e4, (o3 & 0x9F9F9F9Fu) + e4};
            *(LDS u32x4*)(scr + j * 1024 + lane * 16) = y;
          }
          wave_sync();
          const u32 cnt = (lane < 32 ? hd[0] : hd[1]) & 0xFFFFu, id0 = 2u * (lane & 31) - 2u;
          if (lane & 31) {
            if (id0 < cnt) patch_in(scr, sd & 0xFFFFu);
            if (id0 + 1 < cnt) patch_in(scr, sd >> 16);
          }
          if (max(hd[0] & 0xFFFFu, hd[1] & 0xFFFFu) > IN_CAP0) {
            for (int j = 0; j < 3; ++j) {
              const u32 s2 = __builtin_amdgcn_raw_buffer_load_b32(rsP, 256 + (lane >> 5) * 384 + (j * 32 + (lane & 31)) * 4, offE + IU_SIDE, AUX_SYS);
              const u32 id = IN_CAP0 + j * 64 + 2 * (lane & 31);
              if (id < cnt) patch_in(scr, s2 & 0xFFFFu);
              if (id + 1 < cnt) patch_in(scr, s2 >> 16);
            }
          }
          wave_sync();
          #pragma unroll
          for (int j = 0; j < 8; ++j) x[j] = *(LDS u32x4*)(scr + j * 1024 + lane * 16);
          #pragma unroll
          for (int h = 0; h < 2; ++h)
            if (hd[h] >> 31) {
              #pragma unroll
              for (int j = h * 4; j < h * 4 + 4; ++j) x[j] = __builtin_amdgcn_raw_buffer_load_b128(rsP, lane * 16 + j * 1024, par + EX_INR + uid * IU_RAW, AUX_SYS);
            }
        } else {
          LDS char* var = sl + QV;
          const u32 hdr = (u32)__builtin_amdgcn_readfirstlane((int)*(LDS u32*)var);
          const u32 pe = *(LDS unsigned short*)(var + 8 + (lane & 7) * 2);
          const u32 tlen = (u32)__builtin_amdgcn_readfirstlane((int)*(LDS u32*)(var + 4));
          raw = hdr >> 31;
          if (!raw) {
            const u32 nesc = hdr & 0x1FFFu, npat = (hdr >> 13) & 0x1FFu, bias = (4u * ((hdr >> 24) & 31u) - 136u) * 0x01010101u;
            const u32 t0 = QV_SEC + (nesc + 31) / 32 * 8;
            if (t0 * 8 + tlen > QV0 * 8 && lane < (int)(QV_END - QV0) / 16)
              *(LDS u32x4*)(var + QV0 + lane * 16) = __builtin_amdgcn_raw_buffer_load_b128(rsP, lane * 16, offE + QV + QV0, AUX_SYS);
            wave_sync();
            u32 tn = (u32)(uintptr_t)scr + Q_TER;
            for (u32 r0 = 0; r0 < tlen; r0 += 1024) {
              const u32 bit0 = r0 + lane * 16;
              const u32 before = *(LDS unsigned short*)(var + t0 + bit0 / 8 - 2);
              u32 m = bit0 < tlen ? *(LDS unsigned short*)(var + t0 + bit0 / 8) : 0u;
              int prev = 15 - (int)__clz(before);
              if (bit0 == 0) { m &= ~1u; prev = 0; }
              const u32 mine = __popc(m), inc = wave_scan_u32(mine);
              LDS unsigned char* at = (LDS unsigned char*)(uintptr_t)(tn + inc - mine);
              for (; m; m &= m - 1) { const int pl = __builtin_ctz(m); *at++ = (unsigned char)((pl - prev) * 4 - 4); prev = pl; }
              tn += (u32)__builtin_amdgcn_readlane((int)inc, 63);
            }
            tn = (u32)(uintptr_t)scr + Q_TER;
            wave_sync();
            for (u32 r0 = 0; r0 < nesc; r0 += 2048) {
              const u32x2 two = *(LDS u32x2*)(var + QV_SEC + r0 / 4 + lane * 8);
              u32 c2[8], e2[8], zo[8], bits = 0;
              #pragma unroll
              for (int k = 0; k < 8; ++k) {
                const u32 z = two[k >> 2] >> (8 * (k & 3)), z2 = bfi(0x0000000Fu, z, z << 12);
                c2[k] = bfi(0x00030003u, z2, z2 << 6) & 0x03030303u;
                e2[k] = (c2[k] << 2) & (c2[k] << 1) & 0x04040404u;
                zo[k] = bits;
                bits = __builtin_amdgcn_udot4(e2[k], 0x01010101u, bits, false);
              }
              const u32 inc = wave_scan_u32(bits >> 2), b4 = (tn + inc) * 4 - bits;
              tn += (u32)__builtin_amdgcn_readlane((int)inc, 63);
              u32 g0[8], g1[8];
              #pragma unroll
              for (int k = 0; k < 8; ++k) {
                zo[k] = (zo[k] + b4) >> 2;
                const LDS u32* at = (const LDS u32*)(uintptr_t)(zo[k] & ~3u);
                g0[k] = at[0]; g1[k] = at[1];
              }
              __builtin_amdgcn_sched_barrier(0);
              #pragma unroll
              for (int k = 0; k < 8; ++k) {
                const u32 nx = __builtin_amdgcn_alignbyte(g1[k], g0[k], zo[k]);
                zo[k] = __builtin_amdgcn_perm(0u, 0x64686C7Cu, c2[k]) - __builtin_amdgcn_perm(nx, 0u, __umul24(e2[k], 0x404040u) + e2[k]);
              }
              *(LDS u32x4*)(scr + r0 + lane * 32) = u32x4{zo[0], zo[1], zo[2], zo[3]};
              *(LDS u32x4*)(scr + r0 + lane * 32 + 16) = u32x4{zo[4], zo[5], zo[6], zo[7]};
            }
            wave_sync();
            if ((u32)lane < npat && lane < (int)Q_PAT0) patch_short(scr, pe);
            if (npat > Q_PAT0) {
              for (int j = 0; j < 2; ++j) {
                const u32 s2 = __builtin_amdgcn_raw_buffer_load_b32(rsP, lane * 4 + j * 256, offE + QV + QV_END, AUX_SYS);
                const u32 id = Q_PAT0 + j * 128 + 2 * lane;
                if (id < npat) patch_short(scr, s2 & 0xFFFFu);
                if (id + 1 < npat) patch_short(scr, s2 >> 16);
              }
            }
            landed(endp);
            u32x4 c[5];
            #pragma unroll
            for (int j = 0; j < 5; ++j) c[j] = *(LDS u32x4*)(sl + j * 1024 + lane * 16);
            wave_sync();
            ctl_st(rel, n + 1);
            u32 a[32], es[32], qo[32], cnt4 = 0;
            #pragma unroll
            for (int g = 0; g < 4; ++g) {
              u32 k5[5];
              #pragma unroll
              for (int k = 0; k < 5; ++k) { k5[k] = c[(g * 5 + k) >> 2][(g * 5 + k) & 3]; a[g * 8 + k] = k5[k]; }
              a[g * 8 + 5] = bfi(0x80808080u, k5[4] << 3, bfi(0x08080808u, k5[3] >> 1, k5[0] >> 4));
              a[g * 8 + 6] = bfi(0x80808080u, k5[4] << 2, bfi(0x08080808u, k5[3] >> 2, k5[1] >> 4));
              a[g * 8 + 7] = bfi(0x80808080u, k5[4] << 1, bfi(0x08080808u, k5[3] >> 3, k5[2] >> 4));
            }
            #pragma unroll
            for (int d = 0; d < 32; ++d) {
              es[d] = a[d] & (a[d] >> 1) & 0x04040404u;
              qo[d] = cnt4;
              cnt4 = __builtin_amdgcn_udot4(es[d], 0x01010101u, cnt4, false);
            }
            const u32 base4 = ((u32)(uintptr_t)scr + wave_scan_u32(cnt4 >> 2)) * 4 - cnt4;
            wave_sync();
            #pragma unroll
            for (int h = 0; h < 2; ++h) {
              u32 g0[16], g1[16];
              #pragma unroll
              for (int k = 0; k < 16; ++k) {
                qo[h * 16 + k] = (base4 + qo[h * 16 + k]) >> 2;
                const LDS u32* at = (const LDS u32*)(uintptr_t)(qo[h * 16 + k] & ~3u);
                g0[k] = at[0]; g1[k] = at[1];
              }
              __builtin_amdgcn_sched_barrier(0);
              #pragma unroll
              for (int k = 0; k < 16; ++k) {
                const int d = h * 16 + k;
                const u32 nx = __builtin_amdgcn_alignbyte(g1[k], g0[k], qo[d]);
                const u32 sel = __umul24(es[d], 0x404040u) + es[d];
                x[d >> 2][d & 3] = (a[d] & 0x8F8F8F8Fu) + __builtin_amdgcn_perm(nx, 0x7C7C7C7Cu, sel) + bias;
              }
              __builtin_amdgcn_sched_barrier(0);
            }
          } else {
            landed(endp);
            ctl_st(rel, n + 1);
          }
        }
        if (raw) {
          #pragma unroll
          for (int j = 0; j < 8; ++j) x[j] = __builtin_amdgcn_raw_buffer_load_b128(rsP, lane * 16 + j * 1024, par + EX_INR + uid * IU_RAW, AUX_SYS);
        }
        store_raw_unit(x, rsL, dst, lane);
        vm_drain();
        in_place(p.ws, (isx ? 0 : 64) + rg, src * 4 + kb, call, lane);
        continue;
      }
      const int t = i - 512, cg = t / 5, rg = t - cg * 5 + (same ? 0 : 3);
      const u32 obase = (rg * 64 * NN + group_of(src, cg >> 4) * 1024 + (cg & 15) * 64) * 4;
      const u32 olane = (li * NN + lg * 4) * 4;
      if (!ok) {
        const u32x4 nan4 = {0x7FC00000u, 0x7FC00000u, 0x7FC00000u, 0x7FC00000u};
        #pragma unroll
        for (int e = 0; e < 16; ++e) ST128(nan4, rsO, olane + (e & 3) * 64, obase + (e >> 2) * 16 * NN * 4, AUX_SYS);
        continue;
      }
      #pragma unroll
      for (int j = 0; j < 4; ++j) *(LDS u32x4*)(scr + j * 1024 + lane * 16) = u32x4{0u, 0u, 0u, 0u};
      landed(endp - 11);
      LDS char* side = sl + OU_SIDE;
      const u32 hdr = (u32)__builtin_amdgcn_readfirstlane((int)*(LDS u32*)side);
      const u32 hd2 = (u32)__builtin_amdgcn_readfirstlane((int)*(LDS unsigned short*)(side + 4));
      const u32 tl[2] = {*(LDS u32*)(sl + 11264 + lane * 4), *(LDS u32*)(sl + 11520 + lane * 4)};
      const u32 uid = rank * 320 + t;
      if (!(hdr >> 31)) {
        const u32 ns = hd2 & 0xFFu, nf = hd2 >> 8; const int g = (int)((hdr >> 20) & 0xFFu);
        const Side sm = side_map(hdr & 1023u, (hdr >> 10) & 1023u, ns, nf);
        const u32 se = *(LDS unsigned short*)(side + sm.s + lane * 2);
        const u32x2 fe = {*(LDS unsigned short*)(side + sm.f + (lane & 31) * 6),
                          *(LDS unsigned short*)(side + sm.f + (lane & 31) * 6 + 2) | ((u32)*(LDS unsigned short*)(side + sm.f + (lane & 31) * 6 + 4) << 16)};
        if ((u32)lane < ns)
          __hip_atomic_fetch_add((LDS u32*)(scr + ((se >> 5) & 63u) * 64 + (((se >> 11) & 15u) ^ 7u) * 4),
                                 ((se & 7u) << 5) << (se & 24u), __ATOMIC_RELAXED, __HIP_MEMORY_SCOPE_WORKGROUP);
        u32 done = 0;
        for (u32 c = lane; c - lane < max(sm.l[0] - sm.h[0], sm.l[1] - sm.h[1]); c += 64) {
          u32 m[2], k[2], nib[2];
          #pragma unroll
          for (int z = 0; z < 2; ++z) m[z] = c < sm.l[z] - sm.h[z] ? *(LDS unsigned char*)(side + sm.h[z] + c) : 0u;
          const u32 mine = __popc(m[0]) | (__popc(m[1]) << 16), inc = wave_scan_u32(mine);
          k[0] = (done + inc - mine) & 0xFFFFu; k[1] = (done + inc - mine) >> 16;
          #pragma unroll
          for (int z = 0; z < 2; ++z) {
            const u32 nb = ((u32)(uintptr_t)side + sm.l[z]) * 8 + k[z] * 4;
            const LDS u32* at = (const LDS u32*)(uintptr_t)((nb >> 3) & ~3u);
            nib[z] = __builtin_amdgcn_alignbit(at[1], at[0], nb);
          }
          #pragma unroll
          for (int z = 0; z < 2; ++z)
            for (u32 d = c * 8 - k[z]; m[z]; m[z] &= m[z] - 1, --d, nib[z] >>= 4) {
              const u32 hi = (d + (u32)__builtin_ctz(m[z])) & 255u;
              __hip_atomic_fetch_add((LDS u32*)(scr + (hi >> 1) * 32 + (nib[z] & 7u) * 4), (z ? 0x80u : 0x02u) << ((nib[z] & 8u) | (hi & 1u) << 4),
                                     __ATOMIC_RELAXED, __HIP_MEMORY_SCOPE_WORKGROUP);
            }
          done += (u32)__builtin_amdgcn_readlane((int)inc, 63);
        }
        const u32 fgq = (u32)lane < nf ? fe.x >> 8 : 16u;
        u32 th[2] = {0u, 0u};
        const float down = as_f((u32)(127 - 7 - g) << 23);
        u32 seen = ctl_ld(C_LANDED);
        u32x4 cq[11], aq[4];
        if (seen < endp - 8) { landed(endp - 8); }
        #pragma unroll
        for (int j = 0; j < 3; ++j) cq[j] = *(LDS u32x4*)(sl + j * 1024 + lane * 16);
        #pragma unroll
        for (int j = 0; j < 4; ++j) aq[j] = *(LDS u32x4*)(scr + lane * 64 + (j ^ 1) * 16);
        #pragma unroll
        for (int mi = 0; mi < 4; ++mi) {
          if (mi < 3) {
            const u32 need = mi < 2 ? endp - 5 + 3 * mi : endp;
            if (seen < need) { landed(need); seen = ctl_ld(C_LANDED); }
            #pragma unroll
            for (int j = mi * 3 + 3; j < (mi < 2 ? mi * 3 + 6 : 11); ++j) cq[j] = *(LDS u32x4*)(sl + j * 1024 + lane * 16);
            __builtin_amdgcn_sched_barrier(0);
          } else { wave_sync(); ctl_st(C_REL + (n % NSLOT) * 4, n + 1); }
          const u32x4* c = cq + mi * 3;
          const u32x4 adj = aq[mi];
          #pragma unroll
          for (int ni = 0; ni < 4; ++ni) {
            const int k = ni * 3, gq = mi * 4 + ni, be = gq % 7;
            const u32 l0 = gq < 15 ? c[k >> 2][k & 3] : tl[0], l1 = gq < 15 ? c[(k + 1) >> 2][(k + 1) & 3] : tl[1];
            const u32 tp = gq < 14 ? c[(k + 2) >> 2][(k + 2) & 3] : th[gq & 1];
            if (gq < 14) th[gq / 7] = be < 6 ? bfi(0x01010101u << be, tp >> (6 - be), th[gq / 7]) : bfi(0x80808080u, tp << 1, th[gq / 7]);
            const u32 top = bfi(0x40404040u, (tp >> 1) ^ (adj[3 - ni] << 5), tp), low = adj[3 - ni] & 0xE0E0E0E0u;
            const u32 q01 = __builtin_amdgcn_perm(top, low, 0x05010400u), q23 = __builtin_amdgcn_perm(top, low, 0x07030602u);
            float y[4] = {(float)(int)__builtin_amdgcn_perm(q01, l0, 0x05010004u), (float)(int)__builtin_amdgcn_perm(q01, l0, 0x07030206u),
                          (float)(int)__builtin_amdgcn_perm(q23, l1, 0x05010004u), (float)(int)__builtin_amdgcn_perm(q23, l1, 0x07030206u)};
            #pragma unroll
            for (int r = 0; r < 4; ++r) y[r] = y[r] * down * scale;
            if (nf)
              for (u64 em = __ballot(fgq == (u32)gq); em; em &= em - 1) {
                const int from = __builtin_ctzll(em);
                const u32 rel = ((u32)__builtin_amdgcn_readlane((int)fe.x, from) & 255u) - (u32)lane * 4u;
                const float vv = as_f((u32)__builtin_amdgcn_readlane((int)fe.y, from)) * scale;
                #pragma unroll
                for (int r = 0; r < 4; ++r) y[r] = rel == (u32)r ? vv : y[r];
              }
            ST128((u32x4{as_u(y[0]), as_u(y[1]), as_u(y[2]), as_u(y[3])}), rsO, olane + ni * 64, obase + mi * 16 * NN * 4, AUX_SYS);
          }
        }
      } else {
        landed(endp);
        ctl_st(C_REL + (n % NSLOT) * 4, n + 1);
        #pragma unroll
        for (int mi = 0; mi < 4; ++mi) {
          u32x4 r4[4];
          #pragma unroll
          for (int ni = 0; ni < 4; ++ni) r4[ni] = __builtin_amdgcn_raw_buffer_load_b128(rsP, lane * 16 + ni * 1024, par + EX_OUR + uid * OU_RAW + mi * 4096, AUX_SYS);
          #pragma unroll
          for (int ni = 0; ni < 4; ++ni) {
            const f32x4 y = __builtin_bit_cast(f32x4, r4[ni]) * scale;
            ST128(__builtin_bit_cast(u32x4, y), rsO, olane + ni * 64, obase + mi * 16 * NN * 4, AUX_SYS);
          }
        }
      }
    }
    return;
  }

  constexpr int SLOTS = NGEMM / 8;
  const int xc = b & 7, slot = b >> 3, wm = wave >> 1, wn = wave & 1;
  for (int ti = slot; ti < 36; ti += SLOTS) {
    const int j = xc >> 1, ft = ti * 2 + (xc & 1);
    const int c = ti < 24 ? ft / 6 : 4 * (xc & 1) + (ti < 28 ? ti - 24 : (ti - 28) >> 1), cb = j * 8 + c;
    auto row_group = [&](int k) -> int {
      if (ti >= 28) return 8 * rank + 4 * (ti & 1) + k;
      if (ti >= 24) return 8 * ((~rank & 4) | ((j + 1 + min(k, 2)) & 3)) + 3;
      const int t = ti % 3;
      return 8 * (xc & 1 ? (~rank & 4) | ((j + 1 + t) & 3) : (rank & 4) | ((rank + 1 + t) & 3)) + 4 * (xc & 1) + k;
    };
    const int rgw = row_group(wm);
    const u32 deadw = (u32)(uintptr_t)(smem + L_CTL);
    u32 todo = 0xFFFFFFFFu, avail = 0;
    const u32 pick = deadw + 4;
    auto choose = [&](u32 word) {
      if (!avail) {
        const u32 off = OFF_CNT + (lane < 32 ? row_group(lane >> 3) : 64 + cb * 2 + ((lane >> 3) & 1)) * 128 + (lane & 7) * 16;
        const u64 t0 = __builtin_amdgcn_s_memrealtime();
        GLB u32* tmo = (GLB u32*)(p.ws + OFF_TMO);
        for (u32 it = 1;; ++it) {
          asm volatile("" ::: "memory");
          const u32x4 w = __builtin_amdgcn_raw_buffer_load_b128(rsL, off, 0, AUX_SYS);
          #pragma unroll
          for (int k = 0; k < 4; ++k) {
            u64 m = __ballot(w[k] >= call);
            m &= m >> 32; m &= m >> 16; m &= m >> 8;
            avail |= ((u32)m & 255u) << (8 * k);
          }
          avail &= todo;
          if (avail) break;
          if ((it & 15u) == 0u && (__hip_atomic_load(tmo, __ATOMIC_RELAXED, __HIP_MEMORY_SCOPE_AGENT) == call ||
                                   __builtin_amdgcn_s_memrealtime() - t0 > DEADLINE_TICKS)) {
            __hip_atomic_store(tmo, call, __ATOMIC_RELAXED, __HIP_MEMORY_SCOPE_AGENT);
            ctl_st(deadw, 1u);
            avail = todo;
            break;
          }
          __builtin_amdgcn_s_sleep(2);
        }
        asm volatile("" ::: "memory");
      }
      u32 bit = (u32)__builtin_ctz(avail);
      avail &= ~(1u << bit); todo &= ~(1u << bit);
      ctl_st(word, (bit & 7u) * 4 + (bit >> 3));
    };
    const u32 xo0 = OFF_XA + rgw * IU_RAW + wn * 4096, wo0 = OFF_WA + cb * 2 * IU_RAW + wave * 2048;
    auto dma = [&](int k, int ks, u32 st) {
      const u32 xo = xo0 + ks * (64 * IU_RAW), wo = wo0 + ks * (64 * IU_RAW);
      LDS char* dx = smem + st + wave * 4096;
      LDS char* dw = smem + st + 32 * KiB + wave * 2048;
      if (k == 0) __builtin_amdgcn_raw_ptr_buffer_load_lds(rsL, (LDS void*)dx, 16, lane * 16, xo, 0, AUX_SYS);
      if (k == 1) __builtin_amdgcn_raw_ptr_buffer_load_lds(rsL, (LDS void*)dx, 16, lane * 16, xo, 1024, AUX_SYS);
      if (k == 2) __builtin_amdgcn_raw_ptr_buffer_load_lds(rsL, (LDS void*)dx, 16, lane * 16, xo, 2048, AUX_SYS);
      if (k == 3) __builtin_amdgcn_raw_ptr_buffer_load_lds(rsL, (LDS void*)dx, 16, lane * 16, xo, 3072, AUX_SYS);
      if (k == 4) __builtin_amdgcn_raw_ptr_buffer_load_lds(rsL, (LDS void*)dw, 16, lane * 16, wo, 0, AUX_SYS);
      if (k == 5) __builtin_amdgcn_raw_ptr_buffer_load_lds(rsL, (LDS void*)dw, 16, lane * 16, wo, 1024, AUX_SYS);
    };
    f32x4 acc[4][4];
    #pragma unroll
    for (int mi = 0; mi < 4; ++mi)
      #pragma unroll
      for (int ni = 0; ni < 4; ++ni) acc[mi][ni] = f32x4{0.f, 0.f, 0.f, 0.f};
    if (wave == 0) {
      if (ti == slot) ctl_st(deadw, 0u);
      choose(pick + 8); choose(pick + 12);
    }
    asm volatile("s_waitcnt vmcnt(0) lgkmcnt(0)\n\ts_barrier" ::: "memory");
    const int ks0 = (int)ctl_ld(pick + 8), ks1 = (int)ctl_ld(pick + 12);
    #pragma unroll
    for (int k = 0; k < 12; ++k) dma(k % 6, k < 6 ? ks0 : ks1, k / 6 * STAGE);
    u32 st = 0;
    u32 flip = 0;
    i32x8 fa[8], fb[8];
    auto step = [&](i32x8 (&cur)[8], const i32x8 (&prev)[8], bool first, bool more, bool last) {
      if (more && wave == 0) choose(pick + flip);
      if (!last) asm volatile("s_waitcnt vmcnt(6) lgkmcnt(0)\n\ts_barrier" ::: "memory");
      else asm volatile("s_waitcnt vmcnt(0) lgkmcnt(0)\n\ts_barrier" ::: "memory");
      const u32 st2 = st == 0 ? 2 * STAGE : st - STAGE;
      LDS char* ax = smem + st + wm * 8192 + lane * 16;
      LDS char* aw = smem + st + 32 * KiB + wn * 8192 + lane * 16;
      u32x4 h[16];
      auto M = [&](int k) {
        if (!first) acc[k >> 2][k & 3] = __builtin_amdgcn_mfma_scale_f32_16x16x128_f8f6f4(prev[k & 3], prev[4 + (k >> 2)], acc[k >> 2][k & 3], 1, 1, 0, 0, 0, 0);
        __builtin_amdgcn_sched_barrier(0);
      };
      auto R = [&](int k) { h[k] = *(LDS u32x4*)(((k >> 1) & 1 ? ax : aw) + (k >> 2) * 2048 + (k & 1) * 1024); __builtin_amdgcn_sched_barrier(0); };
      u32 kv = 0;
      if (more) kv = *(LDS u32*)(uintptr_t)(pick + flip);
      __builtin_amdgcn_sched_barrier(0);
      M(0); R(0); R(1); M(1);
      const int ksn = __builtin_amdgcn_readfirstlane((int)kv);
      #pragma unroll
      for (int k = 1; k < 7; ++k) {
        M(2 * k); R(2 * k); R(2 * k + 1); M(2 * k + 1);
        if (more) dma(k - 1, ksn, st2);
        __builtin_amdgcn_sched_barrier(0);
      }
      R(14); R(15); M(14); M(15);
      #pragma unroll
      for (int f = 0; f < 4; ++f) {
        cur[f] = __builtin_bit_cast(i32x8, __builtin_shufflevector(h[4 * f], h[4 * f + 1], 0, 1, 2, 3, 4, 5, 6, 7));
        cur[4 + f] = __builtin_bit_cast(i32x8, __builtin_shufflevector(h[4 * f + 2], h[4 * f + 3], 0, 1, 2, 3, 4, 5, 6, 7));
      }
      flip ^= 4u;
      st = st == 2 * STAGE ? 0 : st + STAGE;
    };
    step(fa, fb, true, true, false);
    #pragma unroll 1
    for (int s = 1; s < 29; s += 2) { step(fb, fa, false, true, false); step(fa, fb, false, true, false); }
    step(fb, fa, false, true, false);
    step(fa, fb, false, false, false);
    step(fb, fa, false, false, true);
    #pragma unroll
    for (int k = 0; k < 16; ++k)
      acc[k >> 2][k & 3] = __builtin_amdgcn_mfma_scale_f32_16x16x128_f8f6f4(fb[k & 3], fb[4 + (k >> 2)], acc[k >> 2][k & 3], 1, 1, 0, 0, 0, 0);
    int lz = lane; asm volatile("" : "+v"(lz));
    const bool dead = ctl_ld(deadw) != 0u;
    const int owner = rgw >> 3;
    if (owner == rank) {
      const u32 obase = ((rgw & 7) * 64 * NN + group_of(rank, j) * 1024 + c * 128 + wn * 64) * 4, olane = ((lz & 15) * NN + (lz >> 4) * 4) * 4;
      const float sc = dead ? __builtin_nanf("") : scale;
      #pragma unroll
      for (int mi = 0; mi < 4; ++mi)
        #pragma unroll
        for (int ni = 0; ni < 4; ++ni)
          ST128(__builtin_bit_cast(u32x4, acc[mi][ni] * sc), rsO, olane + ni * 64, obase + mi * 16 * NN * 4, AUX_SYS);
    } else if (!dead && !(ti >= 24 && wm == 3)) {
      const int t = (cb * 2 + wn) * 5 + (rgw & 7) - ((owner >> 2) == (rank >> 2) ? 0 : 3);
      const u32 uid = owner * 320 + t;
      const u32 offE = par + EX_OUE + uid * OU_ENC;
      float sa = 0.f;
      #pragma unroll
      for (int mi = 0; mi < 4; ++mi)
        #pragma unroll
        for (int ni = 0; ni < 4; ++ni) sa += __builtin_fabsf(acc[mi][ni][(mi + ni) & 3]);
      const int g = min(119, max(0, 24 - ((int)((as_u(wave_sum_f32(sa) * (4.2f / 1024.f)) >> 23) & 255u) - 126)));
      LDS char* last1k = smem + wave * 4096;
      LDS char* side = last1k + 512;
      LDS char* pend = last1k + 1536;
      *(LDS u32x2*)(side + lz * 8) = u32x2{0u, 0u};
      u32 npend = 0, mk[4] = {0u, 0u, 0u, 0u};
      auto quad = [&](int gq, u32& l0, u32& l1, u32& tp) {
        u32 qv[4], fr[4], sgn = 0;
        #pragma unroll
        for (int r = 0; r < 4; ++r) {
          const float tt = __builtin_amdgcn_ldexpf(acc[gq >> 2][gq & 3][r], g + 7);
          qv[r] = (u32)f2i_sat(tt);
          fr[r] = as_u(__builtin_amdgcn_fractf(tt));
          sgn = __builtin_amdgcn_bitop3_b32(as_u(tt), qv[r], sgn, 0xBE);
        }
        const u32 p01 = __builtin_amdgcn_perm(qv[1], qv[0], 0x07030400u), p23 = __builtin_amdgcn_perm(qv[3], qv[2], 0x07030400u);
        const u32 low = __builtin_amdgcn_perm(p23, p01, 0x05040100u);
        tp = __builtin_amdgcn_perm(p23, p01, 0x07060302u);
        l0 = __builtin_amdgcn_perm(qv[1], qv[0], 0x06050201u);
        l1 = __builtin_amdgcn_perm(qv[3], qv[2], 0x06050201u);
        mk[gq >> 3] = bfi(0x80808080u >> (gq & 7), (tp ^ (tp << 1)) >> (gq & 7), mk[gq >> 3]);
        mk[2 + (gq >> 3)] = bfi(0x80808080u >> (gq & 7), low >> (gq & 7), mk[2 + (gq >> 3)]);
        const u32 lowest = tp & ~((tp & 0x7F7F7F7Fu) + 0x7F7F7F7Fu) & 0x80808080u;
        const bool odd_one = (fr[0] | fr[1] | fr[2] | fr[3] | (low & 0x7F7F7F7Fu) | lowest | (sgn & 0x80000000u)) != 0u;
        const u64 bm = __ballot(odd_one);
        if (odd_one) {
          const u32 sl = min(63u, npend + __builtin_amdgcn_mbcnt_hi((u32)(bm >> 32), __builtin_amdgcn_mbcnt_lo((u32)bm, 0u)));
          *(LDS f32x4*)(pend + sl * 16) = acc[gq >> 2][gq & 3];
          *(LDS u32*)(pend + 1024 + sl * 4) = gq * 64 + lz;
        }
        npend += __popcll(bm);
      };
      u32 y0, y1, z0, z1, th[2];
      quad(14, y0, y1, th[0]);
      quad(15, z0, z1, th[1]);
      *(LDS u32*)(last1k + lz * 4) = z0;
      *(LDS u32*)(last1k + 256 + lz * 4) = z1;
      #pragma unroll
      for (int mi = 0; mi < 4; ++mi) {
        u32 pk[12];
        #pragma unroll
        for (int ni = 0; ni < 4; ++ni) {
          const int gq = mi * 4 + ni, be = gq % 7;
          if (gq == 14) { pk[6] = y0; pk[7] = y1; break; }
          quad(gq, pk[ni * 3], pk[ni * 3 + 1], pk[ni * 3 + 2]);
          pk[ni * 3 + 2] = bfi(0x40404040u, be < 6 ? th[gq / 7] << (6 - be) : th[gq / 7] >> 1, pk[ni * 3 + 2]);
        }
        #pragma unroll
        for (int j = 0; j < (mi < 3 ? 3 : 2); ++j)
          ST128((u32x4{pk[j * 4], pk[j * 4 + 1], pk[j * 4 + 2], pk[j * 4 + 3]}), rsL, lz * 16 + j * 1024, offE + mi * 3072, AUX_SYS);
      }
      const u32 mine = (__popc(mk[0]) + __popc(mk[1])) | ((__popc(mk[2]) + __popc(mk[3])) << 16);
      const u32 inc = wave_scan_u32(mine), tot = (u32)__builtin_amdgcn_readlane((int)inc, 63);
      const u32 n0 = tot & 0xFFFFu, n1 = tot >> 16;
      Side sm = side_map(n0, n1, 0u, 0u);
      bool raw = npend > 64u || sm.end > 512u;
      u32 ns = 0, nf = 0;
      if (!raw) {
        wave_sync();
        u32 kk[2] = {(inc - mine) & 0xFFFFu, (inc - mine) >> 16};
        #pragma unroll
        for (int w = 0; w < 4; ++w) {
          const u32 hb = sm.h[w >> 1] * 8 + lz * 4 + (w & 1) * 2, lb = sm.l[w >> 1] * 8;
          for (u32 m = mk[w]; m; m &= m - 1) {
            const u32 bp = (u32)__builtin_ctz(m), k = kk[w >> 1]++, ph = hb + (bp >> 4) + k, pl = lb + k * 4;
            __hip_atomic_fetch_or((LDS u32*)(side + ((ph >> 5) << 2)), 1u << (ph & 31u), __ATOMIC_RELAXED, __HIP_MEMORY_SCOPE_WORKGROUP);
            __hip_atomic_fetch_or((LDS u32*)(side + ((pl >> 5) << 2)), (bp & 15u) << (pl & 31u), __ATOMIC_RELAXED, __HIP_MEMORY_SCOPE_WORKGROUP);
          }
        }
        LDS char* fl = last1k + 1024;
        for (u32 p0 = 0; p0 < npend; p0 += 16) {
          const u32 pi = p0 + (lz >> 2), r = lz & 3;
          const bool on = pi < npend;
          const float v = *(LDS float*)(pend + p0 * 16 + lz * 4);
          const u32 pp = *(LDS u32*)(pend + 1024 + pi * 4);
          const float tt = __builtin_amdgcn_ldexpf(v, g + 7);
          const u32 iv = (u32)f2i_sat(tt);
          const bool full = on && (as_u((float)(int)iv) != as_u(tt) || (iv & 31u)), brief = on && !full && (iv & 96u);
          const u64 bs = __ballot(brief), bf = __ballot(full);
          const u32 ss = ns + __builtin_amdgcn_mbcnt_hi((u32)(bs >> 32), __builtin_amdgcn_mbcnt_lo((u32)bs, 0u));
          const u32 sf = nf + __builtin_amdgcn_mbcnt_hi((u32)(bf >> 32), __builtin_amdgcn_mbcnt_lo((u32)bf, 0u));
          if (brief && sm.s + ss * 2 < 512u) *(LDS unsigned short*)(side + sm.s + ss * 2) = (unsigned short)((pp << 5) | (r << 3) | ((iv >> 5) & 3u));
          if (full && sf < OF_CAP) *(LDS u32x2*)(fl + sf * 8) = u32x2{pp * 4 + r, as_u(v)};
          ns += __popcll(bs); nf += __popcll(bf);
        }
        sm = side_map(n0, n1, ns, nf);
        raw = ns > OS_CAP || nf > OF_CAP || sm.end > 512u;
        if (nf && !raw) {
          wave_sync();
          const u32x2 e = *(LDS u32x2*)(fl + (lz & 31) * 8);
          if ((u32)lz < nf) {
            LDS unsigned short* at = (LDS unsigned short*)(side + sm.f + lz * 6);
            at[0] = (unsigned short)e.x; at[1] = (unsigned short)e.y; at[2] = (unsigned short)(e.y >> 16);
          }
        }
      }
      wave_sync();
      if (lz == 0) {
        *(LDS u32*)side = raw ? 0x80000000u : (n0 | (n1 << 10) | ((u32)g << 20));
        *(LDS unsigned short*)(side + 4) = (unsigned short)(ns | (nf << 8));
      }
      const u32 lines = raw ? 1u : (sm.end + 127) / 128;
      wave_sync();
      if ((u32)lz < 32 + lines * 8) ST128(*(LDS u32x4*)(last1k + lz * 16), rsL, lz * 16, offE + 11264, AUX_SYS);
      if (raw) {
        const u32 offR = par + EX_OUR + uid * OU_RAW;
        #pragma unroll
        for (int mi = 0; mi < 4; ++mi)
          #pragma unroll
          for (int ni = 0; ni < 4; ++ni)
            ST128(__builtin_bit_cast(u32x4, acc[mi][ni]), rsL, lz * 16 + ni * 1024, offR + mi * 4096, AUX_SYS);
      }
      vm_drain();
      if (lz == owner)
        __hip_atomic_store((GLB u32*)(peer_ws + OFF_FOUT + (rank * 320 + t) * 128), call << 4 | lines, __ATOMIC_RELAXED, __HIP_MEMORY_SCOPE_SYSTEM);
    }
  }
}

}

extern "C" void kernel_launch(int rank, int nranks, ncclComm_t comm, hipStream_t stream,
                              void* const* d_in, const int* in_sizes, int n_in,
                              void* d_out, int out_size,
                              const int* in_dtypes, int out_dtype,
                              void* const* d_out_all,
                              void* const* d_ws_all, size_t ws_bytes) {
  (void)comm; (void)in_sizes; (void)n_in; (void)out_size; (void)in_dtypes; (void)out_dtype; (void)d_out_all;
  if (nranks != NR || (unsigned)rank >= (unsigned)NR || ws_bytes < WS_NEED) { fprintf(stderr, "unexpected launch shape\n"); return; }
  static unsigned attr_set = 0;
  if (!(attr_set >> rank & 1u)) {
    HIP_CHECK(hipFuncSetAttribute((const void*)gemm_rs_kernel, hipFuncAttributeMaxDynamicSharedMemorySize, (int)LDS_BYTES));
    attr_set |= 1u << rank;
  }
  Params p;
  p.x = (const uint8_t*)d_in[0]; p.w = (const uint8_t*)d_in[1]; p.sx = (const float*)d_in[2]; p.sw = (const float*)d_in[3];
  p.out = (float*)d_out; p.ws_all = (char* const*)d_ws_all; p.ws = (char*)d_ws_all[rank]; p.rank = rank; p.ws_bytes = WS_NEED;
  hipLaunchKernelGGL(gemm_rs_kernel, dim3(NWG), dim3(NTHR), LDS_BYTES, stream, p);
  HIP_CHECK(hipGetLastError());
}
```
